```python
import numpy as np

kernel_name = "gpu_gemm_ar_m4096_k4096_n16384_bf16_reluqe4m3"


ABSMAX_RTOL = 0.09

COMPUTE_DTYPE = "bfloat16"
EPILOGUE = "relu_quant_e4m3"
IN_QUANT = None
OUT_QUANT = "e4m3"
IN_DTYPES = {"x": "bfloat16", "w_mat": "bfloat16"}
WORLD_SIZE = 8

def _to_bf16(x):
    u = np.ascontiguousarray(x, dtype=np.float32).view(np.uint32)
    return ((u + 0x7FFF + ((u >> 16) & 1)) & 0xFFFF0000).view(np.float32)

def setup_inputs():

    x = _to_bf16(np.random.randn(4096, 4096).astype(np.float32))
    w_mat = _to_bf16((np.random.randn(4096, 16384) / np.sqrt(4096)).astype(np.float32))
    return {"x": x, "w_mat": w_mat}

def _snap_fp8(x, mant_bits, max_abs, min_normal):
    a = np.abs(x.astype(np.float64))
    _m, _e = np.frexp(a)
    q = np.where(a >= min_normal, np.ldexp(1.0, _e - 1 - mant_bits),
                 min_normal * 2.0 ** -mant_bits)
    snapped = np.minimum(np.round(a / q) * q, max_abs)
    return (np.sign(x) * snapped).astype(np.float32)
def _quant_dequant(y, amax):
    scale = amax / 448.0
    q = _snap_fp8(y / scale, 3, 448.0, 2.0 ** -6)
    return (q * scale).astype(np.float32)
def _epilogue(y):
    y = np.maximum(y, 0.0)
    return _quant_dequant(y, np.abs(y).max())

def reference(x, w_mat):
    y = (x @ w_mat).astype(np.float32)
    return _epilogue(y)
```

```cpp
#include <hip/hip_runtime.h>
#include <rccl/rccl.h>
#include <cstdio>

#define LAS __attribute__((address_space(3)))
#define GAS __attribute__((address_space(1)))
typedef short bf16x8 __attribute__((ext_vector_type(8)));
typedef float f32x4 __attribute__((ext_vector_type(4)));
typedef unsigned u32x4 __attribute__((ext_vector_type(4)));
typedef unsigned u32x2 __attribute__((ext_vector_type(2)));
typedef __amdgpu_buffer_rsrc_t rsrc_t;

constexpr int NR = 8;
constexpr int M_ = 4096, N_ = 16384;
constexpr int KS = 512;
constexpr int NS = 2048;
constexpr int NT = 64;
constexpr int NGEMM = 128;
constexpr int NWG = 256;

constexpr unsigned OFF_CNT  = 0;
constexpr unsigned OFF_TMO  = 16384;
constexpr unsigned OFF_FLAG = 65536;
constexpr unsigned FLAG_REC = 1024, FLAG_W = 512;
constexpr unsigned OFF_AMAX = 131072;
constexpr unsigned OFF_DONE = OFF_AMAX + 8192;
constexpr unsigned OFF_DBG  = 262144;
constexpr unsigned CTRL_BYTES = 1u << 20;
constexpr unsigned XT_STEP = 512u * 1024u, WT_STEP = 256u * 1024u, BLK = 32768u, HALFB = 16384u;
constexpr unsigned OFF_XT = CTRL_BYTES;
constexpr unsigned XT_BYTES = NT * XT_STEP;
constexpr unsigned OFF_WT = OFF_XT + XT_BYTES;
constexpr unsigned WT_BYTES = NT * WT_STEP;
constexpr unsigned WS_USED = OFF_WT + WT_BYTES;
constexpr unsigned OUT_BYTES = (unsigned)M_ * N_;
constexpr unsigned X_BYTES = (unsigned)M_ * KS * 2, W_BYTES = (unsigned)KS * N_ * 2;

constexpr unsigned long long DEADLINE = 100000000ull;
constexpr int AUX_SYS = 17;
#define GEMM_AUX 17
#define NXW 32
#define NWW 112

constexpr unsigned LDS_STAGE = 8 * HALFB;
constexpr unsigned LDS_FLAG = LDS_STAGE;
constexpr unsigned LDS_RED = LDS_FLAG + 512;
constexpr unsigned LDS_BYTES = LDS_RED + 64;

struct Params {
    const void* x; const void* w;
    void* out[NR]; void* ws[NR];
    int rank; int pad;
};

__device__ __forceinline__ rsrc_t mk_rsrc(const void* p, unsigned bytes) {
    return __builtin_amdgcn_make_buffer_rsrc((void*)p, 0, bytes, 0x00020000);
}
#define WAIT_V(n) asm volatile("s_waitcnt vmcnt(" #n ")" ::: "memory")
#define WAIT_L(n) asm volatile("s_waitcnt lgkmcnt(" #n ")" ::: "memory")
#define BAR __builtin_amdgcn_s_barrier()
#define SCHED __builtin_amdgcn_sched_barrier(0)
#define CFENCE asm volatile("" ::: "memory")

#define STAMP(ws_, wv_, k_) do {} while (0)

__device__ __forceinline__ unsigned ld_agent(const void* p) { return __hip_atomic_load((const GAS unsigned*)p, __ATOMIC_RELAXED, __HIP_MEMORY_SCOPE_AGENT); }
__device__ __forceinline__ void st_agent(void* p, unsigned v) { __hip_atomic_store((GAS unsigned*)p, v, __ATOMIC_RELAXED, __HIP_MEMORY_SCOPE_AGENT); }
__device__ __forceinline__ void st_sys(void* p, unsigned v) { __hip_atomic_store((GAS unsigned*)p, v, __ATOMIC_RELAXED, __HIP_MEMORY_SCOPE_SYSTEM); }

__device__ __forceinline__ bool spin_fail(char* myws, unsigned call, unsigned long long t0) {
    if (__builtin_amdgcn_readfirstlane(ld_agent(myws + OFF_TMO)) == call || __builtin_amdgcn_s_memrealtime() - t0 > DEADLINE) {
        st_agent(myws + OFF_TMO, call);
        return true;
    }
    __builtin_amdgcn_s_sleep(2);
    return false;
}

__device__ __forceinline__ int step_of(int i, int rank) {
    if (i < 8) return rank * 8 + i;
    const int q = i - 8, j = q / 7, p = q - j * 7;
    return (((rank + 1 + p) & 7) << 3) + j;
}

__device__ __forceinline__ void x_load(u32x4 (&v)[8], rsrc_t rx, int j, int g, int lane) {
    const unsigned voff = (unsigned)(lane & 15) * (KS * 2) + (unsigned)(lane >> 4) * 16;
    const unsigned soff = (unsigned)g * 64 * (KS * 2) + (unsigned)j * 128;
#pragma unroll
    for (int r = 0; r < 4; ++r)
#pragma unroll
        for (int kk = 0; kk < 2; ++kk) v[r * 2 + kk] = __builtin_amdgcn_raw_buffer_load_b128(rx, voff + kk * 64, soff + r * 16 * (KS * 2), 0);
}
__device__ __forceinline__ void x_store(const u32x4 (&v)[8], rsrc_t rd, int t, int g, int lane) {
    const unsigned soff = OFF_XT + (unsigned)t * XT_STEP + (unsigned)g * 8192;
#pragma unroll
    for (int f = 0; f < 8; ++f) __builtin_amdgcn_raw_buffer_store_b128(v[f], rd, lane * 16 + (f & 3) * 1024, soff + (f >> 2) * 4096, AUX_SYS);
}
__device__ __forceinline__ void w_load(u32x2 (&v)[16], rsrc_t rw, int j, int d, int ct, int kk, int G, int lane) {
    const unsigned voff = (unsigned)(lane >> 4) * 8 * (N_ * 2) + (unsigned)(lane & 15) * 8;
    const unsigned soff = (unsigned)(j * 64 + kk * 32) * (N_ * 2) + (unsigned)(d * NS + ct * 256 + G * 128) * 2;
#pragma unroll
    for (int jj = 0; jj < 8; ++jj)
#pragma unroll
        for (int g = 0; g < 2; ++g) v[g * 8 + jj] = __builtin_amdgcn_raw_buffer_load_b64(rw, voff + g * 128, soff + jj * (N_ * 2), 0);
}
__device__ __forceinline__ void w_store(const u32x2 (&v)[16], rsrc_t rd, int t, int ct, int kk, int G, int lane) {
    const unsigned soff = OFF_WT + (unsigned)t * WT_STEP + (unsigned)ct * BLK + (unsigned)G * 8192 + (unsigned)kk * 1024;
#pragma unroll
    for (int g = 0; g < 2; ++g)
#pragma unroll
        for (int c = 0; c < 4; ++c) {
            u32x4 o;
#pragma unroll
            for (int dd = 0; dd < 4; ++dd) {
                const unsigned lo = v[g * 8 + 2 * dd][c >> 1], hi = v[g * 8 + 2 * dd + 1][c >> 1];
                o[dd] = __builtin_amdgcn_perm(hi, lo, (c & 1) ? 0x07060302u : 0x05040100u);
            }
            __builtin_amdgcn_raw_buffer_store_b128(o, rd, lane * 16 + (c & 1) * 2048, soff + (c >> 1) * HALFB + g * 4096, AUX_SYS);
        }
}

__device__ __forceinline__ void comm_role(const Params& p, int c, int wid, int lane, unsigned call) {
    const int rank = p.rank;
    char* myws = (char*)p.ws[rank];
    const rsrc_t rx = mk_rsrc(p.x, X_BYTES), rw = mk_rsrc(p.w, W_BYTES);
    const int wv = (NGEMM + c) * 8 + wid; (void)wv;
    STAMP(myws, wv, 0);
    if (wid == 0) {
        constexpr int XSTRIDE = 128 / NXW;
        const int grp = c >> 3;
        if (grp % XSTRIDE == 0) {
            const int me = (grp / XSTRIDE) * 8 + (c & 7);
            char* fdst = (char*)p.ws[lane & 7];
            u32x4 v[8];
            x_load(v, rx, me >> 6, me & 63, lane);
            for (int u = me; u < 512; u += NXW) {
                const int j = u >> 6, g = u & 63, t = rank * 8 + j;
#pragma unroll
                for (int q = 1; q < 8; ++q) {
                    int d = q + me % 7; if (d > 7) d -= 7;
                    d = (rank + d) & 7;
                    x_store(v, mk_rsrc(p.ws[d], WS_USED), t, g, lane);
                }
                const int un = u + NXW < 512 ? u + NXW : u;
                x_load(v, rx, un >> 6, un & 63, lane);
                WAIT_V(0);
                if (lane < 8 && lane != rank) st_sys(fdst + OFF_FLAG + t * FLAG_REC + g * 4, call);
                if (g + NXW >= 64 || NXW >= 64) STAMP(myws, wv, 1 + j);
            }
        }
    } else if (wid == 1) {
        if (c < NWW) {
            u32x2 v[16];
            { const int u = c, j = u / 224, r = u % 224, q = r % 7, e = r / 7; w_load(v, rw, j, (rank + 1 + q) & 7, e >> 2, e & 1, (e >> 1) & 1, lane); }
            for (int u = c; u < 8 * 224; u += NWW) {
                const int j = u / 224, r = u % 224, q = r % 7, e = r / 7, d = (rank + 1 + q) & 7, t = rank * 8 + j;
                w_store(v, mk_rsrc(p.ws[d], WS_USED), t, e >> 2, e & 1, (e >> 1) & 1, lane);
                { const int un = u + NWW < 8 * 224 ? u + NWW : u, j2 = un / 224, r2 = un % 224, q2 = r2 % 7, e2 = r2 / 7;
                  w_load(v, rw, j2, (rank + 1 + q2) & 7, e2 >> 2, e2 & 1, (e2 >> 1) & 1, lane); }
                WAIT_V(0);
                if (lane == 0) st_sys((char*)p.ws[d] + OFF_FLAG + t * FLAG_REC + FLAG_W + e * 4, call);
                if (r + NWW >= 224) STAMP(myws, wv, 1 + j);
            }
        }
    } else {
        const int u = (wid - 2) * 128 + c;
        const rsrc_t rd = mk_rsrc(myws, WS_USED);
        if (u < 256) {
            const int j = u >> 5, e = u & 31, t = rank * 8 + j;
            u32x2 v[16];
            w_load(v, rw, j, rank, e >> 2, e & 1, (e >> 1) & 1, lane);
            w_store(v, rd, t, e >> 2, e & 1, (e >> 1) & 1, lane);
            WAIT_V(0);
            if (lane == 0) st_sys(myws + OFF_FLAG + t * FLAG_REC + FLAG_W + e * 4, call);
        } else {
            const int ux = u - 256, j = ux >> 6, g = ux & 63, t = rank * 8 + j;
            u32x4 v[8];
            x_load(v, rx, j, g, lane);
            x_store(v, rd, t, g, lane);
            WAIT_V(0);
            if (lane == 0) st_sys(myws + OFF_FLAG + t * FLAG_REC + g * 4, call);
        }
        STAMP(myws, wv, 1);
    }
    if (c == 127 - 8) {
        if (wid != rank) {
            const rsrc_t rs = mk_rsrc(myws, WS_USED);
            const unsigned long long t0 = __builtin_amdgcn_s_memrealtime();
            for (;;) {
                CFENCE;
                const u32x2 f = __builtin_amdgcn_raw_buffer_load_b64(rs, lane * 8, OFF_DONE + wid * 512, AUX_SYS);
                if (__builtin_amdgcn_ballot_w64(f.x < call || f.y < call) == 0) break;
                if (spin_fail(myws, call, t0)) break;
            }
            if (__builtin_amdgcn_readfirstlane(ld_agent(myws + OFF_TMO)) == call && lane == 0)
                ((GAS unsigned char*)p.out[rank])[rank * NS] = 0x7f;
        }
        STAMP(myws, wv, 10);
    }
}

__device__ __forceinline__ void gemm_role(const Params& p, LAS unsigned char* lds, int rt, int ct, int wid, int lane, unsigned call) {
    const int rank = p.rank;
    char* myws = (char*)p.ws[rank];
    const int tile = rt * 8 + ct;
    const int wv = tile * 8 + wid; (void)wv;
    const int wr = wid >> 2, wc = wid & 3;
    const rsrc_t rs = mk_rsrc(myws, WS_USED);
    const unsigned voff = (unsigned)wid * 2048 + (unsigned)lane * 16;
    const unsigned voffF = lane < 4 ? (unsigned)(rt * 16 + lane * 4) : FLAG_W + (unsigned)(ct * 16 + (lane & 3) * 4);
    LAS unsigned char* ldsw = lds + wid * 2048;
    LAS unsigned char* lda = lds + wr * 8192 + lane * 16;
    LAS unsigned char* ldb = lds + wc * 4096 + lane * 16;
    const int seqv = step_of(lane, rank);
#define SEQ(i) __builtin_amdgcn_readlane(seqv, (i) < NT ? (i) : NT - 1)
    STAMP(myws, wv, 0);
#define SA(b, h) (((b) * 2 + (h)) * HALFB)
#define SB(b, h) ((4 + (b) * 2 + (h)) * HALFB)
#define STAGE(bufoff, soff) do { \
        __builtin_amdgcn_raw_ptr_buffer_load_lds(rs, (LAS void*)(ldsw + (bufoff)), 16, voff, (soff), 0, GEMM_AUX); \
        __builtin_amdgcn_raw_ptr_buffer_load_lds(rs, (LAS void*)(ldsw + (bufoff)), 16, voff, (soff), 1024, GEMM_AUX); } while (0)
#define LDA(dst, b, h) do { _Pragma("unroll") for (int m = 0; m < 4; ++m) _Pragma("unroll") for (int k = 0; k < 2; ++k) dst[m][k] = *(const LAS bf16x8*)(lda + SA(b, h) + m * 2048 + k * 1024); } while (0)
#define LDB(dst, b, h) do { _Pragma("unroll") for (int n = 0; n < 2; ++n) _Pragma("unroll") for (int k = 0; k < 2; ++k) dst[n][k] = *(const LAS bf16x8*)(ldb + SB(b, h) + n * 2048 + k * 1024); } while (0)
#define MMA(ai, bj, At, Bt) do { __builtin_amdgcn_s_setprio(1); _Pragma("unroll") for (int m = 0; m < 4; ++m) _Pragma("unroll") for (int n = 0; n < 2; ++n) _Pragma("unroll") for (int k = 0; k < 2; ++k) \
        acc[ai][bj][m][n] = __builtin_amdgcn_mfma_f32_16x16x32_bf16(Bt[n][k], At[m][k], acc[ai][bj][m][n], 0, 0, 0); __builtin_amdgcn_s_setprio(0); } while (0)
#define OFFA(t) (OFF_XT + (unsigned)(t) * XT_STEP + (unsigned)rt * BLK)
#define OFFB(t) (OFF_WT + (unsigned)(t) * WT_STEP + (unsigned)ct * BLK)
#define SLOW_WAIT(t) do { const unsigned long long t0_ = __builtin_amdgcn_s_memrealtime(); \
        for (;;) { CFENCE; const unsigned f_ = __builtin_amdgcn_raw_buffer_load_b32(rs, voffF, OFF_FLAG + (unsigned)(t) * FLAG_REC, AUX_SYS); \
            if (__builtin_amdgcn_ballot_w64(f_ < call) == 0) break; if (spin_fail(myws, call, t0_)) break; } } while (0)
#define CHECK(t, s, tn) do { if (wid == 0) { const unsigned f0_ = *(volatile LAS unsigned*)(lds + LDS_FLAG + (s) * 256 + lane * 4); \
            if (__builtin_amdgcn_ballot_w64(f0_ < call) != 0) SLOW_WAIT(t); \
            __builtin_amdgcn_raw_ptr_buffer_load_lds(rs, (LAS void*)(lds + LDS_FLAG + ((s) ^ 1) * 256), 4, voffF, OFF_FLAG + (unsigned)(tn) * FLAG_REC, 0, AUX_SYS); } } while (0)

    f32x4 acc[2][2][4][2];
#pragma unroll
    for (int a = 0; a < 2; ++a)
#pragma unroll
        for (int b = 0; b < 2; ++b)
#pragma unroll
            for (int m = 0; m < 4; ++m)
#pragma unroll
                for (int n = 0; n < 2; ++n) acc[a][b][m][n] = (f32x4){0.f, 0.f, 0.f, 0.f};
    bf16x8 At[4][2], B0[2][2], B1[2][2];

    {
        const int t0 = SEQ(0), t1 = SEQ(1), t2 = SEQ(2);
        if (wid == 0) {
            SLOW_WAIT(t0); SLOW_WAIT(t1);
            __builtin_amdgcn_raw_ptr_buffer_load_lds(rs, (LAS void*)(lds + LDS_FLAG), 4, voffF, OFF_FLAG + (unsigned)t2 * FLAG_REC, 0, AUX_SYS);
        }
        CFENCE; BAR; CFENCE;
        STAMP(myws, wv, 1);
        STAGE(SB(0, 0), OFFB(t0)); STAGE(SA(0, 0), OFFA(t0)); STAGE(SB(0, 1), OFFB(t0) + HALFB); STAGE(SA(0, 1), OFFA(t0) + HALFB);
        if (wr == 1) BAR;
        WAIT_V(4); BAR;
        STAGE(SB(1, 0), OFFB(t1)); STAGE(SA(1, 0), OFFA(t1)); STAGE(SB(1, 1), OFFB(t1) + HALFB);
        WAIT_V(6); BAR;
    }
    for (int i = 0; i < NT; i += 2) {
        const int t1 = SEQ(i + 1), t2 = SEQ(i + 2), t3 = SEQ(i + 3), t4 = SEQ(i + 4);
        const unsigned a1 = OFFA(t1), a2 = OFFA(t2), b2 = OFFB(t2), a3 = OFFA(t3), b3 = OFFB(t3);
        CHECK(t2, 0, t3);
        LDB(B0, 0, 0); SCHED; LDA(At, 0, 0); STAGE(SA(1, 1), a1 + HALFB);
        WAIT_L(8); BAR; WAIT_L(0); MMA(0, 0, At, B0); BAR; SCHED;
        LDB(B1, 0, 1); STAGE(SB(0, 0), b2);
        BAR; WAIT_L(0); MMA(0, 1, At, B1); BAR;
        LDA(At, 0, 1); STAGE(SA(0, 0), a2);
        BAR; WAIT_L(0); MMA(1, 0, At, B0); BAR; SCHED;
        STAGE(SB(0, 1), b2 + HALFB);
        WAIT_V(6); BAR; MMA(1, 1, At, B1); BAR;
        CHECK(t3, 1, t4);
        LDB(B0, 1, 0); SCHED; LDA(At, 1, 0); STAGE(SA(0, 1), a2 + HALFB);
        WAIT_L(8); BAR; WAIT_L(0); MMA(0, 0, At, B0); BAR; SCHED;
        LDB(B1, 1, 1); STAGE(SB(1, 0), b3);
        BAR; WAIT_L(0); MMA(0, 1, At, B1); BAR;
        LDA(At, 1, 1); STAGE(SA(1, 0), a3);
        BAR; WAIT_L(0); MMA(1, 0, At, B0); BAR; SCHED;
        STAGE(SB(1, 1), b3 + HALFB);
        WAIT_V(6); BAR; MMA(1, 1, At, B1); BAR;
    }
    WAIT_V(0);
    if (wr == 0) BAR;
    BAR;
    STAMP(myws, wv, 10);

    float mx = 0.f;
#pragma unroll
    for (int a = 0; a < 2; ++a)
#pragma unroll
        for (int b = 0; b < 2; ++b)
#pragma unroll
            for (int m = 0; m < 4; ++m)
#pragma unroll
                for (int n = 0; n < 2; ++n)
#pragma unroll
                    for (int j = 0; j < 4; ++j) mx = fmaxf(mx, acc[a][b][m][n][j]);
#pragma unroll
    for (int s = 32; s >= 1; s >>= 1) mx = fmaxf(mx, __shfl_xor(mx, s));
    LAS float* red = (LAS float*)(lds + LDS_RED);
    if (lane == 0) red[wid] = mx;
    WAIT_L(0); BAR; CFENCE;
    if (wid == 0) {
        float t = red[lane & 7];
#pragma unroll
        for (int s = 4; s >= 1; s >>= 1) t = fmaxf(t, __shfl_xor(t, s));
        if (lane < 8) {
            GAS unsigned long long* g = (GAS unsigned long long*)((char*)p.ws[lane] + OFF_AMAX + (rank * NGEMM + tile) * 8);
            __hip_atomic_store(g, ((unsigned long long)call << 32) | __float_as_uint(t), __ATOMIC_RELAXED, __HIP_MEMORY_SCOPE_SYSTEM);
        }
    }
    {
        const unsigned long long t0 = __builtin_amdgcn_s_memrealtime();
        const GAS unsigned long long* g = (const GAS unsigned long long*)(myws + OFF_AMAX + wid * 1024 + lane * 16);
        float t;
        for (;;) {
            const unsigned long long g0 = __hip_atomic_load(g, __ATOMIC_RELAXED, __HIP_MEMORY_SCOPE_SYSTEM), g1 = __hip_atomic_load(g + 1, __ATOMIC_RELAXED, __HIP_MEMORY_SCOPE_SYSTEM);
            t = fmaxf(__uint_as_float((unsigned)g0), __uint_as_float((unsigned)g1));
            if (__builtin_amdgcn_ballot_w64((unsigned)(g0 >> 32) < call || (unsigned)(g1 >> 32) < call) == 0) break;
            if (spin_fail(myws, call, t0)) break;
        }
#pragma unroll
        for (int s = 32; s >= 1; s >>= 1) t = fmaxf(t, __shfl_xor(t, s));
        if (lane == 0) red[8 + wid] = t;
    }
    WAIT_L(0); BAR; CFENCE;
    float amax = red[8];
#pragma unroll
    for (int s = 1; s < 8; ++s) amax = fmaxf(amax, red[8 + s]);
    const bool dead = __builtin_amdgcn_readfirstlane(ld_agent(myws + OFF_TMO)) == call;
    const float inv = 448.0f / amax;
    STAMP(myws, wv, 11);

    {
        const int fr = lane & 15, fq = lane >> 4;
#pragma unroll
        for (int a = 0; a < 2; ++a)
#pragma unroll
            for (int m = 0; m < 4; ++m) {
                u32x4 o;
#pragma unroll
                for (int j = 0; j < 4; ++j) {
                    int w = __builtin_amdgcn_cvt_pk_fp8_f32(fmaxf(acc[a][0][m][0][j], 0.f) * inv, fmaxf(acc[a][0][m][1][j], 0.f) * inv, 0, false);
                    w = __builtin_amdgcn_cvt_pk_fp8_f32(fmaxf(acc[a][1][m][0][j], 0.f) * inv, fmaxf(acc[a][1][m][1][j], 0.f) * inv, w, true);
                    o[j] = dead ? 0x7f7f7f7fu : (unsigned)w;
                }
                const int row = a * 128 + wr * 64 + m * 16 + fr;
                *(LAS u32x4*)(lds + row * 256 + (((wc * 4 + fq) ^ fr) << 4)) = o;
            }
    }
    WAIT_L(0); BAR; CFENCE;
    {
        u32x4 o[8];
        const int r0 = wid * 32 + (lane >> 4), ch = lane & 15;
#pragma unroll
        for (int it = 0; it < 8; ++it) { const int row = r0 + it * 4; o[it] = *(const LAS u32x4*)(lds + row * 256 + ((ch ^ (row & 15)) << 4)); }
        const unsigned vo = (unsigned)r0 * N_ + (unsigned)ch * 16, so = (unsigned)rt * 256 * N_ + (unsigned)(rank * NS + ct * 256);
#pragma unroll
        for (int q = 1; q < 8; ++q) {
            int d = q + tile % 7; if (d > 7) d -= 7;
            d = (rank + d) & 7;
            const rsrc_t ro = mk_rsrc(p.out[d], OUT_BYTES);
#pragma unroll
            for (int it = 0; it < 8; ++it) __builtin_amdgcn_raw_buffer_store_b128(o[it], ro, vo, so + it * 4 * N_, AUX_SYS);
        }
        const rsrc_t ro = mk_rsrc(p.out[rank], OUT_BYTES);
#pragma unroll
        for (int it = 0; it < 8; ++it) __builtin_amdgcn_raw_buffer_store_b128(o[it], ro, vo, so + it * 4 * N_, 0);
    }
    WAIT_V(0); BAR; CFENCE;
    if (wid == 0 && lane < 8 && lane != rank) st_sys((char*)p.ws[lane] + OFF_DONE + (rank * NGEMM + tile) * 4, call);
    STAMP(myws, wv, 12);
}

__global__ __launch_bounds__(512, 2) void fused_kernel(Params p) {
    __shared__ __attribute__((aligned(16))) unsigned char lds_arr[LDS_BYTES];
    LAS unsigned char* lds = (LAS unsigned char*)lds_arr;
    const int wid = __builtin_amdgcn_readfirstlane(threadIdx.x >> 6), lane = threadIdx.x & 63;
    const int b = blockIdx.x, xcd = b & 7, idx = b >> 3;
    unsigned call = 0;
    if (lane == 0) call = __hip_atomic_fetch_add((GAS unsigned*)((char*)p.ws[p.rank] + OFF_CNT) + b * 8 + wid, 1u, __ATOMIC_RELAXED, __HIP_MEMORY_SCOPE_AGENT) + 1u;
    call = __builtin_amdgcn_readfirstlane(call);
    if (idx < 16) gemm_role(p, lds, (xcd >> 1) * 4 + (idx >> 2), (xcd & 1) * 4 + (idx & 3), wid, lane, call);
    else comm_role(p, (idx - 16) * 8 + xcd, wid, lane, call);
}

extern "C" void kernel_launch(int rank, int nranks, ncclComm_t comm, hipStream_t stream,
                              void* const* d_in, const int* in_sizes, int n_in,
                              void* d_out, int out_size,
                              const int* in_dtypes, int out_dtype,
                              void* const* d_out_all, void* const* d_ws_all, size_t ws_bytes) {
    (void)comm; (void)in_dtypes; (void)out_dtype; (void)d_out;
    if (nranks != NR || n_in != 2 || in_sizes[0] != M_ * KS || in_sizes[1] != KS * N_ || out_size != M_ * N_ || ws_bytes < WS_USED || (unsigned)rank >= (unsigned)NR) {
        fprintf(stderr, "kernel_launch: unexpected problem shape\n");
        return;
    }
    Params p;
    p.x = d_in[0]; p.w = d_in[1];
    for (int i = 0; i < NR; ++i) { p.out[i] = d_out_all[i]; p.ws[i] = d_ws_all[i]; }
    p.rank = rank; p.pad = 0;
    fused_kernel<<<dim3(NWG), dim3(512), 0, stream>>>(p);
}
```

```cpp
#include <hip/hip_runtime.h>
#include <rccl/rccl.h>
#include <cstdio>

#define LAS __attribute__((address_space(3)))
#define GAS __attribute__((address_space(1)))
typedef short bf16x8 __attribute__((ext_vector_type(8)));
typedef float f32x4 __attribute__((ext_vector_type(4)));
typedef unsigned u32x4 __attribute__((ext_vector_type(4)));
typedef unsigned u32x2 __attribute__((ext_vector_type(2)));
typedef __amdgpu_buffer_rsrc_t rsrc_t;

constexpr int NR = 8;
constexpr int M_ = 4096, N_ = 16384;
constexpr int KS = 512;
constexpr int NS = 2048;
constexpr int NT = 64;
constexpr int NGEMM = 128;
constexpr int NWG = 256;

constexpr unsigned OFF_CNT  = 0;
constexpr unsigned OFF_TMO  = 16384;
constexpr unsigned OFF_FLAG = 65536;
constexpr unsigned FLAG_REC = 1024, FLAG_W = 512;
constexpr unsigned OFF_AMAX = 131072;
constexpr unsigned OFF_DONE = OFF_AMAX + 8192;
constexpr unsigned OFF_TICK = 143360;
constexpr unsigned OFF_PCNT = 147456;
constexpr unsigned OFF_PFLG = 163840;
constexpr unsigned OFF_DBG  = 262144;
constexpr unsigned CTRL_BYTES = 1u << 20;
constexpr unsigned XT_STEP = 512u * 1024u, WT_STEP = 256u * 1024u, BLK = 32768u, HALFB = 16384u;
constexpr unsigned OFF_XT = CTRL_BYTES;
constexpr unsigned XT_BYTES = NT * XT_STEP;
constexpr unsigned OFF_WT = OFF_XT + XT_BYTES;
constexpr unsigned WT_BYTES = NT * WT_STEP;
constexpr unsigned UNIT = 8192;
constexpr unsigned OFF_PX = OFF_WT + WT_BYTES;
constexpr unsigned OFF_PW = OFF_PX + 512 * UNIT;
constexpr unsigned OFF_PH = OFF_PW + 2048 * UNIT;
constexpr unsigned OFF_MSK = OFF_PH + 2560 * 64;
constexpr unsigned OFF_COD = OFF_MSK + 1024 * 1024;
constexpr unsigned WS_USED = OFF_COD + 1024 * 8192;
constexpr unsigned OUT_BYTES = (unsigned)M_ * N_;
constexpr unsigned X_BYTES = (unsigned)M_ * KS * 2, W_BYTES = (unsigned)KS * N_ * 2;

constexpr unsigned long long DEADLINE = 100000000ull;
constexpr int AUX_SYS = 17;
#define GEMM_AUX 17
#define PPP 12
#define PPO 24
#define MASK_W 2
#define OUT_AUX 0
#define LOC_AUX 17

constexpr unsigned LDS_STAGE = 8 * HALFB;
constexpr unsigned LDS_FLAG = LDS_STAGE;
constexpr unsigned LDS_RED = LDS_FLAG + 512;
constexpr unsigned EX_B = 8192, EX_LUT = EX_B + 2 * 7168, EX_MSK = EX_LUT + 256, EX_BYTES = 81920;
constexpr unsigned LDS_BYTES = 2 * EX_BYTES;
static_assert(LDS_RED + 64 <= LDS_BYTES && EX_MSK + ((512 + PPO - 1) / PPO) * 2048 <= EX_BYTES && 7 * PPO <= 256 && 7 * PPP <= 256, "lds");

struct Params {
    const void* x; const void* w;
    void* out[NR]; void* ws[NR];
    int rank; int pad;
};

__device__ __forceinline__ rsrc_t mk_rsrc(const void* p, unsigned bytes) {
    return __builtin_amdgcn_make_buffer_rsrc((void*)p, 0, bytes, 0x00020000);
}
#define WAIT_V(n) asm volatile("s_waitcnt vmcnt(" #n ")" ::: "memory")
#define WAIT_L(n) asm volatile("s_waitcnt lgkmcnt(" #n ")" ::: "memory")
#define BAR __builtin_amdgcn_s_barrier()
#define SCHED __builtin_amdgcn_sched_barrier(0)
#define CFENCE asm volatile("" ::: "memory")

#define STAMP(ws_, wv_, k_) do {} while (0)

__device__ __forceinline__ unsigned ld_agent(const void* p) { return __hip_atomic_load((const GAS unsigned*)p, __ATOMIC_RELAXED, __HIP_MEMORY_SCOPE_AGENT); }
__device__ __forceinline__ void st_agent(void* p, unsigned v) { __hip_atomic_store((GAS unsigned*)p, v, __ATOMIC_RELAXED, __HIP_MEMORY_SCOPE_AGENT); }
__device__ __forceinline__ void st_sys(void* p, unsigned v) { __hip_atomic_store((GAS unsigned*)p, v, __ATOMIC_RELAXED, __HIP_MEMORY_SCOPE_SYSTEM); }

__device__ __forceinline__ bool spin_fail(char* myws, unsigned call, unsigned long long t0) {
    if (__builtin_amdgcn_readfirstlane(ld_agent(myws + OFF_TMO)) == call || __builtin_amdgcn_s_memrealtime() - t0 > DEADLINE) {
        st_agent(myws + OFF_TMO, call);
        return true;
    }
    __builtin_amdgcn_s_sleep(2);
    return false;
}

#define DPP0(x, ctrl, rm) (unsigned)__builtin_amdgcn_update_dpp(0, (int)(x), ctrl, rm, 0xf, false)
__device__ __forceinline__ unsigned wave_scan(unsigned x) {
    x += DPP0(x, 0x111, 0xf); x += DPP0(x, 0x112, 0xf); x += DPP0(x, 0x114, 0xf); x += DPP0(x, 0x118, 0xf);
    x += DPP0(x, 0x142, 0xa); x += DPP0(x, 0x143, 0xc);
    return x;
}
__device__ __forceinline__ unsigned wave_max(unsigned x) {
    x = max(x, DPP0(x, 0x111, 0xf)); x = max(x, DPP0(x, 0x112, 0xf)); x = max(x, DPP0(x, 0x114, 0xf)); x = max(x, DPP0(x, 0x118, 0xf));
    x = max(x, DPP0(x, 0x142, 0xa)); x = max(x, DPP0(x, 0x143, 0xc));
    return __builtin_amdgcn_readlane(x, 63);
}

__device__ __forceinline__ int x_rot(int q) { return (q * 16) / 7; }
__device__ __forceinline__ int step_of(int i, int rank, int rt) {
    if (i < 8) return rank * 8 + i;
    const int z = i - 8, j = z / 7, p = z - j * 7;
    int q0 = 0;
#pragma unroll
    for (int q = 1; q < 7; ++q) if (x_rot(q) <= rt) q0 = q;
    int q = q0 - p; if (q < 0) q += 7;
    return (((rank + 1 + q) & 7) << 3) + j;
}

__device__ __forceinline__ void x_load(u32x4 (&v)[8], rsrc_t rx, int j, int g, int lane) {
    const unsigned voff = (unsigned)(lane & 15) * (KS * 2) + (unsigned)(lane >> 4) * 16;
    const unsigned soff = (unsigned)g * 64 * (KS * 2) + (unsigned)j * 128;
#pragma unroll
    for (int r = 0; r < 4; ++r)
#pragma unroll
        for (int kk = 0; kk < 2; ++kk) v[r * 2 + kk] = __builtin_amdgcn_raw_buffer_load_b128(rx, voff + kk * 64, soff + r * 16 * (KS * 2), 0);
}
__device__ __forceinline__ void w_load(u32x2 (&v)[16], rsrc_t rw, int j, int d, int ct, int kk, int G, int lane) {
    const unsigned voff = (unsigned)(lane >> 4) * 8 * (N_ * 2) + (unsigned)(lane & 15) * 8;
    const unsigned soff = (unsigned)(j * 64 + kk * 32) * (N_ * 2) + (unsigned)(d * NS + ct * 256 + G * 128) * 2;
#pragma unroll
    for (int jj = 0; jj < 8; ++jj)
#pragma unroll
        for (int g = 0; g < 2; ++g) v[g * 8 + jj] = __builtin_amdgcn_raw_buffer_load_b64(rw, voff + g * 128, soff + jj * (N_ * 2), 0);
}
__device__ __forceinline__ void w_transpose(u32x4 (&o)[8], const u32x2 (&v)[16]) {
#pragma unroll
    for (int g = 0; g < 2; ++g)
#pragma unroll
        for (int c = 0; c < 4; ++c)
#pragma unroll
            for (int dd = 0; dd < 4; ++dd)
                o[g * 4 + c][dd] = __builtin_amdgcn_perm(v[g * 8 + 2 * dd + 1][c >> 1], v[g * 8 + 2 * dd][c >> 1], (c & 1) ? 0x07060302u : 0x05040100u);
}
__device__ __forceinline__ unsigned x_img(int t, int g) { return OFF_XT + (unsigned)t * XT_STEP + (unsigned)g * 8192; }
__device__ __forceinline__ unsigned w_img(int t, int e) { return OFF_WT + (unsigned)t * WT_STEP + (unsigned)(e >> 2) * BLK + (unsigned)((e >> 1) & 1) * 8192 + (unsigned)(e & 1) * 1024; }
__device__ __forceinline__ unsigned frag_off(bool isw, unsigned f) { return isw ? ((f >> 1) & 1) * HALFB + (f >> 2) * 4096 + (f & 1) * 2048 : f * 1024; }
__device__ __forceinline__ void img_store(const u32x4 (&v)[8], rsrc_t rd, bool isw, unsigned img, int lane) {
#pragma unroll
    for (int f = 0; f < 8; ++f) __builtin_amdgcn_raw_buffer_store_b128(v[f], rd, lane * 16, img + frag_off(isw, f), LOC_AUX);
}
__device__ __forceinline__ void pack_store(const u32x4 (&v)[8], rsrc_t rs, char* myws, unsigned slot, unsigned hdr, int lane) {
    unsigned lo[16], hi[16];
#pragma unroll
    for (int f = 0; f < 8; ++f)
#pragma unroll
        for (int h = 0; h < 2; ++h) {
            lo[2 * f + h] = __builtin_amdgcn_perm(v[f][2 * h + 1], v[f][2 * h], 0x06040200u);
            hi[2 * f + h] = __builtin_amdgcn_perm(v[f][2 * h + 1], v[f][2 * h], 0x07050301u);
        }
    unsigned top = 0;
#pragma unroll
    for (int b = 0; b < 4; ++b) top = max(top, max((hi[0] >> (8 * b)) & 0x7fu, (hi[1] >> (8 * b)) & 0x7fu));
    top = wave_max(top);
    const unsigned base = top > 7 ? top - 7 : 0, base4 = base * 0x01010101u;
    unsigned nib[8], bad = 0;
#pragma unroll
    for (int f = 0; f < 8; ++f) {
        const unsigned xa = ((hi[2 * f] | 0x80808080u) - base4) ^ 0x80808080u, xb = ((hi[2 * f + 1] | 0x80808080u) - base4) ^ 0x80808080u;
        bad |= xa | xb;
        nib[f] = ((hi[2 * f] >> 4) & 0x08080808u) | (xa & 0x07070707u) | (hi[2 * f + 1] & 0x80808080u) | ((xb & 0x07070707u) << 4);
    }
    unsigned n = 0;
    if (__builtin_amdgcn_ballot_w64((bad & 0xf8f8f8f8u) != 0) != 0) {
#pragma unroll
        for (int r = 0; r < 16; ++r) {
            const unsigned x = (((hi[r] | 0x80808080u) - base4) ^ 0x80808080u) & 0xf8f8f8f8u;
            if (__builtin_amdgcn_ballot_w64(x != 0) != 0) {
#pragma unroll
                for (int b = 0; b < 4; ++b) {
                    const bool e = ((x >> (8 * b)) & 0xffu) != 0;
                    const unsigned long long m = __builtin_amdgcn_ballot_w64(e);
                    const unsigned pos = n + __builtin_amdgcn_mbcnt_hi((unsigned)(m >> 32), __builtin_amdgcn_mbcnt_lo((unsigned)m, 0));
                    if (e && pos < 15) st_sys(myws + hdr + 4 + pos * 4, ((unsigned)((r >> 1) * 1024 + lane * 16 + ((r & 1) * 4 + b) * 2 + 1) << 8) | ((hi[r] >> (8 * b)) & 0xffu));
                    n += __builtin_popcountll(m);
                }
            }
        }
    }
    if (lane == 0) st_sys(myws + hdr, base | (n << 8));
    if (n <= 15) {
#pragma unroll
        for (int r = 0; r < 4; ++r) __builtin_amdgcn_raw_buffer_store_b128((u32x4){lo[4 * r], lo[4 * r + 1], lo[4 * r + 2], lo[4 * r + 3]}, rs, lane * 16 + r * 1024, slot, AUX_SYS);
#pragma unroll
        for (int r = 0; r < 2; ++r) __builtin_amdgcn_raw_buffer_store_b128((u32x4){nib[4 * r], nib[4 * r + 1], nib[4 * r + 2], nib[4 * r + 3]}, rs, lane * 16 + r * 1024, slot + 4096, AUX_SYS);
    } else {
#pragma unroll
        for (int f = 0; f < 8; ++f) __builtin_amdgcn_raw_buffer_store_b128(v[f], rs, lane * 16 + (f & 3) * 1024, slot + (f >> 2) * 4096, AUX_SYS);
    }
}
constexpr unsigned PL_SLOT = 8192 + 256;
struct Puller {
    rsrc_t rp, rs; char* myws; int rank, src, q, lane, seen; unsigned call;
    __device__ __forceinline__ int xg(int i) const { return (i - 32 + x_rot(q) * 4) & 63; }
    __device__ __forceinline__ unsigned unit_id(int u) const { const int j = u / 96, i = u % 96; return i < 32 ? 512 + (rank * 8 + j) * 32 + i : j * 64 + xg(i); }
    __device__ __forceinline__ unsigned flag_of(int u) const { const int j = u / 96, i = u % 96; return OFF_FLAG + (src * 8 + j) * FLAG_REC + (i < 32 ? FLAG_W + i * 4 : xg(i) * 4); }
    __device__ __forceinline__ void issue(LAS unsigned char* b, int u) {
        const int j = u / 96, i = u % 96, need = j * 2 + (i >= 32);
        if (need > seen) {
            const unsigned long long t0 = __builtin_amdgcn_s_memrealtime();
            for (;;) {
                CFENCE;
                const unsigned f = __builtin_amdgcn_raw_buffer_load_b32(rs, 0, OFF_PFLG + (src * 8 + j) * 128 + (i >= 32 ? 0 : 4), AUX_SYS);
                if (__builtin_amdgcn_readfirstlane(f) >= call) break;
                if (spin_fail(myws, call, t0)) break;
            }
            CFENCE;
            seen = need;
        }
        const unsigned id = unit_id(u), slot = OFF_PX + id * UNIT;
        __builtin_amdgcn_raw_ptr_buffer_load_lds(rp, (LAS void*)(b + 8192), 4, (lane & 15) * 4, OFF_PH + id * 64, 0, AUX_SYS);
        __builtin_amdgcn_raw_ptr_buffer_load_lds(rp, (LAS void*)b, 16, lane * 16, slot, 0, AUX_SYS);
        __builtin_amdgcn_raw_ptr_buffer_load_lds(rp, (LAS void*)b, 16, lane * 16, slot, 1024, AUX_SYS);
        __builtin_amdgcn_raw_ptr_buffer_load_lds(rp, (LAS void*)b, 16, lane * 16, slot, 2048, AUX_SYS);
        __builtin_amdgcn_raw_ptr_buffer_load_lds(rp, (LAS void*)b, 16, lane * 16, slot, 3072, AUX_SYS);
        __builtin_amdgcn_raw_ptr_buffer_load_lds(rp, (LAS void*)(b + 4096), 16, lane * 16, slot + 4096, 0, AUX_SYS);
        __builtin_amdgcn_raw_ptr_buffer_load_lds(rp, (LAS void*)(b + 4096), 16, lane * 16, slot + 4096, 1024, AUX_SYS);
    }
    __device__ __forceinline__ void finish(LAS unsigned char* b, int u) {
        const int j = u / 96, i = u % 96, t = src * 8 + j;
        const bool isw = i < 32;
        const unsigned img = isw ? w_img(t, i) : x_img(t, xg(i));
        const unsigned h = *(const LAS unsigned*)(b + 8192 + lane * 4);
        u32x4 v[8];
#pragma unroll
        for (int r = 0; r < 6; ++r) v[r] = *(const LAS u32x4*)(b + r * 1024 + lane * 16);
        const unsigned meta = __builtin_amdgcn_readfirstlane(h), n = meta >> 8;
        if (n <= 15) {
            const unsigned base4 = (meta & 0x7fu) * 0x01010101u;
            u32x4 t_[6];
#pragma unroll
            for (int r = 0; r < 6; ++r) t_[r] = v[r];
#pragma unroll
            for (int f = 0; f < 8; ++f) {
                const unsigned nb = t_[4 + (f >> 2)][f & 3], la = t_[f >> 1][(f & 1) * 2], lb = t_[f >> 1][(f & 1) * 2 + 1];
                const unsigned ha = ((nb << 4) & 0x80808080u) | ((nb & 0x07070707u) + base4), hb = (nb & 0x80808080u) | (((nb >> 4) & 0x07070707u) + base4);
                v[f] = (u32x4){__builtin_amdgcn_perm(ha, la, 0x05010400u), __builtin_amdgcn_perm(ha, la, 0x07030602u), __builtin_amdgcn_perm(hb, lb, 0x05010400u), __builtin_amdgcn_perm(hb, lb, 0x07030602u)};
            }
        } else {
            const unsigned slot = OFF_PX + unit_id(u) * UNIT;
            __builtin_amdgcn_raw_ptr_buffer_load_lds(rp, (LAS void*)(b + 4096), 16, lane * 16, slot + 4096, 2048, AUX_SYS);
            __builtin_amdgcn_raw_ptr_buffer_load_lds(rp, (LAS void*)(b + 4096), 16, lane * 16, slot + 4096, 3072, AUX_SYS);
            WAIT_V(0);
#pragma unroll
            for (int r = 6; r < 8; ++r) v[r] = *(const LAS u32x4*)(b + r * 1024 + lane * 16);
        }
        img_store(v, rs, isw, img, lane);
        if (n - 1u < 15u && lane >= 1 && lane <= n) {
            const unsigned o = h >> 8;
            __builtin_amdgcn_raw_buffer_store_b8((unsigned char)h, rs, frag_off(isw, o >> 10) + (o & 1023u), img, LOC_AUX);
        }
    }
};
static_assert(OFF_PW == OFF_PX + 512 * UNIT, "unit ids run through the x and the w slots");

__device__ __forceinline__ void pull_role(const Params& p, LAS unsigned char* L, int pi, int lane, unsigned call, int wv) {
    Puller P; P.rank = p.rank; P.q = pi % 7; P.src = (p.rank + 1 + P.q) & 7; P.lane = lane; P.seen = -1; P.call = call;
    P.myws = (char*)p.ws[p.rank]; P.rs = mk_rsrc(P.myws, WS_USED); P.rp = mk_rsrc(p.ws[P.src], WS_USED);
    char* tick = P.myws + OFF_TICK + P.q * 128;
    const unsigned tbase = (call - 1) * (768 + 2 * PPP);
#define DRAW(r) { unsigned long long ex_; asm volatile("s_mov_b64 %1, exec\n\ts_mov_b64 exec, 1\n\tglobal_atomic_add %0, %2, %3, %4 sc0\n\ts_mov_b64 exec, %1" \
        : "=&v"(r), "=&s"(ex_) : "v"(0u), "v"(1u), "s"(tick) : "memory"); }
#define TK(x) (int)(__builtin_amdgcn_readfirstlane(x) - tbase)
    unsigned r0, r1, pf = 0;
    DRAW(r0); DRAW(r1);
    asm volatile("s_waitcnt vmcnt(0)" : "+v"(r0), "+v"(r1) :: "memory");
    int u0 = TK(r0), u1 = TK(r1), sl = 0;
    if ((unsigned)u0 < 768u) P.issue(L, u0);
    WAIT_V(0);
    while ((unsigned)u0 < 768u) {
        unsigned r2;
        if ((unsigned)u1 < 768u) { P.issue(L + (sl ^ 1) * PL_SLOT, u1); DRAW(r2); WAIT_V(8); }
        else { DRAW(r2); WAIT_V(0); }
        if (pf != 0 && lane == 0) st_sys(P.myws + pf, call);
        P.finish(L + sl * PL_SLOT, u0);
        asm volatile("s_waitcnt vmcnt(8)" : "+v"(r2) :: "memory");
        pf = P.flag_of(u0);
        u0 = u1; u1 = TK(r2); sl ^= 1;
    }
    WAIT_V(0);
    if (pf != 0 && lane == 0) st_sys(P.myws + pf, call);
    STAMP(P.myws, wv, 9);
}

__device__ __forceinline__ unsigned lut_expand(unsigned nib) { unsigned s = 0, c = 0;
#pragma unroll
    for (int i = 0; i < 4; ++i) { const unsigned b = (nib >> i) & 1u; s |= (b ? c : 0x0cu) << (8 * i); c += b; } return s; }
__device__ __forceinline__ unsigned lut_compact(unsigned nib) { unsigned s = 0x0c0c0c0cu, c = 0;
#pragma unroll
    for (int i = 0; i < 4; ++i) if ((nib >> i) & 1u) { s = (s & ~(0xffu << (8 * c))) | ((unsigned)i << (8 * c)); ++c; } return s; }
__device__ __forceinline__ unsigned popc4(const u32x4& m) { return __builtin_popcount(m[0]) + __builtin_popcount(m[1]) + __builtin_popcount(m[2]) + __builtin_popcount(m[3]); }
__device__ __forceinline__ unsigned nib4(unsigned m, int i) { return (i == 0 ? m << 2 : m >> (4 * i - 2)) & 0x3cu; }
__device__ __forceinline__ unsigned out_voff(int lane) { return (unsigned)(lane & 15) * N_ + (unsigned)(lane >> 4) * 16; }
__device__ __forceinline__ unsigned out_soff(int src, int wt) {
    const int tile = wt >> 3, w = wt & 7;
    return (unsigned)((tile >> 3) * 256 + (w >> 2) * 64) * N_ + (unsigned)(src * NS + (tile & 7) * 256 + (w & 3) * 64);
}
__device__ __forceinline__ constexpr unsigned chunk_soff(int ch) { return (unsigned)((ch >> 2) * 128 + (ch & 3) * 16) * N_; }

__device__ __forceinline__ void expand_role(const Params& p, LAS unsigned char* lds, LAS unsigned char* L, int ei, int lane, unsigned call, int wv) {
    const int rank = p.rank, q = ei % 7, k = ei / 7, src = (rank + 1 + q) & 7;
    char* myws = (char*)p.ws[rank];
    char* pws = (char*)p.ws[src];
    const rsrc_t rs = mk_rsrc(myws, WS_USED), rp = mk_rsrc(pws, WS_USED), ro = mk_rsrc(p.out[rank], OUT_BYTES);
    if (lane < 16) *(LAS unsigned*)(lds + EX_LUT + lane * 4) = lut_expand(lane);
    bool dead = false;
    unsigned long long rdy0 = 0, rdy1 = 0;
#define RDY(tile) (((((tile) & 1) ? rdy1 : rdy0) >> ((tile) >> 1)) & 1)
    for (int s = 0; k + s * PPO < 512; ++s) {
        const int pr = k + s * PPO;
        if (!RDY(pr >> 2) && !dead) {
            const unsigned long long t0 = __builtin_amdgcn_s_memrealtime();
            for (;;) {
                CFENCE;
                const u32x4 g = __builtin_amdgcn_raw_buffer_load_b128(rs, lane * 16, OFF_AMAX + src * 1024, AUX_SYS);
                rdy0 = __builtin_amdgcn_ballot_w64(g[1] >= call); rdy1 = __builtin_amdgcn_ballot_w64(g[3] >= call);
                if (RDY(pr >> 2)) break;
                if (spin_fail(myws, call, t0)) { dead = true; break; }
            }
            CFENCE;
        }
        __builtin_amdgcn_raw_ptr_buffer_load_lds(rp, (LAS void*)(L + EX_MSK + s * 2048), 16, lane * 16, OFF_MSK + pr * 2048, 0, AUX_SYS);
        __builtin_amdgcn_raw_ptr_buffer_load_lds(rp, (LAS void*)(L + EX_MSK + s * 2048), 16, lane * 16, OFF_MSK + pr * 2048, 1024, AUX_SYS);
        asm volatile("s_waitcnt vmcnt(%0)" :: "n"(MASK_W) : "memory");
    }
    WAIT_V(0);
    STAMP(myws, wv, 17);
    rdy0 = rdy1 = 0;
    const int nt = 2 * ((512 - k + PPO - 1) / PPO);
    u32x4 mn; unsigned offn, Ln;
#define EX_PREP(n) { mn = *(const LAS u32x4*)(L + EX_MSK + (n) * 1024 + lane * 16); const unsigned c_ = popc4(mn), s_ = wave_scan(c_); offn = s_ - c_; Ln = __builtin_amdgcn_readlane(s_, 63); }
#define EX_FETCH(n) { const int wt_ = (k + ((n) >> 1) * PPO) * 2 + ((n) & 1); \
        if (!RDY(wt_ >> 3) && !dead) { const unsigned long long t0 = __builtin_amdgcn_s_memrealtime(); \
            for (;;) { CFENCE; const u32x2 g = __builtin_amdgcn_raw_buffer_load_b64(rs, lane * 8, OFF_DONE + src * 512, AUX_SYS); \
                rdy0 = __builtin_amdgcn_ballot_w64(g[0] >= call); rdy1 = __builtin_amdgcn_ballot_w64(g[1] >= call); \
                if (RDY(wt_ >> 3)) break; if (spin_fail(myws, call, t0)) { dead = true; break; } } \
            CFENCE; } \
        const unsigned by_ = (((Ln + 31) >> 5) * 28 + 15) & ~15u; \
        const rsrc_t rc_ = mk_rsrc(pws + OFF_COD + (unsigned)wt_ * 8192, by_); \
        _Pragma("nounroll") for (unsigned r_ = 0; r_ < by_; r_ += 1024) \
            __builtin_amdgcn_raw_ptr_buffer_load_lds(rc_, (LAS void*)(L + EX_B + ((n) & 1) * 7168 + r_), 16, lane * 16, r_, 0, AUX_SYS); }
    EX_PREP(0); EX_FETCH(0);
    STAMP(myws, wv, 18);
    for (int n = 0; n < nt; ++n) {
        const u32x4 m = mn; unsigned off = offn; const unsigned Lc = Ln;
        if (n + 1 < nt) EX_PREP(n + 1);
        if (n == 0) WAIT_V(0); else WAIT_V(8);
        if (n + 1 < nt) EX_FETCH(n + 1);
        LAS unsigned char* B = L + EX_B + (n & 1) * 7168;
#pragma nounroll
        for (unsigned g0 = 0; g0 * 32 < Lc; g0 += 64) {
            const LAS unsigned* sB = (const LAS unsigned*)(B + (g0 + lane) * 28);
            unsigned o[7], d7 = 0;
#pragma unroll
            for (int i = 0; i < 7; ++i) o[i] = sB[i];
#pragma unroll
            for (int i = 0; i < 7; ++i) { d7 |= (o[i] & 0x80808080u) >> (7 - i); o[i] &= 0x7f7f7f7fu; }
            LAS u32x4* dA = (LAS u32x4*)(L + (g0 + lane) * 32);
            dA[0] = (u32x4){o[0], o[1], o[2], o[3]}; dA[1] = (u32x4){o[4], o[5], o[6], d7};
        }
        const int wt = (k + (n >> 1) * PPO) * 2 + (n & 1);
        const unsigned so = out_soff(src, wt), vo = out_voff(lane);
        unsigned sel[32], of[32], w0[32], w1[32];
        off += (unsigned)(size_t)L;
#pragma unroll
        for (int g = 0; g < 32; ++g) {
            const unsigned n4 = nib4(m[g >> 3], g & 7);
            sel[g] = *(const LAS unsigned*)(lds + EX_LUT + n4);
            const LAS unsigned* w = (const LAS unsigned*)(size_t)(off & ~3u);
            w0[g] = w[0]; w1[g] = w[1]; of[g] = off;
            off += __builtin_popcount(n4);
        }
        u32x4 o[8];
#pragma unroll
        for (int g = 0; g < 32; ++g) o[g >> 2][g & 3] = __builtin_amdgcn_perm(0u, __builtin_amdgcn_alignbyte(w1[g], w0[g], of[g]), sel[g]);
        if (dead) {
#pragma unroll
            for (int ch = 0; ch < 8; ++ch) o[ch] = (u32x4){0x7f7f7f7fu, 0x7f7f7f7fu, 0x7f7f7f7fu, 0x7f7f7f7fu};
            CFENCE;
        }
#pragma unroll
        for (int ch = 0; ch < 8; ++ch) __builtin_amdgcn_raw_buffer_store_b128(o[ch], ro, vo, so + chunk_soff(ch), OUT_AUX);
    }
    STAMP(myws, wv, 19);
}

__device__ __forceinline__ void comm_role(const Params& p, LAS unsigned char* lds, int c, int wid, int lane, unsigned call) {
    const int rank = p.rank;
    char* myws = (char*)p.ws[rank];
    const rsrc_t rs = mk_rsrc(myws, WS_USED);
    const int wv = (NGEMM + c) * 8 + wid; (void)wv;
    STAMP(myws, wv, 0);
    if (wid >= 2) {
        const rsrc_t rx = mk_rsrc(p.x, X_BYTES), rw = mk_rsrc(p.w, W_BYTES);
        for (int u = (wid - 2) * 128 + c; u < 2560; u += 768) {
            const int j = u / 320, r = u % 320, t = rank * 8 + j;
            u32x4 v[8];
            if (r >= 224 && r < 288) {
                const int g = r - 224;
                x_load(v, rx, j, g, lane);
                pack_store(v, rs, myws, OFF_PX + (unsigned)(j * 64 + g) * UNIT, OFF_PH + (unsigned)(j * 64 + g) * 64, lane);
                img_store(v, rs, false, x_img(t, g), lane);
                WAIT_V(0);
                unsigned old = 0;
                if (lane == 0) {
                    st_sys(myws + OFF_FLAG + t * FLAG_REC + g * 4, call);
                    old = __hip_atomic_fetch_add((GAS unsigned*)(myws + OFF_PCNT + j * 128), 1u, __ATOMIC_RELAXED, __HIP_MEMORY_SCOPE_AGENT);
                }
                if ((__builtin_amdgcn_readfirstlane(old) & 63) == 63 && lane < 8) st_sys((char*)p.ws[lane] + OFF_PFLG + t * 128, call);
            } else {
                const bool own = r >= 288;
                const int e = own ? r - 288 : r / 7, d = own ? rank : (rank + 1 + r % 7) & 7;
                u32x2 w[16];
                w_load(w, rw, j, d, e >> 2, e & 1, (e >> 1) & 1, lane);
                w_transpose(v, w);
                if (own) {
                    img_store(v, rs, true, w_img(t, e), lane);
                    WAIT_V(0);
                    if (lane == 0) st_sys(myws + OFF_FLAG + t * FLAG_REC + FLAG_W + e * 4, call);
                } else {
                    const unsigned id = (unsigned)((d * 8 + j) * 32 + e);
                    pack_store(v, rs, myws, OFF_PW + id * UNIT, OFF_PH + (512 + id) * 64, lane);
                    WAIT_V(0);
                    unsigned old = 0;
                    if (lane == 0) old = __hip_atomic_fetch_add((GAS unsigned*)(myws + OFF_PCNT + (8 + d * 8 + j) * 128), 1u, __ATOMIC_RELAXED, __HIP_MEMORY_SCOPE_AGENT);
                    if ((__builtin_amdgcn_readfirstlane(old) & 31) == 31 && lane == 0) st_sys((char*)p.ws[d] + OFF_PFLG + t * 128 + 4, call);
                }
            }
            STAMP(myws, wv, 1 + u / 768);
        }
    } else {
        if (c * 2 + wid < 7 * PPP) pull_role(p, lds + wid * EX_BYTES, c * 2 + wid, lane, call, wv);
        if (c * 2 + wid < 7 * PPO) expand_role(p, lds, lds + wid * EX_BYTES, c * 2 + wid, lane, call, wv);
    }
}

__device__ __forceinline__ void gemm_role(const Params& p, LAS unsigned char* lds, int rt, int ct, int wid, int lane, unsigned call) {
    const int rank = p.rank;
    char* myws = (char*)p.ws[rank];
    const int tile = rt * 8 + ct;
    const int wv = tile * 8 + wid; (void)wv;
    const int wr = wid >> 2, wc = wid & 3;
    const rsrc_t rs = mk_rsrc(myws, WS_USED);
    const unsigned voff = (unsigned)wid * 2048 + (unsigned)lane * 16;
    const unsigned voffF = lane < 4 ? (unsigned)(rt * 16 + lane * 4) : FLAG_W + (unsigned)(ct * 16 + (lane & 3) * 4);
    LAS unsigned char* ldsw = lds + wid * 2048;
    LAS unsigned char* lda = lds + wr * 8192 + lane * 16;
    LAS unsigned char* ldb = lds + wc * 4096 + lane * 16;
    const int seqv = step_of(lane, rank, rt);
#define SEQ(i) __builtin_amdgcn_readlane(seqv, (i) < NT ? (i) : NT - 1)
    STAMP(myws, wv, 0);
#define SA(b, h) (((b) * 2 + (h)) * HALFB)
#define SB(b, h) ((4 + (b) * 2 + (h)) * HALFB)
#define STAGE(bufoff, soff) do { \
        __builtin_amdgcn_raw_ptr_buffer_load_lds(rs, (LAS void*)(ldsw + (bufoff)), 16, voff, (soff), 0, GEMM_AUX); \
        __builtin_amdgcn_raw_ptr_buffer_load_lds(rs, (LAS void*)(ldsw + (bufoff)), 16, voff, (soff), 1024, GEMM_AUX); } while (0)
#define LDA(dst, b, h) do { _Pragma("unroll") for (int m = 0; m < 4; ++m) _Pragma("unroll") for (int k = 0; k < 2; ++k) dst[m][k] = *(const LAS bf16x8*)(lda + SA(b, h) + m * 2048 + k * 1024); } while (0)
#define LDB(dst, b, h) do { _Pragma("unroll") for (int n = 0; n < 2; ++n) _Pragma("unroll") for (int k = 0; k < 2; ++k) dst[n][k] = *(const LAS bf16x8*)(ldb + SB(b, h) + n * 2048 + k * 1024); } while (0)
#define MMA(ai, bj, At, Bt) do { __builtin_amdgcn_s_setprio(1); _Pragma("unroll") for (int m = 0; m < 4; ++m) _Pragma("unroll") for (int n = 0; n < 2; ++n) _Pragma("unroll") for (int k = 0; k < 2; ++k) \
        acc[ai][bj][m][n] = __builtin_amdgcn_mfma_f32_16x16x32_bf16(Bt[n][k], At[m][k], acc[ai][bj][m][n], 0, 0, 0); __builtin_amdgcn_s_setprio(0); } while (0)
#define OFFA(t) (OFF_XT + (unsigned)(t) * XT_STEP + (unsigned)rt * BLK)
#define OFFB(t) (OFF_WT + (unsigned)(t) * WT_STEP + (unsigned)ct * BLK)
#define SLOW_WAIT(t) do { const unsigned long long t0_ = __builtin_amdgcn_s_memrealtime(); \
        for (;;) { CFENCE; const unsigned f_ = __builtin_amdgcn_raw_buffer_load_b32(rs, voffF, OFF_FLAG + (unsigned)(t) * FLAG_REC, AUX_SYS); \
            if (__builtin_amdgcn_ballot_w64(f_ < call) == 0) break; if (spin_fail(myws, call, t0_)) break; } } while (0)
#define CHECK(t, s, tn) do { if (wid == 0) { const unsigned f0_ = *(volatile LAS unsigned*)(lds + LDS_FLAG + (s) * 256 + lane * 4); \
            if (__builtin_amdgcn_ballot_w64(f0_ < call) != 0) SLOW_WAIT(t); \
            __builtin_amdgcn_raw_ptr_buffer_load_lds(rs, (LAS void*)(lds + LDS_FLAG + ((s) ^ 1) * 256), 4, voffF, OFF_FLAG + (unsigned)(tn) * FLAG_REC, 0, AUX_SYS); } } while (0)

    f32x4 acc[2][2][4][2];
#pragma unroll
    for (int a = 0; a < 2; ++a)
#pragma unroll
        for (int b = 0; b < 2; ++b)
#pragma unroll
            for (int m = 0; m < 4; ++m)
#pragma unroll
                for (int n = 0; n < 2; ++n) acc[a][b][m][n] = (f32x4){0.f, 0.f, 0.f, 0.f};
    bf16x8 At[4][2], B0[2][2], B1[2][2];

    {
        const int t0 = SEQ(0), t1 = SEQ(1), t2 = SEQ(2);
        if (wid == 0) {
            SLOW_WAIT(t0); SLOW_WAIT(t1);
            __builtin_amdgcn_raw_ptr_buffer_load_lds(rs, (LAS void*)(lds + LDS_FLAG), 4, voffF, OFF_FLAG + (unsigned)t2 * FLAG_REC, 0, AUX_SYS);
        }
        CFENCE; BAR; CFENCE;
        STAMP(myws, wv, 1);
        STAGE(SB(0, 0), OFFB(t0)); STAGE(SA(0, 0), OFFA(t0)); STAGE(SB(0, 1), OFFB(t0) + HALFB); STAGE(SA(0, 1), OFFA(t0) + HALFB);
        if (wr == 1) BAR;
        WAIT_V(4); BAR;
        STAGE(SB(1, 0), OFFB(t1)); STAGE(SA(1, 0), OFFA(t1)); STAGE(SB(1, 1), OFFB(t1) + HALFB);
        WAIT_V(6); BAR;
    }
    for (int i = 0; i < NT; i += 2) {
        const int t1 = SEQ(i + 1), t2 = SEQ(i + 2), t3 = SEQ(i + 3), t4 = SEQ(i + 4);
        const unsigned a1 = OFFA(t1), a2 = OFFA(t2), b2 = OFFB(t2), a3 = OFFA(t3), b3 = OFFB(t3);
        CHECK(t2, 0, t3);
        LDB(B0, 0, 0); SCHED; LDA(At, 0, 0); STAGE(SA(1, 1), a1 + HALFB);
        WAIT_L(8); BAR; WAIT_L(0); MMA(0, 0, At, B0); BAR; SCHED;
        LDB(B1, 0, 1); STAGE(SB(0, 0), b2);
        BAR; WAIT_L(0); MMA(0, 1, At, B1); BAR;
        LDA(At, 0, 1); STAGE(SA(0, 0), a2);
        BAR; WAIT_L(0); MMA(1, 0, At, B0); BAR; SCHED;
        STAGE(SB(0, 1), b2 + HALFB);
        WAIT_V(6); BAR; MMA(1, 1, At, B1); BAR;
        CHECK(t3, 1, t4);
        LDB(B0, 1, 0); SCHED; LDA(At, 1, 0); STAGE(SA(0, 1), a2 + HALFB);
        WAIT_L(8); BAR; WAIT_L(0); MMA(0, 0, At, B0); BAR; SCHED;
        LDB(B1, 1, 1); STAGE(SB(1, 0), b3);
        BAR; WAIT_L(0); MMA(0, 1, At, B1); BAR;
        LDA(At, 1, 1); STAGE(SA(1, 0), a3);
        BAR; WAIT_L(0); MMA(1, 0, At, B0); BAR; SCHED;
        STAGE(SB(1, 1), b3 + HALFB);
        WAIT_V(6); BAR; MMA(1, 1, At, B1); BAR;
    }
    WAIT_V(0);
    if (wr == 0) BAR;
    BAR;
    STAMP(myws, wv, 10);

    const int wt = tile * 8 + wid;
    u32x4 msk;
    float mx = 0.f;
#pragma unroll
    for (int w = 0; w < 4; ++w) {
        unsigned mm = 0;
#pragma unroll
        for (int e = 31; e >= 0; --e) {
            const int ch = 2 * w + (e >> 4);
            const float v = acc[ch >> 2][(e >> 1) & 1][ch & 3][e & 1][(e >> 2) & 3];
            mm = __builtin_amdgcn_alignbit(mm, __float_as_uint(v), 31);
            mx = fmaxf(mx, v);
        }
        msk[w] = ~mm;
    }
    __builtin_amdgcn_raw_buffer_store_b128(msk, rs, lane * 16, OFF_MSK + wt * 1024, AUX_SYS);
    LAS unsigned* red = (LAS unsigned*)(lds + LDS_RED);
    { const unsigned wm = wave_max(__float_as_uint(mx)); if (lane == 0) red[wid] = wm; }
    WAIT_V(0); WAIT_L(0); BAR; CFENCE;
    if (wid == 0) {
        const unsigned t = wave_max(red[lane & 7]);
        if (lane < 8) {
            GAS unsigned long long* g = (GAS unsigned long long*)((char*)p.ws[lane] + OFF_AMAX + (rank * NGEMM + tile) * 8);
            __hip_atomic_store(g, ((unsigned long long)call << 32) | t, __ATOMIC_RELAXED, __HIP_MEMORY_SCOPE_SYSTEM);
        }
    }
    LAS unsigned char* A = lds + wid * 16384;
    LAS unsigned char* B = A + 8192;
    const unsigned cnt = popc4(msk), scan = wave_scan(cnt), Lc = __builtin_amdgcn_readlane(scan, 63);
    unsigned off = scan - cnt;
#pragma unroll
    for (int r = 0; r < 8; ++r) *(LAS u32x4*)(A + r * 1024 + lane * 16) = (u32x4){0u, 0u, 0u, 0u};
    LAS unsigned char* lut = A + 16384 - 64;
    if (lane < 16) *(LAS unsigned*)(lut + lane * 4) = lut_compact(lane);
    unsigned sel[32], of[32];
    off += (unsigned)(size_t)A;
#pragma unroll
    for (int g = 0; g < 32; ++g) {
        const unsigned n4 = nib4(msk[g >> 3], g & 7);
        sel[g] = *(const LAS unsigned*)(lut + n4); of[g] = off;
        off += __builtin_popcount(n4);
    }
    {
        const unsigned long long t0 = __builtin_amdgcn_s_memrealtime();
        unsigned t;
        for (;;) {
            CFENCE;
            const u32x4 g = __builtin_amdgcn_raw_buffer_load_b128(rs, lane * 16, OFF_AMAX + wid * 1024, AUX_SYS);
            t = max(g[0], g[2]);
            if (__builtin_amdgcn_ballot_w64(g[1] < call || g[3] < call) == 0) break;
            if (spin_fail(myws, call, t0)) break;
        }
        t = wave_max(t);
        if (lane == 0) red[8 + wid] = t;
    }
    WAIT_L(0); BAR; CFENCE;
    const float amax = __uint_as_float(wave_max(red[8 + (lane & 7)]));
    const bool dead = __builtin_amdgcn_readfirstlane(ld_agent(myws + OFF_TMO)) == call;
    const float inv = 448.0f / amax;
    STAMP(myws, wv, 11);

    u32x4 qv[8];
#pragma unroll
    for (int ch = 0; ch < 8; ++ch) {
        const int a = ch >> 2, m = ch & 3;
#pragma unroll
        for (int j = 0; j < 4; ++j) {
            int w = __builtin_amdgcn_cvt_pk_fp8_f32(fmaxf(acc[a][0][m][0][j], 0.f) * inv, fmaxf(acc[a][0][m][1][j], 0.f) * inv, 0, false);
            w = __builtin_amdgcn_cvt_pk_fp8_f32(fmaxf(acc[a][1][m][0][j], 0.f) * inv, fmaxf(acc[a][1][m][1][j], 0.f) * inv, w, true);
            const unsigned c = dead ? 0x7f7f7f7fu : (unsigned)w & 0x7f7f7f7fu;
            qv[ch][j] = c;
            const unsigned o_ = of[ch * 4 + j];
            const unsigned long long z = (unsigned long long)__builtin_amdgcn_perm(0u, c, sel[ch * 4 + j]) << ((o_ & 3u) * 8);
            LAS unsigned* d = (LAS unsigned*)(size_t)(o_ & ~3u);
            __hip_atomic_fetch_or(d, (unsigned)z, __ATOMIC_RELAXED, __HIP_MEMORY_SCOPE_WAVEFRONT);
            __hip_atomic_fetch_or(d + 1, (unsigned)(z >> 32), __ATOMIC_RELAXED, __HIP_MEMORY_SCOPE_WAVEFRONT);
        }
    }
    CFENCE;
#pragma nounroll
    for (unsigned g0 = 0; g0 * 32 < Lc; g0 += 64) {
        const LAS u32x4* sA = (const LAS u32x4*)(A + (g0 + lane) * 32);
        const u32x4 d0 = sA[0], d1 = sA[1];
        LAS unsigned* dB = (LAS unsigned*)(B + (g0 + lane) * 28);
#pragma unroll
        for (int i = 0; i < 7; ++i) dB[i] = (i < 4 ? d0[i & 3] : d1[i & 3]) | ((d1[3] << (7 - i)) & 0x80808080u);
    }
    CFENCE;
#pragma nounroll
    for (unsigned r = 0; r < ((Lc + 31) >> 5) * 28; r += 1024)
        __builtin_amdgcn_raw_buffer_store_b128(*(const LAS u32x4*)(B + r + lane * 16), rs, lane * 16, OFF_COD + wt * 8192 + r, AUX_SYS);
    WAIT_V(0); BAR; CFENCE;
    if (wid == 0 && lane < 8 && lane != rank) st_sys((char*)p.ws[lane] + OFF_DONE + (rank * NGEMM + tile) * 4, call);
    STAMP(myws, wv, 12);
    {
        const rsrc_t ro = mk_rsrc(p.out[rank], OUT_BYTES);
        const unsigned so = out_soff(rank, wt), vo = out_voff(lane);
#pragma unroll
        for (int ch = 0; ch < 8; ++ch) __builtin_amdgcn_raw_buffer_store_b128(qv[ch], ro, vo, so + chunk_soff(ch), OUT_AUX);
    }
}

__global__ __launch_bounds__(512, 2) void fused_kernel(Params p) {
    __shared__ __attribute__((aligned(16))) unsigned char lds_arr[LDS_BYTES];
    LAS unsigned char* lds = (LAS unsigned char*)lds_arr;
    const int wid = __builtin_amdgcn_readfirstlane(threadIdx.x >> 6), lane = threadIdx.x & 63;
    const int b = blockIdx.x, xcd = b & 7, idx = b >> 3;
    unsigned call = 0;
    if (lane == 0) call = __hip_atomic_fetch_add((GAS unsigned*)((char*)p.ws[p.rank] + OFF_CNT) + b * 8 + wid, 1u, __ATOMIC_RELAXED, __HIP_MEMORY_SCOPE_AGENT) + 1u;
    call = __builtin_amdgcn_readfirstlane(call);
    if (idx < 16) gemm_role(p, lds, (xcd >> 1) * 4 + (idx >> 2), (xcd & 1) * 4 + (idx & 3), wid, lane, call);
    else comm_role(p, lds, (idx - 16) * 8 + xcd, wid, lane, call);
}

extern "C" void kernel_launch(int rank, int nranks, ncclComm_t comm, hipStream_t stream,
                              void* const* d_in, const int* in_sizes, int n_in,
                              void* d_out, int out_size,
                              const int* in_dtypes, int out_dtype,
                              void* const* d_out_all, void* const* d_ws_all, size_t ws_bytes) {
    (void)comm; (void)in_dtypes; (void)out_dtype; (void)d_out;
    if (nranks != NR || n_in != 2 || in_sizes[0] != M_ * KS || in_sizes[1] != KS * N_ || out_size != M_ * N_ || ws_bytes < WS_USED || (unsigned)rank >= (unsigned)NR) {
        fprintf(stderr, "kernel_launch: unexpected problem shape\n");
        return;
    }
    Params p;
    p.x = d_in[0]; p.w = d_in[1];
    for (int i = 0; i < NR; ++i) { p.out[i] = d_out_all[i]; p.ws[i] = d_ws_all[i]; }
    p.rank = rank; p.pad = 0;
    fused_kernel<<<dim3(NWG), dim3(512), 0, stream>>>(p);
}
```

```cpp
#include <hip/hip_runtime.h>
#include <rccl/rccl.h>
#include <cstdio>

#define LAS __attribute__((address_space(3)))
#define GAS __attribute__((address_space(1)))
typedef short bf16x8 __attribute__((ext_vector_type(8)));
typedef float f32x4 __attribute__((ext_vector_type(4)));
typedef unsigned u32x4 __attribute__((ext_vector_type(4)));
typedef unsigned u32x2 __attribute__((ext_vector_type(2)));
typedef __amdgpu_buffer_rsrc_t rsrc_t;

constexpr int NR = 8;
constexpr int M_ = 4096, N_ = 16384;
constexpr int KS = 512;
constexpr int NS = 2048;
constexpr int NT = 64;
constexpr int NGEMM = 128;
constexpr int NWG = 256;

constexpr unsigned OFF_CNT  = 0;
constexpr unsigned OFF_TMO  = 16384;
constexpr unsigned OFF_FLAG = 65536;
constexpr unsigned FLAG_REC = 1024, FLAG_W = 512;
constexpr unsigned OFF_AMAX = 131072;
constexpr unsigned OFF_TICK = 143360;
constexpr unsigned OFF_PCNT = 147456;
constexpr unsigned OFF_PFLG = 163840;
constexpr unsigned OFF_DONE = 786432;
constexpr unsigned OFF_MF   = OFF_DONE + 65536;
constexpr unsigned OFF_DBG  = 262144;
constexpr unsigned CTRL_BYTES = 1u << 20;
constexpr unsigned XT_STEP = 512u * 1024u, WT_STEP = 256u * 1024u, BLK = 32768u, HALFB = 16384u;
constexpr unsigned OFF_XT = CTRL_BYTES;
constexpr unsigned XT_BYTES = NT * XT_STEP;
constexpr unsigned OFF_WT = OFF_XT + XT_BYTES;
constexpr unsigned WT_BYTES = NT * WT_STEP;
constexpr unsigned UNIT = 9216;
constexpr unsigned OFF_PX = OFF_WT + WT_BYTES;
constexpr unsigned OFF_PW = OFF_PX + 512 * UNIT;
constexpr unsigned OFF_MSK = OFF_PW + 2048 * UNIT;
constexpr unsigned OFF_COD = OFF_MSK + 1024 * 1024;
constexpr unsigned OFF_LM = OFF_COD + 1024 * 8192;
constexpr unsigned WS_USED = OFF_LM + 8 * 1024 * 1024;
constexpr unsigned OUT_BYTES = (unsigned)M_ * N_;
constexpr unsigned X_BYTES = (unsigned)M_ * KS * 2, W_BYTES = (unsigned)KS * N_ * 2;

constexpr unsigned long long DEADLINE = 100000000ull;
constexpr int AUX_SYS = 17;
#define GEMM_AUX 17
#define PPP 14
#define PPO 40
#define PPM 17
#define SCH_A 1
#define SCH_B 60
#define CW 3
#define OUT_AUX 0
#define LOC_AUX 17

constexpr unsigned LDS_STAGE = 8 * HALFB;
constexpr unsigned LDS_FLAG = LDS_STAGE;
constexpr unsigned LDS_RED = LDS_FLAG + 512;
constexpr unsigned EX_B = 8192, EX_M = EX_B + 8192, EX_P = EX_M + 1024, EX_E = EX_P + 1024, EX_BYTES = EX_E + 256, EX_LUT = 0;
constexpr unsigned LDS_BYTES = 2048 + 8 * EX_BYTES;
static_assert(LDS_RED + 128 <= LDS_BYTES && LDS_BYTES <= 163840, "lds");
static_assert(7 * PPO <= 128 * CW && 7 * PPM <= 128 * CW && PPP <= PPM, "waves");

struct Params {
    const void* x; const void* w;
    void* out[NR]; void* ws[NR];
    int rank; int pad;
};

__device__ __forceinline__ rsrc_t mk_rsrc(const void* p, unsigned bytes) {
    return __builtin_amdgcn_make_buffer_rsrc((void*)p, 0, bytes, 0x00020000);
}
#define WAIT_V(n) asm volatile("s_waitcnt vmcnt(" #n ")" ::: "memory")
#define WAIT_L(n) asm volatile("s_waitcnt lgkmcnt(" #n ")" ::: "memory")
#define BAR __builtin_amdgcn_s_barrier()
#define SCHED __builtin_amdgcn_sched_barrier(0)
#define CFENCE asm volatile("" ::: "memory")

#define STAMP(ws_, wv_, k_) do {} while (0)
#define CYC(i) do {} while (0)

__device__ __forceinline__ unsigned ld_agent(const void* p) { return __hip_atomic_load((const GAS unsigned*)p, __ATOMIC_RELAXED, __HIP_MEMORY_SCOPE_AGENT); }
__device__ __forceinline__ void st_agent(void* p, unsigned v) { __hip_atomic_store((GAS unsigned*)p, v, __ATOMIC_RELAXED, __HIP_MEMORY_SCOPE_AGENT); }
__device__ __forceinline__ void st_sys(void* p, unsigned v) { __hip_atomic_store((GAS unsigned*)p, v, __ATOMIC_RELAXED, __HIP_MEMORY_SCOPE_SYSTEM); }

__device__ __forceinline__ bool spin_fail(char* myws, unsigned call, unsigned long long t0) {
    if (__builtin_amdgcn_readfirstlane(ld_agent(myws + OFF_TMO)) == call || __builtin_amdgcn_s_memrealtime() - t0 > DEADLINE) {
        st_agent(myws + OFF_TMO, call);
        return true;
    }
    __builtin_amdgcn_s_sleep(2);
    return false;
}

#define DPP0(x, ctrl, rm) (unsigned)__builtin_amdgcn_update_dpp(0, (int)(x), ctrl, rm, 0xf, false)
__device__ __forceinline__ unsigned wave_scan(unsigned x) {
    x += DPP0(x, 0x111, 0xf); x += DPP0(x, 0x112, 0xf); x += DPP0(x, 0x114, 0xf); x += DPP0(x, 0x118, 0xf);
    x += DPP0(x, 0x142, 0xa); x += DPP0(x, 0x143, 0xc);
    return x;
}
__device__ __forceinline__ unsigned wave_max(unsigned x) {
    x = max(x, DPP0(x, 0x111, 0xf)); x = max(x, DPP0(x, 0x112, 0xf)); x = max(x, DPP0(x, 0x114, 0xf)); x = max(x, DPP0(x, 0x118, 0xf));
    x = max(x, DPP0(x, 0x142, 0xa)); x = max(x, DPP0(x, 0x143, 0xc));
    return __builtin_amdgcn_readlane(x, 63);
}

struct Sched {
    unsigned w[7][64];
    constexpr Sched() : w{} {
        for (int q = 0; q < 7; ++q) {
            int key[136] = {}, id[136] = {}, n = 0;
            for (int j = 0; j < 8; ++j) {
                int lo = 1 << 30;
                for (int rt = 0; rt < 16; ++rt) { const int k = SCH_B * rt + 106 * (7 * j + (q + SCH_A * rt) % 7); key[n] = 2 * k + 1; id[n++] = j * 24 + 8 + rt; lo = k < lo ? k : lo; }
                key[n] = 2 * lo; id[n++] = -1 - j;
            }
            int at = 0;
            for (int o = 0; o < 136; ++o) {
                int best = -1;
                for (int i = 0; i < 136; ++i) if (key[i] >= 0 && (best < 0 || key[i] < key[best])) best = i;
                key[best] = -1;
                const int cnt = id[best] < 0 ? 8 : 1, first = id[best] < 0 ? (-1 - id[best]) * 24 : id[best];
                for (int c = 0; c < cnt; ++c, ++at) w[q][at >> 2] |= (unsigned)(first + c) << (8 * (at & 3));
            }
        }
    }
};
__constant__ const Sched SCHED_TAB;
__device__ __forceinline__ int step_of(int i, int rank, int rt) {
    if (i < 8) return rank * 8 + i;
    const int z = i - 8, j = z / 7, p = z - j * 7, q = (p + 7 * 16 * SCH_A - SCH_A * rt) % 7;
    return (((rank + 1 + q) & 7) << 3) + j;
}

__device__ __forceinline__ void x_load(u32x4 (&v)[8], rsrc_t rx, int j, int g, int lane) {
    const unsigned voff = (unsigned)(lane & 15) * (KS * 2) + (unsigned)(lane >> 4) * 16;
    const unsigned soff = (unsigned)g * 64 * (KS * 2) + (unsigned)j * 128;
#pragma unroll
    for (int r = 0; r < 4; ++r)
#pragma unroll
        for (int kk = 0; kk < 2; ++kk) v[r * 2 + kk] = __builtin_amdgcn_raw_buffer_load_b128(rx, voff + kk * 64, soff + r * 16 * (KS * 2), 0);
}
__device__ __forceinline__ void w_load(u32x2 (&v)[16], rsrc_t rw, int j, int d, int ct, int kk, int G, int lane) {
    const unsigned voff = (unsigned)(lane >> 4) * 8 * (N_ * 2) + (unsigned)(lane & 15) * 8;
    const unsigned soff = (unsigned)(j * 64 + kk * 32) * (N_ * 2) + (unsigned)(d * NS + ct * 256 + G * 128) * 2;
#pragma unroll
    for (int jj = 0; jj < 8; ++jj)
#pragma unroll
        for (int g = 0; g < 2; ++g) v[g * 8 + jj] = __builtin_amdgcn_raw_buffer_load_b64(rw, voff + g * 128, soff + jj * (N_ * 2), 0);
}
__device__ __forceinline__ void w_transpose(u32x4 (&o)[8], const u32x2 (&v)[16]) {
#pragma unroll
    for (int g = 0; g < 2; ++g)
#pragma unroll
        for (int c = 0; c < 4; ++c)
#pragma unroll
            for (int dd = 0; dd < 4; ++dd)
                o[g * 4 + c][dd] = __builtin_amdgcn_perm(v[g * 8 + 2 * dd + 1][c >> 1], v[g * 8 + 2 * dd][c >> 1], (c & 1) ? 0x07060302u : 0x05040100u);
}
__device__ __forceinline__ unsigned x_img(int t, int g) { return OFF_XT + (unsigned)t * XT_STEP + (unsigned)g * 8192; }
__device__ __forceinline__ unsigned w_img(int t, int e) { return OFF_WT + (unsigned)t * WT_STEP + (unsigned)(e >> 2) * BLK + (unsigned)((e >> 1) & 1) * 8192 + (unsigned)(e & 1) * 1024; }
__device__ __forceinline__ unsigned frag_off(bool isw, unsigned f) { return isw ? ((f >> 1) & 1) * HALFB + (f >> 2) * 4096 + (f & 1) * 2048 : f * 1024; }
__device__ __forceinline__ void img_store(const u32x4 (&v)[8], rsrc_t rd, bool isw, unsigned img, int lane) {
#pragma unroll
    for (int f = 0; f < 8; ++f) __builtin_amdgcn_raw_buffer_store_b128(v[f], rd, lane * 16, img + frag_off(isw, f), LOC_AUX);
}
constexpr unsigned H8 = 0x80808080u, MAXESC = 512;
__device__ __forceinline__ unsigned bits16(unsigned m0, unsigned m1, unsigned m2, unsigned m3, int sh) {
    const unsigned a = __builtin_amdgcn_udot4(m0, 0x08040201u, __builtin_amdgcn_udot4(m1, 0x80402010u, 0u, false), false);
    const unsigned b = __builtin_amdgcn_udot4(m2, 0x08040201u, __builtin_amdgcn_udot4(m3, 0x80402010u, 0u, false), false);
    return (a >> sh) | (b << (8 - sh));
}
__device__ __forceinline__ unsigned esc_count(const unsigned (&bm)[4], unsigned& first) {
    const unsigned c = __builtin_popcount(bm[0]) + __builtin_popcount(bm[1]) + __builtin_popcount(bm[2]) + __builtin_popcount(bm[3]), s = wave_scan(c);
    first = s - c;
    return __builtin_amdgcn_readlane(s, 63);
}
__device__ __forceinline__ void esc_list(const unsigned (&bm)[4], unsigned at, LAS unsigned char* list, int lane) {
#pragma unroll
    for (int w = 0; w < 2; ++w) {
        unsigned long long z = ((unsigned long long)bm[2 * w + 1] << 32) | bm[2 * w];
        while (__builtin_amdgcn_ballot_w64(z != 0) != 0) {
            if (z != 0) {
                const unsigned b = __builtin_ctzll(z);
                *(LAS unsigned short*)(list + min(at, MAXESC - 1) * 2) = (unsigned short)(w * 4096 + (b >> 4) * 1024 + lane * 16 + (b & 15));
                ++at; z &= z - 1;
            }
        }
    }
}
constexpr unsigned PK_HDR = 5 * 1024 + 512, PK_RAW = 0xffffu, PK_FIRST = 62;
__device__ __forceinline__ void pack_store(const u32x4 (&v)[8], rsrc_t rs, LAS unsigned char* L, unsigned slot, int lane) {
    unsigned p0[16], e[16], sum = 0;
#pragma unroll
    for (int r = 0; r < 16; ++r) {
        const unsigned lo = __builtin_amdgcn_perm(v[r >> 1][2 * (r & 1) + 1], v[r >> 1][2 * (r & 1)], 0x06040200u);
        const unsigned hi = __builtin_amdgcn_perm(v[r >> 1][2 * (r & 1) + 1], v[r >> 1][2 * (r & 1)], 0x07050301u);
        p0[r] = (hi & H8) | (lo & ~H8);
        e[r] = ((hi << 1) & 0xfefefefeu) | ((lo >> 7) & 0x01010101u);
        sum = __builtin_amdgcn_sad_u8(e[r], 0u, sum);
    }
#pragma unroll
    for (int r = 0; r < 4; ++r) *(LAS u32x4*)(L + r * 1024 + lane * 16) = (u32x4){e[4 * r], e[4 * r + 1], e[4 * r + 2], e[4 * r + 3]};
    const int tot = (int)__builtin_amdgcn_readlane(wave_scan(sum), 63);
    const unsigned base = (unsigned)min(max((tot - 16794) >> 12, 0), 247), base4 = base * 0x01010101u;
    u32x4 p2 = {0u, 0u, 0u, 0u}; u32x2 p1 = {0u, 0u};
    unsigned ms[16];
#pragma unroll
    for (int r = 0; r < 16; ++r) {
        const unsigned z = ((e[r] | H8) - (base4 & ~H8)) ^ ((e[r] ^ ~base4) & H8);
        ms[r] = (((z & 0x78787878u) + 0x78787878u) | z) & H8;
        p2[r >> 2] |= (z & 0x03030303u) << (2 * (r & 3));
        p1[r >> 3] |= (r & 7) >= 2 ? (z & 0x04040404u) << ((r & 7) - 2) : (z & 0x04040404u) >> (2 - (r & 7));
    }
    unsigned bm[4] = {bits16(ms[0], ms[1], ms[2], ms[3], 7) | (bits16(ms[4], ms[5], ms[6], ms[7], 7) << 16),
                      bits16(ms[8], ms[9], ms[10], ms[11], 7) | (bits16(ms[12], ms[13], ms[14], ms[15], 7) << 16), 0u, 0u};
    unsigned at;
    const unsigned n = esc_count(bm, at);
    bool bad = n > 254;
    *(LAS u32x2*)(L + 4096 + lane * 8) = p1;
    if (!bad) {
        esc_list(bm, at, L + 5120, lane);
        CFENCE;
        bool b = false;
#pragma nounroll
        for (unsigned i = lane; i < n; i += 64) {
            const unsigned pos = *(const LAS unsigned short*)(L + 5120 + i * 2), ev = L[pos];
            unsigned d = base - 1 - ev;
            if (ev == base + 8) d = 14; else if (ev == 0) d = 15; else b |= d > 13;
            *(LAS unsigned short*)(L + 4096 + 516 + i * 2) = (unsigned short)(pos | (d << 12));
        }
        bad = __builtin_amdgcn_ballot_w64(b) != 0;
    }
    if (lane == 0) *(LAS unsigned*)(L + 4096 + 512) = base | ((bad ? PK_RAW : n) << 8);
    CFENCE;
    __builtin_amdgcn_raw_buffer_store_b128(*(const LAS u32x4*)(L + 4096 + lane * 16), rs, lane * 16 + 1024, slot + 4096, AUX_SYS);
    if (!bad) {
#pragma unroll
        for (int r = 0; r < 4; ++r) __builtin_amdgcn_raw_buffer_store_b128((u32x4){p0[4 * r], p0[4 * r + 1], p0[4 * r + 2], p0[4 * r + 3]}, rs, lane * 16 + r * 1024, slot, AUX_SYS);
        __builtin_amdgcn_raw_buffer_store_b128(p2, rs, lane * 16, slot + 4096, AUX_SYS);
    } else {
#pragma unroll
        for (int f = 0; f < 8; ++f) __builtin_amdgcn_raw_buffer_store_b128(v[f], rs, lane * 16, slot + (f < 5 ? f : f + 1) * 1024, AUX_SYS);
    }
}
constexpr unsigned PL_SLOT = 8192;
struct Puller {
    rsrc_t rp, rs; char* myws; int rank, src, q, lane; unsigned call, seen, tab;
    __device__ __forceinline__ int unit_of(int tk) const { return (int)((__builtin_amdgcn_readlane(tab, tk >> 4) >> (2 * (tk & 12))) & 0xffu) * 4 + (tk & 3); }
    __device__ __forceinline__ int xg(int i) const { return i - 32; }
    __device__ __forceinline__ unsigned unit_id(int u) const { const int j = u / 96, i = u % 96; return i < 32 ? 512 + (rank * 8 + j) * 32 + i : j * 64 + xg(i); }
    __device__ __forceinline__ unsigned flag_of(int u) const { const int j = u / 96, i = u % 96; return OFF_FLAG + (src * 8 + j) * FLAG_REC + (i < 32 ? FLAG_W + i * 4 : xg(i) * 4); }
    __device__ __forceinline__ void issue(LAS unsigned char* b, int u) {
        const int j = u / 96, i = u % 96, need = j * 2 + (i >= 32);
        if (!((seen >> need) & 1u)) {
            const unsigned long long t0 = __builtin_amdgcn_s_memrealtime();
            for (;;) {
                CFENCE;
                const unsigned f = __builtin_amdgcn_raw_buffer_load_b32(rs, 0, OFF_PFLG + (src * 8 + j) * 128 + (i >= 32 ? 0 : 4), AUX_SYS);
                if (__builtin_amdgcn_readfirstlane(f) >= call) break;
                if (spin_fail(myws, call, t0)) break;
            }
            CFENCE;
            seen |= 1u << need;
        }
        const unsigned id = unit_id(u), slot = OFF_PX + id * UNIT;
        __builtin_amdgcn_raw_ptr_buffer_load_lds(rp, (LAS void*)(b + 4096), 16, lane < 40 ? lane * 16 : 0xf0000000u, slot + 4096, 1024, AUX_SYS);
        __builtin_amdgcn_raw_ptr_buffer_load_lds(rp, (LAS void*)(b + 4096), 16, lane * 16, slot + 4096, 0, AUX_SYS);
        __builtin_amdgcn_raw_ptr_buffer_load_lds(rp, (LAS void*)b, 16, lane * 16, slot, 0, AUX_SYS);
        __builtin_amdgcn_raw_ptr_buffer_load_lds(rp, (LAS void*)b, 16, lane * 16, slot, 1024, AUX_SYS);
        __builtin_amdgcn_raw_ptr_buffer_load_lds(rp, (LAS void*)b, 16, lane * 16, slot, 2048, AUX_SYS);
        __builtin_amdgcn_raw_ptr_buffer_load_lds(rp, (LAS void*)b, 16, lane * 16, slot, 3072, AUX_SYS);
    }
    __device__ __forceinline__ void finish(LAS unsigned char* b, int u) {
        const int j = u / 96, i = u % 96, t = src * 8 + j;
        const bool isw = i < 32;
        const unsigned img = isw ? w_img(t, i) : x_img(t, xg(i));
        const unsigned h = *(const LAS unsigned*)(b + PK_HDR);
        u32x4 v[8];
#pragma unroll
        for (int r = 0; r < 5; ++r) v[r] = *(const LAS u32x4*)(b + r * 1024 + lane * 16);
        const u32x2 p1 = *(const LAS u32x2*)(b + 5120 + lane * 8);
        const unsigned meta = __builtin_amdgcn_readfirstlane(h), n = meta >> 8, base = meta & 0xffu;
        if (n <= 254) {
            if (n > PK_FIRST) {
                __builtin_amdgcn_raw_ptr_buffer_load_lds(rp, (LAS void*)(b + 4096), 16, lane * 16, OFF_PX + unit_id(u) * UNIT + 4096, 1024, AUX_SYS);
                WAIT_V(0);
            }
            const unsigned base4 = base * 0x01010101u;
            const u32x4 p2 = v[4];
            u32x4 p0[4], e[4];
#pragma unroll
            for (int r = 0; r < 16; ++r) {
                const unsigned hb = (r & 7) >= 2 ? p1[r >> 3] >> ((r & 7) - 2) : p1[r >> 3] << (2 - (r & 7));
                e[r >> 2][r & 3] = (((p2[r >> 2] >> (2 * (r & 3))) & 0x03030303u) | (hb & 0x04040404u)) + base4;
            }
#pragma unroll
            for (int r = 0; r < 4; ++r) { p0[r] = v[r]; *(LAS u32x4*)(b + r * 1024 + lane * 16) = e[r]; }
            CFENCE;
#pragma nounroll
            for (unsigned k = lane; k < n; k += 64) {
                const unsigned en = *(const LAS unsigned short*)(b + PK_HDR + 4 + k * 2), d = en >> 12;
                b[en & 0xfffu] = (unsigned char)(d == 14 ? base + 8 : d == 15 ? 0u : base - 1 - d);
            }
            CFENCE;
#pragma unroll
            for (int r = 0; r < 4; ++r) e[r] = *(const LAS u32x4*)(b + r * 1024 + lane * 16);
#pragma unroll
            for (int r = 0; r < 16; ++r) {
                const unsigned pp = p0[r >> 2][r & 3], ee = e[r >> 2][r & 3];
                const unsigned lo = (pp & ~H8) | ((ee << 7) & H8), hi = (pp & H8) | ((ee >> 1) & ~H8);
                v[r >> 1][2 * (r & 1)] = __builtin_amdgcn_perm(hi, lo, 0x05010400u); v[r >> 1][2 * (r & 1) + 1] = __builtin_amdgcn_perm(hi, lo, 0x07030602u);
            }
        } else {
            const unsigned slot = OFF_PX + unit_id(u) * UNIT;
            WAIT_L(0);
            __builtin_amdgcn_raw_ptr_buffer_load_lds(rp, (LAS void*)b, 16, lane * 16, slot + 6144, 0, AUX_SYS);
            __builtin_amdgcn_raw_ptr_buffer_load_lds(rp, (LAS void*)b, 16, lane * 16, slot + 6144, 1024, AUX_SYS);
            __builtin_amdgcn_raw_ptr_buffer_load_lds(rp, (LAS void*)b, 16, lane * 16, slot + 6144, 2048, AUX_SYS);
            WAIT_V(0);
#pragma unroll
            for (int r = 5; r < 8; ++r) v[r] = *(const LAS u32x4*)(b + (r - 5) * 1024 + lane * 16);
        }
        img_store(v, rs, isw, img, lane);
    }
};
static_assert(OFF_PW == OFF_PX + 512 * UNIT, "unit ids run through the x and the w slots");

#define DRAW(r, tick) { unsigned long long ex_; asm volatile("s_mov_b64 %1, exec\n\ts_mov_b64 exec, 1\n\tglobal_atomic_add %0, %2, %3, %4 sc0\n\ts_mov_b64 exec, %1" \
        : "=&v"(r), "=&s"(ex_) : "v"(0u), "v"(1u), "s"(tick) : "memory"); }
#define TAKE(t, r, N) { unsigned t_; asm volatile("s_waitcnt vmcnt(" #N ")\n\tv_readfirstlane_b32 %0, %1" : "=s"(t_) : "v"(r) : "memory"); t = (int)(t_ - tbase); }
constexpr int NU_IN = 768, NU_ALL = NU_IN + 256;

__device__ __forceinline__ void pull_role(const Params& p, LAS unsigned char* L, int pi, int lane, unsigned call, int wv) {
    Puller P; P.rank = p.rank; P.q = pi % 7; P.src = (p.rank + 1 + P.q) & 7; P.lane = lane; P.seen = 0; P.call = call;
    P.tab = SCHED_TAB.w[P.q][lane];
    asm volatile("" : "+v"(P.tab));
    P.myws = (char*)p.ws[p.rank]; P.rs = mk_rsrc(P.myws, WS_USED); P.rp = mk_rsrc(p.ws[P.src], WS_USED);
    char* tick = P.myws + OFF_TICK + P.q * 128;
    const unsigned tbase = (call - 1) * (NU_ALL + 2 * PPM);
    unsigned r0, r1, pf = 0;
    if (pi >= 7 * PPP) {
        const unsigned long long t0 = __builtin_amdgcn_s_memrealtime();
        while ((int)(__builtin_amdgcn_readfirstlane(ld_agent(tick)) - tbase) < NU_IN) { if (spin_fail(P.myws, call, t0)) break; __builtin_amdgcn_s_sleep(32); }
    }
    DRAW(r0, tick); DRAW(r1, tick);
    int u0, u1, sl = 0;
    TAKE(u0, r0, 0); TAKE(u1, r1, 0);
    if ((unsigned)u0 < (unsigned)NU_IN) P.issue(L, P.unit_of(u0));
    WAIT_V(0);
    while ((unsigned)u0 < (unsigned)NU_IN) {
        unsigned r2;
        if ((unsigned)u1 < (unsigned)NU_IN) { P.issue(L + (sl ^ 1) * PL_SLOT, P.unit_of(u1)); DRAW(r2, tick); WAIT_V(7); }
        else { DRAW(r2, tick); WAIT_V(0); }
        if (pf != 0 && lane == 0) st_sys(P.myws + pf, call);
        P.finish(L + sl * PL_SLOT, P.unit_of(u0));
        pf = P.flag_of(P.unit_of(u0));
        u0 = u1; sl ^= 1;
        TAKE(u1, r2, 8);
    }
    WAIT_V(0);
    if (pf != 0 && lane == 0) st_sys(P.myws + pf, call);
    pf = 0;
    STAMP(P.myws, wv, 9);
    int seen = -1;
#define MASK_ISSUE(b, u) { const int mu_ = (u) - NU_IN; \
        if ((mu_ >> 1) != seen) { const unsigned long long t0 = __builtin_amdgcn_s_memrealtime(); \
            for (;;) { CFENCE; const unsigned f = __builtin_amdgcn_raw_buffer_load_b32(P.rs, 0, OFF_AMAX + (P.src * NGEMM + (mu_ >> 1)) * 8 + 4, AUX_SYS); \
                if (__builtin_amdgcn_readfirstlane(f) >= call) break; if (spin_fail(P.myws, call, t0)) break; } \
            CFENCE; seen = mu_ >> 1; } \
        __builtin_amdgcn_raw_ptr_buffer_load_lds(P.rp, (LAS void*)(b), 16, lane * 16, OFF_MSK + mu_ * 4096, 0, AUX_SYS); \
        __builtin_amdgcn_raw_ptr_buffer_load_lds(P.rp, (LAS void*)(b), 16, lane * 16, OFF_MSK + mu_ * 4096, 1024, AUX_SYS); \
        __builtin_amdgcn_raw_ptr_buffer_load_lds(P.rp, (LAS void*)(b), 16, lane * 16, OFF_MSK + mu_ * 4096, 2048, AUX_SYS); \
        __builtin_amdgcn_raw_ptr_buffer_load_lds(P.rp, (LAS void*)(b), 16, lane * 16, OFF_MSK + mu_ * 4096, 3072, AUX_SYS); }
    if ((unsigned)u0 < (unsigned)NU_ALL) MASK_ISSUE(L + sl * PL_SLOT, u0);
    WAIT_V(0);
    STAMP(P.myws, wv, 16);
    while ((unsigned)u0 < (unsigned)NU_ALL) {
        unsigned r2;
        if ((unsigned)u1 < (unsigned)NU_ALL) { MASK_ISSUE(L + (sl ^ 1) * PL_SLOT, u1); DRAW(r2, tick); WAIT_V(5); }
        else { DRAW(r2, tick); WAIT_V(0); }
        if (pf != 0 && lane == 0) st_sys(P.myws + pf, call);
#pragma unroll
        for (int r = 0; r < 4; ++r)
            __builtin_amdgcn_raw_buffer_store_b128(*(const LAS u32x4*)(L + sl * PL_SLOT + r * 1024 + lane * 16), P.rs, lane * 16 + r * 1024, OFF_LM + P.src * 1048576 + (u0 - NU_IN) * 4096, LOC_AUX);
        pf = OFF_MF + (P.src * 256 + u0 - NU_IN) * 4;
        u0 = u1; sl ^= 1;
        TAKE(u1, r2, 4);
    }
    WAIT_V(0);
    if (pf != 0 && lane == 0) st_sys(P.myws + pf, call);
    STAMP(P.myws, wv, 17);
}

__device__ __forceinline__ u32x2 lut_expand(unsigned byte) { u32x2 s = {0u, 0u}; unsigned c = 0;
#pragma unroll
    for (int i = 0; i < 8; ++i) { const unsigned b = (byte >> i) & 1u; s[i >> 2] |= (b ? c : 0x0cu) << (8 * (i & 3)); c += b; } return s; }
__device__ __forceinline__ unsigned lut_compact(unsigned nib) { unsigned s = 0x0c0c0c0cu, c = 0;
#pragma unroll
    for (int i = 0; i < 4; ++i) if ((nib >> i) & 1u) { s = (s & ~(0xffu << (8 * c))) | ((unsigned)i << (8 * c)); ++c; } return s; }
__device__ __forceinline__ unsigned popc4(const u32x4& m) { return __builtin_popcount(m[0]) + __builtin_popcount(m[1]) + __builtin_popcount(m[2]) + __builtin_popcount(m[3]); }
__device__ __forceinline__ unsigned nib4(unsigned m, int i) { return (i == 0 ? m << 2 : m >> (4 * i - 2)) & 0x3cu; }
__device__ __forceinline__ unsigned out_voff(int lane) { return (unsigned)(lane & 15) * N_ + (unsigned)(lane >> 4) * 16; }
__device__ __forceinline__ unsigned out_soff(int src, int wt) {
    const int tile = wt >> 3, w = wt & 7;
    return (unsigned)((tile >> 3) * 256 + (w >> 2) * 64) * N_ + (unsigned)(src * NS + (tile & 7) * 256 + (w & 3) * 64);
}
__device__ __forceinline__ constexpr unsigned chunk_soff(int ch) { return (unsigned)((ch >> 2) * 128 + (ch & 3) * 16) * N_; }

__device__ __forceinline__ void expand_role(const Params& p, LAS unsigned char* lds, LAS unsigned char* L, int ei, int lane, unsigned call, int wv) {
    const int rank = p.rank, q = ei % 7, src = (rank + 1 + q) & 7;
    char* myws = (char*)p.ws[rank];
    char* pws = (char*)p.ws[src];
    const rsrc_t rs = mk_rsrc(myws, WS_USED), ro = mk_rsrc(p.out[rank], OUT_BYTES);
#pragma unroll
    for (int i = 0; i < 4; ++i) *(LAS u32x2*)(lds + EX_LUT + (i * 64 + lane) * 8) = lut_expand(i * 64 + lane);
    char* tick = myws + OFF_TICK + (8 + q) * 128;
    const unsigned tbase = (call - 1) * (1024 + 2 * PPO), tag = call & 0x3ffffu;
    bool dead = false;
#define ENT_OFF(t) ((lane & 2) ? OFF_MF + (src * 256 + ((t) >> 2)) * 4 : OFF_DONE + (src * 1024 + (t)) * 8 + (lane & 1) * 4)
#define ENT_OK(e, x, f) (((e) >> 14) == tag && ((x) >> 22) == (tag & 0x3ffu) && (f) >= call)
#define ENT_FETCH(t) __builtin_amdgcn_raw_ptr_buffer_load_lds(rs, (LAS void*)(L + EX_E), 4, ENT_OFF(t), 0, 0, AUX_SYS)
#define EX_ISSUE(t, e, x, f, n, xo) { unsigned e_ = (e), x_ = (x), f_ = (f); \
        if (!ENT_OK(e_, x_, f_) && !dead) { const unsigned long long t0_ = __builtin_amdgcn_s_memrealtime(); \
            for (;;) { CFENCE; const unsigned g_ = __builtin_amdgcn_raw_buffer_load_b32(rs, ENT_OFF(t), 0, AUX_SYS); \
                e_ = __builtin_amdgcn_readlane(g_, 0); x_ = __builtin_amdgcn_readlane(g_, 1); f_ = __builtin_amdgcn_readlane(g_, 2); \
                if (ENT_OK(e_, x_, f_)) break; if (spin_fail(myws, call, t0_)) { dead = true; break; } __builtin_amdgcn_s_sleep(8); } \
            CFENCE; } \
        n = min(e_ & 0x3fffu, 8192u); xo = x_; \
        const unsigned by_ = min((((x_ >> 21) & 1u) ? n : ((n + 15) >> 4) * 12 + (x_ & 0x3fffu)) + 15 & ~15u, 8192u); \
        const rsrc_t rc_ = mk_rsrc(pws + OFF_COD + (unsigned)(t) * 8192, by_); \
        _Pragma("nounroll") for (unsigned r_ = 0; r_ < by_; r_ += 1024) \
            __builtin_amdgcn_raw_ptr_buffer_load_lds(rc_, (LAS void*)(L + EX_B + r_), 16, lane * 16, r_, 0, AUX_SYS); \
        __builtin_amdgcn_raw_ptr_buffer_load_lds(rs, (LAS void*)(L + EX_M), 16, lane * 16, OFF_LM + src * 1048576 + (t) * 1024, 0, AUX_SYS); }
    unsigned r0, n0 = 0, x0 = 0;
    DRAW(r0, tick);
    int t0;
    TAKE(t0, r0, 0);
    if ((unsigned)t0 < 1024u) EX_ISSUE(t0, 0u, 0u, 0u, n0, x0);
    int t1;
    { unsigned r1; DRAW(r1, tick); TAKE(t1, r1, 0); }
    STAMP(myws, wv, 18);
    while ((unsigned)t0 < 1024u) {
        if ((unsigned)t1 < 1024u) ENT_FETCH(t1);
        const u32x4 m = *(const LAS u32x4*)(L + EX_M + lane * 16);
        if ((x0 >> 21) & 1u) {
#pragma nounroll
            for (unsigned r = 0; r < n0; r += 1024) *(LAS u32x4*)(L + r + lane * 16) = *(const LAS u32x4*)(L + EX_B + r + lane * 16);
        } else {
            const unsigned G = (n0 + 15) >> 4, B4 = ((x0 >> 14) & 0x7fu) * 0x01010101u;
            unsigned bm[4] = {0u, 0u, 0u, 0u};
#define UNFOLD(it0, n) { unsigned o_[n][3]; \
                _Pragma("unroll") for (int i = 0; i < n; ++i) { const LAS unsigned* sB = (const LAS unsigned*)(L + EX_B + ((it0 + i) * 64 + lane) * 12); o_[i][0] = sB[0]; o_[i][1] = sB[1]; o_[i][2] = sB[2]; } \
                SCHED; \
                _Pragma("unroll") for (int i = 0; i < n; ++i) { const int it = it0 + i; const unsigned g = it * 64 + lane; \
                    const unsigned s0 = o_[i][0] & 0x3f3f3f3fu, s1 = o_[i][1] & 0x3f3f3f3fu, s2 = o_[i][2] & 0x3f3f3f3fu; \
                    const unsigned s3 = ((o_[i][0] >> 6) & 0x03030303u) | ((o_[i][1] >> 4) & 0x0c0c0c0cu) | ((o_[i][2] >> 2) & 0x30303030u); \
                    *(LAS u32x4*)(L + g * 16) = (u32x4){s0 + B4, s1 + B4, s2 + B4, s3 + B4}; \
                    const unsigned b_ = bits16((s0 + 0x01010101u) & 0x40404040u, (s1 + 0x01010101u) & 0x40404040u, (s2 + 0x01010101u) & 0x40404040u, (s3 + 0x01010101u) & 0x40404040u, 6); \
                    bm[it >> 1] |= (g < G ? b_ : 0u) << (16 * (it & 1)); } }
            UNFOLD(0, 4);
            if (G > 256) { UNFOLD(4, 1); if (G > 320) UNFOLD(5, 3); }
            CYC(0);
            unsigned at;
            const unsigned ne = min(esc_count(bm, at), MAXESC);
            esc_list(bm, at, L + EX_P, lane);
            CFENCE;
#pragma nounroll
            for (unsigned i = lane; i < ne; i += 64) L[*(const LAS unsigned short*)(L + EX_P + i * 2)] = L[EX_B + G * 12 + i];
            CFENCE;
        }
        const unsigned cnt = popc4(m);
        unsigned off = wave_scan(cnt) - cnt + (unsigned)(size_t)L;
        WAIT_L(0);
        CYC(1);
        unsigned n1 = 0, x1 = 0;
        if ((unsigned)t1 < 1024u) {
            WAIT_V(0);
            const u32x4 ent = *(const LAS u32x4*)(L + EX_E);
            EX_ISSUE(t1, __builtin_amdgcn_readfirstlane(ent[0]), __builtin_amdgcn_readfirstlane(ent[1]), __builtin_amdgcn_readfirstlane(ent[2]), n1, x1);
        }
        unsigned r2;
        DRAW(r2, tick);
        CYC(2);
        const unsigned so = out_soff(src, t0), vo = out_voff(lane);
        u32x2 sel[16]; unsigned of[16], w0[16], w1[16], w2[16];
#pragma unroll
        for (int g = 0; g < 16; ++g) {
            const unsigned b8 = ((g & 3) == 0 ? m[g >> 2] << 3 : m[g >> 2] >> (8 * (g & 3) - 3)) & 0x7f8u;
            sel[g] = *(const LAS u32x2*)(lds + EX_LUT + b8);
            const LAS unsigned* w = (const LAS unsigned*)(size_t)(off & ~3u);
            w0[g] = w[0]; w1[g] = w[1]; w2[g] = w[2]; of[g] = off;
            off += __builtin_popcount(b8);
        }
        SCHED;
        u32x4 o[8];
#pragma unroll
        for (int g = 0; g < 16; ++g) {
            const unsigned a0 = __builtin_amdgcn_alignbyte(w1[g], w0[g], of[g]), a1 = __builtin_amdgcn_alignbyte(w2[g], w1[g], of[g]);
            o[g >> 1][(g & 1) * 2] = __builtin_amdgcn_perm(a1, a0, sel[g][0]); o[g >> 1][(g & 1) * 2 + 1] = __builtin_amdgcn_perm(a1, a0, sel[g][1]);
        }
        CYC(3);
        if (dead) {
#pragma unroll
            for (int ch = 0; ch < 8; ++ch) o[ch] = (u32x4){0x7f7f7f7fu, 0x7f7f7f7fu, 0x7f7f7f7fu, 0x7f7f7f7fu};
            CFENCE;
        }
#pragma unroll
        for (int ch = 0; ch < 8; ++ch) __builtin_amdgcn_raw_buffer_store_b128(o[ch], ro, vo, so + chunk_soff(ch), OUT_AUX);
        CYC(4);
        int t2;
        TAKE(t2, r2, 8);
        CYC(5);
        t0 = t1; n0 = n1; x0 = x1; t1 = t2;
    }
    STAMP(myws, wv, 19);
}

__device__ __forceinline__ void comm_role(const Params& p, LAS unsigned char* lds, int c, int wid, int lane, unsigned call) {
    const int rank = p.rank;
    char* myws = (char*)p.ws[rank];
    const rsrc_t rs = mk_rsrc(myws, WS_USED);
    const int wv = (NGEMM + c) * 8 + wid; (void)wv;
    STAMP(myws, wv, 0);
    if (wid >= 2) {
        const rsrc_t rx = mk_rsrc(p.x, X_BYTES), rw = mk_rsrc(p.w, W_BYTES);
        for (int u = (wid - 2) * 128 + c; u < 2560; u += 768) {
            const int j = u / 320, r = u % 320, t = rank * 8 + j;
            u32x4 v[8];
            if (r >= 224 && r < 288) {
                const int g = r - 224;
                x_load(v, rx, j, g, lane);
                pack_store(v, rs, lds + 2048 + wid * EX_BYTES, OFF_PX + (unsigned)(j * 64 + g) * UNIT, lane);
                img_store(v, rs, false, x_img(t, g), lane);
                WAIT_V(0);
                unsigned old = 0;
                if (lane == 0) {
                    st_sys(myws + OFF_FLAG + t * FLAG_REC + g * 4, call);
                    old = __hip_atomic_fetch_add((GAS unsigned*)(myws + OFF_PCNT + j * 128), 1u, __ATOMIC_RELAXED, __HIP_MEMORY_SCOPE_AGENT);
                }
                if ((__builtin_amdgcn_readfirstlane(old) & 63) == 63 && lane < 8) st_sys((char*)p.ws[lane] + OFF_PFLG + t * 128, call);
            } else {
                const bool own = r >= 288;
                const int e = own ? r - 288 : r / 7, d = own ? rank : (rank + 1 + r % 7) & 7;
                u32x2 w[16];
                w_load(w, rw, j, d, e >> 2, e & 1, (e >> 1) & 1, lane);
                w_transpose(v, w);
                if (own) {
                    img_store(v, rs, true, w_img(t, e), lane);
                    WAIT_V(0);
                    if (lane == 0) st_sys(myws + OFF_FLAG + t * FLAG_REC + FLAG_W + e * 4, call);
                } else {
                    const unsigned id = (unsigned)((d * 8 + j) * 32 + e);
                    pack_store(v, rs, lds + 2048 + wid * EX_BYTES, OFF_PW + id * UNIT, lane);
                    WAIT_V(0);
                    unsigned old = 0;
                    if (lane == 0) old = __hip_atomic_fetch_add((GAS unsigned*)(myws + OFF_PCNT + (8 + d * 8 + j) * 128), 1u, __ATOMIC_RELAXED, __HIP_MEMORY_SCOPE_AGENT);
                    if ((__builtin_amdgcn_readfirstlane(old) & 31) == 31 && lane == 0) st_sys((char*)p.ws[d] + OFF_PFLG + t * 128 + 4, call);
                }
            }
            STAMP(myws, wv, 1 + u / 768);
        }
    }
    if (wid < CW) {
        const int ci = wid * 128 + c;
        if (ci < 7 * PPM) pull_role(p, lds + 2048 + wid * EX_BYTES, ci, lane, call, wv);
        if (ci < 7 * PPO) expand_role(p, lds, lds + 2048 + wid * EX_BYTES, ci, lane, call, wv);
    }
}

__device__ __forceinline__ void gemm_role(const Params& p, LAS unsigned char* lds, int rt, int ct, int wid, int lane, unsigned call) {
    const int rank = p.rank;
    char* myws = (char*)p.ws[rank];
    const int tile = rt * 8 + ct;
    const int wv = tile * 8 + wid; (void)wv;
    const int wr = wid >> 2, wc = wid & 3;
    const rsrc_t rs = mk_rsrc(myws, WS_USED);
    const unsigned voff = (unsigned)wid * 2048 + (unsigned)lane * 16;
    const unsigned voffF = lane < 4 ? (unsigned)(rt * 16 + lane * 4) : FLAG_W + (unsigned)(ct * 16 + (lane & 3) * 4);
    LAS unsigned char* ldsw = lds + wid * 2048;
    LAS unsigned char* lda = lds + wr * 8192 + lane * 16;
    LAS unsigned char* ldb = lds + wc * 4096 + lane * 16;
    const int seqv = step_of(lane, rank, rt);
#define SEQ(i) __builtin_amdgcn_readlane(seqv, (i) < NT ? (i) : NT - 1)
    STAMP(myws, wv, 0);
#define SA(b, h) (((b) * 2 + (h)) * HALFB)
#define SB(b, h) ((4 + (b) * 2 + (h)) * HALFB)
#define STAGE(bufoff, soff) do { \
        __builtin_amdgcn_raw_ptr_buffer_load_lds(rs, (LAS void*)(ldsw + (bufoff)), 16, voff, (soff), 0, GEMM_AUX); \
        __builtin_amdgcn_raw_ptr_buffer_load_lds(rs, (LAS void*)(ldsw + (bufoff)), 16, voff, (soff), 1024, GEMM_AUX); } while (0)
#define LDA(dst, b, h) do { _Pragma("unroll") for (int m = 0; m < 4; ++m) _Pragma("unroll") for (int k = 0; k < 2; ++k) dst[m][k] = *(const LAS bf16x8*)(lda + SA(b, h) + m * 2048 + k * 1024); } while (0)
#define LDB(dst, b, h) do { _Pragma("unroll") for (int n = 0; n < 2; ++n) _Pragma("unroll") for (int k = 0; k < 2; ++k) dst[n][k] = *(const LAS bf16x8*)(ldb + SB(b, h) + n * 2048 + k * 1024); } while (0)
#define MMA(ai, bj, At, Bt) do { __builtin_amdgcn_s_setprio(1); _Pragma("unroll") for (int m = 0; m < 4; ++m) _Pragma("unroll") for (int n = 0; n < 2; ++n) _Pragma("unroll") for (int k = 0; k < 2; ++k) \
        acc[ai][bj][m][n] = __builtin_amdgcn_mfma_f32_16x16x32_bf16(Bt[n][k], At[m][k], acc[ai][bj][m][n], 0, 0, 0); __builtin_amdgcn_s_setprio(0); } while (0)
#define OFFA(t) (OFF_XT + (unsigned)(t) * XT_STEP + (unsigned)rt * BLK)
#define OFFB(t) (OFF_WT + (unsigned)(t) * WT_STEP + (unsigned)ct * BLK)
#define SLOW_WAIT(t) do { const unsigned long long t0_ = __builtin_amdgcn_s_memrealtime(); \
        for (;;) { CFENCE; const unsigned f_ = __builtin_amdgcn_raw_buffer_load_b32(rs, voffF, OFF_FLAG + (unsigned)(t) * FLAG_REC, AUX_SYS); \
            if (__builtin_amdgcn_ballot_w64(f_ < call) == 0) break; if (spin_fail(myws, call, t0_)) break; } } while (0)
#define CHECK(t, s, tn) do { if (wid == 0) { const unsigned f0_ = *(volatile LAS unsigned*)(lds + LDS_FLAG + (s) * 256 + lane * 4); \
            if (__builtin_amdgcn_ballot_w64(f0_ < call) != 0) SLOW_WAIT(t); \
            __builtin_amdgcn_raw_ptr_buffer_load_lds(rs, (LAS void*)(lds + LDS_FLAG + ((s) ^ 1) * 256), 4, voffF, OFF_FLAG + (unsigned)(tn) * FLAG_REC, 0, AUX_SYS); } } while (0)

    f32x4 acc[2][2][4][2];
#pragma unroll
    for (int a = 0; a < 2; ++a)
#pragma unroll
        for (int b = 0; b < 2; ++b)
#pragma unroll
            for (int m = 0; m < 4; ++m)
#pragma unroll
                for (int n = 0; n < 2; ++n) acc[a][b][m][n] = (f32x4){0.f, 0.f, 0.f, 0.f};
    bf16x8 At[4][2], B0[2][2], B1[2][2];

    {
        const int t0 = SEQ(0), t1 = SEQ(1), t2 = SEQ(2);
        if (wid == 0) {
            SLOW_WAIT(t0); SLOW_WAIT(t1);
            __builtin_amdgcn_raw_ptr_buffer_load_lds(rs, (LAS void*)(lds + LDS_FLAG), 4, voffF, OFF_FLAG + (unsigned)t2 * FLAG_REC, 0, AUX_SYS);
        }
        CFENCE; BAR; CFENCE;
        STAMP(myws, wv, 1);
        STAGE(SB(0, 0), OFFB(t0)); STAGE(SA(0, 0), OFFA(t0)); STAGE(SB(0, 1), OFFB(t0) + HALFB); STAGE(SA(0, 1), OFFA(t0) + HALFB);
        if (wr == 1) BAR;
        WAIT_V(4); BAR;
        STAGE(SB(1, 0), OFFB(t1)); STAGE(SA(1, 0), OFFA(t1)); STAGE(SB(1, 1), OFFB(t1) + HALFB);
        WAIT_V(6); BAR;
    }
    for (int i = 0; i < NT; i += 2) {
        const int t1 = SEQ(i + 1), t2 = SEQ(i + 2), t3 = SEQ(i + 3), t4 = SEQ(i + 4);
        const unsigned a1 = OFFA(t1), a2 = OFFA(t2), b2 = OFFB(t2), a3 = OFFA(t3), b3 = OFFB(t3);
        CHECK(t2, 0, t3);
        LDB(B0, 0, 0); SCHED; LDA(At, 0, 0); STAGE(SA(1, 1), a1 + HALFB);
        WAIT_L(8); BAR; WAIT_L(0); MMA(0, 0, At, B0); BAR; SCHED;
        LDB(B1, 0, 1); STAGE(SB(0, 0), b2);
        BAR; WAIT_L(0); MMA(0, 1, At, B1); BAR;
        LDA(At, 0, 1); STAGE(SA(0, 0), a2);
        BAR; WAIT_L(0); MMA(1, 0, At, B0); BAR; SCHED;
        STAGE(SB(0, 1), b2 + HALFB);
        WAIT_V(6); BAR; MMA(1, 1, At, B1); BAR;
        CHECK(t3, 1, t4);
        LDB(B0, 1, 0); SCHED; LDA(At, 1, 0); STAGE(SA(0, 1), a2 + HALFB);
        WAIT_L(8); BAR; WAIT_L(0); MMA(0, 0, At, B0); BAR; SCHED;
        LDB(B1, 1, 1); STAGE(SB(1, 0), b3);
        BAR; WAIT_L(0); MMA(0, 1, At, B1); BAR;
        LDA(At, 1, 1); STAGE(SA(1, 0), a3);
        BAR; WAIT_L(0); MMA(1, 0, At, B0); BAR; SCHED;
        STAGE(SB(1, 1), b3 + HALFB);
        WAIT_V(6); BAR; MMA(1, 1, At, B1); BAR;
    }
    WAIT_V(0);
    if (wr == 0) BAR;
    BAR;
    STAMP(myws, wv, 10);

    const int wt = tile * 8 + wid;
    u32x4 msk;
    float mx = 0.f;
#pragma unroll
    for (int w = 0; w < 4; ++w) {
        unsigned mm = 0;
#pragma unroll
        for (int e = 31; e >= 0; --e) {
            const int ch = 2 * w + (e >> 4);
            const float v = acc[ch >> 2][(e >> 1) & 1][ch & 3][e & 1][(e >> 2) & 3];
            mm = __builtin_amdgcn_alignbit(mm, __float_as_uint(v), 31);
            mx = fmaxf(mx, v);
        }
        msk[w] = ~mm;
    }
    __builtin_amdgcn_raw_buffer_store_b128(msk, rs, lane * 16, OFF_MSK + wt * 1024, AUX_SYS);
    LAS unsigned* red = (LAS unsigned*)(lds + LDS_RED);
    const unsigned wm = wave_max(__float_as_uint(mx));
    if (lane == 0) red[wid] = wm;
    WAIT_V(0); WAIT_L(0); BAR; CFENCE;
    if (wid == 0) {
        const unsigned t = wave_max(red[lane & 7]);
        if (lane < 8) {
            GAS unsigned long long* g = (GAS unsigned long long*)((char*)p.ws[lane] + OFF_AMAX + (rank * NGEMM + tile) * 8);
            __hip_atomic_store(g, ((unsigned long long)call << 32) | t, __ATOMIC_RELAXED, __HIP_MEMORY_SCOPE_SYSTEM);
        }
    }
    LAS unsigned char* A = lds + wid * 16384;
    LAS unsigned char* B = A + 8192;
    const unsigned cnt = popc4(msk), scan = wave_scan(cnt), Lc = __builtin_amdgcn_readlane(scan, 63);
    unsigned off = scan - cnt;
#pragma unroll
    for (int r = 0; r < 8; ++r) *(LAS u32x4*)(A + r * 1024 + lane * 16) = (u32x4){0u, 0u, 0u, 0u};
    LAS unsigned char* lut = A + 16384 - 64;
    if (lane < 16) *(LAS unsigned*)(lut + lane * 4) = lut_compact(lane);
#pragma unroll
    for (int a = 0; a < 2; ++a)
#pragma unroll
        for (int b = 0; b < 2; ++b)
#pragma unroll
            for (int m = 0; m < 4; ++m)
#pragma unroll
                for (int n = 0; n < 2; ++n)
#pragma unroll
                    for (int j = 0; j < 4; ++j) { acc[a][b][m][n][j] = fmaxf(acc[a][b][m][n][j], 0.f); asm volatile("" : "+v"(acc[a][b][m][n][j])); }
    unsigned sel[32], of[32];
    off += (unsigned)(size_t)A;
#pragma unroll
    for (int g = 0; g < 32; ++g) {
        const unsigned n4 = nib4(msk[g >> 3], g & 7);
        sel[g] = *(const LAS unsigned*)(lut + n4); of[g] = off;
        off += __builtin_popcount(n4);
    }
    {
        const unsigned long long t0 = __builtin_amdgcn_s_memrealtime();
        unsigned t;
        for (;;) {
            CFENCE;
            const u32x4 g = __builtin_amdgcn_raw_buffer_load_b128(rs, lane * 16, OFF_AMAX + wid * 1024, AUX_SYS);
            t = max(g[0], g[2]);
            if (__builtin_amdgcn_ballot_w64(g[1] < call || g[3] < call) == 0) break;
            if (spin_fail(myws, call, t0)) break;
        }
        t = wave_max(t);
        if (lane == 0) red[8 + wid] = t;
    }
    WAIT_L(0); BAR; CFENCE;
    const float amax = __uint_as_float(wave_max(red[8 + (lane & 7)]));
    const bool dead = __builtin_amdgcn_readfirstlane(ld_agent(myws + OFF_TMO)) == call;
    const float inv = 448.0f / amax;
    STAMP(myws, wv, 11);

    u32x4 qv[8];
#pragma unroll
    for (int ch = 0; ch < 8; ++ch) {
        const int a = ch >> 2, m = ch & 3;
#pragma unroll
        for (int j = 0; j < 4; ++j) {
            int w = __builtin_amdgcn_cvt_pk_fp8_f32(acc[a][0][m][0][j] * inv, acc[a][0][m][1][j] * inv, 0, false);
            w = __builtin_amdgcn_cvt_pk_fp8_f32(acc[a][1][m][0][j] * inv, acc[a][1][m][1][j] * inv, w, true);
            const unsigned c = dead ? 0x7f7f7f7fu : (unsigned)w & 0x7f7f7f7fu;
            qv[ch][j] = c;
            const unsigned o_ = of[ch * 4 + j];
            const unsigned long long z = (unsigned long long)__builtin_amdgcn_perm(0u, c, sel[ch * 4 + j]) << ((o_ & 3u) * 8);
            LAS unsigned* d = (LAS unsigned*)(size_t)(o_ & ~3u);
            __hip_atomic_fetch_or(d, (unsigned)z, __ATOMIC_RELAXED, __HIP_MEMORY_SCOPE_WAVEFRONT);
            __hip_atomic_fetch_or(d + 1, (unsigned)(z >> 32), __ATOMIC_RELAXED, __HIP_MEMORY_SCOPE_WAVEFRONT);
        }
    }
    CFENCE;
    const unsigned top = dead ? 0x7fu : __builtin_amdgcn_readfirstlane(__builtin_amdgcn_cvt_pk_fp8_f32(__uint_as_float(wm) * inv, 0.f, 0, false)) & 0x7fu;
    const unsigned base = top > 62 ? top - 62 : 0, B4 = base * 0x01010101u, G = (Lc + 15) >> 4;
    if (lane < 16) A[Lc + lane] = (unsigned char)base;
    CFENCE;
    unsigned bm[4] = {0u, 0u, 0u, 0u};
#pragma unroll
    for (int it = 0; it < 8; ++it)
        if (it * 64u < G) {
            const unsigned g = it * 64 + lane;
            if (g < G) {
                const u32x4 d = *(const LAS u32x4*)(A + g * 16);
                unsigned sy[4], ms[4];
#pragma unroll
                for (int i = 0; i < 4; ++i) { const unsigned t = (d[i] | H8) - B4; ms[i] = ~t & H8; sy[i] = (t & 0x3f3f3f3fu) | ((ms[i] >> 1) - (ms[i] >> 7)); }
                LAS unsigned* dB = (LAS unsigned*)(B + g * 12);
#pragma unroll
                for (int i = 0; i < 3; ++i) dB[i] = sy[i] | ((sy[3] << (6 - 2 * i)) & 0xc0c0c0c0u);
                bm[it >> 1] |= bits16(ms[0], ms[1], ms[2], ms[3], 7) << (16 * (it & 1));
            }
        }
    unsigned at;
    const unsigned ne = esc_count(bm, at);
    const bool raw = ne > MAXESC || G * 12 + ne > Lc;
    if (!raw) {
        LAS unsigned char* list = B + 6656;
        esc_list(bm, at, list, lane);
        CFENCE;
#pragma nounroll
        for (unsigned i = lane; i < ne; i += 64) B[G * 12 + i] = A[*(const LAS unsigned short*)(list + i * 2)];
        CFENCE;
    }
    {
        LAS unsigned char* S = raw ? A : B;
#pragma nounroll
        for (unsigned r = 0; r < (raw ? Lc : G * 12 + ne); r += 1024)
            __builtin_amdgcn_raw_buffer_store_b128(*(const LAS u32x4*)(S + r + lane * 16), rs, lane * 16, OFF_COD + wt * 8192 + r, AUX_SYS);
    }
    if (lane == 0) { red[16 + wid] = ((call & 0x3ffffu) << 14) | Lc; red[24 + wid] = (call << 22) | (raw ? 1u << 21 : (base << 14) | ne); }
    WAIT_V(0); WAIT_L(0); BAR; CFENCE;
    if (wid == 0 && (lane >> 3) != rank)
        __hip_atomic_store((GAS unsigned long long*)((char*)p.ws[lane >> 3] + OFF_DONE + (rank * 1024 + tile * 8 + (lane & 7)) * 8),
                           ((unsigned long long)red[24 + (lane & 7)] << 32) | red[16 + (lane & 7)], __ATOMIC_RELAXED, __HIP_MEMORY_SCOPE_SYSTEM);
    STAMP(myws, wv, 12);
    {
        const rsrc_t ro = mk_rsrc(p.out[rank], OUT_BYTES);
        const unsigned so = out_soff(rank, wt), vo = out_voff(lane);
#pragma unroll
        for (int ch = 0; ch < 8; ++ch) __builtin_amdgcn_raw_buffer_store_b128(qv[ch], ro, vo, so + chunk_soff(ch), OUT_AUX);
    }
}

__global__ __launch_bounds__(512, 2) void fused_kernel(Params p) {
    __shared__ __attribute__((aligned(16))) unsigned char lds_arr[LDS_BYTES];
    LAS unsigned char* lds = (LAS unsigned char*)lds_arr;
    const int wid = __builtin_amdgcn_readfirstlane(threadIdx.x >> 6), lane = threadIdx.x & 63;
    const int b = blockIdx.x, xcd = b & 7, idx = b >> 3;
    unsigned call = 0;
    if (lane == 0) call = __hip_atomic_fetch_add((GAS unsigned*)((char*)p.ws[p.rank] + OFF_CNT) + b * 8 + wid, 1u, __ATOMIC_RELAXED, __HIP_MEMORY_SCOPE_AGENT) + 1u;
    call = __builtin_amdgcn_readfirstlane(call);
    if (idx < 16) gemm_role(p, lds, (xcd >> 1) * 4 + (idx >> 2), (xcd & 1) * 4 + (idx & 3), wid, lane, call);
    else comm_role(p, lds, (idx - 16) * 8 + xcd, wid, lane, call);
}

extern "C" void kernel_launch(int rank, int nranks, ncclComm_t comm, hipStream_t stream,
                              void* const* d_in, const int* in_sizes, int n_in,
                              void* d_out, int out_size,
                              const int* in_dtypes, int out_dtype,
                              void* const* d_out_all, void* const* d_ws_all, size_t ws_bytes) {
    (void)comm; (void)in_dtypes; (void)out_dtype; (void)d_out;
    if (nranks != NR || n_in != 2 || in_sizes[0] != M_ * KS || in_sizes[1] != KS * N_ || out_size != M_ * N_ || ws_bytes < WS_USED || (unsigned)rank >= (unsigned)NR) {
        fprintf(stderr, "kernel_launch: unexpected problem shape\n");
        return;
    }
    Params p;
    p.x = d_in[0]; p.w = d_in[1];
    for (int i = 0; i < NR; ++i) { p.out[i] = d_out_all[i]; p.ws[i] = d_ws_all[i]; }
    p.rank = rank; p.pad = 0;
    fused_kernel<<<dim3(NWG), dim3(512), 0, stream>>>(p);
}
```

```cpp
#include <hip/hip_runtime.h>
#include <rccl/rccl.h>
#include <cstdio>

#define LAS __attribute__((address_space(3)))
#define GAS __attribute__((address_space(1)))
typedef short bf16x8 __attribute__((ext_vector_type(8)));
typedef float f32x4 __attribute__((ext_vector_type(4)));
typedef unsigned u32x4 __attribute__((ext_vector_type(4)));
typedef unsigned u32x2 __attribute__((ext_vector_type(2)));
typedef __amdgpu_buffer_rsrc_t rsrc_t;

constexpr int NR = 8;
constexpr int M_ = 4096, N_ = 16384;
constexpr int KS = 512;
constexpr int NS = 2048;
constexpr int NT = 64;
constexpr int NGEMM = 128;
constexpr int NWG = 256;

constexpr unsigned OFF_CNT  = 0;
constexpr unsigned OFF_TMO  = 16384;
constexpr unsigned OFF_FLAG = 65536;
constexpr unsigned FLAG_REC = 1024, FLAG_W = 512;
constexpr unsigned OFF_AMAX = 131072;
constexpr unsigned OFF_TICK = 143360;
constexpr unsigned OFF_PCNT = 147456;
constexpr unsigned OFF_PFLG = 163840;
constexpr unsigned OFF_PUF  = 188416;
constexpr unsigned OFF_MFLG = 176128;
constexpr unsigned OFF_DONE = 786432;
constexpr unsigned OFF_MF   = OFF_DONE + 65536;
constexpr unsigned OFF_DBG  = 262144;
constexpr unsigned CTRL_BYTES = 1u << 20;
constexpr unsigned XT_STEP = 512u * 1024u, WT_STEP = 256u * 1024u, BLK = 32768u, HALFB = 16384u;
constexpr unsigned OFF_XT = CTRL_BYTES;
constexpr unsigned XT_BYTES = NT * XT_STEP;
constexpr unsigned OFF_WT = OFF_XT + XT_BYTES;
constexpr unsigned WT_BYTES = NT * WT_STEP;
constexpr unsigned UNIT = 9216;
constexpr unsigned OFF_PX = OFF_WT + WT_BYTES;
constexpr unsigned OFF_PW = OFF_PX + 512 * UNIT;
constexpr unsigned OFF_MSK = OFF_PW + 2048 * UNIT;
constexpr unsigned OFF_COD = OFF_MSK + 1024 * 1024;
constexpr unsigned OFF_LM = OFF_COD + 1024 * 8192;
constexpr unsigned WS_USED = OFF_LM + 8 * 1024 * 1024;
constexpr unsigned OUT_BYTES = (unsigned)M_ * N_;
constexpr unsigned X_BYTES = (unsigned)M_ * KS * 2, W_BYTES = (unsigned)KS * N_ * 2;

constexpr unsigned long long DEADLINE = 100000000ull;
constexpr int AUX_SYS = 17;
#define GEMM_AUX 17
#define PPP 16
#define PPO 40
#define PPM 17
#define PACE_W 800
#define PACE_W0 800
#define PACE_W1 150
#define PACE_X 100
#define PULL_SERIAL 0
#define AMAX_SLEEP 4
#define NPUF 2
#define PACK_AUX 17
#define IN_AUX 2
#define SCH_A 1
#define SCH_B 70
#define CW 3
#define OUT_AUX 0
#define LOC_AUX 17

constexpr unsigned LDS_STAGE = 8 * HALFB;
constexpr unsigned LDS_FLAG = LDS_STAGE;
constexpr unsigned LDS_RED = LDS_FLAG + 512;
constexpr unsigned EX_B = 8192, EX_M = EX_B + 8192, EX_P = EX_M + 1024, EX_E = EX_P + 1024, EX_BYTES = EX_E + 256, EX_LUT = 0;
constexpr unsigned LDS_BYTES = 2048 + 8 * EX_BYTES;
static_assert(LDS_RED + 160 <= LDS_BYTES && LDS_BYTES <= 163840, "lds");
static_assert(OFF_PUF + 8 * NPUF * 384 <= OFF_DBG, "flags");
static_assert(7 * PPO <= 128 * CW && 7 * PPM <= 128 * CW && PPP <= PPM, "waves");

struct Params {
    const void* x; const void* w;
    void* out[NR]; void* ws[NR];
    int rank; int pad;
};

__device__ __forceinline__ rsrc_t mk_rsrc(const void* p, unsigned bytes) {
    return __builtin_amdgcn_make_buffer_rsrc((void*)p, 0, bytes, 0x00020000);
}
#define WAIT_V(n) asm volatile("s_waitcnt vmcnt(" #n ")" ::: "memory")
#define WAIT_L(n) asm volatile("s_waitcnt lgkmcnt(" #n ")" ::: "memory")
#define BAR __builtin_amdgcn_s_barrier()
#define SCHED __builtin_amdgcn_sched_barrier(0)
#define CFENCE asm volatile("" ::: "memory")

#define STAMP(ws_, wv_, k_) do {} while (0)
#define CYC(i) do {} while (0)

__device__ __forceinline__ unsigned ld_agent(const void* p) { return __hip_atomic_load((const GAS unsigned*)p, __ATOMIC_RELAXED, __HIP_MEMORY_SCOPE_AGENT); }
__device__ __forceinline__ void st_agent(void* p, unsigned v) { __hip_atomic_store((GAS unsigned*)p, v, __ATOMIC_RELAXED, __HIP_MEMORY_SCOPE_AGENT); }
__device__ __forceinline__ void st_sys(void* p, unsigned v) { __hip_atomic_store((GAS unsigned*)p, v, __ATOMIC_RELAXED, __HIP_MEMORY_SCOPE_SYSTEM); }

struct Spin {
    unsigned long long t0; unsigned n;
    __device__ __forceinline__ Spin() : t0(__builtin_amdgcn_s_memrealtime()), n(0) {}
    __device__ __forceinline__ bool fail(char* myws, unsigned call) {
        if ((++n & 15u) == 0 && (__builtin_amdgcn_readfirstlane(ld_agent(myws + OFF_TMO)) == call || __builtin_amdgcn_s_memrealtime() - t0 > DEADLINE)) {
            st_agent(myws + OFF_TMO, call);
            return true;
        }
        __builtin_amdgcn_s_sleep(1);
        return false;
    }
};

#define DPP0(x, ctrl, rm) (unsigned)__builtin_amdgcn_update_dpp(0, (int)(x), ctrl, rm, 0xf, false)
__device__ __forceinline__ unsigned wave_scan(unsigned x) {
    x += DPP0(x, 0x111, 0xf); x += DPP0(x, 0x112, 0xf); x += DPP0(x, 0x114, 0xf); x += DPP0(x, 0x118, 0xf);
    x += DPP0(x, 0x142, 0xa); x += DPP0(x, 0x143, 0xc);
    return x;
}
__device__ __forceinline__ unsigned wave_max(unsigned x) {
    x = max(x, DPP0(x, 0x111, 0xf)); x = max(x, DPP0(x, 0x112, 0xf)); x = max(x, DPP0(x, 0x114, 0xf)); x = max(x, DPP0(x, 0x118, 0xf));
    x = max(x, DPP0(x, 0x142, 0xa)); x = max(x, DPP0(x, 0x143, 0xc));
    return __builtin_amdgcn_readlane(x, 63);
}

struct Sched {
    unsigned w[7][64];
    constexpr Sched() : w{} {
        for (int q = 0; q < 7; ++q) {
            int key[136] = {}, id[136] = {}, n = 0;
            for (int j = 0; j < 8; ++j) {
                int lo = 1 << 30;
                for (int rt = 0; rt < 16; ++rt) { const int k = SCH_B * rt + 106 * (7 * j + (q + SCH_A * rt) % 7); key[n] = 2 * k + 1; id[n++] = j * 24 + 8 + rt; lo = k < lo ? k : lo; }
                key[n] = 2 * lo; id[n++] = -1 - j;
            }
            int at = 0;
            for (int o = 0; o < 136; ++o) {
                int best = -1;
                for (int i = 0; i < 136; ++i) if (key[i] >= 0 && (best < 0 || key[i] < key[best])) best = i;
                key[best] = -1;
                const int cnt = id[best] < 0 ? 8 : 1, first = id[best] < 0 ? (-1 - id[best]) * 24 : id[best];
                for (int c = 0; c < cnt; ++c, ++at) w[q][at >> 2] |= (unsigned)(first + c) << (8 * (at & 3));
            }
        }
    }
};
__constant__ const Sched SCHED_TAB;
__device__ __forceinline__ int step_of(int i, int rank, int rt) {
    if (i < 8) return rank * 8 + i;
    const int z = i - 8, j = z / 7, p = z - j * 7, q = (p + 7 * 16 * SCH_A - SCH_A * rt) % 7;
    return (((rank + 1 + q) & 7) << 3) + j;
}

__device__ __forceinline__ void x_load(u32x4 (&v)[8], rsrc_t rx, int j, int g, int lane) {
    const unsigned voff = (unsigned)(lane & 15) * (KS * 2) + (unsigned)(lane >> 4) * 16;
    const unsigned soff = (unsigned)g * 64 * (KS * 2) + (unsigned)j * 128;
#pragma unroll
    for (int r = 0; r < 4; ++r)
#pragma unroll
        for (int kk = 0; kk < 2; ++kk) v[r * 2 + kk] = __builtin_amdgcn_raw_buffer_load_b128(rx, voff + kk * 64, soff + r * 16 * (KS * 2), IN_AUX);
}
__device__ __forceinline__ void w_load(u32x2 (&v)[16], rsrc_t rw, int j, int d, int ct, int kk, int G, int lane) {
    const unsigned voff = (unsigned)(lane >> 4) * 8 * (N_ * 2) + (unsigned)(lane & 15) * 8;
    const unsigned soff = (unsigned)(j * 64 + kk * 32) * (N_ * 2) + (unsigned)(d * NS + ct * 256 + G * 128) * 2;
#pragma unroll
    for (int jj = 0; jj < 8; ++jj)
#pragma unroll
        for (int g = 0; g < 2; ++g) v[g * 8 + jj] = __builtin_amdgcn_raw_buffer_load_b64(rw, voff + g * 128, soff + jj * (N_ * 2), IN_AUX);
}
__device__ __forceinline__ void w_transpose(u32x4 (&o)[8], const u32x2 (&v)[16]) {
#pragma unroll
    for (int g = 0; g < 2; ++g)
#pragma unroll
        for (int c = 0; c < 4; ++c)
#pragma unroll
            for (int dd = 0; dd < 4; ++dd)
                o[g * 4 + c][dd] = __builtin_amdgcn_perm(v[g * 8 + 2 * dd + 1][c >> 1], v[g * 8 + 2 * dd][c >> 1], (c & 1) ? 0x07060302u : 0x05040100u);
}
__device__ __forceinline__ unsigned x_img(int t, int g) { return OFF_XT + (unsigned)t * XT_STEP + (unsigned)g * 8192; }
__device__ __forceinline__ unsigned w_img(int t, int e) { return OFF_WT + (unsigned)t * WT_STEP + (unsigned)(e >> 2) * BLK + (unsigned)((e >> 1) & 1) * 8192 + (unsigned)(e & 1) * 1024; }
__device__ __forceinline__ unsigned frag_off(bool isw, unsigned f) { return isw ? ((f >> 1) & 1) * HALFB + (f >> 2) * 4096 + (f & 1) * 2048 : f * 1024; }
__device__ __forceinline__ void img_store(const u32x4 (&v)[8], rsrc_t rd, bool isw, unsigned img, int lane) {
#pragma unroll
    for (int f = 0; f < 8; ++f) __builtin_amdgcn_raw_buffer_store_b128(v[f], rd, lane * 16, img + frag_off(isw, f), LOC_AUX);
}
constexpr unsigned H8 = 0x80808080u, MAXESC = 512;
__device__ __forceinline__ unsigned bits16(unsigned m0, unsigned m1, unsigned m2, unsigned m3, int sh) {
    const unsigned a = __builtin_amdgcn_udot4(m0, 0x08040201u, __builtin_amdgcn_udot4(m1, 0x80402010u, 0u, false), false);
    const unsigned b = __builtin_amdgcn_udot4(m2, 0x08040201u, __builtin_amdgcn_udot4(m3, 0x80402010u, 0u, false), false);
    return (a >> sh) | (b << (8 - sh));
}
__device__ __forceinline__ unsigned esc_count(const unsigned (&bm)[4], unsigned& first) {
    const unsigned c = __builtin_popcount(bm[0]) + __builtin_popcount(bm[1]) + __builtin_popcount(bm[2]) + __builtin_popcount(bm[3]), s = wave_scan(c);
    first = s - c;
    return __builtin_amdgcn_readlane(s, 63);
}
__device__ __forceinline__ void esc_list(const unsigned (&bm)[4], unsigned at, LAS unsigned char* list, int lane) {
#pragma unroll
    for (int w = 0; w < 2; ++w) {
        unsigned long long z = ((unsigned long long)bm[2 * w + 1] << 32) | bm[2 * w];
        while (__builtin_amdgcn_ballot_w64(z != 0) != 0) {
            if (z != 0) {
                const unsigned b = __builtin_ctzll(z);
                *(LAS unsigned short*)(list + min(at, MAXESC - 1) * 2) = (unsigned short)(w * 4096 + (b >> 4) * 1024 + lane * 16 + (b & 15));
                ++at; z &= z - 1;
            }
        }
    }
}
constexpr unsigned PK_HDR = 5 * 1024 + 512, PK_RAW = 0xffffu, PK_FIRST = 62;
__device__ __forceinline__ void pack_store(const u32x4 (&v)[8], rsrc_t rs, LAS unsigned char* L, unsigned slot, int lane) {
    unsigned p0[16], e[16], sum = 0;
#pragma unroll
    for (int r = 0; r < 16; ++r) {
        const unsigned lo = __builtin_amdgcn_perm(v[r >> 1][2 * (r & 1) + 1], v[r >> 1][2 * (r & 1)], 0x06040200u);
        const unsigned hi = __builtin_amdgcn_perm(v[r >> 1][2 * (r & 1) + 1], v[r >> 1][2 * (r & 1)], 0x07050301u);
        p0[r] = (hi & H8) | (lo & ~H8);
        e[r] = ((hi << 1) & 0xfefefefeu) | ((lo >> 7) & 0x01010101u);
        sum = __builtin_amdgcn_sad_u8(e[r], 0u, sum);
    }
#pragma unroll
    for (int r = 0; r < 4; ++r) *(LAS u32x4*)(L + r * 1024 + lane * 16) = (u32x4){e[4 * r], e[4 * r + 1], e[4 * r + 2], e[4 * r + 3]};
    const int tot = (int)__builtin_amdgcn_readlane(wave_scan(sum), 63);
    const unsigned base = (unsigned)min(max((tot - 16794) >> 12, 0), 247), base4 = base * 0x01010101u;
    u32x4 p2 = {0u, 0u, 0u, 0u}; u32x2 p1 = {0u, 0u};
    unsigned ms[16];
#pragma unroll
    for (int r = 0; r < 16; ++r) {
        const unsigned z = ((e[r] | H8) - (base4 & ~H8)) ^ ((e[r] ^ ~base4) & H8);
        ms[r] = (((z & 0x78787878u) + 0x78787878u) | z) & H8;
        p2[r >> 2] |= (z & 0x03030303u) << (2 * (r & 3));
        p1[r >> 3] |= (r & 7) >= 2 ? (z & 0x04040404u) << ((r & 7) - 2) : (z & 0x04040404u) >> (2 - (r & 7));
    }
    unsigned bm[4] = {bits16(ms[0], ms[1], ms[2], ms[3], 7) | (bits16(ms[4], ms[5], ms[6], ms[7], 7) << 16),
                      bits16(ms[8], ms[9], ms[10], ms[11], 7) | (bits16(ms[12], ms[13], ms[14], ms[15], 7) << 16), 0u, 0u};
    unsigned at;
    const unsigned n = esc_count(bm, at);
    bool bad = n > 254;
    *(LAS u32x2*)(L + 4096 + lane * 8) = p1;
    if (!bad) {
        esc_list(bm, at, L + 5120, lane);
        CFENCE;
        bool b = false;
#pragma nounroll
        for (unsigned i = lane; i < n; i += 64) {
            const unsigned pos = *(const LAS unsigned short*)(L + 5120 + i * 2), ev = L[pos];
            unsigned d = base - 1 - ev;
            if (ev == base + 8) d = 14; else if (ev == 0) d = 15; else b |= d > 13;
            *(LAS unsigned short*)(L + 4096 + 516 + i * 2) = (unsigned short)(pos | (d << 12));
        }
        bad = __builtin_amdgcn_ballot_w64(b) != 0;
    }
    if (lane == 0) *(LAS unsigned*)(L + 4096 + 512) = base | ((bad ? PK_RAW : n) << 8);
    CFENCE;
    __builtin_amdgcn_raw_buffer_store_b128(*(const LAS u32x4*)(L + 4096 + lane * 16), rs, lane * 16 + 1024, slot + 4096, PACK_AUX);
    if (!bad) {
#pragma unroll
        for (int r = 0; r < 4; ++r) __builtin_amdgcn_raw_buffer_store_b128((u32x4){p0[4 * r], p0[4 * r + 1], p0[4 * r + 2], p0[4 * r + 3]}, rs, lane * 16 + r * 1024, slot, PACK_AUX);
        __builtin_amdgcn_raw_buffer_store_b128(p2, rs, lane * 16, slot + 4096, PACK_AUX);
    } else {
#pragma unroll
        for (int f = 0; f < 8; ++f) __builtin_amdgcn_raw_buffer_store_b128(v[f], rs, lane * 16, slot + (f < 5 ? f : f + 1) * 1024, PACK_AUX);
    }
}
constexpr unsigned PL_SLOT = 8192;
struct Puller {
    rsrc_t rp, rs; char* myws; int rank, src, q, lane; unsigned call, seen, tab;
    __device__ __forceinline__ int unit_of(int tk) const { return (int)((__builtin_amdgcn_readlane(tab, tk >> 4) >> (2 * (tk & 12))) & 0xffu) * 4 + (tk & 3); }
    __device__ __forceinline__ int xg(int i) const { return i - 32; }
    __device__ __forceinline__ unsigned unit_id(int u) const { const int j = u / 96, i = u % 96; return i < 32 ? 512 + (rank * 8 + j) * 32 + i : j * 64 + xg(i); }
    __device__ __forceinline__ unsigned flag_of(int u) const { const int j = u / 96, i = u % 96; return OFF_FLAG + (src * 8 + j) * FLAG_REC + (i < 32 ? FLAG_W + i * 4 : xg(i) * 4); }
    __device__ __forceinline__ void issue(LAS unsigned char* b, int u) {
        const int j = u / 96, i = u % 96, need = j * 2 + (i >= 32);
        if (!((seen >> need) & 1u)) {
            Spin t0;
            for (;;) {
                CFENCE;
                const unsigned sum = OFF_PFLG + (src * 8 + j) * 128 + (i >= 32 ? 0 : 4);
                const unsigned f = __builtin_amdgcn_raw_buffer_load_b32(rs, (lane & 1) && j < NPUF ? OFF_PUF + ((src * NPUF + j) * 96 + i) * 4 : sum, 0, AUX_SYS);
                if (__builtin_amdgcn_readlane(f, 0) >= call) { seen |= 1u << need; break; }
                if (__builtin_amdgcn_readlane(f, 1) >= call) break;
                if (t0.fail(myws, call)) break;
            }
            CFENCE;
        }
        const unsigned id = unit_id(u), slot = OFF_PX + id * UNIT;
        __builtin_amdgcn_raw_ptr_buffer_load_lds(rp, (LAS void*)(b + 4096), 16, lane < 40 ? lane * 16 : 0xf0000000u, slot + 4096, 1024, AUX_SYS);
        __builtin_amdgcn_raw_ptr_buffer_load_lds(rp, (LAS void*)(b + 4096), 16, lane * 16, slot + 4096, 0, AUX_SYS);
        __builtin_amdgcn_raw_ptr_buffer_load_lds(rp, (LAS void*)b, 16, lane * 16, slot, 0, AUX_SYS);
        __builtin_amdgcn_raw_ptr_buffer_load_lds(rp, (LAS void*)b, 16, lane * 16, slot, 1024, AUX_SYS);
        __builtin_amdgcn_raw_ptr_buffer_load_lds(rp, (LAS void*)b, 16, lane * 16, slot, 2048, AUX_SYS);
        __builtin_amdgcn_raw_ptr_buffer_load_lds(rp, (LAS void*)b, 16, lane * 16, slot, 3072, AUX_SYS);
    }
    __device__ __forceinline__ void finish(LAS unsigned char* b, int u) {
        const int j = u / 96, i = u % 96, t = src * 8 + j;
        const bool isw = i < 32;
        const unsigned img = isw ? w_img(t, i) : x_img(t, xg(i));
        const unsigned h = *(const LAS unsigned*)(b + PK_HDR);
        u32x4 v[8];
#pragma unroll
        for (int r = 0; r < 5; ++r) v[r] = *(const LAS u32x4*)(b + r * 1024 + lane * 16);
        const u32x2 p1 = *(const LAS u32x2*)(b + 5120 + lane * 8);
        const unsigned meta = __builtin_amdgcn_readfirstlane(h), n = meta >> 8, base = meta & 0xffu;
        if (n <= 254) {
            if (n > PK_FIRST) {
                __builtin_amdgcn_raw_ptr_buffer_load_lds(rp, (LAS void*)(b + 4096), 16, lane * 16, OFF_PX + unit_id(u) * UNIT + 4096, 1024, AUX_SYS);
                WAIT_V(0);
            }
            const unsigned base4 = base * 0x01010101u;
            const u32x4 p2 = v[4];
            u32x4 p0[4], e[4];
#pragma unroll
            for (int r = 0; r < 16; ++r) {
                const unsigned hb = (r & 7) >= 2 ? p1[r >> 3] >> ((r & 7) - 2) : p1[r >> 3] << (2 - (r & 7));
                e[r >> 2][r & 3] = (((p2[r >> 2] >> (2 * (r & 3))) & 0x03030303u) | (hb & 0x04040404u)) + base4;
            }
#pragma unroll
            for (int r = 0; r < 4; ++r) { p0[r] = v[r]; *(LAS u32x4*)(b + r * 1024 + lane * 16) = e[r]; }
            CFENCE;
#pragma nounroll
            for (unsigned k = lane; k < n; k += 64) {
                const unsigned en = *(const LAS unsigned short*)(b + PK_HDR + 4 + k * 2), d = en >> 12;
                b[en & 0xfffu] = (unsigned char)(d == 14 ? base + 8 : d == 15 ? 0u : base - 1 - d);
            }
            CFENCE;
#pragma unroll
            for (int r = 0; r < 4; ++r) e[r] = *(const LAS u32x4*)(b + r * 1024 + lane * 16);
#pragma unroll
            for (int r = 0; r < 16; ++r) {
                const unsigned pp = p0[r >> 2][r & 3], ee = e[r >> 2][r & 3];
                const unsigned lo = (pp & ~H8) | ((ee << 7) & H8), hi = (pp & H8) | ((ee >> 1) & ~H8);
                v[r >> 1][2 * (r & 1)] = __builtin_amdgcn_perm(hi, lo, 0x05010400u); v[r >> 1][2 * (r & 1) + 1] = __builtin_amdgcn_perm(hi, lo, 0x07030602u);
            }
        } else {
            const unsigned slot = OFF_PX + unit_id(u) * UNIT;
            WAIT_L(0);
            __builtin_amdgcn_raw_ptr_buffer_load_lds(rp, (LAS void*)b, 16, lane * 16, slot + 6144, 0, AUX_SYS);
            __builtin_amdgcn_raw_ptr_buffer_load_lds(rp, (LAS void*)b, 16, lane * 16, slot + 6144, 1024, AUX_SYS);
            __builtin_amdgcn_raw_ptr_buffer_load_lds(rp, (LAS void*)b, 16, lane * 16, slot + 6144, 2048, AUX_SYS);
            WAIT_V(0);
#pragma unroll
            for (int r = 5; r < 8; ++r) v[r] = *(const LAS u32x4*)(b + (r - 5) * 1024 + lane * 16);
        }
        img_store(v, rs, isw, img, lane);
    }
};
static_assert(OFF_PW == OFF_PX + 512 * UNIT, "unit ids run through the x and the w slots");

#define DRAW(r, tick) { unsigned long long ex_; asm volatile("s_mov_b64 %1, exec\n\ts_mov_b64 exec, 1\n\tglobal_atomic_add %0, %2, %3, %4 sc0\n\ts_mov_b64 exec, %1" \
        : "=&v"(r), "=&s"(ex_) : "v"(0u), "v"(1u), "s"(tick) : "memory"); }
#define TAKE(t, r, N) { unsigned t_; asm volatile("s_waitcnt vmcnt(" #N ")\n\tv_readfirstlane_b32 %0, %1" : "=s"(t_) : "v"(r) : "memory"); t = (int)(t_ - tbase); }
constexpr int NU_IN = 768, NU_ALL = NU_IN + 256;

__device__ __forceinline__ void pull_role(const Params& p, LAS unsigned char* L, int pi, int lane, unsigned call, int wv) {
    Puller P; P.rank = p.rank; P.q = pi % 7; P.src = (p.rank + 1 + P.q) & 7; P.lane = lane; P.seen = 0; P.call = call;
    P.tab = SCHED_TAB.w[P.q][lane];
    asm volatile("" : "+v"(P.tab));
    P.myws = (char*)p.ws[p.rank]; P.rs = mk_rsrc(P.myws, WS_USED); P.rp = mk_rsrc(p.ws[P.src], WS_USED);
    char* tick = P.myws + OFF_TICK + P.q * 128;
    const unsigned tbase = (call - 1) * (NU_ALL + 2 * PPM);
    unsigned r0, r1, pf = 0;
    if (pi >= 7 * PPP) {
        Spin t0;
        while ((int)(__builtin_amdgcn_readfirstlane(ld_agent(tick)) - tbase) < NU_IN) { if (t0.fail(P.myws, call)) break; __builtin_amdgcn_s_sleep(32); }
    }
    DRAW(r0, tick); DRAW(r1, tick);
    int u0, u1, sl = 0;
    TAKE(u0, r0, 0); TAKE(u1, r1, 0);
    if ((unsigned)u0 < (unsigned)NU_IN) P.issue(L, P.unit_of(u0));
    WAIT_V(0);
    STAMP(P.myws, wv, 10);
    while ((unsigned)u0 < (unsigned)NU_IN) {
        unsigned r2;
        if ((unsigned)u1 < (unsigned)NU_IN) { P.issue(L + (sl ^ 1) * PL_SLOT, P.unit_of(u1)); DRAW(r2, tick); WAIT_V(7); }
        else { DRAW(r2, tick); WAIT_V(0); }
        if (pf != 0 && lane == 0) st_sys(P.myws + pf, call);
        P.finish(L + sl * PL_SLOT, P.unit_of(u0));
        pf = P.flag_of(P.unit_of(u0));
        u0 = u1; sl ^= 1;
        TAKE(u1, r2, 8);
    }
    WAIT_V(0);
    if (pf != 0 && lane == 0) st_sys(P.myws + pf, call);
    pf = 0;
    STAMP(P.myws, wv, 9);
    unsigned long long seen = 0; int sblk = -1;
#define MASK_ISSUE(b, u) { const int mu_ = (u) - NU_IN; \
        if (!((seen >> (mu_ & 63)) & 1) || (mu_ >> 6) != sblk) { Spin t0; sblk = mu_ >> 6;        \
            for (;;) { CFENCE; const unsigned f = __builtin_amdgcn_raw_buffer_load_b32(P.rs, lane * 4, OFF_MFLG + (P.src * 256 + sblk * 64) * 4, AUX_SYS); \
                seen = __builtin_amdgcn_ballot_w64(f >= call); \
                if ((seen >> (mu_ & 63)) & 1) break; if (t0.fail(P.myws, call)) break; } \
            CFENCE; } \
        __builtin_amdgcn_raw_ptr_buffer_load_lds(P.rp, (LAS void*)(b), 16, lane * 16, OFF_MSK + mu_ * 4096, 0, AUX_SYS); \
        __builtin_amdgcn_raw_ptr_buffer_load_lds(P.rp, (LAS void*)(b), 16, lane * 16, OFF_MSK + mu_ * 4096, 1024, AUX_SYS); \
        __builtin_amdgcn_raw_ptr_buffer_load_lds(P.rp, (LAS void*)(b), 16, lane * 16, OFF_MSK + mu_ * 4096, 2048, AUX_SYS); \
        __builtin_amdgcn_raw_ptr_buffer_load_lds(P.rp, (LAS void*)(b), 16, lane * 16, OFF_MSK + mu_ * 4096, 3072, AUX_SYS); }
    if ((unsigned)u0 < (unsigned)NU_ALL) MASK_ISSUE(L + sl * PL_SLOT, u0);
    WAIT_V(0);
    STAMP(P.myws, wv, 16);
    while ((unsigned)u0 < (unsigned)NU_ALL) {
        unsigned r2;
        if ((unsigned)u1 < (unsigned)NU_ALL) { MASK_ISSUE(L + (sl ^ 1) * PL_SLOT, u1); DRAW(r2, tick); WAIT_V(5); }
        else { DRAW(r2, tick); WAIT_V(0); }
        if (pf != 0 && lane == 0) st_sys(P.myws + pf, call);
#pragma unroll
        for (int r = 0; r < 4; ++r)
            __builtin_amdgcn_raw_buffer_store_b128(*(const LAS u32x4*)(L + sl * PL_SLOT + r * 1024 + lane * 16), P.rs, lane * 16 + r * 1024, OFF_LM + P.src * 1048576 + (u0 - NU_IN) * 4096, LOC_AUX);
        pf = OFF_MF + (P.src * 256 + u0 - NU_IN) * 4;
        u0 = u1; sl ^= 1;
        TAKE(u1, r2, 4);
    }
    WAIT_V(0);
    if (pf != 0 && lane == 0) st_sys(P.myws + pf, call);
    STAMP(P.myws, wv, 17);
}

__device__ __forceinline__ u32x2 lut_expand(unsigned byte) { u32x2 s = {0u, 0u}; unsigned c = 0;
#pragma unroll
    for (int i = 0; i < 8; ++i) { const unsigned b = (byte >> i) & 1u; s[i >> 2] |= (b ? c : 0x0cu) << (8 * (i & 3)); c += b; } return s; }
__device__ __forceinline__ unsigned lut_compact(unsigned nib) { unsigned s = 0x0c0c0c0cu, c = 0;
#pragma unroll
    for (int i = 0; i < 4; ++i) if ((nib >> i) & 1u) { s = (s & ~(0xffu << (8 * c))) | ((unsigned)i << (8 * c)); ++c; } return s; }
__device__ __forceinline__ unsigned popc4(const u32x4& m) { return __builtin_popcount(m[0]) + __builtin_popcount(m[1]) + __builtin_popcount(m[2]) + __builtin_popcount(m[3]); }
__device__ __forceinline__ unsigned nib4(unsigned m, int i) { return (i == 0 ? m << 2 : m >> (4 * i - 2)) & 0x3cu; }
__device__ __forceinline__ unsigned out_voff(int lane) { return (unsigned)(lane & 15) * N_ + (unsigned)(lane >> 4) * 16; }
__device__ __forceinline__ unsigned out_soff(int src, int wt) {
    const int tile = wt >> 3, w = wt & 7;
    return (unsigned)((tile >> 3) * 256 + (w >> 2) * 64) * N_ + (unsigned)(src * NS + (tile & 7) * 256 + (w & 3) * 64);
}
__device__ __forceinline__ constexpr unsigned chunk_soff(int ch) { return (unsigned)((ch >> 2) * 128 + (ch & 3) * 16) * N_; }

__device__ __forceinline__ void expand_role(const Params& p, LAS unsigned char* lds, LAS unsigned char* L, int ei, int lane, unsigned call, int wv) {
    const int rank = p.rank, q = ei % 7, src = (rank + 1 + q) & 7;
    char* myws = (char*)p.ws[rank];
    char* pws = (char*)p.ws[src];
    const rsrc_t rs = mk_rsrc(myws, WS_USED), ro = mk_rsrc(p.out[rank], OUT_BYTES);
#pragma unroll
    for (int i = 0; i < 4; ++i) *(LAS u32x2*)(lds + EX_LUT + (i * 64 + lane) * 8) = lut_expand(i * 64 + lane);
    char* tick = myws + OFF_TICK + (8 + q) * 128;
    const unsigned tbase = (call - 1) * (1024 + 2 * PPO), tag = call & 0x3ffffu;
    bool dead = false;
#define ENT_OFF(t) ((lane & 2) ? OFF_MF + (src * 256 + ((t) >> 2)) * 4 : OFF_DONE + (src * 1024 + (t)) * 8 + (lane & 1) * 4)
#define ENT_OK(e, x, f) (((e) >> 14) == tag && ((x) >> 22) == (tag & 0x3ffu) && (f) >= call)
#define ENT_FETCH(t) __builtin_amdgcn_raw_ptr_buffer_load_lds(rs, (LAS void*)(L + EX_E), 4, ENT_OFF(t), 0, 0, AUX_SYS)
#define EX_ISSUE(t, e, x, f, n, xo) { unsigned e_ = (e), x_ = (x), f_ = (f); \
        if (!ENT_OK(e_, x_, f_) && !dead) { Spin t0_; \
            for (;;) { CFENCE; const unsigned g_ = __builtin_amdgcn_raw_buffer_load_b32(rs, ENT_OFF(t), 0, AUX_SYS); \
                e_ = __builtin_amdgcn_readlane(g_, 0); x_ = __builtin_amdgcn_readlane(g_, 1); f_ = __builtin_amdgcn_readlane(g_, 2); \
                if (ENT_OK(e_, x_, f_)) break; if (t0_.fail(myws, call)) { dead = true; break; } __builtin_amdgcn_s_sleep(8); } \
            CFENCE; } \
        n = min(e_ & 0x3fffu, 8192u); xo = x_; \
        const unsigned by_ = min((((x_ >> 21) & 1u) ? n : ((n + 15) >> 4) * 12 + (x_ & 0x3fffu)) + 15 & ~15u, 8192u); \
        const rsrc_t rc_ = mk_rsrc(pws + OFF_COD + (unsigned)(t) * 8192, by_); \
        _Pragma("nounroll") for (unsigned r_ = 0; r_ < by_; r_ += 1024) \
            __builtin_amdgcn_raw_ptr_buffer_load_lds(rc_, (LAS void*)(L + EX_B + r_), 16, lane * 16, r_, 0, AUX_SYS); \
        __builtin_amdgcn_raw_ptr_buffer_load_lds(rs, (LAS void*)(L + EX_M), 16, lane * 16, OFF_LM + src * 1048576 + (t) * 1024, 0, AUX_SYS); }
    unsigned r0, n0 = 0, x0 = 0;
    DRAW(r0, tick);
    int t0;
    TAKE(t0, r0, 0);
    if ((unsigned)t0 < 1024u) EX_ISSUE(t0, 0u, 0u, 0u, n0, x0);
    int t1;
    { unsigned r1; DRAW(r1, tick); TAKE(t1, r1, 0); }
    STAMP(myws, wv, 18);
    while ((unsigned)t0 < 1024u) {
        if ((unsigned)t1 < 1024u) ENT_FETCH(t1);
        const u32x4 m = *(const LAS u32x4*)(L + EX_M + lane * 16);
        if ((x0 >> 21) & 1u) {
#pragma nounroll
            for (unsigned r = 0; r < n0; r += 1024) *(LAS u32x4*)(L + r + lane * 16) = *(const LAS u32x4*)(L + EX_B + r + lane * 16);
        } else {
            const unsigned G = (n0 + 15) >> 4, B4 = ((x0 >> 14) & 0x7fu) * 0x01010101u;
            unsigned bm[4] = {0u, 0u, 0u, 0u};
#define UNFOLD(it0, n) { unsigned o_[n][3]; \
                _Pragma("unroll") for (int i = 0; i < n; ++i) { const LAS unsigned* sB = (const LAS unsigned*)(L + EX_B + ((it0 + i) * 64 + lane) * 12); o_[i][0] = sB[0]; o_[i][1] = sB[1]; o_[i][2] = sB[2]; } \
                SCHED; \
                _Pragma("unroll") for (int i = 0; i < n; ++i) { const int it = it0 + i; const unsigned g = it * 64 + lane; \
                    const unsigned s0 = o_[i][0] & 0x3f3f3f3fu, s1 = o_[i][1] & 0x3f3f3f3fu, s2 = o_[i][2] & 0x3f3f3f3fu; \
                    const unsigned s3 = ((o_[i][0] >> 6) & 0x03030303u) | ((o_[i][1] >> 4) & 0x0c0c0c0cu) | ((o_[i][2] >> 2) & 0x30303030u); \
                    *(LAS u32x4*)(L + g * 16) = (u32x4){s0 + B4, s1 + B4, s2 + B4, s3 + B4}; \
                    const unsigned b_ = bits16((s0 + 0x01010101u) & 0x40404040u, (s1 + 0x01010101u) & 0x40404040u, (s2 + 0x01010101u) & 0x40404040u, (s3 + 0x01010101u) & 0x40404040u, 6); \
                    bm[it >> 1] |= (g < G ? b_ : 0u) << (16 * (it & 1)); } }
            UNFOLD(0, 4);
            if (G > 256) { UNFOLD(4, 1); if (G > 320) UNFOLD(5, 3); }
            CYC(0);
            unsigned at;
            const unsigned ne = min(esc_count(bm, at), MAXESC);
            esc_list(bm, at, L + EX_P, lane);
            CFENCE;
#pragma nounroll
            for (unsigned i = lane; i < ne; i += 64) L[*(const LAS unsigned short*)(L + EX_P + i * 2)] = L[EX_B + G * 12 + i];
            CFENCE;
        }
        const unsigned cnt = popc4(m);
        unsigned off = wave_scan(cnt) - cnt + (unsigned)(size_t)L;
        WAIT_L(0);
        CYC(1);
        unsigned n1 = 0, x1 = 0;
        if ((unsigned)t1 < 1024u) {
            WAIT_V(0);
            const u32x4 ent = *(const LAS u32x4*)(L + EX_E);
            EX_ISSUE(t1, __builtin_amdgcn_readfirstlane(ent[0]), __builtin_amdgcn_readfirstlane(ent[1]), __builtin_amdgcn_readfirstlane(ent[2]), n1, x1);
        }
        unsigned r2;
        DRAW(r2, tick);
        CYC(2);
        const unsigned so = out_soff(src, t0), vo = out_voff(lane);
        u32x2 sel[16]; unsigned of[16], w0[16], w1[16], w2[16];
#pragma unroll
        for (int g = 0; g < 16; ++g) {
            const unsigned b8 = ((g & 3) == 0 ? m[g >> 2] << 3 : m[g >> 2] >> (8 * (g & 3) - 3)) & 0x7f8u;
            sel[g] = *(const LAS u32x2*)(lds + EX_LUT + b8);
            const LAS unsigned* w = (const LAS unsigned*)(size_t)(off & ~3u);
            w0[g] = w[0]; w1[g] = w[1]; w2[g] = w[2]; of[g] = off;
            off += __builtin_popcount(b8);
        }
        SCHED;
        u32x4 o[8];
#pragma unroll
        for (int g = 0; g < 16; ++g) {
            const unsigned a0 = __builtin_amdgcn_alignbyte(w1[g], w0[g], of[g]), a1 = __builtin_amdgcn_alignbyte(w2[g], w1[g], of[g]);
            o[g >> 1][(g & 1) * 2] = __builtin_amdgcn_perm(a1, a0, sel[g][0]); o[g >> 1][(g & 1) * 2 + 1] = __builtin_amdgcn_perm(a1, a0, sel[g][1]);
        }
        CYC(3);
        if (dead) {
#pragma unroll
            for (int ch = 0; ch < 8; ++ch) o[ch] = (u32x4){0x7f7f7f7fu, 0x7f7f7f7fu, 0x7f7f7f7fu, 0x7f7f7f7fu};
            CFENCE;
        }
#pragma unroll
        for (int ch = 0; ch < 8; ++ch) __builtin_amdgcn_raw_buffer_store_b128(o[ch], ro, vo, so + chunk_soff(ch), OUT_AUX);
        CYC(4);
        int t2;
        TAKE(t2, r2, 8);
        CYC(5);
        t0 = t1; n0 = n1; x0 = x1; t1 = t2;
    }
    STAMP(myws, wv, 19);
}

__device__ __forceinline__ unsigned get_call(unsigned callv) { unsigned c; asm volatile("v_readfirstlane_b32 %0, %1" : "=s"(c) : "v"(callv)); return c + 1u; }

__device__ __forceinline__ void comm_role(const Params& p, LAS unsigned char* lds, int c, int wid, int lane, unsigned callv) {
    const int rank = p.rank;
    char* myws = (char*)p.ws[rank];
    const rsrc_t rs = mk_rsrc(myws, WS_USED);
    const int wv = (NGEMM + c) * 8 + wid; (void)wv;
    STAMP(myws, wv, 0);
    if (wid >= 2) {
        const rsrc_t rx = mk_rsrc(p.x, X_BYTES), rw = mk_rsrc(p.w, W_BYTES);
        const int k0 = (wid - 2) * 128 + c;
        const bool far = k0 < 448;
        const unsigned long long tk0 = __builtin_amdgcn_s_memrealtime();
        for (int u = far ? k0 : k0 - 448; u < (far ? 1792 : 768); u += far ? 448 : 320) {
            const int j = far ? u / 224 : u / 96, r = far ? u % 224 : 224 + u % 96, t = rank * 8 + j;
            const unsigned due = far ? (j > 1 ? j * PACE_W - PACE_W0 : j * PACE_W1) : j * PACE_X;
            while ((unsigned)(__builtin_amdgcn_s_memrealtime() - tk0) < due) __builtin_amdgcn_s_sleep(8);
            u32x4 v[8];
            if (r >= 224 && r < 288) {
                const int g = r - 224;
                x_load(v, rx, j, g, lane);
                pack_store(v, rs, lds + 2048 + wid * EX_BYTES, OFF_PX + (unsigned)(j * 64 + g) * UNIT, lane);
                if (j < NPUF) {
                    WAIT_V(0);
                    if (lane < 8) st_sys((char*)p.ws[lane] + OFF_PUF + ((rank * NPUF + j) * 96 + 32 + g) * 4, get_call(callv));
                    if (j == 0) STAMP(myws, wv, 7);
                }
                img_store(v, rs, false, x_img(t, g), lane);
                WAIT_V(0);
                const unsigned call = get_call(callv);
                unsigned old = 0;
                if (lane == 0) {
                    st_sys(myws + OFF_FLAG + t * FLAG_REC + g * 4, call);
                    old = __hip_atomic_fetch_add((GAS unsigned*)(myws + OFF_PCNT + j * 128), 1u, __ATOMIC_RELAXED, __HIP_MEMORY_SCOPE_AGENT);
                }
                if ((__builtin_amdgcn_readfirstlane(old) & 63) == 63 && lane < 8) st_sys((char*)p.ws[lane] + OFF_PFLG + t * 128, call);
            } else {
                const bool own = r >= 288;
                const int e = own ? r - 288 : r / 7, d = own ? rank : (rank + 1 + r % 7) & 7;
                u32x2 w[16];
                w_load(w, rw, j, d, e >> 2, e & 1, (e >> 1) & 1, lane);
                w_transpose(v, w);
                if (own) {
                    img_store(v, rs, true, w_img(t, e), lane);
                    WAIT_V(0);
                    if (lane == 0) st_sys(myws + OFF_FLAG + t * FLAG_REC + FLAG_W + e * 4, get_call(callv));
                } else {
                    const unsigned id = (unsigned)((d * 8 + j) * 32 + e);
                    pack_store(v, rs, lds + 2048 + wid * EX_BYTES, OFF_PW + id * UNIT, lane);
                    WAIT_V(0);
                    const unsigned call = get_call(callv);
                    if (j < NPUF) { if (lane == 0) st_sys((char*)p.ws[d] + OFF_PUF + ((rank * NPUF + j) * 96 + e) * 4, call); if (j == 0) STAMP(myws, wv, 6); }
                    unsigned old = 0;
                    if (lane == 0) old = __hip_atomic_fetch_add((GAS unsigned*)(myws + OFF_PCNT + (8 + d * 8 + j) * 128), 1u, __ATOMIC_RELAXED, __HIP_MEMORY_SCOPE_AGENT);
                    if ((__builtin_amdgcn_readfirstlane(old) & 31) == 31 && lane == 0) st_sys((char*)p.ws[d] + OFF_PFLG + t * 128 + 4, call);
                }
            }
            STAMP(myws, wv, far ? 1 + (j >> 2) : 3 + (j >> 2));
        }
    }
    if (wid < CW) {
        const unsigned call = get_call(callv);
        const int ci = wid * 128 + c;
        if (ci < 7 * PPM) pull_role(p, lds + 2048 + wid * EX_BYTES, ci, lane, call, wv);
        if (ci < 7 * PPO) expand_role(p, lds, lds + 2048 + wid * EX_BYTES, ci, lane, call, wv);
    }
}

__device__ __forceinline__ void gemm_role(const Params& p, LAS unsigned char* lds, int rt, int ct, int wid, int lane, unsigned callv) {
    const int rank = p.rank;
    char* myws = (char*)p.ws[rank];
    const int tile = rt * 8 + ct;
    STAMP(myws, tile * 8 + wid, 0);
    const unsigned call = get_call(callv);
    const int wv = tile * 8 + wid; (void)wv;
    const int wr = wid >> 2, wc = wid & 3;
    const rsrc_t rs = mk_rsrc(myws, WS_USED);
    const unsigned voff = (unsigned)wid * 2048 + (unsigned)lane * 16;
    const unsigned voffF = lane < 4 ? (unsigned)(rt * 16 + lane * 4) : FLAG_W + (unsigned)(ct * 16 + (lane & 3) * 4);
    LAS unsigned char* ldsw = lds + wid * 2048;
    LAS unsigned char* lda = lds + wr * 8192 + lane * 16;
    LAS unsigned char* ldb = lds + wc * 4096 + lane * 16;
    const int seqv = step_of(lane, rank, rt);
#define SEQ(i) __builtin_amdgcn_readlane(seqv, (i) < NT ? (i) : NT - 1)
#define SA(b, h) (((b) * 2 + (h)) * HALFB)
#define SB(b, h) ((4 + (b) * 2 + (h)) * HALFB)
#define STAGE(bufoff, soff) do { \
        __builtin_amdgcn_raw_ptr_buffer_load_lds(rs, (LAS void*)(ldsw + (bufoff)), 16, voff, (soff), 0, GEMM_AUX); \
        __builtin_amdgcn_raw_ptr_buffer_load_lds(rs, (LAS void*)(ldsw + (bufoff)), 16, voff, (soff), 1024, GEMM_AUX); } while (0)
#define LDA(dst, b, h) do { _Pragma("unroll") for (int m = 0; m < 4; ++m) _Pragma("unroll") for (int k = 0; k < 2; ++k) dst[m][k] = *(const LAS bf16x8*)(lda + SA(b, h) + m * 2048 + k * 1024); } while (0)
#define LDB(dst, b, h) do { _Pragma("unroll") for (int n = 0; n < 2; ++n) _Pragma("unroll") for (int k = 0; k < 2; ++k) dst[n][k] = *(const LAS bf16x8*)(ldb + SB(b, h) + n * 2048 + k * 1024); } while (0)
#define MMA(ai, bj, At, Bt) do { __builtin_amdgcn_s_setprio(1); _Pragma("unroll") for (int m = 0; m < 4; ++m) _Pragma("unroll") for (int n = 0; n < 2; ++n) _Pragma("unroll") for (int k = 0; k < 2; ++k) \
        acc[ai][bj][m][n] = __builtin_amdgcn_mfma_f32_16x16x32_bf16(Bt[n][k], At[m][k], acc[ai][bj][m][n], 0, 0, 0); __builtin_amdgcn_s_setprio(0); } while (0)
#define OFFA(t) (OFF_XT + (unsigned)(t) * XT_STEP + (unsigned)rt * BLK)
#define OFFB(t) (OFF_WT + (unsigned)(t) * WT_STEP + (unsigned)ct * BLK)
#define SLOW_WAIT(t) do { Spin t0_; \
        for (;;) { CFENCE; const unsigned f_ = __builtin_amdgcn_raw_buffer_load_b32(rs, voffF, OFF_FLAG + (unsigned)(t) * FLAG_REC, AUX_SYS); \
            if (__builtin_amdgcn_ballot_w64(f_ < call) == 0) break; if (t0_.fail(myws, call)) break; } } while (0)
#define CHECK(t, s, tn) do { if (wid == 0) { const unsigned f0_ = *(volatile LAS unsigned*)(lds + LDS_FLAG + (s) * 256 + lane * 4); \
            if (__builtin_amdgcn_ballot_w64(f0_ < call) != 0) SLOW_WAIT(t); \
            __builtin_amdgcn_raw_ptr_buffer_load_lds(rs, (LAS void*)(lds + LDS_FLAG + ((s) ^ 1) * 256), 4, voffF, OFF_FLAG + (unsigned)(tn) * FLAG_REC, 0, AUX_SYS); } } while (0)

    f32x4 acc[2][2][4][2];
#pragma unroll
    for (int a = 0; a < 2; ++a)
#pragma unroll
        for (int b = 0; b < 2; ++b)
#pragma unroll
            for (int m = 0; m < 4; ++m)
#pragma unroll
                for (int n = 0; n < 2; ++n) acc[a][b][m][n] = (f32x4){0.f, 0.f, 0.f, 0.f};
    bf16x8 At[4][2], B0[2][2], B1[2][2];

    {
        const int t0 = SEQ(0), t1 = SEQ(1), t2 = SEQ(2);
        if (wid == 0) {
            SLOW_WAIT(t0); SLOW_WAIT(t1);
            __builtin_amdgcn_raw_ptr_buffer_load_lds(rs, (LAS void*)(lds + LDS_FLAG), 4, voffF, OFF_FLAG + (unsigned)t2 * FLAG_REC, 0, AUX_SYS);
        }
        if (wid == 1 && lane < 8) ((LAS unsigned*)(lds + LDS_RED))[32 + lane] = 0u;
        WAIT_L(0); BAR; CFENCE;
        STAMP(myws, wv, 1);
        STAGE(SB(0, 0), OFFB(t0)); STAGE(SA(0, 0), OFFA(t0)); STAGE(SB(0, 1), OFFB(t0) + HALFB); STAGE(SA(0, 1), OFFA(t0) + HALFB);
        if (wr == 1) BAR;
        WAIT_V(4); BAR;
        STAGE(SB(1, 0), OFFB(t1)); STAGE(SA(1, 0), OFFA(t1)); STAGE(SB(1, 1), OFFB(t1) + HALFB);
        WAIT_V(6); BAR;
    }
    for (int i = 0; i < NT; i += 2) {
        const int t1 = SEQ(i + 1), t2 = SEQ(i + 2), t3 = SEQ(i + 3), t4 = SEQ(i + 4);
        const unsigned a1 = OFFA(t1), a2 = OFFA(t2), b2 = OFFB(t2), a3 = OFFA(t3), b3 = OFFB(t3);
        CHECK(t2, 0, t3);
        LDB(B0, 0, 0); SCHED; LDA(At, 0, 0); STAGE(SA(1, 1), a1 + HALFB);
        WAIT_L(8); BAR; WAIT_L(0); MMA(0, 0, At, B0); BAR; SCHED;
        LDB(B1, 0, 1); STAGE(SB(0, 0), b2);
        BAR; WAIT_L(0); MMA(0, 1, At, B1); BAR;
        LDA(At, 0, 1); STAGE(SA(0, 0), a2);
        BAR; WAIT_L(0); MMA(1, 0, At, B0); BAR; SCHED;
        STAGE(SB(0, 1), b2 + HALFB);
        WAIT_V(6); BAR; MMA(1, 1, At, B1); BAR;
        CHECK(t3, 1, t4);
        LDB(B0, 1, 0); SCHED; LDA(At, 1, 0); STAGE(SA(0, 1), a2 + HALFB);
        WAIT_L(8); BAR; WAIT_L(0); MMA(0, 0, At, B0); BAR; SCHED;
        LDB(B1, 1, 1); STAGE(SB(1, 0), b3);
        BAR; WAIT_L(0); MMA(0, 1, At, B1); BAR;
        LDA(At, 1, 1); STAGE(SA(1, 0), a3);
        BAR; WAIT_L(0); MMA(1, 0, At, B0); BAR; SCHED;
        STAGE(SB(1, 1), b3 + HALFB);
        WAIT_V(6); BAR; MMA(1, 1, At, B1); BAR;
    }
    char* pl = (char*)p.ws[lane & 7];
    if (wr == 0) BAR;
    STAMP(myws, wv, 10);

    const int wt = tile * 8 + wid;
    LAS unsigned* red = (LAS unsigned*)(lds + LDS_RED);
    u32x4 msk;
#pragma unroll
    for (int w = 0; w < 4; ++w) {
        unsigned mm = 0;
#pragma unroll
        for (int e = 31; e >= 0; --e) {
            const int ch = 2 * w + (e >> 4);
            mm = __builtin_amdgcn_alignbit(mm, __float_as_uint(acc[ch >> 2][(e >> 1) & 1][ch & 3][e & 1][(e >> 2) & 3]), 31);
        }
        msk[w] = ~mm;
    }
    __builtin_amdgcn_raw_buffer_store_b128(msk, rs, lane * 16, OFF_MSK + wt * 1024, AUX_SYS);
    SCHED;
    float mx = 0.f;
#pragma unroll
    for (int a = 0; a < 2; ++a)
#pragma unroll
        for (int b = 0; b < 2; ++b)
#pragma unroll
            for (int m = 0; m < 4; ++m)
#pragma unroll
                for (int n = 0; n < 2; ++n) mx = fmaxf(fmaxf(fmaxf(fmaxf(mx, acc[a][b][m][n][0]), acc[a][b][m][n][1]), acc[a][b][m][n][2]), acc[a][b][m][n][3]);
    const unsigned wm = wave_max(__float_as_uint(mx));
    {
        unsigned old = 0;
        if (lane == 0) {
            __hip_atomic_fetch_max(red + 32, wm, __ATOMIC_RELAXED, __HIP_MEMORY_SCOPE_WORKGROUP);
            old = __hip_atomic_fetch_add(red + 33, 1u, __ATOMIC_RELAXED, __HIP_MEMORY_SCOPE_WORKGROUP);
        }
        if (__builtin_amdgcn_readfirstlane(old) == 7) {
            const unsigned long long gv = ((unsigned long long)call << 32) | __hip_atomic_load(red + 32, __ATOMIC_RELAXED, __HIP_MEMORY_SCOPE_WORKGROUP);
            char* g = pl + OFF_AMAX + (rank * NGEMM + tile) * 8;
            unsigned long long ex_;
            asm volatile("s_mov_b64 %0, exec\n\ts_mov_b64 exec, 0xff\n\tglobal_store_dwordx2 %1, %2, off sc0 sc1\n\ts_mov_b64 exec, %0\n\ts_waitcnt vmcnt(1)" : "=&s"(ex_) : "v"(g), "v"(gv) : "memory");
        } else WAIT_V(0);
        unsigned old2 = 0;
        if (lane == 0) old2 = __hip_atomic_fetch_add(red + 34 + wr, 1u, __ATOMIC_RELAXED, __HIP_MEMORY_SCOPE_WORKGROUP);
        if (__builtin_amdgcn_readfirstlane(old2) == 3 && lane < 8) st_sys(pl + OFF_MFLG + (rank * 256 + tile * 2 + wr) * 4, call);
    }
#pragma unroll
    for (int a = 0; a < 2; ++a)
#pragma unroll
        for (int b = 0; b < 2; ++b)
#pragma unroll
            for (int m = 0; m < 4; ++m)
#pragma unroll
                for (int n = 0; n < 2; ++n)
#pragma unroll
                    for (int j = 0; j < 4; ++j) { acc[a][b][m][n][j] = fmaxf(acc[a][b][m][n][j], 0.f); if (j == 3) asm volatile("" : "+v"(acc[a][b][m][n])); }
    STAMP(myws, wv, 13);
    BAR; CFENCE;
    LAS unsigned char* A = lds + wid * 16384;
    LAS unsigned char* B = A + 8192;
    const unsigned cnt = popc4(msk), scan = wave_scan(cnt), Lc = __builtin_amdgcn_readlane(scan, 63);
    unsigned off = scan - cnt;
#pragma unroll
    for (int r = 0; r < 8; ++r) *(LAS u32x4*)(A + r * 1024 + lane * 16) = (u32x4){0u, 0u, 0u, 0u};
    LAS unsigned char* lut = A + 16384 - 64;
    if (lane < 16) *(LAS unsigned*)(lut + lane * 4) = lut_compact(lane);
    unsigned sel[32], of[32];
    off += (unsigned)(size_t)A;
#pragma unroll
    for (int g = 0; g < 32; ++g) {
        const unsigned n4 = nib4(msk[g >> 3], g & 7);
        sel[g] = *(const LAS unsigned*)(lut + n4); of[g] = (off & ~3u) | (off << 30);
        asm volatile("" : "+v"(sel[g]), "+v"(of[g]));
        off += __builtin_popcount(n4);
    }
    {
        Spin t0;
        unsigned t, tm = ld_agent(myws + OFF_TMO);
        for (;;) {
            CFENCE;
            const u32x4 g = __builtin_amdgcn_raw_buffer_load_b128(rs, lane * 16, OFF_AMAX + wid * 1024, AUX_SYS);
            t = max(g[0], g[2]);
            if (__builtin_amdgcn_ballot_w64(g[1] < call || g[3] < call) == 0) break;
            if (t0.fail(myws, call)) { tm = call; break; }
            __builtin_amdgcn_s_sleep(AMAX_SLEEP);
        }
        t = wave_max(t);
        if (lane == 0) { red[8 + wid] = t; red[wid] = tm == call; }
    }
    WAIT_L(0); BAR; CFENCE;
    const float amax = __uint_as_float(wave_max(red[8 + (lane & 7)]));
    const bool dead = __builtin_amdgcn_ballot_w64(red[lane & 7] != 0) != 0;
    const float inv = dead ? __builtin_nanf("") : 448.0f / amax;
    STAMP(myws, wv, 11);

    u32x4 qv[8];
#pragma unroll
    for (int ch = 0; ch < 8; ++ch) {
        const int a = ch >> 2, m = ch & 3;
        const f32x4 y0 = acc[a][0][m][0] * inv, y1 = acc[a][0][m][1] * inv, y2 = acc[a][1][m][0] * inv, y3 = acc[a][1][m][1] * inv;
#pragma unroll
        for (int j = 0; j < 4; ++j) {
            int w = __builtin_amdgcn_cvt_pk_fp8_f32(y0[j], y1[j], __float_as_int(y0[j]), false);
            w = __builtin_amdgcn_cvt_pk_fp8_f32(y2[j], y3[j], w, true);
            qv[ch][j] = (unsigned)w;
            const unsigned o_ = of[ch * 4 + j];
            const unsigned long long z = (unsigned long long)__builtin_amdgcn_perm(0u, (unsigned)w, sel[ch * 4 + j]) << (o_ >> 27);
            LAS unsigned* d = (LAS unsigned*)(size_t)(o_ & 0x3ffffu);
            __hip_atomic_fetch_or(d, (unsigned)z, __ATOMIC_RELAXED, __HIP_MEMORY_SCOPE_WAVEFRONT);
            __hip_atomic_fetch_or(d + 1, (unsigned)(z >> 32), __ATOMIC_RELAXED, __HIP_MEMORY_SCOPE_WAVEFRONT);
        }
    }
    CFENCE;
    const unsigned top = __builtin_amdgcn_readfirstlane(__builtin_amdgcn_cvt_pk_fp8_f32(__uint_as_float(wm) * inv, 0.f, 0, false)) & 0x7fu;
    const unsigned base = top > 62 ? top - 62 : 0, B4 = base * 0x01010101u, G = (Lc + 15) >> 4;
    if (lane < 16) A[Lc + lane] = (unsigned char)base;
    CFENCE;
    unsigned bm[4] = {0u, 0u, 0u, 0u};
#define FOLD(it0, n) { u32x4 d_[n]; \
        _Pragma("unroll") for (int i = 0; i < n; ++i) d_[i] = *(const LAS u32x4*)(A + ((it0 + i) * 64 + lane) * 16); \
        SCHED; \
        _Pragma("unroll") for (int i = 0; i < n; ++i) { const int it = it0 + i; const unsigned g = it * 64 + lane; unsigned sy[4], ms[4]; \
            _Pragma("unroll") for (int k = 0; k < 4; ++k) { const unsigned t = (d_[i][k] | H8) - B4; ms[k] = ~t & H8; sy[k] = (t & 0x3f3f3f3fu) | ((ms[k] >> 1) - (ms[k] >> 7)); } \
            LAS unsigned* dB = (LAS unsigned*)(B + g * 12); \
            _Pragma("unroll") for (int k = 0; k < 3; ++k) dB[k] = sy[k] | ((sy[3] << (6 - 2 * k)) & 0xc0c0c0c0u); \
            bm[it >> 1] |= (g < G ? bits16(ms[0], ms[1], ms[2], ms[3], 7) : 0u) << (16 * (it & 1)); } }
    FOLD(0, 4);
    if (G > 256) { FOLD(4, 1); if (G > 320) FOLD(5, 3); }
    unsigned at;
    const unsigned ne = esc_count(bm, at);
    const bool raw = ne > MAXESC || G * 12 + ne > Lc;
    if (!raw) {
        LAS unsigned char* list = B + 6656;
        esc_list(bm, at, list, lane);
        CFENCE;
#pragma nounroll
        for (unsigned i = lane; i < ne; i += 64) B[G * 12 + i] = A[*(const LAS unsigned short*)(list + i * 2)];
        CFENCE;
    }
    {
        LAS unsigned char* S = raw ? A : B;
#pragma nounroll
        for (unsigned r = 0; r < (raw ? Lc : G * 12 + ne); r += 1024)
            __builtin_amdgcn_raw_buffer_store_b128(*(const LAS u32x4*)(S + r + lane * 16), rs, lane * 16, OFF_COD + wt * 8192 + r, AUX_SYS);
    }
    if (lane == 0) { red[16 + wid] = ((call & 0x3ffffu) << 14) | Lc; red[24 + wid] = (call << 22) | (raw ? 1u << 21 : (base << 14) | ne); }
    WAIT_V(0); WAIT_L(0); BAR; CFENCE;
    if (wid == 0 && (lane >> 3) != rank)
        __hip_atomic_store((GAS unsigned long long*)((char*)p.ws[lane >> 3] + OFF_DONE + (rank * 1024 + tile * 8 + (lane & 7)) * 8),
                           ((unsigned long long)red[24 + (lane & 7)] << 32) | red[16 + (lane & 7)], __ATOMIC_RELAXED, __HIP_MEMORY_SCOPE_SYSTEM);
    STAMP(myws, wv, 12);
    {
        const rsrc_t ro = mk_rsrc(p.out[rank], OUT_BYTES);
        const unsigned so = out_soff(rank, wt), vo = out_voff(lane);
#pragma unroll
        for (int ch = 0; ch < 8; ++ch) __builtin_amdgcn_raw_buffer_store_b128(qv[ch], ro, vo, so + chunk_soff(ch), OUT_AUX);
    }
}

__global__ __launch_bounds__(512, 2) void fused_kernel(Params p) {
    __shared__ __attribute__((aligned(16))) unsigned char lds_arr[LDS_BYTES];
    LAS unsigned char* lds = (LAS unsigned char*)lds_arr;
    const int wid = __builtin_amdgcn_readfirstlane(threadIdx.x >> 6), lane = threadIdx.x & 63;
    const int b = blockIdx.x, xcd = b & 7, idx = b >> 3;
    const unsigned callv = (unsigned)__builtin_amdgcn_raw_ptr_buffer_atomic_add_i32(1, mk_rsrc(p.ws[p.rank], WS_USED), lane == 0 ? OFF_CNT + (b * 8 + wid) * 4 : 0xfffffff0u, 0, 0);
    if (idx < 16) gemm_role(p, lds, (xcd >> 1) * 4 + (idx >> 2), (xcd & 1) * 4 + (idx & 3), wid, lane, callv);
    else comm_role(p, lds, (idx - 16) * 8 + xcd, wid, lane, callv);
}

extern "C" void kernel_launch(int rank, int nranks, ncclComm_t comm, hipStream_t stream,
                              void* const* d_in, const int* in_sizes, int n_in,
                              void* d_out, int out_size,
                              const int* in_dtypes, int out_dtype,
                              void* const* d_out_all, void* const* d_ws_all, size_t ws_bytes) {
    (void)comm; (void)in_dtypes; (void)out_dtype; (void)d_out;
    if (nranks != NR || n_in != 2 || in_sizes[0] != M_ * KS || in_sizes[1] != KS * N_ || out_size != M_ * N_ || ws_bytes < WS_USED || (unsigned)rank >= (unsigned)NR) {
        fprintf(stderr, "kernel_launch: unexpected problem shape\n");
        return;
    }
    Params p;
    p.x = d_in[0]; p.w = d_in[1];
    for (int i = 0; i < NR; ++i) { p.out[i] = d_out_all[i]; p.ws[i] = d_ws_all[i]; }
    p.rank = rank; p.pad = 0;
    fused_kernel<<<dim3(NWG), dim3(512), 0, stream>>>(p);
}
```

```cpp
#include <hip/hip_runtime.h>
#include <rccl/rccl.h>
#include <cstdio>

#define LAS __attribute__((address_space(3)))
#define GAS __attribute__((address_space(1)))
typedef short bf16x8 __attribute__((ext_vector_type(8)));
typedef float f32x4 __attribute__((ext_vector_type(4)));
typedef unsigned u32x4 __attribute__((ext_vector_type(4)));
typedef unsigned u32x2 __attribute__((ext_vector_type(2)));
typedef __amdgpu_buffer_rsrc_t rsrc_t;

constexpr int NR = 8;
constexpr int M_ = 4096, N_ = 16384;
constexpr int KS = 512;
constexpr int NS = 2048;
constexpr int NT = 64;
constexpr int NGEMM = 128;
constexpr int NWG = 256;

constexpr unsigned OFF_CNT  = 0;
constexpr unsigned OFF_TMO  = 16384;
constexpr unsigned OFF_FLAG = 65536;
constexpr unsigned FLAG_W = 4096;
constexpr unsigned OFF_AMAX = 131072;
constexpr unsigned OFF_TICK = 143360;
constexpr unsigned OFF_PCNT = 147456;
constexpr unsigned OFF_PFLG = 163840;
constexpr unsigned OFF_PUF  = 188416;
constexpr unsigned OFF_MFLG = 176128;
constexpr unsigned OFF_DONE = 786432;
constexpr unsigned OFF_MF   = OFF_DONE + 65536;
constexpr unsigned OFF_DBG  = 262144;
constexpr unsigned CTRL_BYTES = 1u << 20;
constexpr unsigned XT_STEP = 512u * 1024u, WT_STEP = 256u * 1024u, BLK = 32768u, HALFB = 16384u;
constexpr unsigned OFF_XT = CTRL_BYTES;
constexpr unsigned XT_BYTES = NT * XT_STEP;
constexpr unsigned OFF_WT = OFF_XT + XT_BYTES;
constexpr unsigned WT_BYTES = NT * WT_STEP;
constexpr unsigned UNIT = 9216;
constexpr unsigned OFF_PX = OFF_WT + WT_BYTES;
constexpr unsigned OFF_PW = OFF_PX + 512 * UNIT;
constexpr unsigned OFF_MSK = OFF_PW + 2048 * UNIT;
constexpr unsigned OFF_COD = OFF_MSK + 1024 * 1024;
constexpr unsigned OFF_LM = OFF_COD + 1024 * 8192;
constexpr unsigned WS_USED = OFF_LM + 8 * 1024 * 1024;
constexpr unsigned OUT_BYTES = (unsigned)M_ * N_;
constexpr unsigned X_BYTES = (unsigned)M_ * KS * 2, W_BYTES = (unsigned)KS * N_ * 2;

constexpr unsigned long long DEADLINE = 100000000ull;
constexpr int AUX_SYS = 17;
#define GEMM_AUX 17
#define PPP 16
#define PPO 40
#define PPM 17
#define PACE_W 800
#define PACE_W0 800
#define PACE_W1 150
#define PACE_X 400
#define PACE_O0 100
#define PACE_O 60
#define PULL_SERIAL 0
#define AMAX_SLEEP 4
#define NPUF 2
#define PACK_AUX 17
#define IN_AUX 2
#define SCH_S 100
#define SCH_C 1000
#define PUSH_W 8
#define PUSH_U 3
#define EPI_PRIO 0
#define WT_ORD 0
#define ENT_SLEEP 8
#define SCH_A 1
#define SCH_B 10
#define CW 3
#define OUT_AUX 16
#define LOC_AUX 17

constexpr unsigned LDS_STAGE = 8 * HALFB;
constexpr unsigned LDS_FLAG = LDS_STAGE;
constexpr unsigned LDS_RED = LDS_FLAG + 512;
constexpr unsigned EX_B = 8192, EX_M = EX_B + 8192, EX_P = EX_M + 1024, EX_E = EX_P + 1024, EX_BYTES = EX_E + 256, EX_LUT = 0;
constexpr unsigned LDS_BYTES = 2048 + 8 * EX_BYTES;
static_assert(LDS_RED + 192 <= LDS_BYTES && LDS_BYTES <= 163840, "lds");
static_assert(OFF_PUF + 8 * NPUF * 384 <= OFF_DBG, "flags");
static_assert(LOC_AUX == AUX_SYS, "img_store also writes to peers");
static_assert((PUSH_W * PUSH_U) % 4 == 0 && PUSH_W * PUSH_U <= 32 && 7 * PUSH_W <= NGEMM, "push");
static_assert(7 * PPO <= 128 * CW && 7 * PPM <= 128 * CW && PPP <= PPM, "waves");

struct Params {
    const void* x; const void* w;
    void* out[NR]; void* ws[NR];
    int rank; int pad;
};

__device__ __forceinline__ rsrc_t mk_rsrc(const void* p, unsigned bytes) {
    return __builtin_amdgcn_make_buffer_rsrc((void*)p, 0, bytes, 0x00020000);
}
#define WAIT_V(n) asm volatile("s_waitcnt vmcnt(" #n ")" ::: "memory")
#define WAIT_L(n) asm volatile("s_waitcnt lgkmcnt(" #n ")" ::: "memory")
#define BAR __builtin_amdgcn_s_barrier()
#define SCHED __builtin_amdgcn_sched_barrier(0)
#define CFENCE asm volatile("" ::: "memory")

#define STAMP(ws_, wv_, k_) do {} while (0)
#define CYC(i) do {} while (0)

__device__ __forceinline__ unsigned ld_agent(const void* p) { return __hip_atomic_load((const GAS unsigned*)p, __ATOMIC_RELAXED, __HIP_MEMORY_SCOPE_AGENT); }
__device__ __forceinline__ void st_agent(void* p, unsigned v) { __hip_atomic_store((GAS unsigned*)p, v, __ATOMIC_RELAXED, __HIP_MEMORY_SCOPE_AGENT); }
__device__ __forceinline__ void st_sys(void* p, unsigned v) { __hip_atomic_store((GAS unsigned*)p, v, __ATOMIC_RELAXED, __HIP_MEMORY_SCOPE_SYSTEM); }

__device__ __forceinline__ void st_sys8(void* p, unsigned v) { __hip_atomic_store((GAS unsigned char*)p, (unsigned char)v, __ATOMIC_RELAXED, __HIP_MEMORY_SCOPE_SYSTEM); }
__device__ __forceinline__ unsigned x_flag(int t, int g) { return OFF_FLAG + (g >> 2) * 256 + t * 4 + (g & 3); }
__device__ __forceinline__ unsigned w_flag(int t, int e) { return OFF_FLAG + FLAG_W + (e >> 2) * 256 + t * 4 + (e & 3); }

struct Spin {
    unsigned long long t0; unsigned n;
    __device__ __forceinline__ Spin() : t0(__builtin_amdgcn_s_memrealtime()), n(0) {}
    __device__ __forceinline__ bool fail(char* myws, unsigned call) {
        if ((++n & 15u) == 0 && (__builtin_amdgcn_readfirstlane(ld_agent(myws + OFF_TMO)) == call || __builtin_amdgcn_s_memrealtime() - t0 > DEADLINE)) {
            st_agent(myws + OFF_TMO, call);
            return true;
        }
        __builtin_amdgcn_s_sleep(1);
        return false;
    }
};

#define DPP0(x, ctrl, rm) (unsigned)__builtin_amdgcn_update_dpp(0, (int)(x), ctrl, rm, 0xf, false)
__device__ __forceinline__ unsigned wave_scan(unsigned x) {
    x += DPP0(x, 0x111, 0xf); x += DPP0(x, 0x112, 0xf); x += DPP0(x, 0x114, 0xf); x += DPP0(x, 0x118, 0xf);
    x += DPP0(x, 0x142, 0xa); x += DPP0(x, 0x143, 0xc);
    return x;
}
__device__ __forceinline__ unsigned wave_max(unsigned x) {
    x = max(x, DPP0(x, 0x111, 0xf)); x = max(x, DPP0(x, 0x112, 0xf)); x = max(x, DPP0(x, 0x114, 0xf)); x = max(x, DPP0(x, 0x118, 0xf));
    x = max(x, DPP0(x, 0x142, 0xa)); x = max(x, DPP0(x, 0x143, 0xc));
    return __builtin_amdgcn_readlane(x, 63);
}

struct Sched {
    unsigned w[7][64];
    constexpr Sched() : w{} {
        for (int q = 0; q < 7; ++q) {
            int key[136] = {}, id[136] = {}, n = 0;
            for (int j = 0; j < 8; ++j) {
                int lo = 1 << 30;
                for (int rt = 0; rt < 16; ++rt) { const int k = (SCH_S * rt < SCH_C ? SCH_S * rt : SCH_C) + SCH_B * rt + 106 * (7 * j + (q + SCH_A * rt) % 7); key[n] = 2 * k + 1; id[n++] = j * 24 + 8 + rt; lo = k < lo ? k : lo; }
                key[n] = 2 * lo; id[n++] = -1 - j;
            }
            int at = 0;
            for (int o = 0; o < 136; ++o) {
                int best = -1;
                for (int i = 0; i < 136; ++i) if (key[i] >= 0 && (best < 0 || key[i] < key[best])) best = i;
                key[best] = -1;
                const int skip = id[best] == -1 ? PUSH_W * PUSH_U / 4 : 0;
                const int cnt = id[best] < 0 ? 8 - skip : 1, first = id[best] < 0 ? (-1 - id[best]) * 24 + skip : id[best];
                for (int c = 0; c < cnt; ++c, ++at) w[q][at >> 2] |= (unsigned)(first + c) << (8 * (at & 3));
            }
        }
    }
};
__constant__ const Sched SCHED_TAB;
__device__ __forceinline__ int step_of(int i, int rank, int rt) {
    if (i < 8) return rank * 8 + i;
    const int z = i - 8, j = z / 7, p = z - j * 7, q = (p + 7 * 16 * SCH_A - SCH_A * rt) % 7;
    return (((rank + 1 + q) & 7) << 3) + j;
}

__device__ __forceinline__ void x_load(u32x4 (&v)[8], rsrc_t rx, int j, int g, int lane) {
    const unsigned voff = (unsigned)(lane & 15) * (KS * 2) + (unsigned)(lane >> 4) * 16;
    const unsigned soff = (unsigned)g * 64 * (KS * 2) + (unsigned)j * 128;
#pragma unroll
    for (int r = 0; r < 4; ++r)
#pragma unroll
        for (int kk = 0; kk < 2; ++kk) v[r * 2 + kk] = __builtin_amdgcn_raw_buffer_load_b128(rx, voff + kk * 64, soff + r * 16 * (KS * 2), IN_AUX);
}
__device__ __forceinline__ void w_load(u32x2 (&v)[16], rsrc_t rw, int j, int d, int ct, int kk, int G, int lane) {
    const unsigned voff = (unsigned)(lane >> 4) * 8 * (N_ * 2) + (unsigned)(lane & 15) * 8;
    const unsigned soff = (unsigned)(j * 64 + kk * 32) * (N_ * 2) + (unsigned)(d * NS + ct * 256 + G * 128) * 2;
#pragma unroll
    for (int jj = 0; jj < 8; ++jj)
#pragma unroll
        for (int g = 0; g < 2; ++g) v[g * 8 + jj] = __builtin_amdgcn_raw_buffer_load_b64(rw, voff + g * 128, soff + jj * (N_ * 2), IN_AUX);
}
__device__ __forceinline__ void w_transpose(u32x4 (&o)[8], const u32x2 (&v)[16]) {
#pragma unroll
    for (int g = 0; g < 2; ++g)
#pragma unroll
        for (int c = 0; c < 4; ++c)
#pragma unroll
            for (int dd = 0; dd < 4; ++dd)
                o[g * 4 + c][dd] = __builtin_amdgcn_perm(v[g * 8 + 2 * dd + 1][c >> 1], v[g * 8 + 2 * dd][c >> 1], (c & 1) ? 0x07060302u : 0x05040100u);
}
__device__ __forceinline__ unsigned x_img(int t, int g) { return OFF_XT + (unsigned)t * XT_STEP + (unsigned)g * 8192; }
__device__ __forceinline__ unsigned w_img(int t, int e) { return OFF_WT + (unsigned)t * WT_STEP + (unsigned)(e >> 2) * BLK + (unsigned)((e >> 1) & 1) * 8192 + (unsigned)(e & 1) * 1024; }
__device__ __forceinline__ unsigned frag_off(bool isw, unsigned f) { return isw ? ((f >> 1) & 1) * HALFB + (f >> 2) * 4096 + (f & 1) * 2048 : f * 1024; }
__device__ __forceinline__ void img_store(const u32x4 (&v)[8], rsrc_t rd, bool isw, unsigned img, int lane) {
#pragma unroll
    for (int f = 0; f < 8; ++f) __builtin_amdgcn_raw_buffer_store_b128(v[f], rd, lane * 16, img + frag_off(isw, f), LOC_AUX);
}
constexpr unsigned H8 = 0x80808080u, MAXESC = 512;
__device__ __forceinline__ unsigned bits16(unsigned m0, unsigned m1, unsigned m2, unsigned m3, int sh) {
    const unsigned a = __builtin_amdgcn_udot4(m0, 0x08040201u, __builtin_amdgcn_udot4(m1, 0x80402010u, 0u, false), false);
    const unsigned b = __builtin_amdgcn_udot4(m2, 0x08040201u, __builtin_amdgcn_udot4(m3, 0x80402010u, 0u, false), false);
    return (a >> sh) | (b << (8 - sh));
}
__device__ __forceinline__ unsigned esc_count(const unsigned (&bm)[4], unsigned& first) {
    const unsigned c = __builtin_popcount(bm[0]) + __builtin_popcount(bm[1]) + __builtin_popcount(bm[2]) + __builtin_popcount(bm[3]), s = wave_scan(c);
    first = s - c;
    return __builtin_amdgcn_readlane(s, 63);
}
__device__ __forceinline__ void esc_list(const unsigned (&bm)[4], unsigned at, LAS unsigned char* list, int lane) {
#pragma unroll
    for (int w = 0; w < 2; ++w) {
        unsigned long long z = ((unsigned long long)bm[2 * w + 1] << 32) | bm[2 * w];
        while (__builtin_amdgcn_ballot_w64(z != 0) != 0) {
            if (z != 0) {
                const unsigned b = __builtin_ctzll(z);
                *(LAS unsigned short*)(list + min(at, MAXESC - 1) * 2) = (unsigned short)(w * 4096 + (b >> 4) * 1024 + lane * 16 + (b & 15));
                ++at; z &= z - 1;
            }
        }
    }
}
constexpr unsigned PK_HDR = 5 * 1024 + 512, PK_RAW = 0xffffu, PK_FIRST = 62;
__device__ __forceinline__ void pack_store(const u32x4 (&v)[8], rsrc_t rs, LAS unsigned char* L, unsigned slot, int lane) {
    unsigned p0[16], e[16], sum = 0;
#pragma unroll
    for (int r = 0; r < 16; ++r) {
        const unsigned lo = __builtin_amdgcn_perm(v[r >> 1][2 * (r & 1) + 1], v[r >> 1][2 * (r & 1)], 0x06040200u);
        const unsigned hi = __builtin_amdgcn_perm(v[r >> 1][2 * (r & 1) + 1], v[r >> 1][2 * (r & 1)], 0x07050301u);
        p0[r] = (hi & H8) | (lo & ~H8);
        e[r] = ((hi << 1) & 0xfefefefeu) | ((lo >> 7) & 0x01010101u);
        sum = __builtin_amdgcn_sad_u8(e[r], 0u, sum);
    }
#pragma unroll
    for (int r = 0; r < 4; ++r) *(LAS u32x4*)(L + r * 1024 + lane * 16) = (u32x4){e[4 * r], e[4 * r + 1], e[4 * r + 2], e[4 * r + 3]};
    const int tot = (int)__builtin_amdgcn_readlane(wave_scan(sum), 63);
    const unsigned base = (unsigned)min(max((tot - 16794) >> 12, 0), 247), base4 = base * 0x01010101u;
    u32x4 p2 = {0u, 0u, 0u, 0u}; u32x2 p1 = {0u, 0u};
    unsigned ms[16];
#pragma unroll
    for (int r = 0; r < 16; ++r) {
        const unsigned z = ((e[r] | H8) - (base4 & ~H8)) ^ ((e[r] ^ ~base4) & H8);
        ms[r] = (((z & 0x78787878u) + 0x78787878u) | z) & H8;
        p2[r >> 2] |= (z & 0x03030303u) << (2 * (r & 3));
        p1[r >> 3] |= (r & 7) >= 2 ? (z & 0x04040404u) << ((r & 7) - 2) : (z & 0x04040404u) >> (2 - (r & 7));
    }
    unsigned bm[4] = {bits16(ms[0], ms[1], ms[2], ms[3], 7) | (bits16(ms[4], ms[5], ms[6], ms[7], 7) << 16),
                      bits16(ms[8], ms[9], ms[10], ms[11], 7) | (bits16(ms[12], ms[13], ms[14], ms[15], 7) << 16), 0u, 0u};
    unsigned at;
    const unsigned n = esc_count(bm, at);
    bool bad = n > 254;
    *(LAS u32x2*)(L + 4096 + lane * 8) = p1;
    if (!bad) {
        esc_list(bm, at, L + 5120, lane);
        CFENCE;
        bool b = false;
#pragma nounroll
        for (unsigned i = lane; i < n; i += 64) {
            const unsigned pos = *(const LAS unsigned short*)(L + 5120 + i * 2), ev = L[pos];
            unsigned d = base - 1 - ev;
            if (ev == base + 8) d = 14; else if (ev == 0) d = 15; else b |= d > 13;
            *(LAS unsigned short*)(L + 4096 + 516 + i * 2) = (unsigned short)(pos | (d << 12));
        }
        bad = __builtin_amdgcn_ballot_w64(b) != 0;
    }
    if (lane == 0) *(LAS unsigned*)(L + 4096 + 512) = base | ((bad ? PK_RAW : n) << 8);
    CFENCE;
    __builtin_amdgcn_raw_buffer_store_b128(*(const LAS u32x4*)(L + 4096 + lane * 16), rs, lane * 16 + 1024, slot + 4096, PACK_AUX);
    if (!bad) {
#pragma unroll
        for (int r = 0; r < 4; ++r) __builtin_amdgcn_raw_buffer_store_b128((u32x4){p0[4 * r], p0[4 * r + 1], p0[4 * r + 2], p0[4 * r + 3]}, rs, lane * 16 + r * 1024, slot, PACK_AUX);
        __builtin_amdgcn_raw_buffer_store_b128(p2, rs, lane * 16, slot + 4096, PACK_AUX);
    } else {
#pragma unroll
        for (int f = 0; f < 8; ++f) __builtin_amdgcn_raw_buffer_store_b128(v[f], rs, lane * 16, slot + (f < 5 ? f : f + 1) * 1024, PACK_AUX);
    }
}
constexpr unsigned PL_SLOT = 8192;
struct Puller {
    rsrc_t rp, rs; char* myws; int rank, src, q, lane; unsigned call, seen, tab;
    __device__ __forceinline__ int unit_of(int tk) const { return (int)((__builtin_amdgcn_readlane(tab, tk >> 4) >> (2 * (tk & 12))) & 0xffu) * 4 + (tk & 3); }
    __device__ __forceinline__ int xg(int i) const { return i - 32; }
    __device__ __forceinline__ unsigned unit_id(int u) const { const int j = u / 96, i = u % 96; return i < 32 ? 512 + (rank * 8 + j) * 32 + i : j * 64 + xg(i); }
    __device__ __forceinline__ unsigned flag_of(int u) const { const int j = u / 96, i = u % 96; return i < 32 ? w_flag(src * 8 + j, i) : x_flag(src * 8 + j, xg(i)); }
    __device__ __forceinline__ void issue(LAS unsigned char* b, int u) {
        const int j = u / 96, i = u % 96, need = j * 2 + (i >= 32);
        if (!((seen >> need) & 1u)) {
            Spin t0;
            for (;;) {
                CFENCE;
                const unsigned sum = OFF_PFLG + (src * 8 + j) * 128 + (i >= 32 ? 0 : 4);
                const unsigned f = __builtin_amdgcn_raw_buffer_load_b32(rs, (lane & 1) && j < NPUF ? OFF_PUF + ((src * NPUF + j) * 96 + i) * 4 : sum, 0, AUX_SYS);
                if (__builtin_amdgcn_readlane(f, 0) >= call) { seen |= 1u << need; break; }
                if (__builtin_amdgcn_readlane(f, 1) >= call) break;
                if (t0.fail(myws, call)) break;
            }
            CFENCE;
        }
        const unsigned id = unit_id(u), slot = OFF_PX + id * UNIT;
        __builtin_amdgcn_raw_ptr_buffer_load_lds(rp, (LAS void*)(b + 4096), 16, lane < 40 ? lane * 16 : 0xf0000000u, slot + 4096, 1024, AUX_SYS);
        __builtin_amdgcn_raw_ptr_buffer_load_lds(rp, (LAS void*)(b + 4096), 16, lane * 16, slot + 4096, 0, AUX_SYS);
        __builtin_amdgcn_raw_ptr_buffer_load_lds(rp, (LAS void*)b, 16, lane * 16, slot, 0, AUX_SYS);
        __builtin_amdgcn_raw_ptr_buffer_load_lds(rp, (LAS void*)b, 16, lane * 16, slot, 1024, AUX_SYS);
        __builtin_amdgcn_raw_ptr_buffer_load_lds(rp, (LAS void*)b, 16, lane * 16, slot, 2048, AUX_SYS);
        __builtin_amdgcn_raw_ptr_buffer_load_lds(rp, (LAS void*)b, 16, lane * 16, slot, 3072, AUX_SYS);
    }
    __device__ __forceinline__ void finish(LAS unsigned char* b, int u) {
        const int j = u / 96, i = u % 96, t = src * 8 + j;
        const bool isw = i < 32;
        const unsigned img = isw ? w_img(t, i) : x_img(t, xg(i));
        const unsigned h = *(const LAS unsigned*)(b + PK_HDR);
        u32x4 v[8];
#pragma unroll
        for (int r = 0; r < 5; ++r) v[r] = *(const LAS u32x4*)(b + r * 1024 + lane * 16);
        const u32x2 p1 = *(const LAS u32x2*)(b + 5120 + lane * 8);
        const unsigned meta = __builtin_amdgcn_readfirstlane(h), n = meta >> 8, base = meta & 0xffu;
        if (n <= 254) {
            if (n > PK_FIRST) {
                __builtin_amdgcn_raw_ptr_buffer_load_lds(rp, (LAS void*)(b + 4096), 16, lane * 16, OFF_PX + unit_id(u) * UNIT + 4096, 1024, AUX_SYS);
                WAIT_V(0);
            }
            const unsigned base4 = base * 0x01010101u;
            const u32x4 p2 = v[4];
            u32x4 p0[4], e[4];
#pragma unroll
            for (int r = 0; r < 16; ++r) {
                const unsigned hb = (r & 7) >= 2 ? p1[r >> 3] >> ((r & 7) - 2) : p1[r >> 3] << (2 - (r & 7));
                e[r >> 2][r & 3] = (((p2[r >> 2] >> (2 * (r & 3))) & 0x03030303u) | (hb & 0x04040404u)) + base4;
            }
#pragma unroll
            for (int r = 0; r < 4; ++r) { p0[r] = v[r]; *(LAS u32x4*)(b + r * 1024 + lane * 16) = e[r]; }
            CFENCE;
#pragma nounroll
            for (unsigned k = lane; k < n; k += 64) {
                const unsigned en = *(const LAS unsigned short*)(b + PK_HDR + 4 + k * 2), d = en >> 12;
                b[en & 0xfffu] = (unsigned char)(d == 14 ? base + 8 : d == 15 ? 0u : base - 1 - d);
            }
            CFENCE;
#pragma unroll
            for (int r = 0; r < 4; ++r) e[r] = *(const LAS u32x4*)(b + r * 1024 + lane * 16);
#pragma unroll
            for (int r = 0; r < 16; ++r) {
                const unsigned pp = p0[r >> 2][r & 3], ee = e[r >> 2][r & 3];
                const unsigned lo = (pp & ~H8) | ((ee << 7) & H8), hi = (pp & H8) | ((ee >> 1) & ~H8);
                v[r >> 1][2 * (r & 1)] = __builtin_amdgcn_perm(hi, lo, 0x05010400u); v[r >> 1][2 * (r & 1) + 1] = __builtin_amdgcn_perm(hi, lo, 0x07030602u);
            }
        } else {
            const unsigned slot = OFF_PX + unit_id(u) * UNIT;
            WAIT_L(0);
            __builtin_amdgcn_raw_ptr_buffer_load_lds(rp, (LAS void*)b, 16, lane * 16, slot + 6144, 0, AUX_SYS);
            __builtin_amdgcn_raw_ptr_buffer_load_lds(rp, (LAS void*)b, 16, lane * 16, slot + 6144, 1024, AUX_SYS);
            __builtin_amdgcn_raw_ptr_buffer_load_lds(rp, (LAS void*)b, 16, lane * 16, slot + 6144, 2048, AUX_SYS);
            WAIT_V(0);
#pragma unroll
            for (int r = 5; r < 8; ++r) v[r] = *(const LAS u32x4*)(b + (r - 5) * 1024 + lane * 16);
        }
        img_store(v, rs, isw, img, lane);
    }
};
static_assert(OFF_PW == OFF_PX + 512 * UNIT, "unit ids run through the x and the w slots");

#define DRAW(r, tick) { unsigned long long ex_; asm volatile("s_mov_b64 %1, exec\n\ts_mov_b64 exec, 1\n\tglobal_atomic_add %0, %2, %3, %4 sc0\n\ts_mov_b64 exec, %1" \
        : "=&v"(r), "=&s"(ex_) : "v"(0u), "v"(1u), "s"(tick) : "memory"); }
#define TAKE(t, r, N) { unsigned t_; asm volatile("s_waitcnt vmcnt(" #N ")\n\tv_readfirstlane_b32 %0, %1" : "=s"(t_) : "v"(r) : "memory"); t = (int)(t_ - tbase); }
constexpr int NPUSH = PUSH_W * PUSH_U, NU_IN = 768 - NPUSH, NU_ALL = NU_IN + 256;

__device__ __forceinline__ void pull_role(const Params& p, LAS unsigned char* L, int pi, int lane, unsigned call, int wv) {
    Puller P; P.rank = p.rank; P.q = pi % 7; P.src = (p.rank + 1 + P.q) & 7; P.lane = lane; P.seen = 0; P.call = call;
    P.tab = SCHED_TAB.w[P.q][lane];
    asm volatile("" : "+v"(P.tab));
    P.myws = (char*)p.ws[p.rank]; P.rs = mk_rsrc(P.myws, WS_USED); P.rp = mk_rsrc(p.ws[P.src], WS_USED);
    char* tick = P.myws + OFF_TICK + P.q * 128;
    const unsigned tbase = (call - 1) * (NU_ALL + 2 * PPM);
    unsigned r0, r1, pf = 0;
    if (pi >= 7 * PPP) {
        Spin t0;
        while ((int)(__builtin_amdgcn_readfirstlane(ld_agent(tick)) - tbase) < NU_IN) { if (t0.fail(P.myws, call)) break; __builtin_amdgcn_s_sleep(32); }
    }
    DRAW(r0, tick); DRAW(r1, tick);
    int u0, u1, sl = 0;
    TAKE(u0, r0, 0); TAKE(u1, r1, 0);
    if ((unsigned)u0 < (unsigned)NU_IN) P.issue(L, P.unit_of(u0));
    WAIT_V(0);
    STAMP(P.myws, wv, 10);
    while ((unsigned)u0 < (unsigned)NU_IN) {
        unsigned r2;
        if ((unsigned)u1 < (unsigned)NU_IN) { P.issue(L + (sl ^ 1) * PL_SLOT, P.unit_of(u1)); DRAW(r2, tick); WAIT_V(7); }
        else { DRAW(r2, tick); WAIT_V(0); }
        if (pf != 0 && lane == 0) st_sys8(P.myws + pf, call);
        P.finish(L + sl * PL_SLOT, P.unit_of(u0));
        pf = P.flag_of(P.unit_of(u0));
        u0 = u1; sl ^= 1;
        TAKE(u1, r2, 8);
    }
    WAIT_V(0);
    if (pf != 0 && lane == 0) st_sys8(P.myws + pf, call);
    pf = 0;
    STAMP(P.myws, wv, 9);
    unsigned long long seen = 0; int sblk = -1;
#define MASK_ISSUE(b, u) { const int mu_ = (u) - NU_IN; \
        if (!((seen >> (mu_ & 63)) & 1) || (mu_ >> 6) != sblk) { Spin t0; sblk = mu_ >> 6;        \
            for (;;) { CFENCE; const unsigned f = __builtin_amdgcn_raw_buffer_load_b32(P.rs, lane * 4, OFF_MFLG + (P.src * 256 + sblk * 64) * 4, AUX_SYS); \
                seen = __builtin_amdgcn_ballot_w64(f >= call); \
                if ((seen >> (mu_ & 63)) & 1) break; if (t0.fail(P.myws, call)) break; } \
            CFENCE; } \
        __builtin_amdgcn_raw_ptr_buffer_load_lds(P.rp, (LAS void*)(b), 16, lane * 16, OFF_MSK + mu_ * 4096, 0, AUX_SYS); \
        __builtin_amdgcn_raw_ptr_buffer_load_lds(P.rp, (LAS void*)(b), 16, lane * 16, OFF_MSK + mu_ * 4096, 1024, AUX_SYS); \
        __builtin_amdgcn_raw_ptr_buffer_load_lds(P.rp, (LAS void*)(b), 16, lane * 16, OFF_MSK + mu_ * 4096, 2048, AUX_SYS); \
        __builtin_amdgcn_raw_ptr_buffer_load_lds(P.rp, (LAS void*)(b), 16, lane * 16, OFF_MSK + mu_ * 4096, 3072, AUX_SYS); }
    if ((unsigned)u0 < (unsigned)NU_ALL) MASK_ISSUE(L + sl * PL_SLOT, u0);
    WAIT_V(0);
    STAMP(P.myws, wv, 16);
    while ((unsigned)u0 < (unsigned)NU_ALL) {
        unsigned r2;
        if ((unsigned)u1 < (unsigned)NU_ALL) { MASK_ISSUE(L + (sl ^ 1) * PL_SLOT, u1); DRAW(r2, tick); WAIT_V(5); }
        else { DRAW(r2, tick); WAIT_V(0); }
        if (pf != 0 && lane == 0) st_sys(P.myws + pf, call);
#pragma unroll
        for (int r = 0; r < 4; ++r)
            __builtin_amdgcn_raw_buffer_store_b128(*(const LAS u32x4*)(L + sl * PL_SLOT + r * 1024 + lane * 16), P.rs, lane * 16 + r * 1024, OFF_LM + P.src * 1048576 + (u0 - NU_IN) * 4096, LOC_AUX);
        pf = OFF_MF + (P.src * 256 + u0 - NU_IN) * 4;
        u0 = u1; sl ^= 1;
        TAKE(u1, r2, 4);
    }
    WAIT_V(0);
    if (pf != 0 && lane == 0) st_sys(P.myws + pf, call);
    STAMP(P.myws, wv, 17);
}

__device__ __forceinline__ u32x2 lut_expand(unsigned byte) { u32x2 s = {0u, 0u}; unsigned c = 0;
#pragma unroll
    for (int i = 0; i < 8; ++i) { const unsigned b = (byte >> i) & 1u; s[i >> 2] |= (b ? c : 0x0cu) << (8 * (i & 3)); c += b; } return s; }
__device__ __forceinline__ unsigned lut_compact(unsigned nib) { unsigned s = 0x0c0c0c0cu, c = 0;
#pragma unroll
    for (int i = 0; i < 4; ++i) if ((nib >> i) & 1u) { s = (s & ~(0xffu << (8 * c))) | ((unsigned)i << (8 * c)); ++c; } return s; }
__device__ __forceinline__ unsigned popc4(const u32x4& m) { return __builtin_popcount(m[0]) + __builtin_popcount(m[1]) + __builtin_popcount(m[2]) + __builtin_popcount(m[3]); }
__device__ __forceinline__ unsigned nib4(unsigned m, int i) { return (i == 0 ? m << 2 : m >> (4 * i - 2)) & 0x3cu; }
__device__ __forceinline__ unsigned out_voff(int lane) { return (unsigned)(lane & 15) * N_ + (unsigned)(lane >> 4) * 16; }
__device__ __forceinline__ unsigned out_soff(int src, int wt) {
    const int tile = wt >> 3, w = wt & 7;
    return (unsigned)((tile >> 3) * 256 + (w >> 2) * 64) * N_ + (unsigned)(src * NS + (tile & 7) * 256 + (w & 3) * 64);
}
__device__ __forceinline__ constexpr unsigned chunk_soff(int ch) { return (unsigned)((ch >> 2) * 128 + (ch & 3) * 16) * N_; }

__device__ __forceinline__ void expand_role(const Params& p, LAS unsigned char* lds, LAS unsigned char* L, int ei, int lane, unsigned call, int wv) {
    const int rank = p.rank, q = ei % 7, src = (rank + 1 + q) & 7;
    char* myws = (char*)p.ws[rank];
    char* pws = (char*)p.ws[src];
    const rsrc_t rs = mk_rsrc(myws, WS_USED), ro = mk_rsrc(p.out[rank], OUT_BYTES);
#pragma unroll
    for (int i = 0; i < 4; ++i) *(LAS u32x2*)(lds + EX_LUT + (i * 64 + lane) * 8) = lut_expand(i * 64 + lane);
    char* tick = myws + OFF_TICK + (8 + q) * 128;
    const unsigned tbase = (call - 1) * (1024 + 2 * PPO), tag = call & 0x3ffffu;
    bool dead = false;
#define WT_OF(k) (WT_ORD && (unsigned)(k) < 1024u ? ((k) & ~63) | (((k) & 28) << 1) | (((k) >> 3) & 4) | ((k) & 3) : (k))
#define ENT_OFF(t) ((lane & 2) ? OFF_MF + (src * 256 + ((t) >> 2)) * 4 : OFF_DONE + (src * 1024 + (t)) * 8 + (lane & 1) * 4)
#define ENT_OK(e, x, f) (((e) >> 14) == tag && ((x) >> 22) == (tag & 0x3ffu) && (f) >= call)
#define ENT_FETCH(t) __builtin_amdgcn_raw_ptr_buffer_load_lds(rs, (LAS void*)(L + EX_E), 4, ENT_OFF(t), 0, 0, AUX_SYS)
#define EX_ISSUE(t, e, x, f, n, xo) { unsigned e_ = (e), x_ = (x), f_ = (f); \
        if (!ENT_OK(e_, x_, f_) && !dead) { Spin t0_; \
            for (;;) { CFENCE; const unsigned g_ = __builtin_amdgcn_raw_buffer_load_b32(rs, ENT_OFF(t), 0, AUX_SYS); \
                e_ = __builtin_amdgcn_readlane(g_, 0); x_ = __builtin_amdgcn_readlane(g_, 1); f_ = __builtin_amdgcn_readlane(g_, 2); \
                if (ENT_OK(e_, x_, f_)) break; if (t0_.fail(myws, call)) { dead = true; break; } __builtin_amdgcn_s_sleep(ENT_SLEEP); } \
            CFENCE; } \
        n = min(e_ & 0x3fffu, 8192u); xo = x_; \
        const unsigned by_ = min((((x_ >> 21) & 1u) ? n : ((n + 15) >> 4) * 12 + (x_ & 0x3fffu)) + 15 & ~15u, 8192u); \
        const rsrc_t rc_ = mk_rsrc(pws + OFF_COD + (unsigned)(t) * 8192, by_); \
        _Pragma("nounroll") for (unsigned r_ = 0; r_ < by_; r_ += 1024) \
            __builtin_amdgcn_raw_ptr_buffer_load_lds(rc_, (LAS void*)(L + EX_B + r_), 16, lane * 16, r_, 0, AUX_SYS); \
        __builtin_amdgcn_raw_ptr_buffer_load_lds(rs, (LAS void*)(L + EX_M), 16, lane * 16, OFF_LM + src * 1048576 + (t) * 1024, 0, AUX_SYS); }
    unsigned r0, n0 = 0, x0 = 0;
    DRAW(r0, tick);
    int t0;
    TAKE(t0, r0, 0); t0 = WT_OF(t0);
    if ((unsigned)t0 < 1024u) EX_ISSUE(t0, 0u, 0u, 0u, n0, x0);
    int t1;
    { unsigned r1; DRAW(r1, tick); TAKE(t1, r1, 0); t1 = WT_OF(t1); }
    STAMP(myws, wv, 18);
    while ((unsigned)t0 < 1024u) {
        if ((unsigned)t1 < 1024u) ENT_FETCH(t1);
        const u32x4 m = *(const LAS u32x4*)(L + EX_M + lane * 16);
        if ((x0 >> 21) & 1u) {
#pragma nounroll
            for (unsigned r = 0; r < n0; r += 1024) *(LAS u32x4*)(L + r + lane * 16) = *(const LAS u32x4*)(L + EX_B + r + lane * 16);
        } else {
            const unsigned G = (n0 + 15) >> 4, B4 = ((x0 >> 14) & 0x7fu) * 0x01010101u;
            unsigned bm[4] = {0u, 0u, 0u, 0u};
#define UNFOLD(it0, n) { unsigned o_[n][3]; \
                _Pragma("unroll") for (int i = 0; i < n; ++i) { const LAS unsigned* sB = (const LAS unsigned*)(L + EX_B + ((it0 + i) * 64 + lane) * 12); o_[i][0] = sB[0]; o_[i][1] = sB[1]; o_[i][2] = sB[2]; } \
                SCHED; \
                _Pragma("unroll") for (int i = 0; i < n; ++i) { const int it = it0 + i; const unsigned g = it * 64 + lane; \
                    const unsigned s0 = o_[i][0] & 0x3f3f3f3fu, s1 = o_[i][1] & 0x3f3f3f3fu, s2 = o_[i][2] & 0x3f3f3f3fu; \
                    const unsigned s3 = ((o_[i][0] >> 6) & 0x03030303u) | ((o_[i][1] >> 4) & 0x0c0c0c0cu) | ((o_[i][2] >> 2) & 0x30303030u); \
                    *(LAS u32x4*)(L + g * 16) = (u32x4){s0 + B4, s1 + B4, s2 + B4, s3 + B4}; \
                    const unsigned b_ = bits16((s0 + 0x01010101u) & 0x40404040u, (s1 + 0x01010101u) & 0x40404040u, (s2 + 0x01010101u) & 0x40404040u, (s3 + 0x01010101u) & 0x40404040u, 6); \
                    bm[it >> 1] |= (g < G ? b_ : 0u) << (16 * (it & 1)); } }
            UNFOLD(0, 4);
            if (G > 256) { UNFOLD(4, 1); if (G > 320) UNFOLD(5, 3); }
            CYC(0);
            unsigned at;
            const unsigned ne = min(esc_count(bm, at), MAXESC);
            esc_list(bm, at, L + EX_P, lane);
            CFENCE;
#pragma nounroll
            for (unsigned i = lane; i < ne; i += 64) L[*(const LAS unsigned short*)(L + EX_P + i * 2)] = L[EX_B + G * 12 + i];
            CFENCE;
        }
        const unsigned cnt = popc4(m);
        unsigned off = wave_scan(cnt) - cnt + (unsigned)(size_t)L;
        WAIT_L(0);
        CYC(1);
        unsigned n1 = 0, x1 = 0;
        if ((unsigned)t1 < 1024u) {
            WAIT_V(0);
            const u32x4 ent = *(const LAS u32x4*)(L + EX_E);
            EX_ISSUE(t1, __builtin_amdgcn_readfirstlane(ent[0]), __builtin_amdgcn_readfirstlane(ent[1]), __builtin_amdgcn_readfirstlane(ent[2]), n1, x1);
        }
        unsigned r2;
        DRAW(r2, tick);
        CYC(2);
        const unsigned so = out_soff(src, t0), vo = out_voff(lane);
        u32x2 sel[16]; unsigned of[16], w0[16], w1[16], w2[16];
#pragma unroll
        for (int g = 0; g < 16; ++g) {
            const unsigned b8 = ((g & 3) == 0 ? m[g >> 2] << 3 : m[g >> 2] >> (8 * (g & 3) - 3)) & 0x7f8u;
            sel[g] = *(const LAS u32x2*)(lds + EX_LUT + b8);
            const LAS unsigned* w = (const LAS unsigned*)(size_t)(off & ~3u);
            w0[g] = w[0]; w1[g] = w[1]; w2[g] = w[2]; of[g] = off;
            off += __builtin_popcount(b8);
        }
        SCHED;
        u32x4 o[8];
#pragma unroll
        for (int g = 0; g < 16; ++g) {
            const unsigned a0 = __builtin_amdgcn_alignbyte(w1[g], w0[g], of[g]), a1 = __builtin_amdgcn_alignbyte(w2[g], w1[g], of[g]);
            o[g >> 1][(g & 1) * 2] = __builtin_amdgcn_perm(a1, a0, sel[g][0]); o[g >> 1][(g & 1) * 2 + 1] = __builtin_amdgcn_perm(a1, a0, sel[g][1]);
        }
        CYC(3);
        if (dead) {
#pragma unroll
            for (int ch = 0; ch < 8; ++ch) o[ch] = (u32x4){0x7f7f7f7fu, 0x7f7f7f7fu, 0x7f7f7f7fu, 0x7f7f7f7fu};
            CFENCE;
        }
#pragma unroll
        for (int ch = 0; ch < 8; ++ch) __builtin_amdgcn_raw_buffer_store_b128(o[ch], ro, vo, so + chunk_soff(ch), OUT_AUX);
        CYC(4);
        int t2;
        TAKE(t2, r2, 8); t2 = WT_OF(t2);
        CYC(5);
        t0 = t1; n0 = n1; x0 = x1; t1 = t2;
    }
    STAMP(myws, wv, 19);
}

__device__ __forceinline__ unsigned get_call(unsigned callv) { unsigned c; asm volatile("v_readfirstlane_b32 %0, %1" : "=s"(c) : "v"(callv)); return c + 1u; }

__device__ __forceinline__ void comm_role(const Params& p, LAS unsigned char* lds, int c, int wid, int lane, unsigned callv) {
    const int rank = p.rank;
    char* myws = (char*)p.ws[rank];
    const rsrc_t rs = mk_rsrc(myws, WS_USED);
    const int wv = (NGEMM + c) * 8 + wid; (void)wv;
    STAMP(myws, wv, 0);
    if (wid >= 2) {
        const rsrc_t rx = mk_rsrc(p.x, X_BYTES), rw = mk_rsrc(p.w, W_BYTES);
        const int k0 = (wid - 2) * 128 + c;
        const bool far = k0 < 448;
        const unsigned long long tk0 = __builtin_amdgcn_s_memrealtime();
        for (int u = far ? k0 : k0 - 448; u < (far ? 1792 : 768); u += far ? 448 : 320) {
            const bool ownw = !far && u >= 64 && u < 320;
            const int ux = u < 64 ? u : u - 256;
            const int j = far ? u / 224 : ownw ? (u - 64) >> 5 : ux >> 6, r = far ? u % 224 : ownw ? 288 + ((u - 64) & 31) : 224 + (ux & 63), t = rank * 8 + j;
            if (far && j == 0 && r / 7 < NPUSH) continue;
            const unsigned due = far ? (j > 1 ? j * PACE_W - PACE_W0 : j * PACE_W1) : ownw ? PACE_O0 + j * PACE_O : j * PACE_X;
            while ((unsigned)(__builtin_amdgcn_s_memrealtime() - tk0) < due) __builtin_amdgcn_s_sleep(8);
            u32x4 v[8];
            if (r >= 224 && r < 288) {
                const int g = r - 224;
                x_load(v, rx, j, g, lane);
                pack_store(v, rs, lds + 2048 + wid * EX_BYTES, OFF_PX + (unsigned)(j * 64 + g) * UNIT, lane);
                WAIT_V(0);
                const unsigned call = get_call(callv);
                if (j < NPUF) {
                    if (lane < 8) st_sys((char*)p.ws[lane] + OFF_PUF + ((rank * NPUF + j) * 96 + 32 + g) * 4, call);
                    if (j == 0) STAMP(myws, wv, 7);
                }
                unsigned old = 0;
                if (lane == 0) old = __hip_atomic_fetch_add((GAS unsigned*)(myws + OFF_PCNT + j * 128), 1u, __ATOMIC_RELAXED, __HIP_MEMORY_SCOPE_AGENT);
                if ((__builtin_amdgcn_readfirstlane(old) & 63) == 63 && lane < 8) st_sys((char*)p.ws[lane] + OFF_PFLG + t * 128, call);
            } else {
                const bool own = r >= 288;
                const int e = own ? r - 288 : r / 7, d = own ? rank : (rank + 1 + r % 7) & 7;
                u32x2 w[16];
                w_load(w, rw, j, d, e >> 2, e & 1, (e >> 1) & 1, lane);
                w_transpose(v, w);
                if (own) {
                    img_store(v, rs, true, w_img(t, e), lane);
                    WAIT_V(0);
                    if (lane == 0) st_sys8(myws + w_flag(t, e), get_call(callv));
                } else {
                    const unsigned id = (unsigned)((d * 8 + j) * 32 + e);
                    pack_store(v, rs, lds + 2048 + wid * EX_BYTES, OFF_PW + id * UNIT, lane);
                    WAIT_V(0);
                    const unsigned call = get_call(callv);
                    if (j < NPUF) { if (lane == 0) st_sys((char*)p.ws[d] + OFF_PUF + ((rank * NPUF + j) * 96 + e) * 4, call); if (j == 0) STAMP(myws, wv, 6); }
                    unsigned old = 0;
                    if (lane == 0) old = __hip_atomic_fetch_add((GAS unsigned*)(myws + OFF_PCNT + (8 + d * 8 + j) * 128), 1u, __ATOMIC_RELAXED, __HIP_MEMORY_SCOPE_AGENT);
                    if (__builtin_amdgcn_readfirstlane(old) + 1 == call * (j ? 32 : 32 - NPUSH) && lane == 0) st_sys((char*)p.ws[d] + OFF_PFLG + t * 128 + 4, call);
                }
            }
            STAMP(myws, wv, far ? 1 + (j >> 2) : 3 + (j >> 2));
        }
    }
    if (wid < CW) {
        const unsigned call = get_call(callv);
        const int ci = wid * 128 + c;
        if (ci < 7 * PPM) pull_role(p, lds + 2048 + wid * EX_BYTES, ci, lane, call, wv);
        if (ci < 7 * PPO) expand_role(p, lds, lds + 2048 + wid * EX_BYTES, ci, lane, call, wv);
    }
}

typedef float f32x2 __attribute__((ext_vector_type(2)));
__device__ __forceinline__ f32x4 scale4(const f32x4& v, const f32x2& s) {
    f32x2 a, b;
    asm("v_pk_mul_f32 %0, %1, %2" : "=v"(a) : "v"((f32x2){v[0], v[1]}), "v"(s));
    asm("v_pk_mul_f32 %0, %1, %2" : "=v"(b) : "v"((f32x2){v[2], v[3]}), "v"(s));
    return (f32x4){a[0], a[1], b[0], b[1]};
}
__device__ __forceinline__ void gemm_role(const Params& p, LAS unsigned char* lds, int rt, int ct, int wid, int lane, unsigned callv) {
    const int rank = p.rank;
    char* myws = (char*)p.ws[rank];
    const int tile = rt * 8 + ct;
    STAMP(myws, tile * 8 + wid, 0);
    {
        const int pw = ((rt & 3) * 4 + (ct & 3)) * 8 + (rt >> 2) * 2 + (ct >> 2);
        if (NPUSH && wid == 7 && pw < 7 * PUSH_W) {
            const int d = (rank + 1 + pw % 7) & 7, e0 = pw / 7;
            const rsrc_t rw = mk_rsrc(p.w, W_BYTES), rd = mk_rsrc(p.ws[d], WS_USED);
            u32x2 w[PUSH_U ? PUSH_U : 1][16];
#pragma unroll
            for (int i = 0; i < PUSH_U; ++i) { const int e = e0 + i * PUSH_W; w_load(w[i], rw, 0, d, e >> 2, e & 1, (e >> 1) & 1, lane); }
#pragma unroll
            for (int i = 0; i < PUSH_U; ++i) {
                const int e = e0 + i * PUSH_W;
                u32x4 v[8];
                w_transpose(v, w[i]);
                img_store(v, rd, true, w_img(rank * 8, e), lane);
                WAIT_V(0);
                if (lane == 0) st_sys8((char*)p.ws[d] + w_flag(rank * 8, e), get_call(callv));
            }
            STAMP(myws, tile * 8 + wid, 14);
        }
    }
    if (wid >= 1 && wid <= 4) {
        const int j = tile >> 4, g = (tile & 15) * 4 + wid - 1;
        const unsigned long long tk0 = __builtin_amdgcn_s_memrealtime();
        while ((unsigned)(__builtin_amdgcn_s_memrealtime() - tk0) < PACE_O0 + j * PACE_O) __builtin_amdgcn_s_sleep(4);
        u32x4 v[8];
        x_load(v, mk_rsrc(p.x, X_BYTES), j, g, lane);
        img_store(v, mk_rsrc(myws, WS_USED), false, x_img(rank * 8 + j, g), lane);
        WAIT_V(0);
        if (lane == 0) st_sys8(myws + x_flag(rank * 8 + j, g), get_call(callv));
    }
    const unsigned call = get_call(callv);
    const int wv = tile * 8 + wid; (void)wv;
    const int wr = wid >> 2, wc = wid & 3;
    const rsrc_t rs = mk_rsrc(myws, WS_USED);
    const unsigned voff = (unsigned)wid * 2048 + (unsigned)lane * 16;
    LAS unsigned char* ldsw = lds + wid * 2048;
    LAS unsigned char* lda = lds + wr * 8192 + lane * 16;
    LAS unsigned char* ldb = lds + wc * 4096 + lane * 16;
    const unsigned seqf = (unsigned)step_of(lane, rank, rt) * 4, fx = OFF_FLAG + rt * 256, fw = OFF_FLAG + FLAG_W + ct * 256, c4 = (call & 255u) * 0x01010101u;
    unsigned long long todo = ~0ull, rdy = 0;
    unsigned tl = 0;
    volatile LAS unsigned* tq = (volatile LAS unsigned*)(lds + LDS_RED + 160);
#define SA(b, h) (((b) * 2 + (h)) * HALFB)
#define SB(b, h) ((4 + (b) * 2 + (h)) * HALFB)
#define STAGE(bufoff, soff) do { \
        __builtin_amdgcn_raw_ptr_buffer_load_lds(rs, (LAS void*)(ldsw + (bufoff)), 16, voff, (soff), 0, GEMM_AUX); \
        __builtin_amdgcn_raw_ptr_buffer_load_lds(rs, (LAS void*)(ldsw + (bufoff)), 16, voff, (soff), 1024, GEMM_AUX); } while (0)
#define LDA(dst, b, h) do { _Pragma("unroll") for (int m = 0; m < 4; ++m) _Pragma("unroll") for (int k = 0; k < 2; ++k) dst[m][k] = *(const LAS bf16x8*)(lda + SA(b, h) + m * 2048 + k * 1024); } while (0)
#define LDB(dst, b, h) do { _Pragma("unroll") for (int n = 0; n < 2; ++n) _Pragma("unroll") for (int k = 0; k < 2; ++k) dst[n][k] = *(const LAS bf16x8*)(ldb + SB(b, h) + n * 2048 + k * 1024); } while (0)
#define MMA(ai, bj, At, Bt) do { __builtin_amdgcn_s_setprio(1); _Pragma("unroll") for (int m = 0; m < 4; ++m) _Pragma("unroll") for (int n = 0; n < 2; ++n) _Pragma("unroll") for (int k = 0; k < 2; ++k) \
        acc[ai][bj][m][n] = __builtin_amdgcn_mfma_f32_16x16x32_bf16(Bt[n][k], At[m][k], acc[ai][bj][m][n], 0, 0, 0); __builtin_amdgcn_s_setprio(0); } while (0)
#define OFFA(t) (OFF_XT + (unsigned)(t) * XT_STEP + (unsigned)rt * BLK)
#define OFFB(t) (OFF_WT + (unsigned)(t) * WT_STEP + (unsigned)ct * BLK)
    unsigned la = 0, lb = 0;
#define LOOK do { if (wid == 0) { WAIT_V(8); la = *(const LAS unsigned*)(lds + LDS_FLAG + lane * 4); lb = *(const LAS unsigned*)(lds + LDS_FLAG + 256 + lane * 4); } } while (0)
#define PICK(slot) do { if (wid == 0) { \
            if (todo != 0) { \
                rdy |= __builtin_amdgcn_ballot_w64(la == c4 && lb == c4); \
                if ((rdy & todo) == 0) { Spin t0_; \
                    for (;;) { CFENCE; \
                        const unsigned a_ = __builtin_amdgcn_raw_buffer_load_b32(rs, seqf, fx, AUX_SYS), b_ = __builtin_amdgcn_raw_buffer_load_b32(rs, seqf, fw, AUX_SYS); \
                        rdy |= __builtin_amdgcn_ballot_w64(a_ == c4 && b_ == c4); \
                        if ((rdy & todo) != 0) break; if (t0_.fail(myws, call)) { rdy = ~0ull; break; } } } \
                const int pos_ = __builtin_ctzll(rdy & todo); \
                todo &= ~(1ull << pos_); tl = __builtin_amdgcn_readlane(seqf, pos_) >> 2; \
            } \
            tq[slot] = tl; \
            __builtin_amdgcn_raw_ptr_buffer_load_lds(rs, (LAS void*)(lds + LDS_FLAG), 4, seqf, fx, 0, AUX_SYS); \
            __builtin_amdgcn_raw_ptr_buffer_load_lds(rs, (LAS void*)(lds + LDS_FLAG + 256), 4, seqf, fw, 0, AUX_SYS); \
            SCHED; } } while (0)
#define WAIT_STAGE do { if (wid == 0) { WAIT_V(8); WAIT_L(0); } else WAIT_V(6); } while (0)

    f32x4 acc[2][2][4][2];
#pragma unroll
    for (int a = 0; a < 2; ++a)
#pragma unroll
        for (int b = 0; b < 2; ++b)
#pragma unroll
            for (int m = 0; m < 4; ++m)
#pragma unroll
                for (int n = 0; n < 2; ++n) acc[a][b][m][n] = (f32x4){0.f, 0.f, 0.f, 0.f};
    bf16x8 At[4][2], B0[2][2], B1[2][2];

    unsigned t1;
    {
        la = ~c4; PICK(0); PICK(1); PICK(2);
        if (wid == 1 && lane < 8) ((LAS unsigned*)(lds + LDS_RED))[32 + lane] = 0u;
        WAIT_L(0); BAR; CFENCE;
        STAMP(myws, wv, 1);
        const unsigned t0 = __builtin_amdgcn_readfirstlane(tq[0]);
        t1 = __builtin_amdgcn_readfirstlane(tq[1]);
        STAGE(SB(0, 0), OFFB(t0)); STAGE(SA(0, 0), OFFA(t0)); STAGE(SB(0, 1), OFFB(t0) + HALFB); STAGE(SA(0, 1), OFFA(t0) + HALFB);
        if (wr == 1) BAR;
        WAIT_V(4); BAR;
        STAGE(SB(1, 0), OFFB(t1)); STAGE(SA(1, 0), OFFA(t1)); STAGE(SB(1, 1), OFFB(t1) + HALFB);
        WAIT_V(6); BAR;
    }
    for (int i = 0; i < NT; i += 2) {
        LDB(B0, 0, 0); SCHED; LDA(At, 0, 0); STAGE(SA(1, 1), OFFA(t1) + HALFB);
        WAIT_L(8); BAR; WAIT_L(0);
        const unsigned tv2 = tq[2];
        MMA(0, 0, At, B0); BAR; SCHED;
        const unsigned t2 = __builtin_amdgcn_readfirstlane(tv2), a2 = OFFA(t2), b2 = OFFB(t2);
        LDB(B1, 0, 1); STAGE(SB(0, 0), b2);
        BAR; WAIT_L(0); MMA(0, 1, At, B1); BAR;
        LDA(At, 0, 1); STAGE(SA(0, 0), a2); LOOK;
        BAR; WAIT_L(0); MMA(1, 0, At, B0); BAR; SCHED;
        PICK(3);
        STAGE(SB(0, 1), b2 + HALFB);
        WAIT_STAGE; BAR; MMA(1, 1, At, B1); BAR;
        LDB(B0, 1, 0); SCHED; LDA(At, 1, 0); STAGE(SA(0, 1), a2 + HALFB);
        WAIT_L(8); BAR; WAIT_L(0);
        const unsigned tv3 = tq[3];
        MMA(0, 0, At, B0); BAR; SCHED;
        t1 = __builtin_amdgcn_readfirstlane(tv3);
        const unsigned a3 = OFFA(t1), b3 = OFFB(t1);
        LDB(B1, 1, 1); STAGE(SB(1, 0), b3);
        BAR; WAIT_L(0); MMA(0, 1, At, B1); BAR;
        LDA(At, 1, 1); STAGE(SA(1, 0), a3); LOOK;
        BAR; WAIT_L(0); MMA(1, 0, At, B0); BAR; SCHED;
        PICK(2);
        STAGE(SB(1, 1), b3 + HALFB);
        WAIT_STAGE; BAR; MMA(1, 1, At, B1); BAR;
    }
    char* pl = (char*)p.ws[lane & 7];
    if (wr == 0) BAR;
    STAMP(myws, wv, 10);

    const int wt = tile * 8 + wid;
    LAS unsigned* red = (LAS unsigned*)(lds + LDS_RED);
    u32x4 msk;
#pragma unroll
    for (int w = 0; w < 4; ++w) {
        unsigned mm = 0;
#pragma unroll
        for (int e = 31; e >= 0; --e) {
            const int ch = 2 * w + (e >> 4);
            mm = __builtin_amdgcn_alignbit(mm, __float_as_uint(acc[ch >> 2][(e >> 1) & 1][ch & 3][e & 1][(e >> 2) & 3]), 31);
        }
        msk[w] = ~mm;
    }
    __builtin_amdgcn_raw_buffer_store_b128(msk, rs, lane * 16, OFF_MSK + wt * 1024, AUX_SYS);
    SCHED;
    float mx = 0.f;
#pragma unroll
    for (int a = 0; a < 2; ++a)
#pragma unroll
        for (int b = 0; b < 2; ++b)
#pragma unroll
            for (int m = 0; m < 4; ++m)
#pragma unroll
                for (int n = 0; n < 2; ++n) mx = fmaxf(fmaxf(fmaxf(fmaxf(mx, acc[a][b][m][n][0]), acc[a][b][m][n][1]), acc[a][b][m][n][2]), acc[a][b][m][n][3]);
    const unsigned wm = wave_max(__float_as_uint(mx));
    {
        unsigned old = 0;
        if (lane == 0) {
            __hip_atomic_fetch_max(red + 32, wm, __ATOMIC_RELAXED, __HIP_MEMORY_SCOPE_WORKGROUP);
            old = __hip_atomic_fetch_add(red + 33, 1u, __ATOMIC_RELAXED, __HIP_MEMORY_SCOPE_WORKGROUP);
        }
        if (__builtin_amdgcn_readfirstlane(old) == 7) {
            const unsigned long long gv = ((unsigned long long)call << 32) | __hip_atomic_load(red + 32, __ATOMIC_RELAXED, __HIP_MEMORY_SCOPE_WORKGROUP);
            char* g = pl + OFF_AMAX + (rank * NGEMM + tile) * 8;
            unsigned long long ex_;
            asm volatile("s_mov_b64 %0, exec\n\ts_mov_b64 exec, 0xff\n\tglobal_store_dwordx2 %1, %2, off sc0 sc1\n\ts_mov_b64 exec, %0\n\ts_waitcnt vmcnt(1)" : "=&s"(ex_) : "v"(g), "v"(gv) : "memory");
        } else WAIT_V(0);
        unsigned old2 = 0;
        if (lane == 0) old2 = __hip_atomic_fetch_add(red + 34 + wr, 1u, __ATOMIC_RELAXED, __HIP_MEMORY_SCOPE_WORKGROUP);
        if (__builtin_amdgcn_readfirstlane(old2) == 3 && lane < 8) st_sys(pl + OFF_MFLG + (rank * 256 + tile * 2 + wr) * 4, call);
    }
#pragma unroll
    for (int a = 0; a < 2; ++a)
#pragma unroll
        for (int b = 0; b < 2; ++b)
#pragma unroll
            for (int m = 0; m < 4; ++m)
#pragma unroll
                for (int n = 0; n < 2; ++n)
#pragma unroll
                    for (int j = 0; j < 4; ++j) { acc[a][b][m][n][j] = fmaxf(acc[a][b][m][n][j], 0.f); if (j == 3) asm volatile("" : "+v"(acc[a][b][m][n])); }
    STAMP(myws, wv, 13);
    BAR; CFENCE;
    LAS unsigned char* A = lds + wid * 16384;
    LAS unsigned char* B = A + 8192;
    const unsigned cnt = popc4(msk), scan = wave_scan(cnt), Lc = __builtin_amdgcn_readlane(scan, 63);
    unsigned off = scan - cnt;
#pragma unroll
    for (int r = 0; r < 8; ++r) *(LAS u32x4*)(A + r * 1024 + lane * 16) = (u32x4){0u, 0u, 0u, 0u};
    LAS unsigned char* lut = A + 16384 - 64;
    if (lane < 16) *(LAS unsigned*)(lut + lane * 4) = lut_compact(lane);
    unsigned sel[32], of[32];
    off += (unsigned)(size_t)A;
#pragma unroll
    for (int g = 0; g < 32; ++g) {
        const unsigned n4 = nib4(msk[g >> 3], g & 7);
        sel[g] = *(const LAS unsigned*)(lut + n4); of[g] = (off & ~3u) | (off << 30);
        asm volatile("" : "+v"(sel[g]), "+v"(of[g]));
        off += __builtin_popcount(n4);
    }
    {
        Spin t0;
        unsigned t, tm = ld_agent(myws + OFF_TMO);
        for (;;) {
            CFENCE;
            const u32x4 g = __builtin_amdgcn_raw_buffer_load_b128(rs, lane * 16, OFF_AMAX + wid * 1024, AUX_SYS);
            t = max(g[0], g[2]);
            if (__builtin_amdgcn_ballot_w64(g[1] < call || g[3] < call) == 0) break;
            if (t0.fail(myws, call)) { tm = call; break; }
            __builtin_amdgcn_s_sleep(AMAX_SLEEP);
        }
        t = wave_max(t);
        if (lane == 0) { red[8 + wid] = t; red[wid] = tm == call; }
    }
    WAIT_L(0); BAR; CFENCE;
    const float amax = __uint_as_float(wave_max(red[8 + (lane & 7)]));
    const bool dead = __builtin_amdgcn_ballot_w64(red[lane & 7] != 0) != 0;
    const float inv = dead ? __builtin_nanf("") : 448.0f / amax;
    STAMP(myws, wv, 11);
    if (EPI_PRIO && wr == 0) __builtin_amdgcn_s_setprio(2);

    const f32x2 inv2 = {inv, inv};
    u32x4 qv[8];
#pragma unroll
    for (int ch = 0; ch < 8; ++ch) {
        const int a = ch >> 2, m = ch & 3;
        const f32x4 y0 = scale4(acc[a][0][m][0], inv2), y1 = scale4(acc[a][0][m][1], inv2), y2 = scale4(acc[a][1][m][0], inv2), y3 = scale4(acc[a][1][m][1], inv2);
#pragma unroll
        for (int j = 0; j < 4; ++j) {
            int w = __builtin_amdgcn_cvt_pk_fp8_f32(y0[j], y1[j], __float_as_int(y0[j]), false);
            w = __builtin_amdgcn_cvt_pk_fp8_f32(y2[j], y3[j], w, true);
            qv[ch][j] = (unsigned)w;
            const unsigned o_ = of[ch * 4 + j];
            const unsigned long long z = (unsigned long long)__builtin_amdgcn_perm(0u, (unsigned)w, sel[ch * 4 + j]) << (o_ >> 27);
            LAS unsigned* d = (LAS unsigned*)(size_t)(o_ & 0x3ffffu);
            __hip_atomic_fetch_or(d, (unsigned)z, __ATOMIC_RELAXED, __HIP_MEMORY_SCOPE_WAVEFRONT);
            __hip_atomic_fetch_or(d + 1, (unsigned)(z >> 32), __ATOMIC_RELAXED, __HIP_MEMORY_SCOPE_WAVEFRONT);
        }
    }
    CFENCE;
    const unsigned top = __builtin_amdgcn_readfirstlane(__builtin_amdgcn_cvt_pk_fp8_f32(__uint_as_float(wm) * inv, 0.f, 0, false)) & 0x7fu;
    const unsigned base = top > 62 ? top - 62 : 0, B4 = base * 0x01010101u, G = (Lc + 15) >> 4;
    if (lane < 16) A[Lc + lane] = (unsigned char)base;
    CFENCE;
    unsigned bm[4] = {0u, 0u, 0u, 0u};
#define FOLD(it0, n) { u32x4 d_[n]; \
        _Pragma("unroll") for (int i = 0; i < n; ++i) d_[i] = *(const LAS u32x4*)(A + ((it0 + i) * 64 + lane) * 16); \
        SCHED; \
        _Pragma("unroll") for (int i = 0; i < n; ++i) { const int it = it0 + i; const unsigned g = it * 64 + lane; unsigned sy[4], ms[4]; \
            _Pragma("unroll") for (int k = 0; k < 4; ++k) { const unsigned t = (d_[i][k] | H8) - B4; ms[k] = ~t & H8; sy[k] = (t & 0x3f3f3f3fu) | ((ms[k] >> 1) - (ms[k] >> 7)); } \
            LAS unsigned* dB = (LAS unsigned*)(B + g * 12); \
            _Pragma("unroll") for (int k = 0; k < 3; ++k) dB[k] = sy[k] | ((sy[3] << (6 - 2 * k)) & 0xc0c0c0c0u); \
            bm[it >> 1] |= (g < G ? bits16(ms[0], ms[1], ms[2], ms[3], 7) : 0u) << (16 * (it & 1)); } }
    FOLD(0, 4);
    if (G > 256) { FOLD(4, 1); if (G > 320) FOLD(5, 3); }
    unsigned at;
    const unsigned ne = esc_count(bm, at);
    const bool raw = ne > MAXESC || G * 12 + ne > Lc;
    if (!raw) {
        LAS unsigned char* list = B + 6656;
        esc_list(bm, at, list, lane);
        CFENCE;
#pragma nounroll
        for (unsigned i = lane; i < ne; i += 64) B[G * 12 + i] = A[*(const LAS unsigned short*)(list + i * 2)];
        CFENCE;
    }
    {
        LAS unsigned char* S = raw ? A : B;
#pragma nounroll
        for (unsigned r = 0; r < (raw ? Lc : G * 12 + ne); r += 1024)
            __builtin_amdgcn_raw_buffer_store_b128(*(const LAS u32x4*)(S + r + lane * 16), rs, lane * 16, OFF_COD + wt * 8192 + r, AUX_SYS);
    }
    if (lane == 0) { red[16 + wid] = ((call & 0x3ffffu) << 14) | Lc; red[24 + wid] = (call << 22) | (raw ? 1u << 21 : (base << 14) | ne); }
    WAIT_V(0);
    {
        unsigned old = 0;
        if (lane == 0) old = __hip_atomic_fetch_add(red + 36 + wr, 1u, __ATOMIC_RELAXED, __HIP_MEMORY_SCOPE_WORKGROUP);
        if (__builtin_amdgcn_readfirstlane(old) == 3 && lane < 32 && (lane >> 2) != rank) {
            const int w = wr * 4 + (lane & 3);
            __hip_atomic_store((GAS unsigned long long*)((char*)p.ws[lane >> 2] + OFF_DONE + (rank * 1024 + tile * 8 + w) * 8),
                               ((unsigned long long)((volatile LAS unsigned*)red)[24 + w] << 32) | ((volatile LAS unsigned*)red)[16 + w], __ATOMIC_RELAXED, __HIP_MEMORY_SCOPE_SYSTEM);
        }
    }
    STAMP(myws, wv, 12);
    {
        const rsrc_t ro = mk_rsrc(p.out[rank], OUT_BYTES);
        const unsigned so = out_soff(rank, wt), vo = out_voff(lane);
#pragma unroll
        for (int ch = 0; ch < 8; ++ch) __builtin_amdgcn_raw_buffer_store_b128(qv[ch], ro, vo, so + chunk_soff(ch), OUT_AUX);
    }
}

__global__ __launch_bounds__(512, 2) void fused_kernel(Params p) {
    __shared__ __attribute__((aligned(16))) unsigned char lds_arr[LDS_BYTES];
    LAS unsigned char* lds = (LAS unsigned char*)lds_arr;
    const int wid = __builtin_amdgcn_readfirstlane(threadIdx.x >> 6), lane = threadIdx.x & 63;
    const int b = blockIdx.x, xcd = b & 7, idx = b >> 3;
    const unsigned callv = (unsigned)__builtin_amdgcn_raw_ptr_buffer_atomic_add_i32(1, mk_rsrc(p.ws[p.rank], WS_USED), lane == 0 ? OFF_CNT + (b * 8 + wid) * 4 : 0xfffffff0u, 0, 0);
    if (idx < 16) gemm_role(p, lds, (xcd >> 1) * 4 + (idx >> 2), (xcd & 1) * 4 + (idx & 3), wid, lane, callv);
    else comm_role(p, lds, (idx - 16) * 8 + xcd, wid, lane, callv);
}

extern "C" void kernel_launch(int rank, int nranks, ncclComm_t comm, hipStream_t stream,
                              void* const* d_in, const int* in_sizes, int n_in,
                              void* d_out, int out_size,
                              const int* in_dtypes, int out_dtype,
                              void* const* d_out_all, void* const* d_ws_all, size_t ws_bytes) {
    (void)comm; (void)in_dtypes; (void)out_dtype; (void)d_out;
    if (nranks != NR || n_in != 2 || in_sizes[0] != M_ * KS || in_sizes[1] != KS * N_ || out_size != M_ * N_ || ws_bytes < WS_USED || (unsigned)rank >= (unsigned)NR) {
        fprintf(stderr, "kernel_launch: unexpected problem shape\n");
        return;
    }
    Params p;
    p.x = d_in[0]; p.w = d_in[1];
    for (int i = 0; i < NR; ++i) { p.out[i] = d_out_all[i]; p.ws[i] = d_ws_all[i]; }
    p.rank = rank; p.pad = 0;
    fused_kernel<<<dim3(NWG), dim3(512), 0, stream>>>(p);
}
```

```cpp
#include <hip/hip_runtime.h>
#include <rccl/rccl.h>
#include <cstdio>

#define LAS __attribute__((address_space(3)))
#define GAS __attribute__((address_space(1)))
typedef short bf16x8 __attribute__((ext_vector_type(8)));
typedef float f32x4 __attribute__((ext_vector_type(4)));
typedef unsigned u32x4 __attribute__((ext_vector_type(4)));
typedef unsigned u32x2 __attribute__((ext_vector_type(2)));
typedef __amdgpu_buffer_rsrc_t rsrc_t;

constexpr int NR = 8;
constexpr int M_ = 4096, N_ = 16384;
constexpr int KS = 512;
constexpr int NS = 2048;
constexpr int NT = 64;
constexpr int NGEMM = 128;
constexpr int NWG = 256;

constexpr unsigned OFF_CNT  = 0;
constexpr unsigned OFF_TMO  = 16384;
constexpr unsigned OFF_FLAG = 65536;
constexpr unsigned FLAG_W = 4096;
constexpr unsigned OFF_AMAX = 131072;
constexpr unsigned OFF_TICK = 143360;
constexpr unsigned OFF_PCNT = 147456;
constexpr unsigned OFF_PFLG = 163840;
constexpr unsigned OFF_PUF  = 188416;
constexpr unsigned OFF_MFLG = 176128;
constexpr unsigned OFF_DONE = 786432;
constexpr unsigned OFF_MF   = OFF_DONE + 65536;
constexpr unsigned OFF_DBG  = 262144;
constexpr unsigned CTRL_BYTES = 1u << 20;
constexpr unsigned XT_STEP = 512u * 1024u, WT_STEP = 256u * 1024u, BLK = 32768u, HALFB = 16384u;
constexpr unsigned OFF_XT = CTRL_BYTES;
constexpr unsigned XT_BYTES = NT * XT_STEP;
constexpr unsigned OFF_WT = OFF_XT + XT_BYTES;
constexpr unsigned WT_BYTES = NT * WT_STEP;
constexpr unsigned UNIT = 9216;
constexpr unsigned OFF_PX = OFF_WT + WT_BYTES;
constexpr unsigned OFF_PW = OFF_PX + 512 * UNIT;
constexpr unsigned OFF_MSK = OFF_PW + 2048 * UNIT;
constexpr unsigned OFF_COD = OFF_MSK + 1024 * 1024;
constexpr unsigned OFF_LM = OFF_COD + 1024 * 8192;
constexpr unsigned WS_USED = OFF_LM + 8 * 1024 * 1024;
constexpr unsigned OUT_BYTES = (unsigned)M_ * N_;
constexpr unsigned X_BYTES = (unsigned)M_ * KS * 2, W_BYTES = (unsigned)KS * N_ * 2;

constexpr unsigned long long DEADLINE = 100000000ull;
constexpr int AUX_SYS = 17;
#define GEMM_AUX 17
#define PPP 16
#define PPO 53
#define PPM 17
#define PACE_W 800
#define PACE_W0 800
#define PACE_W1 150
#define PACE_X 400
#define PACE_O0 100
#define PACE_O 60
#define PULL_SERIAL 0
#define AMAX_SLEEP 4
#define NPUF 2
#define PACK_AUX 17
#define INX_AUX 0
#define IN_AUX 2
#define SCH_S 100
#define SCH_C 1000
#define PUSH_W 8
#define PUSH_U 3
#define EPI_PRIO 1
#define WT_ORD 1
#define ENT_SLEEP 8
#define SCH_A 1
#define SCH_B 10
#define CW 4
#define OUT_AUX 16
#define LOC_AUX 17

constexpr unsigned LDS_STAGE = 8 * HALFB;
constexpr unsigned EP_BYTES = 19456;
constexpr unsigned EP_C = 8192, EP_S = EP_C + 5888, EP_L = EP_S + 2080, EP_T = 18432, EP_X = EP_T + 64;
constexpr unsigned LDS_FLAG = 8 * EP_BYTES;
constexpr unsigned LDS_RED = LDS_FLAG + 512;
constexpr unsigned EX_B = 8192, EX_M = EX_B + 8192, EX_P = EX_M + 1024, EX_E = EX_P + 1024, EX_BYTES = EX_E + 256, EX_LUT = 0;
constexpr unsigned LDS_BYTES = LDS_RED + 256;
static_assert(2048 + 8 * EX_BYTES <= LDS_BYTES && EP_BYTES % 1024 == 0 && EP_L + 1024 <= EP_T && EP_X + 256 <= EP_BYTES && LDS_STAGE <= LDS_FLAG && LDS_RED + 192 <= LDS_BYTES && LDS_BYTES <= 163840, "lds");
static_assert(OFF_PUF + 8 * NPUF * 384 <= OFF_DBG, "flags");
static_assert(LOC_AUX == AUX_SYS, "img_store also writes to peers");
static_assert((PUSH_W * PUSH_U) % 4 == 0 && PUSH_W * PUSH_U <= 32 && 7 * PUSH_W <= NGEMM, "push");
static_assert(7 * PPO <= 128 * CW && 7 * PPM <= 128 * CW && PPP <= PPM, "waves");

struct Params {
    const void* x; const void* w;
    void* out[NR]; void* ws[NR];
    int rank; int pad;
};

__device__ __forceinline__ rsrc_t mk_rsrc(const void* p, unsigned bytes) {
    return __builtin_amdgcn_make_buffer_rsrc((void*)p, 0, bytes, 0x00020000);
}
#define WAIT_V(n) asm volatile("s_waitcnt vmcnt(" #n ")" ::: "memory")
#define WAIT_L(n) asm volatile("s_waitcnt lgkmcnt(" #n ")" ::: "memory")
#define BAR __builtin_amdgcn_s_barrier()
#define SCHED __builtin_amdgcn_sched_barrier(0)
#define CFENCE asm volatile("" ::: "memory")

#define STAMP(ws_, wv_, k_) do {} while (0)
#define CYC(i) do {} while (0)

__device__ __forceinline__ unsigned ld_agent(const void* p) { return __hip_atomic_load((const GAS unsigned*)p, __ATOMIC_RELAXED, __HIP_MEMORY_SCOPE_AGENT); }
__device__ __forceinline__ void st_agent(void* p, unsigned v) { __hip_atomic_store((GAS unsigned*)p, v, __ATOMIC_RELAXED, __HIP_MEMORY_SCOPE_AGENT); }
__device__ __forceinline__ void st_sys(void* p, unsigned v) { __hip_atomic_store((GAS unsigned*)p, v, __ATOMIC_RELAXED, __HIP_MEMORY_SCOPE_SYSTEM); }

__device__ __forceinline__ void st_sys8(void* p, unsigned v) { __hip_atomic_store((GAS unsigned char*)p, (unsigned char)v, __ATOMIC_RELAXED, __HIP_MEMORY_SCOPE_SYSTEM); }
__device__ __forceinline__ unsigned x_flag(int t, int g) { return OFF_FLAG + (g >> 2) * 256 + t * 4 + (g & 3); }
__device__ __forceinline__ unsigned w_flag(int t, int e) { return OFF_FLAG + FLAG_W + (e >> 2) * 256 + t * 4 + (e & 3); }

struct Spin {
    unsigned long long t0; unsigned n;
    __device__ __forceinline__ Spin() : t0(__builtin_amdgcn_s_memrealtime()), n(0) {}
    __device__ __forceinline__ bool fail(char* myws, unsigned call) {
        if ((++n & 15u) == 0 && (__builtin_amdgcn_readfirstlane(ld_agent(myws + OFF_TMO)) == call || __builtin_amdgcn_s_memrealtime() - t0 > DEADLINE)) {
            st_agent(myws + OFF_TMO, call);
            return true;
        }
        __builtin_amdgcn_s_sleep(1);
        return false;
    }
};

#define DPP0(x, ctrl, rm) (unsigned)__builtin_amdgcn_update_dpp(0, (int)(x), ctrl, rm, 0xf, false)
__device__ __forceinline__ unsigned wave_scan(unsigned x) {
    x += DPP0(x, 0x111, 0xf); x += DPP0(x, 0x112, 0xf); x += DPP0(x, 0x114, 0xf); x += DPP0(x, 0x118, 0xf);
    x += DPP0(x, 0x142, 0xa); x += DPP0(x, 0x143, 0xc);
    return x;
}
__device__ __forceinline__ unsigned wave_max(unsigned x) {
    x = max(x, DPP0(x, 0x111, 0xf)); x = max(x, DPP0(x, 0x112, 0xf)); x = max(x, DPP0(x, 0x114, 0xf)); x = max(x, DPP0(x, 0x118, 0xf));
    x = max(x, DPP0(x, 0x142, 0xa)); x = max(x, DPP0(x, 0x143, 0xc));
    return __builtin_amdgcn_readlane(x, 63);
}

struct Sched {
    unsigned w[7][64];
    constexpr Sched() : w{} {
        for (int q = 0; q < 7; ++q) {
            int key[136] = {}, id[136] = {}, n = 0;
            for (int j = 0; j < 8; ++j) {
                int lo = 1 << 30;
                for (int rt = 0; rt < 16; ++rt) { const int k = (SCH_S * rt < SCH_C ? SCH_S * rt : SCH_C) + SCH_B * rt + 106 * (7 * j + (q + SCH_A * rt) % 7); key[n] = 2 * k + 1; id[n++] = j * 24 + 8 + rt; lo = k < lo ? k : lo; }
                key[n] = 2 * lo; id[n++] = -1 - j;
            }
            int at = 0;
            for (int o = 0; o < 136; ++o) {
                int best = -1;
                for (int i = 0; i < 136; ++i) if (key[i] >= 0 && (best < 0 || key[i] < key[best])) best = i;
                key[best] = -1;
                const int skip = id[best] == -1 ? PUSH_W * PUSH_U / 4 : 0;
                const int cnt = id[best] < 0 ? 8 - skip : 1, first = id[best] < 0 ? (-1 - id[best]) * 24 + skip : id[best];
                for (int c = 0; c < cnt; ++c, ++at) w[q][at >> 2] |= (unsigned)(first + c) << (8 * (at & 3));
            }
        }
    }
};
__constant__ const Sched SCHED_TAB;
__device__ __forceinline__ int step_of(int i, int rank, int rt) {
    if (i < 8) return rank * 8 + i;
    const int z = i - 8, j = z / 7, p = z - j * 7, q = (p + 7 * 16 * SCH_A - SCH_A * rt) % 7;
    return (((rank + 1 + q) & 7) << 3) + j;
}

__device__ __forceinline__ void x_load(u32x4 (&v)[8], rsrc_t rx, int j, int g, int lane) {
    const unsigned voff = (unsigned)(lane & 15) * (KS * 2) + (unsigned)(lane >> 4) * 16;
    const unsigned soff = (unsigned)g * 64 * (KS * 2) + (unsigned)j * 128;
#pragma unroll
    for (int r = 0; r < 4; ++r)
#pragma unroll
        for (int kk = 0; kk < 2; ++kk) v[r * 2 + kk] = __builtin_amdgcn_raw_buffer_load_b128(rx, voff + kk * 64, soff + r * 16 * (KS * 2), INX_AUX);
}
__device__ __forceinline__ void w_load(u32x2 (&v)[16], rsrc_t rw, int j, int d, int ct, int kk, int G, int lane) {
    const unsigned voff = (unsigned)(lane >> 4) * 8 * (N_ * 2) + (unsigned)(lane & 15) * 8;
    const unsigned soff = (unsigned)(j * 64 + kk * 32) * (N_ * 2) + (unsigned)(d * NS + ct * 256 + G * 128) * 2;
#pragma unroll
    for (int jj = 0; jj < 8; ++jj)
#pragma unroll
        for (int g = 0; g < 2; ++g) v[g * 8 + jj] = __builtin_amdgcn_raw_buffer_load_b64(rw, voff + g * 128, soff + jj * (N_ * 2), IN_AUX);
}
__device__ __forceinline__ void w_transpose(u32x4 (&o)[8], const u32x2 (&v)[16]) {
#pragma unroll
    for (int g = 0; g < 2; ++g)
#pragma unroll
        for (int c = 0; c < 4; ++c)
#pragma unroll
            for (int dd = 0; dd < 4; ++dd)
                o[g * 4 + c][dd] = __builtin_amdgcn_perm(v[g * 8 + 2 * dd + 1][c >> 1], v[g * 8 + 2 * dd][c >> 1], (c & 1) ? 0x07060302u : 0x05040100u);
}
__device__ __forceinline__ unsigned x_img(int t, int g) { return OFF_XT + (unsigned)t * XT_STEP + (unsigned)g * 8192; }
__device__ __forceinline__ unsigned w_img(int t, int e) { return OFF_WT + (unsigned)t * WT_STEP + (unsigned)(e >> 2) * BLK + (unsigned)((e >> 1) & 1) * 8192 + (unsigned)(e & 1) * 1024; }
__device__ __forceinline__ unsigned frag_off(bool isw, unsigned f) { return isw ? ((f >> 1) & 1) * HALFB + (f >> 2) * 4096 + (f & 1) * 2048 : f * 1024; }
__device__ __forceinline__ void img_store(const u32x4 (&v)[8], rsrc_t rd, bool isw, unsigned img, int lane) {
#pragma unroll
    for (int f = 0; f < 8; ++f) __builtin_amdgcn_raw_buffer_store_b128(v[f], rd, lane * 16, img + frag_off(isw, f), LOC_AUX);
}
constexpr unsigned H8 = 0x80808080u, MAXESC = 512;
__device__ __forceinline__ unsigned KS_(unsigned c) { asm("" : "+s"(c)); return c; }
__device__ __forceinline__ unsigned bits16(unsigned m0, unsigned m1, unsigned m2, unsigned m3, int sh) {
    const unsigned a = __builtin_amdgcn_udot4(m0, 0x08040201u, __builtin_amdgcn_udot4(m1, 0x80402010u, 0u, false), false);
    const unsigned b = __builtin_amdgcn_udot4(m2, 0x08040201u, __builtin_amdgcn_udot4(m3, 0x80402010u, 0u, false), false);
    return (a >> sh) | (b << (8 - sh));
}
__device__ __forceinline__ unsigned esc_count(const unsigned (&bm)[4], unsigned& first) {
    const unsigned c = __builtin_popcount(bm[0]) + __builtin_popcount(bm[1]) + __builtin_popcount(bm[2]) + __builtin_popcount(bm[3]), s = wave_scan(c);
    first = s - c;
    return __builtin_amdgcn_readlane(s, 63);
}
__device__ __forceinline__ void esc_list(const unsigned (&bm)[4], unsigned at, LAS unsigned char* list, int lane) {
#pragma unroll
    for (int w = 0; w < 2; ++w) {
        unsigned long long z = ((unsigned long long)bm[2 * w + 1] << 32) | bm[2 * w];
        while (__builtin_amdgcn_ballot_w64(z != 0) != 0) {
            if (z != 0) {
                const unsigned b = __builtin_ctzll(z);
                *(LAS unsigned short*)(list + min(at, MAXESC - 1) * 2) = (unsigned short)(w * 4096 + (b >> 4) * 1024 + lane * 16 + (b & 15));
                ++at; z &= z - 1;
            }
        }
    }
}
constexpr unsigned PK_HDR = 5 * 1024 + 512, PK_RAW = 0xffffu, PK_FIRST = 62;
__device__ __forceinline__ void pack_store(const u32x4 (&v)[8], rsrc_t rs, LAS unsigned char* L, unsigned slot, int lane) {
    unsigned p0[16], e[16], sum = 0;
#pragma unroll
    for (int r = 0; r < 16; ++r) {
        const unsigned lo = __builtin_amdgcn_perm(v[r >> 1][2 * (r & 1) + 1], v[r >> 1][2 * (r & 1)], 0x06040200u);
        const unsigned hi = __builtin_amdgcn_perm(v[r >> 1][2 * (r & 1) + 1], v[r >> 1][2 * (r & 1)], 0x07050301u);
        p0[r] = (hi & H8) | (lo & ~H8);
        e[r] = ((hi << 1) & 0xfefefefeu) | ((lo >> 7) & 0x01010101u);
        sum = __builtin_amdgcn_sad_u8(e[r], 0u, sum);
    }
#pragma unroll
    for (int r = 0; r < 4; ++r) *(LAS u32x4*)(L + r * 1024 + lane * 16) = (u32x4){e[4 * r], e[4 * r + 1], e[4 * r + 2], e[4 * r + 3]};
    const int tot = (int)__builtin_amdgcn_readlane(wave_scan(sum), 63);
    const unsigned base = (unsigned)min(max((tot - 16794) >> 12, 0), 247), base4 = base * 0x01010101u;
    u32x4 p2 = {0u, 0u, 0u, 0u}; u32x2 p1 = {0u, 0u};
    unsigned ms[16];
#pragma unroll
    for (int r = 0; r < 16; ++r) {
        const unsigned z = ((e[r] | H8) - (base4 & ~H8)) ^ ((e[r] ^ ~base4) & H8);
        ms[r] = (((z & 0x78787878u) + 0x78787878u) | z) & H8;
        p2[r >> 2] |= (z & 0x03030303u) << (2 * (r & 3));
        p1[r >> 3] |= (r & 7) >= 2 ? (z & 0x04040404u) << ((r & 7) - 2) : (z & 0x04040404u) >> (2 - (r & 7));
    }
    unsigned bm[4] = {bits16(ms[0], ms[1], ms[2], ms[3], 7) | (bits16(ms[4], ms[5], ms[6], ms[7], 7) << 16),
                      bits16(ms[8], ms[9], ms[10], ms[11], 7) | (bits16(ms[12], ms[13], ms[14], ms[15], 7) << 16), 0u, 0u};
    unsigned at;
    const unsigned n = esc_count(bm, at);
    bool bad = n > 254;
    *(LAS u32x2*)(L + 4096 + lane * 8) = p1;
    if (!bad) {
        esc_list(bm, at, L + 5120, lane);
        CFENCE;
        bool b = false;
#pragma nounroll
        for (unsigned i = lane; i < n; i += 64) {
            const unsigned pos = *(const LAS unsigned short*)(L + 5120 + i * 2), ev = L[pos];
            unsigned d = base - 1 - ev;
            if (ev == base + 8) d = 14; else if (ev == 0) d = 15; else b |= d > 13;
            *(LAS unsigned short*)(L + 4096 + 516 + i * 2) = (unsigned short)(pos | (d << 12));
        }
        bad = __builtin_amdgcn_ballot_w64(b) != 0;
    }
    if (lane == 0) *(LAS unsigned*)(L + 4096 + 512) = base | ((bad ? PK_RAW : n) << 8);
    CFENCE;
    __builtin_amdgcn_raw_buffer_store_b128(*(const LAS u32x4*)(L + 4096 + lane * 16), rs, lane * 16 + 1024, slot + 4096, PACK_AUX);
    if (!bad) {
#pragma unroll
        for (int r = 0; r < 4; ++r) __builtin_amdgcn_raw_buffer_store_b128((u32x4){p0[4 * r], p0[4 * r + 1], p0[4 * r + 2], p0[4 * r + 3]}, rs, lane * 16 + r * 1024, slot, PACK_AUX);
        __builtin_amdgcn_raw_buffer_store_b128(p2, rs, lane * 16, slot + 4096, PACK_AUX);
    } else {
#pragma unroll
        for (int f = 0; f < 8; ++f) __builtin_amdgcn_raw_buffer_store_b128(v[f], rs, lane * 16, slot + (f < 5 ? f : f + 1) * 1024, PACK_AUX);
    }
}
constexpr unsigned PL_SLOT = 8192;
struct Puller {
    rsrc_t rp, rs; char* myws; int rank, src, q, lane; unsigned call, seen, tab;
    __device__ __forceinline__ int unit_of(int tk) const { return (int)((__builtin_amdgcn_readlane(tab, tk >> 4) >> (2 * (tk & 12))) & 0xffu) * 4 + (tk & 3); }
    __device__ __forceinline__ int xg(int i) const { return i - 32; }
    __device__ __forceinline__ unsigned unit_id(int u) const { const int j = u / 96, i = u % 96; return i < 32 ? 512 + (rank * 8 + j) * 32 + i : j * 64 + xg(i); }
    __device__ __forceinline__ unsigned flag_of(int u) const { const int j = u / 96, i = u % 96; return i < 32 ? w_flag(src * 8 + j, i) : x_flag(src * 8 + j, xg(i)); }
    __device__ __forceinline__ void issue(LAS unsigned char* b, int u) {
        const int j = u / 96, i = u % 96, need = j * 2 + (i >= 32);
        if (!((seen >> need) & 1u)) {
            Spin t0;
            for (;;) {
                CFENCE;
                const unsigned sum = OFF_PFLG + (src * 8 + j) * 128 + (i >= 32 ? 0 : 4);
                const unsigned f = __builtin_amdgcn_raw_buffer_load_b32(rs, (lane & 1) && j < NPUF ? OFF_PUF + ((src * NPUF + j) * 96 + i) * 4 : sum, 0, AUX_SYS);
                if (__builtin_amdgcn_readlane(f, 0) >= call) { seen |= 1u << need; break; }
                if (__builtin_amdgcn_readlane(f, 1) >= call) break;
                if (t0.fail(myws, call)) break;
            }
            CFENCE;
        }
        const unsigned id = unit_id(u), slot = OFF_PX + id * UNIT;
        __builtin_amdgcn_raw_ptr_buffer_load_lds(rp, (LAS void*)(b + 4096), 16, lane < 40 ? lane * 16 : 0xf0000000u, slot + 4096, 1024, AUX_SYS);
        __builtin_amdgcn_raw_ptr_buffer_load_lds(rp, (LAS void*)(b + 4096), 16, lane * 16, slot + 4096, 0, AUX_SYS);
        __builtin_amdgcn_raw_ptr_buffer_load_lds(rp, (LAS void*)b, 16, lane * 16, slot, 0, AUX_SYS);
        __builtin_amdgcn_raw_ptr_buffer_load_lds(rp, (LAS void*)b, 16, lane * 16, slot, 1024, AUX_SYS);
        __builtin_amdgcn_raw_ptr_buffer_load_lds(rp, (LAS void*)b, 16, lane * 16, slot, 2048, AUX_SYS);
        __builtin_amdgcn_raw_ptr_buffer_load_lds(rp, (LAS void*)b, 16, lane * 16, slot, 3072, AUX_SYS);
    }
    __device__ __forceinline__ void finish(LAS unsigned char* b, int u) {
        const int j = u / 96, i = u % 96, t = src * 8 + j;
        const bool isw = i < 32;
        const unsigned img = isw ? w_img(t, i) : x_img(t, xg(i));
        const unsigned h = *(const LAS unsigned*)(b + PK_HDR);
        u32x4 v[8];
#pragma unroll
        for (int r = 0; r < 5; ++r) v[r] = *(const LAS u32x4*)(b + r * 1024 + lane * 16);
        const u32x2 p1 = *(const LAS u32x2*)(b + 5120 + lane * 8);
        const unsigned meta = __builtin_amdgcn_readfirstlane(h), n = meta >> 8, base = meta & 0xffu;
        if (n <= 254) {
            if (n > PK_FIRST) {
                __builtin_amdgcn_raw_ptr_buffer_load_lds(rp, (LAS void*)(b + 4096), 16, lane * 16, OFF_PX + unit_id(u) * UNIT + 4096, 1024, AUX_SYS);
                WAIT_V(0);
            }
            const unsigned base4 = base * 0x01010101u;
            const u32x4 p2 = v[4];
            u32x4 p0[4], e[4];
#pragma unroll
            for (int r = 0; r < 16; ++r) {
                const unsigned hb = (r & 7) >= 2 ? p1[r >> 3] >> ((r & 7) - 2) : p1[r >> 3] << (2 - (r & 7));
                e[r >> 2][r & 3] = (((p2[r >> 2] >> (2 * (r & 3))) & KS_(0x03030303u)) | (hb & 0x04040404u)) + base4;
            }
#pragma unroll
            for (int r = 0; r < 4; ++r) { p0[r] = v[r]; *(LAS u32x4*)(b + r * 1024 + lane * 16) = e[r]; }
            CFENCE;
#pragma nounroll
            for (unsigned k = lane; k < n; k += 64) {
                const unsigned en = *(const LAS unsigned short*)(b + PK_HDR + 4 + k * 2), d = en >> 12;
                b[en & 0xfffu] = (unsigned char)(d == 14 ? base + 8 : d == 15 ? 0u : base - 1 - d);
            }
            CFENCE;
#pragma unroll
            for (int r = 0; r < 4; ++r) e[r] = *(const LAS u32x4*)(b + r * 1024 + lane * 16);
#pragma unroll
            for (int r = 0; r < 16; ++r) {
                const unsigned pp = p0[r >> 2][r & 3], ee = e[r >> 2][r & 3];
                const unsigned lo = (pp & ~H8) | ((ee << 7) & H8), hi = (pp & H8) | ((ee >> 1) & ~H8);
                v[r >> 1][2 * (r & 1)] = __builtin_amdgcn_perm(hi, lo, 0x05010400u); v[r >> 1][2 * (r & 1) + 1] = __builtin_amdgcn_perm(hi, lo, 0x07030602u);
            }
        } else {
            const unsigned slot = OFF_PX + unit_id(u) * UNIT;
            WAIT_L(0);
            __builtin_amdgcn_raw_ptr_buffer_load_lds(rp, (LAS void*)b, 16, lane * 16, slot + 6144, 0, AUX_SYS);
            __builtin_amdgcn_raw_ptr_buffer_load_lds(rp, (LAS void*)b, 16, lane * 16, slot + 6144, 1024, AUX_SYS);
            __builtin_amdgcn_raw_ptr_buffer_load_lds(rp, (LAS void*)b, 16, lane * 16, slot + 6144, 2048, AUX_SYS);
            WAIT_V(0);
#pragma unroll
            for (int r = 5; r < 8; ++r) v[r] = *(const LAS u32x4*)(b + (r - 5) * 1024 + lane * 16);
        }
        img_store(v, rs, isw, img, lane);
    }
};
static_assert(OFF_PW == OFF_PX + 512 * UNIT, "unit ids run through the x and the w slots");

#define DRAW(r, tick) { unsigned long long ex_; asm volatile("s_mov_b64 %1, exec\n\ts_mov_b64 exec, 1\n\tglobal_atomic_add %0, %2, %3, %4 sc0\n\ts_mov_b64 exec, %1" \
        : "=&v"(r), "=&s"(ex_) : "v"(0u), "v"(1u), "s"(tick) : "memory"); }
#define TAKE(t, r, N) { unsigned t_; asm volatile("s_waitcnt vmcnt(" #N ")\n\tv_readfirstlane_b32 %0, %1" : "=s"(t_) : "v"(r) : "memory"); t = (int)(t_ - tbase); }
constexpr int NPUSH = PUSH_W * PUSH_U, NU_IN = 768 - NPUSH, NU_ALL = NU_IN + 256;

__device__ __forceinline__ void pull_role(const Params& p, LAS unsigned char* L, int pi, int lane, unsigned call, int wv) {
    Puller P; P.rank = p.rank; P.q = pi % 7; P.src = (p.rank + 1 + P.q) & 7; P.lane = lane; P.seen = 0; P.call = call;
    P.tab = SCHED_TAB.w[P.q][lane];
    asm volatile("" : "+v"(P.tab));
    P.myws = (char*)p.ws[p.rank]; P.rs = mk_rsrc(P.myws, WS_USED); P.rp = mk_rsrc(p.ws[P.src], WS_USED);
    char* tick = P.myws + OFF_TICK + P.q * 128;
    const unsigned tbase = (call - 1) * (NU_ALL + 2 * PPM);
    unsigned r0, r1, pf = 0;
    if (pi >= 7 * PPP) {
        Spin t0;
        while ((int)(__builtin_amdgcn_readfirstlane(ld_agent(tick)) - tbase) < NU_IN) { if (t0.fail(P.myws, call)) break; __builtin_amdgcn_s_sleep(32); }
    }
    DRAW(r0, tick); DRAW(r1, tick);
    int u0, u1, sl = 0;
    TAKE(u0, r0, 0); TAKE(u1, r1, 0);
    if ((unsigned)u0 < (unsigned)NU_IN) P.issue(L, P.unit_of(u0));
    WAIT_V(0);
    STAMP(P.myws, wv, 10);
    while ((unsigned)u0 < (unsigned)NU_IN) {
        unsigned r2;
        if ((unsigned)u1 < (unsigned)NU_IN) { P.issue(L + (sl ^ 1) * PL_SLOT, P.unit_of(u1)); DRAW(r2, tick); WAIT_V(7); }
        else { DRAW(r2, tick); WAIT_V(0); }
        if (pf != 0 && lane == 0) st_sys8(P.myws + pf, call);
        P.finish(L + sl * PL_SLOT, P.unit_of(u0));
        pf = P.flag_of(P.unit_of(u0));
        u0 = u1; sl ^= 1;
        TAKE(u1, r2, 8);
    }
    WAIT_V(0);
    if (pf != 0 && lane == 0) st_sys8(P.myws + pf, call);
    pf = 0;
    STAMP(P.myws, wv, 9);
    unsigned long long seen = 0; int sblk = -1;
#define MASK_ISSUE(b, u) { const int mu_ = (u) - NU_IN; \
        if (!((seen >> (mu_ & 63)) & 1) || (mu_ >> 6) != sblk) { Spin t0; sblk = mu_ >> 6;        \
            for (;;) { CFENCE; const unsigned f = __builtin_amdgcn_raw_buffer_load_b32(P.rs, lane * 4, OFF_MFLG + (P.src * 256 + sblk * 64) * 4, AUX_SYS); \
                seen = __builtin_amdgcn_ballot_w64(f >= call); \
                if ((seen >> (mu_ & 63)) & 1) break; if (t0.fail(P.myws, call)) break; } \
            CFENCE; } \
        __builtin_amdgcn_raw_ptr_buffer_load_lds(P.rp, (LAS void*)(b), 16, lane * 16, OFF_MSK + mu_ * 4096, 0, AUX_SYS); \
        __builtin_amdgcn_raw_ptr_buffer_load_lds(P.rp, (LAS void*)(b), 16, lane * 16, OFF_MSK + mu_ * 4096, 1024, AUX_SYS); \
        __builtin_amdgcn_raw_ptr_buffer_load_lds(P.rp, (LAS void*)(b), 16, lane * 16, OFF_MSK + mu_ * 4096, 2048, AUX_SYS); \
        __builtin_amdgcn_raw_ptr_buffer_load_lds(P.rp, (LAS void*)(b), 16, lane * 16, OFF_MSK + mu_ * 4096, 3072, AUX_SYS); }
    if ((unsigned)u0 < (unsigned)NU_ALL) MASK_ISSUE(L + sl * PL_SLOT, u0);
    WAIT_V(0);
    STAMP(P.myws, wv, 16);
    while ((unsigned)u0 < (unsigned)NU_ALL) {
        unsigned r2;
        if ((unsigned)u1 < (unsigned)NU_ALL) { MASK_ISSUE(L + (sl ^ 1) * PL_SLOT, u1); DRAW(r2, tick); WAIT_V(5); }
        else { DRAW(r2, tick); WAIT_V(0); }
        if (pf != 0 && lane == 0) st_sys(P.myws + pf, call);
#pragma unroll
        for (int r = 0; r < 4; ++r)
            __builtin_amdgcn_raw_buffer_store_b128(*(const LAS u32x4*)(L + sl * PL_SLOT + r * 1024 + lane * 16), P.rs, lane * 16 + r * 1024, OFF_LM + P.src * 1048576 + (u0 - NU_IN) * 4096, LOC_AUX);
        pf = OFF_MF + (P.src * 256 + u0 - NU_IN) * 4;
        u0 = u1; sl ^= 1;
        TAKE(u1, r2, 4);
    }
    WAIT_V(0);
    if (pf != 0 && lane == 0) st_sys(P.myws + pf, call);
    STAMP(P.myws, wv, 17);
}

constexpr unsigned L1_MAX = 4608, L2_MAX = 2048;
__device__ __forceinline__ unsigned off_l2(unsigned G) { return G * 8 + ((G * 2 + 3) & ~3u); }
__device__ __forceinline__ u32x2 lut_expand(unsigned byte) { u32x2 s = {0u, 0u}; unsigned c = 0;
#pragma unroll
    for (int i = 0; i < 8; ++i) { const unsigned b = (byte >> i) & 1u; s[i >> 2] |= (b ? c : 0x0cu) << (8 * (i & 3)); c += b; } return s; }
__device__ __forceinline__ unsigned lut_compact(unsigned nib) { unsigned s = 0x0c0c0c0cu, c = 0;
#pragma unroll
    for (int i = 0; i < 4; ++i) if ((nib >> i) & 1u) { s = (s & ~(0xffu << (8 * c))) | ((unsigned)i << (8 * c)); ++c; } return s; }
__device__ __forceinline__ unsigned popc4(const u32x4& m) { return __builtin_popcount(m[0]) + __builtin_popcount(m[1]) + __builtin_popcount(m[2]) + __builtin_popcount(m[3]); }
__device__ __forceinline__ unsigned nib4(unsigned m, int i) { return (i == 0 ? m << 2 : m >> (4 * i - 2)) & 0x3cu; }
__device__ __forceinline__ unsigned out_voff(int lane) { return (unsigned)(lane & 15) * N_ + (unsigned)(lane >> 4) * 16; }
__device__ __forceinline__ unsigned out_soff(int src, int wt) {
    const int tile = wt >> 3, w = wt & 7;
    return (unsigned)((tile >> 3) * 256 + (w >> 2) * 64) * N_ + (unsigned)(src * NS + (tile & 7) * 256 + (w & 3) * 64);
}
__device__ __forceinline__ constexpr unsigned chunk_soff(int ch) { return (unsigned)((ch >> 2) * 128 + (ch & 3) * 16) * N_; }

__device__ __forceinline__ void expand_role(const Params& p, LAS unsigned char* lds, LAS unsigned char* L, int ei, int lane, unsigned call, int wv) {
    const int rank = p.rank, q = ei % 7, src = (rank + 1 + q) & 7;
    char* myws = (char*)p.ws[rank];
    char* pws = (char*)p.ws[src];
    const rsrc_t rs = mk_rsrc(myws, WS_USED), ro = mk_rsrc(p.out[rank], OUT_BYTES);
#pragma unroll
    for (int i = 0; i < 4; ++i) *(LAS u32x2*)(lds + EX_LUT + (i * 64 + lane) * 8) = lut_expand(i * 64 + lane);
    char* tick = myws + OFF_TICK + (8 + q) * 128;
    const unsigned tbase = (call - 1) * (1024 + 2 * PPO), tag = call & 0x3ffffu;
    bool dead = false;
#define WT_OF(k) (WT_ORD && (unsigned)(k) < 1024u ? ((k) & ~63) | (((k) & 28) << 1) | (((k) >> 3) & 4) | ((k) & 3) : (k))
#define ENT_OFF(t) ((lane & 2) ? OFF_MF + (src * 256 + ((t) >> 2)) * 4 : OFF_DONE + (src * 1024 + (t)) * 8 + (lane & 1) * 4)
#define ENT_OK(e, x, f) (((e) >> 14) == tag && ((x) >> 22) == (tag & 0x3ffu) && (f) >= call)
#define ENT_FETCH(t) __builtin_amdgcn_raw_ptr_buffer_load_lds(rs, (LAS void*)(L + EX_E), 4, ENT_OFF(t), 0, 0, AUX_SYS)
#define EX_ISSUE(t, e, x, f, n, xo) { unsigned e_ = (e), x_ = (x), f_ = (f); \
        if (!ENT_OK(e_, x_, f_) && !dead) { Spin t0_; \
            for (;;) { CFENCE; const unsigned g_ = __builtin_amdgcn_raw_buffer_load_b32(rs, ENT_OFF(t), 0, AUX_SYS); \
                e_ = __builtin_amdgcn_readlane(g_, 0); x_ = __builtin_amdgcn_readlane(g_, 1); f_ = __builtin_amdgcn_readlane(g_, 2); \
                if (ENT_OK(e_, x_, f_)) break; if (t0_.fail(myws, call)) { dead = true; break; } __builtin_amdgcn_s_sleep(ENT_SLEEP); } \
            CFENCE; } \
        n = min(e_ & 0x3fffu, 8192u); xo = x_; \
        const unsigned by_ = min((x_ & 0x3fffu) + 15 & ~15u, 8192u); \
        const rsrc_t rc_ = mk_rsrc(pws + OFF_COD + (unsigned)(t) * 8192, by_); \
        _Pragma("nounroll") for (unsigned r_ = 0; r_ < by_; r_ += 1024) \
            __builtin_amdgcn_raw_ptr_buffer_load_lds(rc_, (LAS void*)(L + EX_B + r_), 16, lane * 16, r_, 0, AUX_SYS); \
        __builtin_amdgcn_raw_ptr_buffer_load_lds(rs, (LAS void*)(L + EX_M), 16, lane * 16, OFF_LM + src * 1048576 + (t) * 1024, 0, AUX_SYS); }
    unsigned r0, n0 = 0, x0 = 0;
    DRAW(r0, tick);
    int t0;
    TAKE(t0, r0, 0); t0 = WT_OF(t0);
    if ((unsigned)t0 < 1024u) EX_ISSUE(t0, 0u, 0u, 0u, n0, x0);
    int t1;
    { unsigned r1; DRAW(r1, tick); TAKE(t1, r1, 0); t1 = WT_OF(t1); }
    STAMP(myws, wv, 18);
    while ((unsigned)t0 < 1024u) {
        if ((unsigned)t1 < 1024u) ENT_FETCH(t1);
        const u32x4 m = *(const LAS u32x4*)(L + EX_M + lane * 16);
        if ((x0 >> 21) & 1u) {
#pragma nounroll
            for (unsigned r = 0; r < n0; r += 1024) *(LAS u32x4*)(L + r + lane * 16) = *(const LAS u32x4*)(L + EX_B + r + lane * 16);
        } else {
            const unsigned G = min((n0 + 15) >> 4, L1_MAX / 16), ec = (x0 >> 14) & 15u;
            const unsigned T1 = (ec << 3) | (((ec - 1) & 15u) << 11) | (((ec - 2) & 15u) << 19) | 0x80000000u;
            const unsigned T2 = (((ec + 1) & 15u) << 3) | (((ec - 3) & 15u) << 11) | (((ec - 4) & 15u) << 19) | 0x80000000u;
            LAS unsigned char* S2 = L + EX_B + 5888;
            unsigned cd[5][4], n7[5][4], cnt2 = 0;
            const unsigned k01 = KS_(0x01010101u), k02 = KS_(0x02020202u), k03 = KS_(0x03030303u), k07 = KS_(0x07070707u), k80 = KS_(H8), kmul = KS_(0x00408102u);
#define LAYER1(i0, n) { u32x2 nn_[n]; unsigned hb_[n]; \
                _Pragma("unroll") for (int i = 0; i < n; ++i) { const unsigned g = (i0 + i) * 64 + lane; nn_[i] = *(const LAS u32x2*)(L + EX_B + g * 8); hb_[i] = *(const LAS unsigned short*)(L + EX_B + G * 8 + g * 2); } \
                SCHED; \
                _Pragma("unroll") for (int i = 0; i < n; ++i) { const unsigned hb = (i0 + i) * 64 + lane < G ? hb_[i] : 0u;        \
                    _Pragma("unroll") for (int k = 0; k < 4; ++k) { const unsigned nb = (k & 1) ? nn_[i][k >> 1] >> 4 : nn_[i][k >> 1]; \
                        const unsigned ex = __builtin_amdgcn_perm(0u, T1, ((nb >> 3) & k01) | (__builtin_amdgcn_ubfe(hb, 4 * k, 4) * kmul & k02)); \
                        cd[i0 + i][k] = (nb & k07) | ex; \
                        n7[i0 + i][k] = __builtin_amdgcn_udot4(ex & k80, 0x08040201u, 0u, false); \
                        cnt2 += __builtin_popcount(n7[i0 + i][k]); } } }
            LAYER1(0, 4);
#pragma unroll
            for (int k = 0; k < 4; ++k) cd[4][k] = n7[4][k] = 0;
            if (G > 256) LAYER1(4, 1);
            CYC(0);
            const unsigned sc2 = wave_scan(cnt2), n2 = min(__builtin_amdgcn_readlane(sc2, 63), L2_MAX), G2 = (n2 + 15) >> 4, o2 = off_l2(G), o3 = o2 + G2 * 4;
            {
                unsigned pk0 = *(const LAS unsigned*)(L + EX_B + o2 + lane * 4), pk1 = *(const LAS unsigned*)(L + EX_B + o2 + 256 + lane * 4);
                pk0 = lane < G2 ? pk0 : 0u; pk1 = 64 + lane < G2 ? pk1 : 0u;
                const unsigned x3 = *(const LAS unsigned*)(L + EX_B + o3 + lane * 4);
                *(LAS u32x2*)(L + EX_P + lane * 8) = (u32x2){(x3 << 3) & 0x78787878u, (x3 >> 1) & 0x78787878u};
                unsigned v2[8], m7[8], cnt3 = 0;
#pragma unroll
                for (int k = 0; k < 8; ++k) {
                    v2[k] = __builtin_amdgcn_perm(0u, T2, ((k < 4 ? pk0 : pk1) >> (2 * (k & 3))) & k03);
                    m7[k] = __builtin_amdgcn_udot4(v2[k] & k80, 0x08040201u, 0u, false);
                    cnt3 += __builtin_popcount(m7[k]);
                }
                unsigned o = wave_scan(cnt3) - cnt3 + (unsigned)(size_t)(L + EX_P), se[8], w0[8], w1[8], of[8];
                CFENCE;
#pragma unroll
                for (int k = 0; k < 8; ++k) {
                    se[k] = *(const LAS unsigned*)(lds + EX_LUT + (m7[k] >> 4));
                    const LAS unsigned* w = (const LAS unsigned*)(size_t)(o & ~3u); w0[k] = w[0]; w1[k] = w[1]; of[k] = o; o += __builtin_popcount(m7[k]);
                }
                SCHED;
#pragma unroll
                for (int k = 0; k < 8; ++k) v2[k] = (v2[k] & ~H8) | __builtin_amdgcn_perm(0u, __builtin_amdgcn_alignbyte(w1[k], w0[k], of[k]), se[k]);
                *(LAS u32x4*)(S2 + lane * 16) = (u32x4){v2[0], v2[1], v2[2], v2[3]};
                *(LAS u32x4*)(S2 + 1024 + lane * 16) = (u32x4){v2[4], v2[5], v2[6], v2[7]};
                CFENCE;
            }
            unsigned o = sc2 - cnt2 + (unsigned)(size_t)S2;
#define MERGE(i0, n) { unsigned se_[n][4], w0_[n][4], w1_[n][4], of_[n][4]; \
                _Pragma("unroll") for (int i = 0; i < n; ++i) _Pragma("unroll") for (int k = 0; k < 4; ++k) { \
                    se_[i][k] = *(const LAS unsigned*)(lds + EX_LUT + (n7[i0 + i][k] >> 4)); \
                    const LAS unsigned* w = (const LAS unsigned*)(size_t)(o & ~3u); w0_[i][k] = w[0]; w1_[i][k] = w[1]; of_[i][k] = o; o += __builtin_popcount(n7[i0 + i][k]); } \
                SCHED; \
                _Pragma("unroll") for (int i = 0; i < n; ++i) { u32x4 v; \
                    _Pragma("unroll") for (int k = 0; k < 4; ++k) v[k] = (cd[i0 + i][k] & ~H8) | __builtin_amdgcn_perm(0u, __builtin_amdgcn_alignbyte(w1_[i][k], w0_[i][k], of_[i][k]), se_[i][k]); \
                    *(LAS u32x4*)(L + ((i0 + i) * 64 + lane) * 16) = v; } }
            MERGE(0, 4);
            if (G > 256) MERGE(4, 1);
            CFENCE;
        }
        const unsigned cnt = popc4(m);
        unsigned off = wave_scan(cnt) - cnt + (unsigned)(size_t)L;
        WAIT_L(0);
        CYC(1);
        unsigned n1 = 0, x1 = 0;
        if ((unsigned)t1 < 1024u) {
            WAIT_V(0);
            const u32x4 ent = *(const LAS u32x4*)(L + EX_E);
            EX_ISSUE(t1, __builtin_amdgcn_readfirstlane(ent[0]), __builtin_amdgcn_readfirstlane(ent[1]), __builtin_amdgcn_readfirstlane(ent[2]), n1, x1);
        }
        unsigned r2;
        DRAW(r2, tick);
        CYC(2);
        const unsigned so = out_soff(src, t0), vo = out_voff(lane);
        u32x2 sel[16]; unsigned of[16], w0[16], w1[16], w2[16];
#pragma unroll
        for (int g = 0; g < 16; ++g) {
            const unsigned b8 = ((g & 3) == 0 ? m[g >> 2] << 3 : m[g >> 2] >> (8 * (g & 3) - 3)) & 0x7f8u;
            sel[g] = *(const LAS u32x2*)(lds + EX_LUT + b8);
            const LAS unsigned* w = (const LAS unsigned*)(size_t)(off & ~3u);
            w0[g] = w[0]; w1[g] = w[1]; w2[g] = w[2]; of[g] = off;
            off += __builtin_popcount(b8);
        }
        SCHED;
        u32x4 o[8];
#pragma unroll
        for (int g = 0; g < 16; ++g) {
            const unsigned a0 = __builtin_amdgcn_alignbyte(w1[g], w0[g], of[g]), a1 = __builtin_amdgcn_alignbyte(w2[g], w1[g], of[g]);
            o[g >> 1][(g & 1) * 2] = __builtin_amdgcn_perm(a1, a0, sel[g][0]); o[g >> 1][(g & 1) * 2 + 1] = __builtin_amdgcn_perm(a1, a0, sel[g][1]);
        }
        CYC(3);
        if (dead) {
#pragma unroll
            for (int ch = 0; ch < 8; ++ch) o[ch] = (u32x4){0x7f7f7f7fu, 0x7f7f7f7fu, 0x7f7f7f7fu, 0x7f7f7f7fu};
            CFENCE;
        }
#pragma unroll
        for (int ch = 0; ch < 8; ++ch) __builtin_amdgcn_raw_buffer_store_b128(o[ch], ro, vo, so + chunk_soff(ch), OUT_AUX);
        CYC(4);
        int t2;
        TAKE(t2, r2, 8); t2 = WT_OF(t2);
        CYC(5);
        t0 = t1; n0 = n1; x0 = x1; t1 = t2;
    }
    STAMP(myws, wv, 19);
}

__device__ __forceinline__ unsigned get_call(unsigned callv) { unsigned c; asm volatile("v_readfirstlane_b32 %0, %1" : "=s"(c) : "v"(callv)); return c + 1u; }

__device__ __forceinline__ void comm_role(const Params& p, LAS unsigned char* lds, int c, int wid, int lane, unsigned callv) {
    const int rank = p.rank;
    char* myws = (char*)p.ws[rank];
    const rsrc_t rs = mk_rsrc(myws, WS_USED);
    const int wv = (NGEMM + c) * 8 + wid; (void)wv;
    STAMP(myws, wv, 0);
    if (wid >= 2) {
        const rsrc_t rx = mk_rsrc(p.x, X_BYTES), rw = mk_rsrc(p.w, W_BYTES);
        const int k0 = (wid - 2) * 128 + c;
        const bool far = k0 < 448;
        const unsigned long long tk0 = __builtin_amdgcn_s_memrealtime();
        for (int u = far ? k0 : k0 - 448; u < (far ? 1792 : 768); u += far ? 448 : 320) {
            const bool ownw = !far && u >= 64 && u < 320;
            const int ux = u < 64 ? u : u - 256;
            const int j = far ? u / 224 : ownw ? (u - 64) >> 5 : ux >> 6, r = far ? u % 224 : ownw ? 288 + ((u - 64) & 31) : 224 + (ux & 63), t = rank * 8 + j;
            if (far && j == 0 && r / 7 < NPUSH) continue;
            const unsigned due = far ? (j > 1 ? j * PACE_W - PACE_W0 : j * PACE_W1) : ownw ? PACE_O0 + j * PACE_O : j * PACE_X;
            while ((unsigned)(__builtin_amdgcn_s_memrealtime() - tk0) < due) __builtin_amdgcn_s_sleep(8);
            u32x4 v[8];
            if (r >= 224 && r < 288) {
                const int g = r - 224;
                x_load(v, rx, j, g, lane);
                pack_store(v, rs, lds + 2048 + wid * EX_BYTES, OFF_PX + (unsigned)(j * 64 + g) * UNIT, lane);
                WAIT_V(0);
                const unsigned call = get_call(callv);
                if (j < NPUF) {
                    if (lane < 8) st_sys((char*)p.ws[lane] + OFF_PUF + ((rank * NPUF + j) * 96 + 32 + g) * 4, call);
                    if (j == 0) STAMP(myws, wv, 7);
                }
                unsigned old = 0;
                if (lane == 0) old = __hip_atomic_fetch_add((GAS unsigned*)(myws + OFF_PCNT + j * 128), 1u, __ATOMIC_RELAXED, __HIP_MEMORY_SCOPE_AGENT);
                if ((__builtin_amdgcn_readfirstlane(old) & 63) == 63 && lane < 8) st_sys((char*)p.ws[lane] + OFF_PFLG + t * 128, call);
            } else {
                const bool own = r >= 288;
                const int e = own ? r - 288 : r / 7, d = own ? rank : (rank + 1 + r % 7) & 7;
                u32x2 w[16];
                w_load(w, rw, j, d, e >> 2, e & 1, (e >> 1) & 1, lane);
                w_transpose(v, w);
                if (own) {
                    img_store(v, rs, true, w_img(t, e), lane);
                    WAIT_V(0);
                    if (lane == 0) st_sys8(myws + w_flag(t, e), get_call(callv));
                } else {
                    const unsigned id = (unsigned)((d * 8 + j) * 32 + e);
                    pack_store(v, rs, lds + 2048 + wid * EX_BYTES, OFF_PW + id * UNIT, lane);
                    WAIT_V(0);
                    const unsigned call = get_call(callv);
                    if (j < NPUF) { if (lane == 0) st_sys((char*)p.ws[d] + OFF_PUF + ((rank * NPUF + j) * 96 + e) * 4, call); if (j == 0) STAMP(myws, wv, 6); }
                    unsigned old = 0;
                    if (lane == 0) old = __hip_atomic_fetch_add((GAS unsigned*)(myws + OFF_PCNT + (8 + d * 8 + j) * 128), 1u, __ATOMIC_RELAXED, __HIP_MEMORY_SCOPE_AGENT);
                    if (__builtin_amdgcn_readfirstlane(old) + 1 == call * (j ? 32 : 32 - NPUSH) && lane == 0) st_sys((char*)p.ws[d] + OFF_PFLG + t * 128 + 4, call);
                }
            }
            STAMP(myws, wv, far ? 1 + (j >> 2) : 3 + (j >> 2));
        }
    }
    if (wid < CW) {
        const unsigned call = get_call(callv);
        const int ci = wid * 128 + c;
        if (ci < 7 * PPM) pull_role(p, lds + 2048 + wid * EX_BYTES, ci, lane, call, wv);
        if (ci < 7 * PPO) expand_role(p, lds, lds + 2048 + wid * EX_BYTES, ci, lane, call, wv);
    }
}

typedef float f32x2 __attribute__((ext_vector_type(2)));
__device__ __forceinline__ f32x4 scale4(const f32x4& v, const f32x2& s) {
    f32x2 a, b;
    asm("v_pk_mul_f32 %0, %1, %2" : "=v"(a) : "v"((f32x2){v[0], v[1]}), "v"(s));
    asm("v_pk_mul_f32 %0, %1, %2" : "=v"(b) : "v"((f32x2){v[2], v[3]}), "v"(s));
    return (f32x4){a[0], a[1], b[0], b[1]};
}
__device__ __forceinline__ void gemm_role(const Params& p, LAS unsigned char* lds, int rt, int ct, int wid, int lane, unsigned callv) {
    const int rank = p.rank;
    char* myws = (char*)p.ws[rank];
    const int tile = rt * 8 + ct;
    STAMP(myws, tile * 8 + wid, 0);
    {
        const int pw = ((rt & 3) * 4 + (ct & 3)) * 8 + (rt >> 2) * 2 + (ct >> 2);
        if (NPUSH && wid == 7 && pw < 7 * PUSH_W) {
            const int d = (rank + 1 + pw % 7) & 7, e0 = pw / 7;
            const rsrc_t rw = mk_rsrc(p.w, W_BYTES), rd = mk_rsrc(p.ws[d], WS_USED);
            u32x2 w[PUSH_U ? PUSH_U : 1][16];
#pragma unroll
            for (int i = 0; i < PUSH_U; ++i) { const int e = e0 + i * PUSH_W; w_load(w[i], rw, 0, d, e >> 2, e & 1, (e >> 1) & 1, lane); }
#pragma unroll
            for (int i = 0; i < PUSH_U; ++i) {
                const int e = e0 + i * PUSH_W;
                u32x4 v[8];
                w_transpose(v, w[i]);
                img_store(v, rd, true, w_img(rank * 8, e), lane);
                WAIT_V(0);
                if (lane == 0) st_sys8((char*)p.ws[d] + w_flag(rank * 8, e), get_call(callv));
            }
            STAMP(myws, tile * 8 + wid, 14);
        }
    }
    if (wid >= 1 && wid <= 4) {
        const int j = tile >> 4, g = (tile & 15) * 4 + wid - 1;
        const unsigned long long tk0 = __builtin_amdgcn_s_memrealtime();
        while ((unsigned)(__builtin_amdgcn_s_memrealtime() - tk0) < PACE_O0 + j * PACE_O) __builtin_amdgcn_s_sleep(4);
        u32x4 v[8];
        x_load(v, mk_rsrc(p.x, X_BYTES), j, g, lane);
        img_store(v, mk_rsrc(myws, WS_USED), false, x_img(rank * 8 + j, g), lane);
        WAIT_V(0);
        if (lane == 0) st_sys8(myws + x_flag(rank * 8 + j, g), get_call(callv));
    }
    const unsigned call = get_call(callv);
    const int wv = tile * 8 + wid; (void)wv;
    const int wr = wid >> 2, wc = wid & 3;
    const rsrc_t rs = mk_rsrc(myws, WS_USED);
    const unsigned voff = (unsigned)wid * 2048 + (unsigned)lane * 16;
    LAS unsigned char* ldsw = lds + wid * 2048;
    LAS unsigned char* lda = lds + wr * 8192 + lane * 16;
    LAS unsigned char* ldb = lds + wc * 4096 + lane * 16;
    const unsigned seqf = (unsigned)step_of(lane, rank, rt) * 4, fx = OFF_FLAG + rt * 256, fw = OFF_FLAG + FLAG_W + ct * 256, c4 = (call & 255u) * 0x01010101u;
    unsigned long long todo = ~0ull, rdy = 0;
    unsigned tl = 0;
    volatile LAS unsigned* tq = (volatile LAS unsigned*)(lds + LDS_RED + 160);
#define SA(b, h) (((b) * 2 + (h)) * HALFB)
#define SB(b, h) ((4 + (b) * 2 + (h)) * HALFB)
#define STAGE(bufoff, soff) do { \
        __builtin_amdgcn_raw_ptr_buffer_load_lds(rs, (LAS void*)(ldsw + (bufoff)), 16, voff, (soff), 0, GEMM_AUX); \
        __builtin_amdgcn_raw_ptr_buffer_load_lds(rs, (LAS void*)(ldsw + (bufoff)), 16, voff, (soff), 1024, GEMM_AUX); } while (0)
#define LDA(dst, b, h) do { _Pragma("unroll") for (int m = 0; m < 4; ++m) _Pragma("unroll") for (int k = 0; k < 2; ++k) dst[m][k] = *(const LAS bf16x8*)(lda + SA(b, h) + m * 2048 + k * 1024); } while (0)
#define LDB(dst, b, h) do { _Pragma("unroll") for (int n = 0; n < 2; ++n) _Pragma("unroll") for (int k = 0; k < 2; ++k) dst[n][k] = *(const LAS bf16x8*)(ldb + SB(b, h) + n * 2048 + k * 1024); } while (0)
#define MMA(ai, bj, At, Bt) do { __builtin_amdgcn_s_setprio(1); _Pragma("unroll") for (int m = 0; m < 4; ++m) _Pragma("unroll") for (int n = 0; n < 2; ++n) _Pragma("unroll") for (int k = 0; k < 2; ++k) \
        acc[ai][bj][m][n] = __builtin_amdgcn_mfma_f32_16x16x32_bf16(Bt[n][k], At[m][k], acc[ai][bj][m][n], 0, 0, 0); __builtin_amdgcn_s_setprio(0); } while (0)
#define OFFA(t) (OFF_XT + (unsigned)(t) * XT_STEP + (unsigned)rt * BLK)
#define OFFB(t) (OFF_WT + (unsigned)(t) * WT_STEP + (unsigned)ct * BLK)
    unsigned la = 0, lb = 0;
#define LOOK do { if (wid == 0) { WAIT_V(8); la = *(const LAS unsigned*)(lds + LDS_FLAG + lane * 4); lb = *(const LAS unsigned*)(lds + LDS_FLAG + 256 + lane * 4); } } while (0)
#define PICK(slot) do { if (wid == 0) { \
            if (todo != 0) { \
                rdy |= __builtin_amdgcn_ballot_w64(la == c4 && lb == c4); \
                if ((rdy & todo) == 0) { Spin t0_; \
                    for (;;) { CFENCE; \
                        const unsigned a_ = __builtin_amdgcn_raw_buffer_load_b32(rs, seqf, fx, AUX_SYS), b_ = __builtin_amdgcn_raw_buffer_load_b32(rs, seqf, fw, AUX_SYS); \
                        rdy |= __builtin_amdgcn_ballot_w64(a_ == c4 && b_ == c4); \
                        if ((rdy & todo) != 0) break; if (t0_.fail(myws, call)) { rdy = ~0ull; break; } } } \
                const int pos_ = __builtin_ctzll(rdy & todo); \
                todo &= ~(1ull << pos_); tl = __builtin_amdgcn_readlane(seqf, pos_) >> 2; \
            } \
            tq[slot] = tl; \
            __builtin_amdgcn_raw_ptr_buffer_load_lds(rs, (LAS void*)(lds + LDS_FLAG), 4, seqf, fx, 0, AUX_SYS); \
            __builtin_amdgcn_raw_ptr_buffer_load_lds(rs, (LAS void*)(lds + LDS_FLAG + 256), 4, seqf, fw, 0, AUX_SYS); \
            SCHED; } } while (0)
#define WAIT_STAGE do { if (wid == 0) { WAIT_V(8); WAIT_L(0); } else WAIT_V(6); } while (0)

    f32x4 acc[2][2][4][2];
#pragma unroll
    for (int a = 0; a < 2; ++a)
#pragma unroll
        for (int b = 0; b < 2; ++b)
#pragma unroll
            for (int m = 0; m < 4; ++m)
#pragma unroll
                for (int n = 0; n < 2; ++n) acc[a][b][m][n] = (f32x4){0.f, 0.f, 0.f, 0.f};
    bf16x8 At[4][2], B0[2][2], B1[2][2];

    unsigned t1;
    {
        la = ~c4; PICK(0); PICK(1); PICK(2);
        if (wid == 1 && lane < 8) ((LAS unsigned*)(lds + LDS_RED))[32 + lane] = 0u;
        WAIT_L(0); BAR; CFENCE;
        STAMP(myws, wv, 1);
        const unsigned t0 = __builtin_amdgcn_readfirstlane(tq[0]);
        t1 = __builtin_amdgcn_readfirstlane(tq[1]);
        STAGE(SB(0, 0), OFFB(t0)); STAGE(SA(0, 0), OFFA(t0)); STAGE(SB(0, 1), OFFB(t0) + HALFB); STAGE(SA(0, 1), OFFA(t0) + HALFB);
        if (wr == 1) BAR;
        WAIT_V(4); BAR;
        STAGE(SB(1, 0), OFFB(t1)); STAGE(SA(1, 0), OFFA(t1)); STAGE(SB(1, 1), OFFB(t1) + HALFB);
        WAIT_V(6); BAR;
    }
    for (int i = 0; i < NT; i += 2) {
        LDB(B0, 0, 0); SCHED; LDA(At, 0, 0); STAGE(SA(1, 1), OFFA(t1) + HALFB);
        WAIT_L(8); BAR; WAIT_L(0);
        const unsigned tv2 = tq[2];
        MMA(0, 0, At, B0); BAR; SCHED;
        const unsigned t2 = __builtin_amdgcn_readfirstlane(tv2), a2 = OFFA(t2), b2 = OFFB(t2);
        LDB(B1, 0, 1); STAGE(SB(0, 0), b2);
        BAR; WAIT_L(0); MMA(0, 1, At, B1); BAR;
        LDA(At, 0, 1); STAGE(SA(0, 0), a2); LOOK;
        BAR; WAIT_L(0); MMA(1, 0, At, B0); BAR; SCHED;
        PICK(3);
        STAGE(SB(0, 1), b2 + HALFB);
        WAIT_STAGE; BAR; MMA(1, 1, At, B1); BAR;
        LDB(B0, 1, 0); SCHED; LDA(At, 1, 0); STAGE(SA(0, 1), a2 + HALFB);
        WAIT_L(8); BAR; WAIT_L(0);
        const unsigned tv3 = tq[3];
        MMA(0, 0, At, B0); BAR; SCHED;
        t1 = __builtin_amdgcn_readfirstlane(tv3);
        const unsigned a3 = OFFA(t1), b3 = OFFB(t1);
        LDB(B1, 1, 1); STAGE(SB(1, 0), b3);
        BAR; WAIT_L(0); MMA(0, 1, At, B1); BAR;
        LDA(At, 1, 1); STAGE(SA(1, 0), a3); LOOK;
        BAR; WAIT_L(0); MMA(1, 0, At, B0); BAR; SCHED;
        PICK(2);
        STAGE(SB(1, 1), b3 + HALFB);
        WAIT_STAGE; BAR; MMA(1, 1, At, B1); BAR;
    }
    char* pl = (char*)p.ws[lane & 7];
    if (wr == 0) BAR;
    STAMP(myws, wv, 10);

    const int wt = tile * 8 + wid;
    LAS unsigned* red = (LAS unsigned*)(lds + LDS_RED);
    u32x4 msk;
#pragma unroll
    for (int w = 0; w < 4; ++w) {
        unsigned mm = 0;
#pragma unroll
        for (int e = 31; e >= 0; --e) {
            const int ch = 2 * w + (e >> 4);
            mm = __builtin_amdgcn_alignbit(mm, __float_as_uint(acc[ch >> 2][(e >> 1) & 1][ch & 3][e & 1][(e >> 2) & 3]), 31);
        }
        msk[w] = ~mm;
    }
    __builtin_amdgcn_raw_buffer_store_b128(msk, rs, lane * 16, OFF_MSK + wt * 1024, AUX_SYS);
    SCHED;
    float mx = 0.f;
#pragma unroll
    for (int a = 0; a < 2; ++a)
#pragma unroll
        for (int b = 0; b < 2; ++b)
#pragma unroll
            for (int m = 0; m < 4; ++m)
#pragma unroll
                for (int n = 0; n < 2; ++n) mx = fmaxf(fmaxf(fmaxf(fmaxf(mx, acc[a][b][m][n][0]), acc[a][b][m][n][1]), acc[a][b][m][n][2]), acc[a][b][m][n][3]);
    const unsigned wm = wave_max(__float_as_uint(mx));
    {
        unsigned old = 0;
        if (lane == 0) {
            __hip_atomic_fetch_max(red + 32, wm, __ATOMIC_RELAXED, __HIP_MEMORY_SCOPE_WORKGROUP);
            old = __hip_atomic_fetch_add(red + 33, 1u, __ATOMIC_RELAXED, __HIP_MEMORY_SCOPE_WORKGROUP);
        }
        if (__builtin_amdgcn_readfirstlane(old) == 7) {
            const unsigned long long gv = ((unsigned long long)call << 32) | __hip_atomic_load(red + 32, __ATOMIC_RELAXED, __HIP_MEMORY_SCOPE_WORKGROUP);
            char* g = pl + OFF_AMAX + (rank * NGEMM + tile) * 8;
            unsigned long long ex_;
            asm volatile("s_mov_b64 %0, exec\n\ts_mov_b64 exec, 0xff\n\tglobal_store_dwordx2 %1, %2, off sc0 sc1\n\ts_mov_b64 exec, %0\n\ts_waitcnt vmcnt(1)" : "=&s"(ex_) : "v"(g), "v"(gv) : "memory");
        } else WAIT_V(0);
        unsigned old2 = 0;
        if (lane == 0) old2 = __hip_atomic_fetch_add(red + 34 + wr, 1u, __ATOMIC_RELAXED, __HIP_MEMORY_SCOPE_WORKGROUP);
        if (__builtin_amdgcn_readfirstlane(old2) == 3 && lane < 8) st_sys(pl + OFF_MFLG + (rank * 256 + tile * 2 + wr) * 4, call);
    }
#pragma unroll
    for (int a = 0; a < 2; ++a)
#pragma unroll
        for (int b = 0; b < 2; ++b)
#pragma unroll
            for (int m = 0; m < 4; ++m)
#pragma unroll
                for (int n = 0; n < 2; ++n)
#pragma unroll
                    for (int j = 0; j < 4; ++j) { acc[a][b][m][n][j] = fmaxf(acc[a][b][m][n][j], 0.f); if (j == 3) asm volatile("" : "+v"(acc[a][b][m][n])); }
    STAMP(myws, wv, 13);
    BAR; CFENCE;
    LAS unsigned char* A = lds + wid * EP_BYTES;
    LAS unsigned char* B = A + EP_C;
    LAS unsigned char* S2 = A + EP_S;
    const unsigned cnt = popc4(msk), scan = wave_scan(cnt), Lc = __builtin_amdgcn_readlane(scan, 63);
    const bool dense = Lc > L1_MAX;
    if (!dense) {
        unsigned o = (unsigned)(size_t)A + (scan - cnt) * 4;
        const unsigned spare = (unsigned)(size_t)(A + EP_X) + lane * 4;
#pragma unroll
        for (int e = 0; e < 128; ++e) {
            const int ch = e >> 4;
            const unsigned bit = __builtin_amdgcn_ubfe(msk[e >> 5], e & 31, 1);
            *(LAS float*)(size_t)(bit ? o ^ ((o >> 4) & 0x30u) : spare) = acc[ch >> 2][(e >> 1) & 1][ch & 3][e & 1][(e >> 2) & 3];
            o += bit * 4;
        }
    }
    float ysum = 0.f;
#pragma unroll
    for (int a = 0; a < 2; ++a)
#pragma unroll
        for (int b = 0; b < 2; ++b)
#pragma unroll
            for (int m = 0; m < 4; ++m) { const f32x4 t = acc[a][b][m][0] + acc[a][b][m][1]; ysum += (t[0] + t[1]) + (t[2] + t[3]); }
#pragma unroll
    for (int sh = 32; sh >= 1; sh >>= 1) ysum += __shfl_xor(ysum, sh);
    ysum *= 1.35f / (float)max(Lc, 1u);
    if (!dense && lane < 16 && Lc + lane < L1_MAX) *(LAS float*)(A + (((Lc + lane) * 4) ^ (((Lc + lane) >> 2) & 0x30u))) = ysum;
    LAS unsigned char* lut = A + EP_T;
    if (lane < 16) *(LAS unsigned*)(lut + lane * 4) = lut_compact(lane);
    {
        Spin t0;
        unsigned t, tm = ld_agent(myws + OFF_TMO);
        for (;;) {
            CFENCE;
            const u32x4 g = __builtin_amdgcn_raw_buffer_load_b128(rs, lane * 16, OFF_AMAX + wid * 1024, AUX_SYS);
            t = max(g[0], g[2]);
            if (__builtin_amdgcn_ballot_w64(g[1] < call || g[3] < call) == 0) break;
            if (t0.fail(myws, call)) { tm = call; break; }
            __builtin_amdgcn_s_sleep(AMAX_SLEEP);
        }
        t = wave_max(t);
        if (lane == 0) { red[8 + wid] = t; red[wid] = tm == call; }
    }
    WAIT_L(0); BAR; CFENCE;
    const float amax = __uint_as_float(wave_max(red[8 + (lane & 7)]));
    const bool dead = __builtin_amdgcn_ballot_w64(red[lane & 7] != 0) != 0;
    const float inv = dead ? __builtin_nanf("") : 448.0f / amax;
    STAMP(myws, wv, 11);
    if (EPI_PRIO == 1 && wr == 0) __builtin_amdgcn_s_setprio(2);
    if (EPI_PRIO == 2 && wr == 1) while (__builtin_amdgcn_readfirstlane(((volatile LAS unsigned*)red)[38]) < 4) __builtin_amdgcn_s_sleep(2);

    const f32x2 inv2 = {inv, inv};
#define QUANT(ch, q) { const int a = (ch) >> 2, m = (ch) & 3; \
        const f32x4 y0 = scale4(acc[a][0][m][0], inv2), y1 = scale4(acc[a][0][m][1], inv2), y2 = scale4(acc[a][1][m][0], inv2), y3 = scale4(acc[a][1][m][1], inv2); \
        _Pragma("unroll") for (int j = 0; j < 4; ++j) \
            q[j] = (unsigned)__builtin_amdgcn_cvt_pk_fp8_f32(y2[j], y3[j], __builtin_amdgcn_cvt_pk_fp8_f32(y0[j], y1[j], __float_as_int(y0[j]), false), true); }
    u32x4 cod[5];
    if (dense) {
#pragma unroll
        for (int r = 0; r < 9; ++r) *(LAS u32x4*)(A + r * 1024 + lane * 16) = (u32x4){0u, 0u, 0u, 0u};
        unsigned o = (unsigned)(size_t)A + scan - cnt;
#pragma unroll
        for (int ch = 0; ch < 8; ++ch) {
            u32x4 q;
            QUANT(ch, q);
#pragma unroll
            for (int j = 0; j < 4; ++j) {
                const unsigned n4 = nib4(msk[ch >> 1], (ch & 1) * 4 + j);
                const unsigned long long z = (unsigned long long)__builtin_amdgcn_perm(0u, q[j], *(const LAS unsigned*)(lut + n4)) << ((o & 3u) * 8);
                LAS unsigned* d = (LAS unsigned*)(size_t)(o & ~3u);
                __hip_atomic_fetch_or(d, (unsigned)z, __ATOMIC_RELAXED, __HIP_MEMORY_SCOPE_WAVEFRONT);
                __hip_atomic_fetch_or(d + 1, (unsigned)(z >> 32), __ATOMIC_RELAXED, __HIP_MEMORY_SCOPE_WAVEFRONT);
                o += __builtin_popcount(n4);
            }
        }
    } else {
#define CODES(i0, n) { f32x4 y_[n][4]; \
            _Pragma("unroll") for (int i = 0; i < n; ++i) _Pragma("unroll") for (int k = 0; k < 4; ++k) y_[i][k] = *(const LAS f32x4*)(A + ((i0 + i) * 64 + (i0 + i < 4 ? lane : lane & 31)) * 64 + ((k ^ (lane >> 2)) & 3) * 16); \
            SCHED; \
            _Pragma("unroll") for (int i = 0; i < n; ++i) _Pragma("unroll") for (int k = 0; k < 4; ++k) { const f32x4 y = scale4(y_[i][k], inv2); \
                cod[i0 + i][k] = (unsigned)__builtin_amdgcn_cvt_pk_fp8_f32(y[2], y[3], __builtin_amdgcn_cvt_pk_fp8_f32(y[0], y[1], __float_as_int(y[0]), false), true); } }
        CODES(0, 2); CODES(2, 2);
        cod[4] = (u32x4){0u, 0u, 0u, 0u};
        if (Lc > 4096) CODES(4, 1);
        CFENCE;
#pragma unroll
        for (int i = 0; i < 5; ++i) *(LAS u32x4*)(A + (i * 64 + lane) * 16) = cod[i];
#pragma unroll
        for (int r = 0; r < 3; ++r) *(LAS u32x4*)(S2 + r * 1024 + lane * 16) = (u32x4){0u, 0u, 0u, 0u};
    }
    CFENCE;
    CYC(0);
    const unsigned ec = (__builtin_amdgcn_readfirstlane(__builtin_amdgcn_cvt_pk_fp8_f32(fminf(ysum * inv, 448.f), 0.f, 0, false)) >> 3) & 15u;
    const unsigned E1 = (ec + 17) * 0x01010101u, G = (Lc + 15) >> 4;
    bool raw = dense;
    unsigned bytes = Lc;
    if (!raw) {
        unsigned s2[5][4], n7[5][4], cnt2 = 0;
        const unsigned k07 = KS_(0x07070707u), k08 = KS_(0x08080808u), k0f = KS_(0x0f0f0f0fu), k10 = KS_(0x10101010u), k80 = KS_(H8);
#define CLASSIFY(i0, n) { const u32x4* d_ = cod + i0; \
            _Pragma("unroll") for (int i = 0; i < n; ++i) { const unsigned g = (i0 + i) * 64 + lane, wgt = g < G ? 0x08040201u : 0u; unsigned nb[4], p_[4]; \
                _Pragma("unroll") for (int k = 0; k < 4; ++k) { \
                    const unsigned e4 = (d_[i][k] >> 3) & k0f, u = E1 - e4, t = u & k08, ui = (u | (t - (t >> 3))) & k07; \
                    p_[k] = __builtin_amdgcn_perm(0x98989898u, 0x10080098u, ui); \
                    s2[i0 + i][k] = e4 | __builtin_amdgcn_perm(0x30302010u, 0x30303000u, ui); \
                    nb[k] = (d_[i][k] & k07) | (p_[k] & k08); \
                    n7[i0 + i][k] = __builtin_amdgcn_udot4(p_[k] & k80, wgt, 0u, false); \
                    cnt2 += __builtin_popcount(n7[i0 + i][k]); } \
                if (g < G) { *(LAS u32x2*)(B + g * 8) = (u32x2){nb[0] | (nb[1] << 4), nb[2] | (nb[3] << 4)}; \
                    *(LAS unsigned short*)(B + G * 8 + g * 2) = (unsigned short)bits16(p_[0] & k10, p_[1] & k10, p_[2] & k10, p_[3] & k10, 4); } } }
        CLASSIFY(0, 4);
#pragma unroll
        for (int k = 0; k < 4; ++k) s2[4][k] = n7[4][k] = 0;
        if (G > 256) CLASSIFY(4, 1);
        CYC(1);
        const unsigned sc2 = wave_scan(cnt2), n2 = __builtin_amdgcn_readlane(sc2, 63), G2 = (n2 + 15) >> 4, o2 = off_l2(G), o3 = o2 + G2 * 4;
        raw = n2 > L2_MAX;
        if (!raw) {
            unsigned o = sc2 - cnt2 + (unsigned)(size_t)S2;
#define SQUEEZE(i0, n) { unsigned se_[n][4]; \
                _Pragma("unroll") for (int i = 0; i < n; ++i) _Pragma("unroll") for (int k = 0; k < 4; ++k) se_[i][k] = *(const LAS unsigned*)(lut + (n7[i0 + i][k] >> 5)); \
                SCHED; \
                _Pragma("unroll") for (int i = 0; i < n; ++i) _Pragma("unroll") for (int k = 0; k < 4; ++k) { \
                    const unsigned long long z = (unsigned long long)__builtin_amdgcn_perm(0u, s2[i0 + i][k], se_[i][k]) << ((o & 3u) * 8); \
                    LAS unsigned* d = (LAS unsigned*)(size_t)(o & ~3u); \
                    __hip_atomic_fetch_or(d, (unsigned)z, __ATOMIC_RELAXED, __HIP_MEMORY_SCOPE_WAVEFRONT); \
                    __hip_atomic_fetch_or(d + 1, (unsigned)(z >> 32), __ATOMIC_RELAXED, __HIP_MEMORY_SCOPE_WAVEFRONT); \
                    o += __builtin_popcount(n7[i0 + i][k]); } }
            SQUEEZE(0, 4);
            if (G > 256) SQUEEZE(4, 1);
            CFENCE;
            CYC(2);
            const u32x4 d0 = *(const LAS u32x4*)(S2 + lane * 16), d1 = *(const LAS u32x4*)(S2 + 1024 + lane * 16);
            if (lane < G2) *(LAS unsigned*)(B + o2 + lane * 4) = ((d0[0] >> 4) & 0x03030303u) | ((d0[1] >> 2) & 0x0c0c0c0cu) | (d0[2] & 0x30303030u) | ((d0[3] << 2) & 0xc0c0c0c0u);
            if (64 + lane < G2) *(LAS unsigned*)(B + o2 + 256 + lane * 4) = ((d1[0] >> 4) & 0x03030303u) | ((d1[1] >> 2) & 0x0c0c0c0cu) | (d1[2] & 0x30303030u) | ((d1[3] << 2) & 0xc0c0c0c0u);
            unsigned m5[8], se[8], cnt3 = 0;
#pragma unroll
            for (int k = 0; k < 8; ++k) {
                const unsigned d = k < 4 ? d0[k & 3] : d1[k & 3];
                m5[k] = __builtin_amdgcn_udot4(d & (d << 1) & 0x20202020u, 0x08040201u, 0u, false);
                se[k] = *(const LAS unsigned*)(lut + (m5[k] >> 3));
                cnt3 += __builtin_popcount(m5[k]);
            }
            const unsigned sc3 = wave_scan(cnt3), n3 = __builtin_amdgcn_readlane(sc3, 63);
            bytes = o3 + ((n3 + 7) >> 3) * 4;
            raw = n3 > MAXESC || bytes > Lc;
            if (!raw) {
                o = sc3 - cnt3 + (unsigned)(size_t)(A + EP_L);
#pragma unroll
                for (int k = 0; k < 8; ++k) {
                    const unsigned long long z = (unsigned long long)__builtin_amdgcn_perm(0u, k < 4 ? d0[k & 3] : d1[k & 3], se[k]) << ((o & 3u) * 8);
                    LAS unsigned* d = (LAS unsigned*)(size_t)(o & ~3u);
                    __hip_atomic_fetch_or(d, (unsigned)z, __ATOMIC_RELAXED, __HIP_MEMORY_SCOPE_WAVEFRONT);
                    __hip_atomic_fetch_or(d + 1, (unsigned)(z >> 32), __ATOMIC_RELAXED, __HIP_MEMORY_SCOPE_WAVEFRONT);
                    o += __builtin_popcount(m5[k]);
                }
                CFENCE;
                const u32x2 x3 = *(const LAS u32x2*)(A + EP_L + lane * 8);
                *(LAS unsigned*)(B + o3 + lane * 4) = (x3[0] & 0x0f0f0f0fu) | ((x3[1] << 4) & 0xf0f0f0f0u);
                CFENCE;
            }
        }
    }
    if (raw) bytes = Lc;
    if (EPI_PRIO == 2 && wr == 0 && lane == 0) __hip_atomic_fetch_add(red + 38, 1u, __ATOMIC_RELAXED, __HIP_MEMORY_SCOPE_WORKGROUP);
    CYC(3);
    {
        LAS unsigned char* S = raw ? A : B;
#pragma nounroll
        for (unsigned r = 0; r < bytes; r += 1024)
            __builtin_amdgcn_raw_buffer_store_b128(*(const LAS u32x4*)(S + r + lane * 16), rs, lane * 16, OFF_COD + wt * 8192 + r, AUX_SYS);
    }
    if (lane == 0) { red[16 + wid] = ((call & 0x3ffffu) << 14) | Lc; red[24 + wid] = (call << 22) | (raw ? 1u << 21 : 0u) | (ec << 14) | bytes; }
    CYC(4);
    WAIT_V(0);
    CYC(5);
    {
        unsigned old = 0;
        if (lane == 0) old = __hip_atomic_fetch_add(red + 36 + wr, 1u, __ATOMIC_RELAXED, __HIP_MEMORY_SCOPE_WORKGROUP);
        if (__builtin_amdgcn_readfirstlane(old) == 3 && lane < 32 && (lane >> 2) != rank) {
            const int w = wr * 4 + (lane & 3);
            __hip_atomic_store((GAS unsigned long long*)((char*)p.ws[lane >> 2] + OFF_DONE + (rank * 1024 + tile * 8 + w) * 8),
                               ((unsigned long long)((volatile LAS unsigned*)red)[24 + w] << 32) | ((volatile LAS unsigned*)red)[16 + w], __ATOMIC_RELAXED, __HIP_MEMORY_SCOPE_SYSTEM);
        }
    }
    STAMP(myws, wv, 12);
    {
        const rsrc_t ro = mk_rsrc(p.out[rank], OUT_BYTES);
        const unsigned so = out_soff(rank, wt), vo = out_voff(lane);
        u32x4 qv[8];
#pragma unroll
        for (int ch = 0; ch < 8; ++ch) QUANT(ch, qv[ch]);
        SCHED;
#pragma unroll
        for (int ch = 0; ch < 8; ++ch) __builtin_amdgcn_raw_buffer_store_b128(qv[ch], ro, vo, so + chunk_soff(ch), OUT_AUX);
    }
}

__global__ __launch_bounds__(512, 2) void fused_kernel(Params p) {
    __shared__ __attribute__((aligned(16))) unsigned char lds_arr[LDS_BYTES];
    LAS unsigned char* lds = (LAS unsigned char*)lds_arr;
    const int wid = __builtin_amdgcn_readfirstlane(threadIdx.x >> 6), lane = threadIdx.x & 63;
    const int b = blockIdx.x, xcd = b & 7, idx = b >> 3;
    const unsigned callv = (unsigned)__builtin_amdgcn_raw_ptr_buffer_atomic_add_i32(1, mk_rsrc(p.ws[p.rank], WS_USED), lane == 0 ? OFF_CNT + (b * 8 + wid) * 4 : 0xfffffff0u, 0, 0);
    if (idx < 16) gemm_role(p, lds, (xcd >> 1) * 4 + (idx >> 2), (xcd & 1) * 4 + (idx & 3), wid, lane, callv);
    else comm_role(p, lds, (idx - 16) * 8 + xcd, wid, lane, callv);
}

extern "C" void kernel_launch(int rank, int nranks, ncclComm_t comm, hipStream_t stream,
                              void* const* d_in, const int* in_sizes, int n_in,
                              void* d_out, int out_size,
                              const int* in_dtypes, int out_dtype,
                              void* const* d_out_all, void* const* d_ws_all, size_t ws_bytes) {
    (void)comm; (void)in_dtypes; (void)out_dtype; (void)d_out;
    if (nranks != NR || n_in != 2 || in_sizes[0] != M_ * KS || in_sizes[1] != KS * N_ || out_size != M_ * N_ || ws_bytes < WS_USED || (unsigned)rank >= (unsigned)NR) {
        fprintf(stderr, "kernel_launch: unexpected problem shape\n");
        return;
    }
    Params p;
    p.x = d_in[0]; p.w = d_in[1];
    for (int i = 0; i < NR; ++i) { p.out[i] = d_out_all[i]; p.ws[i] = d_ws_all[i]; }
    p.rank = rank; p.pad = 0;
    fused_kernel<<<dim3(NWG), dim3(512), 0, stream>>>(p);
}
```

```cpp
#include <hip/hip_runtime.h>
#include <rccl/rccl.h>
#include <cstdio>

#define LAS __attribute__((address_space(3)))
#define GAS __attribute__((address_space(1)))
typedef short bf16x8 __attribute__((ext_vector_type(8)));
typedef float f32x4 __attribute__((ext_vector_type(4)));
typedef unsigned u32x4 __attribute__((ext_vector_type(4)));
typedef unsigned u32x2 __attribute__((ext_vector_type(2)));
typedef __amdgpu_buffer_rsrc_t rsrc_t;

constexpr int NR = 8;
constexpr int M_ = 4096, N_ = 16384;
constexpr int KS = 512;
constexpr int NS = 2048;
constexpr int NT = 64;
constexpr int NGEMM = 128;
constexpr int NWG = 256;

constexpr unsigned OFF_CNT  = 0;
constexpr unsigned OFF_TMO  = 16384;
constexpr unsigned OFF_FLAG = 65536;
constexpr unsigned FLAG_W = 4096;
constexpr unsigned OFF_AMAX = 131072;
constexpr unsigned OFF_TICK = 143360;
constexpr unsigned OFF_PCNT = 147456;
constexpr unsigned OFF_PFLG = 163840;
constexpr unsigned OFF_PUF  = 188416;
constexpr unsigned OFF_MFLG = 176128;
constexpr unsigned OFF_DONE = 786432;
constexpr unsigned OFF_MF   = OFF_DONE + 65536;
constexpr unsigned OFF_DBG  = 262144;
constexpr unsigned CTRL_BYTES = 1u << 20;
constexpr unsigned XT_STEP = 512u * 1024u, WT_STEP = 256u * 1024u, BLK = 32768u, HALFB = 16384u;
constexpr unsigned OFF_XT = CTRL_BYTES;
constexpr unsigned XT_BYTES = NT * XT_STEP;
constexpr unsigned OFF_WT = OFF_XT + XT_BYTES;
constexpr unsigned WT_BYTES = NT * WT_STEP;
constexpr unsigned UNIT = 9216;
constexpr unsigned OFF_PX = OFF_WT + WT_BYTES;
constexpr unsigned OFF_PW = OFF_PX + 512 * UNIT;
constexpr unsigned OFF_MSK = OFF_PW + 2048 * UNIT;
constexpr unsigned OFF_COD = OFF_MSK + 1024 * 1024;
constexpr unsigned OFF_LM = OFF_COD + 1024 * 8192;
constexpr unsigned WS_USED = OFF_LM + 8 * 1024 * 1024;
constexpr unsigned OUT_BYTES = (unsigned)M_ * N_;
constexpr unsigned X_BYTES = (unsigned)M_ * KS * 2, W_BYTES = (unsigned)KS * N_ * 2;

constexpr unsigned long long DEADLINE = 100000000ull;
constexpr int AUX_SYS = 17;
#define GEMM_AUX 17
#define PPP 16
#define PPO 55
#define PPM 17
#define PACE_W 800
#define PACE_W0 800
#define PACE_W1 150
#define PACE_X 400
#define PACE_E0 0
#define PACE_E 70
#define PACE_L0 300
#define PACE_O0 150
#define PACE_O 60
#define PULL_SERIAL 0
#define AMAX_SLEEP 4
#define NPUF 2
#define PACK_AUX 17
#define INX_AUX 0
#define IN_AUX 2
#define SCH_S 100
#define SCH_C 1000
#define PUSH_W 8
#define PUSH_U 3
#define EPI_PRIO 1
#define WT_ORD 1
#define ENT_SLEEP 8
#define AMAX_GATE 24
#define MSK_SLEEP 8
#define POLL_SLEEP 0
#define PUL_SLEEP 4
#define GEMM2 1
#define SCH_A 1
#define SCH_B 10
#define CW 4
#define OUT_AUX 16
#define LOC_AUX 17

constexpr unsigned LDS_STAGE = 8 * HALFB;
constexpr unsigned EP_BYTES = 19456;
constexpr unsigned EP_C = 8192, EP_S = EP_C + 5888, EP_L = EP_S + 2080, EP_T = 18432, EP_X = EP_T + 64;
constexpr unsigned LDS_FLAG = 8 * EP_BYTES;
constexpr unsigned LDS_RED = LDS_FLAG + 512;
constexpr unsigned EX_B = 8192, EX_M = EX_B + 8192, EX_P = EX_M + 1024, EX_E = EX_P + 1024, EX_BYTES = EX_E + 256, EX_LUT = 0;
constexpr unsigned LDS_BYTES = LDS_RED + 256;
static_assert(2048 + 8 * EX_BYTES <= LDS_BYTES && EP_BYTES % 1024 == 0 && EP_L + 1024 <= EP_T && EP_X + 256 <= EP_BYTES && LDS_STAGE <= LDS_FLAG && LDS_RED + 192 <= LDS_BYTES && LDS_BYTES <= 163840, "lds");
static_assert(OFF_PUF + 8 * NPUF * 384 <= OFF_DBG, "flags");
static_assert(LOC_AUX == AUX_SYS, "img_store also writes to peers");
static_assert((PUSH_W * PUSH_U) % 4 == 0 && PUSH_W * PUSH_U <= 32 && 7 * PUSH_W <= NGEMM, "push");
static_assert(7 * PPO <= 128 * CW && 7 * PPM <= 128 * CW && PPP <= PPM, "waves");

struct Args {
    const void* x; const void* w; void* out;
    int rank; int pad;
    void* ws[NR];
};
struct Params {
    const void* x; const void* w; void* out; char* my; char* pl;
    int rank;
    __device__ __forceinline__ char* peer(int k) const {
        const unsigned long long a = (unsigned long long)pl;
        return (char*)(((unsigned long long)(unsigned)__builtin_amdgcn_readlane((unsigned)(a >> 32), k) << 32) | (unsigned)__builtin_amdgcn_readlane((unsigned)a, k));
    }
};

__device__ __forceinline__ rsrc_t mk_rsrc(const void* p, unsigned bytes) {
    return __builtin_amdgcn_make_buffer_rsrc((void*)p, 0, bytes, 0x00020000);
}
#define ST128(v, r, vo, so, aux) do { const u32x4 v_ = (v); __builtin_amdgcn_raw_buffer_store_b128(v_, r, vo, so, aux); asm volatile("s_nop 1" :: "v"(v_) : "memory"); } while (0)
#define ST128_RAW __builtin_amdgcn_raw_buffer_store_b128
#define ST128_PAD4(v) asm volatile("s_nop 1" :: "v"((v)[0]), "v"((v)[1]), "v"((v)[2]), "v"((v)[3]) : "memory")
#define ST128_PAD8(v) asm volatile("s_nop 1" :: "v"((v)[0]), "v"((v)[1]), "v"((v)[2]), "v"((v)[3]), "v"((v)[4]), "v"((v)[5]), "v"((v)[6]), "v"((v)[7]) : "memory")
#define WAIT_V(n) asm volatile("s_waitcnt vmcnt(" #n ")" ::: "memory")
#define WAIT_L(n) asm volatile("s_waitcnt lgkmcnt(" #n ")" ::: "memory")
#define BAR __builtin_amdgcn_s_barrier()
#define SCHED __builtin_amdgcn_sched_barrier(0)
#define CFENCE asm volatile("" ::: "memory")

#define STAMP(ws_, wv_, k_) do {} while (0)
#define CYC(i) do {} while (0)

__device__ __forceinline__ unsigned ld_agent(const void* p) { return __hip_atomic_load((const GAS unsigned*)p, __ATOMIC_RELAXED, __HIP_MEMORY_SCOPE_AGENT); }
__device__ __forceinline__ void st_agent(void* p, unsigned v) { __hip_atomic_store((GAS unsigned*)p, v, __ATOMIC_RELAXED, __HIP_MEMORY_SCOPE_AGENT); }
__device__ __forceinline__ void st_sys(void* p, unsigned v) { __hip_atomic_store((GAS unsigned*)p, v, __ATOMIC_RELAXED, __HIP_MEMORY_SCOPE_SYSTEM); }

__device__ __forceinline__ void st_sys8(void* p, unsigned v) { __hip_atomic_store((GAS unsigned char*)p, (unsigned char)v, __ATOMIC_RELAXED, __HIP_MEMORY_SCOPE_SYSTEM); }
__device__ __forceinline__ unsigned x_flag(int t, int g) { return OFF_FLAG + (g >> 2) * 256 + t * 4 + (g & 3); }
__device__ __forceinline__ unsigned w_flag(int t, int e) { return OFF_FLAG + FLAG_W + (e >> 2) * 256 + t * 4 + (e & 3); }

struct Spin {
    unsigned long long t0; unsigned n;
    __device__ __forceinline__ Spin() : t0(__builtin_amdgcn_s_memrealtime()), n(0) {}
    __device__ __forceinline__ bool fail(char* myws, unsigned call) {
        if ((++n & 15u) == 0 && (__builtin_amdgcn_readfirstlane(ld_agent(myws + OFF_TMO)) == call || __builtin_amdgcn_s_memrealtime() - t0 > DEADLINE)) {
            st_agent(myws + OFF_TMO, call);
            return true;
        }
        __builtin_amdgcn_s_sleep(1);
        return false;
    }
};

#define DPP0(x, ctrl, rm) (unsigned)__builtin_amdgcn_update_dpp(0, (int)(x), ctrl, rm, 0xf, false)
__device__ __forceinline__ unsigned wave_scan(unsigned x) {
    x += DPP0(x, 0x111, 0xf); x += DPP0(x, 0x112, 0xf); x += DPP0(x, 0x114, 0xf); x += DPP0(x, 0x118, 0xf);
    x += DPP0(x, 0x142, 0xa); x += DPP0(x, 0x143, 0xc);
    return x;
}
__device__ __forceinline__ unsigned wave_max(unsigned x) {
    x = max(x, DPP0(x, 0x111, 0xf)); x = max(x, DPP0(x, 0x112, 0xf)); x = max(x, DPP0(x, 0x114, 0xf)); x = max(x, DPP0(x, 0x118, 0xf));
    x = max(x, DPP0(x, 0x142, 0xa)); x = max(x, DPP0(x, 0x143, 0xc));
    return __builtin_amdgcn_readlane(x, 63);
}

struct Sched {
    unsigned w[7][64];
    constexpr Sched() : w{} {
        for (int q = 0; q < 7; ++q) {
            int key[136] = {}, id[136] = {}, n = 0;
            for (int j = 0; j < 8; ++j) {
                int lo = 1 << 30;
                for (int rt = 0; rt < 16; ++rt) { const int k = (SCH_S * rt < SCH_C ? SCH_S * rt : SCH_C) + SCH_B * rt + 106 * (7 * j + (q + SCH_A * rt) % 7); key[n] = 2 * k + 1; id[n++] = j * 24 + 8 + rt; lo = k < lo ? k : lo; }
                key[n] = 2 * lo; id[n++] = -1 - j;
            }
            int at = 0;
            for (int o = 0; o < 136; ++o) {
                int best = -1;
                for (int i = 0; i < 136; ++i) if (key[i] >= 0 && (best < 0 || key[i] < key[best])) best = i;
                key[best] = -1;
                const int skip = id[best] == -1 ? PUSH_W * PUSH_U / 4 : 0;
                const int cnt = id[best] < 0 ? 8 - skip : 1, first = id[best] < 0 ? (-1 - id[best]) * 24 + skip : id[best];
                for (int c = 0; c < cnt; ++c, ++at) w[q][at >> 2] |= (unsigned)(first + c) << (8 * (at & 3));
            }
        }
    }
};
__constant__ const Sched SCHED_TAB;
__device__ __forceinline__ int step_of(int i, int rank, int rt) {
    if (i < 8) return rank * 8 + i;
    const int z = i - 8, j = z / 7, p = z - j * 7, q = (p + 7 * 16 * SCH_A - SCH_A * rt) % 7;
    return (((rank + 1 + q) & 7) << 3) + j;
}

__device__ __forceinline__ void x_load(u32x4 (&v)[8], rsrc_t rx, int j, int g, int lane) {
    const unsigned voff = (unsigned)(lane & 15) * (KS * 2) + (unsigned)(lane >> 4) * 16;
    const unsigned soff = (unsigned)g * 64 * (KS * 2) + (unsigned)j * 128;
#pragma unroll
    for (int r = 0; r < 4; ++r)
#pragma unroll
        for (int kk = 0; kk < 2; ++kk) v[r * 2 + kk] = __builtin_amdgcn_raw_buffer_load_b128(rx, voff + kk * 64, soff + r * 16 * (KS * 2), INX_AUX);
}
__device__ __forceinline__ void w_load(u32x2 (&v)[16], rsrc_t rw, int j, int d, int ct, int kk, int G, int lane) {
    const unsigned voff = (unsigned)(lane >> 4) * 8 * (N_ * 2) + (unsigned)(lane & 15) * 8;
    const unsigned soff = (unsigned)(j * 64 + kk * 32) * (N_ * 2) + (unsigned)(d * NS + ct * 256 + G * 128) * 2;
#pragma unroll
    for (int jj = 0; jj < 8; ++jj)
#pragma unroll
        for (int g = 0; g < 2; ++g) v[g * 8 + jj] = __builtin_amdgcn_raw_buffer_load_b64(rw, voff + g * 128, soff + jj * (N_ * 2), IN_AUX);
}
__device__ __forceinline__ void w_transpose(u32x4 (&o)[8], const u32x2 (&v)[16]) {
#pragma unroll
    for (int g = 0; g < 2; ++g)
#pragma unroll
        for (int c = 0; c < 4; ++c)
#pragma unroll
            for (int dd = 0; dd < 4; ++dd)
                o[g * 4 + c][dd] = __builtin_amdgcn_perm(v[g * 8 + 2 * dd + 1][c >> 1], v[g * 8 + 2 * dd][c >> 1], (c & 1) ? 0x07060302u : 0x05040100u);
}
__device__ __forceinline__ unsigned x_img(int t, int g) { return OFF_XT + (unsigned)t * XT_STEP + (unsigned)g * 8192; }
__device__ __forceinline__ unsigned w_img(int t, int e) { return OFF_WT + (unsigned)t * WT_STEP + (unsigned)(e >> 2) * BLK + (unsigned)((e >> 1) & 1) * 8192 + (unsigned)(e & 1) * 1024; }
__device__ __forceinline__ unsigned frag_off(bool isw, unsigned f) { return isw ? ((f >> 1) & 1) * HALFB + (f >> 2) * 4096 + (f & 1) * 2048 : f * 1024; }
__device__ __forceinline__ void img_store(const u32x4 (&v)[8], rsrc_t rd, bool isw, unsigned img, int lane) {
#pragma unroll
    for (int f = 0; f < 8; ++f) ST128_RAW(v[f], rd, lane * 16, img + frag_off(isw, f), LOC_AUX);
    ST128_PAD8(v);
}
constexpr unsigned H8 = 0x80808080u, MAXESC = 512;
__device__ __forceinline__ unsigned KS_(unsigned c) { asm("" : "+s"(c)); return c; }
__device__ __forceinline__ unsigned bits16(unsigned m0, unsigned m1, unsigned m2, unsigned m3, int sh) {
    const unsigned a = __builtin_amdgcn_udot4(m0, 0x08040201u, __builtin_amdgcn_udot4(m1, 0x80402010u, 0u, false), false);
    const unsigned b = __builtin_amdgcn_udot4(m2, 0x08040201u, __builtin_amdgcn_udot4(m3, 0x80402010u, 0u, false), false);
    return (a >> sh) | (b << (8 - sh));
}
__device__ __forceinline__ unsigned esc_count(const unsigned (&bm)[4], unsigned& first) {
    const unsigned c = __builtin_popcount(bm[0]) + __builtin_popcount(bm[1]) + __builtin_popcount(bm[2]) + __builtin_popcount(bm[3]), s = wave_scan(c);
    first = s - c;
    return __builtin_amdgcn_readlane(s, 63);
}
__device__ __forceinline__ void esc_list(const unsigned (&bm)[4], unsigned at, LAS unsigned char* list, int lane) {
#pragma unroll
    for (int w = 0; w < 2; ++w) {
        unsigned long long z = ((unsigned long long)bm[2 * w + 1] << 32) | bm[2 * w];
        while (__builtin_amdgcn_ballot_w64(z != 0) != 0) {
            if (z != 0) {
                const unsigned b = __builtin_ctzll(z);
                *(LAS unsigned short*)(list + min(at, MAXESC - 1) * 2) = (unsigned short)(w * 4096 + (b >> 4) * 1024 + lane * 16 + (b & 15));
                ++at; z &= z - 1;
            }
        }
    }
}
constexpr unsigned PK_HDR = 5 * 1024 + 512, PK_RAW = 0xffffu, PK_FIRST = 62;
__device__ __forceinline__ void pack_store(const u32x4 (&v)[8], rsrc_t rs, LAS unsigned char* L, unsigned slot, int lane) {
    unsigned p0[16], e[16], sum = 0;
#pragma unroll
    for (int r = 0; r < 16; ++r) {
        const unsigned lo = __builtin_amdgcn_perm(v[r >> 1][2 * (r & 1) + 1], v[r >> 1][2 * (r & 1)], 0x06040200u);
        const unsigned hi = __builtin_amdgcn_perm(v[r >> 1][2 * (r & 1) + 1], v[r >> 1][2 * (r & 1)], 0x07050301u);
        p0[r] = (hi & H8) | (lo & ~H8);
        e[r] = ((hi << 1) & 0xfefefefeu) | ((lo >> 7) & 0x01010101u);
        sum = __builtin_amdgcn_sad_u8(e[r], 0u, sum);
    }
#pragma unroll
    for (int r = 0; r < 4; ++r) *(LAS u32x4*)(L + r * 1024 + lane * 16) = (u32x4){e[4 * r], e[4 * r + 1], e[4 * r + 2], e[4 * r + 3]};
    const int tot = (int)__builtin_amdgcn_readlane(wave_scan(sum), 63);
    const unsigned base = (unsigned)min(max((tot - 16794) >> 12, 0), 247), base4 = base * 0x01010101u;
    u32x4 p2 = {0u, 0u, 0u, 0u}; u32x2 p1 = {0u, 0u};
    unsigned ms[16];
#pragma unroll
    for (int r = 0; r < 16; ++r) {
        const unsigned z = ((e[r] | H8) - (base4 & ~H8)) ^ ((e[r] ^ ~base4) & H8);
        ms[r] = (((z & 0x78787878u) + 0x78787878u) | z) & H8;
        p2[r >> 2] |= (z & 0x03030303u) << (2 * (r & 3));
        p1[r >> 3] |= (r & 7) >= 2 ? (z & 0x04040404u) << ((r & 7) - 2) : (z & 0x04040404u) >> (2 - (r & 7));
    }
    unsigned bm[4] = {bits16(ms[0], ms[1], ms[2], ms[3], 7) | (bits16(ms[4], ms[5], ms[6], ms[7], 7) << 16),
                      bits16(ms[8], ms[9], ms[10], ms[11], 7) | (bits16(ms[12], ms[13], ms[14], ms[15], 7) << 16), 0u, 0u};
    unsigned at;
    const unsigned n = esc_count(bm, at);
    bool bad = n > 254;
    *(LAS u32x2*)(L + 4096 + lane * 8) = p1;
    if (!bad) {
        esc_list(bm, at, L + 5120, lane);
        CFENCE;
        bool b = false;
#pragma nounroll
        for (unsigned i = lane; i < n; i += 64) {
            const unsigned pos = *(const LAS unsigned short*)(L + 5120 + i * 2), ev = L[pos];
            unsigned d = base - 1 - ev;
            if (ev == base + 8) d = 14; else if (ev == 0) d = 15; else b |= d > 13;
            *(LAS unsigned short*)(L + 4096 + 516 + i * 2) = (unsigned short)(pos | (d << 12));
        }
        bad = __builtin_amdgcn_ballot_w64(b) != 0;
    }
    if (lane == 0) *(LAS unsigned*)(L + 4096 + 512) = base | ((bad ? PK_RAW : n) << 8);
    CFENCE;
    ST128(*(const LAS u32x4*)(L + 4096 + lane * 16), rs, lane * 16 + 1024, slot + 4096, PACK_AUX);
    if (!bad) {
#pragma unroll
        for (int r = 0; r < 4; ++r) ST128(((u32x4){p0[4 * r], p0[4 * r + 1], p0[4 * r + 2], p0[4 * r + 3]}), rs, lane * 16 + r * 1024, slot, PACK_AUX);
        ST128(p2, rs, lane * 16, slot + 4096, PACK_AUX);
    } else {
#pragma unroll
        for (int f = 0; f < 8; ++f) ST128(v[f], rs, lane * 16, slot + (f < 5 ? f : f + 1) * 1024, PACK_AUX);
    }
}
constexpr unsigned PL_SLOT = 8192;
struct Puller {
    rsrc_t rp, rs; char* myws; int rank, src, q, lane; unsigned call, seen, tab;
    __device__ __forceinline__ int unit_of(int tk) const { return (int)((__builtin_amdgcn_readlane(tab, tk >> 4) >> (2 * (tk & 12))) & 0xffu) * 4 + (tk & 3); }
    __device__ __forceinline__ int xg(int i) const { return i - 32; }
    __device__ __forceinline__ unsigned unit_id(int u) const { const int j = u / 96, i = u % 96; return i < 32 ? 512 + (rank * 8 + j) * 32 + i : j * 64 + xg(i); }
    __device__ __forceinline__ unsigned flag_of(int u) const { const int j = u / 96, i = u % 96; return i < 32 ? w_flag(src * 8 + j, i) : x_flag(src * 8 + j, xg(i)); }
    __device__ __forceinline__ void issue(LAS unsigned char* b, int u) {
        const int j = u / 96, i = u % 96, need = j * 2 + (i >= 32);
        if (!((seen >> need) & 1u)) {
            Spin t0;
            for (;;) {
                CFENCE;
                const unsigned sum = OFF_PFLG + (src * 8 + j) * 128 + (i >= 32 ? 0 : 4);
                const unsigned f = __builtin_amdgcn_raw_buffer_load_b32(rs, (lane & 1) && j < NPUF ? OFF_PUF + ((src * NPUF + j) * 96 + i) * 4 : sum, 0, AUX_SYS);
                if (__builtin_amdgcn_readlane(f, 0) >= call) { seen |= 1u << need; break; }
                if (__builtin_amdgcn_readlane(f, 1) >= call) break;
                if (t0.fail(myws, call)) break;
                __builtin_amdgcn_s_sleep(PUL_SLEEP);
            }
            CFENCE;
        }
        const unsigned id = unit_id(u), slot = OFF_PX + id * UNIT;
        __builtin_amdgcn_raw_ptr_buffer_load_lds(rp, (LAS void*)(b + 4096), 16, lane < 40 ? lane * 16 : 0xf0000000u, slot + 4096, 1024, AUX_SYS);
        __builtin_amdgcn_raw_ptr_buffer_load_lds(rp, (LAS void*)(b + 4096), 16, lane * 16, slot + 4096, 0, AUX_SYS);
        __builtin_amdgcn_raw_ptr_buffer_load_lds(rp, (LAS void*)b, 16, lane * 16, slot, 0, AUX_SYS);
        __builtin_amdgcn_raw_ptr_buffer_load_lds(rp, (LAS void*)b, 16, lane * 16, slot, 1024, AUX_SYS);
        __builtin_amdgcn_raw_ptr_buffer_load_lds(rp, (LAS void*)b, 16, lane * 16, slot, 2048, AUX_SYS);
        __builtin_amdgcn_raw_ptr_buffer_load_lds(rp, (LAS void*)b, 16, lane * 16, slot, 3072, AUX_SYS);
    }
    __device__ __forceinline__ void finish(LAS unsigned char* b, int u) {
        const int j = u / 96, i = u % 96, t = src * 8 + j;
        const bool isw = i < 32;
        const unsigned img = isw ? w_img(t, i) : x_img(t, xg(i));
        const unsigned h = *(const LAS unsigned*)(b + PK_HDR);
        u32x4 v[8];
#pragma unroll
        for (int r = 0; r < 5; ++r) v[r] = *(const LAS u32x4*)(b + r * 1024 + lane * 16);
        const u32x2 p1 = *(const LAS u32x2*)(b + 5120 + lane * 8);
        const unsigned meta = __builtin_amdgcn_readfirstlane(h), n = meta >> 8, base = meta & 0xffu;
        if (n <= 254) {
            if (n > PK_FIRST) {
                __builtin_amdgcn_raw_ptr_buffer_load_lds(rp, (LAS void*)(b + 4096), 16, lane * 16, OFF_PX + unit_id(u) * UNIT + 4096, 1024, AUX_SYS);
                WAIT_V(0);
            }
            const unsigned base4 = base * 0x01010101u;
            const u32x4 p2 = v[4];
            u32x4 p0[4], e[4];
#pragma unroll
            for (int r = 0; r < 16; ++r) {
                const unsigned hb = (r & 7) >= 2 ? p1[r >> 3] >> ((r & 7) - 2) : p1[r >> 3] << (2 - (r & 7));
                e[r >> 2][r & 3] = (((p2[r >> 2] >> (2 * (r & 3))) & KS_(0x03030303u)) | (hb & 0x04040404u)) + base4;
            }
#pragma unroll
            for (int r = 0; r < 4; ++r) { p0[r] = v[r]; *(LAS u32x4*)(b + r * 1024 + lane * 16) = e[r]; }
            CFENCE;
#pragma nounroll
            for (unsigned k = lane; k < n; k += 64) {
                const unsigned en = *(const LAS unsigned short*)(b + PK_HDR + 4 + k * 2), d = en >> 12;
                b[en & 0xfffu] = (unsigned char)(d == 14 ? base + 8 : d == 15 ? 0u : base - 1 - d);
            }
            CFENCE;
#pragma unroll
            for (int r = 0; r < 4; ++r) e[r] = *(const LAS u32x4*)(b + r * 1024 + lane * 16);
#pragma unroll
            for (int r = 0; r < 16; ++r) {
                const unsigned pp = p0[r >> 2][r & 3], ee = e[r >> 2][r & 3];
                const unsigned lo = (pp & ~H8) | ((ee << 7) & H8), hi = (pp & H8) | ((ee >> 1) & ~H8);
                v[r >> 1][2 * (r & 1)] = __builtin_amdgcn_perm(hi, lo, 0x05010400u); v[r >> 1][2 * (r & 1) + 1] = __builtin_amdgcn_perm(hi, lo, 0x07030602u);
            }
        } else {
            const unsigned slot = OFF_PX + unit_id(u) * UNIT;
            WAIT_L(0);
            __builtin_amdgcn_raw_ptr_buffer_load_lds(rp, (LAS void*)b, 16, lane * 16, slot + 6144, 0, AUX_SYS);
            __builtin_amdgcn_raw_ptr_buffer_load_lds(rp, (LAS void*)b, 16, lane * 16, slot + 6144, 1024, AUX_SYS);
            __builtin_amdgcn_raw_ptr_buffer_load_lds(rp, (LAS void*)b, 16, lane * 16, slot + 6144, 2048, AUX_SYS);
            WAIT_V(0);
#pragma unroll
            for (int r = 5; r < 8; ++r) v[r] = *(const LAS u32x4*)(b + (r - 5) * 1024 + lane * 16);
        }
        img_store(v, rs, isw, img, lane);
    }
};
static_assert(OFF_PW == OFF_PX + 512 * UNIT, "unit ids run through the x and the w slots");

#define DRAW(r, tick) { unsigned long long ex_; asm volatile("s_mov_b64 %1, exec\n\ts_mov_b64 exec, 1\n\tglobal_atomic_add %0, %2, %3, %4 sc0\n\ts_mov_b64 exec, %1" \
        : "=&v"(r), "=&s"(ex_) : "v"(0u), "v"(1u), "s"(tick) : "memory"); }
#define TAKE(t, r, N) { unsigned t_; asm volatile("s_waitcnt vmcnt(" #N ")\n\tv_readfirstlane_b32 %0, %1" : "=s"(t_) : "v"(r) : "memory"); t = (int)(t_ - tbase); }
constexpr int NPUSH = PUSH_W * PUSH_U, NU_IN = 768 - NPUSH, NU_ALL = NU_IN + 256;

__device__ __forceinline__ void pull_role(const Params& p, LAS unsigned char* L, int pi, int lane, unsigned call, int wv) {
    Puller P; P.rank = p.rank; P.q = pi % 7; P.src = (p.rank + 1 + P.q) & 7; P.lane = lane; P.seen = 0; P.call = call;
    P.tab = SCHED_TAB.w[P.q][lane];
    asm volatile("" : "+v"(P.tab));
    P.myws = p.my; P.rs = mk_rsrc(P.myws, WS_USED); P.rp = mk_rsrc(p.peer(1 + P.q), WS_USED);
    char* tick = P.myws + OFF_TICK + P.q * 128;
    const unsigned tbase = (call - 1) * (NU_ALL + 2 * PPM);
    unsigned r0, r1, pf = 0;
    if (pi >= 7 * PPP) {
        Spin t0;
        while ((int)(__builtin_amdgcn_readfirstlane(ld_agent(tick)) - tbase) < NU_IN) { if (t0.fail(P.myws, call)) break; __builtin_amdgcn_s_sleep(32); }
    }
    DRAW(r0, tick); DRAW(r1, tick);
    int u0, u1, sl = 0;
    TAKE(u0, r0, 0); TAKE(u1, r1, 0);
    if ((unsigned)u0 < (unsigned)NU_IN) P.issue(L, P.unit_of(u0));
    WAIT_V(0);
    STAMP(P.myws, wv, 10);
    while ((unsigned)u0 < (unsigned)NU_IN) {
        unsigned r2;
        if ((unsigned)u1 < (unsigned)NU_IN) { P.issue(L + (sl ^ 1) * PL_SLOT, P.unit_of(u1)); DRAW(r2, tick); WAIT_V(7); }
        else { DRAW(r2, tick); WAIT_V(0); }
        if (pf != 0 && lane == 0) st_sys8(P.myws + pf, call);
        P.finish(L + sl * PL_SLOT, P.unit_of(u0));
        pf = P.flag_of(P.unit_of(u0));
        u0 = u1; sl ^= 1;
        TAKE(u1, r2, 8);
    }
    WAIT_V(0);
    if (pf != 0 && lane == 0) st_sys8(P.myws + pf, call);
    pf = 0;
    STAMP(P.myws, wv, 9);
    unsigned long long seen = 0; int sblk = -1;
#define MASK_ISSUE(b, u) { const int mu_ = (u) - NU_IN; \
        if (!((seen >> (mu_ & 63)) & 1) || (mu_ >> 6) != sblk) { Spin t0; sblk = mu_ >> 6;        \
            for (;;) { CFENCE; const unsigned f = __builtin_amdgcn_raw_buffer_load_b32(P.rs, lane * 4, OFF_MFLG + (P.src * 256 + sblk * 64) * 4, AUX_SYS); \
                seen = __builtin_amdgcn_ballot_w64(f >= call); \
                if ((seen >> (mu_ & 63)) & 1) break; if (t0.fail(P.myws, call)) break; __builtin_amdgcn_s_sleep(MSK_SLEEP); } \
            CFENCE; } \
        __builtin_amdgcn_raw_ptr_buffer_load_lds(P.rp, (LAS void*)(b), 16, lane * 16, OFF_MSK + mu_ * 4096, 0, AUX_SYS); \
        __builtin_amdgcn_raw_ptr_buffer_load_lds(P.rp, (LAS void*)(b), 16, lane * 16, OFF_MSK + mu_ * 4096, 1024, AUX_SYS); \
        __builtin_amdgcn_raw_ptr_buffer_load_lds(P.rp, (LAS void*)(b), 16, lane * 16, OFF_MSK + mu_ * 4096, 2048, AUX_SYS); \
        __builtin_amdgcn_raw_ptr_buffer_load_lds(P.rp, (LAS void*)(b), 16, lane * 16, OFF_MSK + mu_ * 4096, 3072, AUX_SYS); }
    if ((unsigned)u0 < (unsigned)NU_ALL) MASK_ISSUE(L + sl * PL_SLOT, u0);
    WAIT_V(0);
    STAMP(P.myws, wv, 16);
    while ((unsigned)u0 < (unsigned)NU_ALL) {
        unsigned r2;
        if ((unsigned)u1 < (unsigned)NU_ALL) { MASK_ISSUE(L + (sl ^ 1) * PL_SLOT, u1); DRAW(r2, tick); WAIT_V(5); }
        else { DRAW(r2, tick); WAIT_V(0); }
        if (pf != 0 && lane == 0) st_sys(P.myws + pf, call);
        u32x4 mv[4];
#pragma unroll
        for (int r = 0; r < 4; ++r) mv[r] = *(const LAS u32x4*)(L + sl * PL_SLOT + r * 1024 + lane * 16);
#pragma unroll
        for (int r = 0; r < 4; ++r) ST128_RAW(mv[r], P.rs, lane * 16 + r * 1024, OFF_LM + P.src * 1048576 + (u0 - NU_IN) * 4096, LOC_AUX);
        ST128_PAD4(mv);
        pf = OFF_MF + (P.src * 256 + u0 - NU_IN) * 4;
        u0 = u1; sl ^= 1;
        TAKE(u1, r2, 4);
    }
    WAIT_V(0);
    if (pf != 0 && lane == 0) st_sys(P.myws + pf, call);
    STAMP(P.myws, wv, 17);
}

constexpr unsigned L1_MAX = 4608, L2_MAX = 2048;
__device__ __forceinline__ unsigned off_l2(unsigned G) { return G * 8 + ((G * 2 + 3) & ~3u); }
__device__ __forceinline__ u32x2 lut_expand(unsigned byte) { u32x2 s = {0u, 0u}; unsigned c = 0;
#pragma unroll
    for (int i = 0; i < 8; ++i) { const unsigned b = (byte >> i) & 1u; s[i >> 2] |= (b ? c : 0x0cu) << (8 * (i & 3)); c += b; } return s; }
__device__ __forceinline__ unsigned lut_compact(unsigned nib) { unsigned s = 0x0c0c0c0cu, c = 0;
#pragma unroll
    for (int i = 0; i < 4; ++i) if ((nib >> i) & 1u) { s = (s & ~(0xffu << (8 * c))) | ((unsigned)i << (8 * c)); ++c; } return s; }
__device__ __forceinline__ unsigned popc4(const u32x4& m) { return __builtin_popcount(m[0]) + __builtin_popcount(m[1]) + __builtin_popcount(m[2]) + __builtin_popcount(m[3]); }
__device__ __forceinline__ unsigned nib4(unsigned m, int i) { return (i == 0 ? m << 2 : m >> (4 * i - 2)) & 0x3cu; }
__device__ __forceinline__ unsigned out_voff(int lane) { return (unsigned)(lane & 15) * N_ + (unsigned)(lane >> 4) * 16; }
__device__ __forceinline__ unsigned out_soff(int src, int wt) {
    const int tile = wt >> 3, w = wt & 7;
    return (unsigned)((tile >> 3) * 256 + (w >> 2) * 64) * N_ + (unsigned)(src * NS + (tile & 7) * 256 + (w & 3) * 64);
}
__device__ __forceinline__ constexpr unsigned chunk_soff(int ch) { return (unsigned)((ch >> 2) * 128 + (ch & 3) * 16) * N_; }

__device__ __forceinline__ void expand_role(const Params& p, LAS unsigned char* lds, LAS unsigned char* L, int ei, int lane, unsigned call, int wv) {
    const int rank = p.rank, q = ei % 7, src = (rank + 1 + q) & 7;
    char* myws = p.my;
    char* pws = p.peer(1 + q);
    const rsrc_t rs = mk_rsrc(myws, WS_USED), ro = mk_rsrc(p.out, OUT_BYTES);
#pragma unroll
    for (int i = 0; i < 4; ++i) *(LAS u32x2*)(lds + EX_LUT + (i * 64 + lane) * 8) = lut_expand(i * 64 + lane);
    char* tick = myws + OFF_TICK + (8 + q) * 128;
    const unsigned tbase = (call - 1) * (1024 + 2 * PPO), tag = call & 0x3ffffu;
    bool dead = false;
#define WT_OF(k) (WT_ORD && (unsigned)(k) < 1024u ? ((k) & ~63) | (((k) & 28) << 1) | (((k) >> 3) & 4) | ((k) & 3) : (k))
#define ENT_OFF(t) ((lane & 2) ? OFF_MF + (src * 256 + ((t) >> 2)) * 4 : OFF_DONE + (src * 1024 + (t)) * 8 + (lane & 1) * 4)
#define ENT_OK(e, x, f) (((e) >> 14) == tag && ((x) >> 22) == (tag & 0x3ffu) && (f) >= call)
#define ENT_FETCH(t) __builtin_amdgcn_raw_ptr_buffer_load_lds(rs, (LAS void*)(L + EX_E), 4, ENT_OFF(t), 0, 0, AUX_SYS)
#define EX_ISSUE(t, e, x, f, n, xo) { unsigned e_ = (e), x_ = (x), f_ = (f); \
        if (!ENT_OK(e_, x_, f_) && !dead) { Spin t0_; \
              \
            for (;;) { CFENCE; const bool had_ = (e_ >> 14) == tag; \
                const unsigned g_ = __builtin_amdgcn_raw_buffer_load_b32(rs, (lane & 2) && !had_ ? 0xf0000000u : ENT_OFF(t), 0, AUX_SYS); \
                e_ = __builtin_amdgcn_readlane(g_, 0); x_ = __builtin_amdgcn_readlane(g_, 1); f_ = __builtin_amdgcn_readlane(g_, 2); \
                if (!had_ && (e_ >> 14) == tag) continue; \
                if (ENT_OK(e_, x_, f_)) break; if (t0_.fail(myws, call)) { dead = true; break; } __builtin_amdgcn_s_sleep(ENT_SLEEP); } \
            CFENCE; } \
        n = min(e_ & 0x3fffu, 8192u); xo = x_; \
        const unsigned by_ = min((x_ & 0x3fffu) + 15 & ~15u, 8192u); \
        const rsrc_t rc_ = mk_rsrc(pws + OFF_COD + (unsigned)(t) * 8192, by_); \
        _Pragma("nounroll") for (unsigned r_ = 0; r_ < by_; r_ += 1024) \
            __builtin_amdgcn_raw_ptr_buffer_load_lds(rc_, (LAS void*)(L + EX_B + r_), 16, lane * 16, r_, 0, AUX_SYS); \
        __builtin_amdgcn_raw_ptr_buffer_load_lds(rs, (LAS void*)(L + EX_M), 16, lane * 16, OFF_LM + src * 1048576 + (t) * 1024, 0, AUX_SYS); }
    unsigned r0, n0 = 0, x0 = 0;
    DRAW(r0, tick);
    int t0;
    TAKE(t0, r0, 0); t0 = WT_OF(t0);
    if ((unsigned)t0 < 1024u) EX_ISSUE(t0, 0u, 0u, 0u, n0, x0);
    int t1;
    { unsigned r1; DRAW(r1, tick); TAKE(t1, r1, 0); t1 = WT_OF(t1); }
    STAMP(myws, wv, 18);
    while ((unsigned)t0 < 1024u) {
        if ((unsigned)t1 < 1024u) ENT_FETCH(t1);
        const u32x4 m = *(const LAS u32x4*)(L + EX_M + lane * 16);
        if ((x0 >> 21) & 1u) {
#pragma nounroll
            for (unsigned r = 0; r < n0; r += 1024) *(LAS u32x4*)(L + r + lane * 16) = *(const LAS u32x4*)(L + EX_B + r + lane * 16);
        } else {
            const unsigned G = min((n0 + 15) >> 4, L1_MAX / 16), ec = (x0 >> 14) & 15u;
            const unsigned T1 = (ec << 3) | (((ec - 1) & 15u) << 11) | (((ec - 2) & 15u) << 19) | 0x80000000u;
            const unsigned T2 = (((ec + 1) & 15u) << 3) | (((ec - 3) & 15u) << 11) | (((ec - 4) & 15u) << 19) | 0x80000000u;
            LAS unsigned char* S2 = L + EX_B + 5888;
            unsigned cd[5][4], n7[5][4], cnt2 = 0;
            const unsigned k01 = KS_(0x01010101u), k02 = KS_(0x02020202u), k03 = KS_(0x03030303u), k07 = KS_(0x07070707u), k80 = KS_(H8), kmul = KS_(0x00408102u);
#define LAYER1(i0, n) { u32x2 nn_[n]; unsigned hb_[n]; \
                _Pragma("unroll") for (int i = 0; i < n; ++i) { const unsigned g = (i0 + i) * 64 + lane; nn_[i] = *(const LAS u32x2*)(L + EX_B + g * 8); hb_[i] = *(const LAS unsigned short*)(L + EX_B + G * 8 + g * 2); } \
                SCHED; \
                _Pragma("unroll") for (int i = 0; i < n; ++i) { const unsigned hb = (i0 + i) * 64 + lane < G ? hb_[i] : 0u;        \
                    _Pragma("unroll") for (int k = 0; k < 4; ++k) { const unsigned nb = (k & 1) ? nn_[i][k >> 1] >> 4 : nn_[i][k >> 1]; \
                        const unsigned ex = __builtin_amdgcn_perm(0u, T1, ((nb >> 3) & k01) | (__builtin_amdgcn_ubfe(hb, 4 * k, 4) * kmul & k02)); \
                        cd[i0 + i][k] = (nb & k07) | ex; \
                        n7[i0 + i][k] = __builtin_amdgcn_udot4(ex & k80, 0x08040201u, 0u, false); \
                        cnt2 += __builtin_popcount(n7[i0 + i][k]); } } }
            LAYER1(0, 4);
#pragma unroll
            for (int k = 0; k < 4; ++k) cd[4][k] = n7[4][k] = 0;
            if (G > 256) LAYER1(4, 1);
            CYC(0);
            const unsigned sc2 = wave_scan(cnt2), n2 = min(__builtin_amdgcn_readlane(sc2, 63), L2_MAX), G2 = (n2 + 15) >> 4, o2 = off_l2(G), o3 = o2 + G2 * 4;
            {
                unsigned pk0 = *(const LAS unsigned*)(L + EX_B + o2 + lane * 4), pk1 = *(const LAS unsigned*)(L + EX_B + o2 + 256 + lane * 4);
                pk0 = lane < G2 ? pk0 : 0u; pk1 = 64 + lane < G2 ? pk1 : 0u;
                const unsigned x3 = *(const LAS unsigned*)(L + EX_B + o3 + lane * 4);
                *(LAS u32x2*)(L + EX_P + lane * 8) = (u32x2){(x3 << 3) & 0x78787878u, (x3 >> 1) & 0x78787878u};
                unsigned v2[8], m7[8], cnt3 = 0;
#pragma unroll
                for (int k = 0; k < 8; ++k) {
                    v2[k] = __builtin_amdgcn_perm(0u, T2, ((k < 4 ? pk0 : pk1) >> (2 * (k & 3))) & k03);
                    m7[k] = __builtin_amdgcn_udot4(v2[k] & k80, 0x08040201u, 0u, false);
                    cnt3 += __builtin_popcount(m7[k]);
                }
                unsigned o = wave_scan(cnt3) - cnt3 + (unsigned)(size_t)(L + EX_P), se[8], w0[8], w1[8], of[8];
                CFENCE;
#pragma unroll
                for (int k = 0; k < 8; ++k) {
                    se[k] = *(const LAS unsigned*)(lds + EX_LUT + (m7[k] >> 4));
                    const LAS unsigned* w = (const LAS unsigned*)(size_t)(o & ~3u); w0[k] = w[0]; w1[k] = w[1]; of[k] = o; o += __builtin_popcount(m7[k]);
                }
                SCHED;
#pragma unroll
                for (int k = 0; k < 8; ++k) v2[k] = (v2[k] & ~H8) | __builtin_amdgcn_perm(0u, __builtin_amdgcn_alignbyte(w1[k], w0[k], of[k]), se[k]);
                *(LAS u32x4*)(S2 + lane * 16) = (u32x4){v2[0], v2[1], v2[2], v2[3]};
                *(LAS u32x4*)(S2 + 1024 + lane * 16) = (u32x4){v2[4], v2[5], v2[6], v2[7]};
                CFENCE;
            }
            unsigned o = sc2 - cnt2 + (unsigned)(size_t)S2;
#define MERGE(i0, n) { unsigned se_[n][4], w0_[n][4], w1_[n][4], of_[n][4]; \
                _Pragma("unroll") for (int i = 0; i < n; ++i) _Pragma("unroll") for (int k = 0; k < 4; ++k) { \
                    se_[i][k] = *(const LAS unsigned*)(lds + EX_LUT + (n7[i0 + i][k] >> 4)); \
                    const LAS unsigned* w = (const LAS unsigned*)(size_t)(o & ~3u); w0_[i][k] = w[0]; w1_[i][k] = w[1]; of_[i][k] = o; o += __builtin_popcount(n7[i0 + i][k]); } \
                SCHED; \
                _Pragma("unroll") for (int i = 0; i < n; ++i) { u32x4 v; \
                    _Pragma("unroll") for (int k = 0; k < 4; ++k) v[k] = (cd[i0 + i][k] & ~H8) | __builtin_amdgcn_perm(0u, __builtin_amdgcn_alignbyte(w1_[i][k], w0_[i][k], of_[i][k]), se_[i][k]); \
                    *(LAS u32x4*)(L + ((i0 + i) * 64 + lane) * 16) = v; } }
            MERGE(0, 4);
            if (G > 256) MERGE(4, 1);
            CFENCE;
        }
        const unsigned cnt = popc4(m);
        unsigned off = wave_scan(cnt) - cnt + (unsigned)(size_t)L;
        WAIT_L(0);
        CYC(1);
        unsigned n1 = 0, x1 = 0;
        if ((unsigned)t1 < 1024u) {
            WAIT_V(0);
            const u32x4 ent = *(const LAS u32x4*)(L + EX_E);
            EX_ISSUE(t1, __builtin_amdgcn_readfirstlane(ent[0]), __builtin_amdgcn_readfirstlane(ent[1]), __builtin_amdgcn_readfirstlane(ent[2]), n1, x1);
        }
        unsigned r2;
        DRAW(r2, tick);
        CYC(2);
        const unsigned so = out_soff(src, t0), vo = out_voff(lane);
        u32x2 sel[16]; unsigned of[16], w0[16], w1[16], w2[16];
#pragma unroll
        for (int g = 0; g < 16; ++g) {
            const unsigned b8 = ((g & 3) == 0 ? m[g >> 2] << 3 : m[g >> 2] >> (8 * (g & 3) - 3)) & 0x7f8u;
            sel[g] = *(const LAS u32x2*)(lds + EX_LUT + b8);
            const LAS unsigned* w = (const LAS unsigned*)(size_t)(off & ~3u);
            w0[g] = w[0]; w1[g] = w[1]; w2[g] = w[2]; of[g] = off;
            off += __builtin_popcount(b8);
        }
        SCHED;
        u32x4 o[8];
#pragma unroll
        for (int g = 0; g < 16; ++g) {
            const unsigned a0 = __builtin_amdgcn_alignbyte(w1[g], w0[g], of[g]), a1 = __builtin_amdgcn_alignbyte(w2[g], w1[g], of[g]);
            o[g >> 1][(g & 1) * 2] = __builtin_amdgcn_perm(a1, a0, sel[g][0]); o[g >> 1][(g & 1) * 2 + 1] = __builtin_amdgcn_perm(a1, a0, sel[g][1]);
        }
        CYC(3);
        if (dead) {
#pragma unroll
            for (int ch = 0; ch < 8; ++ch) o[ch] = (u32x4){0x7f7f7f7fu, 0x7f7f7f7fu, 0x7f7f7f7fu, 0x7f7f7f7fu};
            CFENCE;
        }
#pragma unroll
        for (int ch = 0; ch < 8; ++ch) ST128_RAW(o[ch], ro, vo, so + chunk_soff(ch), OUT_AUX);
        ST128_PAD8(o);
        CYC(4);
        int t2;
        TAKE(t2, r2, 8); t2 = WT_OF(t2);
        CYC(5);
        t0 = t1; n0 = n1; x0 = x1; t1 = t2;
    }
    STAMP(myws, wv, 19);
}

__device__ __forceinline__ unsigned get_call(unsigned callv) { unsigned c; asm volatile("s_nop 0\n\tv_readfirstlane_b32 %0, %1" : "=s"(c) : "v"(callv)); return c + 1u; }

__device__ __forceinline__ void own_x(const Params& p, int j, int g, unsigned due, unsigned long long t0, int lane, unsigned callv) {
    while ((unsigned)(__builtin_amdgcn_s_memrealtime() - t0) < due) __builtin_amdgcn_s_sleep(4);
    u32x4 v[8];
    x_load(v, mk_rsrc(p.x, X_BYTES), j, g, lane);
    img_store(v, mk_rsrc(p.my, WS_USED), false, x_img(p.rank * 8 + j, g), lane);
    WAIT_V(0);
    if (lane == 0) st_sys8(p.my + x_flag(p.rank * 8 + j, g), get_call(callv));
}

__device__ __forceinline__ void comm_role(const Params& p, LAS unsigned char* lds, int c, int wid, int lane, unsigned callv) {
    const int rank = p.rank;
    char* myws = p.my;
    const rsrc_t rs = mk_rsrc(myws, WS_USED);
    const int wv = (NGEMM + c) * 8 + wid; (void)wv;
    STAMP(myws, wv, 0);
    if (wid >= 2) {
        const rsrc_t rx = mk_rsrc(p.x, X_BYTES), rw = mk_rsrc(p.w, W_BYTES);
        const int k0 = (wid - 2) * 128 + c;
        const bool far = k0 < 448;
        const unsigned long long tk0 = __builtin_amdgcn_s_memrealtime();
        for (int u = far ? k0 : k0 - 448; u < (far ? 1792 : 768); u += far ? 448 : 320) {
            const bool ownw = !far && u >= 64 && u < 320;
            const int ux = u < 64 ? u : u - 256;
            const int j = far ? u / 224 : ownw ? (u - 64) >> 5 : ux >> 6, r = far ? u % 224 : ownw ? 288 + ((u - 64) & 31) : 224 + (ux & 63), t = rank * 8 + j;
            if (far && j == 0 && r / 7 < NPUSH) continue;
            const unsigned due = far ? (j > 1 ? j * PACE_W - PACE_W0 : j * PACE_W1) : ownw ? PACE_E0 + j * PACE_E : j * PACE_X;
            while ((unsigned)(__builtin_amdgcn_s_memrealtime() - tk0) < due) __builtin_amdgcn_s_sleep(8);
            u32x4 v[8];
            if (r >= 224 && r < 288) {
                const int g = r - 224;
                x_load(v, rx, j, g, lane);
                pack_store(v, rs, lds + 2048 + wid * EX_BYTES, OFF_PX + (unsigned)(j * 64 + g) * UNIT, lane);
                WAIT_V(0);
                const unsigned call = get_call(callv);
                if (j < NPUF) {
                    if (lane < 8) st_sys(p.pl + OFF_PUF + ((rank * NPUF + j) * 96 + 32 + g) * 4, call);
                    if (j == 0) STAMP(myws, wv, 7);
                }
                unsigned old = 0;
                if (lane == 0) old = __hip_atomic_fetch_add((GAS unsigned*)(myws + OFF_PCNT + j * 128), 1u, __ATOMIC_RELAXED, __HIP_MEMORY_SCOPE_AGENT);
                if ((__builtin_amdgcn_readfirstlane(old) & 63) == 63 && lane < 8) st_sys(p.pl + OFF_PFLG + t * 128, call);
            } else {
                const bool own = r >= 288;
                const int e = own ? r - 288 : r / 7, d = own ? rank : (rank + 1 + r % 7) & 7;
                u32x2 w[16];
                w_load(w, rw, j, d, e >> 2, e & 1, (e >> 1) & 1, lane);
                w_transpose(v, w);
                if (own) {
                    img_store(v, rs, true, w_img(t, e), lane);
                    WAIT_V(0);
                    if (lane == 0) st_sys8(myws + w_flag(t, e), get_call(callv));
                } else {
                    const unsigned id = (unsigned)((d * 8 + j) * 32 + e);
                    pack_store(v, rs, lds + 2048 + wid * EX_BYTES, OFF_PW + id * UNIT, lane);
                    WAIT_V(0);
                    const unsigned call = get_call(callv);
                    if (j < NPUF) { if (lane == 1 + r % 7) st_sys(p.pl + OFF_PUF + ((rank * NPUF + j) * 96 + e) * 4, call); if (j == 0) STAMP(myws, wv, 6); }
                    unsigned old = 0;
                    if (lane == 0) old = __hip_atomic_fetch_add((GAS unsigned*)(myws + OFF_PCNT + (8 + d * 8 + j) * 128), 1u, __ATOMIC_RELAXED, __HIP_MEMORY_SCOPE_AGENT);
                    if (__builtin_amdgcn_readfirstlane(old) + 1 == call * (j ? 32 : 32 - NPUSH) && lane == 1 + r % 7) st_sys(p.pl + OFF_PFLG + t * 128 + 4, call);
                }
            }
            STAMP(myws, wv, far ? 1 + (j >> 2) : 3 + (j >> 2));
        }
    }
    if (wid == 1) {
        const unsigned long long tk0 = __builtin_amdgcn_s_memrealtime();
        if (NPUSH && c < 7 * PUSH_W) {
            const int d = (rank + 1 + c % 7) & 7, e0 = c / 7;
            const rsrc_t rw = mk_rsrc(p.w, W_BYTES), rd = mk_rsrc(p.peer(1 + c % 7), WS_USED);
            u32x2 w[PUSH_U ? PUSH_U : 1][16];
#pragma unroll
            for (int i = 0; i < PUSH_U; ++i) { const int e = e0 + i * PUSH_W; w_load(w[i], rw, 0, d, e >> 2, e & 1, (e >> 1) & 1, lane); }
#pragma unroll
            for (int i = 0; i < PUSH_U; ++i) {
                const int e = e0 + i * PUSH_W;
                u32x4 v[8];
                w_transpose(v, w[i]);
                img_store(v, rd, true, w_img(rank * 8, e), lane);
                WAIT_V(0);
                const unsigned call = get_call(callv);
                if (lane == 1 + c % 7) st_sys8(p.pl + w_flag(rank * 8, e), call);
            }
            STAMP(myws, wv, 14);
        }
        for (int u = c; u < 256; u += 128) own_x(p, u >> 5, 32 + (u & 31), PACE_L0 + (u >> 5) * PACE_O, tk0, lane, callv);
    }
    if (wid < CW) {
        const unsigned call = get_call(callv);
        const int ci = wid * 128 + c;
        if (ci < 7 * PPM) pull_role(p, lds + 2048 + wid * EX_BYTES, ci, lane, call, wv);
        if (ci < 7 * PPO) expand_role(p, lds, lds + 2048 + wid * EX_BYTES, ci, lane, call, wv);
    }
}

typedef float f32x2 __attribute__((ext_vector_type(2)));
__device__ __forceinline__ f32x4 scale4(const f32x4& v, const f32x2& s) {
    f32x2 a, b;
    asm("v_pk_mul_f32 %0, %1, %2" : "=v"(a) : "v"((f32x2){v[0], v[1]}), "v"(s));
    asm("v_pk_mul_f32 %0, %1, %2" : "=v"(b) : "v"((f32x2){v[2], v[3]}), "v"(s));
    return (f32x4){a[0], a[1], b[0], b[1]};
}
__device__ __forceinline__ void gemm_role(const Params& p, LAS unsigned char* lds, int rt, int ct, int wid, int lane, unsigned callv) {
    const int rank = p.rank;
    char* myws = p.my;
    const int tile = rt * 8 + ct;
    STAMP(myws, tile * 8 + wid, 0);
    if (tile >= 64 && wid >= 1 && wid <= 4)
        own_x(p, (tile >> 2) & 7, (3 - (tile >> 5)) * 16 + (tile & 3) * 4 + wid - 1, tile >= 96 ? PACE_E0 + ((tile >> 2) & 7) * PACE_E : PACE_O0 + ((tile >> 2) & 7) * PACE_O,
              __builtin_amdgcn_s_memrealtime(), lane, callv);
    const unsigned call = get_call(callv);
    const int wv = tile * 8 + wid; (void)wv;
    const int wr = wid >> 2, wc = wid & 3;
    const rsrc_t rs = mk_rsrc(myws, WS_USED);
    const unsigned voff = (unsigned)wid * 2048 + (unsigned)lane * 16;
    LAS unsigned char* ldsw = lds + wid * 2048;
    LAS unsigned char* lda = lds + wr * 8192 + lane * 16;
    LAS unsigned char* ldb = lds + wc * 4096 + lane * 16;
    const unsigned seqf = (unsigned)step_of(lane, rank, rt) * 4, fx = OFF_FLAG + rt * 256, fw = OFF_FLAG + FLAG_W + ct * 256, c4 = (call & 255u) * 0x01010101u;
    unsigned long long todo = ~0ull, rdy = 0;
    unsigned tl = 0;
    volatile LAS unsigned* tq = (volatile LAS unsigned*)(lds + LDS_RED + 160);
#define SA(b, h) (((b) * 2 + (h)) * HALFB)
#define SB(b, h) ((4 + (b) * 2 + (h)) * HALFB)
#define STAGE(bufoff, soff) do { \
        __builtin_amdgcn_raw_ptr_buffer_load_lds(rs, (LAS void*)(ldsw + (bufoff)), 16, voff, (soff), 0, GEMM_AUX); \
        __builtin_amdgcn_raw_ptr_buffer_load_lds(rs, (LAS void*)(ldsw + (bufoff)), 16, voff, (soff), 1024, GEMM_AUX); } while (0)
#define LDA(dst, b, h) do { _Pragma("unroll") for (int m = 0; m < 4; ++m) _Pragma("unroll") for (int k = 0; k < 2; ++k) dst[m][k] = *(const LAS bf16x8*)(lda + SA(b, h) + m * 2048 + k * 1024); } while (0)
#define LDB(dst, b, h) do { _Pragma("unroll") for (int n = 0; n < 2; ++n) _Pragma("unroll") for (int k = 0; k < 2; ++k) dst[n][k] = *(const LAS bf16x8*)(ldb + SB(b, h) + n * 2048 + k * 1024); } while (0)
#define MMA(ai, bj, At, Bt) do { __builtin_amdgcn_s_setprio(1); _Pragma("unroll") for (int m = 0; m < 4; ++m) _Pragma("unroll") for (int n = 0; n < 2; ++n) _Pragma("unroll") for (int k = 0; k < 2; ++k) \
        acc[ai][bj][m][n] = __builtin_amdgcn_mfma_f32_16x16x32_bf16(Bt[n][k], At[m][k], acc[ai][bj][m][n], 0, 0, 0); __builtin_amdgcn_s_setprio(0); } while (0)
#define OFFA(t) (OFF_XT + (unsigned)(t) * XT_STEP + (unsigned)rt * BLK)
#define OFFB(t) (OFF_WT + (unsigned)(t) * WT_STEP + (unsigned)ct * BLK)
    unsigned la = 0, lb = 0;
#define LOOK do { if (wid == 0) { WAIT_V(8); la = *(const LAS unsigned*)(lds + LDS_FLAG + lane * 4); lb = *(const LAS unsigned*)(lds + LDS_FLAG + 256 + lane * 4); } } while (0)
#define PICK(slot) do { if (wid == 0) { \
            if (todo != 0) { \
                rdy |= __builtin_amdgcn_ballot_w64(la == c4 && lb == c4); \
                if ((rdy & todo) == 0) { Spin t0_; \
                    for (;;) { CFENCE; \
                        const unsigned a_ = __builtin_amdgcn_raw_buffer_load_b32(rs, seqf, fx, AUX_SYS), b_ = __builtin_amdgcn_raw_buffer_load_b32(rs, seqf, fw, AUX_SYS); \
                        rdy |= __builtin_amdgcn_ballot_w64(a_ == c4 && b_ == c4); \
                        if ((rdy & todo) != 0) break; if (t0_.fail(myws, call)) { rdy = ~0ull; break; } __builtin_amdgcn_s_sleep(POLL_SLEEP); } } \
                const int pos_ = __builtin_ctzll(rdy & todo); \
                todo &= ~(1ull << pos_); tl = __builtin_amdgcn_readlane(seqf, pos_) >> 2; \
            } \
            tq[slot] = tl; \
            const unsigned skip_ = (todo & ~rdy) == 0 ? 0xf0000000u : 0u;        \
            __builtin_amdgcn_raw_ptr_buffer_load_lds(rs, (LAS void*)(lds + LDS_FLAG), 4, seqf, fx + skip_, 0, AUX_SYS); \
            __builtin_amdgcn_raw_ptr_buffer_load_lds(rs, (LAS void*)(lds + LDS_FLAG + 256), 4, seqf, fw + skip_, 0, AUX_SYS); \
            SCHED; } } while (0)
#define WAIT_STAGE do { if (wid == 0) { WAIT_V(8); WAIT_L(0); } else WAIT_V(6); } while (0)

    f32x4 acc[2][2][4][2];
#pragma unroll
    for (int a = 0; a < 2; ++a)
#pragma unroll
        for (int b = 0; b < 2; ++b)
#pragma unroll
            for (int m = 0; m < 4; ++m)
#pragma unroll
                for (int n = 0; n < 2; ++n) acc[a][b][m][n] = (f32x4){0.f, 0.f, 0.f, 0.f};
    bf16x8 At[4][2], B0[2][2], B1[2][2];

#define WAIT2 do { if (wid == 0) WAIT_V(10); else WAIT_V(8); WAIT_L(0); } while (0)
#define STEP2(b) { \
        LOOK; \
        LDB(B0, b, 0); LDA(At, b, 0); LDB(B1, b, 1); \
        PICK(b); \
        STAGE(SA(b ^ 1, 1), OFFA(t1) + HALFB); \
        WAIT2; BAR; \
        const unsigned tv = tq[b]; \
        MMA(0, 0, At, B0); MMA(0, 1, At, B1); BAR; SCHED; \
        t1 = __builtin_amdgcn_readfirstlane(tv); \
        LDA(At, b, 1); \
        STAGE(SB(b, 0), OFFB(t1)); STAGE(SB(b, 1), OFFB(t1) + HALFB); STAGE(SA(b, 0), OFFA(t1)); \
        WAIT2; BAR; \
        MMA(1, 0, At, B0); MMA(1, 1, At, B1); BAR; SCHED; }
    unsigned t1;
    {
        la = ~c4; PICK(2); PICK(3);
        if (wid == 1 && lane < 8) ((LAS unsigned*)(lds + LDS_RED))[32 + lane] = 0u;
        WAIT_L(0); BAR; CFENCE;
        STAMP(myws, wv, 1);
        const unsigned t0 = __builtin_amdgcn_readfirstlane(tq[2]);
        t1 = __builtin_amdgcn_readfirstlane(tq[3]);
        STAGE(SB(0, 0), OFFB(t0)); STAGE(SB(0, 1), OFFB(t0) + HALFB); STAGE(SA(0, 0), OFFA(t0)); STAGE(SA(0, 1), OFFA(t0) + HALFB);
        STAGE(SB(1, 0), OFFB(t1)); STAGE(SB(1, 1), OFFB(t1) + HALFB); STAGE(SA(1, 0), OFFA(t1));
        WAIT_V(8); BAR;
        if (wr == 1) BAR;
    }
    for (int i = 0; i < NT; i += 2) {
        STEP2(0);
        STEP2(1);
    }
    char* pl = p.pl;
    if (wr == 0) BAR;
    STAMP(myws, wv, 10);

    const int wt = tile * 8 + wid;
    LAS unsigned* red = (LAS unsigned*)(lds + LDS_RED);
    u32x4 msk;
#pragma unroll
    for (int w = 0; w < 4; ++w) {
        unsigned mm = 0;
#pragma unroll
        for (int e = 31; e >= 0; --e) {
            const int ch = 2 * w + (e >> 4);
            mm = __builtin_amdgcn_alignbit(mm, __float_as_uint(acc[ch >> 2][(e >> 1) & 1][ch & 3][e & 1][(e >> 2) & 3]), 31);
        }
        msk[w] = ~mm;
    }
    ST128(msk, rs, lane * 16, OFF_MSK + wt * 1024, AUX_SYS);
    SCHED;
    float mx = 0.f;
#pragma unroll
    for (int a = 0; a < 2; ++a)
#pragma unroll
        for (int b = 0; b < 2; ++b)
#pragma unroll
            for (int m = 0; m < 4; ++m)
#pragma unroll
                for (int n = 0; n < 2; ++n) mx = fmaxf(fmaxf(fmaxf(fmaxf(mx, acc[a][b][m][n][0]), acc[a][b][m][n][1]), acc[a][b][m][n][2]), acc[a][b][m][n][3]);
    const unsigned wm = wave_max(__float_as_uint(mx));
    {
        unsigned old = 0;
        if (lane == 0) {
            __hip_atomic_fetch_max(red + 32, wm, __ATOMIC_RELAXED, __HIP_MEMORY_SCOPE_WORKGROUP);
            old = __hip_atomic_fetch_add(red + 33, 1u, __ATOMIC_RELAXED, __HIP_MEMORY_SCOPE_WORKGROUP);
        }
        if (__builtin_amdgcn_readfirstlane(old) == 7) {
            const unsigned long long gv = ((unsigned long long)call << 32) | __hip_atomic_load(red + 32, __ATOMIC_RELAXED, __HIP_MEMORY_SCOPE_WORKGROUP);
            char* g = pl + OFF_AMAX + (rank * NGEMM + tile) * 8;
            unsigned long long ex_;
            asm volatile("s_mov_b64 %0, exec\n\ts_mov_b64 exec, 0xff\n\tglobal_store_dwordx2 %1, %2, off sc0 sc1\n\ts_mov_b64 exec, %0\n\ts_waitcnt vmcnt(1)" : "=&s"(ex_) : "v"(g), "v"(gv) : "memory");
        } else WAIT_V(0);
        unsigned old2 = 0;
        if (lane == 0) old2 = __hip_atomic_fetch_add(red + 34 + wr, 1u, __ATOMIC_RELAXED, __HIP_MEMORY_SCOPE_WORKGROUP);
        if (__builtin_amdgcn_readfirstlane(old2) == 3 && lane < 8) st_sys(pl + OFF_MFLG + (rank * 256 + tile * 2 + wr) * 4, call);
    }
#pragma unroll
    for (int a = 0; a < 2; ++a)
#pragma unroll
        for (int b = 0; b < 2; ++b)
#pragma unroll
            for (int m = 0; m < 4; ++m)
#pragma unroll
                for (int n = 0; n < 2; ++n)
#pragma unroll
                    for (int j = 0; j < 4; ++j) { acc[a][b][m][n][j] = fmaxf(acc[a][b][m][n][j], 0.f); if (j == 3) asm volatile("" : "+v"(acc[a][b][m][n])); }
    STAMP(myws, wv, 13);
    BAR; CFENCE;
    LAS unsigned char* A = lds + wid * EP_BYTES;
    LAS unsigned char* B = A + EP_C;
    LAS unsigned char* S2 = A + EP_S;
    const unsigned cnt = popc4(msk), scan = wave_scan(cnt), Lc = __builtin_amdgcn_readlane(scan, 63);
    const bool dense = Lc > L1_MAX;
    if (!dense) {
        unsigned o = (unsigned)(size_t)A + (scan - cnt) * 4;
        const unsigned spare = (unsigned)(size_t)(A + EP_X) + lane * 4;
#pragma unroll
        for (int e = 0; e < 128; ++e) {
            const int ch = e >> 4;
            const unsigned bit = __builtin_amdgcn_ubfe(msk[e >> 5], e & 31, 1);
            *(LAS float*)(size_t)(bit ? o ^ ((o >> 4) & 0x30u) : spare) = acc[ch >> 2][(e >> 1) & 1][ch & 3][e & 1][(e >> 2) & 3];
            o += bit * 4;
        }
    }
    float ysum = 0.f;
#pragma unroll
    for (int a = 0; a < 2; ++a)
#pragma unroll
        for (int b = 0; b < 2; ++b)
#pragma unroll
            for (int m = 0; m < 4; ++m) { const f32x4 t = acc[a][b][m][0] + acc[a][b][m][1]; ysum += (t[0] + t[1]) + (t[2] + t[3]); }
#pragma unroll
    for (int sh = 32; sh >= 1; sh >>= 1) ysum += __shfl_xor(ysum, sh);
    ysum *= 1.35f / (float)max(Lc, 1u);
    if (!dense && lane < 16 && Lc + lane < L1_MAX) *(LAS float*)(A + (((Lc + lane) * 4) ^ (((Lc + lane) >> 2) & 0x30u))) = ysum;
    LAS unsigned char* lut = A + EP_T;
    if (lane < 16) *(LAS unsigned*)(lut + lane * 4) = lut_compact(lane);
    if (AMAX_GATE) {
        if (wid == 0) {
            Spin t0;
            for (;;) {
                CFENCE;
                const unsigned g = __builtin_amdgcn_raw_buffer_load_b32(rs, (lane & 31) * 8 + 4, OFF_AMAX + (rank * NGEMM + 96) * 8, AUX_SYS);
                if (__builtin_popcountll(__builtin_amdgcn_ballot_w64(g >= call)) >= 2 * AMAX_GATE || t0.fail(myws, call)) break;
                __builtin_amdgcn_s_sleep(8);
            }
        }
        BAR; CFENCE;
    }
    {
        Spin t0;
        unsigned t, tm = ld_agent(myws + OFF_TMO);
        for (;;) {
            CFENCE;
            const u32x4 g = __builtin_amdgcn_raw_buffer_load_b128(rs, lane * 16, OFF_AMAX + wid * 1024, AUX_SYS);
            t = max(g[0], g[2]);
            if (__builtin_amdgcn_ballot_w64(g[1] < call || g[3] < call) == 0) break;
            if (t0.fail(myws, call)) { tm = call; break; }
            __builtin_amdgcn_s_sleep(AMAX_SLEEP);
        }
        t = wave_max(t);
        if (lane == 0) { red[8 + wid] = t; red[wid] = tm == call; }
    }
    WAIT_L(0); BAR; CFENCE;
    const float amax = __uint_as_float(wave_max(red[8 + (lane & 7)]));
    const bool dead = __builtin_amdgcn_ballot_w64(red[lane & 7] != 0) != 0;
    const float inv = dead ? __builtin_nanf("") : 448.0f / amax;
    STAMP(myws, wv, 11);
    if (EPI_PRIO == 1 && wr == 0) __builtin_amdgcn_s_setprio(2);
    if (EPI_PRIO == 2 && wr == 1) while (__builtin_amdgcn_readfirstlane(((volatile LAS unsigned*)red)[38]) < 4) __builtin_amdgcn_s_sleep(2);

    const f32x2 inv2 = {inv, inv};
#define QUANT(ch, q) { const int a = (ch) >> 2, m = (ch) & 3; \
        const f32x4 y0 = scale4(acc[a][0][m][0], inv2), y1 = scale4(acc[a][0][m][1], inv2), y2 = scale4(acc[a][1][m][0], inv2), y3 = scale4(acc[a][1][m][1], inv2); \
        _Pragma("unroll") for (int j = 0; j < 4; ++j) \
            q[j] = (unsigned)__builtin_amdgcn_cvt_pk_fp8_f32(y2[j], y3[j], __builtin_amdgcn_cvt_pk_fp8_f32(y0[j], y1[j], __float_as_int(y0[j]), false), true); }
    u32x4 cod[5];
    if (dense) {
#pragma unroll
        for (int r = 0; r < 9; ++r) *(LAS u32x4*)(A + r * 1024 + lane * 16) = (u32x4){0u, 0u, 0u, 0u};
        unsigned o = (unsigned)(size_t)A + scan - cnt;
#pragma unroll
        for (int ch = 0; ch < 8; ++ch) {
            u32x4 q;
            QUANT(ch, q);
#pragma unroll
            for (int j = 0; j < 4; ++j) {
                const unsigned n4 = nib4(msk[ch >> 1], (ch & 1) * 4 + j);
                const unsigned long long z = (unsigned long long)__builtin_amdgcn_perm(0u, q[j], *(const LAS unsigned*)(lut + n4)) << ((o & 3u) * 8);
                LAS unsigned* d = (LAS unsigned*)(size_t)(o & ~3u);
                __hip_atomic_fetch_or(d, (unsigned)z, __ATOMIC_RELAXED, __HIP_MEMORY_SCOPE_WAVEFRONT);
                __hip_atomic_fetch_or(d + 1, (unsigned)(z >> 32), __ATOMIC_RELAXED, __HIP_MEMORY_SCOPE_WAVEFRONT);
                o += __builtin_popcount(n4);
            }
        }
    } else {
#define CODES(i0, n) { f32x4 y_[n][4]; \
            _Pragma("unroll") for (int i = 0; i < n; ++i) _Pragma("unroll") for (int k = 0; k < 4; ++k) y_[i][k] = *(const LAS f32x4*)(A + ((i0 + i) * 64 + (i0 + i < 4 ? lane : lane & 31)) * 64 + ((k ^ (lane >> 2)) & 3) * 16); \
            SCHED; \
            _Pragma("unroll") for (int i = 0; i < n; ++i) _Pragma("unroll") for (int k = 0; k < 4; ++k) { const f32x4 y = scale4(y_[i][k], inv2); \
                cod[i0 + i][k] = (unsigned)__builtin_amdgcn_cvt_pk_fp8_f32(y[2], y[3], __builtin_amdgcn_cvt_pk_fp8_f32(y[0], y[1], __float_as_int(y[0]), false), true); } }
        CODES(0, 2); CODES(2, 2);
        cod[4] = (u32x4){0u, 0u, 0u, 0u};
        if (Lc > 4096) CODES(4, 1);
        CFENCE;
#pragma unroll
        for (int i = 0; i < 5; ++i) *(LAS u32x4*)(A + (i * 64 + lane) * 16) = cod[i];
#pragma unroll
        for (int r = 0; r < 3; ++r) *(LAS u32x4*)(S2 + r * 1024 + lane * 16) = (u32x4){0u, 0u, 0u, 0u};
    }
    CFENCE;
    CYC(0);
    const unsigned ec = (__builtin_amdgcn_readfirstlane(__builtin_amdgcn_cvt_pk_fp8_f32(fminf(ysum * inv, 448.f), 0.f, 0, false)) >> 3) & 15u;
    const unsigned E1 = (ec + 17) * 0x01010101u, G = (Lc + 15) >> 4;
    bool raw = dense;
    unsigned bytes = Lc;
    if (!raw) {
        unsigned s2[5][4], n7[5][4], cnt2 = 0;
        const unsigned k07 = KS_(0x07070707u), k08 = KS_(0x08080808u), k0f = KS_(0x0f0f0f0fu), k10 = KS_(0x10101010u), k80 = KS_(H8);
#define CLASSIFY(i0, n) { const u32x4* d_ = cod + i0; \
            _Pragma("unroll") for (int i = 0; i < n; ++i) { const unsigned g = (i0 + i) * 64 + lane, wgt = g < G ? 0x08040201u : 0u; unsigned nb[4], p_[4]; \
                _Pragma("unroll") for (int k = 0; k < 4; ++k) { \
                    const unsigned e4 = (d_[i][k] >> 3) & k0f, u = E1 - e4, t = u & k08, ui = (u | (t - (t >> 3))) & k07; \
                    p_[k] = __builtin_amdgcn_perm(0x98989898u, 0x10080098u, ui); \
                    s2[i0 + i][k] = e4 | __builtin_amdgcn_perm(0x30302010u, 0x30303000u, ui); \
                    nb[k] = (d_[i][k] & k07) | (p_[k] & k08); \
                    n7[i0 + i][k] = __builtin_amdgcn_udot4(p_[k] & k80, wgt, 0u, false); \
                    cnt2 += __builtin_popcount(n7[i0 + i][k]); } \
                if (g < G) { *(LAS u32x2*)(B + g * 8) = (u32x2){nb[0] | (nb[1] << 4), nb[2] | (nb[3] << 4)}; \
                    *(LAS unsigned short*)(B + G * 8 + g * 2) = (unsigned short)bits16(p_[0] & k10, p_[1] & k10, p_[2] & k10, p_[3] & k10, 4); } } }
        CLASSIFY(0, 4);
#pragma unroll
        for (int k = 0; k < 4; ++k) s2[4][k] = n7[4][k] = 0;
        if (G > 256) CLASSIFY(4, 1);
        CYC(1);
        const unsigned sc2 = wave_scan(cnt2), n2 = __builtin_amdgcn_readlane(sc2, 63), G2 = (n2 + 15) >> 4, o2 = off_l2(G), o3 = o2 + G2 * 4;
        raw = n2 > L2_MAX;
        if (!raw) {
            unsigned o = sc2 - cnt2 + (unsigned)(size_t)S2;
#define SQUEEZE(i0, n) { unsigned se_[n][4]; \
                _Pragma("unroll") for (int i = 0; i < n; ++i) _Pragma("unroll") for (int k = 0; k < 4; ++k) se_[i][k] = *(const LAS unsigned*)(lut + (n7[i0 + i][k] >> 5)); \
                SCHED; \
                _Pragma("unroll") for (int i = 0; i < n; ++i) _Pragma("unroll") for (int k = 0; k < 4; ++k) { \
                    const unsigned long long z = (unsigned long long)__builtin_amdgcn_perm(0u, s2[i0 + i][k], se_[i][k]) << ((o & 3u) * 8); \
                    LAS unsigned* d = (LAS unsigned*)(size_t)(o & ~3u); \
                    __hip_atomic_fetch_or(d, (unsigned)z, __ATOMIC_RELAXED, __HIP_MEMORY_SCOPE_WAVEFRONT); \
                    __hip_atomic_fetch_or(d + 1, (unsigned)(z >> 32), __ATOMIC_RELAXED, __HIP_MEMORY_SCOPE_WAVEFRONT); \
                    o += __builtin_popcount(n7[i0 + i][k]); } }
            SQUEEZE(0, 4);
            if (G > 256) SQUEEZE(4, 1);
            CFENCE;
            CYC(2);
            const u32x4 d0 = *(const LAS u32x4*)(S2 + lane * 16), d1 = *(const LAS u32x4*)(S2 + 1024 + lane * 16);
            if (lane < G2) *(LAS unsigned*)(B + o2 + lane * 4) = ((d0[0] >> 4) & 0x03030303u) | ((d0[1] >> 2) & 0x0c0c0c0cu) | (d0[2] & 0x30303030u) | ((d0[3] << 2) & 0xc0c0c0c0u);
            if (64 + lane < G2) *(LAS unsigned*)(B + o2 + 256 + lane * 4) = ((d1[0] >> 4) & 0x03030303u) | ((d1[1] >> 2) & 0x0c0c0c0cu) | (d1[2] & 0x30303030u) | ((d1[3] << 2) & 0xc0c0c0c0u);
            unsigned m5[8], se[8], cnt3 = 0;
#pragma unroll
            for (int k = 0; k < 8; ++k) {
                const unsigned d = k < 4 ? d0[k & 3] : d1[k & 3];
                m5[k] = __builtin_amdgcn_udot4(d & (d << 1) & 0x20202020u, 0x08040201u, 0u, false);
                se[k] = *(const LAS unsigned*)(lut + (m5[k] >> 3));
                cnt3 += __builtin_popcount(m5[k]);
            }
            const unsigned sc3 = wave_scan(cnt3), n3 = __builtin_amdgcn_readlane(sc3, 63);
            bytes = o3 + ((n3 + 7) >> 3) * 4;
            raw = n3 > MAXESC || bytes > Lc;
            if (!raw) {
                o = sc3 - cnt3 + (unsigned)(size_t)(A + EP_L);
#pragma unroll
                for (int k = 0; k < 8; ++k) {
                    const unsigned long long z = (unsigned long long)__builtin_amdgcn_perm(0u, k < 4 ? d0[k & 3] : d1[k & 3], se[k]) << ((o & 3u) * 8);
                    LAS unsigned* d = (LAS unsigned*)(size_t)(o & ~3u);
                    __hip_atomic_fetch_or(d, (unsigned)z, __ATOMIC_RELAXED, __HIP_MEMORY_SCOPE_WAVEFRONT);
                    __hip_atomic_fetch_or(d + 1, (unsigned)(z >> 32), __ATOMIC_RELAXED, __HIP_MEMORY_SCOPE_WAVEFRONT);
                    o += __builtin_popcount(m5[k]);
                }
                CFENCE;
                const u32x2 x3 = *(const LAS u32x2*)(A + EP_L + lane * 8);
                *(LAS unsigned*)(B + o3 + lane * 4) = (x3[0] & 0x0f0f0f0fu) | ((x3[1] << 4) & 0xf0f0f0f0u);
                CFENCE;
            }
        }
    }
    if (raw) bytes = Lc;
    if (EPI_PRIO == 2 && wr == 0 && lane == 0) __hip_atomic_fetch_add(red + 38, 1u, __ATOMIC_RELAXED, __HIP_MEMORY_SCOPE_WORKGROUP);
    CYC(3);
    {
        LAS unsigned char* S = raw ? A : B;
#pragma nounroll
        for (unsigned r = 0; r < bytes; r += 1024)
            ST128(*(const LAS u32x4*)(S + r + lane * 16), rs, lane * 16, OFF_COD + wt * 8192 + r, AUX_SYS);
    }
    if (lane == 0) { red[16 + wid] = ((call & 0x3ffffu) << 14) | Lc; red[24 + wid] = (call << 22) | (raw ? 1u << 21 : 0u) | (ec << 14) | bytes; }
    CYC(4);
    WAIT_V(0);
    CYC(5);
    {
        unsigned old = 0;
        if (lane == 0) old = __hip_atomic_fetch_add(red + 36 + wr, 1u, __ATOMIC_RELAXED, __HIP_MEMORY_SCOPE_WORKGROUP);
        if (__builtin_amdgcn_readfirstlane(old) == 3 && lane < 32 && (lane & 7) != 0) {
            const int w = wr * 4 + (lane >> 3);
            __hip_atomic_store((GAS unsigned long long*)(pl + OFF_DONE + (rank * 1024 + tile * 8 + w) * 8),
                               ((unsigned long long)((volatile LAS unsigned*)red)[24 + w] << 32) | ((volatile LAS unsigned*)red)[16 + w], __ATOMIC_RELAXED, __HIP_MEMORY_SCOPE_SYSTEM);
        }
    }
    STAMP(myws, wv, 12);
    {
        const rsrc_t ro = mk_rsrc(p.out, OUT_BYTES);
        const unsigned so = out_soff(rank, wt), vo = out_voff(lane);
        u32x4 qv[8];
#pragma unroll
        for (int ch = 0; ch < 8; ++ch) QUANT(ch, qv[ch]);
#pragma unroll
        for (int ch = 0; ch < 8; ++ch) ST128_RAW(qv[ch], ro, vo, so + chunk_soff(ch), OUT_AUX);
        ST128_PAD8(qv);
    }
}

__global__ __launch_bounds__(512, 2) void fused_kernel(Args a) {
    __shared__ __attribute__((aligned(16))) unsigned char lds_arr[LDS_BYTES];
    LAS unsigned char* lds = (LAS unsigned char*)lds_arr;
    const int wid = __builtin_amdgcn_readfirstlane(threadIdx.x >> 6), lane = threadIdx.x & 63;
    const int b = blockIdx.x, xcd = b & 7, idx = b >> 3;
    Params p;
    p.x = a.x; p.w = a.w; p.out = a.out; p.my = (char*)a.ws[0]; p.rank = a.rank; p.pl = (char*)a.ws[lane & 7];
    asm volatile("" : "+s"(p.x), "+s"(p.w), "+s"(p.out), "+s"(p.my), "+s"(p.rank));
    const unsigned callv = (unsigned)__builtin_amdgcn_raw_ptr_buffer_atomic_add_i32(1, mk_rsrc(p.my, WS_USED), lane == 0 ? OFF_CNT + (b * 8 + wid) * 4 : 0xfffffff0u, 0, 0);
    if (idx < 16) gemm_role(p, lds, (xcd >> 1) * 4 + (idx >> 2), (xcd & 1) * 4 + (idx & 3), wid, lane, callv);
    else comm_role(p, lds, (idx - 16) * 8 + xcd, wid, lane, callv);
}

extern "C" void kernel_launch(int rank, int nranks, ncclComm_t comm, hipStream_t stream,
                              void* const* d_in, const int* in_sizes, int n_in,
                              void* d_out, int out_size,
                              const int* in_dtypes, int out_dtype,
                              void* const* d_out_all, void* const* d_ws_all, size_t ws_bytes) {
    (void)comm; (void)in_dtypes; (void)out_dtype; (void)d_out_all;
    if (nranks != NR || n_in != 2 || in_sizes[0] != M_ * KS || in_sizes[1] != KS * N_ || out_size != M_ * N_ || ws_bytes < WS_USED || (unsigned)rank >= (unsigned)NR) {
        fprintf(stderr, "kernel_launch: unexpected problem shape\n");
        return;
    }
    Args a;
    a.x = d_in[0]; a.w = d_in[1]; a.out = d_out;
    for (int i = 0; i < NR; ++i) a.ws[i] = d_ws_all[(rank + i) % NR];
    a.rank = rank; a.pad = 0;
    fused_kernel<<<dim3(NWG), dim3(512), 0, stream>>>(a);
}
```

```cpp
#include <hip/hip_runtime.h>
#include <rccl/rccl.h>
#include <cstdio>

#define LAS __attribute__((address_space(3)))
#define GAS __attribute__((address_space(1)))
typedef short bf16x8 __attribute__((ext_vector_type(8)));
typedef float f32x4 __attribute__((ext_vector_type(4)));
typedef unsigned u32x4 __attribute__((ext_vector_type(4)));
typedef unsigned u32x2 __attribute__((ext_vector_type(2)));
typedef __amdgpu_buffer_rsrc_t rsrc_t;

constexpr int NR = 8;
constexpr int M_ = 4096, N_ = 16384;
constexpr int KS = 512;
constexpr int NS = 2048;
constexpr int NT = 64;
constexpr int NGEMM = 128;
constexpr int NWG = 256;

constexpr unsigned OFF_CNT  = 0;
constexpr unsigned OFF_TMO  = 16384;
constexpr unsigned OFF_FLAG = 65536;
constexpr unsigned FLAG_W = 4096;
constexpr unsigned OFF_AMAX = 131072;
constexpr unsigned OFF_TICK = 143360;
constexpr unsigned OFF_PCNT = 147456;
constexpr unsigned OFF_PFLG = 163840;
constexpr unsigned OFF_PUF  = 188416;
constexpr unsigned OFF_MFLG = 176128;
constexpr unsigned OFF_DONE = 786432;
constexpr unsigned OFF_MF   = OFF_DONE + 65536;
constexpr unsigned OFF_DBG  = 262144;
constexpr unsigned CTRL_BYTES = 1u << 20;
constexpr unsigned XT_STEP = 512u * 1024u, WT_STEP = 256u * 1024u, BLK = 32768u, HALFB = 16384u;
constexpr unsigned OFF_XT = CTRL_BYTES;
constexpr unsigned XT_BYTES = NT * XT_STEP;
constexpr unsigned OFF_WT = OFF_XT + XT_BYTES;
constexpr unsigned WT_BYTES = NT * WT_STEP;
constexpr unsigned UNIT = 9216;
constexpr unsigned OFF_PX = OFF_WT + WT_BYTES;
constexpr unsigned OFF_PW = OFF_PX + 512 * UNIT;
constexpr unsigned OFF_MSK = OFF_PW + 2048 * UNIT;
constexpr unsigned OFF_COD = OFF_MSK + 1024 * 1024;
constexpr unsigned OFF_LM = OFF_COD + 1024 * 8192;
constexpr unsigned WS_USED = OFF_LM + 8 * 1024 * 1024;
constexpr unsigned OUT_BYTES = (unsigned)M_ * N_;
constexpr unsigned X_BYTES = (unsigned)M_ * KS * 2, W_BYTES = (unsigned)KS * N_ * 2;

constexpr unsigned long long DEADLINE = 100000000ull;
constexpr int AUX_SYS = 17;
#define WS_END 1
constexpr int AUX_NT = 2, AUX_WT = 16;
#define PPP 15
#define PPM 16
#define PPO 55
#define CW 4
#define PUSH_W 8
#define PUSH_U 3
#define RAW_LAST 24
#define STAG 7
#define SCH_S 100
#define SCH_C 1000
#define SCH_B 10
#define SCH_A 1
constexpr int NPUF = 2;
constexpr int PACE_W = 800, PACE_W1 = 150, PACE_X = 400, PACE_E = 70, PACE_O = 60, PACE_O0 = 150, PACE_L0 = 300;

constexpr unsigned LDS_STAGE = 8 * HALFB;
constexpr unsigned EP_BYTES = 19456;
constexpr unsigned EP_C = 8192, EP_S = EP_C + 5888, EP_L = EP_S + 2080, EP_T = 18432, EP_X = EP_T + 64;
constexpr unsigned LDS_FLAG = 8 * EP_BYTES;
constexpr unsigned LDS_RED = LDS_FLAG + 512;
constexpr unsigned EX_B = 8192, EX_M = EX_B + 8192, EX_P = EX_M + 1024, EX_E = EX_P + 1024, EX_BYTES = EX_E + 256, EX_LUT = 0;
constexpr unsigned LDS_BYTES = LDS_RED + 256;
static_assert(2048 + 8 * EX_BYTES <= LDS_BYTES && EP_BYTES % 1024 == 0 && EP_L + 1024 <= EP_T && EP_X + 256 <= EP_BYTES && LDS_STAGE <= LDS_FLAG && LDS_RED + 192 <= LDS_BYTES && LDS_BYTES <= 163840, "lds");
static_assert(OFF_PUF + 8 * NPUF * 384 <= OFF_DBG, "flags");
static_assert((PUSH_W * PUSH_U) % 4 == 0 && PUSH_W * PUSH_U <= 32 && 7 * PUSH_W <= NGEMM, "push");
static_assert(7 * PPO <= 128 * CW && 7 * PPM <= 128 * CW && PPP <= PPM, "waves");

struct Args {
    const void* x; const void* w; void* out;
    int rank; int pad;
    void* ws[NR];
};
struct Params {
    const void* x; const void* w; void* out; char* my; char* pl;
    int rank;
    __device__ __forceinline__ char* peer(int k) const {
        const unsigned long long a = (unsigned long long)pl;
        return (char*)(((unsigned long long)(unsigned)__builtin_amdgcn_readlane((unsigned)(a >> 32), k) << 32) | (unsigned)__builtin_amdgcn_readlane((unsigned)a, k));
    }
};

__device__ __forceinline__ rsrc_t mk_rsrc(const void* p, unsigned bytes) {
    return __builtin_amdgcn_make_buffer_rsrc((void*)p, 0, bytes, 0x00020000);
}
#define ST128(v, r, vo, so, aux) do { const u32x4 v_ = (v); __builtin_amdgcn_raw_buffer_store_b128(v_, r, vo, so, aux); asm volatile("s_nop 1" :: "v"(v_) : "memory"); } while (0)
#define ST128_RAW __builtin_amdgcn_raw_buffer_store_b128
#define ST128_PAD4(v) asm volatile("s_nop 1" :: "v"((v)[0]), "v"((v)[1]), "v"((v)[2]), "v"((v)[3]) : "memory")
#define ST128_PAD8(v) asm volatile("s_nop 1" :: "v"((v)[0]), "v"((v)[1]), "v"((v)[2]), "v"((v)[3]), "v"((v)[4]), "v"((v)[5]), "v"((v)[6]), "v"((v)[7]) : "memory")
#define WAIT_V(n) asm volatile("s_waitcnt vmcnt(" #n ")" ::: "memory")
#define WAIT_L(n) asm volatile("s_waitcnt lgkmcnt(" #n ")" ::: "memory")
#define BAR __builtin_amdgcn_s_barrier()
#define SCHED __builtin_amdgcn_sched_barrier(0)
#define CFENCE asm volatile("" ::: "memory")

#define STAMP(ws_, wv_, k_) do {} while (0)
#define CYC(i) do {} while (0)

__device__ __forceinline__ unsigned ld_agent(const void* p) { return __hip_atomic_load((const GAS unsigned*)p, __ATOMIC_RELAXED, __HIP_MEMORY_SCOPE_AGENT); }
__device__ __forceinline__ void st_agent(void* p, unsigned v) { __hip_atomic_store((GAS unsigned*)p, v, __ATOMIC_RELAXED, __HIP_MEMORY_SCOPE_AGENT); }
__device__ __forceinline__ void st_sys(void* p, unsigned v) { __hip_atomic_store((GAS unsigned*)p, v, __ATOMIC_RELAXED, __HIP_MEMORY_SCOPE_SYSTEM); }

__device__ __forceinline__ void add_sys(void* p) { asm volatile("global_atomic_add %0, %1, off sc1" :: "v"(p), "v"(1u) : "memory"); }
__device__ __forceinline__ unsigned x_flag(int t, int g) { return OFF_FLAG + (g >> 2) * 256 + t * 4; }
__device__ __forceinline__ unsigned w_flag(int t, int e) { return OFF_FLAG + FLAG_W + (e >> 2) * 256 + t * 4; }

struct Spin {
    unsigned long long t0; unsigned n;
    __device__ __forceinline__ Spin() : t0(__builtin_amdgcn_s_memrealtime()), n(0) {}
    __device__ __forceinline__ bool fail(char* myws, unsigned call) {
        if ((++n & 15u) == 0 && (__builtin_amdgcn_readfirstlane(ld_agent(myws + OFF_TMO)) == call || __builtin_amdgcn_s_memrealtime() - t0 > DEADLINE)) {
            st_agent(myws + OFF_TMO, call);
            return true;
        }
        __builtin_amdgcn_s_sleep(1);
        return false;
    }
};

#define DPP0(x, ctrl, rm) (unsigned)__builtin_amdgcn_update_dpp(0, (int)(x), ctrl, rm, 0xf, false)
__device__ __forceinline__ unsigned wave_scan(unsigned x) {
    x += DPP0(x, 0x111, 0xf); x += DPP0(x, 0x112, 0xf); x += DPP0(x, 0x114, 0xf); x += DPP0(x, 0x118, 0xf);
    x += DPP0(x, 0x142, 0xa); x += DPP0(x, 0x143, 0xc);
    return x;
}
__device__ __forceinline__ unsigned wave_max(unsigned x) {
    x = max(x, DPP0(x, 0x111, 0xf)); x = max(x, DPP0(x, 0x112, 0xf)); x = max(x, DPP0(x, 0x114, 0xf)); x = max(x, DPP0(x, 0x118, 0xf));
    x = max(x, DPP0(x, 0x142, 0xa)); x = max(x, DPP0(x, 0x143, 0xc));
    return __builtin_amdgcn_readlane(x, 63);
}

struct Sched {
    unsigned w[7][64];
    constexpr Sched() : w{} {
        for (int q = 0; q < 7; ++q) {
            int key[136] = {}, id[136] = {}, n = 0;
            for (int j = 0; j < 8; ++j) {
                int lo = 1 << 30;
                for (int rt = 0; rt < 16; ++rt) { const int k = (SCH_S * rt < SCH_C ? SCH_S * rt : SCH_C) + SCH_B * rt + 106 * (7 * j + (q + SCH_A * rt) % 7); key[n] = 2 * k + 1; id[n++] = j * 24 + 8 + rt; lo = k < lo ? k : lo; }
                key[n] = 2 * lo; id[n++] = -1 - j;
            }
            int at = 0;
            for (int o = 0; o < 136; ++o) {
                int best = -1;
                for (int i = 0; i < 136; ++i) if (key[i] >= 0 && (best < 0 || key[i] < key[best])) best = i;
                key[best] = -1;
                const int skip = id[best] == -1 ? PUSH_W * PUSH_U / 4 : 0;
                const int cnt = id[best] < 0 ? 8 - skip : 1, first = id[best] < 0 ? (-1 - id[best]) * 24 + skip : id[best];
                for (int c = 0; c < cnt; ++c, ++at) w[q][at >> 2] |= (unsigned)(first + c) << (8 * (at & 3));
            }
        }
    }
};
__constant__ const Sched SCHED_TAB;
__device__ __forceinline__ int step_of(int i, int rank, int rt) {
    if (i < 8) return rank * 8 + i;
    const int z = i - 8, j = z / 7, p = z - j * 7, q = (p + 7 * 16 * SCH_A - SCH_A * rt) % 7;
    return (((rank + 1 + q) & 7) << 3) + j;
}

__device__ __forceinline__ void x_load(u32x4 (&v)[8], rsrc_t rx, int j, int g, int lane) {
    const unsigned voff = (unsigned)(lane & 15) * (KS * 2) + (unsigned)(lane >> 4) * 16;
    const unsigned soff = (unsigned)g * 64 * (KS * 2) + (unsigned)j * 128;
#pragma unroll
    for (int r = 0; r < 4; ++r)
#pragma unroll
        for (int kk = 0; kk < 2; ++kk) v[r * 2 + kk] = __builtin_amdgcn_raw_buffer_load_b128(rx, voff + kk * 64, soff + r * 16 * (KS * 2), 0);
}
__device__ __forceinline__ void w_load(u32x2 (&v)[16], rsrc_t rw, int j, int d, int ct, int kk, int G, int lane) {
    const unsigned voff = (unsigned)(lane >> 4) * 8 * (N_ * 2) + (unsigned)(lane & 15) * 8;
    const unsigned soff = (unsigned)(j * 64 + kk * 32) * (N_ * 2) + (unsigned)(d * NS + ct * 256 + G * 128) * 2;
#pragma unroll
    for (int jj = 0; jj < 8; ++jj)
#pragma unroll
        for (int g = 0; g < 2; ++g) v[g * 8 + jj] = __builtin_amdgcn_raw_buffer_load_b64(rw, voff + g * 128, soff + jj * (N_ * 2), AUX_NT);
}
__device__ __forceinline__ void w_transpose(u32x4 (&o)[8], const u32x2 (&v)[16]) {
#pragma unroll
    for (int g = 0; g < 2; ++g)
#pragma unroll
        for (int c = 0; c < 4; ++c)
#pragma unroll
            for (int dd = 0; dd < 4; ++dd)
                o[g * 4 + c][dd] = __builtin_amdgcn_perm(v[g * 8 + 2 * dd + 1][c >> 1], v[g * 8 + 2 * dd][c >> 1], (c & 1) ? 0x07060302u : 0x05040100u);
}
__device__ __forceinline__ unsigned x_img(int t, int g) { return OFF_XT + (unsigned)t * XT_STEP + (unsigned)g * 8192; }
__device__ __forceinline__ unsigned w_img(int t, int e) { return OFF_WT + (unsigned)t * WT_STEP + (unsigned)(e >> 2) * BLK + (unsigned)((e >> 1) & 1) * 8192 + (unsigned)(e & 1) * 1024; }
__device__ __forceinline__ unsigned frag_off(bool isw, unsigned f) { return isw ? ((f >> 1) & 1) * HALFB + (f >> 2) * 4096 + (f & 1) * 2048 : f * 1024; }
__device__ __forceinline__ void img_store(const u32x4 (&v)[8], rsrc_t rd, bool isw, unsigned img, int lane) {
#pragma unroll
    for (int f = 0; f < 8; ++f) ST128_RAW(v[f], rd, lane * 16, img + frag_off(isw, f), AUX_SYS);
    ST128_PAD8(v);
}
constexpr unsigned H8 = 0x80808080u, MAXESC = 512;
__device__ __forceinline__ unsigned KS_(unsigned c) { asm("" : "+s"(c)); return c; }
__device__ __forceinline__ unsigned bits16(unsigned m0, unsigned m1, unsigned m2, unsigned m3, int sh) {
    const unsigned a = __builtin_amdgcn_udot4(m0, 0x08040201u, __builtin_amdgcn_udot4(m1, 0x80402010u, 0u, false), false);
    const unsigned b = __builtin_amdgcn_udot4(m2, 0x08040201u, __builtin_amdgcn_udot4(m3, 0x80402010u, 0u, false), false);
    return (a >> sh) | (b << (8 - sh));
}
__device__ __forceinline__ unsigned esc_count(const unsigned (&bm)[4], unsigned& first) {
    const unsigned c = __builtin_popcount(bm[0]) + __builtin_popcount(bm[1]) + __builtin_popcount(bm[2]) + __builtin_popcount(bm[3]), s = wave_scan(c);
    first = s - c;
    return __builtin_amdgcn_readlane(s, 63);
}
__device__ __forceinline__ void esc_list(const unsigned (&bm)[4], unsigned at, LAS unsigned char* list, int lane) {
#pragma unroll
    for (int w = 0; w < 2; ++w) {
        unsigned long long z = ((unsigned long long)bm[2 * w + 1] << 32) | bm[2 * w];
        while (__builtin_amdgcn_ballot_w64(z != 0) != 0) {
            if (z != 0) {
                const unsigned b = __builtin_ctzll(z);
                *(LAS unsigned short*)(list + min(at, MAXESC - 1) * 2) = (unsigned short)(w * 4096 + (b >> 4) * 1024 + lane * 16 + (b & 15));
                ++at; z &= z - 1;
            }
        }
    }
}
constexpr unsigned PK_HDR = 5 * 1024 + 512, PK_RAW = 0xffffu, PK_FIRST = 62;
__device__ __forceinline__ void pack_store(const u32x4 (&v)[8], rsrc_t rs, LAS unsigned char* L, unsigned slot, int lane) {
    unsigned p0[16], e[16], sum = 0;
#pragma unroll
    for (int r = 0; r < 16; ++r) {
        const unsigned lo = __builtin_amdgcn_perm(v[r >> 1][2 * (r & 1) + 1], v[r >> 1][2 * (r & 1)], 0x06040200u);
        const unsigned hi = __builtin_amdgcn_perm(v[r >> 1][2 * (r & 1) + 1], v[r >> 1][2 * (r & 1)], 0x07050301u);
        p0[r] = (hi & H8) | (lo & ~H8);
        e[r] = ((hi << 1) & 0xfefefefeu) | ((lo >> 7) & 0x01010101u);
        sum = __builtin_amdgcn_sad_u8(e[r], 0u, sum);
    }
#pragma unroll
    for (int r = 0; r < 4; ++r) *(LAS u32x4*)(L + r * 1024 + lane * 16) = (u32x4){e[4 * r], e[4 * r + 1], e[4 * r + 2], e[4 * r + 3]};
    const int tot = (int)__builtin_amdgcn_readlane(wave_scan(sum), 63);
    const unsigned base = (unsigned)min(max((tot - 16794) >> 12, 0), 247), base4 = base * 0x01010101u;
    u32x4 p2 = {0u, 0u, 0u, 0u}; u32x2 p1 = {0u, 0u};
    unsigned ms[16];
#pragma unroll
    for (int r = 0; r < 16; ++r) {
        const unsigned z = ((e[r] | H8) - (base4 & ~H8)) ^ ((e[r] ^ ~base4) & H8);
        ms[r] = (((z & 0x78787878u) + 0x78787878u) | z) & H8;
        p2[r >> 2] |= (z & 0x03030303u) << (2 * (r & 3));
        p1[r >> 3] |= (r & 7) >= 2 ? (z & 0x04040404u) << ((r & 7) - 2) : (z & 0x04040404u) >> (2 - (r & 7));
    }
    unsigned bm[4] = {bits16(ms[0], ms[1], ms[2], ms[3], 7) | (bits16(ms[4], ms[5], ms[6], ms[7], 7) << 16),
                      bits16(ms[8], ms[9], ms[10], ms[11], 7) | (bits16(ms[12], ms[13], ms[14], ms[15], 7) << 16), 0u, 0u};
    unsigned at;
    const unsigned n = esc_count(bm, at);
    bool bad = n > 254;
    *(LAS u32x2*)(L + 4096 + lane * 8) = p1;
    if (!bad) {
        esc_list(bm, at, L + 5120, lane);
        CFENCE;
        bool b = false;
#pragma nounroll
        for (unsigned i = lane; i < n; i += 64) {
            const unsigned pos = *(const LAS unsigned short*)(L + 5120 + i * 2), ev = L[pos];
            unsigned d = base - 1 - ev;
            if (ev == base + 8) d = 14; else if (ev == 0) d = 15; else b |= d > 13;
            *(LAS unsigned short*)(L + 4096 + 516 + i * 2) = (unsigned short)(pos | (d << 12));
        }
        bad = __builtin_amdgcn_ballot_w64(b) != 0;
    }
    if (lane == 0) *(LAS unsigned*)(L + 4096 + 512) = base | ((bad ? PK_RAW : n) << 8);
    CFENCE;
    ST128(*(const LAS u32x4*)(L + 4096 + lane * 16), rs, lane * 16 + 1024, slot + 4096, AUX_SYS);
    if (!bad) {
#pragma unroll
        for (int r = 0; r < 4; ++r) ST128(((u32x4){p0[4 * r], p0[4 * r + 1], p0[4 * r + 2], p0[4 * r + 3]}), rs, lane * 16 + r * 1024, slot, AUX_SYS);
        ST128(p2, rs, lane * 16, slot + 4096, AUX_SYS);
    } else {
#pragma unroll
        for (int f = 0; f < 8; ++f) ST128(v[f], rs, lane * 16, slot + (f < 5 ? f : f + 1) * 1024, AUX_SYS);
    }
}
constexpr unsigned PL_SLOT = 8192;
struct Puller {
    rsrc_t rp, rs; char* myws; int rank, src, q, lane; unsigned call, seen, tab;
    __device__ __forceinline__ int unit_of(int tk) const { return (int)((__builtin_amdgcn_readlane(tab, tk >> 4) >> (2 * (tk & 12))) & 0xffu) * 4 + (tk & 3); }
    __device__ __forceinline__ int xg(int i) const { return i - 32; }
    __device__ __forceinline__ unsigned unit_id(int u) const { const int j = u / 96, i = u % 96; return i < 32 ? 512 + (rank * 8 + j) * 32 + i : j * 64 + xg(i); }
    __device__ __forceinline__ unsigned flag_of(int u) const { const int j = u / 96, i = u % 96; return i < 32 ? w_flag(src * 8 + j, i) : x_flag(src * 8 + j, xg(i)); }
    __device__ __forceinline__ void issue(LAS unsigned char* b, int u) {
        const int j = u / 96, i = u % 96, need = j * 2 + (i >= 32);
        if (!((seen >> need) & 1u)) {
            Spin t0;
            for (;;) {
                CFENCE;
                const unsigned sum = OFF_PFLG + (src * 8 + j) * 128 + (i >= 32 ? 0 : 4);
                const unsigned f = __builtin_amdgcn_raw_buffer_load_b32(rs, (lane & 1) && j < NPUF ? OFF_PUF + ((src * NPUF + j) * 96 + i) * 4 : sum, 0, AUX_SYS);
                if (__builtin_amdgcn_readlane(f, 0) >= call) { seen |= 1u << need; break; }
                if (__builtin_amdgcn_readlane(f, 1) >= call) break;
                if (t0.fail(myws, call)) break;
                __builtin_amdgcn_s_sleep(4);
            }
            CFENCE;
        }
        const unsigned id = unit_id(u), slot = OFF_PX + id * UNIT;
        __builtin_amdgcn_raw_ptr_buffer_load_lds(rp, (LAS void*)(b + 4096), 16, lane < 40 ? lane * 16 : 0xf0000000u, slot + 4096, 1024, AUX_SYS);
        __builtin_amdgcn_raw_ptr_buffer_load_lds(rp, (LAS void*)(b + 4096), 16, lane * 16, slot + 4096, 0, AUX_SYS);
        __builtin_amdgcn_raw_ptr_buffer_load_lds(rp, (LAS void*)b, 16, lane * 16, slot, 0, AUX_SYS);
        __builtin_amdgcn_raw_ptr_buffer_load_lds(rp, (LAS void*)b, 16, lane * 16, slot, 1024, AUX_SYS);
        __builtin_amdgcn_raw_ptr_buffer_load_lds(rp, (LAS void*)b, 16, lane * 16, slot, 2048, AUX_SYS);
        __builtin_amdgcn_raw_ptr_buffer_load_lds(rp, (LAS void*)b, 16, lane * 16, slot, 3072, AUX_SYS);
    }
    __device__ __forceinline__ void finish(LAS unsigned char* b, int u) {
        const int j = u / 96, i = u % 96, t = src * 8 + j;
        const bool isw = i < 32;
        const unsigned img = isw ? w_img(t, i) : x_img(t, xg(i));
        const unsigned h = *(const LAS unsigned*)(b + PK_HDR);
        u32x4 v[8];
#pragma unroll
        for (int r = 0; r < 5; ++r) v[r] = *(const LAS u32x4*)(b + r * 1024 + lane * 16);
        const u32x2 p1 = *(const LAS u32x2*)(b + 5120 + lane * 8);
        const unsigned meta = __builtin_amdgcn_readfirstlane(h), n = meta >> 8, base = meta & 0xffu;
        if (n <= 254) {
            if (n > PK_FIRST) {
                __builtin_amdgcn_raw_ptr_buffer_load_lds(rp, (LAS void*)(b + 4096), 16, lane * 16, OFF_PX + unit_id(u) * UNIT + 4096, 1024, AUX_SYS);
                WAIT_V(0);
            }
            const unsigned base4 = base * 0x01010101u;
            const u32x4 p2 = v[4];
            u32x4 p0[4], e[4];
#pragma unroll
            for (int r = 0; r < 16; ++r) {
                const unsigned hb = (r & 7) >= 2 ? p1[r >> 3] >> ((r & 7) - 2) : p1[r >> 3] << (2 - (r & 7));
                e[r >> 2][r & 3] = (((p2[r >> 2] >> (2 * (r & 3))) & KS_(0x03030303u)) | (hb & 0x04040404u)) + base4;
            }
#pragma unroll
            for (int r = 0; r < 4; ++r) { p0[r] = v[r]; *(LAS u32x4*)(b + r * 1024 + lane * 16) = e[r]; }
            CFENCE;
#pragma nounroll
            for (unsigned k = lane; k < n; k += 64) {
                const unsigned en = *(const LAS unsigned short*)(b + PK_HDR + 4 + k * 2), d = en >> 12;
                b[en & 0xfffu] = (unsigned char)(d == 14 ? base + 8 : d == 15 ? 0u : base - 1 - d);
            }
            CFENCE;
#pragma unroll
            for (int r = 0; r < 4; ++r) e[r] = *(const LAS u32x4*)(b + r * 1024 + lane * 16);
#pragma unroll
            for (int r = 0; r < 16; ++r) {
                const unsigned pp = p0[r >> 2][r & 3], ee = e[r >> 2][r & 3];
                const unsigned lo = (pp & ~H8) | ((ee << 7) & H8), hi = (pp & H8) | ((ee >> 1) & ~H8);
                v[r >> 1][2 * (r & 1)] = __builtin_amdgcn_perm(hi, lo, 0x05010400u); v[r >> 1][2 * (r & 1) + 1] = __builtin_amdgcn_perm(hi, lo, 0x07030602u);
            }
        } else {
            const unsigned slot = OFF_PX + unit_id(u) * UNIT;
            WAIT_L(0);
            __builtin_amdgcn_raw_ptr_buffer_load_lds(rp, (LAS void*)b, 16, lane * 16, slot + 6144, 0, AUX_SYS);
            __builtin_amdgcn_raw_ptr_buffer_load_lds(rp, (LAS void*)b, 16, lane * 16, slot + 6144, 1024, AUX_SYS);
            __builtin_amdgcn_raw_ptr_buffer_load_lds(rp, (LAS void*)b, 16, lane * 16, slot + 6144, 2048, AUX_SYS);
            WAIT_V(0);
#pragma unroll
            for (int r = 5; r < 8; ++r) v[r] = *(const LAS u32x4*)(b + (r - 5) * 1024 + lane * 16);
        }
        img_store(v, rs, isw, img, lane);
    }
};
static_assert(OFF_PW == OFF_PX + 512 * UNIT, "unit ids run through the x and the w slots");

#define DRAW(r, tick) { unsigned long long ex_; asm volatile("s_mov_b64 %1, exec\n\ts_mov_b64 exec, 1\n\tglobal_atomic_add %0, %2, %3, %4 sc0\n\ts_mov_b64 exec, %1" \
        : "=&v"(r), "=&s"(ex_) : "v"(0u), "v"(1u), "s"(tick) : "memory"); }
#define TAKE(t, r, N) { unsigned t_; asm volatile("s_waitcnt vmcnt(" #N ")\n\tv_readfirstlane_b32 %0, %1" : "=s"(t_) : "v"(r) : "memory"); t = (int)(t_ - tbase); }
constexpr int NPUSH = PUSH_W * PUSH_U, NU_IN = 768 - NPUSH, NU_ALL = NU_IN + 256;

__device__ __forceinline__ void pull_role(const Params& p, LAS unsigned char* L, int pi, int lane, unsigned call, int wv) {
    Puller P; P.rank = p.rank; P.q = pi % 7; P.src = (p.rank + 1 + P.q) & 7; P.lane = lane; P.seen = 0; P.call = call;
    P.tab = SCHED_TAB.w[P.q][lane];
    asm volatile("" : "+v"(P.tab));
    P.myws = p.my; P.rs = mk_rsrc(P.myws, WS_USED); P.rp = mk_rsrc(p.peer(1 + P.q), WS_USED);
    char* tick = P.myws + OFF_TICK + P.q * 128;
    const unsigned tbase = (call - 1) * (NU_ALL + 2 * PPM);
    unsigned r0, r1, pf = 0;
    if (pi >= 7 * PPP) {
        Spin t0;
        while ((int)(__builtin_amdgcn_readfirstlane(ld_agent(tick)) - tbase) < NU_IN) { if (t0.fail(P.myws, call)) break; __builtin_amdgcn_s_sleep(32); }
    }
    DRAW(r0, tick); DRAW(r1, tick);
    int u0, u1, sl = 0;
    TAKE(u0, r0, 0); TAKE(u1, r1, 0);
    if ((unsigned)u0 < (unsigned)NU_IN) P.issue(L, P.unit_of(u0));
    WAIT_V(0);
    STAMP(P.myws, wv, 10);
    while ((unsigned)u0 < (unsigned)NU_IN) {
        unsigned r2;
        if ((unsigned)u1 < (unsigned)NU_IN) { P.issue(L + (sl ^ 1) * PL_SLOT, P.unit_of(u1)); DRAW(r2, tick); WAIT_V(7); }
        else { DRAW(r2, tick); WAIT_V(0); }
        if (pf != 0 && lane == 0) add_sys(P.myws + pf);
        P.finish(L + sl * PL_SLOT, P.unit_of(u0));
        pf = P.flag_of(P.unit_of(u0));
        u0 = u1; sl ^= 1;
        TAKE(u1, r2, 8);
    }
    WAIT_V(0);
    if (pf != 0 && lane == 0) add_sys(P.myws + pf);
    pf = 0;
    STAMP(P.myws, wv, 9);
    unsigned long long seen = 0; int sblk = -1;
#define MASK_ISSUE(b, u) { const int mu_ = (u) - NU_IN; \
        if (!((seen >> (mu_ & 63)) & 1) || (mu_ >> 6) != sblk) { Spin t0; sblk = mu_ >> 6;        \
            for (;;) { CFENCE; const unsigned f = __builtin_amdgcn_raw_buffer_load_b32(P.rs, lane * 4, OFF_MFLG + (P.src * 256 + sblk * 64) * 4, AUX_SYS); \
                seen = __builtin_amdgcn_ballot_w64(f >= call); \
                if ((seen >> (mu_ & 63)) & 1) break; if (t0.fail(P.myws, call)) break; __builtin_amdgcn_s_sleep(8); } \
            CFENCE; } \
        __builtin_amdgcn_raw_ptr_buffer_load_lds(P.rp, (LAS void*)(b), 16, lane * 16, OFF_MSK + mu_ * 4096, 0, AUX_SYS); \
        __builtin_amdgcn_raw_ptr_buffer_load_lds(P.rp, (LAS void*)(b), 16, lane * 16, OFF_MSK + mu_ * 4096, 1024, AUX_SYS); \
        __builtin_amdgcn_raw_ptr_buffer_load_lds(P.rp, (LAS void*)(b), 16, lane * 16, OFF_MSK + mu_ * 4096, 2048, AUX_SYS); \
        __builtin_amdgcn_raw_ptr_buffer_load_lds(P.rp, (LAS void*)(b), 16, lane * 16, OFF_MSK + mu_ * 4096, 3072, AUX_SYS); }
    if ((unsigned)u0 < (unsigned)NU_ALL) MASK_ISSUE(L + sl * PL_SLOT, u0);
    WAIT_V(0);
    STAMP(P.myws, wv, 16);
    while ((unsigned)u0 < (unsigned)NU_ALL) {
        unsigned r2;
        if ((unsigned)u1 < (unsigned)NU_ALL) { MASK_ISSUE(L + (sl ^ 1) * PL_SLOT, u1); DRAW(r2, tick); WAIT_V(5); }
        else { DRAW(r2, tick); WAIT_V(0); }
        if (pf != 0 && lane == 0) st_sys(P.myws + pf, call);
        u32x4 mv[4];
#pragma unroll
        for (int r = 0; r < 4; ++r) mv[r] = *(const LAS u32x4*)(L + sl * PL_SLOT + r * 1024 + lane * 16);
#pragma unroll
        for (int r = 0; r < 4; ++r) ST128_RAW(mv[r], P.rs, lane * 16 + r * 1024, OFF_LM + P.src * 1048576 + (u0 - NU_IN) * 4096, AUX_SYS);
        ST128_PAD4(mv);
        pf = OFF_MF + (P.src * 256 + u0 - NU_IN) * 4;
        u0 = u1; sl ^= 1;
        TAKE(u1, r2, 4);
    }
    WAIT_V(0);
    if (pf != 0 && lane == 0) st_sys(P.myws + pf, call);
    STAMP(P.myws, wv, 17);
}

constexpr unsigned L1_MAX = 4608, L2_MAX = 2048;
__device__ __forceinline__ unsigned off_l2(unsigned G) { return G * 8 + ((G * 2 + 3) & ~3u); }
__device__ __forceinline__ u32x2 lut_expand(unsigned byte) { u32x2 s = {0u, 0u}; unsigned c = 0;
#pragma unroll
    for (int i = 0; i < 8; ++i) { const unsigned b = (byte >> i) & 1u; s[i >> 2] |= (b ? c : 0x0cu) << (8 * (i & 3)); c += b; } return s; }
__device__ __forceinline__ unsigned lut_compact(unsigned nib) { unsigned s = 0x0c0c0c0cu, c = 0;
#pragma unroll
    for (int i = 0; i < 4; ++i) if ((nib >> i) & 1u) { s = (s & ~(0xffu << (8 * c))) | ((unsigned)i << (8 * c)); ++c; } return s; }
__device__ __forceinline__ unsigned popc4(const u32x4& m) { return __builtin_popcount(m[0]) + __builtin_popcount(m[1]) + __builtin_popcount(m[2]) + __builtin_popcount(m[3]); }
__device__ __forceinline__ unsigned nib4(unsigned m, int i) { return (i == 0 ? m << 2 : m >> (4 * i - 2)) & 0x3cu; }
__device__ __forceinline__ unsigned out_voff(int lane) { return (unsigned)(lane & 15) * N_ + (unsigned)(lane >> 4) * 16; }
__device__ __forceinline__ unsigned out_soff(int src, int wt) {
    const int tile = wt >> 3, w = wt & 7;
    return (unsigned)((tile >> 3) * 256 + (w >> 2) * 64) * N_ + (unsigned)(src * NS + (tile & 7) * 256 + (w & 3) * 64);
}
__device__ __forceinline__ constexpr unsigned chunk_soff(int ch) { return (unsigned)((ch >> 2) * 128 + (ch & 3) * 16) * N_; }

__device__ __forceinline__ void expand_role(const Params& p, LAS unsigned char* lds, LAS unsigned char* L, int ei, int lane, unsigned call, int wv) {
    const int rank = p.rank, q = ei % 7, src = (rank + 1 + q) & 7;
    char* myws = p.my;
    char* pws = p.peer(1 + q);
    const rsrc_t rs = mk_rsrc(myws, WS_USED), ro = mk_rsrc(p.out, OUT_BYTES);
#pragma unroll
    for (int i = 0; i < 4; ++i) *(LAS u32x2*)(lds + EX_LUT + (i * 64 + lane) * 8) = lut_expand(i * 64 + lane);
    char* tick = myws + OFF_TICK + (8 + q) * 128;
    const unsigned tbase = (call - 1) * (1024 + 2 * PPO);
    bool dead = false;
#define WT_OF(k) ((unsigned)(k) < 1024u ? ((k) & ~63) | (((k) & 28) << 1) | (((k) >> 3) & 4) | ((k) & 3) : (k))
#define ENT_OFF(t) (lane == 0 ? OFF_DONE + (src * 1024 + ((t) & ~1)) * 8 : OFF_MF + (src * 256 + (((t) >> 2) & ~3)) * 4)
#define ENT_FETCH(t) { if (lane < 2) __builtin_amdgcn_raw_ptr_buffer_load_lds(rs, (LAS void*)(L + EX_E), 16, ENT_OFF(t), 0, 0, AUX_SYS); }
#define EX_ISSUE(t, e, x, f, n, xo) { unsigned e_ = (e), x_ = (x), f_ = (f); \
        if ((e_ < call || f_ < call) && !dead) { Spin t0_; \
              \
            for (;;) { CFENCE; const bool had_ = e_ >= call; \
                const u32x4 g_ = __builtin_amdgcn_raw_buffer_load_b128(rs, lane < (had_ ? 2 : 1) ? ENT_OFF(t) : 0xf0000000u, 0, AUX_SYS); \
                e_ = __builtin_amdgcn_readlane(((t) & 1) ? g_[2] : g_[0], 0); x_ = __builtin_amdgcn_readlane(((t) & 1) ? g_[3] : g_[1], 0); \
                f_ = __builtin_amdgcn_readlane(((t) & 8) ? (((t) & 4) ? g_[3] : g_[2]) : (((t) & 4) ? g_[1] : g_[0]), 1); \
                if (!had_ && e_ >= call) continue; \
                if (e_ >= call && f_ >= call) break; if (t0_.fail(myws, call)) { dead = true; break; } __builtin_amdgcn_s_sleep(8); } \
            CFENCE; \
            if (STAG) { const unsigned long long s0_ = __builtin_amdgcn_s_memrealtime(); while ((unsigned)(__builtin_amdgcn_s_memrealtime() - s0_) < (ei / 7 - PPM) * STAG && ei / 7 > PPM) __builtin_amdgcn_s_sleep(1); } } \
        n = min(x_ & 0x3fffu, 8192u); xo = x_; \
        const unsigned by_ = min(((x_ >> 30) ? n : ((x_ >> 14) & 0xfffu) * 4) + 15 & ~15u, 8192u); \
        const rsrc_t rc_ = mk_rsrc(pws + OFF_COD + (unsigned)(t) * 8192, by_); \
        _Pragma("nounroll") for (unsigned r_ = 0; r_ < by_; r_ += 1024) \
            __builtin_amdgcn_raw_ptr_buffer_load_lds(rc_, (LAS void*)(L + EX_B + r_), 16, lane * 16, r_, 0, AUX_SYS); \
        __builtin_amdgcn_raw_ptr_buffer_load_lds(rs, (LAS void*)(L + EX_M), 16, lane * 16, OFF_LM + src * 1048576 + (t) * 1024, 0, AUX_SYS); }
    unsigned r0, n0 = 0, x0 = 0;
    DRAW(r0, tick);
    int t0;
    TAKE(t0, r0, 0); t0 = WT_OF(t0);
    if ((unsigned)t0 < 1024u) EX_ISSUE(t0, 0u, 0u, 0u, n0, x0);
    int t1;
    { unsigned r1; DRAW(r1, tick); TAKE(t1, r1, 0); t1 = WT_OF(t1); }
    STAMP(myws, wv, 18);
    while ((unsigned)t0 < 1024u) {
        if ((unsigned)t1 < 1024u) ENT_FETCH(t1);
        const u32x4 m = *(const LAS u32x4*)(L + EX_M + lane * 16);
        if (x0 >> 30) {
#pragma nounroll
            for (unsigned r = 0; r < n0; r += 1024) *(LAS u32x4*)(L + r + lane * 16) = *(const LAS u32x4*)(L + EX_B + r + lane * 16);
        } else {
            const unsigned G = min((n0 + 15) >> 4, L1_MAX / 16), ec = (x0 >> 26) & 15u;
            const unsigned T1 = (ec << 3) | (((ec - 1) & 15u) << 11) | (((ec - 2) & 15u) << 19) | 0x80000000u;
            const unsigned T2 = (((ec + 1) & 15u) << 3) | (((ec - 3) & 15u) << 11) | (((ec - 4) & 15u) << 19) | 0x80000000u;
            LAS unsigned char* S2 = L + EX_B + 5888;
            unsigned cd[5][4], n7[5][4], cnt2 = 0;
            const unsigned k01 = KS_(0x01010101u), k02 = KS_(0x02020202u), k03 = KS_(0x03030303u), k07 = KS_(0x07070707u), k80 = KS_(H8), kmul = KS_(0x00408102u);
#define LAYER1(i0, n) { u32x2 nn_[n]; unsigned hb_[n]; \
                _Pragma("unroll") for (int i = 0; i < n; ++i) { const unsigned g = (i0 + i) * 64 + lane; nn_[i] = *(const LAS u32x2*)(L + EX_B + g * 8); hb_[i] = *(const LAS unsigned short*)(L + EX_B + G * 8 + g * 2); } \
                SCHED; \
                _Pragma("unroll") for (int i = 0; i < n; ++i) { const unsigned hb = (i0 + i) * 64 + lane < G ? hb_[i] : 0u;        \
                    _Pragma("unroll") for (int k = 0; k < 4; ++k) { const unsigned nb = (k & 1) ? nn_[i][k >> 1] >> 4 : nn_[i][k >> 1]; \
                        const unsigned ex = __builtin_amdgcn_perm(0u, T1, ((nb >> 3) & k01) | (__builtin_amdgcn_ubfe(hb, 4 * k, 4) * kmul & k02)); \
                        cd[i0 + i][k] = (nb & k07) | ex; \
                        n7[i0 + i][k] = __builtin_amdgcn_udot4(ex & k80, 0x08040201u, 0u, false); \
                        cnt2 += __builtin_popcount(n7[i0 + i][k]); } } }
            LAYER1(0, 4);
#pragma unroll
            for (int k = 0; k < 4; ++k) cd[4][k] = n7[4][k] = 0;
            if (G > 256) LAYER1(4, 1);
            CYC(0);
            const unsigned sc2 = wave_scan(cnt2), n2 = min(__builtin_amdgcn_readlane(sc2, 63), L2_MAX), G2 = (n2 + 15) >> 4, o2 = off_l2(G), o3 = o2 + G2 * 4;
            {
                unsigned pk0 = *(const LAS unsigned*)(L + EX_B + o2 + lane * 4), pk1 = *(const LAS unsigned*)(L + EX_B + o2 + 256 + lane * 4);
                pk0 = lane < G2 ? pk0 : 0u; pk1 = 64 + lane < G2 ? pk1 : 0u;
                const unsigned x3 = *(const LAS unsigned*)(L + EX_B + o3 + lane * 4);
                *(LAS u32x2*)(L + EX_P + lane * 8) = (u32x2){(x3 << 3) & 0x78787878u, (x3 >> 1) & 0x78787878u};
                unsigned v2[8], m7[8], cnt3 = 0;
#pragma unroll
                for (int k = 0; k < 8; ++k) {
                    v2[k] = __builtin_amdgcn_perm(0u, T2, ((k < 4 ? pk0 : pk1) >> (2 * (k & 3))) & k03);
                    m7[k] = __builtin_amdgcn_udot4(v2[k] & k80, 0x08040201u, 0u, false);
                    cnt3 += __builtin_popcount(m7[k]);
                }
                unsigned o = wave_scan(cnt3) - cnt3 + (unsigned)(size_t)(L + EX_P), se[8], w0[8], w1[8], of[8];
                CFENCE;
#pragma unroll
                for (int k = 0; k < 8; ++k) {
                    se[k] = *(const LAS unsigned*)(lds + EX_LUT + (m7[k] >> 4));
                    const LAS unsigned* w = (const LAS unsigned*)(size_t)(o & ~3u); w0[k] = w[0]; w1[k] = w[1]; of[k] = o; o += __builtin_popcount(m7[k]);
                }
                SCHED;
#pragma unroll
                for (int k = 0; k < 8; ++k) v2[k] = (v2[k] & ~H8) | __builtin_amdgcn_perm(0u, __builtin_amdgcn_alignbyte(w1[k], w0[k], of[k]), se[k]);
                *(LAS u32x4*)(S2 + lane * 16) = (u32x4){v2[0], v2[1], v2[2], v2[3]};
                *(LAS u32x4*)(S2 + 1024 + lane * 16) = (u32x4){v2[4], v2[5], v2[6], v2[7]};
                CFENCE;
            }
            unsigned o = sc2 - cnt2 + (unsigned)(size_t)S2;
#define MERGE(i0, n) { unsigned se_[n][4], w0_[n][4], w1_[n][4], of_[n][4]; \
                _Pragma("unroll") for (int i = 0; i < n; ++i) _Pragma("unroll") for (int k = 0; k < 4; ++k) { \
                    se_[i][k] = *(const LAS unsigned*)(lds + EX_LUT + (n7[i0 + i][k] >> 4)); \
                    const LAS unsigned* w = (const LAS unsigned*)(size_t)(o & ~3u); w0_[i][k] = w[0]; w1_[i][k] = w[1]; of_[i][k] = o; o += __builtin_popcount(n7[i0 + i][k]); } \
                SCHED; \
                _Pragma("unroll") for (int i = 0; i < n; ++i) { u32x4 v; \
                    _Pragma("unroll") for (int k = 0; k < 4; ++k) v[k] = (cd[i0 + i][k] & ~H8) | __builtin_amdgcn_perm(0u, __builtin_amdgcn_alignbyte(w1_[i][k], w0_[i][k], of_[i][k]), se_[i][k]); \
                    *(LAS u32x4*)(L + ((i0 + i) * 64 + lane) * 16) = v; } }
            MERGE(0, 4);
            if (G > 256) MERGE(4, 1);
            CFENCE;
        }
        const unsigned cnt = popc4(m);
        unsigned off = wave_scan(cnt) - cnt + (unsigned)(size_t)L;
        WAIT_L(0);
        CYC(1);
        unsigned n1 = 0, x1 = 0;
        if ((unsigned)t1 < 1024u) {
            WAIT_V(0);
            const u32x2 ent = *(const LAS u32x2*)(L + EX_E + (t1 & 1) * 8);
            const unsigned mf = *(const LAS unsigned*)(L + EX_E + 16 + (t1 & 12));
            EX_ISSUE(t1, __builtin_amdgcn_readfirstlane(ent[0]), __builtin_amdgcn_readfirstlane(ent[1]), __builtin_amdgcn_readfirstlane(mf), n1, x1);
        }
        unsigned r2;
        DRAW(r2, tick);
        CYC(2);
        const unsigned so = out_soff(src, t0), vo = out_voff(lane);
        u32x2 sel[16]; unsigned of[16], w0[16], w1[16], w2[16];
#pragma unroll
        for (int g = 0; g < 16; ++g) {
            const unsigned b8 = ((g & 3) == 0 ? m[g >> 2] << 3 : m[g >> 2] >> (8 * (g & 3) - 3)) & 0x7f8u;
            sel[g] = *(const LAS u32x2*)(lds + EX_LUT + b8);
            const LAS unsigned* w = (const LAS unsigned*)(size_t)(off & ~3u);
            w0[g] = w[0]; w1[g] = w[1]; w2[g] = w[2]; of[g] = off;
            off += __builtin_popcount(b8);
        }
        SCHED;
        u32x4 o[8];
#pragma unroll
        for (int g = 0; g < 16; ++g) {
            const unsigned a0 = __builtin_amdgcn_alignbyte(w1[g], w0[g], of[g]), a1 = __builtin_amdgcn_alignbyte(w2[g], w1[g], of[g]);
            o[g >> 1][(g & 1) * 2] = __builtin_amdgcn_perm(a1, a0, sel[g][0]); o[g >> 1][(g & 1) * 2 + 1] = __builtin_amdgcn_perm(a1, a0, sel[g][1]);
        }
        CYC(3);
        if (dead) {
#pragma unroll
            for (int ch = 0; ch < 8; ++ch) o[ch] = (u32x4){0x7f7f7f7fu, 0x7f7f7f7fu, 0x7f7f7f7fu, 0x7f7f7f7fu};
            CFENCE;
        }
#pragma unroll
        for (int ch = 0; ch < 8; ++ch) ST128_RAW(o[ch], ro, vo, so + chunk_soff(ch), AUX_WT);
        ST128_PAD8(o);
        CYC(4);
        int t2;
        TAKE(t2, r2, 8); t2 = WT_OF(t2);
        CYC(5);
        t0 = t1; n0 = n1; x0 = x1; t1 = t2;
    }
    STAMP(myws, wv, 19);
}

__device__ __forceinline__ unsigned get_call(unsigned callv) { unsigned c; asm volatile("s_nop 0\n\tv_readfirstlane_b32 %0, %1" : "=s"(c) : "v"(callv)); return c + 1u; }

__device__ __forceinline__ void own_x(const Params& p, int j, int g, unsigned due, unsigned long long t0, int lane) {
    while ((unsigned)(__builtin_amdgcn_s_memrealtime() - t0) < due) __builtin_amdgcn_s_sleep(4);
    u32x4 v[8];
    x_load(v, mk_rsrc(p.x, X_BYTES), j, g, lane);
    img_store(v, mk_rsrc(p.my, WS_USED), false, x_img(p.rank * 8 + j, g), lane);
    WAIT_V(0);
    if (lane == 0) add_sys(p.my + x_flag(p.rank * 8 + j, g));
}

__device__ __forceinline__ void comm_role(const Params& p, LAS unsigned char* lds, int c, int wid, int lane, unsigned callv) {
    const int rank = p.rank;
    char* myws = p.my;
    const rsrc_t rs = mk_rsrc(myws, WS_USED);
    const int wv = (NGEMM + c) * 8 + wid; (void)wv;
    STAMP(myws, wv, 0);
    if (wid >= 2) {
        const rsrc_t rx = mk_rsrc(p.x, X_BYTES), rw = mk_rsrc(p.w, W_BYTES);
        const int k0 = (wid - 2) * 128 + c;
        const bool far = k0 < 448;
        const unsigned long long tk0 = __builtin_amdgcn_s_memrealtime();
        for (int u = far ? k0 : k0 - 448; u < (far ? 1792 : 768); u += far ? 448 : 320) {
            const bool ownw = !far && u >= 64 && u < 320;
            const int ux = u < 64 ? u : u - 256;
            const int j = far ? u / 224 : ownw ? (u - 64) >> 5 : ux >> 6, r = far ? u % 224 : ownw ? 288 + ((u - 64) & 31) : 224 + (ux & 63), t = rank * 8 + j;
            if (far && j == 0 && r / 7 < NPUSH) continue;
            const unsigned due = far ? (j > 1 ? (j - 1) * PACE_W : j * PACE_W1) : ownw ? j * PACE_E : j * PACE_X;
            while ((unsigned)(__builtin_amdgcn_s_memrealtime() - tk0) < due) __builtin_amdgcn_s_sleep(8);
            u32x4 v[8];
            if (r >= 224 && r < 288) {
                const int g = r - 224;
                x_load(v, rx, j, g, lane);
                pack_store(v, rs, lds + 2048 + wid * EX_BYTES, OFF_PX + (unsigned)(j * 64 + g) * UNIT, lane);
                WAIT_V(0);
                const unsigned call = get_call(callv);
                if (j < NPUF) {
                    if (lane < 8) st_sys(p.pl + OFF_PUF + ((rank * NPUF + j) * 96 + 32 + g) * 4, call);
                    if (j == 0) STAMP(myws, wv, 7);
                }
                unsigned old = 0;
                if (lane == 0) old = __hip_atomic_fetch_add((GAS unsigned*)(myws + OFF_PCNT + j * 128), 1u, __ATOMIC_RELAXED, __HIP_MEMORY_SCOPE_AGENT);
                if ((__builtin_amdgcn_readfirstlane(old) & 63) == 63 && lane < 8) st_sys(p.pl + OFF_PFLG + t * 128, call);
            } else {
                const bool own = r >= 288;
                const int e = own ? r - 288 : r / 7, d = own ? rank : (rank + 1 + r % 7) & 7;
                u32x2 w[16];
                w_load(w, rw, j, d, e >> 2, e & 1, (e >> 1) & 1, lane);
                w_transpose(v, w);
                if (own) {
                    img_store(v, rs, true, w_img(t, e), lane);
                    WAIT_V(0);
                    if (lane == 0) add_sys(myws + w_flag(t, e));
                } else {
                    const unsigned id = (unsigned)((d * 8 + j) * 32 + e);
                    pack_store(v, rs, lds + 2048 + wid * EX_BYTES, OFF_PW + id * UNIT, lane);
                    WAIT_V(0);
                    const unsigned call = get_call(callv);
                    if (j < NPUF) { if (lane == 1 + r % 7) st_sys(p.pl + OFF_PUF + ((rank * NPUF + j) * 96 + e) * 4, call); if (j == 0) STAMP(myws, wv, 6); }
                    unsigned old = 0;
                    if (lane == 0) old = __hip_atomic_fetch_add((GAS unsigned*)(myws + OFF_PCNT + (8 + d * 8 + j) * 128), 1u, __ATOMIC_RELAXED, __HIP_MEMORY_SCOPE_AGENT);
                    if (__builtin_amdgcn_readfirstlane(old) + 1 == call * (j ? 32 : 32 - NPUSH) && lane == 1 + r % 7) st_sys(p.pl + OFF_PFLG + t * 128 + 4, call);
                }
            }
            STAMP(myws, wv, far ? 1 + (j >> 2) : 3 + (j >> 2));
        }
    }
    if (wid == 1) {
        const unsigned long long tk0 = __builtin_amdgcn_s_memrealtime();
        if (NPUSH && c < 7 * PUSH_W) {
            const int d = (rank + 1 + c % 7) & 7, e0 = c / 7;
            const rsrc_t rw = mk_rsrc(p.w, W_BYTES), rd = mk_rsrc(p.peer(1 + c % 7), WS_USED);
            u32x2 w[PUSH_U ? PUSH_U : 1][16];
#pragma unroll
            for (int i = 0; i < PUSH_U; ++i) { const int e = e0 + i * PUSH_W; w_load(w[i], rw, 0, d, e >> 2, e & 1, (e >> 1) & 1, lane); }
#pragma unroll
            for (int i = 0; i < PUSH_U; ++i) {
                const int e = e0 + i * PUSH_W;
                u32x4 v[8];
                w_transpose(v, w[i]);
                img_store(v, rd, true, w_img(rank * 8, e), lane);
                WAIT_V(0);
                if (lane == 1 + c % 7) add_sys(p.pl + w_flag(rank * 8, e));
            }
            STAMP(myws, wv, 14);
        }
        for (int u = c; u < 256; u += 128) own_x(p, u >> 5, 32 + (u & 31), PACE_L0 + (u >> 5) * PACE_O, tk0, lane);
    }
    if (wid < CW) {
        const unsigned call = get_call(callv);
        const int ci = wid * 128 + c;
        if (ci < 7 * PPM) pull_role(p, lds + 2048 + wid * EX_BYTES, ci, lane, call, wv);
        if (ci < 7 * PPO) expand_role(p, lds, lds + 2048 + wid * EX_BYTES, ci, lane, call, wv);
    }
}

typedef float f32x2 __attribute__((ext_vector_type(2)));
__device__ __forceinline__ f32x4 scale4(const f32x4& v, const f32x2& s) {
    f32x2 a, b;
    asm("v_pk_mul_f32 %0, %1, %2" : "=v"(a) : "v"((f32x2){v[0], v[1]}), "v"(s));
    asm("v_pk_mul_f32 %0, %1, %2" : "=v"(b) : "v"((f32x2){v[2], v[3]}), "v"(s));
    return (f32x4){a[0], a[1], b[0], b[1]};
}
__device__ __forceinline__ void gemm_role(const Params& p, LAS unsigned char* lds, int rt, int ct, int wid, int lane, unsigned callv) {
    const int rank = p.rank;
    char* myws = p.my;
    const int tile = rt * 8 + ct;
    STAMP(myws, tile * 8 + wid, 0);
    if (tile >= 64 && wid >= 1 && wid <= 4)
        own_x(p, (tile >> 2) & 7, (3 - (tile >> 5)) * 16 + (tile & 3) * 4 + wid - 1, tile >= 96 ? ((tile >> 2) & 7) * PACE_E : PACE_O0 + ((tile >> 2) & 7) * PACE_O,
              __builtin_amdgcn_s_memrealtime(), lane);
    const unsigned call = get_call(callv);
    const int wv = tile * 8 + wid; (void)wv;
    const int wr = wid >> 2, wc = wid & 3;
    const rsrc_t rs = mk_rsrc(myws, WS_USED);
    const unsigned voff = (unsigned)wid * 2048 + (unsigned)lane * 16;
    LAS unsigned char* ldsw = lds + wid * 2048;
    LAS unsigned char* lda = lds + wr * 8192 + lane * 16;
    LAS unsigned char* ldb = lds + wc * 4096 + lane * 16;
    const unsigned seqf = (unsigned)step_of(lane, rank, rt) * 4, fx = OFF_FLAG + rt * 256, fw = OFF_FLAG + FLAG_W + ct * 256, c4 = call * 4;
    unsigned long long todo = ~0ull, rdy = 0;
    unsigned tl = 0;
    volatile LAS unsigned* tq = (volatile LAS unsigned*)(lds + LDS_RED + 160);
#define SA(b, h) (((b) * 2 + (h)) * HALFB)
#define SB(b, h) ((4 + (b) * 2 + (h)) * HALFB)
#define STAGE(bufoff, soff) do { \
        __builtin_amdgcn_raw_ptr_buffer_load_lds(rs, (LAS void*)(ldsw + (bufoff)), 16, voff, (soff), 0, AUX_SYS); \
        __builtin_amdgcn_raw_ptr_buffer_load_lds(rs, (LAS void*)(ldsw + (bufoff)), 16, voff, (soff), 1024, AUX_SYS); } while (0)
#define LDA(dst, b, h) do { _Pragma("unroll") for (int m = 0; m < 4; ++m) _Pragma("unroll") for (int k = 0; k < 2; ++k) dst[m][k] = *(const LAS bf16x8*)(lda + SA(b, h) + m * 2048 + k * 1024); } while (0)
#define LDB(dst, b, h) do { _Pragma("unroll") for (int n = 0; n < 2; ++n) _Pragma("unroll") for (int k = 0; k < 2; ++k) dst[n][k] = *(const LAS bf16x8*)(ldb + SB(b, h) + n * 2048 + k * 1024); } while (0)
#define MMA(ai, bj, At, Bt) do { __builtin_amdgcn_s_setprio(1); _Pragma("unroll") for (int m = 0; m < 4; ++m) _Pragma("unroll") for (int n = 0; n < 2; ++n) _Pragma("unroll") for (int k = 0; k < 2; ++k) \
        acc[ai][bj][m][n] = __builtin_amdgcn_mfma_f32_16x16x32_bf16(Bt[n][k], At[m][k], acc[ai][bj][m][n], 0, 0, 0); __builtin_amdgcn_s_setprio(0); } while (0)
#define OFFA(t) (OFF_XT + (unsigned)(t) * XT_STEP + (unsigned)rt * BLK)
#define OFFB(t) (OFF_WT + (unsigned)(t) * WT_STEP + (unsigned)ct * BLK)
    unsigned la = 0, lb = 0;
#define LOOK do { if (wid == 0) { WAIT_V(8); la = *(const LAS unsigned*)(lds + LDS_FLAG + lane * 4); lb = *(const LAS unsigned*)(lds + LDS_FLAG + 256 + lane * 4); } } while (0)
#define PICK(slot) do { if (wid == 0) { \
            if (todo != 0) { \
                rdy |= __builtin_amdgcn_ballot_w64(la >= c4 && lb >= c4); \
                if ((rdy & todo) == 0) { Spin t0_; \
                    for (;;) { CFENCE; \
                        const unsigned a_ = __builtin_amdgcn_raw_buffer_load_b32(rs, seqf, fx, AUX_SYS), b_ = __builtin_amdgcn_raw_buffer_load_b32(rs, seqf, fw, AUX_SYS); \
                        rdy |= __builtin_amdgcn_ballot_w64(a_ >= c4 && b_ >= c4); \
                        if ((rdy & todo) != 0) break; if (t0_.fail(myws, call)) { rdy = ~0ull; break; } } } \
                const int pos_ = __builtin_ctzll(rdy & todo); \
                todo &= ~(1ull << pos_); tl = __builtin_amdgcn_readlane(seqf, pos_) >> 2; \
            } \
            tq[slot] = tl; \
            const unsigned skip_ = (todo & ~rdy) == 0 ? 0xf0000000u : 0u;        \
            __builtin_amdgcn_raw_ptr_buffer_load_lds(rs, (LAS void*)(lds + LDS_FLAG), 4, seqf, fx + skip_, 0, AUX_SYS); \
            __builtin_amdgcn_raw_ptr_buffer_load_lds(rs, (LAS void*)(lds + LDS_FLAG + 256), 4, seqf, fw + skip_, 0, AUX_SYS); \
            SCHED; } } while (0)

    f32x4 acc[2][2][4][2];
#pragma unroll
    for (int a = 0; a < 2; ++a)
#pragma unroll
        for (int b = 0; b < 2; ++b)
#pragma unroll
            for (int m = 0; m < 4; ++m)
#pragma unroll
                for (int n = 0; n < 2; ++n) acc[a][b][m][n] = (f32x4){0.f, 0.f, 0.f, 0.f};
    bf16x8 At[4][2], B0[2][2], B1[2][2];

#define WAIT2 do { if (wid == 0) WAIT_V(10); else WAIT_V(8); WAIT_L(0); } while (0)
#define STEP2(b) { \
        LOOK; \
        LDB(B0, b, 0); LDA(At, b, 0); LDB(B1, b, 1); \
        PICK(b); \
        STAGE(SA(b ^ 1, 1), OFFA(t1) + HALFB); \
        WAIT2; BAR; \
        const unsigned tv = tq[b]; \
        MMA(0, 0, At, B0); MMA(0, 1, At, B1); BAR; SCHED; \
        t1 = __builtin_amdgcn_readfirstlane(tv); \
        LDA(At, b, 1); \
        STAGE(SB(b, 0), OFFB(t1)); STAGE(SB(b, 1), OFFB(t1) + HALFB); STAGE(SA(b, 0), OFFA(t1)); \
        WAIT2; BAR; \
        MMA(1, 0, At, B0); MMA(1, 1, At, B1); BAR; SCHED; }
    unsigned t1;
    {
        PICK(2); PICK(3);
        if (wid == 1 && lane < 8) ((LAS unsigned*)(lds + LDS_RED))[32 + lane] = 0u;
        WAIT_L(0); BAR; CFENCE;
        STAMP(myws, wv, 1);
        const unsigned t0 = __builtin_amdgcn_readfirstlane(tq[2]);
        t1 = __builtin_amdgcn_readfirstlane(tq[3]);
        STAGE(SB(0, 0), OFFB(t0)); STAGE(SB(0, 1), OFFB(t0) + HALFB); STAGE(SA(0, 0), OFFA(t0)); STAGE(SA(0, 1), OFFA(t0) + HALFB);
        STAGE(SB(1, 0), OFFB(t1)); STAGE(SB(1, 1), OFFB(t1) + HALFB); STAGE(SA(1, 0), OFFA(t1));
        WAIT_V(8); BAR;
        if (wr == 1) BAR;
    }
    for (int i = 0; i < NT; i += 2) {
        STEP2(0);
        STEP2(1);
    }
    char* pl = p.pl;
    if (wr == 0) BAR;
    STAMP(myws, wv, 10);

    const int wt = tile * 8 + wid;
    LAS unsigned* red = (LAS unsigned*)(lds + LDS_RED);
    u32x4 msk;
#pragma unroll
    for (int w = 0; w < 4; ++w) {
        unsigned mm = 0;
#pragma unroll
        for (int e = 31; e >= 0; --e) {
            const int ch = 2 * w + (e >> 4);
            mm = __builtin_amdgcn_alignbit(mm, __float_as_uint(acc[ch >> 2][(e >> 1) & 1][ch & 3][e & 1][(e >> 2) & 3]), 31);
        }
        msk[w] = ~mm;
    }
    ST128(msk, rs, lane * 16, OFF_MSK + wt * 1024, AUX_SYS);
    SCHED;
    float mx = 0.f;
#pragma unroll
    for (int a = 0; a < 2; ++a)
#pragma unroll
        for (int b = 0; b < 2; ++b)
#pragma unroll
            for (int m = 0; m < 4; ++m)
#pragma unroll
                for (int n = 0; n < 2; ++n) mx = fmaxf(fmaxf(fmaxf(fmaxf(mx, acc[a][b][m][n][0]), acc[a][b][m][n][1]), acc[a][b][m][n][2]), acc[a][b][m][n][3]);
    const unsigned wm = wave_max(__float_as_uint(mx));
    {
        unsigned old = 0;
        if (lane == 0) {
            __hip_atomic_fetch_max(red + 32, wm, __ATOMIC_RELAXED, __HIP_MEMORY_SCOPE_WORKGROUP);
            old = __hip_atomic_fetch_add(red + 33, 1u, __ATOMIC_RELAXED, __HIP_MEMORY_SCOPE_WORKGROUP);
        }
        if (__builtin_amdgcn_readfirstlane(old) == 7) {
            const unsigned long long gv = ((unsigned long long)call << 32) | __hip_atomic_load(red + 32, __ATOMIC_RELAXED, __HIP_MEMORY_SCOPE_WORKGROUP);
            char* g = pl + OFF_AMAX + (rank * NGEMM + tile) * 8;
            unsigned long long ex_;
            asm volatile("s_mov_b64 %0, exec\n\ts_mov_b64 exec, 0xff\n\tglobal_store_dwordx2 %1, %2, off sc0 sc1\n\ts_mov_b64 exec, %0\n\ts_waitcnt vmcnt(1)" : "=&s"(ex_) : "v"(g), "v"(gv) : "memory");
        } else WAIT_V(0);
        unsigned old2 = 0;
        if (lane == 0) old2 = __hip_atomic_fetch_add(red + 34 + wr, 1u, __ATOMIC_RELAXED, __HIP_MEMORY_SCOPE_WORKGROUP);
        if (__builtin_amdgcn_readfirstlane(old2) == 3 && lane < 8) st_sys(pl + OFF_MFLG + (rank * 256 + tile * 2 + wr) * 4, call);
    }
#pragma unroll
    for (int a = 0; a < 2; ++a)
#pragma unroll
        for (int b = 0; b < 2; ++b)
#pragma unroll
            for (int m = 0; m < 4; ++m)
#pragma unroll
                for (int n = 0; n < 2; ++n)
#pragma unroll
                    for (int j = 0; j < 4; ++j) { acc[a][b][m][n][j] = fmaxf(acc[a][b][m][n][j], 0.f); if (j == 3) asm volatile("" : "+v"(acc[a][b][m][n])); }
    STAMP(myws, wv, 13);
    BAR; CFENCE;
    LAS unsigned char* A = lds + wid * EP_BYTES;
    LAS unsigned char* B = A + EP_C;
    LAS unsigned char* S2 = A + EP_S;
    const unsigned cnt = popc4(msk), scan = wave_scan(cnt), Lc = __builtin_amdgcn_readlane(scan, 63);
    const bool dense = Lc > L1_MAX;
    if (!dense) {
        unsigned o = (unsigned)(size_t)A + (scan - cnt) * 4;
        const unsigned spare = (unsigned)(size_t)(A + EP_X) + lane * 4;
#pragma unroll
        for (int e = 0; e < 128; ++e) {
            const int ch = e >> 4;
            const unsigned bit = __builtin_amdgcn_ubfe(msk[e >> 5], e & 31, 1);
            *(LAS float*)(size_t)(bit ? o ^ ((o >> 4) & 0x30u) : spare) = acc[ch >> 2][(e >> 1) & 1][ch & 3][e & 1][(e >> 2) & 3];
            o += bit * 4;
        }
    }
    float ysum = 0.f;
#pragma unroll
    for (int a = 0; a < 2; ++a)
#pragma unroll
        for (int b = 0; b < 2; ++b)
#pragma unroll
            for (int m = 0; m < 4; ++m) { const f32x4 t = acc[a][b][m][0] + acc[a][b][m][1]; ysum += (t[0] + t[1]) + (t[2] + t[3]); }
#pragma unroll
    for (int sh = 32; sh >= 1; sh >>= 1) ysum += __shfl_xor(ysum, sh);
    ysum *= 1.35f / (float)max(Lc, 1u);
    if (!dense && lane < 16 && Lc + lane < L1_MAX) *(LAS float*)(A + (((Lc + lane) * 4) ^ (((Lc + lane) >> 2) & 0x30u))) = ysum;
    LAS unsigned char* lut = A + EP_T;
    if (lane < 16) *(LAS unsigned*)(lut + lane * 4) = lut_compact(lane);
    {
        if (wid == 0) {
            Spin t0;
            for (;;) {
                CFENCE;
                const unsigned g = __builtin_amdgcn_raw_buffer_load_b32(rs, (lane & 31) * 8 + 4, OFF_AMAX + (rank * NGEMM + 96) * 8, AUX_SYS);
                if (__builtin_popcountll(__builtin_amdgcn_ballot_w64(g >= call)) >= 48 || t0.fail(myws, call)) break;
                __builtin_amdgcn_s_sleep(8);
            }
        }
        BAR; CFENCE;
    }
    {
        Spin t0;
        unsigned t, tm = ld_agent(myws + OFF_TMO);
        for (;;) {
            CFENCE;
            const u32x4 g = __builtin_amdgcn_raw_buffer_load_b128(rs, lane * 16, OFF_AMAX + wid * 1024, AUX_SYS);
            t = max(g[0], g[2]);
            if (__builtin_amdgcn_ballot_w64(g[1] < call || g[3] < call) == 0) break;
            if (t0.fail(myws, call)) { tm = call; break; }
            __builtin_amdgcn_s_sleep(4);
        }
        t = wave_max(t);
        if (lane == 0) { red[8 + wid] = t; red[wid] = tm == call; }
    }
    WAIT_L(0); BAR; CFENCE;
    const float amax = __uint_as_float(wave_max(red[8 + (lane & 7)]));
    const bool dead = __builtin_amdgcn_ballot_w64(red[lane & 7] != 0) != 0;
    const float inv = dead ? __builtin_nanf("") : 448.0f / amax;
    STAMP(myws, wv, 11);
    if (wr == 0) __builtin_amdgcn_s_setprio(2);

    const f32x2 inv2 = {inv, inv};
#define QUANT(ch, q) { const int a = (ch) >> 2, m = (ch) & 3; \
        const f32x4 y0 = scale4(acc[a][0][m][0], inv2), y1 = scale4(acc[a][0][m][1], inv2), y2 = scale4(acc[a][1][m][0], inv2), y3 = scale4(acc[a][1][m][1], inv2); \
        _Pragma("unroll") for (int j = 0; j < 4; ++j) \
            q[j] = (unsigned)__builtin_amdgcn_cvt_pk_fp8_f32(y2[j], y3[j], __builtin_amdgcn_cvt_pk_fp8_f32(y0[j], y1[j], __float_as_int(y0[j]), false), true); }
    u32x4 cod[5];
    if (dense) {
#pragma unroll
        for (int r = 0; r < 9; ++r) *(LAS u32x4*)(A + r * 1024 + lane * 16) = (u32x4){0u, 0u, 0u, 0u};
        unsigned o = (unsigned)(size_t)A + scan - cnt;
#pragma unroll
        for (int ch = 0; ch < 8; ++ch) {
            u32x4 q;
            QUANT(ch, q);
#pragma unroll
            for (int j = 0; j < 4; ++j) {
                const unsigned n4 = nib4(msk[ch >> 1], (ch & 1) * 4 + j);
                const unsigned long long z = (unsigned long long)__builtin_amdgcn_perm(0u, q[j], *(const LAS unsigned*)(lut + n4)) << ((o & 3u) * 8);
                LAS unsigned* d = (LAS unsigned*)(size_t)(o & ~3u);
                __hip_atomic_fetch_or(d, (unsigned)z, __ATOMIC_RELAXED, __HIP_MEMORY_SCOPE_WAVEFRONT);
                __hip_atomic_fetch_or(d + 1, (unsigned)(z >> 32), __ATOMIC_RELAXED, __HIP_MEMORY_SCOPE_WAVEFRONT);
                o += __builtin_popcount(n4);
            }
        }
    } else {
#define CODES(i0, n) { f32x4 y_[n][4]; \
            _Pragma("unroll") for (int i = 0; i < n; ++i) _Pragma("unroll") for (int k = 0; k < 4; ++k) y_[i][k] = *(const LAS f32x4*)(A + ((i0 + i) * 64 + (i0 + i < 4 ? lane : lane & 31)) * 64 + ((k ^ (lane >> 2)) & 3) * 16); \
            SCHED; \
            _Pragma("unroll") for (int i = 0; i < n; ++i) _Pragma("unroll") for (int k = 0; k < 4; ++k) { const f32x4 y = scale4(y_[i][k], inv2); \
                cod[i0 + i][k] = (unsigned)__builtin_amdgcn_cvt_pk_fp8_f32(y[2], y[3], __builtin_amdgcn_cvt_pk_fp8_f32(y[0], y[1], __float_as_int(y[0]), false), true); } }
        CODES(0, 2); CODES(2, 2);
        cod[4] = (u32x4){0u, 0u, 0u, 0u};
        if (Lc > 4096) CODES(4, 1);
        CFENCE;
#pragma unroll
        for (int i = 0; i < 5; ++i) *(LAS u32x4*)(A + (i * 64 + lane) * 16) = cod[i];
#pragma unroll
        for (int r = 0; r < 3; ++r) *(LAS u32x4*)(S2 + r * 1024 + lane * 16) = (u32x4){0u, 0u, 0u, 0u};
    }
    CFENCE;
    CYC(0);
    const unsigned ec = (__builtin_amdgcn_readfirstlane(__builtin_amdgcn_cvt_pk_fp8_f32(fminf(ysum * inv, 448.f), 0.f, 0, false)) >> 3) & 15u;
    const unsigned E1 = (ec + 17) * 0x01010101u, G = (Lc + 15) >> 4;
    bool raw = dense || (rt * 2 + wr) * 32 + ct * 4 + wc >= 1024 - RAW_LAST;
    unsigned bytes = Lc;
    if (!raw) {
        unsigned s2[5][4], n7[5][4], cnt2 = 0;
        const unsigned k07 = KS_(0x07070707u), k08 = KS_(0x08080808u), k0f = KS_(0x0f0f0f0fu), k10 = KS_(0x10101010u), k80 = KS_(H8);
#define CLASSIFY(i0, n) { const u32x4* d_ = cod + i0; \
            _Pragma("unroll") for (int i = 0; i < n; ++i) { const unsigned g = (i0 + i) * 64 + lane, wgt = g < G ? 0x08040201u : 0u; unsigned nb[4], p_[4]; \
                _Pragma("unroll") for (int k = 0; k < 4; ++k) { \
                    const unsigned e4 = (d_[i][k] >> 3) & k0f, u = E1 - e4, t = u & k08, ui = (u | (t - (t >> 3))) & k07; \
                    p_[k] = __builtin_amdgcn_perm(0x98989898u, 0x10080098u, ui); \
                    s2[i0 + i][k] = e4 | __builtin_amdgcn_perm(0x30302010u, 0x30303000u, ui); \
                    nb[k] = (d_[i][k] & k07) | (p_[k] & k08); \
                    n7[i0 + i][k] = __builtin_amdgcn_udot4(p_[k] & k80, wgt, 0u, false); \
                    cnt2 += __builtin_popcount(n7[i0 + i][k]); } \
                if (g < G) { *(LAS u32x2*)(B + g * 8) = (u32x2){nb[0] | (nb[1] << 4), nb[2] | (nb[3] << 4)}; \
                    *(LAS unsigned short*)(B + G * 8 + g * 2) = (unsigned short)bits16(p_[0] & k10, p_[1] & k10, p_[2] & k10, p_[3] & k10, 4); } } }
        CLASSIFY(0, 4);
#pragma unroll
        for (int k = 0; k < 4; ++k) s2[4][k] = n7[4][k] = 0;
        if (G > 256) CLASSIFY(4, 1);
        CYC(1);
        const unsigned sc2 = wave_scan(cnt2), n2 = __builtin_amdgcn_readlane(sc2, 63), G2 = (n2 + 15) >> 4, o2 = off_l2(G), o3 = o2 + G2 * 4;
        raw = n2 > L2_MAX;
        if (!raw) {
            unsigned o = sc2 - cnt2 + (unsigned)(size_t)S2;
#define SQUEEZE(i0, n) { unsigned se_[n][4]; \
                _Pragma("unroll") for (int i = 0; i < n; ++i) _Pragma("unroll") for (int k = 0; k < 4; ++k) se_[i][k] = *(const LAS unsigned*)(lut + (n7[i0 + i][k] >> 5)); \
                SCHED; \
                _Pragma("unroll") for (int i = 0; i < n; ++i) _Pragma("unroll") for (int k = 0; k < 4; ++k) { \
                    const unsigned long long z = (unsigned long long)__builtin_amdgcn_perm(0u, s2[i0 + i][k], se_[i][k]) << ((o & 3u) * 8); \
                    LAS unsigned* d = (LAS unsigned*)(size_t)(o & ~3u); \
                    __hip_atomic_fetch_or(d, (unsigned)z, __ATOMIC_RELAXED, __HIP_MEMORY_SCOPE_WAVEFRONT); \
                    __hip_atomic_fetch_or(d + 1, (unsigned)(z >> 32), __ATOMIC_RELAXED, __HIP_MEMORY_SCOPE_WAVEFRONT); \
                    o += __builtin_popcount(n7[i0 + i][k]); } }
            SQUEEZE(0, 4);
            if (G > 256) SQUEEZE(4, 1);
            CFENCE;
            CYC(2);
            const u32x4 d0 = *(const LAS u32x4*)(S2 + lane * 16), d1 = *(const LAS u32x4*)(S2 + 1024 + lane * 16);
            if (lane < G2) *(LAS unsigned*)(B + o2 + lane * 4) = ((d0[0] >> 4) & 0x03030303u) | ((d0[1] >> 2) & 0x0c0c0c0cu) | (d0[2] & 0x30303030u) | ((d0[3] << 2) & 0xc0c0c0c0u);
            if (64 + lane < G2) *(LAS unsigned*)(B + o2 + 256 + lane * 4) = ((d1[0] >> 4) & 0x03030303u) | ((d1[1] >> 2) & 0x0c0c0c0cu) | (d1[2] & 0x30303030u) | ((d1[3] << 2) & 0xc0c0c0c0u);
            unsigned m5[8], se[8], cnt3 = 0;
#pragma unroll
            for (int k = 0; k < 8; ++k) {
                const unsigned d = k < 4 ? d0[k & 3] : d1[k & 3];
                m5[k] = __builtin_amdgcn_udot4(d & (d << 1) & 0x20202020u, 0x08040201u, 0u, false);
                se[k] = *(const LAS unsigned*)(lut + (m5[k] >> 3));
                cnt3 += __builtin_popcount(m5[k]);
            }
            const unsigned sc3 = wave_scan(cnt3), n3 = __builtin_amdgcn_readlane(sc3, 63);
            bytes = o3 + ((n3 + 7) >> 3) * 4;
            raw = n3 > MAXESC || bytes > Lc;
            if (!raw) {
                o = sc3 - cnt3 + (unsigned)(size_t)(A + EP_L);
#pragma unroll
                for (int k = 0; k < 8; ++k) {
                    const unsigned long long z = (unsigned long long)__builtin_amdgcn_perm(0u, k < 4 ? d0[k & 3] : d1[k & 3], se[k]) << ((o & 3u) * 8);
                    LAS unsigned* d = (LAS unsigned*)(size_t)(o & ~3u);
                    __hip_atomic_fetch_or(d, (unsigned)z, __ATOMIC_RELAXED, __HIP_MEMORY_SCOPE_WAVEFRONT);
                    __hip_atomic_fetch_or(d + 1, (unsigned)(z >> 32), __ATOMIC_RELAXED, __HIP_MEMORY_SCOPE_WAVEFRONT);
                    o += __builtin_popcount(m5[k]);
                }
                CFENCE;
                const u32x2 x3 = *(const LAS u32x2*)(A + EP_L + lane * 8);
                *(LAS unsigned*)(B + o3 + lane * 4) = (x3[0] & 0x0f0f0f0fu) | ((x3[1] << 4) & 0xf0f0f0f0u);
                CFENCE;
            }
        }
    }
    if (raw) bytes = Lc;
    CYC(3);
    {
        LAS unsigned char* S = raw ? A : B;
#pragma nounroll
        for (unsigned r = 0; r < bytes; r += 1024)
            ST128(*(const LAS u32x4*)(S + r + lane * 16), rs, lane * 16, OFF_COD + wt * 8192 + r, AUX_SYS);
    }
    if (lane == 0) red[24 + wid] = (raw ? 1u << 30 : (bytes >> 2) << 14) | (ec << 26) | Lc;
    CYC(4);
    WAIT_V(0);
    CYC(5);
    {
        unsigned old = 0;
        if (lane == 0) old = __hip_atomic_fetch_add(red + 36 + wr, 1u, __ATOMIC_RELAXED, __HIP_MEMORY_SCOPE_WORKGROUP);
        if (__builtin_amdgcn_readfirstlane(old) == 3 && lane < 32 && (lane & 7) != 0) {
            const int w = wr * 4 + (lane >> 3);
            __hip_atomic_store((GAS unsigned long long*)(pl + OFF_DONE + (rank * 1024 + tile * 8 + w) * 8),
                               ((unsigned long long)((volatile LAS unsigned*)red)[24 + w] << 32) | call, __ATOMIC_RELAXED, __HIP_MEMORY_SCOPE_SYSTEM);
        }
    }
    STAMP(myws, wv, 12);
    {
        const rsrc_t ro = mk_rsrc(p.out, OUT_BYTES);
        const unsigned so = out_soff(rank, wt), vo = out_voff(lane);
        u32x4 qv[8];
#pragma unroll
        for (int ch = 0; ch < 8; ++ch) QUANT(ch, qv[ch]);
#pragma unroll
        for (int ch = 0; ch < 8; ++ch) ST128_RAW(qv[ch], ro, vo, so + chunk_soff(ch), AUX_WT);
        ST128_PAD8(qv);
    }
}

__global__ __launch_bounds__(512, 2) void fused_kernel(Args a) {
    __shared__ __attribute__((aligned(16))) unsigned char lds_arr[LDS_BYTES];
    LAS unsigned char* lds = (LAS unsigned char*)lds_arr;
    const int wid = __builtin_amdgcn_readfirstlane(threadIdx.x >> 6), lane = threadIdx.x & 63;
    const int b = blockIdx.x, xcd = b & 7, idx = b >> 3;
    Params p;
    p.x = a.x; p.w = a.w; p.out = a.out; p.my = (char*)a.ws[0]; p.rank = a.rank; p.pl = (char*)a.ws[lane & 7];
    asm volatile("" : "+s"(p.x), "+s"(p.w), "+s"(p.out), "+s"(p.my), "+s"(p.rank));
    const unsigned callv = (unsigned)__builtin_amdgcn_raw_ptr_buffer_atomic_add_i32(1, mk_rsrc(p.my, WS_USED), lane == 0 ? OFF_CNT + (b * 8 + wid) * 4 : 0xfffffff0u, 0, 0);
    if (idx < 16) gemm_role(p, lds, (xcd >> 1) * 4 + (idx >> 2), (xcd & 1) * 4 + (idx & 3), wid, lane, callv);
    else comm_role(p, lds, (idx - 16) * 8 + xcd, wid, lane, callv);
}

extern "C" void kernel_launch(int rank, int nranks, ncclComm_t comm, hipStream_t stream,
                              void* const* d_in, const int* in_sizes, int n_in,
                              void* d_out, int out_size,
                              const int* in_dtypes, int out_dtype,
                              void* const* d_out_all, void* const* d_ws_all, size_t ws_bytes) {
    (void)comm; (void)in_dtypes; (void)out_dtype; (void)d_out_all;
    if (nranks != NR || n_in != 2 || in_sizes[0] != M_ * KS || in_sizes[1] != KS * N_ || out_size != M_ * N_ || ws_bytes < WS_USED || (unsigned)rank >= (unsigned)NR) {
        fprintf(stderr, "kernel_launch: unexpected problem shape\n");
        return;
    }
    Args a;
    a.x = d_in[0]; a.w = d_in[1]; a.out = d_out;
    const size_t shift = (ws_bytes - WS_USED) & ~(size_t)0x1fffff;
    for (int i = 0; i < NR; ++i) a.ws[i] = (char*)d_ws_all[(rank + i) % NR] + shift;
    a.rank = rank; a.pad = 0;
    fused_kernel<<<dim3(NWG), dim3(512), 0, stream>>>(a);
}
```

```cpp
#include <hip/hip_runtime.h>
#include <rccl/rccl.h>
#include <cstdio>

#define LAS __attribute__((address_space(3)))
#define GAS __attribute__((address_space(1)))
typedef short bf16x8 __attribute__((ext_vector_type(8)));
typedef float f32x4 __attribute__((ext_vector_type(4)));
typedef unsigned u32x4 __attribute__((ext_vector_type(4)));
typedef unsigned u32x2 __attribute__((ext_vector_type(2)));
typedef __amdgpu_buffer_rsrc_t rsrc_t;

constexpr int NR = 8;
constexpr int M_ = 4096, N_ = 16384;
constexpr int KS = 512;
constexpr int NS = 2048;
constexpr int NT = 64;
constexpr int NGEMM = 128;
constexpr int NWG = 256;

constexpr unsigned OFF_CNT  = 0;
constexpr unsigned OFF_TMO  = 16384;
constexpr unsigned OFF_FLAG = 65536;
constexpr unsigned FLAG_W = 4096;
constexpr unsigned OFF_AMAX = 131072;
constexpr unsigned OFF_TICK = 143360;
constexpr unsigned OFF_PCNT = 147456;
constexpr unsigned OFF_PFLG = 163840;
constexpr unsigned OFF_PUF  = 188416;
constexpr unsigned OFF_MFLG = 176128;
constexpr unsigned OFF_DONE = 786432;
constexpr unsigned OFF_MF   = OFF_DONE + 65536;
constexpr unsigned OFF_DBG  = 262144;
constexpr unsigned CTRL_BYTES = 1u << 20;
constexpr unsigned XT_STEP = 512u * 1024u, WT_STEP = 256u * 1024u, BLK = 32768u, HALFB = 16384u;
constexpr unsigned OFF_XT = CTRL_BYTES;
constexpr unsigned XT_BYTES = NT * XT_STEP;
constexpr unsigned OFF_WT = OFF_XT + XT_BYTES;
constexpr unsigned WT_BYTES = NT * WT_STEP;
constexpr unsigned UNIT = 9216;
constexpr unsigned OFF_PX = OFF_WT + WT_BYTES;
constexpr unsigned OFF_PW = OFF_PX + 512 * UNIT;
constexpr unsigned OFF_MSK = OFF_PW + 2048 * UNIT;
constexpr unsigned OFF_COD = OFF_MSK + 1024 * 1024;
constexpr unsigned OFF_LM = OFF_COD + 1024 * 8192;
constexpr unsigned WS_USED = OFF_LM + 8 * 1024 * 1024;
constexpr unsigned OUT_BYTES = (unsigned)M_ * N_;
constexpr unsigned X_BYTES = (unsigned)M_ * KS * 2, W_BYTES = (unsigned)KS * N_ * 2;

constexpr unsigned long long DEADLINE = 100000000ull;
constexpr int AUX_SYS = 17;
#define WS_END 1
constexpr int AUX_NT = 2, AUX_WT = 16;
#define PPP 15
#define PPM 16
#define PPO 55
#define CW 4
#define PUSH_W 8
#define PUSH_U 3
#define RAW_LAST 24
#define STAG 7
#define DW_AUX 0
#define SCH_S 100
#define SCH_C 1000
#define SCH_B 10
#define SCH_A 1
constexpr int NPUF = 2;
constexpr int PACE_W = 800, PACE_W1 = 150, PACE_X = 400, PACE_E = 70, PACE_O = 60, PACE_O0 = 150, PACE_L0 = 300;

constexpr unsigned LDS_STAGE = 8 * HALFB;
constexpr unsigned EP_BYTES = 19456;
constexpr unsigned EP_C = 8192, EP_S = EP_C + 5888, EP_L = EP_S + 2080, EP_T = 18432, EP_X = EP_T + 64;
constexpr unsigned LDS_FLAG = 8 * EP_BYTES;
constexpr unsigned LDS_RED = LDS_FLAG + 512;
constexpr unsigned EX_B = 8192, EX_M = EX_B + 8192, EX_P = EX_M + 1024, EX_E = EX_P + 1024, EX_BYTES = EX_E + 256, EX_LUT = 0;
constexpr unsigned LDS_BYTES = LDS_RED + 256;
static_assert(2048 + 8 * EX_BYTES <= LDS_BYTES && EP_BYTES % 1024 == 0 && EP_L + 1024 <= EP_T && EP_X + 256 <= EP_BYTES && LDS_STAGE <= LDS_FLAG && LDS_RED + 192 <= LDS_BYTES && LDS_BYTES <= 163840, "lds");
static_assert(OFF_PUF + 8 * NPUF * 384 <= OFF_DBG, "flags");
static_assert((PUSH_W * PUSH_U) % 4 == 0 && PUSH_W * PUSH_U <= 32 && 7 * PUSH_W <= NGEMM, "push");
static_assert(7 * PPO <= 128 * CW && 7 * PPM <= 128 * CW && PPP <= PPM, "waves");

struct Args {
    const void* x; const void* w; void* out;
    int rank; int pad;
    void* ws[NR];
};
struct Params {
    const void* x; const void* w; void* out; char* my; char* pl;
    int rank;
    __device__ __forceinline__ char* peer(int k) const {
        const unsigned long long a = (unsigned long long)pl;
        return (char*)(((unsigned long long)(unsigned)__builtin_amdgcn_readlane((unsigned)(a >> 32), k) << 32) | (unsigned)__builtin_amdgcn_readlane((unsigned)a, k));
    }
};

__device__ __forceinline__ rsrc_t mk_rsrc(const void* p, unsigned bytes) {
    return __builtin_amdgcn_make_buffer_rsrc((void*)p, 0, bytes, 0x00020000);
}
#define ST128(v, r, vo, so, aux) do { const u32x4 v_ = (v); __builtin_amdgcn_raw_buffer_store_b128(v_, r, vo, so, aux); asm volatile("s_nop 1" :: "v"(v_) : "memory"); } while (0)
#define ST128_RAW __builtin_amdgcn_raw_buffer_store_b128
#define ST128_PAD4(v) asm volatile("s_nop 1" :: "v"((v)[0]), "v"((v)[1]), "v"((v)[2]), "v"((v)[3]) : "memory")
#define ST128_PAD8(v) asm volatile("s_nop 1" :: "v"((v)[0]), "v"((v)[1]), "v"((v)[2]), "v"((v)[3]), "v"((v)[4]), "v"((v)[5]), "v"((v)[6]), "v"((v)[7]) : "memory")
#define WAIT_V(n) asm volatile("s_waitcnt vmcnt(" #n ")" ::: "memory")
#define WAIT_L(n) asm volatile("s_waitcnt lgkmcnt(" #n ")" ::: "memory")
#define BAR __builtin_amdgcn_s_barrier()
#define SCHED __builtin_amdgcn_sched_barrier(0)
#define CFENCE asm volatile("" ::: "memory")

#define STAMP(ws_, wv_, k_) do {} while (0)
#define CYC(i) do {} while (0)

__device__ __forceinline__ unsigned ld_agent(const void* p) { return __hip_atomic_load((const GAS unsigned*)p, __ATOMIC_RELAXED, __HIP_MEMORY_SCOPE_AGENT); }
__device__ __forceinline__ void st_agent(void* p, unsigned v) { __hip_atomic_store((GAS unsigned*)p, v, __ATOMIC_RELAXED, __HIP_MEMORY_SCOPE_AGENT); }
__device__ __forceinline__ void st_sys(void* p, unsigned v) { __hip_atomic_store((GAS unsigned*)p, v, __ATOMIC_RELAXED, __HIP_MEMORY_SCOPE_SYSTEM); }

__device__ __forceinline__ void add_sys(void* p) { asm volatile("global_atomic_add %0, %1, off sc1" :: "v"(p), "v"(1u) : "memory"); }
__device__ __forceinline__ unsigned x_flag(int t, int g) { return OFF_FLAG + (g >> 2) * 256 + t * 4; }
__device__ __forceinline__ unsigned w_flag(int t, int e) { return OFF_FLAG + FLAG_W + (e >> 2) * 256 + t * 4; }

struct Spin {
    unsigned long long t0; unsigned n;
    __device__ __forceinline__ Spin() : t0(__builtin_amdgcn_s_memrealtime()), n(0) {}
    __device__ __forceinline__ bool fail(char* myws, unsigned call) {
        if ((++n & 15u) == 0 && (__builtin_amdgcn_readfirstlane(ld_agent(myws + OFF_TMO)) == call || __builtin_amdgcn_s_memrealtime() - t0 > DEADLINE)) {
            st_agent(myws + OFF_TMO, call);
            return true;
        }
        __builtin_amdgcn_s_sleep(1);
        return false;
    }
};

#define DPP0(x, ctrl, rm) (unsigned)__builtin_amdgcn_update_dpp(0, (int)(x), ctrl, rm, 0xf, false)
__device__ __forceinline__ unsigned wave_scan(unsigned x) {
    x += DPP0(x, 0x111, 0xf); x += DPP0(x, 0x112, 0xf); x += DPP0(x, 0x114, 0xf); x += DPP0(x, 0x118, 0xf);
    x += DPP0(x, 0x142, 0xa); x += DPP0(x, 0x143, 0xc);
    return x;
}
__device__ __forceinline__ unsigned wave_max(unsigned x) {
    x = max(x, DPP0(x, 0x111, 0xf)); x = max(x, DPP0(x, 0x112, 0xf)); x = max(x, DPP0(x, 0x114, 0xf)); x = max(x, DPP0(x, 0x118, 0xf));
    x = max(x, DPP0(x, 0x142, 0xa)); x = max(x, DPP0(x, 0x143, 0xc));
    return __builtin_amdgcn_readlane(x, 63);
}

struct Sched {
    unsigned w[7][64];
    constexpr Sched() : w{} {
        for (int q = 0; q < 7; ++q) {
            int key[136] = {}, id[136] = {}, n = 0;
            for (int j = 0; j < 8; ++j) {
                int lo = 1 << 30;
                for (int rt = 0; rt < 16; ++rt) { const int k = (SCH_S * rt < SCH_C ? SCH_S * rt : SCH_C) + SCH_B * rt + 106 * (7 * j + (q + SCH_A * rt) % 7); key[n] = 2 * k + 1; id[n++] = j * 24 + 8 + rt; lo = k < lo ? k : lo; }
                key[n] = 2 * lo; id[n++] = -1 - j;
            }
            int at = 0;
            for (int o = 0; o < 136; ++o) {
                int best = -1;
                for (int i = 0; i < 136; ++i) if (key[i] >= 0 && (best < 0 || key[i] < key[best])) best = i;
                key[best] = -1;
                const int skip = id[best] == -1 ? PUSH_W * PUSH_U / 4 : 0;
                const int cnt = id[best] < 0 ? 8 - skip : 1, first = id[best] < 0 ? (-1 - id[best]) * 24 + skip : id[best];
                for (int c = 0; c < cnt; ++c, ++at) w[q][at >> 2] |= (unsigned)(first + c) << (8 * (at & 3));
            }
        }
    }
};
__constant__ const Sched SCHED_TAB;
__device__ __forceinline__ int step_of(int i, int rank, int rt) {
    if (i < 8) return rank * 8 + i;
    const int z = i - 8, j = z / 7, p = z - j * 7, q = (p + 7 * 16 * SCH_A - SCH_A * rt) % 7;
    return (((rank + 1 + q) & 7) << 3) + j;
}

__device__ __forceinline__ void x_load(u32x4 (&v)[8], rsrc_t rx, int j, int g, int lane) {
    const unsigned voff = (unsigned)(lane & 15) * (KS * 2) + (unsigned)(lane >> 4) * 16;
    const unsigned soff = (unsigned)g * 64 * (KS * 2) + (unsigned)j * 128;
#pragma unroll
    for (int r = 0; r < 4; ++r)
#pragma unroll
        for (int kk = 0; kk < 2; ++kk) v[r * 2 + kk] = __builtin_amdgcn_raw_buffer_load_b128(rx, voff + kk * 64, soff + r * 16 * (KS * 2), 0);
}
template <int AUX = AUX_NT> __device__ __forceinline__ void w_load(u32x2 (&v)[16], rsrc_t rw, int j, int d, int ct, int kk, int G, int lane) {
    const unsigned voff = (unsigned)(lane >> 4) * 8 * (N_ * 2) + (unsigned)(lane & 15) * 8;
    const unsigned soff = (unsigned)(j * 64 + kk * 32) * (N_ * 2) + (unsigned)(d * NS + ct * 256 + G * 128) * 2;
#pragma unroll
    for (int jj = 0; jj < 8; ++jj)
#pragma unroll
        for (int g = 0; g < 2; ++g) v[g * 8 + jj] = __builtin_amdgcn_raw_buffer_load_b64(rw, voff + g * 128, soff + jj * (N_ * 2), AUX);
}
__device__ __forceinline__ void w_transpose(u32x4 (&o)[8], const u32x2 (&v)[16]) {
#pragma unroll
    for (int g = 0; g < 2; ++g)
#pragma unroll
        for (int c = 0; c < 4; ++c)
#pragma unroll
            for (int dd = 0; dd < 4; ++dd)
                o[g * 4 + c][dd] = __builtin_amdgcn_perm(v[g * 8 + 2 * dd + 1][c >> 1], v[g * 8 + 2 * dd][c >> 1], (c & 1) ? 0x07060302u : 0x05040100u);
}
__device__ __forceinline__ unsigned x_img(int t, int g) { return OFF_XT + (unsigned)t * XT_STEP + (unsigned)g * 8192; }
__device__ __forceinline__ unsigned w_img(int t, int e) { return OFF_WT + (unsigned)t * WT_STEP + (unsigned)(e >> 2) * BLK + (unsigned)((e >> 1) & 1) * 8192 + (unsigned)(e & 1) * 1024; }
__device__ __forceinline__ unsigned frag_off(bool isw, unsigned f) { return isw ? ((f >> 1) & 1) * HALFB + (f >> 2) * 4096 + (f & 1) * 2048 : f * 1024; }
__device__ __forceinline__ void img_store(const u32x4 (&v)[8], rsrc_t rd, bool isw, unsigned img, int lane) {
#pragma unroll
    for (int f = 0; f < 8; ++f) ST128_RAW(v[f], rd, lane * 16, img + frag_off(isw, f), AUX_SYS);
    ST128_PAD8(v);
}
constexpr unsigned H8 = 0x80808080u, MAXESC = 512;
__device__ __forceinline__ unsigned KS_(unsigned c) { asm("" : "+s"(c)); return c; }
__device__ __forceinline__ unsigned bits16(unsigned m0, unsigned m1, unsigned m2, unsigned m3, int sh) {
    const unsigned a = __builtin_amdgcn_udot4(m0, 0x08040201u, __builtin_amdgcn_udot4(m1, 0x80402010u, 0u, false), false);
    const unsigned b = __builtin_amdgcn_udot4(m2, 0x08040201u, __builtin_amdgcn_udot4(m3, 0x80402010u, 0u, false), false);
    return (a >> sh) | (b << (8 - sh));
}
__device__ __forceinline__ unsigned esc_count(const unsigned (&bm)[4], unsigned& first) {
    const unsigned c = __builtin_popcount(bm[0]) + __builtin_popcount(bm[1]) + __builtin_popcount(bm[2]) + __builtin_popcount(bm[3]), s = wave_scan(c);
    first = s - c;
    return __builtin_amdgcn_readlane(s, 63);
}
__device__ __forceinline__ void esc_list(const unsigned (&bm)[4], unsigned at, LAS unsigned char* list, int lane) {
#pragma unroll
    for (int w = 0; w < 2; ++w) {
        unsigned long long z = ((unsigned long long)bm[2 * w + 1] << 32) | bm[2 * w];
        while (__builtin_amdgcn_ballot_w64(z != 0) != 0) {
            if (z != 0) {
                const unsigned b = __builtin_ctzll(z);
                *(LAS unsigned short*)(list + min(at, MAXESC - 1) * 2) = (unsigned short)(w * 4096 + (b >> 4) * 1024 + lane * 16 + (b & 15));
                ++at; z &= z - 1;
            }
        }
    }
}
constexpr unsigned PK_HDR = 5 * 1024 + 512, PK_RAW = 0xffffu, PK_FIRST = 62;
__device__ __forceinline__ void pack_store(const u32x4 (&v)[8], rsrc_t rs, LAS unsigned char* L, unsigned slot, int lane) {
    unsigned p0[16], e[16], sum = 0;
#pragma unroll
    for (int r = 0; r < 16; ++r) {
        const unsigned lo = __builtin_amdgcn_perm(v[r >> 1][2 * (r & 1) + 1], v[r >> 1][2 * (r & 1)], 0x06040200u);
        const unsigned hi = __builtin_amdgcn_perm(v[r >> 1][2 * (r & 1) + 1], v[r >> 1][2 * (r & 1)], 0x07050301u);
        p0[r] = (hi & H8) | (lo & ~H8);
        e[r] = ((hi << 1) & 0xfefefefeu) | ((lo >> 7) & 0x01010101u);
        sum = __builtin_amdgcn_sad_u8(e[r], 0u, sum);
    }
#pragma unroll
    for (int r = 0; r < 4; ++r) *(LAS u32x4*)(L + r * 1024 + lane * 16) = (u32x4){e[4 * r], e[4 * r + 1], e[4 * r + 2], e[4 * r + 3]};
    const int tot = (int)__builtin_amdgcn_readlane(wave_scan(sum), 63);
    const unsigned base = (unsigned)min(max((tot - 16794) >> 12, 0), 247), base4 = base * 0x01010101u;
    u32x4 p2 = {0u, 0u, 0u, 0u}; u32x2 p1 = {0u, 0u};
    unsigned ms[16];
#pragma unroll
    for (int r = 0; r < 16; ++r) {
        const unsigned z = ((e[r] | H8) - (base4 & ~H8)) ^ ((e[r] ^ ~base4) & H8);
        ms[r] = (((z & 0x78787878u) + 0x78787878u) | z) & H8;
        p2[r >> 2] |= (z & 0x03030303u) << (2 * (r & 3));
        p1[r >> 3] |= (r & 7) >= 2 ? (z & 0x04040404u) << ((r & 7) - 2) : (z & 0x04040404u) >> (2 - (r & 7));
    }
    unsigned bm[4] = {bits16(ms[0], ms[1], ms[2], ms[3], 7) | (bits16(ms[4], ms[5], ms[6], ms[7], 7) << 16),
                      bits16(ms[8], ms[9], ms[10], ms[11], 7) | (bits16(ms[12], ms[13], ms[14], ms[15], 7) << 16), 0u, 0u};
    unsigned at;
    const unsigned n = esc_count(bm, at);
    bool bad = n > 254;
    *(LAS u32x2*)(L + 4096 + lane * 8) = p1;
    if (!bad) {
        esc_list(bm, at, L + 5120, lane);
        CFENCE;
        bool b = false;
#pragma nounroll
        for (unsigned i = lane; i < n; i += 64) {
            const unsigned pos = *(const LAS unsigned short*)(L + 5120 + i * 2), ev = L[pos];
            unsigned d = base - 1 - ev;
            if (ev == base + 8) d = 14; else if (ev == 0) d = 15; else b |= d > 13;
            *(LAS unsigned short*)(L + 4096 + 516 + i * 2) = (unsigned short)(pos | (d << 12));
        }
        bad = __builtin_amdgcn_ballot_w64(b) != 0;
    }
    if (lane == 0) *(LAS unsigned*)(L + 4096 + 512) = base | ((bad ? PK_RAW : n) << 8);
    CFENCE;
    ST128(*(const LAS u32x4*)(L + 4096 + lane * 16), rs, lane * 16 + 1024, slot + 4096, AUX_SYS);
    if (!bad) {
#pragma unroll
        for (int r = 0; r < 4; ++r) ST128(((u32x4){p0[4 * r], p0[4 * r + 1], p0[4 * r + 2], p0[4 * r + 3]}), rs, lane * 16 + r * 1024, slot, AUX_SYS);
        ST128(p2, rs, lane * 16, slot + 4096, AUX_SYS);
    } else {
#pragma unroll
        for (int f = 0; f < 8; ++f) ST128(v[f], rs, lane * 16, slot + (f < 5 ? f : f + 1) * 1024, AUX_SYS);
    }
}
constexpr unsigned PL_SLOT = 8192;
struct Puller {
    rsrc_t rp, rs; char* myws; int rank, src, q, lane; unsigned call, seen, tab;
    __device__ __forceinline__ int unit_of(int tk) const { return (int)((__builtin_amdgcn_readlane(tab, tk >> 4) >> (2 * (tk & 12))) & 0xffu) * 4 + (tk & 3); }
    __device__ __forceinline__ int xg(int i) const { return i - 32; }
    __device__ __forceinline__ unsigned unit_id(int u) const { const int j = u / 96, i = u % 96; return i < 32 ? 512 + (rank * 8 + j) * 32 + i : j * 64 + xg(i); }
    __device__ __forceinline__ unsigned flag_of(int u) const { const int j = u / 96, i = u % 96; return i < 32 ? w_flag(src * 8 + j, i) : x_flag(src * 8 + j, xg(i)); }
    __device__ __forceinline__ void issue(LAS unsigned char* b, int u) {
        const int j = u / 96, i = u % 96, need = j * 2 + (i >= 32);
        if (!((seen >> need) & 1u)) {
            Spin t0;
            for (;;) {
                CFENCE;
                const unsigned sum = OFF_PFLG + (src * 8 + j) * 128 + (i >= 32 ? 0 : 4);
                const unsigned f = __builtin_amdgcn_raw_buffer_load_b32(rs, (lane & 1) && j < NPUF ? OFF_PUF + ((src * NPUF + j) * 96 + i) * 4 : sum, 0, AUX_SYS);
                if (__builtin_amdgcn_readlane(f, 0) >= call) { seen |= 1u << need; break; }
                if (__builtin_amdgcn_readlane(f, 1) >= call) break;
                if (t0.fail(myws, call)) break;
                __builtin_amdgcn_s_sleep(4);
            }
            CFENCE;
        }
        const unsigned id = unit_id(u), slot = OFF_PX + id * UNIT;
        __builtin_amdgcn_raw_ptr_buffer_load_lds(rp, (LAS void*)(b + 4096), 16, lane < 40 ? lane * 16 : 0xf0000000u, slot + 4096, 1024, AUX_SYS);
        __builtin_amdgcn_raw_ptr_buffer_load_lds(rp, (LAS void*)(b + 4096), 16, lane * 16, slot + 4096, 0, AUX_SYS);
        __builtin_amdgcn_raw_ptr_buffer_load_lds(rp, (LAS void*)b, 16, lane * 16, slot, 0, AUX_SYS);
        __builtin_amdgcn_raw_ptr_buffer_load_lds(rp, (LAS void*)b, 16, lane * 16, slot, 1024, AUX_SYS);
        __builtin_amdgcn_raw_ptr_buffer_load_lds(rp, (LAS void*)b, 16, lane * 16, slot, 2048, AUX_SYS);
        __builtin_amdgcn_raw_ptr_buffer_load_lds(rp, (LAS void*)b, 16, lane * 16, slot, 3072, AUX_SYS);
    }
    __device__ __forceinline__ void finish(LAS unsigned char* b, int u) {
        const int j = u / 96, i = u % 96, t = src * 8 + j;
        const bool isw = i < 32;
        const unsigned img = isw ? w_img(t, i) : x_img(t, xg(i));
        const unsigned h = *(const LAS unsigned*)(b + PK_HDR);
        u32x4 v[8];
#pragma unroll
        for (int r = 0; r < 5; ++r) v[r] = *(const LAS u32x4*)(b + r * 1024 + lane * 16);
        const u32x2 p1 = *(const LAS u32x2*)(b + 5120 + lane * 8);
        const unsigned meta = __builtin_amdgcn_readfirstlane(h), n = meta >> 8, base = meta & 0xffu;
        if (n <= 254) {
            if (n > PK_FIRST) {
                __builtin_amdgcn_raw_ptr_buffer_load_lds(rp, (LAS void*)(b + 4096), 16, lane * 16, OFF_PX + unit_id(u) * UNIT + 4096, 1024, AUX_SYS);
                WAIT_V(0);
            }
            const unsigned base4 = base * 0x01010101u;
            const u32x4 p2 = v[4];
            u32x4 p0[4], e[4];
#pragma unroll
            for (int r = 0; r < 16; ++r) {
                const unsigned hb = (r & 7) >= 2 ? p1[r >> 3] >> ((r & 7) - 2) : p1[r >> 3] << (2 - (r & 7));
                e[r >> 2][r & 3] = (((p2[r >> 2] >> (2 * (r & 3))) & KS_(0x03030303u)) | (hb & 0x04040404u)) + base4;
            }
#pragma unroll
            for (int r = 0; r < 4; ++r) { p0[r] = v[r]; *(LAS u32x4*)(b + r * 1024 + lane * 16) = e[r]; }
            CFENCE;
#pragma nounroll
            for (unsigned k = lane; k < n; k += 64) {
                const unsigned en = *(const LAS unsigned short*)(b + PK_HDR + 4 + k * 2), d = en >> 12;
                b[en & 0xfffu] = (unsigned char)(d == 14 ? base + 8 : d == 15 ? 0u : base - 1 - d);
            }
            CFENCE;
#pragma unroll
            for (int r = 0; r < 4; ++r) e[r] = *(const LAS u32x4*)(b + r * 1024 + lane * 16);
#pragma unroll
            for (int r = 0; r < 16; ++r) {
                const unsigned pp = p0[r >> 2][r & 3], ee = e[r >> 2][r & 3];
                const unsigned lo = (pp & ~H8) | ((ee << 7) & H8), hi = (pp & H8) | ((ee >> 1) & ~H8);
                v[r >> 1][2 * (r & 1)] = __builtin_amdgcn_perm(hi, lo, 0x05010400u); v[r >> 1][2 * (r & 1) + 1] = __builtin_amdgcn_perm(hi, lo, 0x07030602u);
            }
        } else {
            const unsigned slot = OFF_PX + unit_id(u) * UNIT;
            WAIT_L(0);
            __builtin_amdgcn_raw_ptr_buffer_load_lds(rp, (LAS void*)b, 16, lane * 16, slot + 6144, 0, AUX_SYS);
            __builtin_amdgcn_raw_ptr_buffer_load_lds(rp, (LAS void*)b, 16, lane * 16, slot + 6144, 1024, AUX_SYS);
            __builtin_amdgcn_raw_ptr_buffer_load_lds(rp, (LAS void*)b, 16, lane * 16, slot + 6144, 2048, AUX_SYS);
            WAIT_V(0);
#pragma unroll
            for (int r = 5; r < 8; ++r) v[r] = *(const LAS u32x4*)(b + (r - 5) * 1024 + lane * 16);
        }
        img_store(v, rs, isw, img, lane);
    }
};
static_assert(OFF_PW == OFF_PX + 512 * UNIT, "unit ids run through the x and the w slots");

#define DRAW(r, tick) { unsigned long long ex_; asm volatile("s_mov_b64 %1, exec\n\ts_mov_b64 exec, 1\n\tglobal_atomic_add %0, %2, %3, %4 sc0\n\ts_mov_b64 exec, %1" \
        : "=&v"(r), "=&s"(ex_) : "v"(0u), "v"(1u), "s"(tick) : "memory"); }
#define TAKE(t, r, N) { unsigned t_; asm volatile("s_waitcnt vmcnt(" #N ")\n\tv_readfirstlane_b32 %0, %1" : "=s"(t_) : "v"(r) : "memory"); t = (int)(t_ - tbase); }
constexpr int NPUSH = PUSH_W * PUSH_U, NU_IN = 768 - NPUSH, NU_ALL = NU_IN + 256;

__device__ __forceinline__ void pull_role(const Params& p, LAS unsigned char* L, int pi, int lane, unsigned call, int wv) {
    Puller P; P.rank = p.rank; P.q = pi % 7; P.src = (p.rank + 1 + P.q) & 7; P.lane = lane; P.seen = 0; P.call = call;
    P.tab = SCHED_TAB.w[P.q][lane];
    asm volatile("" : "+v"(P.tab));
    P.myws = p.my; P.rs = mk_rsrc(P.myws, WS_USED); P.rp = mk_rsrc(p.peer(1 + P.q), WS_USED);
    char* tick = P.myws + OFF_TICK + P.q * 128;
    const unsigned tbase = (call - 1) * (NU_ALL + 2 * PPM);
    unsigned r0, r1, pf = 0;
    if (pi >= 7 * PPP) {
        Spin t0;
        while ((int)(__builtin_amdgcn_readfirstlane(ld_agent(tick)) - tbase) < NU_IN) { if (t0.fail(P.myws, call)) break; __builtin_amdgcn_s_sleep(32); }
    }
    DRAW(r0, tick); DRAW(r1, tick);
    int u0, u1, sl = 0;
    TAKE(u0, r0, 0); TAKE(u1, r1, 0);
    if ((unsigned)u0 < (unsigned)NU_IN) P.issue(L, P.unit_of(u0));
    WAIT_V(0);
    STAMP(P.myws, wv, 10);
    while ((unsigned)u0 < (unsigned)NU_IN) {
        unsigned r2;
        if ((unsigned)u1 < (unsigned)NU_IN) { P.issue(L + (sl ^ 1) * PL_SLOT, P.unit_of(u1)); DRAW(r2, tick); WAIT_V(7); }
        else { DRAW(r2, tick); WAIT_V(0); }
        if (pf != 0 && lane == 0) add_sys(P.myws + pf);
        P.finish(L + sl * PL_SLOT, P.unit_of(u0));
        pf = P.flag_of(P.unit_of(u0));
        u0 = u1; sl ^= 1;
        TAKE(u1, r2, 8);
    }
    WAIT_V(0);
    if (pf != 0 && lane == 0) add_sys(P.myws + pf);
    pf = 0;
    STAMP(P.myws, wv, 9);
    unsigned long long seen = 0; int sblk = -1;
#define MASK_ISSUE(b, u) { const int mu_ = (u) - NU_IN; \
        if (!((seen >> (mu_ & 63)) & 1) || (mu_ >> 6) != sblk) { Spin t0; sblk = mu_ >> 6;        \
            for (;;) { CFENCE; const unsigned f = __builtin_amdgcn_raw_buffer_load_b32(P.rs, lane * 4, OFF_MFLG + (P.src * 256 + sblk * 64) * 4, AUX_SYS); \
                seen = __builtin_amdgcn_ballot_w64(f >= call); \
                if ((seen >> (mu_ & 63)) & 1) break; if (t0.fail(P.myws, call)) break; __builtin_amdgcn_s_sleep(8); } \
            CFENCE; } \
        __builtin_amdgcn_raw_ptr_buffer_load_lds(P.rp, (LAS void*)(b), 16, lane * 16, OFF_MSK + mu_ * 4096, 0, AUX_SYS); \
        __builtin_amdgcn_raw_ptr_buffer_load_lds(P.rp, (LAS void*)(b), 16, lane * 16, OFF_MSK + mu_ * 4096, 1024, AUX_SYS); \
        __builtin_amdgcn_raw_ptr_buffer_load_lds(P.rp, (LAS void*)(b), 16, lane * 16, OFF_MSK + mu_ * 4096, 2048, AUX_SYS); \
        __builtin_amdgcn_raw_ptr_buffer_load_lds(P.rp, (LAS void*)(b), 16, lane * 16, OFF_MSK + mu_ * 4096, 3072, AUX_SYS); }
    if ((unsigned)u0 < (unsigned)NU_ALL) MASK_ISSUE(L + sl * PL_SLOT, u0);
    WAIT_V(0);
    STAMP(P.myws, wv, 16);
    while ((unsigned)u0 < (unsigned)NU_ALL) {
        unsigned r2;
        if ((unsigned)u1 < (unsigned)NU_ALL) { MASK_ISSUE(L + (sl ^ 1) * PL_SLOT, u1); DRAW(r2, tick); WAIT_V(5); }
        else { DRAW(r2, tick); WAIT_V(0); }
        if (pf != 0 && lane == 0) st_sys(P.myws + pf, call);
        u32x4 mv[4];
#pragma unroll
        for (int r = 0; r < 4; ++r) mv[r] = *(const LAS u32x4*)(L + sl * PL_SLOT + r * 1024 + lane * 16);
#pragma unroll
        for (int r = 0; r < 4; ++r) ST128_RAW(mv[r], P.rs, lane * 16 + r * 1024, OFF_LM + P.src * 1048576 + (u0 - NU_IN) * 4096, AUX_SYS);
        ST128_PAD4(mv);
        pf = OFF_MF + (P.src * 256 + u0 - NU_IN) * 4;
        u0 = u1; sl ^= 1;
        TAKE(u1, r2, 4);
    }
    WAIT_V(0);
    if (pf != 0 && lane == 0) st_sys(P.myws + pf, call);
    STAMP(P.myws, wv, 17);
}

constexpr unsigned L1_MAX = 4608, L2_MAX = 2048;
__device__ __forceinline__ unsigned off_l2(unsigned G) { return G * 8 + ((G * 2 + 3) & ~3u); }
__device__ __forceinline__ u32x2 lut_expand(unsigned byte) { u32x2 s = {0u, 0u}; unsigned c = 0;
#pragma unroll
    for (int i = 0; i < 8; ++i) { const unsigned b = (byte >> i) & 1u; s[i >> 2] |= (b ? c : 0x0cu) << (8 * (i & 3)); c += b; } return s; }
__device__ __forceinline__ unsigned lut_compact(unsigned nib) { unsigned s = 0x0c0c0c0cu, c = 0;
#pragma unroll
    for (int i = 0; i < 4; ++i) if ((nib >> i) & 1u) { s = (s & ~(0xffu << (8 * c))) | ((unsigned)i << (8 * c)); ++c; } return s; }
__device__ __forceinline__ unsigned popc4(const u32x4& m) { return __builtin_popcount(m[0]) + __builtin_popcount(m[1]) + __builtin_popcount(m[2]) + __builtin_popcount(m[3]); }
__device__ __forceinline__ unsigned nib4(unsigned m, int i) { return (i == 0 ? m << 2 : m >> (4 * i - 2)) & 0x3cu; }
__device__ __forceinline__ unsigned out_voff(int lane) { return (unsigned)(lane & 15) * N_ + (unsigned)(lane >> 4) * 16; }
__device__ __forceinline__ unsigned out_soff(int src, int wt) {
    const int tile = wt >> 3, w = wt & 7;
    return (unsigned)((tile >> 3) * 256 + (w >> 2) * 64) * N_ + (unsigned)(src * NS + (tile & 7) * 256 + (w & 3) * 64);
}
__device__ __forceinline__ constexpr unsigned chunk_soff(int ch) { return (unsigned)((ch >> 2) * 128 + (ch & 3) * 16) * N_; }

__device__ __forceinline__ void expand_role(const Params& p, LAS unsigned char* lds, LAS unsigned char* L, int ei, int lane, unsigned call, int wv) {
    const int rank = p.rank, q = ei % 7, src = (rank + 1 + q) & 7;
    char* myws = p.my;
    char* pws = p.peer(1 + q);
    const rsrc_t rs = mk_rsrc(myws, WS_USED), ro = mk_rsrc(p.out, OUT_BYTES);
#pragma unroll
    for (int i = 0; i < 4; ++i) *(LAS u32x2*)(lds + EX_LUT + (i * 64 + lane) * 8) = lut_expand(i * 64 + lane);
    char* tick = myws + OFF_TICK + (8 + q) * 128;
    const unsigned tbase = (call - 1) * (1024 + 2 * PPO);
    bool dead = false;
#define WT_OF(k) ((unsigned)(k) < 1024u ? ((k) & ~63) | (((k) & 28) << 1) | (((k) >> 3) & 4) | ((k) & 3) : (k))
#define ENT_OFF(t) (lane == 0 ? OFF_DONE + (src * 1024 + ((t) & ~1)) * 8 : OFF_MF + (src * 256 + (((t) >> 2) & ~3)) * 4)
#define ENT_FETCH(t) { if (lane < 2) __builtin_amdgcn_raw_ptr_buffer_load_lds(rs, (LAS void*)(L + EX_E), 16, ENT_OFF(t), 0, 0, AUX_SYS); }
#define EX_ISSUE(t, e, x, f, n, xo) { unsigned e_ = (e), x_ = (x), f_ = (f); \
        if ((e_ < call || f_ < call) && !dead) { Spin t0_; \
              \
            for (;;) { CFENCE; const bool had_ = e_ >= call; \
                const u32x4 g_ = __builtin_amdgcn_raw_buffer_load_b128(rs, lane < (had_ ? 2 : 1) ? ENT_OFF(t) : 0xf0000000u, 0, AUX_SYS); \
                e_ = __builtin_amdgcn_readlane(((t) & 1) ? g_[2] : g_[0], 0); x_ = __builtin_amdgcn_readlane(((t) & 1) ? g_[3] : g_[1], 0); \
                f_ = __builtin_amdgcn_readlane(((t) & 8) ? (((t) & 4) ? g_[3] : g_[2]) : (((t) & 4) ? g_[1] : g_[0]), 1); \
                if (!had_ && e_ >= call) continue; \
                if (e_ >= call && f_ >= call) break; if (t0_.fail(myws, call)) { dead = true; break; } __builtin_amdgcn_s_sleep(8); } \
            CFENCE; \
            if (STAG) { const unsigned long long s0_ = __builtin_amdgcn_s_memrealtime(); while ((unsigned)(__builtin_amdgcn_s_memrealtime() - s0_) < (ei / 7 - PPM) * STAG && ei / 7 > PPM) __builtin_amdgcn_s_sleep(1); } } \
        n = min(x_ & 0x3fffu, 8192u); xo = x_; \
        const unsigned by_ = min(((x_ >> 30) ? n : ((x_ >> 14) & 0xfffu) * 4) + 15 & ~15u, 8192u); \
        const rsrc_t rc_ = mk_rsrc(pws + OFF_COD + (unsigned)(t) * 8192, by_); \
        _Pragma("nounroll") for (unsigned r_ = 0; r_ < by_; r_ += 1024) \
            __builtin_amdgcn_raw_ptr_buffer_load_lds(rc_, (LAS void*)(L + EX_B + r_), 16, lane * 16, r_, 0, AUX_SYS); \
        __builtin_amdgcn_raw_ptr_buffer_load_lds(rs, (LAS void*)(L + EX_M), 16, lane * 16, OFF_LM + src * 1048576 + (t) * 1024, 0, AUX_SYS); }
    unsigned r0, n0 = 0, x0 = 0;
    DRAW(r0, tick);
    int t0;
    TAKE(t0, r0, 0); t0 = WT_OF(t0);
    if ((unsigned)t0 < 1024u) EX_ISSUE(t0, 0u, 0u, 0u, n0, x0);
    int t1;
    { unsigned r1; DRAW(r1, tick); TAKE(t1, r1, 0); t1 = WT_OF(t1); }
    STAMP(myws, wv, 18);
    while ((unsigned)t0 < 1024u) {
        if ((unsigned)t1 < 1024u) ENT_FETCH(t1);
        const u32x4 m = *(const LAS u32x4*)(L + EX_M + lane * 16);
        if (x0 >> 30) {
#pragma nounroll
            for (unsigned r = 0; r < n0; r += 1024) *(LAS u32x4*)(L + r + lane * 16) = *(const LAS u32x4*)(L + EX_B + r + lane * 16);
        } else {
            const unsigned G = min((n0 + 15) >> 4, L1_MAX / 16), ec = (x0 >> 26) & 15u;
            const unsigned T1 = (ec << 3) | (((ec - 1) & 15u) << 11) | (((ec - 2) & 15u) << 19) | 0x80000000u;
            const unsigned T2 = (((ec + 1) & 15u) << 3) | (((ec - 3) & 15u) << 11) | (((ec - 4) & 15u) << 19) | 0x80000000u;
            LAS unsigned char* S2 = L + EX_B + 5888;
            unsigned cd[5][4], n7[5][4], cnt2 = 0;
            const unsigned k01 = KS_(0x01010101u), k02 = KS_(0x02020202u), k03 = KS_(0x03030303u), k07 = KS_(0x07070707u), k80 = KS_(H8), kmul = KS_(0x00408102u);
#define LAYER1(i0, n) { u32x2 nn_[n]; unsigned hb_[n]; \
                _Pragma("unroll") for (int i = 0; i < n; ++i) { const unsigned g = (i0 + i) * 64 + lane; nn_[i] = *(const LAS u32x2*)(L + EX_B + g * 8); hb_[i] = *(const LAS unsigned short*)(L + EX_B + G * 8 + g * 2); } \
                SCHED; \
                _Pragma("unroll") for (int i = 0; i < n; ++i) { const unsigned hb = (i0 + i) * 64 + lane < G ? hb_[i] : 0u;        \
                    _Pragma("unroll") for (int k = 0; k < 4; ++k) { const unsigned nb = (k & 1) ? nn_[i][k >> 1] >> 4 : nn_[i][k >> 1]; \
                        const unsigned ex = __builtin_amdgcn_perm(0u, T1, ((nb >> 3) & k01) | (__builtin_amdgcn_ubfe(hb, 4 * k, 4) * kmul & k02)); \
                        cd[i0 + i][k] = (nb & k07) | ex; \
                        n7[i0 + i][k] = __builtin_amdgcn_udot4(ex & k80, 0x08040201u, 0u, false); \
                        cnt2 += __builtin_popcount(n7[i0 + i][k]); } } }
            LAYER1(0, 4);
#pragma unroll
            for (int k = 0; k < 4; ++k) cd[4][k] = n7[4][k] = 0;
            if (G > 256) LAYER1(4, 1);
            CYC(0);
            const unsigned sc2 = wave_scan(cnt2), n2 = min(__builtin_amdgcn_readlane(sc2, 63), L2_MAX), G2 = (n2 + 15) >> 4, o2 = off_l2(G), o3 = o2 + G2 * 4;
            {
                unsigned pk0 = *(const LAS unsigned*)(L + EX_B + o2 + lane * 4), pk1 = *(const LAS unsigned*)(L + EX_B + o2 + 256 + lane * 4);
                pk0 = lane < G2 ? pk0 : 0u; pk1 = 64 + lane < G2 ? pk1 : 0u;
                const unsigned x3 = *(const LAS unsigned*)(L + EX_B + o3 + lane * 4);
                *(LAS u32x2*)(L + EX_P + lane * 8) = (u32x2){(x3 << 3) & 0x78787878u, (x3 >> 1) & 0x78787878u};
                unsigned v2[8], m7[8], cnt3 = 0;
#pragma unroll
                for (int k = 0; k < 8; ++k) {
                    v2[k] = __builtin_amdgcn_perm(0u, T2, ((k < 4 ? pk0 : pk1) >> (2 * (k & 3))) & k03);
                    m7[k] = __builtin_amdgcn_udot4(v2[k] & k80, 0x08040201u, 0u, false);
                    cnt3 += __builtin_popcount(m7[k]);
                }
                unsigned o = wave_scan(cnt3) - cnt3 + (unsigned)(size_t)(L + EX_P), se[8], w0[8], w1[8], of[8];
                CFENCE;
#pragma unroll
                for (int k = 0; k < 8; ++k) {
                    se[k] = *(const LAS unsigned*)(lds + EX_LUT + (m7[k] >> 4));
                    const LAS unsigned* w = (const LAS unsigned*)(size_t)(o & ~3u); w0[k] = w[0]; w1[k] = w[1]; of[k] = o; o += __builtin_popcount(m7[k]);
                }
                SCHED;
#pragma unroll
                for (int k = 0; k < 8; ++k) v2[k] = (v2[k] & ~H8) | __builtin_amdgcn_perm(0u, __builtin_amdgcn_alignbyte(w1[k], w0[k], of[k]), se[k]);
                *(LAS u32x4*)(S2 + lane * 16) = (u32x4){v2[0], v2[1], v2[2], v2[3]};
                *(LAS u32x4*)(S2 + 1024 + lane * 16) = (u32x4){v2[4], v2[5], v2[6], v2[7]};
                CFENCE;
            }
            unsigned o = sc2 - cnt2 + (unsigned)(size_t)S2;
#define MERGE(i0, n) { unsigned se_[n][4], w0_[n][4], w1_[n][4], of_[n][4]; \
                _Pragma("unroll") for (int i = 0; i < n; ++i) _Pragma("unroll") for (int k = 0; k < 4; ++k) { \
                    se_[i][k] = *(const LAS unsigned*)(lds + EX_LUT + (n7[i0 + i][k] >> 4)); \
                    const LAS unsigned* w = (const LAS unsigned*)(size_t)(o & ~3u); w0_[i][k] = w[0]; w1_[i][k] = w[1]; of_[i][k] = o; o += __builtin_popcount(n7[i0 + i][k]); } \
                SCHED; \
                _Pragma("unroll") for (int i = 0; i < n; ++i) { u32x4 v; \
                    _Pragma("unroll") for (int k = 0; k < 4; ++k) v[k] = (cd[i0 + i][k] & ~H8) | __builtin_amdgcn_perm(0u, __builtin_amdgcn_alignbyte(w1_[i][k], w0_[i][k], of_[i][k]), se_[i][k]); \
                    *(LAS u32x4*)(L + ((i0 + i) * 64 + lane) * 16) = v; } }
            MERGE(0, 4);
            if (G > 256) MERGE(4, 1);
            CFENCE;
        }
        const unsigned cnt = popc4(m);
        unsigned off = wave_scan(cnt) - cnt + (unsigned)(size_t)L;
        WAIT_L(0);
        CYC(1);
        unsigned n1 = 0, x1 = 0;
        if ((unsigned)t1 < 1024u) {
            WAIT_V(0);
            const u32x2 ent = *(const LAS u32x2*)(L + EX_E + (t1 & 1) * 8);
            const unsigned mf = *(const LAS unsigned*)(L + EX_E + 16 + (t1 & 12));
            EX_ISSUE(t1, __builtin_amdgcn_readfirstlane(ent[0]), __builtin_amdgcn_readfirstlane(ent[1]), __builtin_amdgcn_readfirstlane(mf), n1, x1);
        }
        unsigned r2;
        DRAW(r2, tick);
        CYC(2);
        const unsigned so = out_soff(src, t0), vo = out_voff(lane);
        u32x2 sel[16]; unsigned of[16], w0[16], w1[16], w2[16];
#pragma unroll
        for (int g = 0; g < 16; ++g) {
            const unsigned b8 = ((g & 3) == 0 ? m[g >> 2] << 3 : m[g >> 2] >> (8 * (g & 3) - 3)) & 0x7f8u;
            sel[g] = *(const LAS u32x2*)(lds + EX_LUT + b8);
            const LAS unsigned* w = (const LAS unsigned*)(size_t)(off & ~3u);
            w0[g] = w[0]; w1[g] = w[1]; w2[g] = w[2]; of[g] = off;
            off += __builtin_popcount(b8);
        }
        SCHED;
        u32x4 o[8];
#pragma unroll
        for (int g = 0; g < 16; ++g) {
            const unsigned a0 = __builtin_amdgcn_alignbyte(w1[g], w0[g], of[g]), a1 = __builtin_amdgcn_alignbyte(w2[g], w1[g], of[g]);
            o[g >> 1][(g & 1) * 2] = __builtin_amdgcn_perm(a1, a0, sel[g][0]); o[g >> 1][(g & 1) * 2 + 1] = __builtin_amdgcn_perm(a1, a0, sel[g][1]);
        }
        CYC(3);
        if (dead) {
#pragma unroll
            for (int ch = 0; ch < 8; ++ch) o[ch] = (u32x4){0x7f7f7f7fu, 0x7f7f7f7fu, 0x7f7f7f7fu, 0x7f7f7f7fu};
            CFENCE;
        }
#pragma unroll
        for (int ch = 0; ch < 8; ++ch) ST128_RAW(o[ch], ro, vo, so + chunk_soff(ch), AUX_WT);
        ST128_PAD8(o);
        CYC(4);
        int t2;
        TAKE(t2, r2, 8); t2 = WT_OF(t2);
        CYC(5);
        t0 = t1; n0 = n1; x0 = x1; t1 = t2;
    }
    STAMP(myws, wv, 19);
}

__device__ __forceinline__ unsigned get_call(unsigned callv) { unsigned c; asm volatile("s_nop 0\n\tv_readfirstlane_b32 %0, %1" : "=s"(c) : "v"(callv)); return c + 1u; }

__device__ __forceinline__ void own_x(const Params& p, int j, int g, unsigned due, unsigned long long t0, int lane) {
    if (j < 2) return;
    while ((unsigned)(__builtin_amdgcn_s_memrealtime() - t0) < due) __builtin_amdgcn_s_sleep(4);
    u32x4 v[8];
    x_load(v, mk_rsrc(p.x, X_BYTES), j, g, lane);
    img_store(v, mk_rsrc(p.my, WS_USED), false, x_img(p.rank * 8 + j, g), lane);
    WAIT_V(0);
    if (lane == 0) add_sys(p.my + x_flag(p.rank * 8 + j, g));
}

__device__ __forceinline__ void comm_role(const Params& p, LAS unsigned char* lds, int c, int wid, int lane, unsigned callv) {
    const int rank = p.rank;
    char* myws = p.my;
    const rsrc_t rs = mk_rsrc(myws, WS_USED);
    const int wv = (NGEMM + c) * 8 + wid; (void)wv;
    STAMP(myws, wv, 0);
    if (wid >= 2) {
        const rsrc_t rx = mk_rsrc(p.x, X_BYTES), rw = mk_rsrc(p.w, W_BYTES);
        const int k0 = (wid - 2) * 128 + c;
        const bool far = k0 < 448;
        const unsigned long long tk0 = __builtin_amdgcn_s_memrealtime();
        for (int u = far ? k0 : k0 - 448; u < (far ? 1792 : 768); u += far ? 448 : 320) {
            const bool ownw = !far && u >= 64 && u < 320;
            const int ux = u < 64 ? u : u - 256;
            const int j = far ? u / 224 : ownw ? (u - 64) >> 5 : ux >> 6, r = far ? u % 224 : ownw ? 288 + ((u - 64) & 31) : 224 + (ux & 63), t = rank * 8 + j;
            if (far && j == 0 && r / 7 < NPUSH) continue;
            if (ownw && j < 2) continue;
            const unsigned due = far ? (j > 1 ? (j - 1) * PACE_W : j * PACE_W1) : ownw ? j * PACE_E : j * PACE_X;
            while ((unsigned)(__builtin_amdgcn_s_memrealtime() - tk0) < due) __builtin_amdgcn_s_sleep(8);
            u32x4 v[8];
            if (r >= 224 && r < 288) {
                const int g = r - 224;
                x_load(v, rx, j, g, lane);
                pack_store(v, rs, lds + 2048 + wid * EX_BYTES, OFF_PX + (unsigned)(j * 64 + g) * UNIT, lane);
                WAIT_V(0);
                const unsigned call = get_call(callv);
                if (j < NPUF) {
                    if (lane < 8) st_sys(p.pl + OFF_PUF + ((rank * NPUF + j) * 96 + 32 + g) * 4, call);
                    if (j == 0) STAMP(myws, wv, 7);
                }
                unsigned old = 0;
                if (lane == 0) old = __hip_atomic_fetch_add((GAS unsigned*)(myws + OFF_PCNT + j * 128), 1u, __ATOMIC_RELAXED, __HIP_MEMORY_SCOPE_AGENT);
                if ((__builtin_amdgcn_readfirstlane(old) & 63) == 63 && lane < 8) st_sys(p.pl + OFF_PFLG + t * 128, call);
            } else {
                const bool own = r >= 288;
                const int e = own ? r - 288 : r / 7, d = own ? rank : (rank + 1 + r % 7) & 7;
                u32x2 w[16];
                w_load(w, rw, j, d, e >> 2, e & 1, (e >> 1) & 1, lane);
                w_transpose(v, w);
                if (own) {
                    img_store(v, rs, true, w_img(t, e), lane);
                    WAIT_V(0);
                    if (lane == 0) add_sys(myws + w_flag(t, e));
                } else {
                    const unsigned id = (unsigned)((d * 8 + j) * 32 + e);
                    pack_store(v, rs, lds + 2048 + wid * EX_BYTES, OFF_PW + id * UNIT, lane);
                    WAIT_V(0);
                    const unsigned call = get_call(callv);
                    if (j < NPUF) { if (lane == 1 + r % 7) st_sys(p.pl + OFF_PUF + ((rank * NPUF + j) * 96 + e) * 4, call); if (j == 0) STAMP(myws, wv, 6); }
                    unsigned old = 0;
                    if (lane == 0) old = __hip_atomic_fetch_add((GAS unsigned*)(myws + OFF_PCNT + (8 + d * 8 + j) * 128), 1u, __ATOMIC_RELAXED, __HIP_MEMORY_SCOPE_AGENT);
                    if (__builtin_amdgcn_readfirstlane(old) + 1 == call * (j ? 32 : 32 - NPUSH) && lane == 1 + r % 7) st_sys(p.pl + OFF_PFLG + t * 128 + 4, call);
                }
            }
            STAMP(myws, wv, far ? 1 + (j >> 2) : 3 + (j >> 2));
        }
    }
    if (wid == 1) {
        const unsigned long long tk0 = __builtin_amdgcn_s_memrealtime();
        if (NPUSH && c < 7 * PUSH_W) {
            const int d = (rank + 1 + c % 7) & 7, e0 = c / 7;
            const rsrc_t rw = mk_rsrc(p.w, W_BYTES), rd = mk_rsrc(p.peer(1 + c % 7), WS_USED);
            u32x2 w[PUSH_U ? PUSH_U : 1][16];
#pragma unroll
            for (int i = 0; i < PUSH_U; ++i) { const int e = e0 + i * PUSH_W; w_load(w[i], rw, 0, d, e >> 2, e & 1, (e >> 1) & 1, lane); }
#pragma unroll
            for (int i = 0; i < PUSH_U; ++i) {
                const int e = e0 + i * PUSH_W;
                u32x4 v[8];
                w_transpose(v, w[i]);
                img_store(v, rd, true, w_img(rank * 8, e), lane);
                WAIT_V(0);
                if (lane == 1 + c % 7) add_sys(p.pl + w_flag(rank * 8, e));
            }
            STAMP(myws, wv, 14);
        }
        for (int u = c; u < 256; u += 128) own_x(p, u >> 5, 32 + (u & 31), PACE_L0 + (u >> 5) * PACE_O, tk0, lane);
    }
    if (wid < CW) {
        const unsigned call = get_call(callv);
        const int ci = wid * 128 + c;
        if (ci < 7 * PPM) pull_role(p, lds + 2048 + wid * EX_BYTES, ci, lane, call, wv);
        if (ci < 7 * PPO) expand_role(p, lds, lds + 2048 + wid * EX_BYTES, ci, lane, call, wv);
    }
}

typedef float f32x2 __attribute__((ext_vector_type(2)));
__device__ __forceinline__ f32x4 scale4(const f32x4& v, const f32x2& s) {
    f32x2 a, b;
    asm("v_pk_mul_f32 %0, %1, %2" : "=v"(a) : "v"((f32x2){v[0], v[1]}), "v"(s));
    asm("v_pk_mul_f32 %0, %1, %2" : "=v"(b) : "v"((f32x2){v[2], v[3]}), "v"(s));
    return (f32x4){a[0], a[1], b[0], b[1]};
}
__device__ __forceinline__ void gemm_role(const Params& p, LAS unsigned char* lds, int rt, int ct, int wid, int lane, unsigned callv) {
    const int rank = p.rank;
    char* myws = p.my;
    const int tile = rt * 8 + ct;
    STAMP(myws, tile * 8 + wid, 0);
    if (tile >= 64 && wid >= 1 && wid <= 4)
        own_x(p, (tile >> 2) & 7, (3 - (tile >> 5)) * 16 + (tile & 3) * 4 + wid - 1, tile >= 96 ? ((tile >> 2) & 7) * PACE_E : PACE_O0 + ((tile >> 2) & 7) * PACE_O,
              __builtin_amdgcn_s_memrealtime(), lane);
    const int wr = wid >> 2, wc = wid & 3;
    LAS unsigned char* dst = lds + 2 * HALFB + lane * 16 + (wr == 0 ? wid * 8192 : 4 * HALFB + (wc >> 1) * 8192 + (wc & 1) * 1024);
    u32x4 dv[8];
    {
        u32x4 v0[8];
        if (wr == 0) {
            x_load(v0, mk_rsrc(p.x, X_BYTES), 0, rt * 4 + wid, lane); x_load(dv, mk_rsrc(p.x, X_BYTES), 1, rt * 4 + wid, lane);
        } else {
            u32x2 w[2][16];
            w_load<DW_AUX>(w[0], mk_rsrc(p.w, W_BYTES), 0, rank, ct, wc & 1, wc >> 1, lane); w_load<DW_AUX>(w[1], mk_rsrc(p.w, W_BYTES), 1, rank, ct, wc & 1, wc >> 1, lane);
            w_transpose(v0, w[0]); w_transpose(dv, w[1]);
        }
#pragma unroll
        for (int f = 0; f < 8; ++f) *(LAS u32x4*)(dst + frag_off(wr == 1, f)) = v0[f];
    }
    const unsigned call = get_call(callv);
    const int wv = tile * 8 + wid; (void)wv;
    const rsrc_t rs = mk_rsrc(myws, WS_USED);
    const unsigned voff = (unsigned)wid * 2048 + (unsigned)lane * 16;
    LAS unsigned char* ldsw = lds + wid * 2048;
    LAS unsigned char* lda = lds + wr * 8192 + lane * 16;
    LAS unsigned char* ldb = lds + wc * 4096 + lane * 16;
    const unsigned seqf = (unsigned)step_of(lane, rank, rt) * 4, fx = OFF_FLAG + rt * 256, fw = OFF_FLAG + FLAG_W + ct * 256, c4 = call * 4;
    unsigned long long todo = ~0ull, rdy = 0;
    unsigned tl = 0;
    volatile LAS unsigned* tq = (volatile LAS unsigned*)(lds + LDS_RED + 160);
#define SA(b, h) (((b) * 2 + (h)) * HALFB)
#define SB(b, h) ((4 + (b) * 2 + (h)) * HALFB)
#define STAGE(bufoff, soff) do { \
        __builtin_amdgcn_raw_ptr_buffer_load_lds(rs, (LAS void*)(ldsw + (bufoff)), 16, voff, (soff), 0, AUX_SYS); \
        __builtin_amdgcn_raw_ptr_buffer_load_lds(rs, (LAS void*)(ldsw + (bufoff)), 16, voff, (soff), 1024, AUX_SYS); } while (0)
#define LDA(dst, b, h) do { _Pragma("unroll") for (int m = 0; m < 4; ++m) _Pragma("unroll") for (int k = 0; k < 2; ++k) dst[m][k] = *(const LAS bf16x8*)(lda + SA(b, h) + m * 2048 + k * 1024); } while (0)
#define LDB(dst, b, h) do { _Pragma("unroll") for (int n = 0; n < 2; ++n) _Pragma("unroll") for (int k = 0; k < 2; ++k) dst[n][k] = *(const LAS bf16x8*)(ldb + SB(b, h) + n * 2048 + k * 1024); } while (0)
#define MMA(ai, bj, At, Bt) do { __builtin_amdgcn_s_setprio(1); _Pragma("unroll") for (int m = 0; m < 4; ++m) _Pragma("unroll") for (int n = 0; n < 2; ++n) _Pragma("unroll") for (int k = 0; k < 2; ++k) \
        acc[ai][bj][m][n] = __builtin_amdgcn_mfma_f32_16x16x32_bf16(Bt[n][k], At[m][k], acc[ai][bj][m][n], 0, 0, 0); __builtin_amdgcn_s_setprio(0); } while (0)
#define OFFA(t) (OFF_XT + (unsigned)(t) * XT_STEP + (unsigned)rt * BLK)
#define OFFB(t) (OFF_WT + (unsigned)(t) * WT_STEP + (unsigned)ct * BLK)
    unsigned la = 0, lb = 0;
#define LOOK do { if (wid == 0) { WAIT_V(8); la = *(const LAS unsigned*)(lds + LDS_FLAG + lane * 4); lb = *(const LAS unsigned*)(lds + LDS_FLAG + 256 + lane * 4); } } while (0)
#define PICK(slot) do { if (wid == 0) { \
            if (todo != 0) { \
                rdy |= __builtin_amdgcn_ballot_w64(la >= c4 && lb >= c4); \
                if ((rdy & todo) == 0) { Spin t0_; \
                    for (;;) { CFENCE; \
                        const unsigned a_ = __builtin_amdgcn_raw_buffer_load_b32(rs, seqf, fx, AUX_SYS), b_ = __builtin_amdgcn_raw_buffer_load_b32(rs, seqf, fw, AUX_SYS); \
                        rdy |= __builtin_amdgcn_ballot_w64(a_ >= c4 && b_ >= c4); \
                        if ((rdy & todo) != 0) break; if (t0_.fail(myws, call)) { rdy = ~0ull; break; } } } \
                const int pos_ = __builtin_ctzll(rdy & todo); \
                todo &= ~(1ull << pos_); tl = __builtin_amdgcn_readlane(seqf, pos_) >> 2; \
            } \
            tq[slot] = tl; \
            const unsigned skip_ = (todo & ~rdy) == 0 ? 0xf0000000u : 0u;        \
            __builtin_amdgcn_raw_ptr_buffer_load_lds(rs, (LAS void*)(lds + LDS_FLAG), 4, seqf, fx + skip_, 0, AUX_SYS); \
            __builtin_amdgcn_raw_ptr_buffer_load_lds(rs, (LAS void*)(lds + LDS_FLAG + 256), 4, seqf, fw + skip_, 0, AUX_SYS); \
            SCHED; } } while (0)

    f32x4 acc[2][2][4][2];
#pragma unroll
    for (int a = 0; a < 2; ++a)
#pragma unroll
        for (int b = 0; b < 2; ++b)
#pragma unroll
            for (int m = 0; m < 4; ++m)
#pragma unroll
                for (int n = 0; n < 2; ++n) acc[a][b][m][n] = (f32x4){0.f, 0.f, 0.f, 0.f};
    bf16x8 At[4][2], B0[2][2], B1[2][2];

#define WAIT2 do { if (wid == 0) WAIT_V(10); else WAIT_V(8); WAIT_L(0); } while (0)
#define STEP2(b) { \
        LOOK; \
        LDB(B0, b, 0); LDA(At, b, 0); LDB(B1, b, 1); \
        PICK(b); \
        STAGE(SA(b ^ 1, 1), OFFA(t1) + HALFB); \
        WAIT2; BAR; \
        const unsigned tv = tq[b]; \
        MMA(0, 0, At, B0); MMA(0, 1, At, B1); BAR; SCHED; \
        t1 = __builtin_amdgcn_readfirstlane(tv); \
        LDA(At, b, 1); \
        STAGE(SB(b, 0), OFFB(t1)); STAGE(SB(b, 1), OFFB(t1) + HALFB); STAGE(SA(b, 0), OFFA(t1)); \
        WAIT2; BAR; \
        MMA(1, 0, At, B0); MMA(1, 1, At, B1); BAR; SCHED; }
    unsigned t1;
    {
#define PLAIN_STEP { LDB(B0, 1, 0); LDA(At, 1, 0); LDB(B1, 1, 1); MMA(0, 0, At, B0); MMA(0, 1, At, B1); LDA(At, 1, 1); MMA(1, 0, At, B0); MMA(1, 1, At, B1); }
        todo = ~3ull;
        if (wid == 1 && lane < 8) ((LAS unsigned*)(lds + LDS_RED))[32 + lane] = 0u;
        WAIT_L(0); BAR; CFENCE;
        PLAIN_STEP;
        WAIT_L(0); BAR; CFENCE;
#pragma unroll
        for (int f = 0; f < 8; ++f) *(LAS u32x4*)(dst + frag_off(wr == 1, f)) = dv[f];
        PICK(2);
        WAIT_L(0); BAR; CFENCE;
        STAMP(myws, wv, 1);
        const unsigned t0 = __builtin_amdgcn_readfirstlane(tq[2]);
        STAGE(SB(0, 0), OFFB(t0)); STAGE(SB(0, 1), OFFB(t0) + HALFB); STAGE(SA(0, 0), OFFA(t0)); STAGE(SA(0, 1), OFFA(t0) + HALFB);
        SCHED;
        PLAIN_STEP;
        PICK(3);
        WAIT_L(0); BAR; CFENCE;
        STAMP(myws, wv, 14);
        t1 = __builtin_amdgcn_readfirstlane(tq[3]);
        STAGE(SB(1, 0), OFFB(t1)); STAGE(SB(1, 1), OFFB(t1) + HALFB); STAGE(SA(1, 0), OFFA(t1));
        WAIT_V(6); BAR;
        if (wr == 1) BAR;
    }
    for (int i = 2; i < NT; i += 2) {
        STEP2(0);
        STEP2(1);
    }
    char* pl = p.pl;
    if (wr == 0) BAR;
    STAMP(myws, wv, 10);

    const int wt = tile * 8 + wid;
    LAS unsigned* red = (LAS unsigned*)(lds + LDS_RED);
    u32x4 msk;
#pragma unroll
    for (int w = 0; w < 4; ++w) {
        unsigned mm = 0;
#pragma unroll
        for (int e = 31; e >= 0; --e) {
            const int ch = 2 * w + (e >> 4);
            mm = __builtin_amdgcn_alignbit(mm, __float_as_uint(acc[ch >> 2][(e >> 1) & 1][ch & 3][e & 1][(e >> 2) & 3]), 31);
        }
        msk[w] = ~mm;
    }
    ST128(msk, rs, lane * 16, OFF_MSK + wt * 1024, AUX_SYS);
    SCHED;
    float mx = 0.f;
#pragma unroll
    for (int a = 0; a < 2; ++a)
#pragma unroll
        for (int b = 0; b < 2; ++b)
#pragma unroll
            for (int m = 0; m < 4; ++m)
#pragma unroll
                for (int n = 0; n < 2; ++n) mx = fmaxf(fmaxf(fmaxf(fmaxf(mx, acc[a][b][m][n][0]), acc[a][b][m][n][1]), acc[a][b][m][n][2]), acc[a][b][m][n][3]);
    const unsigned wm = wave_max(__float_as_uint(mx));
    {
        unsigned old = 0;
        if (lane == 0) {
            __hip_atomic_fetch_max(red + 32, wm, __ATOMIC_RELAXED, __HIP_MEMORY_SCOPE_WORKGROUP);
            old = __hip_atomic_fetch_add(red + 33, 1u, __ATOMIC_RELAXED, __HIP_MEMORY_SCOPE_WORKGROUP);
        }
        if (__builtin_amdgcn_readfirstlane(old) == 7) {
            const unsigned long long gv = ((unsigned long long)call << 32) | __hip_atomic_load(red + 32, __ATOMIC_RELAXED, __HIP_MEMORY_SCOPE_WORKGROUP);
            char* g = pl + OFF_AMAX + (rank * NGEMM + tile) * 8;
            unsigned long long ex_;
            asm volatile("s_mov_b64 %0, exec\n\ts_mov_b64 exec, 0xff\n\tglobal_store_dwordx2 %1, %2, off sc0 sc1\n\ts_mov_b64 exec, %0\n\ts_waitcnt vmcnt(1)" : "=&s"(ex_) : "v"(g), "v"(gv) : "memory");
        } else WAIT_V(0);
        unsigned old2 = 0;
        if (lane == 0) old2 = __hip_atomic_fetch_add(red + 34 + wr, 1u, __ATOMIC_RELAXED, __HIP_MEMORY_SCOPE_WORKGROUP);
        if (__builtin_amdgcn_readfirstlane(old2) == 3 && lane < 8) st_sys(pl + OFF_MFLG + (rank * 256 + tile * 2 + wr) * 4, call);
    }
#pragma unroll
    for (int a = 0; a < 2; ++a)
#pragma unroll
        for (int b = 0; b < 2; ++b)
#pragma unroll
            for (int m = 0; m < 4; ++m)
#pragma unroll
                for (int n = 0; n < 2; ++n)
#pragma unroll
                    for (int j = 0; j < 4; ++j) { acc[a][b][m][n][j] = fmaxf(acc[a][b][m][n][j], 0.f); if (j == 3) asm volatile("" : "+v"(acc[a][b][m][n])); }
    STAMP(myws, wv, 13);
    BAR; CFENCE;
    LAS unsigned char* A = lds + wid * EP_BYTES;
    LAS unsigned char* B = A + EP_C;
    LAS unsigned char* S2 = A + EP_S;
    const unsigned cnt = popc4(msk), scan = wave_scan(cnt), Lc = __builtin_amdgcn_readlane(scan, 63);
    const bool dense = Lc > L1_MAX;
    if (!dense) {
        unsigned o = (unsigned)(size_t)A + (scan - cnt) * 4;
        const unsigned spare = (unsigned)(size_t)(A + EP_X) + lane * 4;
#pragma unroll
        for (int e = 0; e < 128; ++e) {
            const int ch = e >> 4;
            const unsigned bit = __builtin_amdgcn_ubfe(msk[e >> 5], e & 31, 1);
            *(LAS float*)(size_t)(bit ? o ^ ((o >> 4) & 0x30u) : spare) = acc[ch >> 2][(e >> 1) & 1][ch & 3][e & 1][(e >> 2) & 3];
            o += bit * 4;
        }
    }
    float ysum = 0.f;
#pragma unroll
    for (int a = 0; a < 2; ++a)
#pragma unroll
        for (int b = 0; b < 2; ++b)
#pragma unroll
            for (int m = 0; m < 4; ++m) { const f32x4 t = acc[a][b][m][0] + acc[a][b][m][1]; ysum += (t[0] + t[1]) + (t[2] + t[3]); }
#pragma unroll
    for (int sh = 32; sh >= 1; sh >>= 1) ysum += __shfl_xor(ysum, sh);
    ysum *= 1.35f / (float)max(Lc, 1u);
    if (!dense && lane < 16 && Lc + lane < L1_MAX) *(LAS float*)(A + (((Lc + lane) * 4) ^ (((Lc + lane) >> 2) & 0x30u))) = ysum;
    LAS unsigned char* lut = A + EP_T;
    if (lane < 16) *(LAS unsigned*)(lut + lane * 4) = lut_compact(lane);
    {
        if (wid == 0) {
            Spin t0;
            for (;;) {
                CFENCE;
                const unsigned g = __builtin_amdgcn_raw_buffer_load_b32(rs, (lane & 31) * 8 + 4, OFF_AMAX + (rank * NGEMM + 96) * 8, AUX_SYS);
                if (__builtin_popcountll(__builtin_amdgcn_ballot_w64(g >= call)) >= 48 || t0.fail(myws, call)) break;
                __builtin_amdgcn_s_sleep(8);
            }
        }
        BAR; CFENCE;
    }
    {
        Spin t0;
        unsigned t, tm = ld_agent(myws + OFF_TMO);
        for (;;) {
            CFENCE;
            const u32x4 g = __builtin_amdgcn_raw_buffer_load_b128(rs, lane * 16, OFF_AMAX + wid * 1024, AUX_SYS);
            t = max(g[0], g[2]);
            if (__builtin_amdgcn_ballot_w64(g[1] < call || g[3] < call) == 0) break;
            if (t0.fail(myws, call)) { tm = call; break; }
            __builtin_amdgcn_s_sleep(4);
        }
        t = wave_max(t);
        if (lane == 0) { red[8 + wid] = t; red[wid] = tm == call; }
    }
    WAIT_L(0); BAR; CFENCE;
    const float amax = __uint_as_float(wave_max(red[8 + (lane & 7)]));
    const bool dead = __builtin_amdgcn_ballot_w64(red[lane & 7] != 0) != 0;
    const float inv = dead ? __builtin_nanf("") : 448.0f / amax;
    STAMP(myws, wv, 11);
    if (wr == 0) __builtin_amdgcn_s_setprio(2);

    const f32x2 inv2 = {inv, inv};
#define QUANT(ch, q) { const int a = (ch) >> 2, m = (ch) & 3; \
        const f32x4 y0 = scale4(acc[a][0][m][0], inv2), y1 = scale4(acc[a][0][m][1], inv2), y2 = scale4(acc[a][1][m][0], inv2), y3 = scale4(acc[a][1][m][1], inv2); \
        _Pragma("unroll") for (int j = 0; j < 4; ++j) \
            q[j] = (unsigned)__builtin_amdgcn_cvt_pk_fp8_f32(y2[j], y3[j], __builtin_amdgcn_cvt_pk_fp8_f32(y0[j], y1[j], __float_as_int(y0[j]), false), true); }
    u32x4 cod[5];
    if (dense) {
#pragma unroll
        for (int r = 0; r < 9; ++r) *(LAS u32x4*)(A + r * 1024 + lane * 16) = (u32x4){0u, 0u, 0u, 0u};
        unsigned o = (unsigned)(size_t)A + scan - cnt;
#pragma unroll
        for (int ch = 0; ch < 8; ++ch) {
            u32x4 q;
            QUANT(ch, q);
#pragma unroll
            for (int j = 0; j < 4; ++j) {
                const unsigned n4 = nib4(msk[ch >> 1], (ch & 1) * 4 + j);
                const unsigned long long z = (unsigned long long)__builtin_amdgcn_perm(0u, q[j], *(const LAS unsigned*)(lut + n4)) << ((o & 3u) * 8);
                LAS unsigned* d = (LAS unsigned*)(size_t)(o & ~3u);
                __hip_atomic_fetch_or(d, (unsigned)z, __ATOMIC_RELAXED, __HIP_MEMORY_SCOPE_WAVEFRONT);
                __hip_atomic_fetch_or(d + 1, (unsigned)(z >> 32), __ATOMIC_RELAXED, __HIP_MEMORY_SCOPE_WAVEFRONT);
                o += __builtin_popcount(n4);
            }
        }
    } else {
#define CODES(i0, n) { f32x4 y_[n][4]; \
            _Pragma("unroll") for (int i = 0; i < n; ++i) _Pragma("unroll") for (int k = 0; k < 4; ++k) y_[i][k] = *(const LAS f32x4*)(A + ((i0 + i) * 64 + (i0 + i < 4 ? lane : lane & 31)) * 64 + ((k ^ (lane >> 2)) & 3) * 16); \
            SCHED; \
            _Pragma("unroll") for (int i = 0; i < n; ++i) _Pragma("unroll") for (int k = 0; k < 4; ++k) { const f32x4 y = scale4(y_[i][k], inv2); \
                cod[i0 + i][k] = (unsigned)__builtin_amdgcn_cvt_pk_fp8_f32(y[2], y[3], __builtin_amdgcn_cvt_pk_fp8_f32(y[0], y[1], __float_as_int(y[0]), false), true); } }
        CODES(0, 2); CODES(2, 2);
        cod[4] = (u32x4){0u, 0u, 0u, 0u};
        if (Lc > 4096) CODES(4, 1);
        CFENCE;
#pragma unroll
        for (int i = 0; i < 5; ++i) *(LAS u32x4*)(A + (i * 64 + lane) * 16) = cod[i];
#pragma unroll
        for (int r = 0; r < 3; ++r) *(LAS u32x4*)(S2 + r * 1024 + lane * 16) = (u32x4){0u, 0u, 0u, 0u};
    }
    CFENCE;
    CYC(0);
    const unsigned ec = (__builtin_amdgcn_readfirstlane(__builtin_amdgcn_cvt_pk_fp8_f32(fminf(ysum * inv, 448.f), 0.f, 0, false)) >> 3) & 15u;
    const unsigned E1 = (ec + 17) * 0x01010101u, G = (Lc + 15) >> 4;
    bool raw = dense || (rt * 2 + wr) * 32 + ct * 4 + wc >= 1024 - RAW_LAST;
    unsigned bytes = Lc;
    if (!raw) {
        unsigned s2[5][4], n7[5][4], cnt2 = 0;
        const unsigned k07 = KS_(0x07070707u), k08 = KS_(0x08080808u), k0f = KS_(0x0f0f0f0fu), k10 = KS_(0x10101010u), k80 = KS_(H8);
#define CLASSIFY(i0, n) { const u32x4* d_ = cod + i0; \
            _Pragma("unroll") for (int i = 0; i < n; ++i) { const unsigned g = (i0 + i) * 64 + lane, wgt = g < G ? 0x08040201u : 0u; unsigned nb[4], p_[4]; \
                _Pragma("unroll") for (int k = 0; k < 4; ++k) { \
                    const unsigned e4 = (d_[i][k] >> 3) & k0f, u = E1 - e4, t = u & k08, ui = (u | (t - (t >> 3))) & k07; \
                    p_[k] = __builtin_amdgcn_perm(0x98989898u, 0x10080098u, ui); \
                    s2[i0 + i][k] = e4 | __builtin_amdgcn_perm(0x30302010u, 0x30303000u, ui); \
                    nb[k] = (d_[i][k] & k07) | (p_[k] & k08); \
                    n7[i0 + i][k] = __builtin_amdgcn_udot4(p_[k] & k80, wgt, 0u, false); \
                    cnt2 += __builtin_popcount(n7[i0 + i][k]); } \
                if (g < G) { *(LAS u32x2*)(B + g * 8) = (u32x2){nb[0] | (nb[1] << 4), nb[2] | (nb[3] << 4)}; \
                    *(LAS unsigned short*)(B + G * 8 + g * 2) = (unsigned short)bits16(p_[0] & k10, p_[1] & k10, p_[2] & k10, p_[3] & k10, 4); } } }
        CLASSIFY(0, 4);
#pragma unroll
        for (int k = 0; k < 4; ++k) s2[4][k] = n7[4][k] = 0;
        if (G > 256) CLASSIFY(4, 1);
        CYC(1);
        const unsigned sc2 = wave_scan(cnt2), n2 = __builtin_amdgcn_readlane(sc2, 63), G2 = (n2 + 15) >> 4, o2 = off_l2(G), o3 = o2 + G2 * 4;
        raw = n2 > L2_MAX;
        if (!raw) {
            unsigned o = sc2 - cnt2 + (unsigned)(size_t)S2;
#define SQUEEZE(i0, n) { unsigned se_[n][4]; \
                _Pragma("unroll") for (int i = 0; i < n; ++i) _Pragma("unroll") for (int k = 0; k < 4; ++k) se_[i][k] = *(const LAS unsigned*)(lut + (n7[i0 + i][k] >> 5)); \
                SCHED; \
                _Pragma("unroll") for (int i = 0; i < n; ++i) _Pragma("unroll") for (int k = 0; k < 4; ++k) { \
                    const unsigned long long z = (unsigned long long)__builtin_amdgcn_perm(0u, s2[i0 + i][k], se_[i][k]) << ((o & 3u) * 8); \
                    LAS unsigned* d = (LAS unsigned*)(size_t)(o & ~3u); \
                    __hip_atomic_fetch_or(d, (unsigned)z, __ATOMIC_RELAXED, __HIP_MEMORY_SCOPE_WAVEFRONT); \
                    __hip_atomic_fetch_or(d + 1, (unsigned)(z >> 32), __ATOMIC_RELAXED, __HIP_MEMORY_SCOPE_WAVEFRONT); \
                    o += __builtin_popcount(n7[i0 + i][k]); } }
            SQUEEZE(0, 4);
            if (G > 256) SQUEEZE(4, 1);
            CFENCE;
            CYC(2);
            const u32x4 d0 = *(const LAS u32x4*)(S2 + lane * 16), d1 = *(const LAS u32x4*)(S2 + 1024 + lane * 16);
            if (lane < G2) *(LAS unsigned*)(B + o2 + lane * 4) = ((d0[0] >> 4) & 0x03030303u) | ((d0[1] >> 2) & 0x0c0c0c0cu) | (d0[2] & 0x30303030u) | ((d0[3] << 2) & 0xc0c0c0c0u);
            if (64 + lane < G2) *(LAS unsigned*)(B + o2 + 256 + lane * 4) = ((d1[0] >> 4) & 0x03030303u) | ((d1[1] >> 2) & 0x0c0c0c0cu) | (d1[2] & 0x30303030u) | ((d1[3] << 2) & 0xc0c0c0c0u);
            unsigned m5[8], se[8], cnt3 = 0;
#pragma unroll
            for (int k = 0; k < 8; ++k) {
                const unsigned d = k < 4 ? d0[k & 3] : d1[k & 3];
                m5[k] = __builtin_amdgcn_udot4(d & (d << 1) & 0x20202020u, 0x08040201u, 0u, false);
                se[k] = *(const LAS unsigned*)(lut + (m5[k] >> 3));
                cnt3 += __builtin_popcount(m5[k]);
            }
            const unsigned sc3 = wave_scan(cnt3), n3 = __builtin_amdgcn_readlane(sc3, 63);
            bytes = o3 + ((n3 + 7) >> 3) * 4;
            raw = n3 > MAXESC || bytes > Lc;
            if (!raw) {
                o = sc3 - cnt3 + (unsigned)(size_t)(A + EP_L);
#pragma unroll
                for (int k = 0; k < 8; ++k) {
                    const unsigned long long z = (unsigned long long)__builtin_amdgcn_perm(0u, k < 4 ? d0[k & 3] : d1[k & 3], se[k]) << ((o & 3u) * 8);
                    LAS unsigned* d = (LAS unsigned*)(size_t)(o & ~3u);
                    __hip_atomic_fetch_or(d, (unsigned)z, __ATOMIC_RELAXED, __HIP_MEMORY_SCOPE_WAVEFRONT);
                    __hip_atomic_fetch_or(d + 1, (unsigned)(z >> 32), __ATOMIC_RELAXED, __HIP_MEMORY_SCOPE_WAVEFRONT);
                    o += __builtin_popcount(m5[k]);
                }
                CFENCE;
                const u32x2 x3 = *(const LAS u32x2*)(A + EP_L + lane * 8);
                *(LAS unsigned*)(B + o3 + lane * 4) = (x3[0] & 0x0f0f0f0fu) | ((x3[1] << 4) & 0xf0f0f0f0u);
                CFENCE;
            }
        }
    }
    if (raw) bytes = Lc;
    CYC(3);
    {
        LAS unsigned char* S = raw ? A : B;
#pragma nounroll
        for (unsigned r = 0; r < bytes; r += 1024)
            ST128(*(const LAS u32x4*)(S + r + lane * 16), rs, lane * 16, OFF_COD + wt * 8192 + r, AUX_SYS);
    }
    if (lane == 0) red[24 + wid] = (raw ? 1u << 30 : (bytes >> 2) << 14) | (ec << 26) | Lc;
    CYC(4);
    WAIT_V(0);
    CYC(5);
    {
        unsigned old = 0;
        if (lane == 0) old = __hip_atomic_fetch_add(red + 36 + wr, 1u, __ATOMIC_RELAXED, __HIP_MEMORY_SCOPE_WORKGROUP);
        if (__builtin_amdgcn_readfirstlane(old) == 3 && lane < 32 && (lane & 7) != 0) {
            const int w = wr * 4 + (lane >> 3);
            __hip_atomic_store((GAS unsigned long long*)(pl + OFF_DONE + (rank * 1024 + tile * 8 + w) * 8),
                               ((unsigned long long)((volatile LAS unsigned*)red)[24 + w] << 32) | call, __ATOMIC_RELAXED, __HIP_MEMORY_SCOPE_SYSTEM);
        }
    }
    STAMP(myws, wv, 12);
    {
        const rsrc_t ro = mk_rsrc(p.out, OUT_BYTES);
        const unsigned so = out_soff(rank, wt), vo = out_voff(lane);
        u32x4 qv[8];
#pragma unroll
        for (int ch = 0; ch < 8; ++ch) QUANT(ch, qv[ch]);
#pragma unroll
        for (int ch = 0; ch < 8; ++ch) ST128_RAW(qv[ch], ro, vo, so + chunk_soff(ch), AUX_WT);
        ST128_PAD8(qv);
    }
}

__global__ __launch_bounds__(512, 2) void fused_kernel(Args a) {
    __shared__ __attribute__((aligned(16))) unsigned char lds_arr[LDS_BYTES];
    LAS unsigned char* lds = (LAS unsigned char*)lds_arr;
    const int wid = __builtin_amdgcn_readfirstlane(threadIdx.x >> 6), lane = threadIdx.x & 63;
    const int b = blockIdx.x, xcd = b & 7, idx = b >> 3;
    Params p;
    p.x = a.x; p.w = a.w; p.out = a.out; p.my = (char*)a.ws[0]; p.rank = a.rank; p.pl = (char*)a.ws[lane & 7];
    asm volatile("" : "+s"(p.x), "+s"(p.w), "+s"(p.out), "+s"(p.my), "+s"(p.rank));
    const unsigned callv = (unsigned)__builtin_amdgcn_raw_ptr_buffer_atomic_add_i32(1, mk_rsrc(p.my, WS_USED), lane == 0 ? OFF_CNT + (b * 8 + wid) * 4 : 0xfffffff0u, 0, 0);
    if (idx < 16) gemm_role(p, lds, (xcd >> 1) * 4 + (idx >> 2), (xcd & 1) * 4 + (idx & 3), wid, lane, callv);
    else comm_role(p, lds, (idx - 16) * 8 + xcd, wid, lane, callv);
}

extern "C" void kernel_launch(int rank, int nranks, ncclComm_t comm, hipStream_t stream,
                              void* const* d_in, const int* in_sizes, int n_in,
                              void* d_out, int out_size,
                              const int* in_dtypes, int out_dtype,
                              void* const* d_out_all, void* const* d_ws_all, size_t ws_bytes) {
    (void)comm; (void)in_dtypes; (void)out_dtype; (void)d_out_all;
    if (nranks != NR || n_in != 2 || in_sizes[0] != M_ * KS || in_sizes[1] != KS * N_ || out_size != M_ * N_ || ws_bytes < WS_USED || (unsigned)rank >= (unsigned)NR) {
        fprintf(stderr, "kernel_launch: unexpected problem shape\n");
        return;
    }
    Args a;
    a.x = d_in[0]; a.w = d_in[1]; a.out = d_out;
    const size_t shift = (ws_bytes - WS_USED) & ~(size_t)0x1fffff;
    for (int i = 0; i < NR; ++i) a.ws[i] = (char*)d_ws_all[(rank + i) % NR] + shift;
    a.rank = rank; a.pad = 0;
    fused_kernel<<<dim3(NWG), dim3(512), 0, stream>>>(a);
}
```

```cpp
#include <hip/hip_runtime.h>
#include <rccl/rccl.h>
#include <cstdio>

#define LAS __attribute__((address_space(3)))
#define GAS __attribute__((address_space(1)))
typedef short bf16x8 __attribute__((ext_vector_type(8)));
typedef float f32x4 __attribute__((ext_vector_type(4)));
typedef unsigned u32x4 __attribute__((ext_vector_type(4)));
typedef unsigned u32x2 __attribute__((ext_vector_type(2)));
typedef __amdgpu_buffer_rsrc_t rsrc_t;

constexpr int NR = 8;
constexpr int M_ = 4096, N_ = 16384;
constexpr int KS = 512;
constexpr int NS = 2048;
constexpr int NT = 64;
constexpr int NGEMM = 128;
constexpr int NWG = 256;

constexpr unsigned OFF_CNT  = 0;
constexpr unsigned OFF_TMO  = 16384;
constexpr unsigned OFF_FLAG = 65536;
constexpr unsigned FLAG_W = 4096;
constexpr unsigned OFF_AMAX = 131072;
constexpr unsigned OFF_TICK = 143360;
constexpr unsigned OFF_PCNT = 147456;
constexpr unsigned OFF_PFLG = 163840;
constexpr unsigned OFF_PUF  = 188416;
constexpr unsigned OFF_MFLG = 176128;
constexpr unsigned OFF_DONE = 786432;
constexpr unsigned OFF_MF   = OFF_DONE + 65536;
constexpr unsigned OFF_DBG  = 262144;
constexpr unsigned CTRL_BYTES = 1u << 20;
constexpr unsigned XT_STEP = 512u * 1024u, WT_STEP = 256u * 1024u, BLK = 32768u, HALFB = 16384u;
constexpr unsigned OFF_XT = CTRL_BYTES;
constexpr unsigned XT_BYTES = NT * XT_STEP;
constexpr unsigned OFF_WT = OFF_XT + XT_BYTES;
constexpr unsigned WT_BYTES = NT * WT_STEP;
constexpr unsigned UNIT = 9216;
constexpr unsigned OFF_PX = OFF_WT + WT_BYTES;
constexpr unsigned OFF_PW = OFF_PX + 512 * UNIT;
constexpr unsigned OFF_MSK = OFF_PW + 2048 * UNIT;
constexpr unsigned OFF_COD = OFF_MSK + 1024 * 1024;
constexpr unsigned OFF_LM = OFF_COD + 1024 * 8192;
constexpr unsigned WS_USED = OFF_LM + 8 * 1024 * 1024;
constexpr unsigned OUT_BYTES = (unsigned)M_ * N_;
constexpr unsigned X_BYTES = (unsigned)M_ * KS * 2, W_BYTES = (unsigned)KS * N_ * 2;

constexpr unsigned long long DEADLINE = 100000000ull;
constexpr int AUX_SYS = 17;
#define WS_END 1
constexpr int AUX_NT = 2, AUX_WT = 16;
#define PPP 14
#define PPM 14
#define TEAM 4
#define PPO 55
#define CW 4
#define PUSH_W 8
#define PUSH_U 3
#define RAW_LAST 24
#define STAG 7
#define DW_AUX 0
#define DIR_E 8
#define DIR_D 250
#define PE_ 70
#define SCH_S 100
#define SCH_C 1000
#define SCH_B 10
#define SCH_A 1
constexpr int NPUF = 2;
constexpr int PACE_W = 800, PACE_W1 = 150, PACE_X = 400, PACE_E = PE_, PACE_O = 60, PACE_O0 = 150, PACE_L0 = 300;

constexpr unsigned LDS_STAGE = 8 * HALFB;
constexpr unsigned EP_BYTES = 19456;
constexpr unsigned EP_C = 8192, EP_S = EP_C + 5888, EP_L = EP_S + 2080, EP_T = 18432, EP_X = EP_T + 64;
constexpr unsigned LDS_FLAG = 8 * EP_BYTES;
constexpr unsigned LDS_RED = LDS_FLAG + 512;
constexpr unsigned EX_B = 8192, EX_M = EX_B + 8192, EX_P = EX_M + 1024, EX_E = EX_P + 1024, EX_BYTES = EX_E + 256, EX_LUT = 0, EX_MRG = 2048, EX_TOK = 2176, EX_0 = 2304;
constexpr unsigned LDS_BYTES = LDS_RED + 256;
static_assert(EX_0 + 8 * EX_BYTES <= LDS_BYTES && EP_BYTES % 1024 == 0 && EP_L + 1024 <= EP_T && EP_X + 256 <= EP_BYTES && LDS_STAGE <= LDS_FLAG && LDS_RED + 192 <= LDS_BYTES && LDS_BYTES <= 163840, "lds");
static_assert(OFF_PUF + 8 * NPUF * 384 <= OFF_DBG, "flags");
static_assert((PUSH_W * PUSH_U) % 4 == 0 && PUSH_W * PUSH_U <= 32 && 7 * PUSH_W <= NGEMM, "push");
static_assert(7 * PPO <= 128 * CW && 7 * PPM <= 128 && PPP <= PPM && TEAM <= CW, "waves");

struct Args {
    const void* x; const void* w; void* out;
    int rank; int pad;
    void* ws[NR];
};
struct Params {
    const void* x; const void* w; void* out; char* my; char* pl;
    int rank;
    __device__ __forceinline__ char* peer(int k) const {
        const unsigned long long a = (unsigned long long)pl;
        return (char*)(((unsigned long long)(unsigned)__builtin_amdgcn_readlane((unsigned)(a >> 32), k) << 32) | (unsigned)__builtin_amdgcn_readlane((unsigned)a, k));
    }
};

__device__ __forceinline__ rsrc_t mk_rsrc(const void* p, unsigned bytes) {
    return __builtin_amdgcn_make_buffer_rsrc((void*)p, 0, bytes, 0x00020000);
}
#define ST128(v, r, vo, so, aux) do { const u32x4 v_ = (v); __builtin_amdgcn_raw_buffer_store_b128(v_, r, vo, so, aux); asm volatile("s_nop 1" :: "v"(v_) : "memory"); } while (0)
#define ST128_RAW __builtin_amdgcn_raw_buffer_store_b128
#define ST128_PAD4(v) asm volatile("s_nop 1" :: "v"((v)[0]), "v"((v)[1]), "v"((v)[2]), "v"((v)[3]) : "memory")
#define ST128_PAD8(v) asm volatile("s_nop 1" :: "v"((v)[0]), "v"((v)[1]), "v"((v)[2]), "v"((v)[3]), "v"((v)[4]), "v"((v)[5]), "v"((v)[6]), "v"((v)[7]) : "memory")
#define WAIT_V(n) asm volatile("s_waitcnt vmcnt(" #n ")" ::: "memory")
#define WAIT_L(n) asm volatile("s_waitcnt lgkmcnt(" #n ")" ::: "memory")
#define BAR __builtin_amdgcn_s_barrier()
#define SCHED __builtin_amdgcn_sched_barrier(0)
#define CFENCE asm volatile("" ::: "memory")

#define STAMP(ws_, wv_, k_) do {} while (0)
#define CYC(i) do {} while (0)

__device__ __forceinline__ unsigned ld_agent(const void* p) { return __hip_atomic_load((const GAS unsigned*)p, __ATOMIC_RELAXED, __HIP_MEMORY_SCOPE_AGENT); }
__device__ __forceinline__ void st_agent(void* p, unsigned v) { __hip_atomic_store((GAS unsigned*)p, v, __ATOMIC_RELAXED, __HIP_MEMORY_SCOPE_AGENT); }
__device__ __forceinline__ void st_sys(void* p, unsigned v) { __hip_atomic_store((GAS unsigned*)p, v, __ATOMIC_RELAXED, __HIP_MEMORY_SCOPE_SYSTEM); }

__device__ __forceinline__ void add_sys(void* p) { asm volatile("global_atomic_add %0, %1, off sc1" :: "v"(p), "v"(1u) : "memory"); }
__device__ __forceinline__ unsigned x_flag(int t, int g) { return OFF_FLAG + (g >> 2) * 256 + t * 4; }
__device__ __forceinline__ unsigned w_flag(int t, int e) { return OFF_FLAG + FLAG_W + (e >> 2) * 256 + t * 4; }

struct Spin {
    unsigned long long t0; unsigned n;
    __device__ __forceinline__ Spin() : t0(__builtin_amdgcn_s_memrealtime()), n(0) {}
    __device__ __forceinline__ bool fail(char* myws, unsigned call) {
        if ((++n & 15u) == 0 && (__builtin_amdgcn_readfirstlane(ld_agent(myws + OFF_TMO)) == call || __builtin_amdgcn_s_memrealtime() - t0 > DEADLINE)) {
            st_agent(myws + OFF_TMO, call);
            return true;
        }
        __builtin_amdgcn_s_sleep(1);
        return false;
    }
};

#define DPP0(x, ctrl, rm) (unsigned)__builtin_amdgcn_update_dpp(0, (int)(x), ctrl, rm, 0xf, false)
__device__ __forceinline__ unsigned wave_scan(unsigned x) {
    x += DPP0(x, 0x111, 0xf); x += DPP0(x, 0x112, 0xf); x += DPP0(x, 0x114, 0xf); x += DPP0(x, 0x118, 0xf);
    x += DPP0(x, 0x142, 0xa); x += DPP0(x, 0x143, 0xc);
    return x;
}
__device__ __forceinline__ unsigned wave_max(unsigned x) {
    x = max(x, DPP0(x, 0x111, 0xf)); x = max(x, DPP0(x, 0x112, 0xf)); x = max(x, DPP0(x, 0x114, 0xf)); x = max(x, DPP0(x, 0x118, 0xf));
    x = max(x, DPP0(x, 0x142, 0xa)); x = max(x, DPP0(x, 0x143, 0xc));
    return __builtin_amdgcn_readlane(x, 63);
}

struct Sched {
    unsigned w[7][64];
    constexpr Sched() : w{} {
        for (int q = 0; q < 7; ++q) {
            int key[136] = {}, id[136] = {}, n = 0;
            for (int j = 0; j < 8; ++j) {
                int lo = 1 << 30;
                for (int rt = 0; rt < 16; ++rt) { const int k = (SCH_S * rt < SCH_C ? SCH_S * rt : SCH_C) + SCH_B * rt + 106 * (7 * j + (q + SCH_A * rt) % 7); key[n] = 2 * k + 1; id[n++] = j * 24 + 8 + rt; lo = k < lo ? k : lo; }
                key[n] = 2 * lo; id[n++] = -1 - j;
            }
            int at = 0;
            for (int o = 0; o < 136; ++o) {
                int best = -1;
                for (int i = 0; i < 136; ++i) if (key[i] >= 0 && (best < 0 || key[i] < key[best])) best = i;
                key[best] = -1;
                const int skip = id[best] == -1 ? PUSH_W * PUSH_U / 4 : 0;
                const int cnt = id[best] < 0 ? 8 - skip : 1, first = id[best] < 0 ? (-1 - id[best]) * 24 + skip : id[best];
                for (int c = 0; c < cnt; ++c, ++at) w[q][at >> 2] |= (unsigned)(first + c) << (8 * (at & 3));
            }
        }
    }
};
__constant__ const Sched SCHED_TAB;
__device__ __forceinline__ int step_of(int i, int rank, int rt) {
    if (i < 8) return rank * 8 + i;
    const int z = i - 8, j = z / 7, p = z - j * 7, q = (p + 7 * 16 * SCH_A - SCH_A * rt) % 7;
    return (((rank + 1 + q) & 7) << 3) + j;
}

__device__ __forceinline__ void x_load(u32x4 (&v)[8], rsrc_t rx, int j, int g, int lane) {
    const unsigned voff = (unsigned)(lane & 15) * (KS * 2) + (unsigned)(lane >> 4) * 16;
    const unsigned soff = (unsigned)g * 64 * (KS * 2) + (unsigned)j * 128;
#pragma unroll
    for (int r = 0; r < 4; ++r)
#pragma unroll
        for (int kk = 0; kk < 2; ++kk) v[r * 2 + kk] = __builtin_amdgcn_raw_buffer_load_b128(rx, voff + kk * 64, soff + r * 16 * (KS * 2), 0);
}
template <int AUX = AUX_NT> __device__ __forceinline__ void w_load(u32x2 (&v)[16], rsrc_t rw, int j, int d, int ct, int kk, int G, int lane) {
    const unsigned voff = (unsigned)(lane >> 4) * 8 * (N_ * 2) + (unsigned)(lane & 15) * 8;
    const unsigned soff = (unsigned)(j * 64 + kk * 32) * (N_ * 2) + (unsigned)(d * NS + ct * 256 + G * 128) * 2;
#pragma unroll
    for (int jj = 0; jj < 8; ++jj)
#pragma unroll
        for (int g = 0; g < 2; ++g) v[g * 8 + jj] = __builtin_amdgcn_raw_buffer_load_b64(rw, voff + g * 128, soff + jj * (N_ * 2), AUX);
}
__device__ __forceinline__ void w_transpose(u32x4 (&o)[8], const u32x2 (&v)[16]) {
#pragma unroll
    for (int g = 0; g < 2; ++g)
#pragma unroll
        for (int c = 0; c < 4; ++c)
#pragma unroll
            for (int dd = 0; dd < 4; ++dd)
                o[g * 4 + c][dd] = __builtin_amdgcn_perm(v[g * 8 + 2 * dd + 1][c >> 1], v[g * 8 + 2 * dd][c >> 1], (c & 1) ? 0x07060302u : 0x05040100u);
}
__device__ __forceinline__ unsigned x_img(int t, int g) { return OFF_XT + (unsigned)t * XT_STEP + (unsigned)g * 8192; }
__device__ __forceinline__ unsigned w_img(int t, int e) { return OFF_WT + (unsigned)t * WT_STEP + (unsigned)(e >> 2) * BLK + (unsigned)((e >> 1) & 1) * 8192 + (unsigned)(e & 1) * 1024; }
__device__ __forceinline__ unsigned frag_off(bool isw, unsigned f) { return isw ? ((f >> 1) & 1) * HALFB + (f >> 2) * 4096 + (f & 1) * 2048 : f * 1024; }
__device__ __forceinline__ void img_store(const u32x4 (&v)[8], rsrc_t rd, bool isw, unsigned img, int lane) {
#pragma unroll
    for (int f = 0; f < 8; ++f) ST128_RAW(v[f], rd, lane * 16, img + frag_off(isw, f), AUX_SYS);
    ST128_PAD8(v);
}
constexpr unsigned H8 = 0x80808080u, MAXESC = 512;
__device__ __forceinline__ unsigned KS_(unsigned c) { asm("" : "+s"(c)); return c; }
__device__ __forceinline__ unsigned bits16(unsigned m0, unsigned m1, unsigned m2, unsigned m3, int sh) {
    const unsigned a = __builtin_amdgcn_udot4(m0, 0x08040201u, __builtin_amdgcn_udot4(m1, 0x80402010u, 0u, false), false);
    const unsigned b = __builtin_amdgcn_udot4(m2, 0x08040201u, __builtin_amdgcn_udot4(m3, 0x80402010u, 0u, false), false);
    return (a >> sh) | (b << (8 - sh));
}
__device__ __forceinline__ u32x2 lut_expand(unsigned byte) { u32x2 s = {0u, 0u}; unsigned c = 0;
#pragma unroll
    for (int i = 0; i < 8; ++i) { const unsigned b = (byte >> i) & 1u; s[i >> 2] |= (b ? c : 0x0cu) << (8 * (i & 3)); c += b; } return s; }
__device__ __forceinline__ unsigned lut_merge(unsigned nib) { unsigned s = 0, c = 0;
#pragma unroll
    for (int i = 0; i < 4; ++i) { const unsigned b = (nib >> i) & 1u; s |= (b ? c : 4u + i) << (8 * i); c += b; } return s; }
template <int N> __device__ __forceinline__ void merge(unsigned* x, const unsigned* n8, unsigned& o, const LAS unsigned char* mrg) {
    unsigned se[N], w0[N], w1[N], of[N];
#pragma unroll
    for (int k = 0; k < N; ++k) { se[k] = *(const LAS unsigned*)(mrg + n8[k]); const LAS unsigned* w = (const LAS unsigned*)(size_t)(o & ~3u); w0[k] = w[0]; w1[k] = w[1]; of[k] = o; o += __builtin_popcount(n8[k]); }
    SCHED;
#pragma unroll
    for (int k = 0; k < N; ++k) x[k] = __builtin_amdgcn_perm(x[k], __builtin_amdgcn_alignbyte(w1[k], w0[k], of[k]), se[k]);
}
__device__ __forceinline__ unsigned lut_compact(unsigned nib) { unsigned s = 0x0c0c0c0cu, c = 0;
#pragma unroll
    for (int i = 0; i < 4; ++i) if ((nib >> i) & 1u) { s = (s & ~(0xffu << (8 * c))) | ((unsigned)i << (8 * c)); ++c; } return s; }
__device__ __forceinline__ void squeeze(unsigned x, unsigned sel, unsigned o) {
    const unsigned long long z = (unsigned long long)__builtin_amdgcn_perm(0u, x, sel) << ((o & 3u) * 8);
    LAS unsigned* d = (LAS unsigned*)(size_t)(o & ~3u);
    __hip_atomic_fetch_or(d, (unsigned)z, __ATOMIC_RELAXED, __HIP_MEMORY_SCOPE_WAVEFRONT);
    __hip_atomic_fetch_or(d + 1, (unsigned)(z >> 32), __ATOMIC_RELAXED, __HIP_MEMORY_SCOPE_WAVEFRONT);
}
constexpr unsigned PK_HDR = 5 * 1024, PK_RAW = 0x80000000u, PK_FIRST = 384;
__device__ __forceinline__ unsigned clamp7(unsigned z) {
    const unsigned o = (((z & KS_(0x78787878u)) + 0x78787878u) | z) & H8;
    return (z | ((o >> 4) - (o >> 7))) & 0x07070707u;
}
__device__ __forceinline__ void pack_store(const u32x4 (&v)[8], rsrc_t rs, LAS unsigned char* L, unsigned slot, int lane) {
#pragma unroll
    for (int r = 0; r < 3; ++r) *(LAS u32x4*)(L + r * 1024 + lane * 16) = (u32x4){0u, 0u, 0u, 0u};
    if (lane < 16) *(LAS unsigned*)(L + 4096 + lane * 4) = lut_compact(lane);
    unsigned p0[16], z[16], n7[16], sum = 0, cnt = 0;
#pragma unroll
    for (int r = 0; r < 16; ++r) {
        const unsigned lo = __builtin_amdgcn_perm(v[r >> 1][2 * (r & 1) + 1], v[r >> 1][2 * (r & 1)], 0x06040200u);
        const unsigned hi = __builtin_amdgcn_perm(v[r >> 1][2 * (r & 1) + 1], v[r >> 1][2 * (r & 1)], 0x07050301u);
        p0[r] = (hi & H8) | (lo & ~H8);
        z[r] = ((hi << 1) & 0xfefefefeu) | ((lo >> 7) & 0x01010101u);
        sum = __builtin_amdgcn_sad_u8(z[r], 0u, sum);
    }
    const unsigned ec = min(max((__builtin_amdgcn_readlane(wave_scan(sum), 63) + 7782u) >> 12, 19u), 253u), C = (ec + 1) * 0x01010101u;
    u32x4 p2 = {0u, 0u, 0u, 0u};
#pragma unroll
    for (int r = 0; r < 16; ++r) {
        z[r] = ((C | H8) - (z[r] & KS_(~H8))) ^ ((C ^ ~z[r]) & H8);
        const unsigned c = __builtin_amdgcn_perm(0x83838383u, 0x02010083u, clamp7(z[r]));
        p2[r >> 2] |= (c & KS_(0x03030303u)) << (2 * (r & 3));
        n7[r] = __builtin_amdgcn_udot4(c & KS_(H8), 0x08040201u, 0u, false);
        cnt += __builtin_popcount(n7[r]);
    }
    const unsigned sc = wave_scan(cnt), n2 = __builtin_amdgcn_readlane(sc, 63), G2 = (n2 + 15) >> 4;
    bool bad = n2 > 2048;
    unsigned bytes = 0;
    if (!bad) {
        unsigned se[16], o = sc - cnt + (unsigned)(size_t)L;
#pragma unroll
        for (int r = 0; r < 16; ++r) se[r] = *(const LAS unsigned*)(L + 4096 + (n7[r] >> 5));
#pragma unroll
        for (int r = 0; r < 16; ++r) { squeeze(z[r], se[r], o); o += __builtin_popcount(n7[r]); }
        CFENCE;
        const u32x4 d0 = *(const LAS u32x4*)(L + lane * 16), d1 = *(const LAS u32x4*)(L + 1024 + lane * 16);
        unsigned pk[2] = {0u, 0u}, m7[8], cnt3 = 0;
#pragma unroll
        for (int k = 0; k < 8; ++k) {
            const unsigned c = __builtin_amdgcn_perm(0x83830201u, 0x83838300u, clamp7(k < 4 ? d0[k & 3] : d1[k & 3]));
            pk[k >> 2] |= (c & KS_(0x03030303u)) << (2 * (k & 3));
            m7[k] = __builtin_amdgcn_udot4(c & KS_(H8), 0x08040201u, 0u, false);
            se[k] = *(const LAS unsigned*)(L + 4096 + (m7[k] >> 5));
            cnt3 += __builtin_popcount(m7[k]);
        }
        if (lane < G2) *(LAS unsigned*)(L + 3076 + lane * 4) = pk[0];
        if (64 + lane < G2) *(LAS unsigned*)(L + 3076 + 256 + lane * 4) = pk[1];
        const unsigned sc3 = wave_scan(cnt3), n3 = __builtin_amdgcn_readlane(sc3, 63);
        bad = n3 > 512;
        bytes = 4 + G2 * 4 + ((n3 + 7) >> 3) * 4;
        if (!bad) {
            o = sc3 - cnt3 + (unsigned)(size_t)(L + 2304);
#pragma unroll
            for (int k = 0; k < 8; ++k) { squeeze((k < 4 ? d0[k & 3] : d1[k & 3]) ^ 0x06060606u, se[k], o); o += __builtin_popcount(m7[k]); }
            CFENCE;
            const u32x2 x = *(const LAS u32x2*)(L + 2304 + lane * 8);
            unsigned nb[2], b = 0;
#pragma unroll
            for (int k = 0; k < 2; ++k) {
                const unsigned y = x[k] ^ 0x06060606u, ff = ((y & ~H8) + 0x01010101u) & y & H8;
                b |= (((y & ~H8) + 0x6b6b6b6bu) | y) & H8 & ~ff;
                nb[k] = ((y + 0x0a0a0a0au - ((ff >> 7) * 10u)) | ((ff >> 3) - (ff >> 7))) & 0x0f0f0f0fu;
            }
            *(LAS unsigned*)(L + 3076 + G2 * 4 + lane * 4) = nb[0] | (nb[1] << 4);
            bad = __builtin_amdgcn_ballot_w64(b != 0) != 0;
        }
    }
    if (lane == 0) *(LAS unsigned*)(L + 3072) = bad ? PK_RAW : ec | (bytes << 8);
    CFENCE;
    ST128(*(const LAS u32x4*)(L + 3072 + lane * 16), rs, lane * 16 + 1024, slot + 4096, AUX_SYS);
    if (!bad) {
#pragma unroll
        for (int r = 0; r < 4; ++r) ST128(((u32x4){p0[4 * r], p0[4 * r + 1], p0[4 * r + 2], p0[4 * r + 3]}), rs, lane * 16 + r * 1024, slot, AUX_SYS);
        ST128(p2, rs, lane * 16, slot + 4096, AUX_SYS);
    } else {
#pragma unroll
        for (int f = 0; f < 8; ++f) ST128(v[f], rs, lane * 16, slot + (f < 5 ? f : f + 1) * 1024, AUX_SYS);
    }
}
struct Puller {
    rsrc_t rp, rs; char* myws; LAS unsigned char* lut; int rank, src, q, lane; unsigned call, seen, tab;
    __device__ __forceinline__ int unit_of(int tk) const { return (int)((__builtin_amdgcn_readlane(tab, tk >> 4) >> (2 * (tk & 12))) & 0xffu) * 4 + (tk & 3); }
    __device__ __forceinline__ int xg(int i) const { return i - 32; }
    __device__ __forceinline__ unsigned unit_id(int u) const { const int j = u / 96, i = u % 96; return i < 32 ? 512 + (rank * 8 + j) * 32 + i : j * 64 + xg(i); }
    __device__ __forceinline__ unsigned flag_of(int u) const { const int j = u / 96, i = u % 96; return i < 32 ? w_flag(src * 8 + j, i) : x_flag(src * 8 + j, xg(i)); }
    __device__ __forceinline__ void ready(int u) {
        const int j = u / 96, i = u % 96, need = j * 2 + (i >= 32);
        if (!((seen >> need) & 1u)) {
            Spin t0;
            for (;;) {
                CFENCE;
                const unsigned sum = OFF_PFLG + (src * 8 + j) * 128 + (i >= 32 ? 0 : 4);
                const unsigned f = __builtin_amdgcn_raw_buffer_load_b32(rs, (lane & 1) && j < NPUF ? OFF_PUF + ((src * NPUF + j) * 96 + i) * 4 : sum, 0, AUX_SYS);
                if (__builtin_amdgcn_readlane(f, 0) >= call) { seen |= 1u << need; break; }
                if (__builtin_amdgcn_readlane(f, 1) >= call) break;
                if (t0.fail(myws, call)) break;
                __builtin_amdgcn_s_sleep(4);
            }
            CFENCE;
        }
    }
    __device__ __forceinline__ void issue(LAS unsigned char* b, int u) {
        const unsigned id = unit_id(u), slot = OFF_PX + id * UNIT;
        __builtin_amdgcn_raw_ptr_buffer_load_lds(rp, (LAS void*)(b + 4096), 16, lane < PK_FIRST / 16 ? lane * 16 : 0xf0000000u, slot + 4096, 1024, AUX_SYS);
        __builtin_amdgcn_raw_ptr_buffer_load_lds(rp, (LAS void*)(b + 4096), 16, lane * 16, slot + 4096, 0, AUX_SYS);
        __builtin_amdgcn_raw_ptr_buffer_load_lds(rp, (LAS void*)b, 16, lane * 16, slot, 0, AUX_SYS);
        __builtin_amdgcn_raw_ptr_buffer_load_lds(rp, (LAS void*)b, 16, lane * 16, slot, 1024, AUX_SYS);
        __builtin_amdgcn_raw_ptr_buffer_load_lds(rp, (LAS void*)b, 16, lane * 16, slot, 2048, AUX_SYS);
        __builtin_amdgcn_raw_ptr_buffer_load_lds(rp, (LAS void*)b, 16, lane * 16, slot, 3072, AUX_SYS);
    }
    __device__ __forceinline__ void finish(LAS unsigned char* b, int u) {
        const int j = u / 96, i = u % 96, t = src * 8 + j;
        const bool isw = i < 32;
        const unsigned img = isw ? w_img(t, i) : x_img(t, xg(i));
        const unsigned h = *(const LAS unsigned*)(b + PK_HDR);
        u32x4 v[8];
#pragma unroll
        for (int r = 0; r < 5; ++r) v[r] = *(const LAS u32x4*)(b + r * 1024 + lane * 16);
        const unsigned meta = __builtin_amdgcn_readfirstlane(h), ec = meta & 0xffu;
        if (!(meta & PK_RAW)) {
            if ((meta >> 8) > PK_FIRST) {
                __builtin_amdgcn_raw_ptr_buffer_load_lds(rp, (LAS void*)(b + 4096), 16, lane * 16, OFF_PX + unit_id(u) * UNIT + 4096, 1024, AUX_SYS);
                WAIT_V(0);
            }
            const unsigned T1 = ec | ((ec - 1) << 8) | ((ec - 2) << 16), T2 = (ec + 1) | ((ec - 3) << 8) | ((ec - 4) << 16), k01 = KS_(0x01010101u), k03 = KS_(0x03030303u);
            const u32x4 p2 = v[4];
            unsigned pk0 = *(const LAS unsigned*)(b + PK_HDR + 4 + lane * 4), pk1 = *(const LAS unsigned*)(b + PK_HDR + 260 + lane * 4);
            u32x4 p0[4];
            unsigned e[16], n8[16], cnt = 0;
#pragma unroll
            for (int w = 0; w < 4; ++w) {
                const unsigned ot = p2[w] & (p2[w] >> 1) & KS_(0x55555555u);
                cnt += __builtin_popcount(ot);
#pragma unroll
                for (int k = 0; k < 4; ++k) {
                    e[4 * w + k] = __builtin_amdgcn_perm(0u, T1, (p2[w] >> (2 * k)) & k03);
                    n8[4 * w + k] = __builtin_amdgcn_udot4((ot >> (2 * k)) & k01, 0x40201008u, 0u, false);
                }
            }
            const unsigned sc = wave_scan(cnt), G2 = min((__builtin_amdgcn_readlane(sc, 63) + 15) >> 4, 128u);
            const unsigned x3 = *(const LAS unsigned*)(b + PK_HDR + 4 + G2 * 4 + lane * 4), B5 = (ec - 5) * 0x01010101u;
#pragma unroll
            for (int r = 0; r < 4; ++r) p0[r] = v[r];
            {
                const unsigned d0 = x3 & 0x0f0f0f0fu, d1 = (x3 >> 4) & 0x0f0f0f0fu, t0 = (d0 + k01) & 0x10101010u, t1 = (d1 + k01) & 0x10101010u;
                *(LAS u32x2*)(b + lane * 8) = (u32x2){B5 + t0 + (t0 >> 2) + (t0 >> 3) - d0, B5 + t1 + (t1 >> 2) + (t1 >> 3) - d1};
            }
            pk0 = lane < G2 ? pk0 : 0u; pk1 = 64 + lane < G2 ? pk1 : 0u;
            const unsigned ot0 = pk0 & (pk0 >> 1) & KS_(0x55555555u), ot1 = pk1 & (pk1 >> 1) & KS_(0x55555555u), cnt3 = __builtin_popcount(ot0) + __builtin_popcount(ot1);
            unsigned o = wave_scan(cnt3) - cnt3 + (unsigned)(size_t)b;
            CFENCE;
#define IN_L2(pk, ot, at) { unsigned v2[4], m8[4]; \
                _Pragma("unroll") for (int k = 0; k < 4; ++k) { v2[k] = __builtin_amdgcn_perm(0u, T2, (pk >> (2 * k)) & k03); m8[k] = __builtin_amdgcn_udot4((ot >> (2 * k)) & k01, 0x40201008u, 0u, false); } \
                merge<4>(v2, m8, o, lut); \
                *(LAS u32x4*)(b + at + lane * 16) = (u32x4){v2[0], v2[1], v2[2], v2[3]}; }
            IN_L2(pk0, ot0, 6144);
            if (G2 > 64) IN_L2(pk1, ot1, 7168);
            CFENCE;
            o = sc - cnt + (unsigned)(size_t)(b + 6144);
            merge<16>(e, n8, o, lut);
#pragma unroll
            for (int r = 0; r < 16; ++r) {
                const unsigned pp = p0[r >> 2][r & 3], ee = e[r];
                const unsigned lo = (pp & ~H8) | ((ee << 7) & H8), hi = (pp & H8) | ((ee >> 1) & ~H8);
                v[r >> 1][2 * (r & 1)] = __builtin_amdgcn_perm(hi, lo, 0x05010400u); v[r >> 1][2 * (r & 1) + 1] = __builtin_amdgcn_perm(hi, lo, 0x07030602u);
            }
        } else {
            const unsigned slot = OFF_PX + unit_id(u) * UNIT;
            WAIT_L(0);
            __builtin_amdgcn_raw_ptr_buffer_load_lds(rp, (LAS void*)b, 16, lane * 16, slot + 6144, 0, AUX_SYS);
            __builtin_amdgcn_raw_ptr_buffer_load_lds(rp, (LAS void*)b, 16, lane * 16, slot + 6144, 1024, AUX_SYS);
            __builtin_amdgcn_raw_ptr_buffer_load_lds(rp, (LAS void*)b, 16, lane * 16, slot + 6144, 2048, AUX_SYS);
            WAIT_V(0);
#pragma unroll
            for (int r = 5; r < 8; ++r) v[r] = *(const LAS u32x4*)(b + (r - 5) * 1024 + lane * 16);
        }
        img_store(v, rs, isw, img, lane);
    }
};
static_assert(OFF_PW == OFF_PX + 512 * UNIT, "unit ids run through the x and the w slots");

#define DRAW(r, tick) { unsigned long long ex_; asm volatile("s_mov_b64 %1, exec\n\ts_mov_b64 exec, 1\n\tglobal_atomic_add %0, %2, %3, %4 sc0\n\ts_mov_b64 exec, %1" \
        : "=&v"(r), "=&s"(ex_) : "v"(0u), "v"(1u), "s"(tick) : "memory"); }
#define TAKE(t, r, N) { unsigned t_; asm volatile("s_waitcnt vmcnt(" #N ")\n\tv_readfirstlane_b32 %0, %1" : "=s"(t_) : "v"(r) : "memory"); t = (int)(t_ - tbase); }
constexpr int NPUSH = PUSH_W * PUSH_U, NU_IN = 768 - NPUSH, NU_ALL = NU_IN + 256;

__device__ __forceinline__ void pull_role(const Params& p, LAS unsigned char* lds, LAS unsigned char* L, int c, int lane, unsigned call, int wv) {
    Puller P; P.rank = p.rank; P.q = c % 7; P.src = (p.rank + 1 + P.q) & 7; P.lane = lane; P.seen = 0; P.call = call; P.lut = lds + EX_MRG;
    P.tab = SCHED_TAB.w[P.q][lane];
    asm volatile("" : "+v"(P.tab));
    P.myws = p.my; P.rs = mk_rsrc(P.myws, WS_USED); P.rp = mk_rsrc(p.peer(1 + P.q), WS_USED);
    char* tick = P.myws + OFF_TICK + P.q * 128;
    const unsigned tbase = (call - 1) * (NU_ALL + TEAM * PPM);
    LAS unsigned* token = (LAS unsigned*)(lds + EX_TOK);
#define TOKEN_GET { for (;;) { unsigned o_ = 1; if (lane == 0) o_ = __hip_atomic_exchange(token, 1u, __ATOMIC_RELAXED, __HIP_MEMORY_SCOPE_WORKGROUP); \
            if (__builtin_amdgcn_readfirstlane(o_) == 0) break; __builtin_amdgcn_s_sleep(1); } }
#define TOKEN_PUT { if (lane == 0) __hip_atomic_store(token, 0u, __ATOMIC_RELAXED, __HIP_MEMORY_SCOPE_WORKGROUP); }
    if (c >= 7 * PPP) {
        Spin t0;
        while ((int)(__builtin_amdgcn_readfirstlane(ld_agent(tick)) - tbase) < NU_IN) { if (t0.fail(P.myws, call)) break; __builtin_amdgcn_s_sleep(32); }
    }
    unsigned r;
    int u;
    DRAW(r, tick); TAKE(u, r, 0);
    while ((unsigned)u < (unsigned)NU_IN) {
        const int un = P.unit_of(u);
        P.ready(un);
        TOKEN_GET;
        P.issue(L, un);
        WAIT_V(0);
        TOKEN_PUT;
        DRAW(r, tick);
        P.finish(L, un);
        TAKE(u, r, 0);
        if (lane == 0) add_sys(P.myws + P.flag_of(un));
    }
    STAMP(P.myws, wv, 9);
    unsigned long long seen = 0; int sblk = -1;
    while ((unsigned)u < (unsigned)NU_ALL) {
        const int mu = u - NU_IN;
        if (!((seen >> (mu & 63)) & 1) || (mu >> 6) != sblk) {
            Spin t0; sblk = mu >> 6;
            for (;;) {
                CFENCE;
                seen = __builtin_amdgcn_ballot_w64(__builtin_amdgcn_raw_buffer_load_b32(P.rs, lane * 4, OFF_MFLG + (P.src * 256 + sblk * 64) * 4, AUX_SYS) >= call);
                if (((seen >> (mu & 63)) & 1) || t0.fail(P.myws, call)) break;
                __builtin_amdgcn_s_sleep(8);
            }
            CFENCE;
        }
        TOKEN_GET;
        __builtin_amdgcn_raw_ptr_buffer_load_lds(P.rp, (LAS void*)L, 16, lane * 16, OFF_MSK + mu * 4096, 0, AUX_SYS);
        __builtin_amdgcn_raw_ptr_buffer_load_lds(P.rp, (LAS void*)L, 16, lane * 16, OFF_MSK + mu * 4096, 1024, AUX_SYS);
        __builtin_amdgcn_raw_ptr_buffer_load_lds(P.rp, (LAS void*)L, 16, lane * 16, OFF_MSK + mu * 4096, 2048, AUX_SYS);
        __builtin_amdgcn_raw_ptr_buffer_load_lds(P.rp, (LAS void*)L, 16, lane * 16, OFF_MSK + mu * 4096, 3072, AUX_SYS);
        WAIT_V(0);
        TOKEN_PUT;
        DRAW(r, tick);
        u32x4 mv[4];
#pragma unroll
        for (int k = 0; k < 4; ++k) mv[k] = *(const LAS u32x4*)(L + k * 1024 + lane * 16);
#pragma unroll
        for (int k = 0; k < 4; ++k) ST128_RAW(mv[k], P.rs, lane * 16 + k * 1024, OFF_LM + P.src * 1048576 + mu * 4096, AUX_SYS);
        ST128_PAD4(mv);
        TAKE(u, r, 0);
        if (lane == 0) st_sys(P.myws + OFF_MF + (P.src * 256 + mu) * 4, call);
    }
    STAMP(P.myws, wv, 17);
}

constexpr unsigned L1_MAX = 4608, L2_MAX = 2048;
__device__ __forceinline__ unsigned off_l2(unsigned G) { return G * 8 + ((G * 2 + 3) & ~3u); }
__device__ __forceinline__ unsigned popc4(const u32x4& m) { return __builtin_popcount(m[0]) + __builtin_popcount(m[1]) + __builtin_popcount(m[2]) + __builtin_popcount(m[3]); }
__device__ __forceinline__ unsigned nib4(unsigned m, int i) { return (i == 0 ? m << 2 : m >> (4 * i - 2)) & 0x3cu; }
__device__ __forceinline__ unsigned out_voff(int lane) { return (unsigned)(lane & 15) * N_ + (unsigned)(lane >> 4) * 16; }
__device__ __forceinline__ unsigned out_soff(int src, int wt) {
    const int tile = wt >> 3, w = wt & 7;
    return (unsigned)((tile >> 3) * 256 + (w >> 2) * 64) * N_ + (unsigned)(src * NS + (tile & 7) * 256 + (w & 3) * 64);
}
__device__ __forceinline__ constexpr unsigned chunk_soff(int ch) { return (unsigned)((ch >> 2) * 128 + (ch & 3) * 16) * N_; }

__device__ __forceinline__ void expand_role(const Params& p, LAS unsigned char* lds, LAS unsigned char* L, int ei, int lane, unsigned call, int wv) {
    const int rank = p.rank, q = ei % 7, src = (rank + 1 + q) & 7;
    char* myws = p.my;
    char* pws = p.peer(1 + q);
    const rsrc_t rs = mk_rsrc(myws, WS_USED), ro = mk_rsrc(p.out, OUT_BYTES);
    char* tick = myws + OFF_TICK + (8 + q) * 128;
    const unsigned tbase = (call - 1) * (1024 + 2 * PPO);
    bool dead = false;
#define WT_OF(k) ((unsigned)(k) < 1024u ? ((k) & ~63) | (((k) & 28) << 1) | (((k) >> 3) & 4) | ((k) & 3) : (k))
#define ENT_OFF(t) (lane == 0 ? OFF_DONE + (src * 1024 + ((t) & ~1)) * 8 : OFF_MF + (src * 256 + (((t) >> 2) & ~3)) * 4)
#define ENT_FETCH(t) { if (lane < 2) __builtin_amdgcn_raw_ptr_buffer_load_lds(rs, (LAS void*)(L + EX_E), 16, ENT_OFF(t), 0, 0, AUX_SYS); }
#define EX_ISSUE(t, e, x, f, n, xo) { unsigned e_ = (e), x_ = (x), f_ = (f); \
        if ((e_ < call || f_ < call) && !dead) { Spin t0_; \
              \
            for (;;) { CFENCE; const bool had_ = e_ >= call; \
                const u32x4 g_ = __builtin_amdgcn_raw_buffer_load_b128(rs, lane < (had_ ? 2 : 1) ? ENT_OFF(t) : 0xf0000000u, 0, AUX_SYS); \
                e_ = __builtin_amdgcn_readlane(((t) & 1) ? g_[2] : g_[0], 0); x_ = __builtin_amdgcn_readlane(((t) & 1) ? g_[3] : g_[1], 0); \
                f_ = __builtin_amdgcn_readlane(((t) & 8) ? (((t) & 4) ? g_[3] : g_[2]) : (((t) & 4) ? g_[1] : g_[0]), 1); \
                if (!had_ && e_ >= call) continue; \
                if (e_ >= call && f_ >= call) break; if (t0_.fail(myws, call)) { dead = true; break; } __builtin_amdgcn_s_sleep(8); } \
            CFENCE; \
            if (STAG) { const unsigned long long s0_ = __builtin_amdgcn_s_memrealtime(); while ((unsigned)(__builtin_amdgcn_s_memrealtime() - s0_) < (ei / 7 - PPM) * STAG && ei / 7 > PPM) __builtin_amdgcn_s_sleep(1); } } \
        n = min(x_ & 0x3fffu, 8192u); xo = x_; \
        const unsigned by_ = min(((x_ >> 30) ? n : ((x_ >> 14) & 0xfffu) * 4) + 15 & ~15u, 8192u); \
        const rsrc_t rc_ = mk_rsrc(pws + OFF_COD + (unsigned)(t) * 8192, by_); \
        _Pragma("nounroll") for (unsigned r_ = 0; r_ < by_; r_ += 1024) \
            __builtin_amdgcn_raw_ptr_buffer_load_lds(rc_, (LAS void*)(L + EX_B + r_), 16, lane * 16, r_, 0, AUX_SYS); \
        __builtin_amdgcn_raw_ptr_buffer_load_lds(rs, (LAS void*)(L + EX_M), 16, lane * 16, OFF_LM + src * 1048576 + (t) * 1024, 0, AUX_SYS); }
    unsigned r0, n0 = 0, x0 = 0;
    DRAW(r0, tick);
    int t0;
    TAKE(t0, r0, 0); t0 = WT_OF(t0);
    if ((unsigned)t0 < 1024u) EX_ISSUE(t0, 0u, 0u, 0u, n0, x0);
    int t1;
    { unsigned r1; DRAW(r1, tick); TAKE(t1, r1, 0); t1 = WT_OF(t1); }
    STAMP(myws, wv, 18);
    while ((unsigned)t0 < 1024u) {
        if ((unsigned)t1 < 1024u) ENT_FETCH(t1);
        const u32x4 m = *(const LAS u32x4*)(L + EX_M + lane * 16);
        if (x0 >> 30) {
#pragma nounroll
            for (unsigned r = 0; r < n0; r += 1024) *(LAS u32x4*)(L + r + lane * 16) = *(const LAS u32x4*)(L + EX_B + r + lane * 16);
        } else {
            const unsigned G = min((n0 + 15) >> 4, L1_MAX / 16), ec = (x0 >> 26) & 15u;
            const unsigned T1 = (ec << 3) | (((ec - 1) & 15u) << 11) | (((ec - 2) & 15u) << 19) | 0x80000000u;
            const unsigned T2 = (((ec + 1) & 15u) << 3) | (((ec - 3) & 15u) << 11) | (((ec - 4) & 15u) << 19) | 0x80000000u;
            LAS unsigned char* S2 = L + EX_B + 5888;
            unsigned mn[20], ex[20], n8[20], cnt2 = 0;
            const unsigned k01 = KS_(0x01010101u), k02 = KS_(0x02020202u), k03 = KS_(0x03030303u), k07 = KS_(0x07070707u), k80 = KS_(H8), kmul = KS_(0x00408102u);
#define LAYER1(i0, n) { u32x2 nn_[n]; unsigned hb_[n]; \
                _Pragma("unroll") for (int i = 0; i < n; ++i) { const unsigned g = (i0 + i) * 64 + lane; nn_[i] = *(const LAS u32x2*)(L + EX_B + g * 8); hb_[i] = *(const LAS unsigned short*)(L + EX_B + G * 8 + g * 2); } \
                SCHED; \
                _Pragma("unroll") for (int i = 0; i < n; ++i) { const unsigned hb = (i0 + i) * 64 + lane < G ? hb_[i] : 0u;        \
                    _Pragma("unroll") for (int k = 0; k < 4; ++k) { const int z = 4 * (i0 + i) + k; mn[z] = (k & 1) ? nn_[i][k >> 1] >> 4 : nn_[i][k >> 1]; \
                        ex[z] = __builtin_amdgcn_perm(0u, T1, ((mn[z] >> 3) & k01) | (__builtin_amdgcn_ubfe(hb, 4 * k, 4) * kmul & k02)); \
                        n8[z] = __builtin_amdgcn_udot4(ex[z] & k80, 0x08040201u, 0u, false) >> 4; \
                        cnt2 += __builtin_popcount(n8[z]); } } }
            LAYER1(0, 4);
#pragma unroll
            for (int k = 16; k < 20; ++k) mn[k] = ex[k] = n8[k] = 0;
            if (G > 256) LAYER1(4, 1);
            CYC(0);
            const unsigned sc2 = wave_scan(cnt2), n2 = min(__builtin_amdgcn_readlane(sc2, 63), L2_MAX), G2 = (n2 + 15) >> 4, o2 = off_l2(G), o3 = o2 + G2 * 4;
            {
                unsigned pk0 = *(const LAS unsigned*)(L + EX_B + o2 + lane * 4), pk1 = *(const LAS unsigned*)(L + EX_B + o2 + 256 + lane * 4);
                pk0 = lane < G2 ? pk0 : 0u; pk1 = 64 + lane < G2 ? pk1 : 0u;
                const unsigned x3 = *(const LAS unsigned*)(L + EX_B + o3 + lane * 4);
                *(LAS u32x2*)(L + EX_P + lane * 8) = (u32x2){(x3 << 3) & 0x78787878u, (x3 >> 1) & 0x78787878u};
                const unsigned ot0 = pk0 & (pk0 >> 1) & KS_(0x55555555u), ot1 = pk1 & (pk1 >> 1) & KS_(0x55555555u), cnt3 = __builtin_popcount(ot0) + __builtin_popcount(ot1);
                unsigned v2[8], m8[8];
#pragma unroll
                for (int k = 0; k < 8; ++k) {
                    v2[k] = __builtin_amdgcn_perm(0u, T2, ((k < 4 ? pk0 : pk1) >> (2 * (k & 3))) & k03);
                    m8[k] = __builtin_amdgcn_udot4(((k < 4 ? ot0 : ot1) >> (2 * (k & 3))) & k01, 0x40201008u, 0u, false);
                }
                unsigned o = wave_scan(cnt3) - cnt3 + (unsigned)(size_t)(L + EX_P);
                CFENCE;
                merge<8>(v2, m8, o, lds + EX_MRG);
                *(LAS u32x4*)(S2 + lane * 16) = (u32x4){v2[0], v2[1], v2[2], v2[3]};
                *(LAS u32x4*)(S2 + 1024 + lane * 16) = (u32x4){v2[4], v2[5], v2[6], v2[7]};
                CFENCE;
            }
            unsigned o = sc2 - cnt2 + (unsigned)(size_t)S2;
#define CODES_OUT(i) *(LAS u32x4*)(L + ((i) * 64 + lane) * 16) = (u32x4){(mn[4 * (i)] & k07) | ex[4 * (i)], (mn[4 * (i) + 1] & k07) | ex[4 * (i) + 1], (mn[4 * (i) + 2] & k07) | ex[4 * (i) + 2], (mn[4 * (i) + 3] & k07) | ex[4 * (i) + 3]}
            merge<16>(ex, n8, o, lds + EX_MRG);
            CODES_OUT(0); CODES_OUT(1); CODES_OUT(2); CODES_OUT(3);
            if (G > 256) { merge<4>(ex + 16, n8 + 16, o, lds + EX_MRG); CODES_OUT(4); }
            CFENCE;
        }
        const unsigned cnt = popc4(m);
        unsigned off = wave_scan(cnt) - cnt + (unsigned)(size_t)L;
        WAIT_L(0);
        CYC(1);
        unsigned n1 = 0, x1 = 0;
        if ((unsigned)t1 < 1024u) {
            WAIT_V(0);
            const u32x2 ent = *(const LAS u32x2*)(L + EX_E + (t1 & 1) * 8);
            const unsigned mf = *(const LAS unsigned*)(L + EX_E + 16 + (t1 & 12));
            EX_ISSUE(t1, __builtin_amdgcn_readfirstlane(ent[0]), __builtin_amdgcn_readfirstlane(ent[1]), __builtin_amdgcn_readfirstlane(mf), n1, x1);
        }
        unsigned r2;
        DRAW(r2, tick);
        CYC(2);
        const unsigned so = out_soff(src, t0), vo = out_voff(lane);
        u32x2 sel[16]; unsigned of[16], w0[16], w1[16], w2[16];
#pragma unroll
        for (int g = 0; g < 16; ++g) {
            const unsigned b8 = ((g & 3) == 0 ? m[g >> 2] << 3 : m[g >> 2] >> (8 * (g & 3) - 3)) & 0x7f8u;
            sel[g] = *(const LAS u32x2*)(lds + EX_LUT + b8);
            const LAS unsigned* w = (const LAS unsigned*)(size_t)(off & ~3u);
            w0[g] = w[0]; w1[g] = w[1]; w2[g] = w[2]; of[g] = off;
            off += __builtin_popcount(b8);
        }
        SCHED;
        u32x4 o[8];
#pragma unroll
        for (int g = 0; g < 16; ++g) {
            const unsigned a0 = __builtin_amdgcn_alignbyte(w1[g], w0[g], of[g]), a1 = __builtin_amdgcn_alignbyte(w2[g], w1[g], of[g]);
            o[g >> 1][(g & 1) * 2] = __builtin_amdgcn_perm(a1, a0, sel[g][0]); o[g >> 1][(g & 1) * 2 + 1] = __builtin_amdgcn_perm(a1, a0, sel[g][1]);
        }
        CYC(3);
        if (dead) {
#pragma unroll
            for (int ch = 0; ch < 8; ++ch) o[ch] = (u32x4){0x7f7f7f7fu, 0x7f7f7f7fu, 0x7f7f7f7fu, 0x7f7f7f7fu};
            CFENCE;
        }
#pragma unroll
        for (int ch = 0; ch < 8; ++ch) ST128_RAW(o[ch], ro, vo, so + chunk_soff(ch), AUX_WT);
        ST128_PAD8(o);
        CYC(4);
        int t2;
        TAKE(t2, r2, 8); t2 = WT_OF(t2);
        CYC(5);
        t0 = t1; n0 = n1; x0 = x1; t1 = t2;
    }
    STAMP(myws, wv, 19);
}

__device__ __forceinline__ unsigned get_call(unsigned callv) { unsigned c; asm volatile("s_nop 0\n\tv_readfirstlane_b32 %0, %1" : "=s"(c) : "v"(callv)); return c + 1u; }

__device__ __forceinline__ void own_x(const Params& p, int j, int g, unsigned due, unsigned long long t0, int lane) {
    if (j < 2) return;
    while ((unsigned)(__builtin_amdgcn_s_memrealtime() - t0) < due) __builtin_amdgcn_s_sleep(4);
    u32x4 v[8];
    x_load(v, mk_rsrc(p.x, X_BYTES), j, g, lane);
    img_store(v, mk_rsrc(p.my, WS_USED), false, x_img(p.rank * 8 + j, g), lane);
    WAIT_V(0);
    if (lane == 0) add_sys(p.my + x_flag(p.rank * 8 + j, g));
}

__device__ __forceinline__ void comm_role(const Params& p, LAS unsigned char* lds, int c, int wid, int lane, unsigned callv) {
    const int rank = p.rank;
    char* myws = p.my;
    const rsrc_t rs = mk_rsrc(myws, WS_USED);
    const int wv = (NGEMM + c) * 8 + wid; (void)wv;
    STAMP(myws, wv, 0);
    if (wid == 0 && lane == 0) *(volatile LAS unsigned*)(lds + EX_TOK) = 0u;
    if (wid == 0 && lane < 16) *(LAS unsigned*)(lds + EX_MRG + lane * 8) = lut_merge(lane);
    if (wid == 2 || wid == 3) { *(LAS u32x2*)(lds + EX_LUT + ((wid & 1) * 128 + lane) * 8) = lut_expand((wid & 1) * 128 + lane); *(LAS u32x2*)(lds + EX_LUT + ((wid & 1) * 128 + 64 + lane) * 8) = lut_expand((wid & 1) * 128 + 64 + lane); }
    WAIT_L(0); BAR; CFENCE;
    if (wid >= 4) {
        const rsrc_t rx = mk_rsrc(p.x, X_BYTES), rw = mk_rsrc(p.w, W_BYTES);
        const bool far = wid < 6;
        const unsigned long long tk0 = __builtin_amdgcn_s_memrealtime();
        for (int u = (wid & 1) * 128 + c + (far ? 7 * NPUSH : 0); u < (far ? 1792 : 704); u += 256) {
            const bool ownw = !far && u >= 64 && u < 256;
            const int ux = u < 64 ? u : u - 192;
            const int j = far ? u / 224 : ownw ? u >> 5 : ux >> 6, r = far ? u % 224 : ownw ? 288 + (u & 31) : 224 + (ux & 63), t = rank * 8 + j;
            const unsigned due = far ? (j > 1 ? (j - 1) * PACE_W : j * PACE_W1) : ownw ? (j - 2) * PACE_E : j * PACE_X;
            while ((unsigned)(__builtin_amdgcn_s_memrealtime() - tk0) < due) __builtin_amdgcn_s_sleep(8);
            u32x4 v[8];
            if (r >= 224 && r < 288) {
                const int g = r - 224;
                x_load(v, rx, j, g, lane);
                pack_store(v, rs, lds + EX_0 + wid * EX_BYTES, OFF_PX + (unsigned)(j * 64 + g) * UNIT, lane);
                WAIT_V(0);
                const unsigned call = get_call(callv);
                if (j < NPUF) {
                    if (lane < 8) st_sys(p.pl + OFF_PUF + ((rank * NPUF + j) * 96 + 32 + g) * 4, call);
                    if (j == 0) STAMP(myws, wv, 7);
                }
                unsigned old = 0;
                if (lane == 0) old = __hip_atomic_fetch_add((GAS unsigned*)(myws + OFF_PCNT + j * 128), 1u, __ATOMIC_RELAXED, __HIP_MEMORY_SCOPE_AGENT);
                if ((__builtin_amdgcn_readfirstlane(old) & 63) == 63 && lane < 8) st_sys(p.pl + OFF_PFLG + t * 128, call);
            } else {
                const bool own = r >= 288;
                const int e = own ? r - 288 : r / 7, d = own ? rank : (rank + 1 + r % 7) & 7;
                u32x2 w[16];
                w_load(w, rw, j, d, e >> 2, e & 1, (e >> 1) & 1, lane);
                w_transpose(v, w);
                if (own) {
                    img_store(v, rs, true, w_img(t, e), lane);
                    WAIT_V(0);
                    if (lane == 0) add_sys(myws + w_flag(t, e));
                } else {
                    const unsigned id = (unsigned)((d * 8 + j) * 32 + e);
                    pack_store(v, rs, lds + EX_0 + wid * EX_BYTES, OFF_PW + id * UNIT, lane);
                    WAIT_V(0);
                    const unsigned call = get_call(callv);
                    if (j < NPUF) { if (lane == 1 + r % 7) st_sys(p.pl + OFF_PUF + ((rank * NPUF + j) * 96 + e) * 4, call); if (j == 0) STAMP(myws, wv, 6); }
                    unsigned old = 0;
                    if (lane == 0) old = __hip_atomic_fetch_add((GAS unsigned*)(myws + OFF_PCNT + (8 + d * 8 + j) * 128), 1u, __ATOMIC_RELAXED, __HIP_MEMORY_SCOPE_AGENT);
                    if (__builtin_amdgcn_readfirstlane(old) + 1 == call * (j ? 32 : 32 - NPUSH) && lane == 1 + r % 7) st_sys(p.pl + OFF_PFLG + t * 128 + 4, call);
                }
            }
            STAMP(myws, wv, far ? 1 + (j >> 2) : 3 + (j >> 2));
        }
    }
    if (wid == 1) {
        const unsigned long long tk0 = __builtin_amdgcn_s_memrealtime();
        if (NPUSH && c < 7 * PUSH_W) {
            const int d = (rank + 1 + c % 7) & 7, e0 = c / 7;
            const rsrc_t rw = mk_rsrc(p.w, W_BYTES), rd = mk_rsrc(p.peer(1 + c % 7), WS_USED);
            u32x2 w[PUSH_U ? PUSH_U : 1][16];
#pragma unroll
            for (int i = 0; i < PUSH_U; ++i) { const int e = e0 + i * PUSH_W; w_load(w[i], rw, 0, d, e >> 2, e & 1, (e >> 1) & 1, lane); }
#pragma unroll
            for (int i = 0; i < PUSH_U; ++i) {
                const int e = e0 + i * PUSH_W;
                u32x4 v[8];
                w_transpose(v, w[i]);
                img_store(v, rd, true, w_img(rank * 8, e), lane);
                WAIT_V(0);
                if (lane == 1 + c % 7) add_sys(p.pl + w_flag(rank * 8, e));
            }
            STAMP(myws, wv, 14);
        }
        for (int u = c; u < 256; u += 128) own_x(p, u >> 5, 32 + (u & 31), PACE_L0 + (u >> 5) * PACE_O, tk0, lane);
    }
    if (wid < CW) {
        const unsigned call = get_call(callv);
        const int ci = wid * 128 + c;
        if (wid < TEAM && c < 7 * PPM) pull_role(p, lds, lds + EX_0 + wid * EX_BYTES, c, lane, call, wv);
        if (ci < 7 * PPO) expand_role(p, lds, lds + EX_0 + wid * EX_BYTES, ci, lane, call, wv);
    }
}

typedef float f32x2 __attribute__((ext_vector_type(2)));
__device__ __forceinline__ f32x4 scale4(const f32x4& v, const f32x2& s) {
    f32x2 a, b;
    asm("v_pk_mul_f32 %0, %1, %2" : "=v"(a) : "v"((f32x2){v[0], v[1]}), "v"(s));
    asm("v_pk_mul_f32 %0, %1, %2" : "=v"(b) : "v"((f32x2){v[2], v[3]}), "v"(s));
    return (f32x4){a[0], a[1], b[0], b[1]};
}
__device__ __forceinline__ void gemm_role(const Params& p, LAS unsigned char* lds, int rt, int ct, int wid, int lane, unsigned callv) {
    const int rank = p.rank;
    char* myws = p.my;
    const int tile = rt * 8 + ct;
    STAMP(myws, tile * 8 + wid, 0);
    if (tile >= 64 && wid >= 1 && wid <= 4)
        own_x(p, (tile >> 2) & 7, (3 - (tile >> 5)) * 16 + (tile & 3) * 4 + wid - 1, tile >= 96 ? (((tile >> 2) & 7) - 2) * PACE_E : PACE_O0 + ((tile >> 2) & 7) * PACE_O,
              __builtin_amdgcn_s_memrealtime(), lane);
    const int wr = wid >> 2, wc = wid & 3;
    LAS unsigned char* dst = lds + 2 * HALFB + lane * 16 + (wr == 0 ? wid * 8192 : 4 * HALFB + (wc >> 1) * 8192 + (wc & 1) * 1024);
    u32x4 dv[8];
    if (rt >= DIR_E) { const unsigned long long t0_ = __builtin_amdgcn_s_memrealtime(); while ((unsigned)(__builtin_amdgcn_s_memrealtime() - t0_) < DIR_D) __builtin_amdgcn_s_sleep(4); }
    {
        u32x4 v0[8];
        if (wr == 0) {
            x_load(v0, mk_rsrc(p.x, X_BYTES), 0, rt * 4 + wid, lane); x_load(dv, mk_rsrc(p.x, X_BYTES), 1, rt * 4 + wid, lane);
        } else {
            u32x2 w[2][16];
            w_load<DW_AUX>(w[0], mk_rsrc(p.w, W_BYTES), 0, rank, ct, wc & 1, wc >> 1, lane); w_load<DW_AUX>(w[1], mk_rsrc(p.w, W_BYTES), 1, rank, ct, wc & 1, wc >> 1, lane);
            w_transpose(v0, w[0]);
#pragma unroll
            for (int i = 0; i < 8; ++i) dv[i] = (u32x4){w[1][2 * i][0], w[1][2 * i][1], w[1][2 * i + 1][0], w[1][2 * i + 1][1]};
        }
#pragma unroll
        for (int f = 0; f < 8; ++f) *(LAS u32x4*)(dst + frag_off(wr == 1, f)) = v0[f];
    }
    const unsigned call = get_call(callv);
    const int wv = tile * 8 + wid; (void)wv;
    const rsrc_t rs = mk_rsrc(myws, WS_USED);
    const unsigned voff = (unsigned)wid * 2048 + (unsigned)lane * 16;
    LAS unsigned char* ldsw = lds + wid * 2048;
    LAS unsigned char* lda = lds + wr * 8192 + lane * 16;
    LAS unsigned char* ldb = lds + wc * 4096 + lane * 16;
    const unsigned seqf = (unsigned)step_of(lane, rank, rt) * 4, fx = OFF_FLAG + rt * 256, fw = OFF_FLAG + FLAG_W + ct * 256, c4 = call * 4;
    unsigned long long todo = ~0ull, rdy = 0;
    unsigned tl = 0;
    volatile LAS unsigned* tq = (volatile LAS unsigned*)(lds + LDS_RED + 160);
#define SA(b, h) (((b) * 2 + (h)) * HALFB)
#define SB(b, h) ((4 + (b) * 2 + (h)) * HALFB)
#define STAGE(bufoff, soff) do { \
        __builtin_amdgcn_raw_ptr_buffer_load_lds(rs, (LAS void*)(ldsw + (bufoff)), 16, voff, (soff), 0, AUX_SYS); \
        __builtin_amdgcn_raw_ptr_buffer_load_lds(rs, (LAS void*)(ldsw + (bufoff)), 16, voff, (soff), 1024, AUX_SYS); } while (0)
#define LDA(dst, b, h) do { _Pragma("unroll") for (int m = 0; m < 4; ++m) _Pragma("unroll") for (int k = 0; k < 2; ++k) dst[m][k] = *(const LAS bf16x8*)(lda + SA(b, h) + m * 2048 + k * 1024); } while (0)
#define LDB(dst, b, h) do { _Pragma("unroll") for (int n = 0; n < 2; ++n) _Pragma("unroll") for (int k = 0; k < 2; ++k) dst[n][k] = *(const LAS bf16x8*)(ldb + SB(b, h) + n * 2048 + k * 1024); } while (0)
#define MMA(ai, bj, At, Bt) do { __builtin_amdgcn_s_setprio(1); _Pragma("unroll") for (int m = 0; m < 4; ++m) _Pragma("unroll") for (int n = 0; n < 2; ++n) _Pragma("unroll") for (int k = 0; k < 2; ++k) \
        acc[ai][bj][m][n] = __builtin_amdgcn_mfma_f32_16x16x32_bf16(Bt[n][k], At[m][k], acc[ai][bj][m][n], 0, 0, 0); __builtin_amdgcn_s_setprio(0); } while (0)
#define OFFA(t) (OFF_XT + (unsigned)(t) * XT_STEP + (unsigned)rt * BLK)
#define OFFB(t) (OFF_WT + (unsigned)(t) * WT_STEP + (unsigned)ct * BLK)
    unsigned la = 0, lb = 0;
#define LOOK do { if (wid == 0) { WAIT_V(8); la = *(const LAS unsigned*)(lds + LDS_FLAG + lane * 4); lb = *(const LAS unsigned*)(lds + LDS_FLAG + 256 + lane * 4); } } while (0)
#define PICK(slot) do { if (wid == 0) { \
            if (todo != 0) { \
                rdy |= __builtin_amdgcn_ballot_w64(la >= c4 && lb >= c4); \
                if ((rdy & todo) == 0) { Spin t0_; \
                    for (;;) { CFENCE; \
                        const unsigned a_ = __builtin_amdgcn_raw_buffer_load_b32(rs, seqf, fx, AUX_SYS), b_ = __builtin_amdgcn_raw_buffer_load_b32(rs, seqf, fw, AUX_SYS); \
                        rdy |= __builtin_amdgcn_ballot_w64(a_ >= c4 && b_ >= c4); \
                        if ((rdy & todo) != 0) break; if (t0_.fail(myws, call)) { rdy = ~0ull; break; } } } \
                const int pos_ = __builtin_ctzll(rdy & todo); \
                todo &= ~(1ull << pos_); tl = __builtin_amdgcn_readlane(seqf, pos_) >> 2; \
            } \
            tq[slot] = tl; \
            const unsigned skip_ = (todo & ~rdy) == 0 ? 0xf0000000u : 0u;        \
            __builtin_amdgcn_raw_ptr_buffer_load_lds(rs, (LAS void*)(lds + LDS_FLAG), 4, seqf, fx + skip_, 0, AUX_SYS); \
            __builtin_amdgcn_raw_ptr_buffer_load_lds(rs, (LAS void*)(lds + LDS_FLAG + 256), 4, seqf, fw + skip_, 0, AUX_SYS); \
            SCHED; } } while (0)

    f32x4 acc[2][2][4][2];
#pragma unroll
    for (int a = 0; a < 2; ++a)
#pragma unroll
        for (int b = 0; b < 2; ++b)
#pragma unroll
            for (int m = 0; m < 4; ++m)
#pragma unroll
                for (int n = 0; n < 2; ++n) acc[a][b][m][n] = (f32x4){0.f, 0.f, 0.f, 0.f};
    bf16x8 At[4][2], B0[2][2], B1[2][2];

#define WAIT2 do { if (wid == 0) WAIT_V(10); else WAIT_V(8); WAIT_L(0); } while (0)
#define STEP2(b) { \
        LOOK; \
        LDB(B0, b, 0); LDA(At, b, 0); LDB(B1, b, 1); \
        PICK(b); \
        STAGE(SA(b ^ 1, 1), OFFA(t1) + HALFB); \
        WAIT2; BAR; \
        const unsigned tv = tq[b]; \
        MMA(0, 0, At, B0); MMA(0, 1, At, B1); BAR; SCHED; \
        t1 = __builtin_amdgcn_readfirstlane(tv); \
        LDA(At, b, 1); \
        STAGE(SB(b, 0), OFFB(t1)); STAGE(SB(b, 1), OFFB(t1) + HALFB); STAGE(SA(b, 0), OFFA(t1)); \
        WAIT2; BAR; \
        MMA(1, 0, At, B0); MMA(1, 1, At, B1); BAR; SCHED; }
    unsigned t1;
    {
#define PLAIN_STEP { LDB(B0, 1, 0); LDA(At, 1, 0); LDB(B1, 1, 1); MMA(0, 0, At, B0); MMA(0, 1, At, B1); LDA(At, 1, 1); MMA(1, 0, At, B0); MMA(1, 1, At, B1); }
        todo = ~3ull;
        if (wid == 1 && lane < 8) ((LAS unsigned*)(lds + LDS_RED))[32 + lane] = 0u;
        WAIT_L(0); BAR; CFENCE;
        if (wid == 0) {
            __builtin_amdgcn_raw_ptr_buffer_load_lds(rs, (LAS void*)(lds + LDS_FLAG), 4, seqf, fx, 0, AUX_SYS);
            __builtin_amdgcn_raw_ptr_buffer_load_lds(rs, (LAS void*)(lds + LDS_FLAG + 256), 4, seqf, fw, 0, AUX_SYS);
        }
        PLAIN_STEP;
        SCHED;
        if (wr == 1) {
            u32x2 w[16];
#pragma unroll
            for (int i = 0; i < 8; ++i) { w[2 * i] = (u32x2){dv[i][0], dv[i][1]}; w[2 * i + 1] = (u32x2){dv[i][2], dv[i][3]}; }
            w_transpose(dv, w);
        }
        WAIT_V(0); LOOK;
        WAIT_L(0); BAR; CFENCE;
#pragma unroll
        for (int f = 0; f < 8; ++f) *(LAS u32x4*)(dst + frag_off(wr == 1, f)) = dv[f];
        PICK(2);
        WAIT_L(0); BAR; CFENCE;
        STAMP(myws, wv, 1);
        const unsigned t0 = __builtin_amdgcn_readfirstlane(tq[2]);
        STAGE(SB(0, 0), OFFB(t0)); STAGE(SB(0, 1), OFFB(t0) + HALFB); STAGE(SA(0, 0), OFFA(t0)); STAGE(SA(0, 1), OFFA(t0) + HALFB);
        SCHED;
        PLAIN_STEP;
        LOOK; PICK(3);
        WAIT_L(0); BAR; CFENCE;
        STAMP(myws, wv, 14);
        t1 = __builtin_amdgcn_readfirstlane(tq[3]);
        STAGE(SB(1, 0), OFFB(t1)); STAGE(SB(1, 1), OFFB(t1) + HALFB); STAGE(SA(1, 0), OFFA(t1));
        WAIT_V(6); BAR;
        if (wr == 1) BAR;
    }
    for (int i = 2; i < NT; i += 2) {
        STEP2(0);
        STEP2(1);
    }
    char* pl = p.pl;
    if (wr == 0) BAR;
    STAMP(myws, wv, 10);

    const int wt = tile * 8 + wid;
    LAS unsigned* red = (LAS unsigned*)(lds + LDS_RED);
    u32x4 msk;
#pragma unroll
    for (int w = 0; w < 4; ++w) {
        unsigned mm = 0;
#pragma unroll
        for (int e = 31; e >= 0; --e) {
            const int ch = 2 * w + (e >> 4);
            mm = __builtin_amdgcn_alignbit(mm, __float_as_uint(acc[ch >> 2][(e >> 1) & 1][ch & 3][e & 1][(e >> 2) & 3]), 31);
        }
        msk[w] = ~mm;
    }
    ST128(msk, rs, lane * 16, OFF_MSK + wt * 1024, AUX_SYS);
    SCHED;
    float mx = 0.f;
#pragma unroll
    for (int a = 0; a < 2; ++a)
#pragma unroll
        for (int b = 0; b < 2; ++b)
#pragma unroll
            for (int m = 0; m < 4; ++m)
#pragma unroll
                for (int n = 0; n < 2; ++n) mx = fmaxf(fmaxf(fmaxf(fmaxf(mx, acc[a][b][m][n][0]), acc[a][b][m][n][1]), acc[a][b][m][n][2]), acc[a][b][m][n][3]);
    const unsigned wm = wave_max(__float_as_uint(mx));
    {
        unsigned old = 0;
        if (lane == 0) {
            __hip_atomic_fetch_max(red + 32, wm, __ATOMIC_RELAXED, __HIP_MEMORY_SCOPE_WORKGROUP);
            old = __hip_atomic_fetch_add(red + 33, 1u, __ATOMIC_RELAXED, __HIP_MEMORY_SCOPE_WORKGROUP);
        }
        if (__builtin_amdgcn_readfirstlane(old) == 7) {
            const unsigned long long gv = ((unsigned long long)call << 32) | __hip_atomic_load(red + 32, __ATOMIC_RELAXED, __HIP_MEMORY_SCOPE_WORKGROUP);
            char* g = pl + OFF_AMAX + (rank * NGEMM + tile) * 8;
            unsigned long long ex_;
            asm volatile("s_mov_b64 %0, exec\n\ts_mov_b64 exec, 0xff\n\tglobal_store_dwordx2 %1, %2, off sc0 sc1\n\ts_mov_b64 exec, %0\n\ts_waitcnt vmcnt(1)" : "=&s"(ex_) : "v"(g), "v"(gv) : "memory");
        } else WAIT_V(0);
        unsigned old2 = 0;
        if (lane == 0) old2 = __hip_atomic_fetch_add(red + 34 + wr, 1u, __ATOMIC_RELAXED, __HIP_MEMORY_SCOPE_WORKGROUP);
        if (__builtin_amdgcn_readfirstlane(old2) == 3 && lane < 8) st_sys(pl + OFF_MFLG + (rank * 256 + tile * 2 + wr) * 4, call);
    }
#pragma unroll
    for (int a = 0; a < 2; ++a)
#pragma unroll
        for (int b = 0; b < 2; ++b)
#pragma unroll
            for (int m = 0; m < 4; ++m)
#pragma unroll
                for (int n = 0; n < 2; ++n)
#pragma unroll
                    for (int j = 0; j < 4; ++j) { acc[a][b][m][n][j] = fmaxf(acc[a][b][m][n][j], 0.f); if (j == 3) asm volatile("" : "+v"(acc[a][b][m][n])); }
    STAMP(myws, wv, 13);
    BAR; CFENCE;
    LAS unsigned char* A = lds + wid * EP_BYTES;
    LAS unsigned char* B = A + EP_C;
    LAS unsigned char* S2 = A + EP_S;
    const unsigned cnt = popc4(msk), scan = wave_scan(cnt), Lc = __builtin_amdgcn_readlane(scan, 63);
    const bool dense = Lc > L1_MAX;
    if (!dense) {
        unsigned o = (unsigned)(size_t)A + (scan - cnt) * 4;
        const unsigned spare = (unsigned)(size_t)(A + EP_X) + lane * 4;
#pragma unroll
        for (int e = 0; e < 128; ++e) {
            const int ch = e >> 4;
            const unsigned bit = __builtin_amdgcn_ubfe(msk[e >> 5], e & 31, 1);
            *(LAS float*)(size_t)(bit ? o ^ ((o >> 4) & 0x30u) : spare) = acc[ch >> 2][(e >> 1) & 1][ch & 3][e & 1][(e >> 2) & 3];
            o += bit * 4;
        }
    }
    float ysum = 0.f;
#pragma unroll
    for (int a = 0; a < 2; ++a)
#pragma unroll
        for (int b = 0; b < 2; ++b)
#pragma unroll
            for (int m = 0; m < 4; ++m) { const f32x4 t = acc[a][b][m][0] + acc[a][b][m][1]; ysum += (t[0] + t[1]) + (t[2] + t[3]); }
#pragma unroll
    for (int sh = 32; sh >= 1; sh >>= 1) ysum += __shfl_xor(ysum, sh);
    ysum *= 1.35f / (float)max(Lc, 1u);
    if (!dense && lane < 16 && Lc + lane < L1_MAX) *(LAS float*)(A + (((Lc + lane) * 4) ^ (((Lc + lane) >> 2) & 0x30u))) = ysum;
    LAS unsigned char* lut = A + EP_T;
    if (lane < 16) *(LAS unsigned*)(lut + lane * 4) = lut_compact(lane);
    {
        if (wid == 0) {
            Spin t0;
            for (;;) {
                CFENCE;
                const unsigned g = __builtin_amdgcn_raw_buffer_load_b32(rs, (lane & 31) * 8 + 4, OFF_AMAX + (rank * NGEMM + 96) * 8, AUX_SYS);
                if (__builtin_popcountll(__builtin_amdgcn_ballot_w64(g >= call)) >= 48 || t0.fail(myws, call)) break;
                __builtin_amdgcn_s_sleep(8);
            }
        }
        BAR; CFENCE;
    }
    {
        Spin t0;
        unsigned t, tm = ld_agent(myws + OFF_TMO);
        for (;;) {
            CFENCE;
            const u32x4 g = __builtin_amdgcn_raw_buffer_load_b128(rs, lane * 16, OFF_AMAX + wid * 1024, AUX_SYS);
            t = max(g[0], g[2]);
            if (__builtin_amdgcn_ballot_w64(g[1] < call || g[3] < call) == 0) break;
            if (t0.fail(myws, call)) { tm = call; break; }
            __builtin_amdgcn_s_sleep(4);
        }
        t = wave_max(t);
        if (lane == 0) { red[8 + wid] = t; red[wid] = tm == call; }
    }
    WAIT_L(0); BAR; CFENCE;
    const float amax = __uint_as_float(wave_max(red[8 + (lane & 7)]));
    const bool dead = __builtin_amdgcn_ballot_w64(red[lane & 7] != 0) != 0;
    const float inv = dead ? __builtin_nanf("") : 448.0f / amax;
    STAMP(myws, wv, 11);
    if (wr == 0) __builtin_amdgcn_s_setprio(2);

    const f32x2 inv2 = {inv, inv};
#define QUANT(ch, q) { const int a = (ch) >> 2, m = (ch) & 3; \
        const f32x4 y0 = scale4(acc[a][0][m][0], inv2), y1 = scale4(acc[a][0][m][1], inv2), y2 = scale4(acc[a][1][m][0], inv2), y3 = scale4(acc[a][1][m][1], inv2); \
        _Pragma("unroll") for (int j = 0; j < 4; ++j) \
            q[j] = (unsigned)__builtin_amdgcn_cvt_pk_fp8_f32(y2[j], y3[j], __builtin_amdgcn_cvt_pk_fp8_f32(y0[j], y1[j], __float_as_int(y0[j]), false), true); }
    u32x4 cod[5];
    if (dense) {
#pragma unroll
        for (int r = 0; r < 9; ++r) *(LAS u32x4*)(A + r * 1024 + lane * 16) = (u32x4){0u, 0u, 0u, 0u};
        unsigned o = (unsigned)(size_t)A + scan - cnt;
#pragma unroll
        for (int ch = 0; ch < 8; ++ch) {
            u32x4 q;
            QUANT(ch, q);
#pragma unroll
            for (int j = 0; j < 4; ++j) {
                const unsigned n4 = nib4(msk[ch >> 1], (ch & 1) * 4 + j);
                const unsigned long long z = (unsigned long long)__builtin_amdgcn_perm(0u, q[j], *(const LAS unsigned*)(lut + n4)) << ((o & 3u) * 8);
                LAS unsigned* d = (LAS unsigned*)(size_t)(o & ~3u);
                __hip_atomic_fetch_or(d, (unsigned)z, __ATOMIC_RELAXED, __HIP_MEMORY_SCOPE_WAVEFRONT);
                __hip_atomic_fetch_or(d + 1, (unsigned)(z >> 32), __ATOMIC_RELAXED, __HIP_MEMORY_SCOPE_WAVEFRONT);
                o += __builtin_popcount(n4);
            }
        }
    } else {
#define CODES(i0, n) { f32x4 y_[n][4]; \
            _Pragma("unroll") for (int i = 0; i < n; ++i) _Pragma("unroll") for (int k = 0; k < 4; ++k) y_[i][k] = *(const LAS f32x4*)(A + ((i0 + i) * 64 + (i0 + i < 4 ? lane : lane & 31)) * 64 + ((k ^ (lane >> 2)) & 3) * 16); \
            SCHED; \
            _Pragma("unroll") for (int i = 0; i < n; ++i) _Pragma("unroll") for (int k = 0; k < 4; ++k) { const f32x4 y = scale4(y_[i][k], inv2); \
                cod[i0 + i][k] = (unsigned)__builtin_amdgcn_cvt_pk_fp8_f32(y[2], y[3], __builtin_amdgcn_cvt_pk_fp8_f32(y[0], y[1], __float_as_int(y[0]), false), true); } }
        CODES(0, 2); CODES(2, 2);
        cod[4] = (u32x4){0u, 0u, 0u, 0u};
        if (Lc > 4096) CODES(4, 1);
        CFENCE;
#pragma unroll
        for (int i = 0; i < 5; ++i) *(LAS u32x4*)(A + (i * 64 + lane) * 16) = cod[i];
#pragma unroll
        for (int r = 0; r < 3; ++r) *(LAS u32x4*)(S2 + r * 1024 + lane * 16) = (u32x4){0u, 0u, 0u, 0u};
    }
    CFENCE;
    CYC(0);
    const unsigned ec = (__builtin_amdgcn_readfirstlane(__builtin_amdgcn_cvt_pk_fp8_f32(fminf(ysum * inv, 448.f), 0.f, 0, false)) >> 3) & 15u;
    const unsigned E1 = (ec + 17) * 0x01010101u, G = (Lc + 15) >> 4;
    bool raw = dense || (rt * 2 + wr) * 32 + ct * 4 + wc >= 1024 - RAW_LAST;
    unsigned bytes = Lc;
    if (!raw) {
        unsigned s2[5][4], n7[5][4], cnt2 = 0;
        const unsigned k07 = KS_(0x07070707u), k08 = KS_(0x08080808u), k0f = KS_(0x0f0f0f0fu), k10 = KS_(0x10101010u), k80 = KS_(H8);
#define CLASSIFY(i0, n) { const u32x4* d_ = cod + i0; \
            _Pragma("unroll") for (int i = 0; i < n; ++i) { const unsigned g = (i0 + i) * 64 + lane, wgt = g < G ? 0x08040201u : 0u; unsigned nb[4], p_[4]; \
                _Pragma("unroll") for (int k = 0; k < 4; ++k) { \
                    const unsigned e4 = (d_[i][k] >> 3) & k0f, u = E1 - e4, t = u & k08, ui = (u | (t - (t >> 3))) & k07; \
                    p_[k] = __builtin_amdgcn_perm(0x98989898u, 0x10080098u, ui); \
                    s2[i0 + i][k] = e4 | __builtin_amdgcn_perm(0x30302010u, 0x30303000u, ui); \
                    nb[k] = (d_[i][k] & k07) | (p_[k] & k08); \
                    n7[i0 + i][k] = __builtin_amdgcn_udot4(p_[k] & k80, wgt, 0u, false); \
                    cnt2 += __builtin_popcount(n7[i0 + i][k]); } \
                if (g < G) { *(LAS u32x2*)(B + g * 8) = (u32x2){nb[0] | (nb[1] << 4), nb[2] | (nb[3] << 4)}; \
                    *(LAS unsigned short*)(B + G * 8 + g * 2) = (unsigned short)bits16(p_[0] & k10, p_[1] & k10, p_[2] & k10, p_[3] & k10, 4); } } }
        CLASSIFY(0, 4);
#pragma unroll
        for (int k = 0; k < 4; ++k) s2[4][k] = n7[4][k] = 0;
        if (G > 256) CLASSIFY(4, 1);
        CYC(1);
        const unsigned sc2 = wave_scan(cnt2), n2 = __builtin_amdgcn_readlane(sc2, 63), G2 = (n2 + 15) >> 4, o2 = off_l2(G), o3 = o2 + G2 * 4;
        raw = n2 > L2_MAX;
        if (!raw) {
            unsigned o = sc2 - cnt2 + (unsigned)(size_t)S2;
#define SQUEEZE(i0, n) { unsigned se_[n][4]; \
                _Pragma("unroll") for (int i = 0; i < n; ++i) _Pragma("unroll") for (int k = 0; k < 4; ++k) se_[i][k] = *(const LAS unsigned*)(lut + (n7[i0 + i][k] >> 5)); \
                SCHED; \
                _Pragma("unroll") for (int i = 0; i < n; ++i) _Pragma("unroll") for (int k = 0; k < 4; ++k) { \
                    const unsigned long long z = (unsigned long long)__builtin_amdgcn_perm(0u, s2[i0 + i][k], se_[i][k]) << ((o & 3u) * 8); \
                    LAS unsigned* d = (LAS unsigned*)(size_t)(o & ~3u); \
                    __hip_atomic_fetch_or(d, (unsigned)z, __ATOMIC_RELAXED, __HIP_MEMORY_SCOPE_WAVEFRONT); \
                    __hip_atomic_fetch_or(d + 1, (unsigned)(z >> 32), __ATOMIC_RELAXED, __HIP_MEMORY_SCOPE_WAVEFRONT); \
                    o += __builtin_popcount(n7[i0 + i][k]); } }
            SQUEEZE(0, 4);
            if (G > 256) SQUEEZE(4, 1);
            CFENCE;
            CYC(2);
            const u32x4 d0 = *(const LAS u32x4*)(S2 + lane * 16), d1 = *(const LAS u32x4*)(S2 + 1024 + lane * 16);
            if (lane < G2) *(LAS unsigned*)(B + o2 + lane * 4) = ((d0[0] >> 4) & 0x03030303u) | ((d0[1] >> 2) & 0x0c0c0c0cu) | (d0[2] & 0x30303030u) | ((d0[3] << 2) & 0xc0c0c0c0u);
            if (64 + lane < G2) *(LAS unsigned*)(B + o2 + 256 + lane * 4) = ((d1[0] >> 4) & 0x03030303u) | ((d1[1] >> 2) & 0x0c0c0c0cu) | (d1[2] & 0x30303030u) | ((d1[3] << 2) & 0xc0c0c0c0u);
            unsigned m5[8], se[8], cnt3 = 0;
#pragma unroll
            for (int k = 0; k < 8; ++k) {
                const unsigned d = k < 4 ? d0[k & 3] : d1[k & 3];
                m5[k] = __builtin_amdgcn_udot4(d & (d << 1) & 0x20202020u, 0x08040201u, 0u, false);
                se[k] = *(const LAS unsigned*)(lut + (m5[k] >> 3));
                cnt3 += __builtin_popcount(m5[k]);
            }
            const unsigned sc3 = wave_scan(cnt3), n3 = __builtin_amdgcn_readlane(sc3, 63);
            bytes = o3 + ((n3 + 7) >> 3) * 4;
            raw = n3 > MAXESC || bytes > Lc;
            if (!raw) {
                o = sc3 - cnt3 + (unsigned)(size_t)(A + EP_L);
#pragma unroll
                for (int k = 0; k < 8; ++k) {
                    const unsigned long long z = (unsigned long long)__builtin_amdgcn_perm(0u, k < 4 ? d0[k & 3] : d1[k & 3], se[k]) << ((o & 3u) * 8);
                    LAS unsigned* d = (LAS unsigned*)(size_t)(o & ~3u);
                    __hip_atomic_fetch_or(d, (unsigned)z, __ATOMIC_RELAXED, __HIP_MEMORY_SCOPE_WAVEFRONT);
                    __hip_atomic_fetch_or(d + 1, (unsigned)(z >> 32), __ATOMIC_RELAXED, __HIP_MEMORY_SCOPE_WAVEFRONT);
                    o += __builtin_popcount(m5[k]);
                }
                CFENCE;
                const u32x2 x3 = *(const LAS u32x2*)(A + EP_L + lane * 8);
                *(LAS unsigned*)(B + o3 + lane * 4) = (x3[0] & 0x0f0f0f0fu) | ((x3[1] << 4) & 0xf0f0f0f0u);
                CFENCE;
            }
        }
    }
    if (raw) bytes = Lc;
    CYC(3);
    {
        LAS unsigned char* S = raw ? A : B;
#pragma nounroll
        for (unsigned r = 0; r < bytes; r += 1024)
            ST128(*(const LAS u32x4*)(S + r + lane * 16), rs, lane * 16, OFF_COD + wt * 8192 + r, AUX_SYS);
    }
    if (lane == 0) red[24 + wid] = (raw ? 1u << 30 : (bytes >> 2) << 14) | (ec << 26) | Lc;
    CYC(4);
    WAIT_V(0);
    CYC(5);
    {
        unsigned old = 0;
        if (lane == 0) old = __hip_atomic_fetch_add(red + 36 + wr, 1u, __ATOMIC_RELAXED, __HIP_MEMORY_SCOPE_WORKGROUP);
        if (__builtin_amdgcn_readfirstlane(old) == 3 && lane < 32 && (lane & 7) != 0) {
            const int w = wr * 4 + (lane >> 3);
            __hip_atomic_store((GAS unsigned long long*)(pl + OFF_DONE + (rank * 1024 + tile * 8 + w) * 8),
                               ((unsigned long long)((volatile LAS unsigned*)red)[24 + w] << 32) | call, __ATOMIC_RELAXED, __HIP_MEMORY_SCOPE_SYSTEM);
        }
    }
    STAMP(myws, wv, 12);
    {
        const rsrc_t ro = mk_rsrc(p.out, OUT_BYTES);
        const unsigned so = out_soff(rank, wt), vo = out_voff(lane);
        u32x4 qv[8];
#pragma unroll
        for (int ch = 0; ch < 8; ++ch) QUANT(ch, qv[ch]);
#pragma unroll
        for (int ch = 0; ch < 8; ++ch) ST128_RAW(qv[ch], ro, vo, so + chunk_soff(ch), AUX_WT);
        ST128_PAD8(qv);
    }
}

__global__ __launch_bounds__(512, 2) void fused_kernel(Args a) {
    __shared__ __attribute__((aligned(16))) unsigned char lds_arr[LDS_BYTES];
    LAS unsigned char* lds = (LAS unsigned char*)lds_arr;
    const int wid = __builtin_amdgcn_readfirstlane(threadIdx.x >> 6), lane = threadIdx.x & 63;
    const int b = blockIdx.x, xcd = b & 7, idx = b >> 3;
    Params p;
    p.x = a.x; p.w = a.w; p.out = a.out; p.my = (char*)a.ws[0]; p.rank = a.rank; p.pl = (char*)a.ws[lane & 7];
    asm volatile("" : "+s"(p.x), "+s"(p.w), "+s"(p.out), "+s"(p.my), "+s"(p.rank));
    const unsigned callv = (unsigned)__builtin_amdgcn_raw_ptr_buffer_atomic_add_i32(1, mk_rsrc(p.my, WS_USED), lane == 0 ? OFF_CNT + (b * 8 + wid) * 4 : 0xfffffff0u, 0, 0);
    if (idx < 16) gemm_role(p, lds, (xcd >> 1) * 4 + (idx >> 2), (xcd & 1) * 4 + (idx & 3), wid, lane, callv);
    else comm_role(p, lds, (idx - 16) * 8 + xcd, wid, lane, callv);
}

extern "C" void kernel_launch(int rank, int nranks, ncclComm_t comm, hipStream_t stream,
                              void* const* d_in, const int* in_sizes, int n_in,
                              void* d_out, int out_size,
                              const int* in_dtypes, int out_dtype,
                              void* const* d_out_all, void* const* d_ws_all, size_t ws_bytes) {
    (void)comm; (void)in_dtypes; (void)out_dtype; (void)d_out_all;
    if (nranks != NR || n_in != 2 || in_sizes[0] != M_ * KS || in_sizes[1] != KS * N_ || out_size != M_ * N_ || ws_bytes < WS_USED || (unsigned)rank >= (unsigned)NR) {
        fprintf(stderr, "kernel_launch: unexpected problem shape\n");
        return;
    }
    Args a;
    a.x = d_in[0]; a.w = d_in[1]; a.out = d_out;
    const size_t shift = (ws_bytes - WS_USED) & ~(size_t)0x1fffff;
    for (int i = 0; i < NR; ++i) a.ws[i] = (char*)d_ws_all[(rank + i) % NR] + shift;
    a.rank = rank; a.pad = 0;
    fused_kernel<<<dim3(NWG), dim3(512), 0, stream>>>(a);
}
```

```cpp
#include <hip/hip_runtime.h>
#include <rccl/rccl.h>
#include <cstdio>

#define LAS __attribute__((address_space(3)))
#define GAS __attribute__((address_space(1)))
typedef short bf16x8 __attribute__((ext_vector_type(8)));
typedef float f32x4 __attribute__((ext_vector_type(4)));
typedef unsigned u32x4 __attribute__((ext_vector_type(4)));
typedef unsigned u32x2 __attribute__((ext_vector_type(2)));
typedef __amdgpu_buffer_rsrc_t rsrc_t;

constexpr int NR = 8;
constexpr int M_ = 4096, N_ = 16384;
constexpr int KS = 512;
constexpr int NS = 2048;
constexpr int NT = 64;
constexpr int NGEMM = 128;
constexpr int NWG = 256;

constexpr unsigned OFF_CNT  = 0;
constexpr unsigned OFF_TMO  = 16384;
constexpr unsigned OFF_FLAG = 65536;
constexpr unsigned FLAG_W = 4096;
constexpr unsigned OFF_AMAX = 131072;
constexpr unsigned OFF_TICK = 143360;
constexpr unsigned OFF_PCNT = 147456;
constexpr unsigned OFF_PFLG = 163840;
constexpr unsigned OFF_PUF  = 188416;
constexpr unsigned OFF_MFLG = 176128;
constexpr unsigned OFF_DONE = 786432;
constexpr unsigned OFF_MF   = OFF_DONE + 65536;
constexpr unsigned OFF_DBG  = 262144;
constexpr unsigned CTRL_BYTES = 1u << 20;
constexpr unsigned XT_STEP = 512u * 1024u, WT_STEP = 256u * 1024u, BLK = 32768u, HALFB = 16384u;
constexpr unsigned OFF_XT = CTRL_BYTES;
constexpr unsigned XT_BYTES = NT * XT_STEP;
constexpr unsigned OFF_WT = OFF_XT + XT_BYTES;
constexpr unsigned WT_BYTES = NT * WT_STEP;
constexpr unsigned UNIT = 9216;
constexpr unsigned OFF_PX = OFF_WT + WT_BYTES;
constexpr unsigned OFF_PW = OFF_PX + 512 * UNIT;
constexpr unsigned OFF_MSK = OFF_PW + 2048 * UNIT;
constexpr unsigned OFF_COD = OFF_MSK + 1024 * 1024;
constexpr unsigned OFF_LM = OFF_COD + 1024 * 8192;
constexpr unsigned WS_USED = OFF_LM + 8 * 1024 * 1024;
constexpr unsigned OUT_BYTES = (unsigned)M_ * N_;
constexpr unsigned X_BYTES = (unsigned)M_ * KS * 2, W_BYTES = (unsigned)KS * N_ * 2;

constexpr unsigned long long DEADLINE = 100000000ull;
constexpr int AUX_SYS = 17;
#define WS_END 1
constexpr int AUX_NT = 2, AUX_WT = 16;
#define PPP 14
#define PPM 14
#define TEAM 4
#define PPO 55
#define CW 4
#define PUSH_W 8
#define PUSH_U 3
#define SCH_S 100
#define SCH_C 1000
#define SCH_B 10
#define SCH_A 1
constexpr int RAW_LAST = 24;
constexpr int STAG = 7;
constexpr int NPUF = 2;
constexpr int PACE_W = 800, PACE_W1 = 150, PACE_X = 400, PACE_E = 70, PACE_O = 60, PACE_O0 = 150, PACE_L0 = 300;

constexpr unsigned LDS_STAGE = 8 * HALFB;
constexpr unsigned EP_BYTES = 19456;
constexpr unsigned EP_C = 8192, EP_S = EP_C + 5888, EP_L = EP_S + 2080, EP_T = 18432, EP_X = EP_T + 64;
constexpr unsigned LDS_FLAG = 8 * EP_BYTES;
constexpr unsigned LDS_RED = LDS_FLAG + 512;
constexpr unsigned EX_B = 8192, EX_M = EX_B + 8192, EX_P = EX_M + 1024, EX_E = EX_P + 1024, EX_BYTES = EX_E + 256, EX_LUT = 0, EX_MRG = 2048, EX_TOK = 2176, EX_0 = 2304;
constexpr unsigned LDS_BYTES = LDS_RED + 256;
static_assert(EX_0 + 8 * EX_BYTES <= LDS_BYTES && EP_BYTES % 1024 == 0 && EP_L + 1024 <= EP_T && EP_X + 256 <= EP_BYTES && LDS_STAGE <= LDS_FLAG && LDS_RED + 192 <= LDS_BYTES && LDS_BYTES <= 163840, "lds");
static_assert(OFF_PUF + 8 * NPUF * 384 <= OFF_DBG, "flags");
static_assert((PUSH_W * PUSH_U) % 4 == 0 && PUSH_W * PUSH_U <= 32 && 7 * PUSH_W <= NGEMM, "push");
static_assert(7 * PPO <= 128 * CW && 7 * PPM <= 128 && PPP <= PPM && TEAM <= CW, "waves");

struct Args {
    const void* x; const void* w; void* out;
    int rank; int pad;
    void* ws[NR];
};
struct Params {
    const void* x; const void* w; void* out; char* my; char* pl;
    int rank;
    __device__ __forceinline__ char* peer(int k) const {
        const unsigned long long a = (unsigned long long)pl;
        return (char*)(((unsigned long long)(unsigned)__builtin_amdgcn_readlane((unsigned)(a >> 32), k) << 32) | (unsigned)__builtin_amdgcn_readlane((unsigned)a, k));
    }
};

__device__ __forceinline__ rsrc_t mk_rsrc(const void* p, unsigned bytes) {
    return __builtin_amdgcn_make_buffer_rsrc((void*)p, 0, bytes, 0x00020000);
}
#define ST128(v, r, vo, so, aux) do { const u32x4 v_ = (v); __builtin_amdgcn_raw_buffer_store_b128(v_, r, vo, so, aux); asm volatile("s_nop 1" :: "v"(v_) : "memory"); } while (0)
#define ST128_RAW __builtin_amdgcn_raw_buffer_store_b128
#define ST128_PAD4(v) asm volatile("s_nop 1" :: "v"((v)[0]), "v"((v)[1]), "v"((v)[2]), "v"((v)[3]) : "memory")
#define ST128_PAD8(v) asm volatile("s_nop 1" :: "v"((v)[0]), "v"((v)[1]), "v"((v)[2]), "v"((v)[3]), "v"((v)[4]), "v"((v)[5]), "v"((v)[6]), "v"((v)[7]) : "memory")
#define WAIT_V(n) asm volatile("s_waitcnt vmcnt(" #n ")" ::: "memory")
#define WAIT_L(n) asm volatile("s_waitcnt lgkmcnt(" #n ")" ::: "memory")
#define BAR __builtin_amdgcn_s_barrier()
#define SCHED __builtin_amdgcn_sched_barrier(0)
#define CFENCE asm volatile("" ::: "memory")

#define STAMP(ws_, wv_, k_) do {} while (0)
#define CYC(i) do {} while (0)

__device__ __forceinline__ unsigned ld_agent(const void* p) { return __hip_atomic_load((const GAS unsigned*)p, __ATOMIC_RELAXED, __HIP_MEMORY_SCOPE_AGENT); }
__device__ __forceinline__ void st_agent(void* p, unsigned v) { __hip_atomic_store((GAS unsigned*)p, v, __ATOMIC_RELAXED, __HIP_MEMORY_SCOPE_AGENT); }
__device__ __forceinline__ void st_sys(void* p, unsigned v) { __hip_atomic_store((GAS unsigned*)p, v, __ATOMIC_RELAXED, __HIP_MEMORY_SCOPE_SYSTEM); }

__device__ __forceinline__ void add_sys(void* p) { asm volatile("global_atomic_add %0, %1, off sc1" :: "v"(p), "v"(1u) : "memory"); }
__device__ __forceinline__ unsigned x_flag(int t, int g) { return OFF_FLAG + (g >> 2) * 256 + t * 4; }
__device__ __forceinline__ unsigned w_flag(int t, int e) { return OFF_FLAG + FLAG_W + (e >> 2) * 256 + t * 4; }

struct Spin {
    unsigned long long t0; unsigned n;
    __device__ __forceinline__ Spin() : t0(__builtin_amdgcn_s_memrealtime()), n(0) {}
    __device__ __forceinline__ bool fail(char* myws, unsigned call) {
        if ((++n & 15u) == 0 && (__builtin_amdgcn_readfirstlane(ld_agent(myws + OFF_TMO)) == call || __builtin_amdgcn_s_memrealtime() - t0 > DEADLINE)) {
            st_agent(myws + OFF_TMO, call);
            return true;
        }
        __builtin_amdgcn_s_sleep(1);
        return false;
    }
};

#define DPP0(x, ctrl, rm) (unsigned)__builtin_amdgcn_update_dpp(0, (int)(x), ctrl, rm, 0xf, false)
__device__ __forceinline__ unsigned wave_scan(unsigned x) {
    x += DPP0(x, 0x111, 0xf); x += DPP0(x, 0x112, 0xf); x += DPP0(x, 0x114, 0xf); x += DPP0(x, 0x118, 0xf);
    x += DPP0(x, 0x142, 0xa); x += DPP0(x, 0x143, 0xc);
    return x;
}
__device__ __forceinline__ unsigned wave_max(unsigned x) {
    x = max(x, DPP0(x, 0x111, 0xf)); x = max(x, DPP0(x, 0x112, 0xf)); x = max(x, DPP0(x, 0x114, 0xf)); x = max(x, DPP0(x, 0x118, 0xf));
    x = max(x, DPP0(x, 0x142, 0xa)); x = max(x, DPP0(x, 0x143, 0xc));
    return __builtin_amdgcn_readlane(x, 63);
}

struct Sched {
    unsigned w[7][64];
    constexpr Sched() : w{} {
        for (int q = 0; q < 7; ++q) {
            int key[136] = {}, id[136] = {}, n = 0;
            for (int j = 0; j < 8; ++j) {
                int lo = 1 << 30;
                for (int rt = 0; rt < 16; ++rt) { const int k = (SCH_S * rt < SCH_C ? SCH_S * rt : SCH_C) + SCH_B * rt + 106 * (7 * j + (q + SCH_A * rt) % 7); key[n] = 2 * k + 1; id[n++] = j * 24 + 8 + rt; lo = k < lo ? k : lo; }
                key[n] = 2 * lo; id[n++] = -1 - j;
            }
            int at = 0;
            for (int o = 0; o < 136; ++o) {
                int best = -1;
                for (int i = 0; i < 136; ++i) if (key[i] >= 0 && (best < 0 || key[i] < key[best])) best = i;
                key[best] = -1;
                const int skip = id[best] == -1 ? PUSH_W * PUSH_U / 4 : 0;
                const int cnt = id[best] < 0 ? 8 - skip : 1, first = id[best] < 0 ? (-1 - id[best]) * 24 + skip : id[best];
                for (int c = 0; c < cnt; ++c, ++at) w[q][at >> 2] |= (unsigned)(first + c) << (8 * (at & 3));
            }
        }
    }
};
__constant__ const Sched SCHED_TAB;
__device__ __forceinline__ int step_of(int i, int rank, int rt) {
    if (i < 8) return rank * 8 + i;
    const int z = i - 8, j = z / 7, p = z - j * 7, q = (p + 7 * 16 * SCH_A - SCH_A * rt) % 7;
    return (((rank + 1 + q) & 7) << 3) + j;
}

__device__ __forceinline__ void x_load(u32x4 (&v)[8], rsrc_t rx, int j, int g, int lane) {
    const unsigned voff = (unsigned)(lane & 15) * (KS * 2) + (unsigned)(lane >> 4) * 16;
    const unsigned soff = (unsigned)g * 64 * (KS * 2) + (unsigned)j * 128;
#pragma unroll
    for (int r = 0; r < 4; ++r)
#pragma unroll
        for (int kk = 0; kk < 2; ++kk) v[r * 2 + kk] = __builtin_amdgcn_raw_buffer_load_b128(rx, voff + kk * 64, soff + r * 16 * (KS * 2), 0);
}
template <int AUX = AUX_NT> __device__ __forceinline__ void w_load(u32x2 (&v)[16], rsrc_t rw, int j, int d, int ct, int kk, int G, int lane) {
    const unsigned voff = (unsigned)(lane >> 4) * 8 * (N_ * 2) + (unsigned)(lane & 15) * 8;
    const unsigned soff = (unsigned)(j * 64 + kk * 32) * (N_ * 2) + (unsigned)(d * NS + ct * 256 + G * 128) * 2;
#pragma unroll
    for (int jj = 0; jj < 8; ++jj)
#pragma unroll
        for (int g = 0; g < 2; ++g) v[g * 8 + jj] = __builtin_amdgcn_raw_buffer_load_b64(rw, voff + g * 128, soff + jj * (N_ * 2), AUX);
}
__device__ __forceinline__ void w_transpose(u32x4 (&o)[8], const u32x2 (&v)[16]) {
#pragma unroll
    for (int g = 0; g < 2; ++g)
#pragma unroll
        for (int c = 0; c < 4; ++c)
#pragma unroll
            for (int dd = 0; dd < 4; ++dd)
                o[g * 4 + c][dd] = __builtin_amdgcn_perm(v[g * 8 + 2 * dd + 1][c >> 1], v[g * 8 + 2 * dd][c >> 1], (c & 1) ? 0x07060302u : 0x05040100u);
}
__device__ __forceinline__ unsigned x_img(int t, int g) { return OFF_XT + (unsigned)t * XT_STEP + (unsigned)g * 8192; }
__device__ __forceinline__ unsigned w_img(int t, int e) { return OFF_WT + (unsigned)t * WT_STEP + (unsigned)(e >> 2) * BLK + (unsigned)((e >> 1) & 1) * 8192 + (unsigned)(e & 1) * 1024; }
__device__ __forceinline__ unsigned frag_off(bool isw, unsigned f) { return isw ? ((f >> 1) & 1) * HALFB + (f >> 2) * 4096 + (f & 1) * 2048 : f * 1024; }
__device__ __forceinline__ void img_store(const u32x4 (&v)[8], rsrc_t rd, bool isw, unsigned img, int lane) {
#pragma unroll
    for (int f = 0; f < 8; ++f) ST128_RAW(v[f], rd, lane * 16, img + frag_off(isw, f), AUX_SYS);
    ST128_PAD8(v);
}
constexpr unsigned H8 = 0x80808080u, MAXESC = 512;
__device__ __forceinline__ unsigned KS_(unsigned c) { asm("" : "+s"(c)); return c; }
__device__ __forceinline__ unsigned bits16(unsigned m0, unsigned m1, unsigned m2, unsigned m3, int sh) {
    const unsigned a = __builtin_amdgcn_udot4(m0, 0x08040201u, __builtin_amdgcn_udot4(m1, 0x80402010u, 0u, false), false);
    const unsigned b = __builtin_amdgcn_udot4(m2, 0x08040201u, __builtin_amdgcn_udot4(m3, 0x80402010u, 0u, false), false);
    return (a >> sh) | (b << (8 - sh));
}
__device__ __forceinline__ u32x2 lut_expand(unsigned byte) { u32x2 s = {0u, 0u}; unsigned c = 0;
#pragma unroll
    for (int i = 0; i < 8; ++i) { const unsigned b = (byte >> i) & 1u; s[i >> 2] |= (b ? c : 0x0cu) << (8 * (i & 3)); c += b; } return s; }
__device__ __forceinline__ unsigned lut_merge(unsigned nib) { unsigned s = 0, c = 0;
#pragma unroll
    for (int i = 0; i < 4; ++i) { const unsigned b = (nib >> i) & 1u; s |= (b ? c : 4u + i) << (8 * i); c += b; } return s; }
template <int N> __device__ __forceinline__ void merge(unsigned* x, const unsigned* n8, unsigned& o, const LAS unsigned char* mrg) {
    unsigned se[N], w0[N], w1[N], of[N];
#pragma unroll
    for (int k = 0; k < N; ++k) { se[k] = *(const LAS unsigned*)(mrg + n8[k]); const LAS unsigned* w = (const LAS unsigned*)(size_t)(o & ~3u); w0[k] = w[0]; w1[k] = w[1]; of[k] = o; o += __builtin_popcount(n8[k]); }
    SCHED;
#pragma unroll
    for (int k = 0; k < N; ++k) x[k] = __builtin_amdgcn_perm(x[k], __builtin_amdgcn_alignbyte(w1[k], w0[k], of[k]), se[k]);
}
__device__ __forceinline__ unsigned lut_compact(unsigned nib) { unsigned s = 0x0c0c0c0cu, c = 0;
#pragma unroll
    for (int i = 0; i < 4; ++i) if ((nib >> i) & 1u) { s = (s & ~(0xffu << (8 * c))) | ((unsigned)i << (8 * c)); ++c; } return s; }
__device__ __forceinline__ void squeeze(unsigned x, unsigned sel, unsigned o) {
    const unsigned long long z = (unsigned long long)__builtin_amdgcn_perm(0u, x, sel) << ((o & 3u) * 8);
    LAS unsigned* d = (LAS unsigned*)(size_t)(o & ~3u);
    __hip_atomic_fetch_or(d, (unsigned)z, __ATOMIC_RELAXED, __HIP_MEMORY_SCOPE_WAVEFRONT);
    __hip_atomic_fetch_or(d + 1, (unsigned)(z >> 32), __ATOMIC_RELAXED, __HIP_MEMORY_SCOPE_WAVEFRONT);
}
constexpr unsigned PK_HDR = 5 * 1024, PK_RAW = 0x80000000u, PK_FIRST = 384;
__device__ __forceinline__ unsigned clamp7(unsigned z) {
    const unsigned o = (((z & KS_(0x78787878u)) + 0x78787878u) | z) & H8;
    return (z | ((o >> 4) - (o >> 7))) & 0x07070707u;
}
__device__ __forceinline__ void pack_store(const u32x4 (&v)[8], rsrc_t rs, LAS unsigned char* L, unsigned slot, int lane) {
#pragma unroll
    for (int r = 0; r < 3; ++r) *(LAS u32x4*)(L + r * 1024 + lane * 16) = (u32x4){0u, 0u, 0u, 0u};
    if (lane < 16) *(LAS unsigned*)(L + 4096 + lane * 4) = lut_compact(lane);
    unsigned p0[16], z[16], n7[16], sum = 0, cnt = 0;
#pragma unroll
    for (int r = 0; r < 16; ++r) {
        const unsigned lo = __builtin_amdgcn_perm(v[r >> 1][2 * (r & 1) + 1], v[r >> 1][2 * (r & 1)], 0x06040200u);
        const unsigned hi = __builtin_amdgcn_perm(v[r >> 1][2 * (r & 1) + 1], v[r >> 1][2 * (r & 1)], 0x07050301u);
        p0[r] = (hi & H8) | (lo & ~H8);
        z[r] = ((hi << 1) & 0xfefefefeu) | ((lo >> 7) & 0x01010101u);
        sum = __builtin_amdgcn_sad_u8(z[r], 0u, sum);
    }
    const unsigned ec = min(max((__builtin_amdgcn_readlane(wave_scan(sum), 63) + 7782u) >> 12, 19u), 253u), C = (ec + 1) * 0x01010101u;
    u32x4 p2 = {0u, 0u, 0u, 0u};
#pragma unroll
    for (int r = 0; r < 16; ++r) {
        z[r] = ((C | H8) - (z[r] & KS_(~H8))) ^ ((C ^ ~z[r]) & H8);
        const unsigned c = __builtin_amdgcn_perm(0x83838383u, 0x02010083u, clamp7(z[r]));
        p2[r >> 2] |= (c & KS_(0x03030303u)) << (2 * (r & 3));
        n7[r] = __builtin_amdgcn_udot4(c & KS_(H8), 0x08040201u, 0u, false);
        cnt += __builtin_popcount(n7[r]);
    }
    const unsigned sc = wave_scan(cnt), n2 = __builtin_amdgcn_readlane(sc, 63), G2 = (n2 + 15) >> 4;
    bool bad = n2 > 2048;
    unsigned bytes = 0;
    if (!bad) {
        unsigned se[16], o = sc - cnt + (unsigned)(size_t)L;
#pragma unroll
        for (int r = 0; r < 16; ++r) se[r] = *(const LAS unsigned*)(L + 4096 + (n7[r] >> 5));
#pragma unroll
        for (int r = 0; r < 16; ++r) { squeeze(z[r], se[r], o); o += __builtin_popcount(n7[r]); }
        CFENCE;
        const u32x4 d0 = *(const LAS u32x4*)(L + lane * 16), d1 = *(const LAS u32x4*)(L + 1024 + lane * 16);
        unsigned pk[2] = {0u, 0u}, m7[8], cnt3 = 0;
#pragma unroll
        for (int k = 0; k < 8; ++k) {
            const unsigned c = __builtin_amdgcn_perm(0x83830201u, 0x83838300u, clamp7(k < 4 ? d0[k & 3] : d1[k & 3]));
            pk[k >> 2] |= (c & KS_(0x03030303u)) << (2 * (k & 3));
            m7[k] = __builtin_amdgcn_udot4(c & KS_(H8), 0x08040201u, 0u, false);
            se[k] = *(const LAS unsigned*)(L + 4096 + (m7[k] >> 5));
            cnt3 += __builtin_popcount(m7[k]);
        }
        if (lane < G2) *(LAS unsigned*)(L + 3076 + lane * 4) = pk[0];
        if (64 + lane < G2) *(LAS unsigned*)(L + 3076 + 256 + lane * 4) = pk[1];
        const unsigned sc3 = wave_scan(cnt3), n3 = __builtin_amdgcn_readlane(sc3, 63);
        bad = n3 > 512;
        bytes = 4 + G2 * 4 + ((n3 + 7) >> 3) * 4;
        if (!bad) {
            o = sc3 - cnt3 + (unsigned)(size_t)(L + 2304);
#pragma unroll
            for (int k = 0; k < 8; ++k) { squeeze((k < 4 ? d0[k & 3] : d1[k & 3]) ^ 0x06060606u, se[k], o); o += __builtin_popcount(m7[k]); }
            CFENCE;
            const u32x2 x = *(const LAS u32x2*)(L + 2304 + lane * 8);
            unsigned nb[2], b = 0;
#pragma unroll
            for (int k = 0; k < 2; ++k) {
                const unsigned y = x[k] ^ 0x06060606u, ff = ((y & ~H8) + 0x01010101u) & y & H8;
                b |= (((y & ~H8) + 0x6b6b6b6bu) | y) & H8 & ~ff;
                nb[k] = ((y + 0x0a0a0a0au - ((ff >> 7) * 10u)) | ((ff >> 3) - (ff >> 7))) & 0x0f0f0f0fu;
            }
            *(LAS unsigned*)(L + 3076 + G2 * 4 + lane * 4) = nb[0] | (nb[1] << 4);
            bad = __builtin_amdgcn_ballot_w64(b != 0) != 0;
        }
    }
    if (lane == 0) *(LAS unsigned*)(L + 3072) = bad ? PK_RAW : ec | (bytes << 8);
    CFENCE;
    ST128(*(const LAS u32x4*)(L + 3072 + lane * 16), rs, lane * 16 + 1024, slot + 4096, AUX_SYS);
    if (!bad) {
#pragma unroll
        for (int r = 0; r < 4; ++r) ST128(((u32x4){p0[4 * r], p0[4 * r + 1], p0[4 * r + 2], p0[4 * r + 3]}), rs, lane * 16 + r * 1024, slot, AUX_SYS);
        ST128(p2, rs, lane * 16, slot + 4096, AUX_SYS);
    } else {
#pragma unroll
        for (int f = 0; f < 8; ++f) ST128(v[f], rs, lane * 16, slot + (f < 5 ? f : f + 1) * 1024, AUX_SYS);
    }
}
struct Puller {
    rsrc_t rp, rs; char* myws; LAS unsigned char* lut; int rank, src, q, lane; unsigned call, seen, tab;
    __device__ __forceinline__ int unit_of(int tk) const { return (int)((__builtin_amdgcn_readlane(tab, tk >> 4) >> (2 * (tk & 12))) & 0xffu) * 4 + (tk & 3); }
    __device__ __forceinline__ int xg(int i) const { return i - 32; }
    __device__ __forceinline__ unsigned unit_id(int u) const { const int j = u / 96, i = u % 96; return i < 32 ? 512 + (rank * 8 + j) * 32 + i : j * 64 + xg(i); }
    __device__ __forceinline__ unsigned flag_of(int u) const { const int j = u / 96, i = u % 96; return i < 32 ? w_flag(src * 8 + j, i) : x_flag(src * 8 + j, xg(i)); }
    __device__ __forceinline__ void ready(int u) {
        const int j = u / 96, i = u % 96, need = j * 2 + (i >= 32);
        if (!((seen >> need) & 1u)) {
            Spin t0;
            for (;;) {
                CFENCE;
                const unsigned sum = OFF_PFLG + (src * 8 + j) * 128 + (i >= 32 ? 0 : 4);
                const unsigned f = __builtin_amdgcn_raw_buffer_load_b32(rs, (lane & 1) && j < NPUF ? OFF_PUF + ((src * NPUF + j) * 96 + i) * 4 : sum, 0, AUX_SYS);
                if (__builtin_amdgcn_readlane(f, 0) >= call) { seen |= 1u << need; break; }
                if (__builtin_amdgcn_readlane(f, 1) >= call) break;
                if (t0.fail(myws, call)) break;
                __builtin_amdgcn_s_sleep(4);
            }
            CFENCE;
        }
    }
    __device__ __forceinline__ void issue(LAS unsigned char* b, int u) {
        const unsigned id = unit_id(u), slot = OFF_PX + id * UNIT;
        __builtin_amdgcn_raw_ptr_buffer_load_lds(rp, (LAS void*)(b + 4096), 16, lane < PK_FIRST / 16 ? lane * 16 : 0xf0000000u, slot + 4096, 1024, AUX_SYS);
        __builtin_amdgcn_raw_ptr_buffer_load_lds(rp, (LAS void*)(b + 4096), 16, lane * 16, slot + 4096, 0, AUX_SYS);
        __builtin_amdgcn_raw_ptr_buffer_load_lds(rp, (LAS void*)b, 16, lane * 16, slot, 0, AUX_SYS);
        __builtin_amdgcn_raw_ptr_buffer_load_lds(rp, (LAS void*)b, 16, lane * 16, slot, 1024, AUX_SYS);
        __builtin_amdgcn_raw_ptr_buffer_load_lds(rp, (LAS void*)b, 16, lane * 16, slot, 2048, AUX_SYS);
        __builtin_amdgcn_raw_ptr_buffer_load_lds(rp, (LAS void*)b, 16, lane * 16, slot, 3072, AUX_SYS);
    }
    __device__ __forceinline__ void finish(LAS unsigned char* b, int u) {
        const int j = u / 96, i = u % 96, t = src * 8 + j;
        const bool isw = i < 32;
        const unsigned img = isw ? w_img(t, i) : x_img(t, xg(i));
        const unsigned h = *(const LAS unsigned*)(b + PK_HDR);
        u32x4 v[8];
#pragma unroll
        for (int r = 0; r < 5; ++r) v[r] = *(const LAS u32x4*)(b + r * 1024 + lane * 16);
        const unsigned meta = __builtin_amdgcn_readfirstlane(h), ec = meta & 0xffu;
        if (!(meta & PK_RAW)) {
            if ((meta >> 8) > PK_FIRST) {
                __builtin_amdgcn_raw_ptr_buffer_load_lds(rp, (LAS void*)(b + 4096), 16, lane * 16, OFF_PX + unit_id(u) * UNIT + 4096, 1024, AUX_SYS);
                WAIT_V(0);
            }
            const unsigned T1 = ec | ((ec - 1) << 8) | ((ec - 2) << 16), T2 = (ec + 1) | ((ec - 3) << 8) | ((ec - 4) << 16), k01 = KS_(0x01010101u), k03 = KS_(0x03030303u);
            const u32x4 p2 = v[4];
            unsigned pk0 = *(const LAS unsigned*)(b + PK_HDR + 4 + lane * 4), pk1 = *(const LAS unsigned*)(b + PK_HDR + 260 + lane * 4);
            u32x4 p0[4];
            unsigned e[16], n8[16], cnt = 0;
#pragma unroll
            for (int w = 0; w < 4; ++w) {
                const unsigned ot = p2[w] & (p2[w] >> 1) & KS_(0x55555555u);
                cnt += __builtin_popcount(ot);
#pragma unroll
                for (int k = 0; k < 4; ++k) {
                    e[4 * w + k] = __builtin_amdgcn_perm(0u, T1, (p2[w] >> (2 * k)) & k03);
                    n8[4 * w + k] = __builtin_amdgcn_udot4((ot >> (2 * k)) & k01, 0x40201008u, 0u, false);
                }
            }
            const unsigned sc = wave_scan(cnt), G2 = min((__builtin_amdgcn_readlane(sc, 63) + 15) >> 4, 128u);
            const unsigned x3 = *(const LAS unsigned*)(b + PK_HDR + 4 + G2 * 4 + lane * 4), B5 = (ec - 5) * 0x01010101u;
#pragma unroll
            for (int r = 0; r < 4; ++r) p0[r] = v[r];
            {
                const unsigned d0 = x3 & 0x0f0f0f0fu, d1 = (x3 >> 4) & 0x0f0f0f0fu, t0 = (d0 + k01) & 0x10101010u, t1 = (d1 + k01) & 0x10101010u;
                *(LAS u32x2*)(b + lane * 8) = (u32x2){B5 + t0 + (t0 >> 2) + (t0 >> 3) - d0, B5 + t1 + (t1 >> 2) + (t1 >> 3) - d1};
            }
            pk0 = lane < G2 ? pk0 : 0u; pk1 = 64 + lane < G2 ? pk1 : 0u;
            const unsigned ot0 = pk0 & (pk0 >> 1) & KS_(0x55555555u), ot1 = pk1 & (pk1 >> 1) & KS_(0x55555555u), cnt3 = __builtin_popcount(ot0) + __builtin_popcount(ot1);
            unsigned o = wave_scan(cnt3) - cnt3 + (unsigned)(size_t)b;
            CFENCE;
#define IN_L2(pk, ot, at) { unsigned v2[4], m8[4]; \
                _Pragma("unroll") for (int k = 0; k < 4; ++k) { v2[k] = __builtin_amdgcn_perm(0u, T2, (pk >> (2 * k)) & k03); m8[k] = __builtin_amdgcn_udot4((ot >> (2 * k)) & k01, 0x40201008u, 0u, false); } \
                merge<4>(v2, m8, o, lut); \
                *(LAS u32x4*)(b + at + lane * 16) = (u32x4){v2[0], v2[1], v2[2], v2[3]}; }
            IN_L2(pk0, ot0, 6144);
            if (G2 > 64) IN_L2(pk1, ot1, 7168);
            CFENCE;
            o = sc - cnt + (unsigned)(size_t)(b + 6144);
            merge<16>(e, n8, o, lut);
#pragma unroll
            for (int r = 0; r < 16; ++r) {
                const unsigned pp = p0[r >> 2][r & 3], ee = e[r];
                const unsigned lo = (pp & ~H8) | ((ee << 7) & H8), hi = (pp & H8) | ((ee >> 1) & ~H8);
                v[r >> 1][2 * (r & 1)] = __builtin_amdgcn_perm(hi, lo, 0x05010400u); v[r >> 1][2 * (r & 1) + 1] = __builtin_amdgcn_perm(hi, lo, 0x07030602u);
            }
        } else {
            const unsigned slot = OFF_PX + unit_id(u) * UNIT;
            WAIT_L(0);
            __builtin_amdgcn_raw_ptr_buffer_load_lds(rp, (LAS void*)b, 16, lane * 16, slot + 6144, 0, AUX_SYS);
            __builtin_amdgcn_raw_ptr_buffer_load_lds(rp, (LAS void*)b, 16, lane * 16, slot + 6144, 1024, AUX_SYS);
            __builtin_amdgcn_raw_ptr_buffer_load_lds(rp, (LAS void*)b, 16, lane * 16, slot + 6144, 2048, AUX_SYS);
            WAIT_V(0);
#pragma unroll
            for (int r = 5; r < 8; ++r) v[r] = *(const LAS u32x4*)(b + (r - 5) * 1024 + lane * 16);
        }
        img_store(v, rs, isw, img, lane);
    }
};
static_assert(OFF_PW == OFF_PX + 512 * UNIT, "unit ids run through the x and the w slots");

#define DRAW(r, tick) { unsigned long long ex_; asm volatile("s_mov_b64 %1, exec\n\ts_mov_b64 exec, 1\n\tglobal_atomic_add %0, %2, %3, %4 sc0\n\ts_mov_b64 exec, %1" \
        : "=&v"(r), "=&s"(ex_) : "v"(0u), "v"(1u), "s"(tick) : "memory"); }
#define TAKE(t, r, N) { unsigned t_; asm volatile("s_waitcnt vmcnt(" #N ")\n\tv_readfirstlane_b32 %0, %1" : "=s"(t_) : "v"(r) : "memory"); t = (int)(t_ - tbase); }
constexpr int NPUSH = PUSH_W * PUSH_U, NU_IN = 768 - NPUSH, NU_ALL = NU_IN + 256;

__device__ __forceinline__ void pull_role(const Params& p, LAS unsigned char* lds, LAS unsigned char* L, int c, int lane, unsigned call, int wv) {
    Puller P; P.rank = p.rank; P.q = c % 7; P.src = (p.rank + 1 + P.q) & 7; P.lane = lane; P.seen = 0; P.call = call; P.lut = lds + EX_MRG;
    P.tab = SCHED_TAB.w[P.q][lane];
    asm volatile("" : "+v"(P.tab));
    P.myws = p.my; P.rs = mk_rsrc(P.myws, WS_USED); P.rp = mk_rsrc(p.peer(1 + P.q), WS_USED);
    char* tick = P.myws + OFF_TICK + P.q * 128;
    const unsigned tbase = (call - 1) * (NU_ALL + TEAM * PPM);
    LAS unsigned* token = (LAS unsigned*)(lds + EX_TOK);
#define TOKEN_GET { for (;;) { unsigned o_ = 1; if (lane == 0) o_ = __hip_atomic_exchange(token, 1u, __ATOMIC_RELAXED, __HIP_MEMORY_SCOPE_WORKGROUP); \
            if (__builtin_amdgcn_readfirstlane(o_) == 0) break; __builtin_amdgcn_s_sleep(1); } }
#define TOKEN_PUT { if (lane == 0) __hip_atomic_store(token, 0u, __ATOMIC_RELAXED, __HIP_MEMORY_SCOPE_WORKGROUP); }
    if (c >= 7 * PPP) {
        Spin t0;
        while ((int)(__builtin_amdgcn_readfirstlane(ld_agent(tick)) - tbase) < NU_IN) { if (t0.fail(P.myws, call)) break; __builtin_amdgcn_s_sleep(32); }
    }
    unsigned r;
    int u;
    DRAW(r, tick); TAKE(u, r, 0);
    while ((unsigned)u < (unsigned)NU_IN) {
        const int un = P.unit_of(u);
        P.ready(un);
        TOKEN_GET;
        P.issue(L, un);
        WAIT_V(0);
        TOKEN_PUT;
        DRAW(r, tick);
        P.finish(L, un);
        TAKE(u, r, 0);
        if (lane == 0) add_sys(P.myws + P.flag_of(un));
    }
    STAMP(P.myws, wv, 9);
    unsigned long long seen = 0; int sblk = -1;
    while ((unsigned)u < (unsigned)NU_ALL) {
        const int mu = u - NU_IN;
        if (!((seen >> (mu & 63)) & 1) || (mu >> 6) != sblk) {
            Spin t0; sblk = mu >> 6;
            for (;;) {
                CFENCE;
                seen = __builtin_amdgcn_ballot_w64(__builtin_amdgcn_raw_buffer_load_b32(P.rs, lane * 4, OFF_MFLG + (P.src * 256 + sblk * 64) * 4, AUX_SYS) >= call);
                if (((seen >> (mu & 63)) & 1) || t0.fail(P.myws, call)) break;
                __builtin_amdgcn_s_sleep(8);
            }
            CFENCE;
        }
        TOKEN_GET;
        __builtin_amdgcn_raw_ptr_buffer_load_lds(P.rp, (LAS void*)L, 16, lane * 16, OFF_MSK + mu * 4096, 0, AUX_SYS);
        __builtin_amdgcn_raw_ptr_buffer_load_lds(P.rp, (LAS void*)L, 16, lane * 16, OFF_MSK + mu * 4096, 1024, AUX_SYS);
        __builtin_amdgcn_raw_ptr_buffer_load_lds(P.rp, (LAS void*)L, 16, lane * 16, OFF_MSK + mu * 4096, 2048, AUX_SYS);
        __builtin_amdgcn_raw_ptr_buffer_load_lds(P.rp, (LAS void*)L, 16, lane * 16, OFF_MSK + mu * 4096, 3072, AUX_SYS);
        WAIT_V(0);
        TOKEN_PUT;
        DRAW(r, tick);
        u32x4 mv[4];
#pragma unroll
        for (int k = 0; k < 4; ++k) mv[k] = *(const LAS u32x4*)(L + k * 1024 + lane * 16);
#pragma unroll
        for (int k = 0; k < 4; ++k) ST128_RAW(mv[k], P.rs, lane * 16 + k * 1024, OFF_LM + P.src * 1048576 + mu * 4096, AUX_SYS);
        ST128_PAD4(mv);
        TAKE(u, r, 0);
        if (lane == 0) st_sys(P.myws + OFF_MF + (P.src * 256 + mu) * 4, call);
    }
    STAMP(P.myws, wv, 17);
}

constexpr unsigned L1_MAX = 4608, L2_MAX = 2048;
__device__ __forceinline__ unsigned off_l2(unsigned G) { return G * 8 + ((G * 2 + 3) & ~3u); }
__device__ __forceinline__ unsigned popc4(const u32x4& m) { return __builtin_popcount(m[0]) + __builtin_popcount(m[1]) + __builtin_popcount(m[2]) + __builtin_popcount(m[3]); }
__device__ __forceinline__ unsigned nib4(unsigned m, int i) { return (i == 0 ? m << 2 : m >> (4 * i - 2)) & 0x3cu; }
__device__ __forceinline__ unsigned out_voff(int lane) { return (unsigned)(lane & 15) * N_ + (unsigned)(lane >> 4) * 16; }
__device__ __forceinline__ unsigned out_soff(int src, int wt) {
    const int tile = wt >> 3, w = wt & 7;
    return (unsigned)((tile >> 3) * 256 + (w >> 2) * 64) * N_ + (unsigned)(src * NS + (tile & 7) * 256 + (w & 3) * 64);
}
__device__ __forceinline__ constexpr unsigned chunk_soff(int ch) { return (unsigned)((ch >> 2) * 128 + (ch & 3) * 16) * N_; }

__device__ __forceinline__ void expand_role(const Params& p, LAS unsigned char* lds, LAS unsigned char* L, int ei, int lane, unsigned call, int wv) {
    const int rank = p.rank, q = ei % 7, src = (rank + 1 + q) & 7;
    char* myws = p.my;
    char* pws = p.peer(1 + q);
    const rsrc_t rs = mk_rsrc(myws, WS_USED), ro = mk_rsrc(p.out, OUT_BYTES);
    char* tick = myws + OFF_TICK + (8 + q) * 128;
    const unsigned tbase = (call - 1) * (1024 + 2 * PPO);
    bool dead = false;
#define WT_OF(k) ((unsigned)(k) < 1024u ? ((k) & ~63) | (((k) & 28) << 1) | (((k) >> 3) & 4) | ((k) & 3) : (k))
#define ENT_OFF(t) (lane == 0 ? OFF_DONE + (src * 1024 + ((t) & ~1)) * 8 : OFF_MF + (src * 256 + (((t) >> 2) & ~3)) * 4)
#define ENT_FETCH(t) { if (lane < 2) __builtin_amdgcn_raw_ptr_buffer_load_lds(rs, (LAS void*)(L + EX_E), 16, ENT_OFF(t), 0, 0, AUX_SYS); }
#define EX_ISSUE(t, e, x, f, n, xo) { unsigned e_ = (e), x_ = (x), f_ = (f); \
        if ((e_ < call || f_ < call) && !dead) { Spin t0_; \
              \
            for (;;) { CFENCE; const bool had_ = e_ >= call; \
                const u32x4 g_ = __builtin_amdgcn_raw_buffer_load_b128(rs, lane < (had_ ? 2 : 1) ? ENT_OFF(t) : 0xf0000000u, 0, AUX_SYS); \
                e_ = __builtin_amdgcn_readlane(((t) & 1) ? g_[2] : g_[0], 0); x_ = __builtin_amdgcn_readlane(((t) & 1) ? g_[3] : g_[1], 0); \
                f_ = __builtin_amdgcn_readlane(((t) & 8) ? (((t) & 4) ? g_[3] : g_[2]) : (((t) & 4) ? g_[1] : g_[0]), 1); \
                if (!had_ && e_ >= call) continue; \
                if (e_ >= call && f_ >= call) break; if (t0_.fail(myws, call)) { dead = true; break; } __builtin_amdgcn_s_sleep(8); } \
            CFENCE; \
            { const unsigned long long s0_ = __builtin_amdgcn_s_memrealtime(); while ((unsigned)(__builtin_amdgcn_s_memrealtime() - s0_) < (ei / 7 - PPM) * STAG && ei / 7 > PPM) __builtin_amdgcn_s_sleep(1); } } \
        n = min(x_ & 0x3fffu, 8192u); xo = x_; \
        const unsigned by_ = min(((x_ >> 30) ? n : ((x_ >> 14) & 0xfffu) * 4) + 15 & ~15u, 8192u); \
        const rsrc_t rc_ = mk_rsrc(pws + OFF_COD + (unsigned)(t) * 8192, by_); \
        _Pragma("nounroll") for (unsigned r_ = 0; r_ < by_; r_ += 1024) \
            __builtin_amdgcn_raw_ptr_buffer_load_lds(rc_, (LAS void*)(L + EX_B + r_), 16, lane * 16, r_, 0, AUX_SYS); \
        __builtin_amdgcn_raw_ptr_buffer_load_lds(rs, (LAS void*)(L + EX_M), 16, lane * 16, OFF_LM + src * 1048576 + (t) * 1024, 0, AUX_SYS); }
    unsigned r0, n0 = 0, x0 = 0;
    DRAW(r0, tick);
    int t0;
    TAKE(t0, r0, 0); t0 = WT_OF(t0);
    if ((unsigned)t0 < 1024u) EX_ISSUE(t0, 0u, 0u, 0u, n0, x0);
    int t1;
    { unsigned r1; DRAW(r1, tick); TAKE(t1, r1, 0); t1 = WT_OF(t1); }
    STAMP(myws, wv, 18);
    while ((unsigned)t0 < 1024u) {
        if ((unsigned)t1 < 1024u) ENT_FETCH(t1);
        const u32x4 m = *(const LAS u32x4*)(L + EX_M + lane * 16);
        if (x0 >> 30) {
#pragma nounroll
            for (unsigned r = 0; r < n0; r += 1024) *(LAS u32x4*)(L + r + lane * 16) = *(const LAS u32x4*)(L + EX_B + r + lane * 16);
        } else {
            const unsigned G = min((n0 + 15) >> 4, L1_MAX / 16), ec = (x0 >> 26) & 15u;
            const unsigned T1 = (ec << 3) | (((ec - 1) & 15u) << 11) | (((ec - 2) & 15u) << 19) | 0x80000000u;
            const unsigned T2 = (((ec + 1) & 15u) << 3) | (((ec - 3) & 15u) << 11) | (((ec - 4) & 15u) << 19) | 0x80000000u;
            LAS unsigned char* S2 = L + EX_B + 5888;
            unsigned mn[20], ex[20], n8[20], cnt2 = 0;
            const unsigned k01 = KS_(0x01010101u), k02 = KS_(0x02020202u), k03 = KS_(0x03030303u), k07 = KS_(0x07070707u), k80 = KS_(H8), kmul = KS_(0x00408102u);
#define LAYER1(i0, n) { u32x2 nn_[n]; unsigned hb_[n]; \
                _Pragma("unroll") for (int i = 0; i < n; ++i) { const unsigned g = (i0 + i) * 64 + lane; nn_[i] = *(const LAS u32x2*)(L + EX_B + g * 8); hb_[i] = *(const LAS unsigned short*)(L + EX_B + G * 8 + g * 2); } \
                SCHED; \
                _Pragma("unroll") for (int i = 0; i < n; ++i) { const unsigned hb = (i0 + i) * 64 + lane < G ? hb_[i] : 0u;        \
                    _Pragma("unroll") for (int k = 0; k < 4; ++k) { const int z = 4 * (i0 + i) + k; mn[z] = (k & 1) ? nn_[i][k >> 1] >> 4 : nn_[i][k >> 1]; \
                        ex[z] = __builtin_amdgcn_perm(0u, T1, ((mn[z] >> 3) & k01) | (__builtin_amdgcn_ubfe(hb, 4 * k, 4) * kmul & k02)); \
                        n8[z] = __builtin_amdgcn_udot4(ex[z] & k80, 0x08040201u, 0u, false) >> 4; \
                        cnt2 += __builtin_popcount(n8[z]); } } }
            LAYER1(0, 4);
#pragma unroll
            for (int k = 16; k < 20; ++k) mn[k] = ex[k] = n8[k] = 0;
            if (G > 256) LAYER1(4, 1);
            CYC(0);
            const unsigned sc2 = wave_scan(cnt2), n2 = min(__builtin_amdgcn_readlane(sc2, 63), L2_MAX), G2 = (n2 + 15) >> 4, o2 = off_l2(G), o3 = o2 + G2 * 4;
            {
                unsigned pk0 = *(const LAS unsigned*)(L + EX_B + o2 + lane * 4), pk1 = *(const LAS unsigned*)(L + EX_B + o2 + 256 + lane * 4);
                pk0 = lane < G2 ? pk0 : 0u; pk1 = 64 + lane < G2 ? pk1 : 0u;
                const unsigned x3 = *(const LAS unsigned*)(L + EX_B + o3 + lane * 4);
                *(LAS u32x2*)(L + EX_P + lane * 8) = (u32x2){(x3 << 3) & 0x78787878u, (x3 >> 1) & 0x78787878u};
                const unsigned ot0 = pk0 & (pk0 >> 1) & KS_(0x55555555u), ot1 = pk1 & (pk1 >> 1) & KS_(0x55555555u), cnt3 = __builtin_popcount(ot0) + __builtin_popcount(ot1);
                unsigned v2[8], m8[8];
#pragma unroll
                for (int k = 0; k < 8; ++k) {
                    v2[k] = __builtin_amdgcn_perm(0u, T2, ((k < 4 ? pk0 : pk1) >> (2 * (k & 3))) & k03);
                    m8[k] = __builtin_amdgcn_udot4(((k < 4 ? ot0 : ot1) >> (2 * (k & 3))) & k01, 0x40201008u, 0u, false);
                }
                unsigned o = wave_scan(cnt3) - cnt3 + (unsigned)(size_t)(L + EX_P);
                CFENCE;
                merge<8>(v2, m8, o, lds + EX_MRG);
                *(LAS u32x4*)(S2 + lane * 16) = (u32x4){v2[0], v2[1], v2[2], v2[3]};
                *(LAS u32x4*)(S2 + 1024 + lane * 16) = (u32x4){v2[4], v2[5], v2[6], v2[7]};
                CFENCE;
            }
            unsigned o = sc2 - cnt2 + (unsigned)(size_t)S2;
#define CODES_OUT(i) *(LAS u32x4*)(L + ((i) * 64 + lane) * 16) = (u32x4){(mn[4 * (i)] & k07) | ex[4 * (i)], (mn[4 * (i) + 1] & k07) | ex[4 * (i) + 1], (mn[4 * (i) + 2] & k07) | ex[4 * (i) + 2], (mn[4 * (i) + 3] & k07) | ex[4 * (i) + 3]}
            merge<16>(ex, n8, o, lds + EX_MRG);
            CODES_OUT(0); CODES_OUT(1); CODES_OUT(2); CODES_OUT(3);
            if (G > 256) { merge<4>(ex + 16, n8 + 16, o, lds + EX_MRG); CODES_OUT(4); }
            CFENCE;
        }
        const unsigned cnt = popc4(m);
        unsigned off = wave_scan(cnt) - cnt + (unsigned)(size_t)L;
        WAIT_L(0);
        CYC(1);
        unsigned n1 = 0, x1 = 0;
        if ((unsigned)t1 < 1024u) {
            WAIT_V(0);
            const u32x2 ent = *(const LAS u32x2*)(L + EX_E + (t1 & 1) * 8);
            const unsigned mf = *(const LAS unsigned*)(L + EX_E + 16 + (t1 & 12));
            EX_ISSUE(t1, __builtin_amdgcn_readfirstlane(ent[0]), __builtin_amdgcn_readfirstlane(ent[1]), __builtin_amdgcn_readfirstlane(mf), n1, x1);
        }
        unsigned r2;
        DRAW(r2, tick);
        CYC(2);
        const unsigned so = out_soff(src, t0), vo = out_voff(lane);
        u32x2 sel[16]; unsigned of[16], w0[16], w1[16], w2[16];
#pragma unroll
        for (int g = 0; g < 16; ++g) {
            const unsigned b8 = ((g & 3) == 0 ? m[g >> 2] << 3 : m[g >> 2] >> (8 * (g & 3) - 3)) & 0x7f8u;
            sel[g] = *(const LAS u32x2*)(lds + EX_LUT + b8);
            const LAS unsigned* w = (const LAS unsigned*)(size_t)(off & ~3u);
            w0[g] = w[0]; w1[g] = w[1]; w2[g] = w[2]; of[g] = off;
            off += __builtin_popcount(b8);
        }
        SCHED;
        u32x4 o[8];
#pragma unroll
        for (int g = 0; g < 16; ++g) {
            const unsigned a0 = __builtin_amdgcn_alignbyte(w1[g], w0[g], of[g]), a1 = __builtin_amdgcn_alignbyte(w2[g], w1[g], of[g]);
            o[g >> 1][(g & 1) * 2] = __builtin_amdgcn_perm(a1, a0, sel[g][0]); o[g >> 1][(g & 1) * 2 + 1] = __builtin_amdgcn_perm(a1, a0, sel[g][1]);
        }
        CYC(3);
        if (dead) {
#pragma unroll
            for (int ch = 0; ch < 8; ++ch) o[ch] = (u32x4){0x7f7f7f7fu, 0x7f7f7f7fu, 0x7f7f7f7fu, 0x7f7f7f7fu};
            CFENCE;
        }
#pragma unroll
        for (int ch = 0; ch < 8; ++ch) ST128_RAW(o[ch], ro, vo, so + chunk_soff(ch), AUX_WT);
        ST128_PAD8(o);
        CYC(4);
        int t2;
        TAKE(t2, r2, 8); t2 = WT_OF(t2);
        CYC(5);
        t0 = t1; n0 = n1; x0 = x1; t1 = t2;
    }
    STAMP(myws, wv, 19);
}

__device__ __forceinline__ unsigned get_call(unsigned callv) { unsigned c; asm volatile("s_nop 0\n\tv_readfirstlane_b32 %0, %1" : "=s"(c) : "v"(callv)); return c + 1u; }

__device__ __forceinline__ void own_x(const Params& p, int j, int g, unsigned due, unsigned long long t0, int lane) {
    if (j < 2) return;
    while ((unsigned)(__builtin_amdgcn_s_memrealtime() - t0) < due) __builtin_amdgcn_s_sleep(4);
    u32x4 v[8];
    x_load(v, mk_rsrc(p.x, X_BYTES), j, g, lane);
    img_store(v, mk_rsrc(p.my, WS_USED), false, x_img(p.rank * 8 + j, g), lane);
    WAIT_V(0);
    if (lane == 0) add_sys(p.my + x_flag(p.rank * 8 + j, g));
}

__device__ __forceinline__ void comm_role(const Params& p, LAS unsigned char* lds, int c, int wid, int lane, unsigned callv) {
    const int rank = p.rank;
    char* myws = p.my;
    const rsrc_t rs = mk_rsrc(myws, WS_USED);
    const int wv = (NGEMM + c) * 8 + wid; (void)wv;
    STAMP(myws, wv, 0);
    if (wid == 0 && lane == 0) *(volatile LAS unsigned*)(lds + EX_TOK) = 0u;
    if (wid == 0 && lane < 16) *(LAS unsigned*)(lds + EX_MRG + lane * 8) = lut_merge(lane);
    if (wid == 2 || wid == 3) { *(LAS u32x2*)(lds + EX_LUT + ((wid & 1) * 128 + lane) * 8) = lut_expand((wid & 1) * 128 + lane); *(LAS u32x2*)(lds + EX_LUT + ((wid & 1) * 128 + 64 + lane) * 8) = lut_expand((wid & 1) * 128 + 64 + lane); }
    WAIT_L(0); BAR; CFENCE;
    if (wid >= 4) {
        const rsrc_t rx = mk_rsrc(p.x, X_BYTES), rw = mk_rsrc(p.w, W_BYTES);
        const bool far = wid < 6;
        const unsigned long long tk0 = __builtin_amdgcn_s_memrealtime();
        for (int u = (wid & 1) * 128 + c + (far ? 7 * NPUSH : 0); u < (far ? 1792 : 704); u += 256) {
            const bool ownw = !far && u >= 64 && u < 256;
            const int ux = u < 64 ? u : u - 192;
            const int j = far ? u / 224 : ownw ? u >> 5 : ux >> 6, r = far ? u % 224 : ownw ? 288 + (u & 31) : 224 + (ux & 63), t = rank * 8 + j;
            const unsigned due = far ? (j > 1 ? (j - 1) * PACE_W : j * PACE_W1) : ownw ? (j - 2) * PACE_E : j * PACE_X;
            while ((unsigned)(__builtin_amdgcn_s_memrealtime() - tk0) < due) __builtin_amdgcn_s_sleep(8);
            u32x4 v[8];
            if (r >= 224 && r < 288) {
                const int g = r - 224;
                x_load(v, rx, j, g, lane);
                pack_store(v, rs, lds + EX_0 + wid * EX_BYTES, OFF_PX + (unsigned)(j * 64 + g) * UNIT, lane);
                WAIT_V(0);
                const unsigned call = get_call(callv);
                if (j < NPUF) {
                    if (lane < 8) st_sys(p.pl + OFF_PUF + ((rank * NPUF + j) * 96 + 32 + g) * 4, call);
                    if (j == 0) STAMP(myws, wv, 7);
                }
                unsigned old = 0;
                if (lane == 0) old = __hip_atomic_fetch_add((GAS unsigned*)(myws + OFF_PCNT + j * 128), 1u, __ATOMIC_RELAXED, __HIP_MEMORY_SCOPE_AGENT);
                if ((__builtin_amdgcn_readfirstlane(old) & 63) == 63 && lane < 8) st_sys(p.pl + OFF_PFLG + t * 128, call);
            } else {
                const bool own = r >= 288;
                const int e = own ? r - 288 : r / 7, d = own ? rank : (rank + 1 + r % 7) & 7;
                u32x2 w[16];
                w_load(w, rw, j, d, e >> 2, e & 1, (e >> 1) & 1, lane);
                w_transpose(v, w);
                if (own) {
                    img_store(v, rs, true, w_img(t, e), lane);
                    WAIT_V(0);
                    if (lane == 0) add_sys(myws + w_flag(t, e));
                } else {
                    const unsigned id = (unsigned)((d * 8 + j) * 32 + e);
                    pack_store(v, rs, lds + EX_0 + wid * EX_BYTES, OFF_PW + id * UNIT, lane);
                    WAIT_V(0);
                    const unsigned call = get_call(callv);
                    if (j < NPUF) { if (lane == 1 + r % 7) st_sys(p.pl + OFF_PUF + ((rank * NPUF + j) * 96 + e) * 4, call); if (j == 0) STAMP(myws, wv, 6); }
                    unsigned old = 0;
                    if (lane == 0) old = __hip_atomic_fetch_add((GAS unsigned*)(myws + OFF_PCNT + (8 + d * 8 + j) * 128), 1u, __ATOMIC_RELAXED, __HIP_MEMORY_SCOPE_AGENT);
                    if (__builtin_amdgcn_readfirstlane(old) + 1 == call * (j ? 32 : 32 - NPUSH) && lane == 1 + r % 7) st_sys(p.pl + OFF_PFLG + t * 128 + 4, call);
                }
            }
            STAMP(myws, wv, far ? 1 + (j >> 2) : 3 + (j >> 2));
        }
    }
    if (wid == 1) {
        const unsigned long long tk0 = __builtin_amdgcn_s_memrealtime();
        if (NPUSH && c < 7 * PUSH_W) {
            const int d = (rank + 1 + c % 7) & 7, e0 = c / 7;
            const rsrc_t rw = mk_rsrc(p.w, W_BYTES), rd = mk_rsrc(p.peer(1 + c % 7), WS_USED);
            u32x2 w[PUSH_U ? PUSH_U : 1][16];
#pragma unroll
            for (int i = 0; i < PUSH_U; ++i) { const int e = e0 + i * PUSH_W; w_load(w[i], rw, 0, d, e >> 2, e & 1, (e >> 1) & 1, lane); }
#pragma unroll
            for (int i = 0; i < PUSH_U; ++i) {
                const int e = e0 + i * PUSH_W;
                u32x4 v[8];
                w_transpose(v, w[i]);
                img_store(v, rd, true, w_img(rank * 8, e), lane);
                WAIT_V(0);
                if (lane == 1 + c % 7) add_sys(p.pl + w_flag(rank * 8, e));
            }
            STAMP(myws, wv, 14);
        }
        for (int u = c; u < 256; u += 128) own_x(p, u >> 5, 32 + (u & 31), PACE_L0 + (u >> 5) * PACE_O, tk0, lane);
    }
    if (wid < CW) {
        const unsigned call = get_call(callv);
        const int ci = wid * 128 + c;
        if (wid < TEAM && c < 7 * PPM) pull_role(p, lds, lds + EX_0 + wid * EX_BYTES, c, lane, call, wv);
        if (ci < 7 * PPO) expand_role(p, lds, lds + EX_0 + wid * EX_BYTES, ci, lane, call, wv);
    }
}

typedef float f32x2 __attribute__((ext_vector_type(2)));
__device__ __forceinline__ f32x4 scale4(const f32x4& v, const f32x2& s) {
    f32x2 a, b;
    asm("v_pk_mul_f32 %0, %1, %2" : "=v"(a) : "v"((f32x2){v[0], v[1]}), "v"(s));
    asm("v_pk_mul_f32 %0, %1, %2" : "=v"(b) : "v"((f32x2){v[2], v[3]}), "v"(s));
    return (f32x4){a[0], a[1], b[0], b[1]};
}
__device__ __forceinline__ void gemm_role(const Params& p, LAS unsigned char* lds, int rt, int ct, int wid, int lane, unsigned callv) {
    const int rank = p.rank;
    char* myws = p.my;
    const int tile = rt * 8 + ct;
    STAMP(myws, tile * 8 + wid, 0);
    if (tile >= 64 && wid >= 1 && wid <= 4)
        own_x(p, (tile >> 2) & 7, (3 - (tile >> 5)) * 16 + (tile & 3) * 4 + wid - 1, tile >= 96 ? (((tile >> 2) & 7) - 2) * PACE_E : PACE_O0 + ((tile >> 2) & 7) * PACE_O,
              __builtin_amdgcn_s_memrealtime(), lane);
    const int wr = wid >> 2, wc = wid & 3;
    LAS unsigned char* dst = lds + 2 * HALFB + lane * 16 + (wr == 0 ? wid * 8192 : 4 * HALFB + (wc >> 1) * 8192 + (wc & 1) * 1024);
    u32x4 dv[8];
    if (rt >= 8) { const unsigned long long t0_ = __builtin_amdgcn_s_memrealtime(); while ((unsigned)(__builtin_amdgcn_s_memrealtime() - t0_) < 250) __builtin_amdgcn_s_sleep(4); }
    {
        u32x4 v0[8];
        if (wr == 0) {
            x_load(v0, mk_rsrc(p.x, X_BYTES), 0, rt * 4 + wid, lane); x_load(dv, mk_rsrc(p.x, X_BYTES), 1, rt * 4 + wid, lane);
        } else {
            u32x2 w[2][16];
            w_load<0>(w[0], mk_rsrc(p.w, W_BYTES), 0, rank, ct, wc & 1, wc >> 1, lane); w_load<0>(w[1], mk_rsrc(p.w, W_BYTES), 1, rank, ct, wc & 1, wc >> 1, lane);
            w_transpose(v0, w[0]);
#pragma unroll
            for (int i = 0; i < 8; ++i) dv[i] = (u32x4){w[1][2 * i][0], w[1][2 * i][1], w[1][2 * i + 1][0], w[1][2 * i + 1][1]};
        }
#pragma unroll
        for (int f = 0; f < 8; ++f) *(LAS u32x4*)(dst + frag_off(wr == 1, f)) = v0[f];
    }
    const unsigned call = get_call(callv);
    const int wv = tile * 8 + wid; (void)wv;
    const rsrc_t rs = mk_rsrc(myws, WS_USED);
    const unsigned voff = (unsigned)wid * 2048 + (unsigned)lane * 16;
    LAS unsigned char* ldsw = lds + wid * 2048;
    LAS unsigned char* lda = lds + wr * 8192 + lane * 16;
    LAS unsigned char* ldb = lds + wc * 4096 + lane * 16;
    const unsigned seqf = (unsigned)step_of(lane, rank, rt) * 4, fx = OFF_FLAG + rt * 256, fw = OFF_FLAG + FLAG_W + ct * 256, c4 = call * 4;
    unsigned long long todo = ~0ull, rdy = 0;
    unsigned tl = 0;
    volatile LAS unsigned* tq = (volatile LAS unsigned*)(lds + LDS_RED + 160);
#define SA(b, h) (((b) * 2 + (h)) * HALFB)
#define SB(b, h) ((4 + (b) * 2 + (h)) * HALFB)
#define STAGE(bufoff, soff) do { \
        __builtin_amdgcn_raw_ptr_buffer_load_lds(rs, (LAS void*)(ldsw + (bufoff)), 16, voff, (soff), 0, AUX_SYS); \
        __builtin_amdgcn_raw_ptr_buffer_load_lds(rs, (LAS void*)(ldsw + (bufoff)), 16, voff, (soff), 1024, AUX_SYS); } while (0)
#define LDA(dst, b, h) do { _Pragma("unroll") for (int m = 0; m < 4; ++m) _Pragma("unroll") for (int k = 0; k < 2; ++k) dst[m][k] = *(const LAS bf16x8*)(lda + SA(b, h) + m * 2048 + k * 1024); } while (0)
#define LDB(dst, b, h) do { _Pragma("unroll") for (int n = 0; n < 2; ++n) _Pragma("unroll") for (int k = 0; k < 2; ++k) dst[n][k] = *(const LAS bf16x8*)(ldb + SB(b, h) + n * 2048 + k * 1024); } while (0)
#define MMA(ai, bj, At, Bt) do { __builtin_amdgcn_s_setprio(1); _Pragma("unroll") for (int m = 0; m < 4; ++m) _Pragma("unroll") for (int n = 0; n < 2; ++n) _Pragma("unroll") for (int k = 0; k < 2; ++k) \
        acc[ai][bj][m][n] = __builtin_amdgcn_mfma_f32_16x16x32_bf16(Bt[n][k], At[m][k], acc[ai][bj][m][n], 0, 0, 0); __builtin_amdgcn_s_setprio(0); } while (0)
#define OFFA(t) (OFF_XT + (unsigned)(t) * XT_STEP + (unsigned)rt * BLK)
#define OFFB(t) (OFF_WT + (unsigned)(t) * WT_STEP + (unsigned)ct * BLK)
    unsigned la = 0, lb = 0;
#define LOOK do { if (wid == 0) { WAIT_V(8); la = *(const LAS unsigned*)(lds + LDS_FLAG + lane * 4); lb = *(const LAS unsigned*)(lds + LDS_FLAG + 256 + lane * 4); } } while (0)
#define PICK(slot) do { if (wid == 0) { \
            if (todo != 0) { \
                rdy |= __builtin_amdgcn_ballot_w64(la >= c4 && lb >= c4); \
                if ((rdy & todo) == 0) { Spin t0_; \
                    for (;;) { CFENCE; \
                        const unsigned a_ = __builtin_amdgcn_raw_buffer_load_b32(rs, seqf, fx, AUX_SYS), b_ = __builtin_amdgcn_raw_buffer_load_b32(rs, seqf, fw, AUX_SYS); \
                        rdy |= __builtin_amdgcn_ballot_w64(a_ >= c4 && b_ >= c4); \
                        if ((rdy & todo) != 0) break; if (t0_.fail(myws, call)) { rdy = ~0ull; break; } } } \
                const int pos_ = __builtin_ctzll(rdy & todo); \
                todo &= ~(1ull << pos_); tl = __builtin_amdgcn_readlane(seqf, pos_) >> 2; \
            } \
            tq[slot] = tl; \
            const unsigned skip_ = (todo & ~rdy) == 0 ? 0xf0000000u : 0u;        \
            __builtin_amdgcn_raw_ptr_buffer_load_lds(rs, (LAS void*)(lds + LDS_FLAG), 4, seqf, fx + skip_, 0, AUX_SYS); \
            __builtin_amdgcn_raw_ptr_buffer_load_lds(rs, (LAS void*)(lds + LDS_FLAG + 256), 4, seqf, fw + skip_, 0, AUX_SYS); \
            SCHED; } } while (0)

    f32x4 acc[2][2][4][2];
#pragma unroll
    for (int a = 0; a < 2; ++a)
#pragma unroll
        for (int b = 0; b < 2; ++b)
#pragma unroll
            for (int m = 0; m < 4; ++m)
#pragma unroll
                for (int n = 0; n < 2; ++n) acc[a][b][m][n] = (f32x4){0.f, 0.f, 0.f, 0.f};
    bf16x8 At[4][2], B0[2][2], B1[2][2];

#define WAIT2 do { if (wid == 0) WAIT_V(10); else WAIT_V(8); WAIT_L(0); } while (0)
#define STEP2(b) { \
        LOOK; \
        LDB(B0, b, 0); LDA(At, b, 0); LDB(B1, b, 1); \
        PICK(b); \
        STAGE(SA(b ^ 1, 1), OFFA(t1) + HALFB); \
        WAIT2; BAR; \
        const unsigned tv = tq[b]; \
        MMA(0, 0, At, B0); MMA(0, 1, At, B1); BAR; SCHED; \
        t1 = __builtin_amdgcn_readfirstlane(tv); \
        LDA(At, b, 1); \
        STAGE(SB(b, 0), OFFB(t1)); STAGE(SB(b, 1), OFFB(t1) + HALFB); STAGE(SA(b, 0), OFFA(t1)); \
        WAIT2; BAR; \
        MMA(1, 0, At, B0); MMA(1, 1, At, B1); BAR; SCHED; }
    unsigned t1;
    {
#define PLAIN_STEP { LDB(B0, 1, 0); LDA(At, 1, 0); LDB(B1, 1, 1); MMA(0, 0, At, B0); MMA(0, 1, At, B1); LDA(At, 1, 1); MMA(1, 0, At, B0); MMA(1, 1, At, B1); }
        todo = ~3ull;
        if (wid == 1 && lane < 8) ((LAS unsigned*)(lds + LDS_RED))[32 + lane] = 0u;
        WAIT_L(0); BAR; CFENCE;
        if (wid == 0) {
            __builtin_amdgcn_raw_ptr_buffer_load_lds(rs, (LAS void*)(lds + LDS_FLAG), 4, seqf, fx, 0, AUX_SYS);
            __builtin_amdgcn_raw_ptr_buffer_load_lds(rs, (LAS void*)(lds + LDS_FLAG + 256), 4, seqf, fw, 0, AUX_SYS);
        }
        PLAIN_STEP;
        SCHED;
        if (wr == 1) {
            u32x2 w[16];
#pragma unroll
            for (int i = 0; i < 8; ++i) { w[2 * i] = (u32x2){dv[i][0], dv[i][1]}; w[2 * i + 1] = (u32x2){dv[i][2], dv[i][3]}; }
            w_transpose(dv, w);
        }
        WAIT_V(0); LOOK;
        WAIT_L(0); BAR; CFENCE;
#pragma unroll
        for (int f = 0; f < 8; ++f) *(LAS u32x4*)(dst + frag_off(wr == 1, f)) = dv[f];
        PICK(2);
        WAIT_L(0); BAR; CFENCE;
        STAMP(myws, wv, 1);
        const unsigned t0 = __builtin_amdgcn_readfirstlane(tq[2]);
        STAGE(SB(0, 0), OFFB(t0)); STAGE(SB(0, 1), OFFB(t0) + HALFB); STAGE(SA(0, 0), OFFA(t0)); STAGE(SA(0, 1), OFFA(t0) + HALFB);
        SCHED;
        PLAIN_STEP;
        LOOK; PICK(3);
        WAIT_L(0); BAR; CFENCE;
        STAMP(myws, wv, 14);
        t1 = __builtin_amdgcn_readfirstlane(tq[3]);
        STAGE(SB(1, 0), OFFB(t1)); STAGE(SB(1, 1), OFFB(t1) + HALFB); STAGE(SA(1, 0), OFFA(t1));
        WAIT_V(6); BAR;
        if (wr == 1) BAR;
    }
    for (int i = 2; i < NT; i += 2) {
        STEP2(0);
        STEP2(1);
    }
    char* pl = p.pl;
    if (wr == 0) BAR;
    STAMP(myws, wv, 10);

    const int wt = tile * 8 + wid;
    LAS unsigned* red = (LAS unsigned*)(lds + LDS_RED);
    u32x4 msk;
#pragma unroll
    for (int w = 0; w < 4; ++w) {
        unsigned mm = 0;
#pragma unroll
        for (int e = 31; e >= 0; --e) {
            const int ch = 2 * w + (e >> 4);
            mm = __builtin_amdgcn_alignbit(mm, __float_as_uint(acc[ch >> 2][(e >> 1) & 1][ch & 3][e & 1][(e >> 2) & 3]), 31);
        }
        msk[w] = ~mm;
    }
    ST128(msk, rs, lane * 16, OFF_MSK + wt * 1024, AUX_SYS);
    SCHED;
    float mx = 0.f;
#pragma unroll
    for (int a = 0; a < 2; ++a)
#pragma unroll
        for (int b = 0; b < 2; ++b)
#pragma unroll
            for (int m = 0; m < 4; ++m)
#pragma unroll
                for (int n = 0; n < 2; ++n) mx = fmaxf(fmaxf(fmaxf(fmaxf(mx, acc[a][b][m][n][0]), acc[a][b][m][n][1]), acc[a][b][m][n][2]), acc[a][b][m][n][3]);
    const unsigned wm = wave_max(__float_as_uint(mx));
    {
        unsigned old = 0;
        if (lane == 0) {
            __hip_atomic_fetch_max(red + 32, wm, __ATOMIC_RELAXED, __HIP_MEMORY_SCOPE_WORKGROUP);
            old = __hip_atomic_fetch_add(red + 33, 1u, __ATOMIC_RELAXED, __HIP_MEMORY_SCOPE_WORKGROUP);
        }
        if (__builtin_amdgcn_readfirstlane(old) == 7) {
            const unsigned long long gv = ((unsigned long long)call << 32) | __hip_atomic_load(red + 32, __ATOMIC_RELAXED, __HIP_MEMORY_SCOPE_WORKGROUP);
            char* g = pl + OFF_AMAX + (rank * NGEMM + tile) * 8;
            unsigned long long ex_;
            asm volatile("s_mov_b64 %0, exec\n\ts_mov_b64 exec, 0xff\n\tglobal_store_dwordx2 %1, %2, off sc0 sc1\n\ts_mov_b64 exec, %0\n\ts_waitcnt vmcnt(1)" : "=&s"(ex_) : "v"(g), "v"(gv) : "memory");
        } else WAIT_V(0);
        unsigned old2 = 0;
        if (lane == 0) old2 = __hip_atomic_fetch_add(red + 34 + wr, 1u, __ATOMIC_RELAXED, __HIP_MEMORY_SCOPE_WORKGROUP);
        if (__builtin_amdgcn_readfirstlane(old2) == 3 && lane < 8) st_sys(pl + OFF_MFLG + (rank * 256 + tile * 2 + wr) * 4, call);
    }
#pragma unroll
    for (int a = 0; a < 2; ++a)
#pragma unroll
        for (int b = 0; b < 2; ++b)
#pragma unroll
            for (int m = 0; m < 4; ++m)
#pragma unroll
                for (int n = 0; n < 2; ++n)
#pragma unroll
                    for (int j = 0; j < 4; ++j) { acc[a][b][m][n][j] = fmaxf(acc[a][b][m][n][j], 0.f); if (j == 3) asm volatile("" : "+v"(acc[a][b][m][n])); }
    STAMP(myws, wv, 13);
    BAR; CFENCE;
    LAS unsigned char* A = lds + wid * EP_BYTES;
    LAS unsigned char* B = A + EP_C;
    LAS unsigned char* S2 = A + EP_S;
    const unsigned cnt = popc4(msk), scan = wave_scan(cnt), Lc = __builtin_amdgcn_readlane(scan, 63);
    const bool dense = Lc > L1_MAX;
    if (!dense) {
        unsigned o = (unsigned)(size_t)A + (scan - cnt) * 4;
        const unsigned spare = (unsigned)(size_t)(A + EP_X) + lane * 4;
#pragma unroll
        for (int e = 0; e < 128; ++e) {
            const int ch = e >> 4;
            const unsigned bit = __builtin_amdgcn_ubfe(msk[e >> 5], e & 31, 1);
            *(LAS float*)(size_t)(bit ? o ^ ((o >> 4) & 0x30u) : spare) = acc[ch >> 2][(e >> 1) & 1][ch & 3][e & 1][(e >> 2) & 3];
            o += bit * 4;
        }
    }
    float ysum = 0.f;
#pragma unroll
    for (int a = 0; a < 2; ++a)
#pragma unroll
        for (int b = 0; b < 2; ++b)
#pragma unroll
            for (int m = 0; m < 4; ++m) { const f32x4 t = acc[a][b][m][0] + acc[a][b][m][1]; ysum += (t[0] + t[1]) + (t[2] + t[3]); }
#pragma unroll
    for (int sh = 32; sh >= 1; sh >>= 1) ysum += __shfl_xor(ysum, sh);
    ysum *= 1.35f / (float)max(Lc, 1u);
    if (!dense && lane < 16 && Lc + lane < L1_MAX) *(LAS float*)(A + (((Lc + lane) * 4) ^ (((Lc + lane) >> 2) & 0x30u))) = ysum;
    LAS unsigned char* lut = A + EP_T;
    if (lane < 16) *(LAS unsigned*)(lut + lane * 4) = lut_compact(lane);
    {
        if (wid == 0) {
            Spin t0;
            for (;;) {
                CFENCE;
                const unsigned g = __builtin_amdgcn_raw_buffer_load_b32(rs, (lane & 31) * 8 + 4, OFF_AMAX + (rank * NGEMM + 96) * 8, AUX_SYS);
                if (__builtin_popcountll(__builtin_amdgcn_ballot_w64(g >= call)) >= 48 || t0.fail(myws, call)) break;
                __builtin_amdgcn_s_sleep(8);
            }
        }
        BAR; CFENCE;
    }
    {
        Spin t0;
        unsigned t, tm = ld_agent(myws + OFF_TMO);
        for (;;) {
            CFENCE;
            const u32x4 g = __builtin_amdgcn_raw_buffer_load_b128(rs, lane * 16, OFF_AMAX + wid * 1024, AUX_SYS);
            t = max(g[0], g[2]);
            if (__builtin_amdgcn_ballot_w64(g[1] < call || g[3] < call) == 0) break;
            if (t0.fail(myws, call)) { tm = call; break; }
            __builtin_amdgcn_s_sleep(4);
        }
        t = wave_max(t);
        if (lane == 0) { red[8 + wid] = t; red[wid] = tm == call; }
    }
    WAIT_L(0); BAR; CFENCE;
    const float amax = __uint_as_float(wave_max(red[8 + (lane & 7)]));
    const bool dead = __builtin_amdgcn_ballot_w64(red[lane & 7] != 0) != 0;
    const float inv = dead ? __builtin_nanf("") : 448.0f / amax;
    STAMP(myws, wv, 11);
    if (wr == 0) __builtin_amdgcn_s_setprio(2);

    const f32x2 inv2 = {inv, inv};
#define QUANT(ch, q) { const int a = (ch) >> 2, m = (ch) & 3; \
        const f32x4 y0 = scale4(acc[a][0][m][0], inv2), y1 = scale4(acc[a][0][m][1], inv2), y2 = scale4(acc[a][1][m][0], inv2), y3 = scale4(acc[a][1][m][1], inv2); \
        _Pragma("unroll") for (int j = 0; j < 4; ++j) \
            q[j] = (unsigned)__builtin_amdgcn_cvt_pk_fp8_f32(y2[j], y3[j], __builtin_amdgcn_cvt_pk_fp8_f32(y0[j], y1[j], __float_as_int(y0[j]), false), true); }
    u32x4 cod[5];
    if (dense) {
#pragma unroll
        for (int r = 0; r < 9; ++r) *(LAS u32x4*)(A + r * 1024 + lane * 16) = (u32x4){0u, 0u, 0u, 0u};
        unsigned o = (unsigned)(size_t)A + scan - cnt;
#pragma unroll
        for (int ch = 0; ch < 8; ++ch) {
            u32x4 q;
            QUANT(ch, q);
#pragma unroll
            for (int j = 0; j < 4; ++j) {
                const unsigned n4 = nib4(msk[ch >> 1], (ch & 1) * 4 + j);
                const unsigned long long z = (unsigned long long)__builtin_amdgcn_perm(0u, q[j], *(const LAS unsigned*)(lut + n4)) << ((o & 3u) * 8);
                LAS unsigned* d = (LAS unsigned*)(size_t)(o & ~3u);
                __hip_atomic_fetch_or(d, (unsigned)z, __ATOMIC_RELAXED, __HIP_MEMORY_SCOPE_WAVEFRONT);
                __hip_atomic_fetch_or(d + 1, (unsigned)(z >> 32), __ATOMIC_RELAXED, __HIP_MEMORY_SCOPE_WAVEFRONT);
                o += __builtin_popcount(n4);
            }
        }
    } else {
#define CODES(i0, n) { f32x4 y_[n][4]; \
            _Pragma("unroll") for (int i = 0; i < n; ++i) _Pragma("unroll") for (int k = 0; k < 4; ++k) y_[i][k] = *(const LAS f32x4*)(A + ((i0 + i) * 64 + (i0 + i < 4 ? lane : lane & 31)) * 64 + ((k ^ (lane >> 2)) & 3) * 16); \
            SCHED; \
            _Pragma("unroll") for (int i = 0; i < n; ++i) _Pragma("unroll") for (int k = 0; k < 4; ++k) { const f32x4 y = scale4(y_[i][k], inv2); \
                cod[i0 + i][k] = (unsigned)__builtin_amdgcn_cvt_pk_fp8_f32(y[2], y[3], __builtin_amdgcn_cvt_pk_fp8_f32(y[0], y[1], __float_as_int(y[0]), false), true); } }
        CODES(0, 2); CODES(2, 2);
        cod[4] = (u32x4){0u, 0u, 0u, 0u};
        if (Lc > 4096) CODES(4, 1);
        CFENCE;
#pragma unroll
        for (int i = 0; i < 5; ++i) *(LAS u32x4*)(A + (i * 64 + lane) * 16) = cod[i];
#pragma unroll
        for (int r = 0; r < 3; ++r) *(LAS u32x4*)(S2 + r * 1024 + lane * 16) = (u32x4){0u, 0u, 0u, 0u};
    }
    CFENCE;
    CYC(0);
    const unsigned ec = (__builtin_amdgcn_readfirstlane(__builtin_amdgcn_cvt_pk_fp8_f32(fminf(ysum * inv, 448.f), 0.f, 0, false)) >> 3) & 15u;
    const unsigned E1 = (ec + 17) * 0x01010101u, G = (Lc + 15) >> 4;
    bool raw = dense || (rt * 2 + wr) * 32 + ct * 4 + wc >= 1024 - RAW_LAST;
    unsigned bytes = Lc;
    if (!raw) {
        unsigned s2[5][4], n7[5][4], cnt2 = 0;
        const unsigned k07 = KS_(0x07070707u), k08 = KS_(0x08080808u), k0f = KS_(0x0f0f0f0fu), k10 = KS_(0x10101010u), k80 = KS_(H8);
#define CLASSIFY(i0, n) { const u32x4* d_ = cod + i0; \
            _Pragma("unroll") for (int i = 0; i < n; ++i) { const unsigned g = (i0 + i) * 64 + lane, wgt = g < G ? 0x08040201u : 0u; unsigned nb[4], p_[4]; \
                _Pragma("unroll") for (int k = 0; k < 4; ++k) { \
                    const unsigned e4 = (d_[i][k] >> 3) & k0f, u = E1 - e4, t = u & k08, ui = (u | (t - (t >> 3))) & k07; \
                    p_[k] = __builtin_amdgcn_perm(0x98989898u, 0x10080098u, ui); \
                    s2[i0 + i][k] = e4 | __builtin_amdgcn_perm(0x30302010u, 0x30303000u, ui); \
                    nb[k] = (d_[i][k] & k07) | (p_[k] & k08); \
                    n7[i0 + i][k] = __builtin_amdgcn_udot4(p_[k] & k80, wgt, 0u, false); \
                    cnt2 += __builtin_popcount(n7[i0 + i][k]); } \
                if (g < G) { *(LAS u32x2*)(B + g * 8) = (u32x2){nb[0] | (nb[1] << 4), nb[2] | (nb[3] << 4)}; \
                    *(LAS unsigned short*)(B + G * 8 + g * 2) = (unsigned short)bits16(p_[0] & k10, p_[1] & k10, p_[2] & k10, p_[3] & k10, 4); } } }
        CLASSIFY(0, 4);
#pragma unroll
        for (int k = 0; k < 4; ++k) s2[4][k] = n7[4][k] = 0;
        if (G > 256) CLASSIFY(4, 1);
        CYC(1);
        const unsigned sc2 = wave_scan(cnt2), n2 = __builtin_amdgcn_readlane(sc2, 63), G2 = (n2 + 15) >> 4, o2 = off_l2(G), o3 = o2 + G2 * 4;
        raw = n2 > L2_MAX;
        if (!raw) {
            unsigned o = sc2 - cnt2 + (unsigned)(size_t)S2;
#define SQUEEZE(i0, n) { unsigned se_[n][4]; \
                _Pragma("unroll") for (int i = 0; i < n; ++i) _Pragma("unroll") for (int k = 0; k < 4; ++k) se_[i][k] = *(const LAS unsigned*)(lut + (n7[i0 + i][k] >> 5)); \
                SCHED; \
                _Pragma("unroll") for (int i = 0; i < n; ++i) _Pragma("unroll") for (int k = 0; k < 4; ++k) { \
                    const unsigned long long z = (unsigned long long)__builtin_amdgcn_perm(0u, s2[i0 + i][k], se_[i][k]) << ((o & 3u) * 8); \
                    LAS unsigned* d = (LAS unsigned*)(size_t)(o & ~3u); \
                    __hip_atomic_fetch_or(d, (unsigned)z, __ATOMIC_RELAXED, __HIP_MEMORY_SCOPE_WAVEFRONT); \
                    __hip_atomic_fetch_or(d + 1, (unsigned)(z >> 32), __ATOMIC_RELAXED, __HIP_MEMORY_SCOPE_WAVEFRONT); \
                    o += __builtin_popcount(n7[i0 + i][k]); } }
            SQUEEZE(0, 4);
            if (G > 256) SQUEEZE(4, 1);
            CFENCE;
            CYC(2);
            const u32x4 d0 = *(const LAS u32x4*)(S2 + lane * 16), d1 = *(const LAS u32x4*)(S2 + 1024 + lane * 16);
            if (lane < G2) *(LAS unsigned*)(B + o2 + lane * 4) = ((d0[0] >> 4) & 0x03030303u) | ((d0[1] >> 2) & 0x0c0c0c0cu) | (d0[2] & 0x30303030u) | ((d0[3] << 2) & 0xc0c0c0c0u);
            if (64 + lane < G2) *(LAS unsigned*)(B + o2 + 256 + lane * 4) = ((d1[0] >> 4) & 0x03030303u) | ((d1[1] >> 2) & 0x0c0c0c0cu) | (d1[2] & 0x30303030u) | ((d1[3] << 2) & 0xc0c0c0c0u);
            unsigned m5[8], se[8], cnt3 = 0;
#pragma unroll
            for (int k = 0; k < 8; ++k) {
                const unsigned d = k < 4 ? d0[k & 3] : d1[k & 3];
                m5[k] = __builtin_amdgcn_udot4(d & (d << 1) & 0x20202020u, 0x08040201u, 0u, false);
                se[k] = *(const LAS unsigned*)(lut + (m5[k] >> 3));
                cnt3 += __builtin_popcount(m5[k]);
            }
            const unsigned sc3 = wave_scan(cnt3), n3 = __builtin_amdgcn_readlane(sc3, 63);
            bytes = o3 + ((n3 + 7) >> 3) * 4;
            raw = n3 > MAXESC || bytes > Lc;
            if (!raw) {
                o = sc3 - cnt3 + (unsigned)(size_t)(A + EP_L);
#pragma unroll
                for (int k = 0; k < 8; ++k) {
                    const unsigned long long z = (unsigned long long)__builtin_amdgcn_perm(0u, k < 4 ? d0[k & 3] : d1[k & 3], se[k]) << ((o & 3u) * 8);
                    LAS unsigned* d = (LAS unsigned*)(size_t)(o & ~3u);
                    __hip_atomic_fetch_or(d, (unsigned)z, __ATOMIC_RELAXED, __HIP_MEMORY_SCOPE_WAVEFRONT);
                    __hip_atomic_fetch_or(d + 1, (unsigned)(z >> 32), __ATOMIC_RELAXED, __HIP_MEMORY_SCOPE_WAVEFRONT);
                    o += __builtin_popcount(m5[k]);
                }
                CFENCE;
                const u32x2 x3 = *(const LAS u32x2*)(A + EP_L + lane * 8);
                *(LAS unsigned*)(B + o3 + lane * 4) = (x3[0] & 0x0f0f0f0fu) | ((x3[1] << 4) & 0xf0f0f0f0u);
                CFENCE;
            }
        }
    }
    if (raw) bytes = Lc;
    CYC(3);
    {
        LAS unsigned char* S = raw ? A : B;
#pragma nounroll
        for (unsigned r = 0; r < bytes; r += 1024)
            ST128(*(const LAS u32x4*)(S + r + lane * 16), rs, lane * 16, OFF_COD + wt * 8192 + r, AUX_SYS);
    }
    if (lane == 0) red[24 + wid] = (raw ? 1u << 30 : (bytes >> 2) << 14) | (ec << 26) | Lc;
    CYC(4);
    WAIT_V(0);
    CYC(5);
    {
        unsigned old = 0;
        if (lane == 0) old = __hip_atomic_fetch_add(red + 36 + wr, 1u, __ATOMIC_RELAXED, __HIP_MEMORY_SCOPE_WORKGROUP);
        if (__builtin_amdgcn_readfirstlane(old) == 3 && lane < 32 && (lane & 7) != 0) {
            const int w = wr * 4 + (lane >> 3);
            __hip_atomic_store((GAS unsigned long long*)(pl + OFF_DONE + (rank * 1024 + tile * 8 + w) * 8),
                               ((unsigned long long)((volatile LAS unsigned*)red)[24 + w] << 32) | call, __ATOMIC_RELAXED, __HIP_MEMORY_SCOPE_SYSTEM);
        }
    }
    STAMP(myws, wv, 12);
    {
        const rsrc_t ro = mk_rsrc(p.out, OUT_BYTES);
        const unsigned so = out_soff(rank, wt), vo = out_voff(lane);
        u32x4 qv[8];
#pragma unroll
        for (int ch = 0; ch < 8; ++ch) QUANT(ch, qv[ch]);
#pragma unroll
        for (int ch = 0; ch < 8; ++ch) ST128_RAW(qv[ch], ro, vo, so + chunk_soff(ch), AUX_WT);
        ST128_PAD8(qv);
    }
}

__global__ __launch_bounds__(512, 2) void fused_kernel(Args a) {
    __shared__ __attribute__((aligned(16))) unsigned char lds_arr[LDS_BYTES];
    LAS unsigned char* lds = (LAS unsigned char*)lds_arr;
    const int wid = __builtin_amdgcn_readfirstlane(threadIdx.x >> 6), lane = threadIdx.x & 63;
    const int b = blockIdx.x, xcd = b & 7, idx = b >> 3;
    Params p;
    p.x = a.x; p.w = a.w; p.out = a.out; p.my = (char*)a.ws[0]; p.rank = a.rank; p.pl = (char*)a.ws[lane & 7];
    asm volatile("" : "+s"(p.x), "+s"(p.w), "+s"(p.out), "+s"(p.my), "+s"(p.rank));
    const unsigned callv = (unsigned)__builtin_amdgcn_raw_ptr_buffer_atomic_add_i32(1, mk_rsrc(p.my, WS_USED), lane == 0 ? OFF_CNT + (b * 8 + wid) * 4 : 0xfffffff0u, 0, 0);
    if (idx < 16) gemm_role(p, lds, (xcd >> 1) * 4 + (idx >> 2), (xcd & 1) * 4 + (idx & 3), wid, lane, callv);
    else comm_role(p, lds, (idx - 16) * 8 + xcd, wid, lane, callv);
}

extern "C" void kernel_launch(int rank, int nranks, ncclComm_t comm, hipStream_t stream,
                              void* const* d_in, const int* in_sizes, int n_in,
                              void* d_out, int out_size,
                              const int* in_dtypes, int out_dtype,
                              void* const* d_out_all, void* const* d_ws_all, size_t ws_bytes) {
    (void)comm; (void)in_dtypes; (void)out_dtype; (void)d_out_all;
    if (nranks != NR || n_in != 2 || in_sizes[0] != M_ * KS || in_sizes[1] != KS * N_ || out_size != M_ * N_ || ws_bytes < WS_USED || (unsigned)rank >= (unsigned)NR) {
        fprintf(stderr, "kernel_launch: unexpected problem shape\n");
        return;
    }
    Args a;
    a.x = d_in[0]; a.w = d_in[1]; a.out = d_out;
    const size_t shift = (ws_bytes - WS_USED) & ~(size_t)0x1fffff;
    for (int i = 0; i < NR; ++i) a.ws[i] = (char*)d_ws_all[(rank + i) % NR] + shift;
    a.rank = rank; a.pad = 0;
    fused_kernel<<<dim3(NWG), dim3(512), 0, stream>>>(a);
}
```

```cpp
#include <hip/hip_runtime.h>
#include <rccl/rccl.h>
#include <cstdio>

#define LAS __attribute__((address_space(3)))
#define GAS __attribute__((address_space(1)))
typedef short bf16x8 __attribute__((ext_vector_type(8)));
typedef float f32x4 __attribute__((ext_vector_type(4)));
typedef unsigned u32x4 __attribute__((ext_vector_type(4)));
typedef unsigned u32x2 __attribute__((ext_vector_type(2)));
typedef __amdgpu_buffer_rsrc_t rsrc_t;

constexpr int NR = 8;
constexpr int M_ = 4096, N_ = 16384;
constexpr int KS = 512;
constexpr int NS = 2048;
constexpr int NT = 64;
constexpr int NGEMM = 128;
constexpr int NWG = 256;

constexpr unsigned OFF_CNT  = 0;
constexpr unsigned OFF_TMO  = 16384;
constexpr unsigned OFF_FLAG = 65536;
constexpr unsigned FLAG_W = 4096;
constexpr unsigned OFF_AMAX = 131072;
constexpr unsigned OFF_TICK = 143360;
constexpr unsigned OFF_PCNT = 147456;
constexpr unsigned OFF_PFLG = 163840;
constexpr unsigned OFF_PUF  = 188416;
constexpr unsigned OFF_MFLG = 176128;
constexpr unsigned OFF_DONE = 786432;
constexpr unsigned OFF_MF   = OFF_DONE + 65536;
constexpr unsigned OFF_DBG  = 262144;
constexpr unsigned CTRL_BYTES = 1u << 20;
constexpr unsigned XT_STEP = 512u * 1024u, WT_STEP = 256u * 1024u, BLK = 32768u, HALFB = 16384u;
constexpr unsigned OFF_XT = CTRL_BYTES;
constexpr unsigned XT_BYTES = NT * XT_STEP;
constexpr unsigned OFF_WT = OFF_XT + XT_BYTES;
constexpr unsigned WT_BYTES = NT * WT_STEP;
constexpr unsigned UNIT = 9216;
constexpr unsigned OFF_PX = OFF_WT + WT_BYTES;
constexpr unsigned OFF_PW = OFF_PX + 512 * UNIT;
constexpr unsigned OFF_MSK = OFF_PW + 2048 * UNIT;
constexpr unsigned OFF_COD = OFF_MSK + 1024 * 1024;
constexpr unsigned OFF_LM = OFF_COD + 1024 * 8192;
constexpr unsigned WS_USED = OFF_LM + 8 * 1024 * 1024;
constexpr unsigned OUT_BYTES = (unsigned)M_ * N_;
constexpr unsigned X_BYTES = (unsigned)M_ * KS * 2, W_BYTES = (unsigned)KS * N_ * 2;

constexpr unsigned long long DEADLINE = 100000000ull;
constexpr int AUX_SYS = 17;
#define WS_END 1
constexpr int AUX_NT = 2, AUX_WT = 16;
#define PPP 14
#define TEAM 4
#define PPO 55
#define CW 4
#define PUSH_W 8
#define PUSH_U 3
#define SCH_R0 1
#define SCH_R1 6
#define SCH_S 100
#define SCH_C 1000
#define SCH_B 10
#define SCH_A 1
constexpr int RAW_LAST = 24;
constexpr int STAG = 7;
constexpr int NPUF = 2;
constexpr int PACE_W = 800, PACE_W1 = 150, PACE_X = 400, PACE_E = 70, PACE_R = 120;

constexpr unsigned LDS_STAGE = 9 * HALFB;
constexpr unsigned EP_BYTES = 19456;
constexpr unsigned EP_C = 8192, EP_S = EP_C + 5888, EP_L = EP_S + 2080, EP_T = 18432, EP_X = EP_T + 64;
constexpr unsigned LDS_FLAG = 8 * EP_BYTES;
constexpr unsigned LDS_RED = LDS_FLAG + 512;
constexpr unsigned EX_B = 8192, EX_M = EX_B + 8192, EX_P = EX_M + 1024, EX_E = EX_P + 1024, EX_BYTES = EX_E + 256, EX_LUT = 0, EX_MRG = 2048, EX_TOK = 2176, EX_0 = 2304;
constexpr unsigned LDS_BYTES = LDS_RED + 256;
static_assert(EX_0 + 8 * EX_BYTES <= LDS_BYTES && EP_BYTES % 1024 == 0 && EP_L + 1024 <= EP_T && EP_X + 256 <= EP_BYTES && LDS_STAGE <= LDS_FLAG && LDS_RED + 192 <= LDS_BYTES && LDS_BYTES <= 163840, "lds");
static_assert(OFF_PUF + 8 * NPUF * 384 <= OFF_DBG, "flags");
static_assert((PUSH_W * PUSH_U) % 4 == 0 && PUSH_W * PUSH_U <= 32 && 7 * PUSH_W <= NGEMM, "push");
static_assert(7 * PPO <= 128 * CW && 7 * PPP <= 128 && PPP >= 8 && TEAM <= CW, "waves");

struct Args {
    const void* x; const void* w; void* out;
    int rank; int pad;
    void* ws[NR];
};
struct Params {
    const void* x; const void* w; void* out; char* my; char* pl;
    int rank;
    __device__ __forceinline__ char* peer(int k) const {
        const unsigned long long a = (unsigned long long)pl;
        return (char*)(((unsigned long long)(unsigned)__builtin_amdgcn_readlane((unsigned)(a >> 32), k) << 32) | (unsigned)__builtin_amdgcn_readlane((unsigned)a, k));
    }
};

__device__ __forceinline__ rsrc_t mk_rsrc(const void* p, unsigned bytes) {
    return __builtin_amdgcn_make_buffer_rsrc((void*)p, 0, bytes, 0x00020000);
}
#define ST128(v, r, vo, so, aux) do { const u32x4 v_ = (v); __builtin_amdgcn_raw_buffer_store_b128(v_, r, vo, so, aux); asm volatile("s_nop 1" :: "v"(v_) : "memory"); } while (0)
#define ST128_RAW __builtin_amdgcn_raw_buffer_store_b128
#define ST128_PAD4(v) asm volatile("s_nop 1" :: "v"((v)[0]), "v"((v)[1]), "v"((v)[2]), "v"((v)[3]) : "memory")
#define ST128_PAD8(v) asm volatile("s_nop 1" :: "v"((v)[0]), "v"((v)[1]), "v"((v)[2]), "v"((v)[3]), "v"((v)[4]), "v"((v)[5]), "v"((v)[6]), "v"((v)[7]) : "memory")
#define WAIT_V(n) asm volatile("s_waitcnt vmcnt(" #n ")" ::: "memory")
#define WAIT_L(n) asm volatile("s_waitcnt lgkmcnt(" #n ")" ::: "memory")
#define BAR __builtin_amdgcn_s_barrier()
#define SCHED __builtin_amdgcn_sched_barrier(0)
#define CFENCE asm volatile("" ::: "memory")

#define STAMP(ws_, wv_, k_) do {} while (0)
#define CYC(i) do {} while (0)

__device__ __forceinline__ unsigned ld_agent(const void* p) { return __hip_atomic_load((const GAS unsigned*)p, __ATOMIC_RELAXED, __HIP_MEMORY_SCOPE_AGENT); }
__device__ __forceinline__ void st_agent(void* p, unsigned v) { __hip_atomic_store((GAS unsigned*)p, v, __ATOMIC_RELAXED, __HIP_MEMORY_SCOPE_AGENT); }
__device__ __forceinline__ void st_sys(void* p, unsigned v) { __hip_atomic_store((GAS unsigned*)p, v, __ATOMIC_RELAXED, __HIP_MEMORY_SCOPE_SYSTEM); }

__device__ __forceinline__ void add_sys(void* p) { asm volatile("global_atomic_add %0, %1, off sc1" :: "v"(p), "v"(1u) : "memory"); }
__device__ __forceinline__ unsigned x_flag(int t, int g) { return OFF_FLAG + (g >> 2) * 256 + t * 4; }
__device__ __forceinline__ unsigned w_flag(int t, int e) { return OFF_FLAG + FLAG_W + (e >> 2) * 256 + t * 4; }

struct Spin {
    unsigned long long t0; unsigned n;
    __device__ __forceinline__ Spin() : t0(__builtin_amdgcn_s_memrealtime()), n(0) {}
    __device__ __forceinline__ bool fail(char* myws, unsigned call) {
        if ((++n & 15u) == 0 && (__builtin_amdgcn_readfirstlane(ld_agent(myws + OFF_TMO)) == call || __builtin_amdgcn_s_memrealtime() - t0 > DEADLINE)) {
            st_agent(myws + OFF_TMO, call);
            return true;
        }
        __builtin_amdgcn_s_sleep(1);
        return false;
    }
};

#define DPP0(x, ctrl, rm) (unsigned)__builtin_amdgcn_update_dpp(0, (int)(x), ctrl, rm, 0xf, false)
__device__ __forceinline__ unsigned wave_scan(unsigned x) {
    x += DPP0(x, 0x111, 0xf); x += DPP0(x, 0x112, 0xf); x += DPP0(x, 0x114, 0xf); x += DPP0(x, 0x118, 0xf);
    x += DPP0(x, 0x142, 0xa); x += DPP0(x, 0x143, 0xc);
    return x;
}
__device__ __forceinline__ unsigned wave_max(unsigned x) {
    x = max(x, DPP0(x, 0x111, 0xf)); x = max(x, DPP0(x, 0x112, 0xf)); x = max(x, DPP0(x, 0x114, 0xf)); x = max(x, DPP0(x, 0x118, 0xf));
    x = max(x, DPP0(x, 0x142, 0xa)); x = max(x, DPP0(x, 0x143, 0xc));
    return __builtin_amdgcn_readlane(x, 63);
}

struct Sched {
    unsigned w[7][64];
    constexpr Sched() : w{} {
        for (int q = 0; q < 7; ++q) {
            int key[136] = {}, id[136] = {}, n = 0;
            for (int j = 0; j < 8; ++j) {
                int lo = 1 << 30;
                for (int rt = 0; rt < 16; ++rt) { const int k = ((SCH_S * rt < SCH_C ? SCH_S * rt : SCH_C) + SCH_B * rt) * (j + SCH_R0 < SCH_R1 ? j + SCH_R0 : SCH_R1) / SCH_R1 + 106 * (7 * j + (q + SCH_A * rt) % 7); key[n] = 2 * k + 1; id[n++] = j * 24 + 8 + rt; lo = k < lo ? k : lo; }
                key[n] = 2 * lo; id[n++] = -1 - j;
            }
            int at = 0;
            for (int o = 0; o < 136; ++o) {
                int best = -1;
                for (int i = 0; i < 136; ++i) if (key[i] >= 0 && (best < 0 || key[i] < key[best])) best = i;
                key[best] = -1;
                const int skip = id[best] == -1 ? PUSH_W * PUSH_U / 4 : 0;
                const int cnt = id[best] < 0 ? 8 - skip : 1, first = id[best] < 0 ? (-1 - id[best]) * 24 + skip : id[best];
                for (int c = 0; c < cnt; ++c, ++at) w[q][at >> 2] |= (unsigned)(first + c) << (8 * (at & 3));
            }
        }
    }
};
__constant__ const Sched SCHED_TAB;
__device__ __forceinline__ int step_of(int i, int rank, int rt) {
    if (i < 8) return rank * 8 + i;
    const int z = i - 8, j = z / 7, p = z - j * 7, q = (p + 7 * 16 * SCH_A - SCH_A * rt) % 7;
    return (((rank + 1 + q) & 7) << 3) + j;
}

__device__ __forceinline__ void x_load(u32x4 (&v)[8], rsrc_t rx, int j, int g, int lane) {
    const unsigned voff = (unsigned)(lane & 15) * (KS * 2) + (unsigned)(lane >> 4) * 16;
    const unsigned soff = (unsigned)g * 64 * (KS * 2) + (unsigned)j * 128;
#pragma unroll
    for (int r = 0; r < 4; ++r)
#pragma unroll
        for (int kk = 0; kk < 2; ++kk) v[r * 2 + kk] = __builtin_amdgcn_raw_buffer_load_b128(rx, voff + kk * 64, soff + r * 16 * (KS * 2), 0);
}
template <int AUX = AUX_NT> __device__ __forceinline__ void w_load(u32x2 (&v)[16], rsrc_t rw, int j, int d, int ct, int kk, int G, int lane) {
    const unsigned voff = (unsigned)(lane >> 4) * 8 * (N_ * 2) + (unsigned)(lane & 15) * 8;
    const unsigned soff = (unsigned)(j * 64 + kk * 32) * (N_ * 2) + (unsigned)(d * NS + ct * 256 + G * 128) * 2;
#pragma unroll
    for (int jj = 0; jj < 8; ++jj)
#pragma unroll
        for (int g = 0; g < 2; ++g) v[g * 8 + jj] = __builtin_amdgcn_raw_buffer_load_b64(rw, voff + g * 128, soff + jj * (N_ * 2), AUX);
}
__device__ __forceinline__ void w_transpose(u32x4 (&o)[8], const u32x2 (&v)[16]) {
#pragma unroll
    for (int g = 0; g < 2; ++g)
#pragma unroll
        for (int c = 0; c < 4; ++c)
#pragma unroll
            for (int dd = 0; dd < 4; ++dd)
                o[g * 4 + c][dd] = __builtin_amdgcn_perm(v[g * 8 + 2 * dd + 1][c >> 1], v[g * 8 + 2 * dd][c >> 1], (c & 1) ? 0x07060302u : 0x05040100u);
}
__device__ __forceinline__ unsigned x_img(int t, int g) { return OFF_XT + (unsigned)t * XT_STEP + (unsigned)g * 8192; }
__device__ __forceinline__ unsigned w_img(int t, int e) { return OFF_WT + (unsigned)t * WT_STEP + (unsigned)(e >> 2) * BLK + (unsigned)((e >> 1) & 1) * 8192 + (unsigned)(e & 1) * 1024; }
__device__ __forceinline__ unsigned frag_off(bool isw, unsigned f) { return isw ? ((f >> 1) & 1) * HALFB + (f >> 2) * 4096 + (f & 1) * 2048 : f * 1024; }
__device__ __forceinline__ void img_store(const u32x4 (&v)[8], rsrc_t rd, bool isw, unsigned img, int lane) {
#pragma unroll
    for (int f = 0; f < 8; ++f) ST128_RAW(v[f], rd, lane * 16, img + frag_off(isw, f), AUX_SYS);
    ST128_PAD8(v);
}
constexpr unsigned H8 = 0x80808080u, MAXESC = 512;
__device__ __forceinline__ unsigned KS_(unsigned c) { asm("" : "+s"(c)); return c; }
__device__ __forceinline__ unsigned bits16(unsigned m0, unsigned m1, unsigned m2, unsigned m3, int sh) {
    const unsigned a = __builtin_amdgcn_udot4(m0, 0x08040201u, __builtin_amdgcn_udot4(m1, 0x80402010u, 0u, false), false);
    const unsigned b = __builtin_amdgcn_udot4(m2, 0x08040201u, __builtin_amdgcn_udot4(m3, 0x80402010u, 0u, false), false);
    return (a >> sh) | (b << (8 - sh));
}
__device__ __forceinline__ u32x2 lut_expand(unsigned byte) { u32x2 s = {0u, 0u}; unsigned c = 0;
#pragma unroll
    for (int i = 0; i < 8; ++i) { const unsigned b = (byte >> i) & 1u; s[i >> 2] |= (b ? c : 0x0cu) << (8 * (i & 3)); c += b; } return s; }
__device__ __forceinline__ unsigned lut_merge(unsigned nib) { unsigned s = 0, c = 0;
#pragma unroll
    for (int i = 0; i < 4; ++i) { const unsigned b = (nib >> i) & 1u; s |= (b ? c : 4u + i) << (8 * i); c += b; } return s; }
template <int N> __device__ __forceinline__ void merge(unsigned* x, const unsigned* n8, unsigned& o, const LAS unsigned char* mrg) {
    unsigned se[N], w0[N], w1[N], of[N];
#pragma unroll
    for (int k = 0; k < N; ++k) { se[k] = *(const LAS unsigned*)(mrg + n8[k]); const LAS unsigned* w = (const LAS unsigned*)(size_t)(o & ~3u); w0[k] = w[0]; w1[k] = w[1]; of[k] = o; o += __builtin_popcount(n8[k]); }
    SCHED;
#pragma unroll
    for (int k = 0; k < N; ++k) x[k] = __builtin_amdgcn_perm(x[k], __builtin_amdgcn_alignbyte(w1[k], w0[k], of[k]), se[k]);
}
__device__ __forceinline__ unsigned lut_compact(unsigned nib) { unsigned s = 0x0c0c0c0cu, c = 0;
#pragma unroll
    for (int i = 0; i < 4; ++i) if ((nib >> i) & 1u) { s = (s & ~(0xffu << (8 * c))) | ((unsigned)i << (8 * c)); ++c; } return s; }
__device__ __forceinline__ void squeeze(unsigned x, unsigned sel, unsigned o) {
    const unsigned long long z = (unsigned long long)__builtin_amdgcn_perm(0u, x, sel) << ((o & 3u) * 8);
    LAS unsigned* d = (LAS unsigned*)(size_t)(o & ~3u);
    __hip_atomic_fetch_or(d, (unsigned)z, __ATOMIC_RELAXED, __HIP_MEMORY_SCOPE_WAVEFRONT);
    __hip_atomic_fetch_or(d + 1, (unsigned)(z >> 32), __ATOMIC_RELAXED, __HIP_MEMORY_SCOPE_WAVEFRONT);
}
constexpr unsigned PK_HDR = 5 * 1024, PK_RAW = 0x80000000u, PK_FIRST = 384;
__device__ __forceinline__ unsigned clamp7(unsigned z) {
    const unsigned o = (((z & KS_(0x78787878u)) + 0x78787878u) | z) & H8;
    return (z | ((o >> 4) - (o >> 7))) & 0x07070707u;
}
__device__ __forceinline__ void pack_store(const u32x4 (&v)[8], rsrc_t rs, LAS unsigned char* L, unsigned slot, int lane) {
#pragma unroll
    for (int r = 0; r < 3; ++r) *(LAS u32x4*)(L + r * 1024 + lane * 16) = (u32x4){0u, 0u, 0u, 0u};
    if (lane < 16) *(LAS unsigned*)(L + 4096 + lane * 4) = lut_compact(lane);
    unsigned p0[16], z[16], n7[16], sum = 0, cnt = 0;
#pragma unroll
    for (int r = 0; r < 16; ++r) {
        const unsigned lo = __builtin_amdgcn_perm(v[r >> 1][2 * (r & 1) + 1], v[r >> 1][2 * (r & 1)], 0x06040200u);
        const unsigned hi = __builtin_amdgcn_perm(v[r >> 1][2 * (r & 1) + 1], v[r >> 1][2 * (r & 1)], 0x07050301u);
        p0[r] = (hi & H8) | (lo & ~H8);
        z[r] = ((hi << 1) & 0xfefefefeu) | ((lo >> 7) & 0x01010101u);
        sum = __builtin_amdgcn_sad_u8(z[r], 0u, sum);
    }
    const unsigned ec = min(max((__builtin_amdgcn_readlane(wave_scan(sum), 63) + 7782u) >> 12, 19u), 253u), C = (ec + 1) * 0x01010101u;
    u32x4 p2 = {0u, 0u, 0u, 0u};
#pragma unroll
    for (int r = 0; r < 16; ++r) {
        z[r] = ((C | H8) - (z[r] & KS_(~H8))) ^ ((C ^ ~z[r]) & H8);
        const unsigned c = __builtin_amdgcn_perm(0x83838383u, 0x02010083u, clamp7(z[r]));
        p2[r >> 2] |= (c & KS_(0x03030303u)) << (2 * (r & 3));
        n7[r] = __builtin_amdgcn_udot4(c & KS_(H8), 0x08040201u, 0u, false);
        cnt += __builtin_popcount(n7[r]);
    }
    const unsigned sc = wave_scan(cnt), n2 = __builtin_amdgcn_readlane(sc, 63), G2 = (n2 + 15) >> 4;
    bool bad = n2 > 2048;
    unsigned bytes = 0;
    if (!bad) {
        unsigned se[16], o = sc - cnt + (unsigned)(size_t)L;
#pragma unroll
        for (int r = 0; r < 16; ++r) se[r] = *(const LAS unsigned*)(L + 4096 + (n7[r] >> 5));
#pragma unroll
        for (int r = 0; r < 16; ++r) { squeeze(z[r], se[r], o); o += __builtin_popcount(n7[r]); }
        CFENCE;
        const u32x4 d0 = *(const LAS u32x4*)(L + lane * 16), d1 = *(const LAS u32x4*)(L + 1024 + lane * 16);
        unsigned pk[2] = {0u, 0u}, m7[8], cnt3 = 0;
#pragma unroll
        for (int k = 0; k < 8; ++k) {
            const unsigned c = __builtin_amdgcn_perm(0x83830201u, 0x83838300u, clamp7(k < 4 ? d0[k & 3] : d1[k & 3]));
            pk[k >> 2] |= (c & KS_(0x03030303u)) << (2 * (k & 3));
            m7[k] = __builtin_amdgcn_udot4(c & KS_(H8), 0x08040201u, 0u, false);
            se[k] = *(const LAS unsigned*)(L + 4096 + (m7[k] >> 5));
            cnt3 += __builtin_popcount(m7[k]);
        }
        if (lane < G2) *(LAS unsigned*)(L + 3076 + lane * 4) = pk[0];
        if (64 + lane < G2) *(LAS unsigned*)(L + 3076 + 256 + lane * 4) = pk[1];
        const unsigned sc3 = wave_scan(cnt3), n3 = __builtin_amdgcn_readlane(sc3, 63);
        bad = n3 > 512;
        bytes = 4 + G2 * 4 + ((n3 + 7) >> 3) * 4;
        if (!bad) {
            o = sc3 - cnt3 + (unsigned)(size_t)(L + 2304);
#pragma unroll
            for (int k = 0; k < 8; ++k) { squeeze((k < 4 ? d0[k & 3] : d1[k & 3]) ^ 0x06060606u, se[k], o); o += __builtin_popcount(m7[k]); }
            CFENCE;
            const u32x2 x = *(const LAS u32x2*)(L + 2304 + lane * 8);
            unsigned nb[2], b = 0;
#pragma unroll
            for (int k = 0; k < 2; ++k) {
                const unsigned y = x[k] ^ 0x06060606u, ff = ((y & ~H8) + 0x01010101u) & y & H8;
                b |= (((y & ~H8) + 0x6b6b6b6bu) | y) & H8 & ~ff;
                nb[k] = ((y + 0x0a0a0a0au - ((ff >> 7) * 10u)) | ((ff >> 3) - (ff >> 7))) & 0x0f0f0f0fu;
            }
            *(LAS unsigned*)(L + 3076 + G2 * 4 + lane * 4) = nb[0] | (nb[1] << 4);
            bad = __builtin_amdgcn_ballot_w64(b != 0) != 0;
        }
    }
    if (lane == 0) *(LAS unsigned*)(L + 3072) = bad ? PK_RAW : ec | (bytes << 8);
    CFENCE;
    ST128(*(const LAS u32x4*)(L + 3072 + lane * 16), rs, lane * 16 + 1024, slot + 4096, AUX_SYS);
    if (!bad) {
#pragma unroll
        for (int r = 0; r < 4; ++r) ST128(((u32x4){p0[4 * r], p0[4 * r + 1], p0[4 * r + 2], p0[4 * r + 3]}), rs, lane * 16 + r * 1024, slot, AUX_SYS);
        ST128(p2, rs, lane * 16, slot + 4096, AUX_SYS);
    } else {
#pragma unroll
        for (int f = 0; f < 8; ++f) ST128(v[f], rs, lane * 16, slot + (f < 5 ? f : f + 1) * 1024, AUX_SYS);
    }
}
struct Puller {
    rsrc_t rp, rs; char* myws; LAS unsigned char* lut; int rank, src, q, lane; unsigned call, seen, tab;
    __device__ __forceinline__ int unit_of(int tk) const { return (int)((__builtin_amdgcn_readlane(tab, tk >> 4) >> (2 * (tk & 12))) & 0xffu) * 4 + (tk & 3); }
    __device__ __forceinline__ int xg(int i) const { return i - 32; }
    __device__ __forceinline__ unsigned unit_id(int u) const { const int j = u / 96, i = u % 96; return i < 32 ? 512 + (rank * 8 + j) * 32 + i : j * 64 + xg(i); }
    __device__ __forceinline__ unsigned flag_of(int u) const { const int j = u / 96, i = u % 96; return i < 32 ? w_flag(src * 8 + j, i) : x_flag(src * 8 + j, xg(i)); }
    __device__ __forceinline__ void ready(int u) {
        const int j = u / 96, i = u % 96, need = j * 2 + (i >= 32);
        if (!((seen >> need) & 1u)) {
            Spin t0;
            for (;;) {
                CFENCE;
                const unsigned sum = OFF_PFLG + (src * 8 + j) * 128 + (i >= 32 ? 0 : 4);
                const unsigned f = __builtin_amdgcn_raw_buffer_load_b32(rs, (lane & 1) && j < NPUF ? OFF_PUF + ((src * NPUF + j) * 96 + i) * 4 : sum, 0, AUX_SYS);
                if (__builtin_amdgcn_readlane(f, 0) >= call) { seen |= 1u << need; break; }
                if (__builtin_amdgcn_readlane(f, 1) >= call) break;
                if (t0.fail(myws, call)) break;
                __builtin_amdgcn_s_sleep(4);
            }
            CFENCE;
        }
    }
    __device__ __forceinline__ void issue(LAS unsigned char* b, int u) {
        const unsigned id = unit_id(u), slot = OFF_PX + id * UNIT;
        __builtin_amdgcn_raw_ptr_buffer_load_lds(rp, (LAS void*)(b + 4096), 16, lane < PK_FIRST / 16 ? lane * 16 : 0xf0000000u, slot + 4096, 1024, AUX_SYS);
        __builtin_amdgcn_raw_ptr_buffer_load_lds(rp, (LAS void*)(b + 4096), 16, lane * 16, slot + 4096, 0, AUX_SYS);
        __builtin_amdgcn_raw_ptr_buffer_load_lds(rp, (LAS void*)b, 16, lane * 16, slot, 0, AUX_SYS);
        __builtin_amdgcn_raw_ptr_buffer_load_lds(rp, (LAS void*)b, 16, lane * 16, slot, 1024, AUX_SYS);
        __builtin_amdgcn_raw_ptr_buffer_load_lds(rp, (LAS void*)b, 16, lane * 16, slot, 2048, AUX_SYS);
        __builtin_amdgcn_raw_ptr_buffer_load_lds(rp, (LAS void*)b, 16, lane * 16, slot, 3072, AUX_SYS);
    }
    __device__ __forceinline__ void finish(LAS unsigned char* b, int u) {
        const int j = u / 96, i = u % 96, t = src * 8 + j;
        const bool isw = i < 32;
        const unsigned img = isw ? w_img(t, i) : x_img(t, xg(i));
        const unsigned h = *(const LAS unsigned*)(b + PK_HDR);
        u32x4 v[8];
#pragma unroll
        for (int r = 0; r < 5; ++r) v[r] = *(const LAS u32x4*)(b + r * 1024 + lane * 16);
        const unsigned meta = __builtin_amdgcn_readfirstlane(h), ec = meta & 0xffu;
        if (!(meta & PK_RAW)) {
            if ((meta >> 8) > PK_FIRST) {
                __builtin_amdgcn_raw_ptr_buffer_load_lds(rp, (LAS void*)(b + 4096), 16, lane * 16, OFF_PX + unit_id(u) * UNIT + 4096, 1024, AUX_SYS);
                WAIT_V(0);
            }
            const unsigned T1 = ec | ((ec - 1) << 8) | ((ec - 2) << 16), T2 = (ec + 1) | ((ec - 3) << 8) | ((ec - 4) << 16), k01 = KS_(0x01010101u), k03 = KS_(0x03030303u);
            const u32x4 p2 = v[4];
            unsigned pk0 = *(const LAS unsigned*)(b + PK_HDR + 4 + lane * 4), pk1 = *(const LAS unsigned*)(b + PK_HDR + 260 + lane * 4);
            u32x4 p0[4];
            unsigned e[16], n8[16], cnt = 0;
#pragma unroll
            for (int w = 0; w < 4; ++w) {
                const unsigned ot = p2[w] & (p2[w] >> 1) & KS_(0x55555555u);
                cnt += __builtin_popcount(ot);
#pragma unroll
                for (int k = 0; k < 4; ++k) {
                    e[4 * w + k] = __builtin_amdgcn_perm(0u, T1, (p2[w] >> (2 * k)) & k03);
                    n8[4 * w + k] = __builtin_amdgcn_udot4((ot >> (2 * k)) & k01, 0x40201008u, 0u, false);
                }
            }
            const unsigned sc = wave_scan(cnt), G2 = min((__builtin_amdgcn_readlane(sc, 63) + 15) >> 4, 128u);
            const unsigned x3 = *(const LAS unsigned*)(b + PK_HDR + 4 + G2 * 4 + lane * 4), B5 = (ec - 5) * 0x01010101u;
#pragma unroll
            for (int r = 0; r < 4; ++r) p0[r] = v[r];
            {
                const unsigned d0 = x3 & 0x0f0f0f0fu, d1 = (x3 >> 4) & 0x0f0f0f0fu, t0 = (d0 + k01) & 0x10101010u, t1 = (d1 + k01) & 0x10101010u;
                *(LAS u32x2*)(b + lane * 8) = (u32x2){B5 + t0 + (t0 >> 2) + (t0 >> 3) - d0, B5 + t1 + (t1 >> 2) + (t1 >> 3) - d1};
            }
            pk0 = lane < G2 ? pk0 : 0u; pk1 = 64 + lane < G2 ? pk1 : 0u;
            const unsigned ot0 = pk0 & (pk0 >> 1) & KS_(0x55555555u), ot1 = pk1 & (pk1 >> 1) & KS_(0x55555555u), cnt3 = __builtin_popcount(ot0) + __builtin_popcount(ot1);
            unsigned o = wave_scan(cnt3) - cnt3 + (unsigned)(size_t)b;
            CFENCE;
#define IN_L2(pk, ot, at) { unsigned v2[4], m8[4]; \
                _Pragma("unroll") for (int k = 0; k < 4; ++k) { v2[k] = __builtin_amdgcn_perm(0u, T2, (pk >> (2 * k)) & k03); m8[k] = __builtin_amdgcn_udot4((ot >> (2 * k)) & k01, 0x40201008u, 0u, false); } \
                merge<4>(v2, m8, o, lut); \
                *(LAS u32x4*)(b + at + lane * 16) = (u32x4){v2[0], v2[1], v2[2], v2[3]}; }
            IN_L2(pk0, ot0, 6144);
            if (G2 > 64) IN_L2(pk1, ot1, 7168);
            CFENCE;
            o = sc - cnt + (unsigned)(size_t)(b + 6144);
            merge<16>(e, n8, o, lut);
#pragma unroll
            for (int r = 0; r < 16; ++r) {
                const unsigned pp = p0[r >> 2][r & 3], ee = e[r];
                const unsigned lo = (pp & ~H8) | ((ee << 7) & H8), hi = (pp & H8) | ((ee >> 1) & ~H8);
                v[r >> 1][2 * (r & 1)] = __builtin_amdgcn_perm(hi, lo, 0x05010400u); v[r >> 1][2 * (r & 1) + 1] = __builtin_amdgcn_perm(hi, lo, 0x07030602u);
            }
        } else {
            const unsigned slot = OFF_PX + unit_id(u) * UNIT;
            WAIT_L(0);
            __builtin_amdgcn_raw_ptr_buffer_load_lds(rp, (LAS void*)b, 16, lane * 16, slot + 6144, 0, AUX_SYS);
            __builtin_amdgcn_raw_ptr_buffer_load_lds(rp, (LAS void*)b, 16, lane * 16, slot + 6144, 1024, AUX_SYS);
            __builtin_amdgcn_raw_ptr_buffer_load_lds(rp, (LAS void*)b, 16, lane * 16, slot + 6144, 2048, AUX_SYS);
            WAIT_V(0);
#pragma unroll
            for (int r = 5; r < 8; ++r) v[r] = *(const LAS u32x4*)(b + (r - 5) * 1024 + lane * 16);
        }
        img_store(v, rs, isw, img, lane);
    }
};
static_assert(OFF_PW == OFF_PX + 512 * UNIT, "unit ids run through the x and the w slots");

#define DRAW(r, tick) { unsigned long long ex_; asm volatile("s_mov_b64 %1, exec\n\ts_mov_b64 exec, 1\n\tglobal_atomic_add %0, %2, %3, %4 sc0\n\ts_mov_b64 exec, %1" \
        : "=&v"(r), "=&s"(ex_) : "v"(0u), "v"(1u), "s"(tick) : "memory"); }
#define TAKE(t, r, N) { unsigned t_; asm volatile("s_waitcnt vmcnt(" #N ")\n\tv_readfirstlane_b32 %0, %1" : "=s"(t_) : "v"(r) : "memory"); t = (int)(t_ - tbase); }
constexpr int NPUSH = PUSH_W * PUSH_U, NU_IN = 768 - NPUSH;

__device__ __forceinline__ void pull_role(const Params& p, LAS unsigned char* lds, LAS unsigned char* L, int c, int lane, unsigned call, int wv) {
    Puller P; P.rank = p.rank; P.q = c % 7; P.src = (p.rank + 1 + P.q) & 7; P.lane = lane; P.seen = 0; P.call = call; P.lut = lds + EX_MRG;
    P.tab = SCHED_TAB.w[P.q][lane];
    asm volatile("" : "+v"(P.tab));
    P.myws = p.my; P.rs = mk_rsrc(P.myws, WS_USED); P.rp = mk_rsrc(p.peer(1 + P.q), WS_USED);
    char* tick = P.myws + OFF_TICK + P.q * 128;
    const unsigned tbase = (call - 1) * (NU_IN + TEAM * PPP);
    LAS unsigned* token = (LAS unsigned*)(lds + EX_TOK);
#define TOKEN_GET { unsigned o_ = 0; if (lane == 0) o_ = __hip_atomic_fetch_add(token, 1u, __ATOMIC_RELAXED, __HIP_MEMORY_SCOPE_WORKGROUP); \
        turn = __builtin_amdgcn_readfirstlane(o_); \
        while (__builtin_amdgcn_readfirstlane(*(volatile LAS unsigned*)(token + 1)) != turn) __builtin_amdgcn_s_sleep(1); }
#define TOKEN_PUT { if (lane == 0) *(volatile LAS unsigned*)(token + 1) = turn + 1; }
    unsigned turn;
    unsigned r;
    int u;
    DRAW(r, tick); TAKE(u, r, 0);
    while ((unsigned)u < (unsigned)NU_IN) {
        const int un = P.unit_of(u);
        P.ready(un);
        TOKEN_GET;
        P.issue(L, un);
        WAIT_V(0);
        TOKEN_PUT;
        P.finish(L, un);
        DRAW(r, tick);
        TAKE(u, r, 0);
        if (lane == 0) add_sys(P.myws + P.flag_of(un));
    }
    STAMP(P.myws, wv, 9);
    volatile LAS unsigned* mst = (volatile LAS unsigned*)(token + 2);
    const int k0 = c / 7;
    const unsigned all = (1u << ((256 - k0 + PPP - 1) / PPP)) - 1u;
    for (;;) {
        TOKEN_GET;
        const unsigned td = all & ~__builtin_amdgcn_readfirstlane(mst[0]);
        unsigned rd = __builtin_amdgcn_readfirstlane(mst[1]);
        if (td == 0) { TOKEN_PUT; break; }
        if ((rd & td) == 0) {
            Spin t0;
            const unsigned at = __builtin_amdgcn_inverse_ballot_w64(td & ((td & -td) * 255u)) ? (k0 + lane * PPP) * 4 : 0xf0000000u;
            for (;;) {
                CFENCE;
                rd |= (unsigned)__builtin_amdgcn_ballot_w64(__builtin_amdgcn_raw_buffer_load_b32(P.rs, at, OFF_MFLG + P.src * 1024, AUX_SYS) >= call);
                if ((rd & td) != 0) break;
                if (t0.fail(P.myws, call)) { rd = ~0u; break; }
                __builtin_amdgcn_s_sleep(8);
            }
            CFENCE;
        }
        const int mi = __builtin_ctz(rd & td), mu = k0 + mi * PPP;
        if (lane == 0) { mst[0] = (all & ~td) | (1u << mi); mst[1] = rd; }
        __builtin_amdgcn_raw_ptr_buffer_load_lds(P.rp, (LAS void*)L, 16, lane * 16, OFF_MSK + mu * 4096, 0, AUX_SYS);
        __builtin_amdgcn_raw_ptr_buffer_load_lds(P.rp, (LAS void*)L, 16, lane * 16, OFF_MSK + mu * 4096, 1024, AUX_SYS);
        __builtin_amdgcn_raw_ptr_buffer_load_lds(P.rp, (LAS void*)L, 16, lane * 16, OFF_MSK + mu * 4096, 2048, AUX_SYS);
        __builtin_amdgcn_raw_ptr_buffer_load_lds(P.rp, (LAS void*)L, 16, lane * 16, OFF_MSK + mu * 4096, 3072, AUX_SYS);
        WAIT_V(0);
        TOKEN_PUT;
        u32x4 mv[4];
#pragma unroll
        for (int k = 0; k < 4; ++k) mv[k] = *(const LAS u32x4*)(L + k * 1024 + lane * 16);
#pragma unroll
        for (int k = 0; k < 4; ++k) ST128_RAW(mv[k], P.rs, lane * 16 + k * 1024, OFF_LM + P.src * 1048576 + mu * 4096, AUX_SYS);
        ST128_PAD4(mv);
        WAIT_V(0);
        if (lane == 0) st_sys(P.myws + OFF_MF + (P.src * 256 + mu) * 4, call);
    }
    STAMP(P.myws, wv, 17);
}

constexpr unsigned L1_MAX = 4608, L2_MAX = 2048;
__device__ __forceinline__ unsigned off_l2(unsigned G) { return G * 8 + ((G * 2 + 3) & ~3u); }
__device__ __forceinline__ unsigned popc4(const u32x4& m) { return __builtin_popcount(m[0]) + __builtin_popcount(m[1]) + __builtin_popcount(m[2]) + __builtin_popcount(m[3]); }
__device__ __forceinline__ unsigned nib4(unsigned m, int i) { return (i == 0 ? m << 2 : m >> (4 * i - 2)) & 0x3cu; }
__device__ __forceinline__ unsigned out_voff(int lane) { return (unsigned)(lane & 15) * N_ + (unsigned)(lane >> 4) * 16; }
__device__ __forceinline__ unsigned out_soff(int src, int wt) {
    const int tile = wt >> 3, w = wt & 7;
    return (unsigned)((tile >> 3) * 256 + (w >> 2) * 64) * N_ + (unsigned)(src * NS + (tile & 7) * 256 + (w & 3) * 64);
}
__device__ __forceinline__ constexpr unsigned chunk_soff(int ch) { return (unsigned)((ch >> 2) * 128 + (ch & 3) * 16) * N_; }

__device__ __forceinline__ void expand_role(const Params& p, LAS unsigned char* lds, LAS unsigned char* L, int ei, int lane, unsigned call, int wv) {
    const int rank = p.rank, q = ei % 7, src = (rank + 1 + q) & 7;
    char* myws = p.my;
    char* pws = p.peer(1 + q);
    const rsrc_t rs = mk_rsrc(myws, WS_USED), ro = mk_rsrc(p.out, OUT_BYTES);
    char* tick = myws + OFF_TICK + (8 + q) * 128;
    const unsigned tbase = (call - 1) * (1024 + 2 * PPO);
    bool dead = false;
#define WT_OF(k) ((unsigned)(k) < 1024u ? ((k) & ~63) | (((k) & 28) << 1) | (((k) >> 3) & 4) | ((k) & 3) : (k))
#define ENT_OFF(t) (lane == 0 ? OFF_DONE + (src * 1024 + ((t) & ~1)) * 8 : OFF_MF + (src * 256 + (((t) >> 2) & ~3)) * 4)
#define ENT_FETCH(t) { if (lane < 2) __builtin_amdgcn_raw_ptr_buffer_load_lds(rs, (LAS void*)(L + EX_E), 16, ENT_OFF(t), 0, 0, AUX_SYS); }
#define EX_ISSUE(t, e, x, f, n, xo) { unsigned e_ = (e), x_ = (x), f_ = (f); \
        if ((e_ < call || f_ < call) && !dead) { Spin t0_; \
              \
            for (;;) { CFENCE; const bool had_ = e_ >= call; \
                const u32x4 g_ = __builtin_amdgcn_raw_buffer_load_b128(rs, lane < (had_ ? 2 : 1) ? ENT_OFF(t) : 0xf0000000u, 0, AUX_SYS); \
                e_ = __builtin_amdgcn_readlane(((t) & 1) ? g_[2] : g_[0], 0); x_ = __builtin_amdgcn_readlane(((t) & 1) ? g_[3] : g_[1], 0); \
                f_ = __builtin_amdgcn_readlane(((t) & 8) ? (((t) & 4) ? g_[3] : g_[2]) : (((t) & 4) ? g_[1] : g_[0]), 1); \
                if (!had_ && e_ >= call) continue; \
                if (e_ >= call && f_ >= call) break; if (t0_.fail(myws, call)) { dead = true; break; } __builtin_amdgcn_s_sleep(8); } \
            CFENCE; \
            { const unsigned long long s0_ = __builtin_amdgcn_s_memrealtime(); while ((unsigned)(__builtin_amdgcn_s_memrealtime() - s0_) < (ei / 7 - PPP) * STAG && ei / 7 > PPP) __builtin_amdgcn_s_sleep(1); } } \
        n = min(x_ & 0x3fffu, 8192u); xo = x_; \
        const unsigned by_ = min(((x_ >> 30) ? n : ((x_ >> 14) & 0xfffu) * 4) + 15 & ~15u, 8192u); \
        const rsrc_t rc_ = mk_rsrc(pws + OFF_COD + (unsigned)(t) * 8192, by_); \
        _Pragma("nounroll") for (unsigned r_ = 0; r_ < by_; r_ += 1024) \
            __builtin_amdgcn_raw_ptr_buffer_load_lds(rc_, (LAS void*)(L + EX_B + r_), 16, lane * 16, r_, 0, AUX_SYS); \
        __builtin_amdgcn_raw_ptr_buffer_load_lds(rs, (LAS void*)(L + EX_M), 16, lane * 16, OFF_LM + src * 1048576 + (t) * 1024, 0, AUX_SYS); }
    unsigned r0, n0 = 0, x0 = 0;
    DRAW(r0, tick);
    int t0;
    TAKE(t0, r0, 0); t0 = WT_OF(t0);
    if ((unsigned)t0 < 1024u) EX_ISSUE(t0, 0u, 0u, 0u, n0, x0);
    int t1;
    { unsigned r1; DRAW(r1, tick); TAKE(t1, r1, 0); t1 = WT_OF(t1); }
    STAMP(myws, wv, 18);
    while ((unsigned)t0 < 1024u) {
        if ((unsigned)t1 < 1024u) ENT_FETCH(t1);
        const u32x4 m = *(const LAS u32x4*)(L + EX_M + lane * 16);
        if (x0 >> 30) {
#pragma nounroll
            for (unsigned r = 0; r < n0; r += 1024) *(LAS u32x4*)(L + r + lane * 16) = *(const LAS u32x4*)(L + EX_B + r + lane * 16);
        } else {
            const unsigned G = min((n0 + 15) >> 4, L1_MAX / 16), ec = (x0 >> 26) & 15u;
            const unsigned T1 = (ec << 3) | (((ec - 1) & 15u) << 11) | (((ec - 2) & 15u) << 19) | 0x80000000u;
            const unsigned T2 = (((ec + 1) & 15u) << 3) | (((ec - 3) & 15u) << 11) | (((ec - 4) & 15u) << 19) | 0x80000000u;
            LAS unsigned char* S2 = L + EX_B + 5888;
            unsigned mn[20], ex[20], n8[20], cnt2 = 0;
            const unsigned k01 = KS_(0x01010101u), k02 = KS_(0x02020202u), k03 = KS_(0x03030303u), k07 = KS_(0x07070707u), k80 = KS_(H8), kmul = KS_(0x00408102u);
#define LAYER1(i0, n) { u32x2 nn_[n]; unsigned hb_[n]; \
                _Pragma("unroll") for (int i = 0; i < n; ++i) { const unsigned g = (i0 + i) * 64 + lane; nn_[i] = *(const LAS u32x2*)(L + EX_B + g * 8); hb_[i] = *(const LAS unsigned short*)(L + EX_B + G * 8 + g * 2); } \
                SCHED; \
                _Pragma("unroll") for (int i = 0; i < n; ++i) { const unsigned hb = (i0 + i) * 64 + lane < G ? hb_[i] : 0u;        \
                    _Pragma("unroll") for (int k = 0; k < 4; ++k) { const int z = 4 * (i0 + i) + k; mn[z] = (k & 1) ? nn_[i][k >> 1] >> 4 : nn_[i][k >> 1]; \
                        ex[z] = __builtin_amdgcn_perm(0u, T1, ((mn[z] >> 3) & k01) | (__builtin_amdgcn_ubfe(hb, 4 * k, 4) * kmul & k02)); \
                        n8[z] = __builtin_amdgcn_udot4(ex[z] & k80, 0x08040201u, 0u, false) >> 4; \
                        cnt2 += __builtin_popcount(n8[z]); } } }
            LAYER1(0, 4);
#pragma unroll
            for (int k = 16; k < 20; ++k) mn[k] = ex[k] = n8[k] = 0;
            if (G > 256) LAYER1(4, 1);
            CYC(0);
            const unsigned sc2 = wave_scan(cnt2), n2 = min(__builtin_amdgcn_readlane(sc2, 63), L2_MAX), G2 = (n2 + 15) >> 4, o2 = off_l2(G), o3 = o2 + G2 * 4;
            {
                unsigned pk0 = *(const LAS unsigned*)(L + EX_B + o2 + lane * 4), pk1 = *(const LAS unsigned*)(L + EX_B + o2 + 256 + lane * 4);
                pk0 = lane < G2 ? pk0 : 0u; pk1 = 64 + lane < G2 ? pk1 : 0u;
                const unsigned x3 = *(const LAS unsigned*)(L + EX_B + o3 + lane * 4);
                *(LAS u32x2*)(L + EX_P + lane * 8) = (u32x2){(x3 << 3) & 0x78787878u, (x3 >> 1) & 0x78787878u};
                const unsigned ot0 = pk0 & (pk0 >> 1) & KS_(0x55555555u), ot1 = pk1 & (pk1 >> 1) & KS_(0x55555555u), cnt3 = __builtin_popcount(ot0) + __builtin_popcount(ot1);
                unsigned v2[8], m8[8];
#pragma unroll
                for (int k = 0; k < 8; ++k) {
                    v2[k] = __builtin_amdgcn_perm(0u, T2, ((k < 4 ? pk0 : pk1) >> (2 * (k & 3))) & k03);
                    m8[k] = __builtin_amdgcn_udot4(((k < 4 ? ot0 : ot1) >> (2 * (k & 3))) & k01, 0x40201008u, 0u, false);
                }
                unsigned o = wave_scan(cnt3) - cnt3 + (unsigned)(size_t)(L + EX_P);
                CFENCE;
                merge<8>(v2, m8, o, lds + EX_MRG);
                *(LAS u32x4*)(S2 + lane * 16) = (u32x4){v2[0], v2[1], v2[2], v2[3]};
                *(LAS u32x4*)(S2 + 1024 + lane * 16) = (u32x4){v2[4], v2[5], v2[6], v2[7]};
                CFENCE;
            }
            unsigned o = sc2 - cnt2 + (unsigned)(size_t)S2;
#define CODES_OUT(i) *(LAS u32x4*)(L + ((i) * 64 + lane) * 16) = (u32x4){(mn[4 * (i)] & k07) | ex[4 * (i)], (mn[4 * (i) + 1] & k07) | ex[4 * (i) + 1], (mn[4 * (i) + 2] & k07) | ex[4 * (i) + 2], (mn[4 * (i) + 3] & k07) | ex[4 * (i) + 3]}
            merge<16>(ex, n8, o, lds + EX_MRG);
            CODES_OUT(0); CODES_OUT(1); CODES_OUT(2); CODES_OUT(3);
            if (G > 256) { merge<4>(ex + 16, n8 + 16, o, lds + EX_MRG); CODES_OUT(4); }
            CFENCE;
        }
        const unsigned cnt = popc4(m);
        unsigned off = wave_scan(cnt) - cnt + (unsigned)(size_t)L;
        WAIT_L(0);
        CYC(1);
        unsigned n1 = 0, x1 = 0;
        if ((unsigned)t1 < 1024u) {
            WAIT_V(0);
            const u32x2 ent = *(const LAS u32x2*)(L + EX_E + (t1 & 1) * 8);
            const unsigned mf = *(const LAS unsigned*)(L + EX_E + 16 + (t1 & 12));
            EX_ISSUE(t1, __builtin_amdgcn_readfirstlane(ent[0]), __builtin_amdgcn_readfirstlane(ent[1]), __builtin_amdgcn_readfirstlane(mf), n1, x1);
        }
        unsigned r2;
        DRAW(r2, tick);
        CYC(2);
        const unsigned so = out_soff(src, t0), vo = out_voff(lane);
        u32x2 sel[16]; unsigned of[16], w0[16], w1[16], w2[16];
#pragma unroll
        for (int g = 0; g < 16; ++g) {
            const unsigned b8 = ((g & 3) == 0 ? m[g >> 2] << 3 : m[g >> 2] >> (8 * (g & 3) - 3)) & 0x7f8u;
            sel[g] = *(const LAS u32x2*)(lds + EX_LUT + b8);
            const LAS unsigned* w = (const LAS unsigned*)(size_t)(off & ~3u);
            w0[g] = w[0]; w1[g] = w[1]; w2[g] = w[2]; of[g] = off;
            off += __builtin_popcount(b8);
        }
        SCHED;
        u32x4 o[8];
#pragma unroll
        for (int g = 0; g < 16; ++g) {
            const unsigned a0 = __builtin_amdgcn_alignbyte(w1[g], w0[g], of[g]), a1 = __builtin_amdgcn_alignbyte(w2[g], w1[g], of[g]);
            o[g >> 1][(g & 1) * 2] = __builtin_amdgcn_perm(a1, a0, sel[g][0]); o[g >> 1][(g & 1) * 2 + 1] = __builtin_amdgcn_perm(a1, a0, sel[g][1]);
        }
        CYC(3);
        if (dead) {
#pragma unroll
            for (int ch = 0; ch < 8; ++ch) o[ch] = (u32x4){0x7f7f7f7fu, 0x7f7f7f7fu, 0x7f7f7f7fu, 0x7f7f7f7fu};
            CFENCE;
        }
#pragma unroll
        for (int ch = 0; ch < 8; ++ch) ST128_RAW(o[ch], ro, vo, so + chunk_soff(ch), AUX_WT);
        ST128_PAD8(o);
        CYC(4);
        int t2;
        TAKE(t2, r2, 8); t2 = WT_OF(t2);
        CYC(5);
        t0 = t1; n0 = n1; x0 = x1; t1 = t2;
    }
    STAMP(myws, wv, 19);
}

__device__ __forceinline__ unsigned get_call(unsigned callv) { unsigned c; asm volatile("s_nop 0\n\tv_readfirstlane_b32 %0, %1" : "=s"(c) : "v"(callv)); return c + 1u; }

__device__ __forceinline__ void own_x(const Params& p, int j, int g, unsigned due, unsigned long long t0, int lane) {
    while ((unsigned)(__builtin_amdgcn_s_memrealtime() - t0) < due) __builtin_amdgcn_s_sleep(4);
    u32x4 v[8];
    x_load(v, mk_rsrc(p.x, X_BYTES), j, g, lane);
    img_store(v, mk_rsrc(p.my, WS_USED), false, x_img(p.rank * 8 + j, g), lane);
    WAIT_V(0);
    if (lane == 0) add_sys(p.my + x_flag(p.rank * 8 + j, g));
}

__device__ __forceinline__ void comm_role(const Params& p, LAS unsigned char* lds, int c, int wid, int lane, unsigned callv) {
    const int rank = p.rank;
    char* myws = p.my;
    const rsrc_t rs = mk_rsrc(myws, WS_USED);
    const int wv = (NGEMM + c) * 8 + wid; (void)wv;
    STAMP(myws, wv, 0);
    if (wid == 0 && lane < 4) *(volatile LAS unsigned*)(lds + EX_TOK + lane * 4) = 0u;
    if (wid == 0 && lane < 16) *(LAS unsigned*)(lds + EX_MRG + lane * 8) = lut_merge(lane);
    if (wid == 2 || wid == 3) { *(LAS u32x2*)(lds + EX_LUT + ((wid & 1) * 128 + lane) * 8) = lut_expand((wid & 1) * 128 + lane); *(LAS u32x2*)(lds + EX_LUT + ((wid & 1) * 128 + 64 + lane) * 8) = lut_expand((wid & 1) * 128 + 64 + lane); }
    WAIT_L(0); BAR; CFENCE;
    if (wid >= 4) {
        const rsrc_t rx = mk_rsrc(p.x, X_BYTES), rw = mk_rsrc(p.w, W_BYTES);
        const bool far = wid < 6;
        const unsigned long long tk0 = __builtin_amdgcn_s_memrealtime();
        for (int u = (wid & 1) * 128 + c + (far ? 7 * NPUSH : 0); u < (far ? 1792 : 704); u += 256) {
            const bool ownw = !far && u >= 64 && u < 256;
            const int ux = u < 64 ? u : u - 192;
            const int j = far ? u / 224 : ownw ? u >> 5 : ux >> 6, r = far ? u % 224 : ownw ? 288 + (u & 31) : 224 + (ux & 63), t = rank * 8 + j;
            const unsigned due = far ? (j > 1 ? (j - 1) * PACE_W : j * PACE_W1) : ownw ? (j - 2) * PACE_E : j * PACE_X;
            while ((unsigned)(__builtin_amdgcn_s_memrealtime() - tk0) < due) __builtin_amdgcn_s_sleep(8);
            u32x4 v[8];
            if (r >= 224 && r < 288) {
                const int g = r - 224;
                x_load(v, rx, j, g, lane);
                pack_store(v, rs, lds + EX_0 + wid * EX_BYTES, OFF_PX + (unsigned)(j * 64 + g) * UNIT, lane);
                WAIT_V(0);
                const unsigned call = get_call(callv);
                if (j < NPUF) {
                    if (lane < 8) st_sys(p.pl + OFF_PUF + ((rank * NPUF + j) * 96 + 32 + g) * 4, call);
                    if (j == 0) STAMP(myws, wv, 7);
                }
                unsigned old = 0;
                if (lane == 0) old = __hip_atomic_fetch_add((GAS unsigned*)(myws + OFF_PCNT + j * 128), 1u, __ATOMIC_RELAXED, __HIP_MEMORY_SCOPE_AGENT);
                if ((__builtin_amdgcn_readfirstlane(old) & 63) == 63 && lane < 8) st_sys(p.pl + OFF_PFLG + t * 128, call);
            } else {
                const bool own = r >= 288;
                const int e = own ? r - 288 : r / 7, d = own ? rank : (rank + 1 + r % 7) & 7;
                u32x2 w[16];
                w_load(w, rw, j, d, e >> 2, e & 1, (e >> 1) & 1, lane);
                w_transpose(v, w);
                if (own) {
                    img_store(v, rs, true, w_img(t, e), lane);
                    WAIT_V(0);
                    if (lane == 0) add_sys(myws + w_flag(t, e));
                } else {
                    const unsigned id = (unsigned)((d * 8 + j) * 32 + e);
                    pack_store(v, rs, lds + EX_0 + wid * EX_BYTES, OFF_PW + id * UNIT, lane);
                    WAIT_V(0);
                    const unsigned call = get_call(callv);
                    if (j < NPUF) { if (lane == 1 + r % 7) st_sys(p.pl + OFF_PUF + ((rank * NPUF + j) * 96 + e) * 4, call); if (j == 0) STAMP(myws, wv, 6); }
                    unsigned old = 0;
                    if (lane == 0) old = __hip_atomic_fetch_add((GAS unsigned*)(myws + OFF_PCNT + (8 + d * 8 + j) * 128), 1u, __ATOMIC_RELAXED, __HIP_MEMORY_SCOPE_AGENT);
                    if (__builtin_amdgcn_readfirstlane(old) + 1 == call * (j ? 32 : 32 - NPUSH) && lane == 1 + r % 7) st_sys(p.pl + OFF_PFLG + t * 128 + 4, call);
                }
            }
            STAMP(myws, wv, far ? 1 + (j >> 2) : 3 + (j >> 2));
        }
    }
    if (wid == 2 || wid == 3) {
        const unsigned long long tk0 = __builtin_amdgcn_s_memrealtime();
        for (int u = (wid & 1) * 128 + c; u < 384; u += 256) own_x(p, 2 + u % 96 / 16, u / 96 * 16 + u % 16, u / 96 * PACE_R + u % 96 / 16 * PACE_E, tk0, lane);
    }
    if (wid == 1) {
        if (NPUSH && c < 7 * PUSH_W) {
            const int d = (rank + 1 + c % 7) & 7, e0 = c / 7;
            const rsrc_t rw = mk_rsrc(p.w, W_BYTES), rd = mk_rsrc(p.peer(1 + c % 7), WS_USED);
            u32x2 w[PUSH_U ? PUSH_U : 1][16];
#pragma unroll
            for (int i = 0; i < PUSH_U; ++i) { const int e = e0 + i * PUSH_W; w_load(w[i], rw, 0, d, e >> 2, e & 1, (e >> 1) & 1, lane); }
#pragma unroll
            for (int i = 0; i < PUSH_U; ++i) {
                const int e = e0 + i * PUSH_W;
                u32x4 v[8];
                w_transpose(v, w[i]);
                img_store(v, rd, true, w_img(rank * 8, e), lane);
                WAIT_V(0);
                if (lane == 1 + c % 7) add_sys(p.pl + w_flag(rank * 8, e));
            }
            STAMP(myws, wv, 14);
        }
    }
    if (wid < CW) {
        const unsigned call = get_call(callv);
        const int ci = wid * 128 + c;
        if (wid < TEAM && c < 7 * PPP) pull_role(p, lds, lds + EX_0 + wid * EX_BYTES, c, lane, call, wv);
        if (ci < 7 * PPO) expand_role(p, lds, lds + EX_0 + wid * EX_BYTES, ci, lane, call, wv);
    }
}

typedef float f32x2 __attribute__((ext_vector_type(2)));
__device__ __forceinline__ f32x4 scale4(const f32x4& v, const f32x2& s) {
    f32x2 a, b;
    asm("v_pk_mul_f32 %0, %1, %2" : "=v"(a) : "v"((f32x2){v[0], v[1]}), "v"(s));
    asm("v_pk_mul_f32 %0, %1, %2" : "=v"(b) : "v"((f32x2){v[2], v[3]}), "v"(s));
    return (f32x4){a[0], a[1], b[0], b[1]};
}
__device__ __forceinline__ void gemm_role(const Params& p, LAS unsigned char* lds, int rt, int ct, int wid, int lane, unsigned callv) {
    const int rank = p.rank;
    char* myws = p.my;
    const int tile = rt * 8 + ct;
    STAMP(myws, tile * 8 + wid, 0);
    const int wr = wid >> 2, wc = wid & 3;
    LAS unsigned char* dst = lds + lane * 16 + (wr == 0 ? 2 * HALFB + wid * 8192 : 5 * HALFB + (wc >> 1) * 8192 + (wc & 1) * 1024);
#define DIR_PUT(v) { _Pragma("unroll") for (int f = 0; f < 8; ++f) *(LAS u32x4*)(dst + (wr == 0 ? f * 1024 : ((f >> 1) & 1) * 3 * HALFB + (f >> 2) * 4096 + (f & 1) * 2048)) = v[f]; }
    u32x4 dv[8];
    if (rt >= 8) { const unsigned long long t0_ = __builtin_amdgcn_s_memrealtime(); while ((unsigned)(__builtin_amdgcn_s_memrealtime() - t0_) < 150) __builtin_amdgcn_s_sleep(4); }
    {
        u32x4 v0[8];
        if (wr == 0) {
            x_load(v0, mk_rsrc(p.x, X_BYTES), 0, rt * 4 + wid, lane); x_load(dv, mk_rsrc(p.x, X_BYTES), 1, rt * 4 + wid, lane);
        } else {
            u32x2 w[2][16];
            w_load<0>(w[0], mk_rsrc(p.w, W_BYTES), 0, rank, ct, wc & 1, wc >> 1, lane); w_load<0>(w[1], mk_rsrc(p.w, W_BYTES), 1, rank, ct, wc & 1, wc >> 1, lane);
            w_transpose(v0, w[0]);
#pragma unroll
            for (int i = 0; i < 8; ++i) dv[i] = (u32x4){w[1][2 * i][0], w[1][2 * i][1], w[1][2 * i + 1][0], w[1][2 * i + 1][1]};
        }
        DIR_PUT(v0);
    }
    const unsigned call = get_call(callv);
    const int wv = tile * 8 + wid; (void)wv;
    const rsrc_t rs = mk_rsrc(myws, WS_USED);
    const unsigned voffA = (wc >> 1) * HALFB + wr * 8192 + (wc & 1) * 4096, voffB = wr * HALFB + wc * 4096, vl = lane * 16;
    unsigned lda = (unsigned)(size_t)lds + wr * 8192 + lane * 16, ldb = (unsigned)(size_t)lds + 4 * HALFB + wc * 4096 + lane * 16;
    asm volatile("" : "+v"(lda), "+v"(ldb));
    const unsigned seqf = (unsigned)step_of(lane, rank, rt) * 4, fx = OFF_FLAG + rt * 256, fw = OFF_FLAG + FLAG_W + ct * 256, c4 = call * 4;
    unsigned long long todo = ~0ull, rdy = 0;
    unsigned tl = 0;
    volatile LAS unsigned* tq = (volatile LAS unsigned*)(lds + LDS_RED + 160);
#define STAGE4(at, soff) do { \
        __builtin_amdgcn_raw_ptr_buffer_load_lds(rs, (LAS void*)(lds + (at)), 16, vl, (soff), 0, AUX_SYS); \
        __builtin_amdgcn_raw_ptr_buffer_load_lds(rs, (LAS void*)(lds + (at)), 16, vl, (soff), 1024, AUX_SYS); \
        __builtin_amdgcn_raw_ptr_buffer_load_lds(rs, (LAS void*)(lds + (at)), 16, vl, (soff), 2048, AUX_SYS); \
        __builtin_amdgcn_raw_ptr_buffer_load_lds(rs, (LAS void*)(lds + (at)), 16, vl, (soff), 3072, AUX_SYS); } while (0)
#define LDA(dst, b, h) do { _Pragma("unroll") for (int m = 0; m < 4; ++m) _Pragma("unroll") for (int k = 0; k < 2; ++k) dst[m][k] = *(const LAS bf16x8*)(size_t)(lda + ((b) * 2 + (h)) * HALFB + m * 2048 + k * 1024); } while (0)
#define LDB(dst, at) do { _Pragma("unroll") for (int n = 0; n < 2; ++n) _Pragma("unroll") for (int k = 0; k < 2; ++k) dst[n][k] = *(const LAS bf16x8*)(size_t)((at) + n * 2048 + k * 1024); } while (0)
#define MMA(ai, bj, At, Bt) do { __builtin_amdgcn_s_setprio(1); _Pragma("unroll") for (int m = 0; m < 4; ++m) _Pragma("unroll") for (int n = 0; n < 2; ++n) _Pragma("unroll") for (int k = 0; k < 2; ++k) \
        acc[ai][bj][m][n] = __builtin_amdgcn_mfma_f32_16x16x32_bf16(Bt[n][k], At[m][k], acc[ai][bj][m][n], 0, 0, 0); __builtin_amdgcn_s_setprio(0); } while (0)
#define OFFA(t) (OFF_XT + (unsigned)(t) * XT_STEP + (unsigned)rt * BLK)
#define OFFB(t) (OFF_WT + (unsigned)(t) * WT_STEP + (unsigned)ct * BLK)
    unsigned la = 0, lb = 0;
#define LOOK do { if (wid == 0) { la = *(const LAS unsigned*)(lds + LDS_FLAG + lane * 4); lb = *(const LAS unsigned*)(lds + LDS_FLAG + 256 + lane * 4); } } while (0)
#define PICK(slot) do { if (wid == 0) { \
            if (todo != 0) { \
                rdy |= __builtin_amdgcn_ballot_w64(la >= c4 && lb >= c4); \
                if ((rdy & todo) == 0) { Spin t0_; \
                    for (;;) { CFENCE; \
                        const unsigned a_ = __builtin_amdgcn_raw_buffer_load_b32(rs, seqf, fx, AUX_SYS), b_ = __builtin_amdgcn_raw_buffer_load_b32(rs, seqf, fw, AUX_SYS); \
                        rdy |= __builtin_amdgcn_ballot_w64(a_ >= c4 && b_ >= c4); \
                        if ((rdy & todo) != 0) break; if (t0_.fail(myws, call)) { rdy = ~0ull; break; } } } \
                const int pos_ = __builtin_ctzll(rdy & todo); \
                todo &= ~(1ull << pos_); tl = __builtin_amdgcn_readlane(seqf, pos_) >> 2; \
            } \
            tq[slot] = tl; \
            const unsigned skip_ = (todo & ~rdy) == 0 ? 0xf0000000u : 0u;        \
            __builtin_amdgcn_raw_ptr_buffer_load_lds(rs, (LAS void*)(lds + LDS_FLAG), 4, seqf, fx + skip_, 0, AUX_SYS); \
            __builtin_amdgcn_raw_ptr_buffer_load_lds(rs, (LAS void*)(lds + LDS_FLAG + 256), 4, seqf, fw + skip_, 0, AUX_SYS); \
            SCHED; } } while (0)

    f32x4 acc[2][2][4][2];
#pragma unroll
    for (int a = 0; a < 2; ++a)
#pragma unroll
        for (int b = 0; b < 2; ++b)
#pragma unroll
            for (int m = 0; m < 4; ++m)
#pragma unroll
                for (int n = 0; n < 2; ++n) acc[a][b][m][n] = (f32x4){0.f, 0.f, 0.f, 0.f};
    bf16x8 A0[4][2], A1[4][2], B0[2][2], B1[2][2];

#define STEP(b) { \
        const unsigned tw = tq[b];        \
        LOOK; \
        LDB(B0, ldb + (b) * HALFB); LDA(A0, b, 0); LDB(B1, ldb + h1 * HALFB); LDA(A1, b, 1); \
        SCHED; \
        PICK(b); \
        STAGE4(((b) ^ 1) * 2 * HALFB + voffA, OFFA(t1) + voffA); \
        SCHED; \
        const unsigned hn = h1 == 4 ? 2 : h1 + 1; \
        { const unsigned lb_ = (4 + (wr ? (hn == 4 ? 2 : hn + 1) : (b) ^ 1)) * HALFB, gb_ = OFFB(wr ? __builtin_amdgcn_readfirstlane(tw) : t1) + wr * HALFB; \
          STAGE4(lb_ + wc * 4096, gb_ + wc * 4096); } \
        h1 = hn; \
        WAIT_L(0); BAR; \
        const unsigned tv = tq[b]; \
        MMA(0, 0, A0, B0); MMA(0, 1, A0, B1); MMA(1, 0, A1, B0); MMA(1, 1, A1, B1); \
        WAIT_V(0); BAR; SCHED; \
        t1 = __builtin_amdgcn_readfirstlane(tv); }
    unsigned t1, h1 = 2;
    {
#define PLAIN_STEP { LDB(B0, ldb + HALFB); LDA(A0, 1, 0); LDB(B1, ldb + 4 * HALFB); MMA(0, 0, A0, B0); MMA(0, 1, A0, B1); LDA(A0, 1, 1); MMA(1, 0, A0, B0); MMA(1, 1, A0, B1); }
        todo = ~3ull;
        if (wid == 1 && lane < 8) ((LAS unsigned*)(lds + LDS_RED))[32 + lane] = 0u;
        WAIT_L(0); BAR; CFENCE;
        if (wid == 0) {
            __builtin_amdgcn_raw_ptr_buffer_load_lds(rs, (LAS void*)(lds + LDS_FLAG), 4, seqf, fx, 0, AUX_SYS);
            __builtin_amdgcn_raw_ptr_buffer_load_lds(rs, (LAS void*)(lds + LDS_FLAG + 256), 4, seqf, fw, 0, AUX_SYS);
        }
        PLAIN_STEP;
        SCHED;
        if (wr == 1) {
            u32x2 w[16];
#pragma unroll
            for (int i = 0; i < 8; ++i) { w[2 * i] = (u32x2){dv[i][0], dv[i][1]}; w[2 * i + 1] = (u32x2){dv[i][2], dv[i][3]}; }
            w_transpose(dv, w);
        }
        WAIT_V(0); LOOK;
        WAIT_L(0); BAR; CFENCE;
        DIR_PUT(dv);
        PICK(2); PICK(1);
        WAIT_L(0); BAR; CFENCE;
        STAMP(myws, wv, 1);
        const unsigned t0 = __builtin_amdgcn_readfirstlane(tq[2]);
        t1 = __builtin_amdgcn_readfirstlane(tq[1]);
        STAGE4(voffA, OFFA(t0) + voffA);
        STAGE4((4 + 2 * wr) * HALFB + wc * 4096, OFFB(t0) + voffB);
        if (wr == 1) STAGE4(7 * HALFB + wc * 4096, OFFB(t1) + voffB);
        SCHED;
        PLAIN_STEP;
        WAIT_V(0); WAIT_L(0); BAR; CFENCE;
        STAMP(myws, wv, 14);
        if (wr == 1) BAR;
    }
    for (int i = 2; i < NT; i += 2) {
        STEP(0);
        STEP(1);
    }
    char* pl = p.pl;
    if (wr == 0) BAR;
    STAMP(myws, wv, 10);

    const int wt = tile * 8 + wid;
    LAS unsigned* red = (LAS unsigned*)(lds + LDS_RED);
    u32x4 msk;
#pragma unroll
    for (int w = 0; w < 4; ++w) {
        unsigned mm = 0;
#pragma unroll
        for (int e = 31; e >= 0; --e) {
            const int ch = 2 * w + (e >> 4);
            mm = __builtin_amdgcn_alignbit(mm, __float_as_uint(acc[ch >> 2][(e >> 1) & 1][ch & 3][e & 1][(e >> 2) & 3]), 31);
        }
        msk[w] = ~mm;
    }
    ST128(msk, rs, lane * 16, OFF_MSK + wt * 1024, AUX_SYS);
    SCHED;
    float mx = 0.f;
#pragma unroll
    for (int a = 0; a < 2; ++a)
#pragma unroll
        for (int b = 0; b < 2; ++b)
#pragma unroll
            for (int m = 0; m < 4; ++m)
#pragma unroll
                for (int n = 0; n < 2; ++n) mx = fmaxf(fmaxf(fmaxf(fmaxf(mx, acc[a][b][m][n][0]), acc[a][b][m][n][1]), acc[a][b][m][n][2]), acc[a][b][m][n][3]);
    const unsigned wm = wave_max(__float_as_uint(mx));
    {
        unsigned old = 0;
        if (lane == 0) {
            __hip_atomic_fetch_max(red + 32, wm, __ATOMIC_RELAXED, __HIP_MEMORY_SCOPE_WORKGROUP);
            old = __hip_atomic_fetch_add(red + 33, 1u, __ATOMIC_RELAXED, __HIP_MEMORY_SCOPE_WORKGROUP);
        }
        if (__builtin_amdgcn_readfirstlane(old) == 7) {
            const unsigned long long gv = ((unsigned long long)call << 32) | __hip_atomic_load(red + 32, __ATOMIC_RELAXED, __HIP_MEMORY_SCOPE_WORKGROUP);
            char* g = pl + OFF_AMAX + (rank * NGEMM + tile) * 8;
            unsigned long long ex_;
            asm volatile("s_mov_b64 %0, exec\n\ts_mov_b64 exec, 0xff\n\tglobal_store_dwordx2 %1, %2, off sc0 sc1\n\ts_mov_b64 exec, %0\n\ts_waitcnt vmcnt(1)" : "=&s"(ex_) : "v"(g), "v"(gv) : "memory");
        } else WAIT_V(0);
        unsigned old2 = 0;
        if (lane == 0) old2 = __hip_atomic_fetch_add(red + 34 + wr, 1u, __ATOMIC_RELAXED, __HIP_MEMORY_SCOPE_WORKGROUP);
        if (__builtin_amdgcn_readfirstlane(old2) == 3 && lane < 8) st_sys(pl + OFF_MFLG + (rank * 256 + tile * 2 + wr) * 4, call);
    }
#pragma unroll
    for (int a = 0; a < 2; ++a)
#pragma unroll
        for (int b = 0; b < 2; ++b)
#pragma unroll
            for (int m = 0; m < 4; ++m)
#pragma unroll
                for (int n = 0; n < 2; ++n)
#pragma unroll
                    for (int j = 0; j < 4; ++j) { acc[a][b][m][n][j] = fmaxf(acc[a][b][m][n][j], 0.f); if (j == 3) asm volatile("" : "+v"(acc[a][b][m][n])); }
    STAMP(myws, wv, 13);
    BAR; CFENCE;
    LAS unsigned char* A = lds + wid * EP_BYTES;
    LAS unsigned char* B = A + EP_C;
    LAS unsigned char* S2 = A + EP_S;
    const unsigned cnt = popc4(msk), scan = wave_scan(cnt), Lc = __builtin_amdgcn_readlane(scan, 63);
    const bool dense = Lc > L1_MAX;
    if (!dense) {
        unsigned o = (unsigned)(size_t)A + (scan - cnt) * 4;
        const unsigned spare = (unsigned)(size_t)(A + EP_X) + lane * 4;
#pragma unroll
        for (int e = 0; e < 128; ++e) {
            const int ch = e >> 4;
            const unsigned bit = __builtin_amdgcn_ubfe(msk[e >> 5], e & 31, 1);
            *(LAS float*)(size_t)(bit ? o ^ ((o >> 4) & 0x30u) : spare) = acc[ch >> 2][(e >> 1) & 1][ch & 3][e & 1][(e >> 2) & 3];
            o += bit * 4;
        }
    }
    float ysum = 0.f;
#pragma unroll
    for (int a = 0; a < 2; ++a)
#pragma unroll
        for (int b = 0; b < 2; ++b)
#pragma unroll
            for (int m = 0; m < 4; ++m) { const f32x4 t = acc[a][b][m][0] + acc[a][b][m][1]; ysum += (t[0] + t[1]) + (t[2] + t[3]); }
#pragma unroll
    for (int sh = 32; sh >= 1; sh >>= 1) ysum += __shfl_xor(ysum, sh);
    ysum *= 1.35f / (float)max(Lc, 1u);
    if (!dense && lane < 16 && Lc + lane < L1_MAX) *(LAS float*)(A + (((Lc + lane) * 4) ^ (((Lc + lane) >> 2) & 0x30u))) = ysum;
    LAS unsigned char* lut = A + EP_T;
    if (lane < 16) *(LAS unsigned*)(lut + lane * 4) = lut_compact(lane);
    {
        if (wid == 0) {
            Spin t0;
            for (;;) {
                CFENCE;
                const unsigned g = __builtin_amdgcn_raw_buffer_load_b32(rs, (lane & 31) * 8 + 4, OFF_AMAX + (rank * NGEMM + 96) * 8, AUX_SYS);
                if (__builtin_popcountll(__builtin_amdgcn_ballot_w64(g >= call)) >= 48 || t0.fail(myws, call)) break;
                __builtin_amdgcn_s_sleep(8);
            }
        }
        BAR; CFENCE;
    }
    {
        Spin t0;
        unsigned t, tm = ld_agent(myws + OFF_TMO);
        for (;;) {
            CFENCE;
            const u32x4 g = __builtin_amdgcn_raw_buffer_load_b128(rs, lane * 16, OFF_AMAX + wid * 1024, AUX_SYS);
            t = max(g[0], g[2]);
            if (__builtin_amdgcn_ballot_w64(g[1] < call || g[3] < call) == 0) break;
            if (t0.fail(myws, call)) { tm = call; break; }
            __builtin_amdgcn_s_sleep(4);
        }
        t = wave_max(t);
        if (lane == 0) { red[8 + wid] = t; red[wid] = tm == call; }
    }
    WAIT_L(0); BAR; CFENCE;
    const float amax = __uint_as_float(wave_max(red[8 + (lane & 7)]));
    const bool dead = __builtin_amdgcn_ballot_w64(red[lane & 7] != 0) != 0;
    const float inv = dead ? __builtin_nanf("") : 448.0f / amax;
    STAMP(myws, wv, 11);
    if (wr == 0) __builtin_amdgcn_s_setprio(2);

    const f32x2 inv2 = {inv, inv};
#define QUANT(ch, q) { const int a = (ch) >> 2, m = (ch) & 3; \
        const f32x4 y0 = scale4(acc[a][0][m][0], inv2), y1 = scale4(acc[a][0][m][1], inv2), y2 = scale4(acc[a][1][m][0], inv2), y3 = scale4(acc[a][1][m][1], inv2); \
        _Pragma("unroll") for (int j = 0; j < 4; ++j) \
            q[j] = (unsigned)__builtin_amdgcn_cvt_pk_fp8_f32(y2[j], y3[j], __builtin_amdgcn_cvt_pk_fp8_f32(y0[j], y1[j], __float_as_int(y0[j]), false), true); }
    u32x4 cod[5];
    if (dense) {
#pragma unroll
        for (int r = 0; r < 9; ++r) *(LAS u32x4*)(A + r * 1024 + lane * 16) = (u32x4){0u, 0u, 0u, 0u};
        unsigned o = (unsigned)(size_t)A + scan - cnt;
#pragma unroll
        for (int ch = 0; ch < 8; ++ch) {
            u32x4 q;
            QUANT(ch, q);
#pragma unroll
            for (int j = 0; j < 4; ++j) {
                const unsigned n4 = nib4(msk[ch >> 1], (ch & 1) * 4 + j);
                const unsigned long long z = (unsigned long long)__builtin_amdgcn_perm(0u, q[j], *(const LAS unsigned*)(lut + n4)) << ((o & 3u) * 8);
                LAS unsigned* d = (LAS unsigned*)(size_t)(o & ~3u);
                __hip_atomic_fetch_or(d, (unsigned)z, __ATOMIC_RELAXED, __HIP_MEMORY_SCOPE_WAVEFRONT);
                __hip_atomic_fetch_or(d + 1, (unsigned)(z >> 32), __ATOMIC_RELAXED, __HIP_MEMORY_SCOPE_WAVEFRONT);
                o += __builtin_popcount(n4);
            }
        }
    } else {
#define CODES(i0, n) { f32x4 y_[n][4]; \
            _Pragma("unroll") for (int i = 0; i < n; ++i) _Pragma("unroll") for (int k = 0; k < 4; ++k) y_[i][k] = *(const LAS f32x4*)(A + ((i0 + i) * 64 + (i0 + i < 4 ? lane : lane & 31)) * 64 + ((k ^ (lane >> 2)) & 3) * 16); \
            SCHED; \
            _Pragma("unroll") for (int i = 0; i < n; ++i) _Pragma("unroll") for (int k = 0; k < 4; ++k) { const f32x4 y = scale4(y_[i][k], inv2); \
                cod[i0 + i][k] = (unsigned)__builtin_amdgcn_cvt_pk_fp8_f32(y[2], y[3], __builtin_amdgcn_cvt_pk_fp8_f32(y[0], y[1], __float_as_int(y[0]), false), true); } }
        CODES(0, 2); CODES(2, 2);
        cod[4] = (u32x4){0u, 0u, 0u, 0u};
        if (Lc > 4096) CODES(4, 1);
        CFENCE;
#pragma unroll
        for (int i = 0; i < 5; ++i) *(LAS u32x4*)(A + (i * 64 + lane) * 16) = cod[i];
#pragma unroll
        for (int r = 0; r < 3; ++r) *(LAS u32x4*)(S2 + r * 1024 + lane * 16) = (u32x4){0u, 0u, 0u, 0u};
    }
    CFENCE;
    CYC(0);
    const unsigned ec = (__builtin_amdgcn_readfirstlane(__builtin_amdgcn_cvt_pk_fp8_f32(fminf(ysum * inv, 448.f), 0.f, 0, false)) >> 3) & 15u;
    const unsigned E1 = (ec + 17) * 0x01010101u, G = (Lc + 15) >> 4;
    bool raw = dense || (rt * 2 + wr) * 32 + ct * 4 + wc >= 1024 - RAW_LAST;
    unsigned bytes = Lc;
    if (!raw) {
        unsigned s2[5][4], n7[5][4], cnt2 = 0;
        const unsigned k07 = KS_(0x07070707u), k08 = KS_(0x08080808u), k0f = KS_(0x0f0f0f0fu), k10 = KS_(0x10101010u), k80 = KS_(H8);
#define CLASSIFY(i0, n) { const u32x4* d_ = cod + i0; \
            _Pragma("unroll") for (int i = 0; i < n; ++i) { const unsigned g = (i0 + i) * 64 + lane, wgt = g < G ? 0x08040201u : 0u; unsigned nb[4], p_[4]; \
                _Pragma("unroll") for (int k = 0; k < 4; ++k) { \
                    const unsigned e4 = (d_[i][k] >> 3) & k0f, u = E1 - e4, t = u & k08, ui = (u | (t - (t >> 3))) & k07; \
                    p_[k] = __builtin_amdgcn_perm(0x98989898u, 0x10080098u, ui); \
                    s2[i0 + i][k] = e4 | __builtin_amdgcn_perm(0x30302010u, 0x30303000u, ui); \
                    nb[k] = (d_[i][k] & k07) | (p_[k] & k08); \
                    n7[i0 + i][k] = __builtin_amdgcn_udot4(p_[k] & k80, wgt, 0u, false); \
                    cnt2 += __builtin_popcount(n7[i0 + i][k]); } \
                if (g < G) { *(LAS u32x2*)(B + g * 8) = (u32x2){nb[0] | (nb[1] << 4), nb[2] | (nb[3] << 4)}; \
                    *(LAS unsigned short*)(B + G * 8 + g * 2) = (unsigned short)bits16(p_[0] & k10, p_[1] & k10, p_[2] & k10, p_[3] & k10, 4); } } }
        CLASSIFY(0, 4);
#pragma unroll
        for (int k = 0; k < 4; ++k) s2[4][k] = n7[4][k] = 0;
        if (G > 256) CLASSIFY(4, 1);
        CYC(1);
        const unsigned sc2 = wave_scan(cnt2), n2 = __builtin_amdgcn_readlane(sc2, 63), G2 = (n2 + 15) >> 4, o2 = off_l2(G), o3 = o2 + G2 * 4;
        raw = n2 > L2_MAX;
        if (!raw) {
            unsigned o = sc2 - cnt2 + (unsigned)(size_t)S2;
#define SQUEEZE(i0, n) { unsigned se_[n][4]; \
                _Pragma("unroll") for (int i = 0; i < n; ++i) _Pragma("unroll") for (int k = 0; k < 4; ++k) se_[i][k] = *(const LAS unsigned*)(lut + (n7[i0 + i][k] >> 5)); \
                SCHED; \
                _Pragma("unroll") for (int i = 0; i < n; ++i) _Pragma("unroll") for (int k = 0; k < 4; ++k) { \
                    const unsigned long long z = (unsigned long long)__builtin_amdgcn_perm(0u, s2[i0 + i][k], se_[i][k]) << ((o & 3u) * 8); \
                    LAS unsigned* d = (LAS unsigned*)(size_t)(o & ~3u); \
                    __hip_atomic_fetch_or(d, (unsigned)z, __ATOMIC_RELAXED, __HIP_MEMORY_SCOPE_WAVEFRONT); \
                    __hip_atomic_fetch_or(d + 1, (unsigned)(z >> 32), __ATOMIC_RELAXED, __HIP_MEMORY_SCOPE_WAVEFRONT); \
                    o += __builtin_popcount(n7[i0 + i][k]); } }
            SQUEEZE(0, 4);
            if (G > 256) SQUEEZE(4, 1);
            CFENCE;
            CYC(2);
            const u32x4 d0 = *(const LAS u32x4*)(S2 + lane * 16), d1 = *(const LAS u32x4*)(S2 + 1024 + lane * 16);
            if (lane < G2) *(LAS unsigned*)(B + o2 + lane * 4) = ((d0[0] >> 4) & 0x03030303u) | ((d0[1] >> 2) & 0x0c0c0c0cu) | (d0[2] & 0x30303030u) | ((d0[3] << 2) & 0xc0c0c0c0u);
            if (64 + lane < G2) *(LAS unsigned*)(B + o2 + 256 + lane * 4) = ((d1[0] >> 4) & 0x03030303u) | ((d1[1] >> 2) & 0x0c0c0c0cu) | (d1[2] & 0x30303030u) | ((d1[3] << 2) & 0xc0c0c0c0u);
            unsigned m5[8], se[8], cnt3 = 0;
#pragma unroll
            for (int k = 0; k < 8; ++k) {
                const unsigned d = k < 4 ? d0[k & 3] : d1[k & 3];
                m5[k] = __builtin_amdgcn_udot4(d & (d << 1) & 0x20202020u, 0x08040201u, 0u, false);
                se[k] = *(const LAS unsigned*)(lut + (m5[k] >> 3));
                cnt3 += __builtin_popcount(m5[k]);
            }
            const unsigned sc3 = wave_scan(cnt3), n3 = __builtin_amdgcn_readlane(sc3, 63);
            bytes = o3 + ((n3 + 7) >> 3) * 4;
            raw = n3 > MAXESC || bytes > Lc;
            if (!raw) {
                o = sc3 - cnt3 + (unsigned)(size_t)(A + EP_L);
#pragma unroll
                for (int k = 0; k < 8; ++k) {
                    const unsigned long long z = (unsigned long long)__builtin_amdgcn_perm(0u, k < 4 ? d0[k & 3] : d1[k & 3], se[k]) << ((o & 3u) * 8);
                    LAS unsigned* d = (LAS unsigned*)(size_t)(o & ~3u);
                    __hip_atomic_fetch_or(d, (unsigned)z, __ATOMIC_RELAXED, __HIP_MEMORY_SCOPE_WAVEFRONT);
                    __hip_atomic_fetch_or(d + 1, (unsigned)(z >> 32), __ATOMIC_RELAXED, __HIP_MEMORY_SCOPE_WAVEFRONT);
                    o += __builtin_popcount(m5[k]);
                }
                CFENCE;
                const u32x2 x3 = *(const LAS u32x2*)(A + EP_L + lane * 8);
                *(LAS unsigned*)(B + o3 + lane * 4) = (x3[0] & 0x0f0f0f0fu) | ((x3[1] << 4) & 0xf0f0f0f0u);
                CFENCE;
            }
        }
    }
    if (raw) bytes = Lc;
    CYC(3);
    {
        LAS unsigned char* S = raw ? A : B;
#pragma nounroll
        for (unsigned r = 0; r < bytes; r += 1024)
            ST128(*(const LAS u32x4*)(S + r + lane * 16), rs, lane * 16, OFF_COD + wt * 8192 + r, AUX_SYS);
    }
    if (lane == 0) red[24 + wid] = (raw ? 1u << 30 : (bytes >> 2) << 14) | (ec << 26) | Lc;
    CYC(4);
    WAIT_V(0);
    CYC(5);
    {
        unsigned old = 0;
        if (lane == 0) old = __hip_atomic_fetch_add(red + 36 + wr, 1u, __ATOMIC_RELAXED, __HIP_MEMORY_SCOPE_WORKGROUP);
        if (__builtin_amdgcn_readfirstlane(old) == 3 && lane < 32 && (lane & 7) != 0) {
            const int w = wr * 4 + (lane >> 3);
            __hip_atomic_store((GAS unsigned long long*)(pl + OFF_DONE + (rank * 1024 + tile * 8 + w) * 8),
                               ((unsigned long long)((volatile LAS unsigned*)red)[24 + w] << 32) | call, __ATOMIC_RELAXED, __HIP_MEMORY_SCOPE_SYSTEM);
        }
    }
    STAMP(myws, wv, 12);
    {
        const rsrc_t ro = mk_rsrc(p.out, OUT_BYTES);
        const unsigned so = out_soff(rank, wt), vo = out_voff(lane);
        u32x4 qv[8];
#pragma unroll
        for (int ch = 0; ch < 8; ++ch) QUANT(ch, qv[ch]);
#pragma unroll
        for (int ch = 0; ch < 8; ++ch) ST128_RAW(qv[ch], ro, vo, so + chunk_soff(ch), AUX_WT);
        ST128_PAD8(qv);
    }
}

__global__ __launch_bounds__(512, 2) void fused_kernel(Args a) {
    __shared__ __attribute__((aligned(16))) unsigned char lds_arr[LDS_BYTES];
    LAS unsigned char* lds = (LAS unsigned char*)lds_arr;
    const int wid = __builtin_amdgcn_readfirstlane(threadIdx.x >> 6), lane = threadIdx.x & 63;
    const int b = blockIdx.x, xcd = b & 7, idx = b >> 3;
    Params p;
    p.x = a.x; p.w = a.w; p.out = a.out; p.my = (char*)a.ws[0]; p.rank = a.rank; p.pl = (char*)a.ws[lane & 7];
    asm volatile("" : "+s"(p.x), "+s"(p.w), "+s"(p.out), "+s"(p.my), "+s"(p.rank));
    const unsigned callv = (unsigned)__builtin_amdgcn_raw_ptr_buffer_atomic_add_i32(1, mk_rsrc(p.my, WS_USED), lane == 0 ? OFF_CNT + (b * 8 + wid) * 4 : 0xfffffff0u, 0, 0);
    if (idx < 16) gemm_role(p, lds, (idx >> 2) * 4 + (xcd >> 1), (xcd & 1) * 4 + (idx & 3), wid, lane, callv);
    else comm_role(p, lds, (idx - 16) * 8 + xcd, wid, lane, callv);
}

extern "C" void kernel_launch(int rank, int nranks, ncclComm_t comm, hipStream_t stream,
                              void* const* d_in, const int* in_sizes, int n_in,
                              void* d_out, int out_size,
                              const int* in_dtypes, int out_dtype,
                              void* const* d_out_all, void* const* d_ws_all, size_t ws_bytes) {
    (void)comm; (void)in_dtypes; (void)out_dtype; (void)d_out_all;
    if (nranks != NR || n_in != 2 || in_sizes[0] != M_ * KS || in_sizes[1] != KS * N_ || out_size != M_ * N_ || ws_bytes < WS_USED || (unsigned)rank >= (unsigned)NR) {
        fprintf(stderr, "kernel_launch: unexpected problem shape\n");
        return;
    }
    Args a;
    a.x = d_in[0]; a.w = d_in[1]; a.out = d_out;
    const size_t shift = (ws_bytes - WS_USED) & ~(size_t)0x1fffff;
    for (int i = 0; i < NR; ++i) a.ws[i] = (char*)d_ws_all[(rank + i) % NR] + shift;
    a.rank = rank; a.pad = 0;
    fused_kernel<<<dim3(NWG), dim3(512), 0, stream>>>(a);
}
```
